```python
import numpy as np

kernel_name = "gpu_gemm_ar_m4096_k4096_n16384_e5m2e5m2_dequant"


ABSMAX_RTOL = 0.05

COMPUTE_DTYPE = "float32"
EPILOGUE = "dequant"
IN_QUANT = None
OUT_QUANT = None
IN_FP8 = "e5m2e5m2"
IN_DTYPES = {"x": "e5m2", "w_mat": "e5m2",
             "scale_x": "float32", "scale_w": "float32"}
WORLD_SIZE = 8

def _snap_fp8(x, mant_bits, max_abs, min_normal):
    a = np.abs(x.astype(np.float64))
    _m, _e = np.frexp(a)
    q = np.where(a >= min_normal, np.ldexp(1.0, _e - 1 - mant_bits),
                 min_normal * 2.0 ** -mant_bits)
    snapped = np.minimum(np.round(a / q) * q, max_abs)
    return (np.sign(x) * snapped).astype(np.float32)

def setup_inputs():

    x = _snap_fp8(np.random.randn(4096, 4096).astype(np.float32), 2, 57344.0, 2.0 ** -14)
    w_mat = _snap_fp8(np.random.randn(4096, 16384).astype(np.float32), 2, 57344.0, 2.0 ** -14)
    scale_x = (0.02 * (1.0 + np.random.rand(1))).astype(np.float32)
    scale_w = (0.02 * (1.0 + np.random.rand(1))).astype(np.float32)
    return {"x": x, "w_mat": w_mat, "scale_x": scale_x, "scale_w": scale_w}

def reference(x, w_mat, scale_x, scale_w):
    acc = (x @ w_mat).astype(np.float32)
    y = (acc * (scale_x * scale_w)).astype(np.float32)
    return y
```

```cpp
#include <hip/hip_runtime.h>
#include <rccl/rccl.h>
#include <cstdio>

#define GEMM_ASM_TEXT \
  "s_mov_b64 s[36:37], %[rs]\n" \
  "s_mov_b32 s38, 0x5000000\n" \
  "s_mov_b32 s39, 0x00020000\n" \
  "s_mov_b64 s[40:41], %[ro]\n" \
  "s_mov_b32 s42, 0\n" \
  "s_mov_b32 s43, 0x00020000\n" \
  "s_mov_b64 s[40:41], %[ro]\n" \
  "s_mov_b32 s42, 0\n" \
  "s_mov_b32 s43, 0x00020000\n" \
  "s_mov_b32 s72, 0x10000000\n" \
  "s_mov_b32 s69, %[m0b]\n" \
  "s_mov_b32 s75, %[xb]\n" \
  "s_mov_b32 s78, %[sc]\n" \
  "s_mov_b64 s[80:81], %[prog]\n" \
  "s_mov_b64 s[44:45], %[prog]\n" \
  "s_mov_b32 s46, %[pdn]\n" \
  "s_mov_b32 s47, 0x00020000\n" \
  "s_mov_b32 s68, 0\n" \
  "s_mov_b32 s79, 0\n" \
  "s_mov_b32 s86, 0\n" \
  "s_mov_b32 s87, 0\n" \
  "s_mov_b32 s88, 0x171\n" \
  "s_mov_b32 s48, 0\n" \
  "s_mov_b32 s49, 0\n" \
  "s_mov_b32 s50, 0\n" \
  "s_mov_b32 s51, 0\n" \
  "s_mov_b32 s48, 0\n" \
  "s_mov_b32 s49, 0\n" \
  "s_mov_b32 s50, 0\n" \
  "s_mov_b32 s51, 0\n" \
  "s_mov_b32 s56, 0x80000\n" \
  "s_mov_b32 s57, 0x80000\n" \
  "s_mov_b32 s58, 0x80000\n" \
  "s_mov_b32 s59, 0x80000\n" \
  "s_mov_b32 s60, 0x80000\n" \
  "s_mov_b32 s61, 0x80000\n" \
  "s_mov_b32 s62, 0x80000\n" \
  "s_mov_b32 s63, 0x80000\n" \
  "s_mov_b32 s64, 0x80000\n" \
  "s_getpc_b64 s[94:95]\n" \
  "Lpc_%=:\n" \
  "s_add_u32 s90, s94, Lpollr_%=-Lpc_%=\n" \
  "s_addc_u32 s91, s95, 0\n" \
  "s_memrealtime s[82:83]\n" \
  "v_mbcnt_lo_u32_b32 v160, -1, 0\n" \
  "v_mbcnt_hi_u32_b32 v160, -1, v160\n" \
  "v_and_b32 v167, 15, v160\n" \
  "v_lshlrev_b32 v167, 16, v167\n" \
  "v_and_b32 v216, 48, v160\n" \
  "v_add_u32 v167, v167, v216\n" \
  "v_and_b32 v222, 15, v160\n" \
  "v_mul_u32_u24 v222, 768, v222\n" \
  "v_add_u32 v222, v222, v216\n" \
  "v_mov_b32 v224, %[sg0]\n" \
  "s_mov_b64 s[52:53], %[sb]\n" \
  "s_mov_b32 s54, 0x300000\n" \
  "s_mov_b32 s55, 0x00020000\n" \
  "v_mov_b32 v168, 0\n" \
  "v_lshlrev_b32 v160, 4, v160\n" \
  "v_add_u32 v162, s69, v160\n" \
  "v_add_u32 v161, 0x10000, v160\n" \
  "v_mov_b32 v218, 0\n" \
  "v_mov_b32 v219, 0\n" \
  "v_lshlrev_b32 v184, 3, v162\n" \
  "v_add_u32 v185, 0, v184\n" \
  "v_min_u32 v185, Ldone_%=-Lpc_%=-4, v185\n" \
  "global_load_dword v186, v185, s[94:95]\n" \
  "v_add_u32 v185, 32768, v184\n" \
  "v_min_u32 v185, Ldone_%=-Lpc_%=-4, v185\n" \
  "global_load_dword v187, v185, s[94:95]\n" \
  "v_add_u32 v185, 65536, v184\n" \
  "v_min_u32 v185, Ldone_%=-Lpc_%=-4, v185\n" \
  "global_load_dword v188, v185, s[94:95]\n" \
  "v_add_u32 v185, 98304, v184\n" \
  "v_min_u32 v185, Ldone_%=-Lpc_%=-4, v185\n" \
  "global_load_dword v189, v185, s[94:95]\n" \
  "v_mov_b32 v169, %[t0]\n" \
  "v_mov_b32 v170, %[t1]\n" \
  "v_mov_b32 v171, %[t2]\n" \
  "v_mov_b32 v172, %[t3]\n" \
  "v_mov_b32 v173, %[t4]\n" \
  "v_mov_b32 v174, %[t5]\n" \
  "v_mov_b32 v175, %[t6]\n" \
  "v_mov_b32 v176, %[t7]\n" \
  "v_mov_b32 v177, %[t8]\n" \
  "v_mov_b32 v178, %[t9]\n" \
  "v_mov_b32 v179, %[t10]\n" \
  "v_mov_b32 v180, %[t11]\n" \
  "v_mov_b32 v181, %[t12]\n" \
  "v_mov_b32 v182, %[t13]\n" \
  "v_mov_b32 v183, %[t14]\n" \
  "s_waitcnt lgkmcnt(0)\n" \
  "s_mov_b32 s70, 0\n" \
  "s_nop 0\n" \
  "v_readlane_b32 s67, v175, s70\n" \
  "v_readlane_b32 s65, v178, s70\n" \
  "s_nop 3\n" \
  "v_readlane_b32 s76, v169, s70\n" \
  "v_readlane_b32 s77, v172, s70\n" \
  "s_nop 3\n" \
  "v_readlane_b32 s71, v181, s70\n" \
  "s_add_u32 s70, s70, 1\n" \
  "v_add_u32 v163, s76, v162\n" \
  "s_nop 3\n" \
  "v_add_u32 v164, 4096, v163\n" \
  "s_cmp_gt_u32 s67, s68\n" \
  "s_cbranch_scc1 Lp1_%=\n" \
  "Lq1_%=:\n" \
  "s_nop 3\n" \
  "s_bfe_u32 m0, s65, 0x40018\n" \
  "s_nop 0\n" \
  "s_movreld_b32 s56, s77\n" \
  "s_barrier\n" \
  "s_add_u32 m0, s69, 0\n" \
  "s_nop 0\n" \
  "buffer_load_dwordx4 v163, s[36:39], s56 offen lds\n" \
  "s_add_u32 m0, s69, 4096\n" \
  "s_nop 0\n" \
  "buffer_load_dwordx4 v163, s[36:39], s57 offen lds\n" \
  "s_add_u32 m0, s69, 8192\n" \
  "s_nop 0\n" \
  "buffer_load_dwordx4 v163, s[36:39], s58 offen lds\n" \
  "s_add_u32 m0, s69, 12288\n" \
  "s_nop 0\n" \
  "buffer_load_dwordx4 v163, s[36:39], s59 offen lds\n" \
  "s_add_u32 m0, s69, 16384\n" \
  "s_nop 0\n" \
  "buffer_load_dwordx4 v163, s[36:39], s60 offen lds\n" \
  "s_add_u32 m0, s69, 20480\n" \
  "s_nop 0\n" \
  "buffer_load_dwordx4 v163, s[36:39], s61 offen lds\n" \
  "s_add_u32 m0, s69, 24576\n" \
  "s_nop 0\n" \
  "buffer_load_dwordx4 v163, s[36:39], s62 offen lds\n" \
  "s_add_u32 m0, s69, 28672\n" \
  "s_nop 0\n" \
  "buffer_load_dwordx4 v163, s[36:39], s63 offen lds\n" \
  "buffer_load_dwordx4 v[0:3], v163, s[36:39], s75 offen offset:0\n" \
  "buffer_load_dwordx4 v[4:7], v163, s[36:39], s75 offen offset:1024\n" \
  "buffer_load_dwordx4 v[8:11], v163, s[36:39], s75 offen offset:2048\n" \
  "buffer_load_dwordx4 v[12:15], v163, s[36:39], s75 offen offset:3072\n" \
  "buffer_load_dwordx4 v[16:19], v164, s[36:39], s75 offen offset:0\n" \
  "buffer_load_dwordx4 v[20:23], v164, s[36:39], s75 offen offset:1024\n" \
  "buffer_load_dwordx4 v[24:27], v164, s[36:39], s75 offen offset:2048\n" \
  "buffer_load_dwordx4 v[28:31], v164, s[36:39], s75 offen offset:3072\n" \
  "buffer_store_dwordx4 v[184:187], v167, s[40:43], s48 offen offset:0 nt\n" \
  "buffer_store_dwordx4 v[184:187], v167, s[40:43], s48 offen offset:0 nt\n" \
  "v_readlane_b32 s67, v175, s70\n" \
  "v_readlane_b32 s65, v178, s70\n" \
  "s_nop 3\n" \
  "v_readlane_b32 s76, v169, s70\n" \
  "v_readlane_b32 s77, v172, s70\n" \
  "s_nop 3\n" \
  "v_readlane_b32 s71, v181, s70\n" \
  "s_add_u32 s70, s70, 1\n" \
  "v_add_u32 v163, s76, v162\n" \
  "s_nop 3\n" \
  "v_add_u32 v164, 4096, v163\n" \
  "s_cmp_gt_u32 s67, s68\n" \
  "s_cbranch_scc1 Lp2_%=\n" \
  "Lq2_%=:\n" \
  "s_nop 3\n" \
  "s_bfe_u32 m0, s65, 0x40018\n" \
  "s_nop 0\n" \
  "s_movreld_b32 s56, s77\n" \
  "s_barrier\n" \
  "s_add_u32 m0, s69, 32768\n" \
  "s_nop 0\n" \
  "buffer_load_dwordx4 v163, s[36:39], s56 offen lds\n" \
  "s_add_u32 m0, s69, 36864\n" \
  "s_nop 0\n" \
  "buffer_load_dwordx4 v163, s[36:39], s57 offen lds\n" \
  "s_add_u32 m0, s69, 40960\n" \
  "s_nop 0\n" \
  "buffer_load_dwordx4 v163, s[36:39], s58 offen lds\n" \
  "s_add_u32 m0, s69, 45056\n" \
  "s_nop 0\n" \
  "buffer_load_dwordx4 v163, s[36:39], s59 offen lds\n" \
  "s_add_u32 m0, s69, 49152\n" \
  "s_nop 0\n" \
  "buffer_load_dwordx4 v163, s[36:39], s60 offen lds\n" \
  "s_add_u32 m0, s69, 53248\n" \
  "s_nop 0\n" \
  "buffer_load_dwordx4 v163, s[36:39], s61 offen lds\n" \
  "s_add_u32 m0, s69, 57344\n" \
  "s_nop 0\n" \
  "buffer_load_dwordx4 v163, s[36:39], s62 offen lds\n" \
  "s_add_u32 m0, s69, 61440\n" \
  "s_nop 0\n" \
  "buffer_load_dwordx4 v163, s[36:39], s63 offen lds\n" \
  "buffer_load_dwordx4 v[32:35], v163, s[36:39], s75 offen offset:0\n" \
  "buffer_load_dwordx4 v[36:39], v163, s[36:39], s75 offen offset:1024\n" \
  "buffer_load_dwordx4 v[40:43], v163, s[36:39], s75 offen offset:2048\n" \
  "buffer_load_dwordx4 v[44:47], v163, s[36:39], s75 offen offset:3072\n" \
  "buffer_load_dwordx4 v[48:51], v164, s[36:39], s75 offen offset:0\n" \
  "buffer_load_dwordx4 v[52:55], v164, s[36:39], s75 offen offset:1024\n" \
  "buffer_load_dwordx4 v[56:59], v164, s[36:39], s75 offen offset:2048\n" \
  "buffer_load_dwordx4 v[60:63], v164, s[36:39], s75 offen offset:3072\n" \
  "buffer_store_dwordx4 v[184:187], v167, s[40:43], s48 offen offset:0 nt\n" \
  "buffer_store_dwordx4 v[184:187], v167, s[40:43], s48 offen offset:0 nt\n" \
  "v_readlane_b32 s67, v175, s70\n" \
  "v_readlane_b32 s65, v178, s70\n" \
  "s_nop 3\n" \
  "v_readlane_b32 s76, v169, s70\n" \
  "v_readlane_b32 s77, v172, s70\n" \
  "s_nop 3\n" \
  "v_readlane_b32 s71, v181, s70\n" \
  "s_add_u32 s70, s70, 1\n" \
  "v_add_u32 v163, s76, v162\n" \
  "s_nop 3\n" \
  "v_add_u32 v164, 4096, v163\n" \
  "s_cmp_gt_u32 s67, s68\n" \
  "s_cbranch_scc1 Lp3_%=\n" \
  "Lq3_%=:\n" \
  "s_nop 3\n" \
  "s_bfe_u32 m0, s65, 0x40018\n" \
  "s_nop 0\n" \
  "s_movreld_b32 s56, s77\n" \
  "s_barrier\n" \
  "s_add_u32 m0, s69, 65536\n" \
  "s_nop 0\n" \
  "buffer_load_dwordx4 v163, s[36:39], s56 offen lds\n" \
  "s_add_u32 m0, s69, 69632\n" \
  "s_nop 0\n" \
  "buffer_load_dwordx4 v163, s[36:39], s57 offen lds\n" \
  "s_add_u32 m0, s69, 73728\n" \
  "s_nop 0\n" \
  "buffer_load_dwordx4 v163, s[36:39], s58 offen lds\n" \
  "s_add_u32 m0, s69, 77824\n" \
  "s_nop 0\n" \
  "buffer_load_dwordx4 v163, s[36:39], s59 offen lds\n" \
  "s_add_u32 m0, s69, 81920\n" \
  "s_nop 0\n" \
  "buffer_load_dwordx4 v163, s[36:39], s60 offen lds\n" \
  "s_add_u32 m0, s69, 86016\n" \
  "s_nop 0\n" \
  "buffer_load_dwordx4 v163, s[36:39], s61 offen lds\n" \
  "s_add_u32 m0, s69, 90112\n" \
  "s_nop 0\n" \
  "buffer_load_dwordx4 v163, s[36:39], s62 offen lds\n" \
  "s_add_u32 m0, s69, 94208\n" \
  "s_nop 0\n" \
  "buffer_load_dwordx4 v163, s[36:39], s63 offen lds\n" \
  "buffer_load_dwordx4 v[64:67], v163, s[36:39], s75 offen offset:0\n" \
  "buffer_load_dwordx4 v[68:71], v163, s[36:39], s75 offen offset:1024\n" \
  "buffer_load_dwordx4 v[72:75], v163, s[36:39], s75 offen offset:2048\n" \
  "buffer_load_dwordx4 v[76:79], v163, s[36:39], s75 offen offset:3072\n" \
  "buffer_load_dwordx4 v[80:83], v164, s[36:39], s75 offen offset:0\n" \
  "buffer_load_dwordx4 v[84:87], v164, s[36:39], s75 offen offset:1024\n" \
  "buffer_load_dwordx4 v[88:91], v164, s[36:39], s75 offen offset:2048\n" \
  "buffer_load_dwordx4 v[92:95], v164, s[36:39], s75 offen offset:3072\n" \
  "buffer_store_dwordx4 v[184:187], v167, s[40:43], s48 offen offset:0 nt\n" \
  "buffer_store_dwordx4 v[184:187], v167, s[40:43], s48 offen offset:0 nt\n" \
  "v_readlane_b32 s67, v175, s70\n" \
  "v_readlane_b32 s65, v178, s70\n" \
  "s_nop 3\n" \
  "v_readlane_b32 s76, v169, s70\n" \
  "v_readlane_b32 s77, v172, s70\n" \
  "s_nop 3\n" \
  "v_readlane_b32 s71, v181, s70\n" \
  "s_add_u32 s70, s70, 1\n" \
  "v_add_u32 v163, s76, v162\n" \
  "s_nop 3\n" \
  "v_add_u32 v164, 4096, v163\n" \
  "s_cmp_gt_u32 s67, s68\n" \
  "s_cbranch_scc1 Lp4_%=\n" \
  "Lq4_%=:\n" \
  "s_nop 3\n" \
  "s_bfe_u32 m0, s65, 0x40018\n" \
  "s_nop 0\n" \
  "s_movreld_b32 s56, s77\n" \
  "s_nop 3\n" \
  "Lm0_%=:\n" \
  "s_bitcmp1_b32 s65, 31\n" \
  "s_cbranch_scc1 Ldone_%=\n" \
  "s_bitcmp1_b32 s65, 29\n" \
  "s_cbranch_scc1 Lrd_%=\n" \
  "s_and_b32 s98, s65, 15\n" \
  "s_bitcmp1_b32 s65, 10\n" \
  "s_cselect_b32 s74, 9, 0\n" \
  "s_add_u32 s98, s98, s74\n" \
  "s_mul_i32 s98, s98, Lpre0_0_1_%=-Lpre0_0_0_%=\n" \
  "s_add_u32 s98, s98, Lpre0_0_0_%=-Lpc_%=\n" \
  "s_add_u32 s98, s98, s94\n" \
  "s_addc_u32 s99, s95, 0\n" \
  "s_bfe_u32 s74, s65, 0x40004\n" \
  "s_bfe_u32 s96, s65, 0x30014\n" \
  "s_mul_i32 s96, s96, Lev0_1_%=-Lev0_0_%=\n" \
  "s_add_u32 s96, s96, Lev0_0_%=-Lpc_%=\n" \
  "s_add_u32 s96, s96, s94\n" \
  "s_addc_u32 s97, s95, 0\n" \
  "s_waitcnt vmcnt(36) lgkmcnt(0)\n" \
  "s_barrier\n" \
  "s_bitcmp1_b32 s65, 19\n" \
  "s_cbranch_scc1 Lme0_%=\n" \
  "Lmer0_%=:\n" \
  "s_bitcmp1_b32 s65, 8\n" \
  "s_cbranch_scc0 Lz0_%=\n" \
  "v_mfma_f32_16x16x128_f8f6f4 a[224:227], v[144:151], v[96:103], a[224:227] cbsz:1 blgp:1\n" \
  "s_add_u32 m0, s69, 126976\n" \
  "ds_read_b128 v[128:131], v160 offset:0\n" \
  "ds_read_b128 v[132:135], v160 offset:1024\n" \
  "v_mfma_f32_16x16x128_f8f6f4 a[228:231], v[144:151], v[104:111], a[228:231] cbsz:1 blgp:1\n" \
  "buffer_load_dwordx4 v163, s[36:39], s63 offen lds\n" \
  "v_mfma_f32_16x16x128_f8f6f4 a[232:235], v[144:151], v[112:119], a[232:235] cbsz:1 blgp:1\n" \
  "v_mfma_f32_16x16x128_f8f6f4 a[236:239], v[144:151], v[120:127], a[236:239] cbsz:1 blgp:1\n" \
  "v_mfma_f32_16x16x128_f8f6f4 a[240:243], v[152:159], v[96:103], a[240:243] cbsz:1 blgp:1\n" \
  "ds_read_b128 v[136:139], v160 offset:2048\n" \
  "ds_read_b128 v[140:143], v160 offset:3072\n" \
  "v_mfma_f32_16x16x128_f8f6f4 a[244:247], v[152:159], v[104:111], a[244:247] cbsz:1 blgp:1\n" \
  "v_mfma_f32_16x16x128_f8f6f4 a[248:251], v[152:159], v[112:119], a[248:251] cbsz:1 blgp:1\n" \
  "v_mfma_f32_16x16x128_f8f6f4 a[252:255], v[152:159], v[120:127], a[252:255] cbsz:1 blgp:1\n" \
  "Lzr0_%=:\n" \
  "s_bitcmp1_b32 s65, 9\n" \
  "s_cbranch_scc1 Lpj0_%=\n" \
  "Ls0_1_%=:\n" \
  "s_waitcnt lgkmcnt(2)\n" \
  "v_mfma_f32_16x16x128_f8f6f4 a[0:3], v[128:135], v[0:7], a[0:3] cbsz:1 blgp:1\n" \
  "s_add_u32 m0, s69, 98304\n" \
  "ds_read_b128 v[144:147], v160 offset:4096\n" \
  "ds_read_b128 v[148:151], v160 offset:5120\n" \
  "v_mfma_f32_16x16x128_f8f6f4 a[4:7], v[128:135], v[8:15], a[4:7] cbsz:1 blgp:1\n" \
  "buffer_load_dwordx4 v163, s[36:39], s56 offen lds\n" \
  "v_mfma_f32_16x16x128_f8f6f4 a[8:11], v[128:135], v[16:23], a[8:11] cbsz:1 blgp:1\n" \
  "v_mfma_f32_16x16x128_f8f6f4 a[12:15], v[128:135], v[24:31], a[12:15] cbsz:1 blgp:1\n" \
  "s_waitcnt lgkmcnt(2)\n" \
  "v_mfma_f32_16x16x128_f8f6f4 a[16:19], v[136:143], v[0:7], a[16:19] cbsz:1 blgp:1\n" \
  "ds_read_b128 v[152:155], v160 offset:6144\n" \
  "ds_read_b128 v[156:159], v160 offset:7168\n" \
  "v_mfma_f32_16x16x128_f8f6f4 a[20:23], v[136:143], v[8:15], a[20:23] cbsz:1 blgp:1\n" \
  "v_mfma_f32_16x16x128_f8f6f4 a[24:27], v[136:143], v[16:23], a[24:27] cbsz:1 blgp:1\n" \
  "v_mfma_f32_16x16x128_f8f6f4 a[28:31], v[136:143], v[24:31], a[28:31] cbsz:1 blgp:1\n" \
  "s_cmp_eq_u32 s74, 1\n" \
  "s_cbranch_scc1 Lsuf0_0_1_%=\n" \
  "Ls0_2_%=:\n" \
  "s_waitcnt lgkmcnt(2)\n" \
  "v_mfma_f32_16x16x128_f8f6f4 a[32:35], v[144:151], v[0:7], a[32:35] cbsz:1 blgp:1\n" \
  "s_add_u32 m0, s69, 102400\n" \
  "ds_read_b128 v[128:131], v160 offset:8192\n" \
  "ds_read_b128 v[132:135], v160 offset:9216\n" \
  "v_mfma_f32_16x16x128_f8f6f4 a[36:39], v[144:151], v[8:15], a[36:39] cbsz:1 blgp:1\n" \
  "buffer_load_dwordx4 v163, s[36:39], s57 offen lds\n" \
  "v_mfma_f32_16x16x128_f8f6f4 a[40:43], v[144:151], v[16:23], a[40:43] cbsz:1 blgp:1\n" \
  "v_mfma_f32_16x16x128_f8f6f4 a[44:47], v[144:151], v[24:31], a[44:47] cbsz:1 blgp:1\n" \
  "s_waitcnt lgkmcnt(2)\n" \
  "v_mfma_f32_16x16x128_f8f6f4 a[48:51], v[152:159], v[0:7], a[48:51] cbsz:1 blgp:1\n" \
  "ds_read_b128 v[136:139], v160 offset:10240\n" \
  "ds_read_b128 v[140:143], v160 offset:11264\n" \
  "v_mfma_f32_16x16x128_f8f6f4 a[52:55], v[152:159], v[8:15], a[52:55] cbsz:1 blgp:1\n" \
  "v_mfma_f32_16x16x128_f8f6f4 a[56:59], v[152:159], v[16:23], a[56:59] cbsz:1 blgp:1\n" \
  "v_mfma_f32_16x16x128_f8f6f4 a[60:63], v[152:159], v[24:31], a[60:63] cbsz:1 blgp:1\n" \
  "s_cmp_eq_u32 s74, 2\n" \
  "s_cbranch_scc1 Lsuf0_0_2_%=\n" \
  "Ls0_3_%=:\n" \
  "s_waitcnt lgkmcnt(2)\n" \
  "v_mfma_f32_16x16x128_f8f6f4 a[64:67], v[128:135], v[0:7], a[64:67] cbsz:1 blgp:1\n" \
  "s_add_u32 m0, s69, 106496\n" \
  "ds_read_b128 v[144:147], v160 offset:12288\n" \
  "ds_read_b128 v[148:151], v160 offset:13312\n" \
  "v_mfma_f32_16x16x128_f8f6f4 a[68:71], v[128:135], v[8:15], a[68:71] cbsz:1 blgp:1\n" \
  "buffer_load_dwordx4 v163, s[36:39], s58 offen lds\n" \
  "v_mfma_f32_16x16x128_f8f6f4 a[72:75], v[128:135], v[16:23], a[72:75] cbsz:1 blgp:1\n" \
  "v_mfma_f32_16x16x128_f8f6f4 a[76:79], v[128:135], v[24:31], a[76:79] cbsz:1 blgp:1\n" \
  "s_waitcnt lgkmcnt(2)\n" \
  "v_mfma_f32_16x16x128_f8f6f4 a[80:83], v[136:143], v[0:7], a[80:83] cbsz:1 blgp:1\n" \
  "ds_read_b128 v[152:155], v160 offset:14336\n" \
  "ds_read_b128 v[156:159], v160 offset:15360\n" \
  "v_mfma_f32_16x16x128_f8f6f4 a[84:87], v[136:143], v[8:15], a[84:87] cbsz:1 blgp:1\n" \
  "v_mfma_f32_16x16x128_f8f6f4 a[88:91], v[136:143], v[16:23], a[88:91] cbsz:1 blgp:1\n" \
  "v_mfma_f32_16x16x128_f8f6f4 a[92:95], v[136:143], v[24:31], a[92:95] cbsz:1 blgp:1\n" \
  "s_cmp_eq_u32 s74, 3\n" \
  "s_cbranch_scc1 Lsuf0_0_3_%=\n" \
  "Ls0_4_%=:\n" \
  "s_waitcnt lgkmcnt(2)\n" \
  "v_mfma_f32_16x16x128_f8f6f4 a[96:99], v[144:151], v[0:7], a[96:99] cbsz:1 blgp:1\n" \
  "s_add_u32 m0, s69, 110592\n" \
  "ds_read_b128 v[128:131], v160 offset:16384\n" \
  "ds_read_b128 v[132:135], v160 offset:17408\n" \
  "v_mfma_f32_16x16x128_f8f6f4 a[100:103], v[144:151], v[8:15], a[100:103] cbsz:1 blgp:1\n" \
  "buffer_load_dwordx4 v163, s[36:39], s59 offen lds\n" \
  "v_mfma_f32_16x16x128_f8f6f4 a[104:107], v[144:151], v[16:23], a[104:107] cbsz:1 blgp:1\n" \
  "v_mfma_f32_16x16x128_f8f6f4 a[108:111], v[144:151], v[24:31], a[108:111] cbsz:1 blgp:1\n" \
  "s_waitcnt lgkmcnt(2)\n" \
  "v_mfma_f32_16x16x128_f8f6f4 a[112:115], v[152:159], v[0:7], a[112:115] cbsz:1 blgp:1\n" \
  "ds_read_b128 v[136:139], v160 offset:18432\n" \
  "ds_read_b128 v[140:143], v160 offset:19456\n" \
  "v_mfma_f32_16x16x128_f8f6f4 a[116:119], v[152:159], v[8:15], a[116:119] cbsz:1 blgp:1\n" \
  "v_mfma_f32_16x16x128_f8f6f4 a[120:123], v[152:159], v[16:23], a[120:123] cbsz:1 blgp:1\n" \
  "v_mfma_f32_16x16x128_f8f6f4 a[124:127], v[152:159], v[24:31], a[124:127] cbsz:1 blgp:1\n" \
  "s_cmp_eq_u32 s74, 4\n" \
  "s_cbranch_scc1 Lsuf0_0_4_%=\n" \
  "Ls0_5_%=:\n" \
  "s_waitcnt lgkmcnt(2)\n" \
  "v_mfma_f32_16x16x128_f8f6f4 a[128:131], v[128:135], v[0:7], a[128:131] cbsz:1 blgp:1\n" \
  "s_add_u32 m0, s69, 114688\n" \
  "ds_read_b128 v[144:147], v160 offset:20480\n" \
  "ds_read_b128 v[148:151], v160 offset:21504\n" \
  "v_mfma_f32_16x16x128_f8f6f4 a[132:135], v[128:135], v[8:15], a[132:135] cbsz:1 blgp:1\n" \
  "buffer_load_dwordx4 v163, s[36:39], s60 offen lds\n" \
  "v_mfma_f32_16x16x128_f8f6f4 a[136:139], v[128:135], v[16:23], a[136:139] cbsz:1 blgp:1\n" \
  "v_readfirstlane_b32 s89, v218\n" \
  "s_nop 1\n" \
  "v_mfma_f32_16x16x128_f8f6f4 a[140:143], v[128:135], v[24:31], a[140:143] cbsz:1 blgp:1\n" \
  "buffer_load_dwordx4 v[100:103], v163, s[36:39], s75 offen offset:1024\n" \
  "s_bitcmp1_b32 s65, 18\n" \
  "s_cselect_b32 s42, s72, 0\n" \
  "s_mov_b32 s48, s71\n" \
  "s_bitcmp1_b32 s65, 28\n" \
  "s_cselect_b32 s73, 3, 0x100\n" \
  "s_lshl_b32 s73, s73, 12\n" \
  "s_add_u32 s49, s48, s73\n" \
  "s_add_u32 s50, s49, s73\n" \
  "s_add_u32 s51, s50, s73\n" \
  "s_cmp_eq_u32 s70, 64\n" \
  "s_cbranch_scc1 Lp5_%=\n" \
  "Lq5_%=:\n" \
  "s_waitcnt lgkmcnt(2)\n" \
  "v_mfma_f32_16x16x128_f8f6f4 a[144:147], v[136:143], v[0:7], a[144:147] cbsz:1 blgp:1\n" \
  "ds_read_b128 v[152:155], v160 offset:22528\n" \
  "ds_read_b128 v[156:159], v160 offset:23552\n" \
  "v_mfma_f32_16x16x128_f8f6f4 a[148:151], v[136:143], v[8:15], a[148:151] cbsz:1 blgp:1\n" \
  "buffer_load_dwordx4 v[96:99], v163, s[36:39], s75 offen offset:0\n" \
  "v_mfma_f32_16x16x128_f8f6f4 a[152:155], v[136:143], v[16:23], a[152:155] cbsz:1 blgp:1\n" \
  "s_max_u32 s68, s68, s89\n" \
  "v_mfma_f32_16x16x128_f8f6f4 a[156:159], v[136:143], v[24:31], a[156:159] cbsz:1 blgp:1\n" \
  "v_readlane_b32 s67, v175, s70\n" \
  "v_readlane_b32 s66, v178, s70\n" \
  "v_readlane_b32 s76, v169, s70\n" \
  "v_readlane_b32 s77, v172, s70\n" \
  "s_add_u32 s70, s70, 1\n" \
  "v_add_u32 v165, s76, v162\n" \
  "s_cmp_eq_u32 s74, 5\n" \
  "s_cbranch_scc1 Lsuf0_0_5_%=\n" \
  "Ls0_6_%=:\n" \
  "s_waitcnt lgkmcnt(2)\n" \
  "v_mfma_f32_16x16x128_f8f6f4 a[160:163], v[144:151], v[0:7], a[160:163] cbsz:1 blgp:1\n" \
  "s_add_u32 m0, s69, 118784\n" \
  "ds_read_b128 v[128:131], v160 offset:24576\n" \
  "ds_read_b128 v[132:135], v160 offset:25600\n" \
  "v_mfma_f32_16x16x128_f8f6f4 a[164:167], v[144:151], v[8:15], a[164:167] cbsz:1 blgp:1\n" \
  "buffer_load_dwordx4 v163, s[36:39], s61 offen lds\n" \
  "v_mfma_f32_16x16x128_f8f6f4 a[168:171], v[144:151], v[16:23], a[168:171] cbsz:1 blgp:1\n" \
  "v_mfma_f32_16x16x128_f8f6f4 a[172:175], v[144:151], v[24:31], a[172:175] cbsz:1 blgp:1\n" \
  "buffer_load_dwordx4 v[108:111], v163, s[36:39], s75 offen offset:3072\n" \
  "v_add_u32 v166, 4096, v165\n" \
  "s_cmp_gt_u32 s67, s68\n" \
  "s_cbranch_scc1 Lp6_%=\n" \
  "Lq6_%=:\n" \
  "s_waitcnt lgkmcnt(2)\n" \
  "v_mfma_f32_16x16x128_f8f6f4 a[176:179], v[152:159], v[0:7], a[176:179] cbsz:1 blgp:1\n" \
  "ds_read_b128 v[136:139], v160 offset:26624\n" \
  "ds_read_b128 v[140:143], v160 offset:27648\n" \
  "v_mfma_f32_16x16x128_f8f6f4 a[180:183], v[152:159], v[8:15], a[180:183] cbsz:1 blgp:1\n" \
  "buffer_load_dwordx4 v[104:107], v163, s[36:39], s75 offen offset:2048\n" \
  "v_mfma_f32_16x16x128_f8f6f4 a[184:187], v[152:159], v[16:23], a[184:187] cbsz:1 blgp:1\n" \
  "v_mfma_f32_16x16x128_f8f6f4 a[188:191], v[152:159], v[24:31], a[188:191] cbsz:1 blgp:1\n" \
  "buffer_load_dwordx4 v[112:115], v164, s[36:39], s75 offen offset:0\n" \
  "s_cmp_eq_u32 s74, 6\n" \
  "s_cbranch_scc1 Lsuf0_0_6_%=\n" \
  "Ls0_7_%=:\n" \
  "s_waitcnt lgkmcnt(2)\n" \
  "v_mfma_f32_16x16x128_f8f6f4 a[192:195], v[128:135], v[0:7], a[192:195] cbsz:1 blgp:1\n" \
  "s_add_u32 m0, s69, 122880\n" \
  "ds_read_b128 v[144:147], v160 offset:28672\n" \
  "ds_read_b128 v[148:151], v160 offset:29696\n" \
  "v_mfma_f32_16x16x128_f8f6f4 a[196:199], v[128:135], v[8:15], a[196:199] cbsz:1 blgp:1\n" \
  "buffer_load_dwordx4 v163, s[36:39], s62 offen lds\n" \
  "v_mfma_f32_16x16x128_f8f6f4 a[200:203], v[128:135], v[16:23], a[200:203] cbsz:1 blgp:1\n" \
  "v_mfma_f32_16x16x128_f8f6f4 a[204:207], v[128:135], v[24:31], a[204:207] cbsz:1 blgp:1\n" \
  "buffer_load_dwordx4 v[120:123], v164, s[36:39], s75 offen offset:2048\n" \
  "s_and_b32 s73, s66, 0xf00001ff\n" \
  "s_waitcnt lgkmcnt(2)\n" \
  "v_mfma_f32_16x16x128_f8f6f4 a[208:211], v[136:143], v[0:7], a[208:211] cbsz:1 blgp:1\n" \
  "ds_read_b128 v[152:155], v160 offset:30720\n" \
  "ds_read_b128 v[156:159], v160 offset:31744\n" \
  "v_mfma_f32_16x16x128_f8f6f4 a[212:215], v[136:143], v[8:15], a[212:215] cbsz:1 blgp:1\n" \
  "buffer_load_dwordx4 v[116:119], v164, s[36:39], s75 offen offset:1024\n" \
  "v_mfma_f32_16x16x128_f8f6f4 a[216:219], v[136:143], v[16:23], a[216:219] cbsz:1 blgp:1\n" \
  "v_mul_f32 v184, s78, v184\n" \
  "v_mul_f32 v185, s78, v185\n" \
  "v_mul_f32 v186, s78, v186\n" \
  "v_mul_f32 v187, s78, v187\n" \
  "s_bitcmp1_b32 s65, 28\n" \
  "s_cbranch_scc1 Lp7_%=\n" \
  "buffer_store_dwordx4 v[184:187], v167, s[40:43], s48 offen offset:0 nt\n" \
  "Lq7_%=:\n" \
  "v_mul_f32 v200, s78, v200\n" \
  "v_mul_f32 v201, s78, v201\n" \
  "v_mul_f32 v202, s78, v202\n" \
  "v_mul_f32 v203, s78, v203\n" \
  "s_bitcmp1_b32 s65, 28\n" \
  "s_cbranch_scc1 Lp8_%=\n" \
  "buffer_store_dwordx4 v[200:203], v167, s[40:43], s48 offen offset:64 nt\n" \
  "Lq8_%=:\n" \
  "v_mfma_f32_16x16x128_f8f6f4 a[220:223], v[136:143], v[24:31], a[220:223] cbsz:1 blgp:1\n" \
  "buffer_load_dwordx4 v[124:127], v164, s[36:39], s75 offen offset:3072\n" \
  "s_bfe_u32 m0, s66, 0x40018\n" \
  "s_nop 0\n" \
  "s_movreld_b32 s56, s77\n" \
  "Ls0_8_%=:\n" \
  "s_cmp_eq_u32 s73, s88\n" \
  "s_cbranch_scc1 Lf1_%=\n" \
  "Lm1_%=:\n" \
  "s_bitcmp1_b32 s66, 31\n" \
  "s_cbranch_scc1 Ldone_%=\n" \
  "s_and_b32 s98, s66, 15\n" \
  "s_bitcmp1_b32 s66, 10\n" \
  "s_cselect_b32 s74, 9, 0\n" \
  "s_add_u32 s98, s98, s74\n" \
  "s_mul_i32 s98, s98, Lpre1_0_1_%=-Lpre1_0_0_%=\n" \
  "s_add_u32 s98, s98, Lpre1_0_0_%=-Lpc_%=\n" \
  "s_add_u32 s98, s98, s94\n" \
  "s_addc_u32 s99, s95, 0\n" \
  "s_bfe_u32 s74, s66, 0x40004\n" \
  "s_waitcnt vmcnt(36) lgkmcnt(0)\n" \
  "s_barrier\n" \
  "s_bitcmp1_b32 s66, 8\n" \
  "s_cbranch_scc0 Lz1_%=\n" \
  "v_mfma_f32_16x16x128_f8f6f4 a[224:227], v[144:151], v[0:7], a[224:227] cbsz:1 blgp:1\n" \
  "s_add_u32 m0, s69, 28672\n" \
  "ds_read_b128 v[128:131], v160 offset:32768\n" \
  "ds_read_b128 v[132:135], v160 offset:33792\n" \
  "v_mfma_f32_16x16x128_f8f6f4 a[228:231], v[144:151], v[8:15], a[228:231] cbsz:1 blgp:1\n" \
  "buffer_load_dwordx4 v165, s[36:39], s63 offen lds\n" \
  "v_mfma_f32_16x16x128_f8f6f4 a[232:235], v[144:151], v[16:23], a[232:235] cbsz:1 blgp:1\n" \
  "v_mfma_f32_16x16x128_f8f6f4 a[236:239], v[144:151], v[24:31], a[236:239] cbsz:1 blgp:1\n" \
  "v_mfma_f32_16x16x128_f8f6f4 a[240:243], v[152:159], v[0:7], a[240:243] cbsz:1 blgp:1\n" \
  "ds_read_b128 v[136:139], v160 offset:34816\n" \
  "ds_read_b128 v[140:143], v160 offset:35840\n" \
  "v_mfma_f32_16x16x128_f8f6f4 a[244:247], v[152:159], v[8:15], a[244:247] cbsz:1 blgp:1\n" \
  "v_mfma_f32_16x16x128_f8f6f4 a[248:251], v[152:159], v[16:23], a[248:251] cbsz:1 blgp:1\n" \
  "v_mfma_f32_16x16x128_f8f6f4 a[252:255], v[152:159], v[24:31], a[252:255] cbsz:1 blgp:1\n" \
  "Lzr1_%=:\n" \
  "s_bitcmp1_b32 s66, 9\n" \
  "s_cbranch_scc1 Lpj1_%=\n" \
  "Ls1_1_%=:\n" \
  "s_waitcnt lgkmcnt(2)\n" \
  "v_mfma_f32_16x16x128_f8f6f4 a[0:3], v[128:135], v[32:39], a[0:3] cbsz:1 blgp:1\n" \
  "s_add_u32 m0, s69, 0\n" \
  "ds_read_b128 v[144:147], v160 offset:36864\n" \
  "ds_read_b128 v[148:151], v160 offset:37888\n" \
  "v_mfma_f32_16x16x128_f8f6f4 a[4:7], v[128:135], v[40:47], a[4:7] cbsz:1 blgp:1\n" \
  "buffer_load_dwordx4 v165, s[36:39], s56 offen lds\n" \
  "v_mfma_f32_16x16x128_f8f6f4 a[8:11], v[128:135], v[48:55], a[8:11] cbsz:1 blgp:1\n" \
  "v_mfma_f32_16x16x128_f8f6f4 a[12:15], v[128:135], v[56:63], a[12:15] cbsz:1 blgp:1\n" \
  "s_waitcnt lgkmcnt(2)\n" \
  "v_mfma_f32_16x16x128_f8f6f4 a[16:19], v[136:143], v[32:39], a[16:19] cbsz:1 blgp:1\n" \
  "ds_read_b128 v[152:155], v160 offset:38912\n" \
  "ds_read_b128 v[156:159], v160 offset:39936\n" \
  "v_mfma_f32_16x16x128_f8f6f4 a[20:23], v[136:143], v[40:47], a[20:23] cbsz:1 blgp:1\n" \
  "v_mfma_f32_16x16x128_f8f6f4 a[24:27], v[136:143], v[48:55], a[24:27] cbsz:1 blgp:1\n" \
  "v_mfma_f32_16x16x128_f8f6f4 a[28:31], v[136:143], v[56:63], a[28:31] cbsz:1 blgp:1\n" \
  "s_cmp_eq_u32 s74, 1\n" \
  "s_cbranch_scc1 Lsuf1_0_1_%=\n" \
  "Ls1_2_%=:\n" \
  "s_waitcnt lgkmcnt(2)\n" \
  "v_mfma_f32_16x16x128_f8f6f4 a[32:35], v[144:151], v[32:39], a[32:35] cbsz:1 blgp:1\n" \
  "s_add_u32 m0, s69, 4096\n" \
  "ds_read_b128 v[128:131], v160 offset:40960\n" \
  "ds_read_b128 v[132:135], v160 offset:41984\n" \
  "v_mfma_f32_16x16x128_f8f6f4 a[36:39], v[144:151], v[40:47], a[36:39] cbsz:1 blgp:1\n" \
  "buffer_load_dwordx4 v165, s[36:39], s57 offen lds\n" \
  "v_mfma_f32_16x16x128_f8f6f4 a[40:43], v[144:151], v[48:55], a[40:43] cbsz:1 blgp:1\n" \
  "v_mfma_f32_16x16x128_f8f6f4 a[44:47], v[144:151], v[56:63], a[44:47] cbsz:1 blgp:1\n" \
  "s_waitcnt lgkmcnt(2)\n" \
  "v_mfma_f32_16x16x128_f8f6f4 a[48:51], v[152:159], v[32:39], a[48:51] cbsz:1 blgp:1\n" \
  "ds_read_b128 v[136:139], v160 offset:43008\n" \
  "ds_read_b128 v[140:143], v160 offset:44032\n" \
  "v_mfma_f32_16x16x128_f8f6f4 a[52:55], v[152:159], v[40:47], a[52:55] cbsz:1 blgp:1\n" \
  "v_mfma_f32_16x16x128_f8f6f4 a[56:59], v[152:159], v[48:55], a[56:59] cbsz:1 blgp:1\n" \
  "v_mfma_f32_16x16x128_f8f6f4 a[60:63], v[152:159], v[56:63], a[60:63] cbsz:1 blgp:1\n" \
  "s_cmp_eq_u32 s74, 2\n" \
  "s_cbranch_scc1 Lsuf1_0_2_%=\n" \
  "Ls1_3_%=:\n" \
  "s_waitcnt lgkmcnt(2)\n" \
  "v_mfma_f32_16x16x128_f8f6f4 a[64:67], v[128:135], v[32:39], a[64:67] cbsz:1 blgp:1\n" \
  "s_add_u32 m0, s69, 8192\n" \
  "ds_read_b128 v[144:147], v160 offset:45056\n" \
  "ds_read_b128 v[148:151], v160 offset:46080\n" \
  "v_mfma_f32_16x16x128_f8f6f4 a[68:71], v[128:135], v[40:47], a[68:71] cbsz:1 blgp:1\n" \
  "buffer_load_dwordx4 v165, s[36:39], s58 offen lds\n" \
  "v_mfma_f32_16x16x128_f8f6f4 a[72:75], v[128:135], v[48:55], a[72:75] cbsz:1 blgp:1\n" \
  "v_mfma_f32_16x16x128_f8f6f4 a[76:79], v[128:135], v[56:63], a[76:79] cbsz:1 blgp:1\n" \
  "s_waitcnt lgkmcnt(2)\n" \
  "v_mfma_f32_16x16x128_f8f6f4 a[80:83], v[136:143], v[32:39], a[80:83] cbsz:1 blgp:1\n" \
  "ds_read_b128 v[152:155], v160 offset:47104\n" \
  "ds_read_b128 v[156:159], v160 offset:48128\n" \
  "v_mfma_f32_16x16x128_f8f6f4 a[84:87], v[136:143], v[40:47], a[84:87] cbsz:1 blgp:1\n" \
  "v_mfma_f32_16x16x128_f8f6f4 a[88:91], v[136:143], v[48:55], a[88:91] cbsz:1 blgp:1\n" \
  "v_mfma_f32_16x16x128_f8f6f4 a[92:95], v[136:143], v[56:63], a[92:95] cbsz:1 blgp:1\n" \
  "s_cmp_eq_u32 s74, 3\n" \
  "s_cbranch_scc1 Lsuf1_0_3_%=\n" \
  "Ls1_4_%=:\n" \
  "s_waitcnt lgkmcnt(2)\n" \
  "v_mfma_f32_16x16x128_f8f6f4 a[96:99], v[144:151], v[32:39], a[96:99] cbsz:1 blgp:1\n" \
  "s_add_u32 m0, s69, 12288\n" \
  "ds_read_b128 v[128:131], v160 offset:49152\n" \
  "ds_read_b128 v[132:135], v160 offset:50176\n" \
  "v_mfma_f32_16x16x128_f8f6f4 a[100:103], v[144:151], v[40:47], a[100:103] cbsz:1 blgp:1\n" \
  "buffer_load_dwordx4 v165, s[36:39], s59 offen lds\n" \
  "v_mfma_f32_16x16x128_f8f6f4 a[104:107], v[144:151], v[48:55], a[104:107] cbsz:1 blgp:1\n" \
  "v_mfma_f32_16x16x128_f8f6f4 a[108:111], v[144:151], v[56:63], a[108:111] cbsz:1 blgp:1\n" \
  "s_waitcnt lgkmcnt(2)\n" \
  "v_mfma_f32_16x16x128_f8f6f4 a[112:115], v[152:159], v[32:39], a[112:115] cbsz:1 blgp:1\n" \
  "ds_read_b128 v[136:139], v160 offset:51200\n" \
  "ds_read_b128 v[140:143], v160 offset:52224\n" \
  "v_mfma_f32_16x16x128_f8f6f4 a[116:119], v[152:159], v[40:47], a[116:119] cbsz:1 blgp:1\n" \
  "v_mfma_f32_16x16x128_f8f6f4 a[120:123], v[152:159], v[48:55], a[120:123] cbsz:1 blgp:1\n" \
  "v_mfma_f32_16x16x128_f8f6f4 a[124:127], v[152:159], v[56:63], a[124:127] cbsz:1 blgp:1\n" \
  "s_cmp_eq_u32 s74, 4\n" \
  "s_cbranch_scc1 Lsuf1_0_4_%=\n" \
  "Ls1_5_%=:\n" \
  "s_waitcnt lgkmcnt(2)\n" \
  "v_mfma_f32_16x16x128_f8f6f4 a[128:131], v[128:135], v[32:39], a[128:131] cbsz:1 blgp:1\n" \
  "s_add_u32 m0, s69, 16384\n" \
  "ds_read_b128 v[144:147], v160 offset:53248\n" \
  "ds_read_b128 v[148:151], v160 offset:54272\n" \
  "v_mfma_f32_16x16x128_f8f6f4 a[132:135], v[128:135], v[40:47], a[132:135] cbsz:1 blgp:1\n" \
  "buffer_load_dwordx4 v165, s[36:39], s60 offen lds\n" \
  "v_mfma_f32_16x16x128_f8f6f4 a[136:139], v[128:135], v[48:55], a[136:139] cbsz:1 blgp:1\n" \
  "v_mfma_f32_16x16x128_f8f6f4 a[140:143], v[128:135], v[56:63], a[140:143] cbsz:1 blgp:1\n" \
  "buffer_load_dwordx4 v[4:7], v165, s[36:39], s75 offen offset:1024\n" \
  "s_waitcnt lgkmcnt(2)\n" \
  "v_mfma_f32_16x16x128_f8f6f4 a[144:147], v[136:143], v[32:39], a[144:147] cbsz:1 blgp:1\n" \
  "ds_read_b128 v[152:155], v160 offset:55296\n" \
  "ds_read_b128 v[156:159], v160 offset:56320\n" \
  "v_mfma_f32_16x16x128_f8f6f4 a[148:151], v[136:143], v[40:47], a[148:151] cbsz:1 blgp:1\n" \
  "buffer_load_dwordx4 v[0:3], v165, s[36:39], s75 offen offset:0\n" \
  "v_mfma_f32_16x16x128_f8f6f4 a[152:155], v[136:143], v[48:55], a[152:155] cbsz:1 blgp:1\n" \
  "buffer_load_dword v218, off, s[44:47], 0 sc1\n" \
  "v_mfma_f32_16x16x128_f8f6f4 a[156:159], v[136:143], v[56:63], a[156:159] cbsz:1 blgp:1\n" \
  "v_readlane_b32 s67, v175, s70\n" \
  "v_readlane_b32 s65, v178, s70\n" \
  "v_readlane_b32 s76, v169, s70\n" \
  "v_readlane_b32 s77, v172, s70\n" \
  "s_add_u32 s70, s70, 1\n" \
  "v_add_u32 v163, s76, v162\n" \
  "s_cmp_eq_u32 s74, 5\n" \
  "s_cbranch_scc1 Lsuf1_0_5_%=\n" \
  "Ls1_6_%=:\n" \
  "s_waitcnt lgkmcnt(2)\n" \
  "v_mfma_f32_16x16x128_f8f6f4 a[160:163], v[144:151], v[32:39], a[160:163] cbsz:1 blgp:1\n" \
  "s_add_u32 m0, s69, 20480\n" \
  "ds_read_b128 v[128:131], v160 offset:57344\n" \
  "ds_read_b128 v[132:135], v160 offset:58368\n" \
  "v_mfma_f32_16x16x128_f8f6f4 a[164:167], v[144:151], v[40:47], a[164:167] cbsz:1 blgp:1\n" \
  "buffer_load_dwordx4 v165, s[36:39], s61 offen lds\n" \
  "v_mfma_f32_16x16x128_f8f6f4 a[168:171], v[144:151], v[48:55], a[168:171] cbsz:1 blgp:1\n" \
  "v_mfma_f32_16x16x128_f8f6f4 a[172:175], v[144:151], v[56:63], a[172:175] cbsz:1 blgp:1\n" \
  "buffer_load_dwordx4 v[12:15], v165, s[36:39], s75 offen offset:3072\n" \
  "v_add_u32 v164, 4096, v163\n" \
  "s_cmp_gt_u32 s67, s68\n" \
  "s_cbranch_scc1 Lp9_%=\n" \
  "Lq9_%=:\n" \
  "s_waitcnt lgkmcnt(2)\n" \
  "v_mfma_f32_16x16x128_f8f6f4 a[176:179], v[152:159], v[32:39], a[176:179] cbsz:1 blgp:1\n" \
  "ds_read_b128 v[136:139], v160 offset:59392\n" \
  "ds_read_b128 v[140:143], v160 offset:60416\n" \
  "v_mfma_f32_16x16x128_f8f6f4 a[180:183], v[152:159], v[40:47], a[180:183] cbsz:1 blgp:1\n" \
  "buffer_load_dwordx4 v[8:11], v165, s[36:39], s75 offen offset:2048\n" \
  "v_mfma_f32_16x16x128_f8f6f4 a[184:187], v[152:159], v[48:55], a[184:187] cbsz:1 blgp:1\n" \
  "v_mul_f32 v188, s78, v188\n" \
  "v_mul_f32 v189, s78, v189\n" \
  "v_mul_f32 v190, s78, v190\n" \
  "v_mul_f32 v191, s78, v191\n" \
  "s_bitcmp1_b32 s66, 28\n" \
  "s_cbranch_scc1 Lp10_%=\n" \
  "buffer_store_dwordx4 v[188:191], v167, s[40:43], s49 offen offset:0 nt\n" \
  "Lq10_%=:\n" \
  "v_mfma_f32_16x16x128_f8f6f4 a[188:191], v[152:159], v[56:63], a[188:191] cbsz:1 blgp:1\n" \
  "buffer_load_dwordx4 v[16:19], v166, s[36:39], s75 offen offset:0\n" \
  "s_cmp_eq_u32 s74, 6\n" \
  "s_cbranch_scc1 Lsuf1_0_6_%=\n" \
  "Ls1_7_%=:\n" \
  "s_waitcnt lgkmcnt(2)\n" \
  "v_mfma_f32_16x16x128_f8f6f4 a[192:195], v[128:135], v[32:39], a[192:195] cbsz:1 blgp:1\n" \
  "s_add_u32 m0, s69, 24576\n" \
  "ds_read_b128 v[144:147], v160 offset:61440\n" \
  "ds_read_b128 v[148:151], v160 offset:62464\n" \
  "v_mfma_f32_16x16x128_f8f6f4 a[196:199], v[128:135], v[40:47], a[196:199] cbsz:1 blgp:1\n" \
  "buffer_load_dwordx4 v165, s[36:39], s62 offen lds\n" \
  "v_mfma_f32_16x16x128_f8f6f4 a[200:203], v[128:135], v[48:55], a[200:203] cbsz:1 blgp:1\n" \
  "v_mul_f32 v204, s78, v204\n" \
  "v_mul_f32 v205, s78, v205\n" \
  "v_mul_f32 v206, s78, v206\n" \
  "v_mul_f32 v207, s78, v207\n" \
  "s_bitcmp1_b32 s66, 28\n" \
  "s_cbranch_scc1 Lp11_%=\n" \
  "buffer_store_dwordx4 v[204:207], v167, s[40:43], s49 offen offset:64 nt\n" \
  "Lq11_%=:\n" \
  "v_mfma_f32_16x16x128_f8f6f4 a[204:207], v[128:135], v[56:63], a[204:207] cbsz:1 blgp:1\n" \
  "buffer_load_dwordx4 v[24:27], v166, s[36:39], s75 offen offset:2048\n" \
  "s_and_b32 s73, s65, 0xf00001ff\n" \
  "s_waitcnt lgkmcnt(2)\n" \
  "v_mfma_f32_16x16x128_f8f6f4 a[208:211], v[136:143], v[32:39], a[208:211] cbsz:1 blgp:1\n" \
  "ds_read_b128 v[152:155], v160 offset:63488\n" \
  "ds_read_b128 v[156:159], v160 offset:64512\n" \
  "v_mfma_f32_16x16x128_f8f6f4 a[212:215], v[136:143], v[40:47], a[212:215] cbsz:1 blgp:1\n" \
  "buffer_load_dwordx4 v[20:23], v166, s[36:39], s75 offen offset:1024\n" \
  "v_mfma_f32_16x16x128_f8f6f4 a[216:219], v[136:143], v[48:55], a[216:219] cbsz:1 blgp:1\n" \
  "s_bfe_u32 m0, s65, 0x40018\n" \
  "s_nop 0\n" \
  "s_movreld_b32 s56, s77\n" \
  "v_mfma_f32_16x16x128_f8f6f4 a[220:223], v[136:143], v[56:63], a[220:223] cbsz:1 blgp:1\n" \
  "buffer_load_dwordx4 v[28:31], v166, s[36:39], s75 offen offset:3072\n" \
  "Ls1_8_%=:\n" \
  "s_cmp_eq_u32 s73, s88\n" \
  "s_cbranch_scc1 Lf2_%=\n" \
  "Lm2_%=:\n" \
  "s_bitcmp1_b32 s65, 31\n" \
  "s_cbranch_scc1 Ldone_%=\n" \
  "s_and_b32 s98, s65, 15\n" \
  "s_bitcmp1_b32 s65, 10\n" \
  "s_cselect_b32 s74, 9, 0\n" \
  "s_add_u32 s98, s98, s74\n" \
  "s_mul_i32 s98, s98, Lpre2_0_1_%=-Lpre2_0_0_%=\n" \
  "s_add_u32 s98, s98, Lpre2_0_0_%=-Lpc_%=\n" \
  "s_add_u32 s98, s98, s94\n" \
  "s_addc_u32 s99, s95, 0\n" \
  "s_bfe_u32 s74, s65, 0x40004\n" \
  "s_waitcnt vmcnt(36) lgkmcnt(0)\n" \
  "s_barrier\n" \
  "s_bitcmp1_b32 s65, 8\n" \
  "s_cbranch_scc0 Lz2_%=\n" \
  "v_mfma_f32_16x16x128_f8f6f4 a[224:227], v[144:151], v[32:39], a[224:227] cbsz:1 blgp:1\n" \
  "s_add_u32 m0, s69, 61440\n" \
  "ds_read_b128 v[128:131], v161 offset:0\n" \
  "ds_read_b128 v[132:135], v161 offset:1024\n" \
  "v_mfma_f32_16x16x128_f8f6f4 a[228:231], v[144:151], v[40:47], a[228:231] cbsz:1 blgp:1\n" \
  "buffer_load_dwordx4 v163, s[36:39], s63 offen lds\n" \
  "v_mfma_f32_16x16x128_f8f6f4 a[232:235], v[144:151], v[48:55], a[232:235] cbsz:1 blgp:1\n" \
  "v_mfma_f32_16x16x128_f8f6f4 a[236:239], v[144:151], v[56:63], a[236:239] cbsz:1 blgp:1\n" \
  "v_mfma_f32_16x16x128_f8f6f4 a[240:243], v[152:159], v[32:39], a[240:243] cbsz:1 blgp:1\n" \
  "ds_read_b128 v[136:139], v161 offset:2048\n" \
  "ds_read_b128 v[140:143], v161 offset:3072\n" \
  "v_mfma_f32_16x16x128_f8f6f4 a[244:247], v[152:159], v[40:47], a[244:247] cbsz:1 blgp:1\n" \
  "v_mfma_f32_16x16x128_f8f6f4 a[248:251], v[152:159], v[48:55], a[248:251] cbsz:1 blgp:1\n" \
  "v_mfma_f32_16x16x128_f8f6f4 a[252:255], v[152:159], v[56:63], a[252:255] cbsz:1 blgp:1\n" \
  "Lzr2_%=:\n" \
  "s_bitcmp1_b32 s65, 9\n" \
  "s_cbranch_scc1 Lpj2_%=\n" \
  "Ls2_1_%=:\n" \
  "s_waitcnt lgkmcnt(2)\n" \
  "v_mfma_f32_16x16x128_f8f6f4 a[0:3], v[128:135], v[64:71], a[0:3] cbsz:1 blgp:1\n" \
  "s_add_u32 m0, s69, 32768\n" \
  "ds_read_b128 v[144:147], v161 offset:4096\n" \
  "ds_read_b128 v[148:151], v161 offset:5120\n" \
  "v_mfma_f32_16x16x128_f8f6f4 a[4:7], v[128:135], v[72:79], a[4:7] cbsz:1 blgp:1\n" \
  "buffer_load_dwordx4 v163, s[36:39], s56 offen lds\n" \
  "v_mfma_f32_16x16x128_f8f6f4 a[8:11], v[128:135], v[80:87], a[8:11] cbsz:1 blgp:1\n" \
  "v_mfma_f32_16x16x128_f8f6f4 a[12:15], v[128:135], v[88:95], a[12:15] cbsz:1 blgp:1\n" \
  "s_waitcnt lgkmcnt(2)\n" \
  "v_mfma_f32_16x16x128_f8f6f4 a[16:19], v[136:143], v[64:71], a[16:19] cbsz:1 blgp:1\n" \
  "ds_read_b128 v[152:155], v161 offset:6144\n" \
  "ds_read_b128 v[156:159], v161 offset:7168\n" \
  "v_mfma_f32_16x16x128_f8f6f4 a[20:23], v[136:143], v[72:79], a[20:23] cbsz:1 blgp:1\n" \
  "v_mfma_f32_16x16x128_f8f6f4 a[24:27], v[136:143], v[80:87], a[24:27] cbsz:1 blgp:1\n" \
  "v_mfma_f32_16x16x128_f8f6f4 a[28:31], v[136:143], v[88:95], a[28:31] cbsz:1 blgp:1\n" \
  "s_cmp_eq_u32 s74, 1\n" \
  "s_cbranch_scc1 Lsuf2_0_1_%=\n" \
  "Ls2_2_%=:\n" \
  "s_waitcnt lgkmcnt(2)\n" \
  "v_mfma_f32_16x16x128_f8f6f4 a[32:35], v[144:151], v[64:71], a[32:35] cbsz:1 blgp:1\n" \
  "s_add_u32 m0, s69, 36864\n" \
  "ds_read_b128 v[128:131], v161 offset:8192\n" \
  "ds_read_b128 v[132:135], v161 offset:9216\n" \
  "v_mfma_f32_16x16x128_f8f6f4 a[36:39], v[144:151], v[72:79], a[36:39] cbsz:1 blgp:1\n" \
  "buffer_load_dwordx4 v163, s[36:39], s57 offen lds\n" \
  "v_mfma_f32_16x16x128_f8f6f4 a[40:43], v[144:151], v[80:87], a[40:43] cbsz:1 blgp:1\n" \
  "v_mfma_f32_16x16x128_f8f6f4 a[44:47], v[144:151], v[88:95], a[44:47] cbsz:1 blgp:1\n" \
  "s_waitcnt lgkmcnt(2)\n" \
  "v_mfma_f32_16x16x128_f8f6f4 a[48:51], v[152:159], v[64:71], a[48:51] cbsz:1 blgp:1\n" \
  "ds_read_b128 v[136:139], v161 offset:10240\n" \
  "ds_read_b128 v[140:143], v161 offset:11264\n" \
  "v_mfma_f32_16x16x128_f8f6f4 a[52:55], v[152:159], v[72:79], a[52:55] cbsz:1 blgp:1\n" \
  "v_mfma_f32_16x16x128_f8f6f4 a[56:59], v[152:159], v[80:87], a[56:59] cbsz:1 blgp:1\n" \
  "v_mfma_f32_16x16x128_f8f6f4 a[60:63], v[152:159], v[88:95], a[60:63] cbsz:1 blgp:1\n" \
  "s_cmp_eq_u32 s74, 2\n" \
  "s_cbranch_scc1 Lsuf2_0_2_%=\n" \
  "Ls2_3_%=:\n" \
  "s_waitcnt lgkmcnt(2)\n" \
  "v_mfma_f32_16x16x128_f8f6f4 a[64:67], v[128:135], v[64:71], a[64:67] cbsz:1 blgp:1\n" \
  "s_add_u32 m0, s69, 40960\n" \
  "ds_read_b128 v[144:147], v161 offset:12288\n" \
  "ds_read_b128 v[148:151], v161 offset:13312\n" \
  "v_mfma_f32_16x16x128_f8f6f4 a[68:71], v[128:135], v[72:79], a[68:71] cbsz:1 blgp:1\n" \
  "buffer_load_dwordx4 v163, s[36:39], s58 offen lds\n" \
  "v_mfma_f32_16x16x128_f8f6f4 a[72:75], v[128:135], v[80:87], a[72:75] cbsz:1 blgp:1\n" \
  "v_mfma_f32_16x16x128_f8f6f4 a[76:79], v[128:135], v[88:95], a[76:79] cbsz:1 blgp:1\n" \
  "s_waitcnt lgkmcnt(2)\n" \
  "v_mfma_f32_16x16x128_f8f6f4 a[80:83], v[136:143], v[64:71], a[80:83] cbsz:1 blgp:1\n" \
  "ds_read_b128 v[152:155], v161 offset:14336\n" \
  "ds_read_b128 v[156:159], v161 offset:15360\n" \
  "v_mfma_f32_16x16x128_f8f6f4 a[84:87], v[136:143], v[72:79], a[84:87] cbsz:1 blgp:1\n" \
  "v_mfma_f32_16x16x128_f8f6f4 a[88:91], v[136:143], v[80:87], a[88:91] cbsz:1 blgp:1\n" \
  "v_mfma_f32_16x16x128_f8f6f4 a[92:95], v[136:143], v[88:95], a[92:95] cbsz:1 blgp:1\n" \
  "s_cmp_eq_u32 s74, 3\n" \
  "s_cbranch_scc1 Lsuf2_0_3_%=\n" \
  "Ls2_4_%=:\n" \
  "s_waitcnt lgkmcnt(2)\n" \
  "v_mfma_f32_16x16x128_f8f6f4 a[96:99], v[144:151], v[64:71], a[96:99] cbsz:1 blgp:1\n" \
  "s_add_u32 m0, s69, 45056\n" \
  "ds_read_b128 v[128:131], v161 offset:16384\n" \
  "ds_read_b128 v[132:135], v161 offset:17408\n" \
  "v_mfma_f32_16x16x128_f8f6f4 a[100:103], v[144:151], v[72:79], a[100:103] cbsz:1 blgp:1\n" \
  "buffer_load_dwordx4 v163, s[36:39], s59 offen lds\n" \
  "v_mfma_f32_16x16x128_f8f6f4 a[104:107], v[144:151], v[80:87], a[104:107] cbsz:1 blgp:1\n" \
  "v_mfma_f32_16x16x128_f8f6f4 a[108:111], v[144:151], v[88:95], a[108:111] cbsz:1 blgp:1\n" \
  "s_waitcnt lgkmcnt(2)\n" \
  "v_mfma_f32_16x16x128_f8f6f4 a[112:115], v[152:159], v[64:71], a[112:115] cbsz:1 blgp:1\n" \
  "ds_read_b128 v[136:139], v161 offset:18432\n" \
  "ds_read_b128 v[140:143], v161 offset:19456\n" \
  "v_mfma_f32_16x16x128_f8f6f4 a[116:119], v[152:159], v[72:79], a[116:119] cbsz:1 blgp:1\n" \
  "v_mfma_f32_16x16x128_f8f6f4 a[120:123], v[152:159], v[80:87], a[120:123] cbsz:1 blgp:1\n" \
  "v_mfma_f32_16x16x128_f8f6f4 a[124:127], v[152:159], v[88:95], a[124:127] cbsz:1 blgp:1\n" \
  "s_cmp_eq_u32 s74, 4\n" \
  "s_cbranch_scc1 Lsuf2_0_4_%=\n" \
  "Ls2_5_%=:\n" \
  "s_waitcnt lgkmcnt(2)\n" \
  "v_mfma_f32_16x16x128_f8f6f4 a[128:131], v[128:135], v[64:71], a[128:131] cbsz:1 blgp:1\n" \
  "s_add_u32 m0, s69, 49152\n" \
  "ds_read_b128 v[144:147], v161 offset:20480\n" \
  "ds_read_b128 v[148:151], v161 offset:21504\n" \
  "v_mfma_f32_16x16x128_f8f6f4 a[132:135], v[128:135], v[72:79], a[132:135] cbsz:1 blgp:1\n" \
  "buffer_load_dwordx4 v163, s[36:39], s60 offen lds\n" \
  "v_mfma_f32_16x16x128_f8f6f4 a[136:139], v[128:135], v[80:87], a[136:139] cbsz:1 blgp:1\n" \
  "v_mfma_f32_16x16x128_f8f6f4 a[140:143], v[128:135], v[88:95], a[140:143] cbsz:1 blgp:1\n" \
  "buffer_load_dwordx4 v[36:39], v163, s[36:39], s75 offen offset:1024\n" \
  "s_waitcnt lgkmcnt(2)\n" \
  "v_mfma_f32_16x16x128_f8f6f4 a[144:147], v[136:143], v[64:71], a[144:147] cbsz:1 blgp:1\n" \
  "ds_read_b128 v[152:155], v161 offset:22528\n" \
  "ds_read_b128 v[156:159], v161 offset:23552\n" \
  "v_mfma_f32_16x16x128_f8f6f4 a[148:151], v[136:143], v[72:79], a[148:151] cbsz:1 blgp:1\n" \
  "buffer_load_dwordx4 v[32:35], v163, s[36:39], s75 offen offset:0\n" \
  "v_mfma_f32_16x16x128_f8f6f4 a[152:155], v[136:143], v[80:87], a[152:155] cbsz:1 blgp:1\n" \
  "v_mfma_f32_16x16x128_f8f6f4 a[156:159], v[136:143], v[88:95], a[156:159] cbsz:1 blgp:1\n" \
  "v_readlane_b32 s67, v175, s70\n" \
  "v_readlane_b32 s66, v178, s70\n" \
  "v_readlane_b32 s76, v169, s70\n" \
  "v_readlane_b32 s77, v172, s70\n" \
  "s_add_u32 s70, s70, 1\n" \
  "v_add_u32 v165, s76, v162\n" \
  "s_cmp_eq_u32 s74, 5\n" \
  "s_cbranch_scc1 Lsuf2_0_5_%=\n" \
  "Ls2_6_%=:\n" \
  "s_waitcnt lgkmcnt(2)\n" \
  "v_mfma_f32_16x16x128_f8f6f4 a[160:163], v[144:151], v[64:71], a[160:163] cbsz:1 blgp:1\n" \
  "s_add_u32 m0, s69, 53248\n" \
  "ds_read_b128 v[128:131], v161 offset:24576\n" \
  "ds_read_b128 v[132:135], v161 offset:25600\n" \
  "v_mfma_f32_16x16x128_f8f6f4 a[164:167], v[144:151], v[72:79], a[164:167] cbsz:1 blgp:1\n" \
  "buffer_load_dwordx4 v163, s[36:39], s61 offen lds\n" \
  "v_mfma_f32_16x16x128_f8f6f4 a[168:171], v[144:151], v[80:87], a[168:171] cbsz:1 blgp:1\n" \
  "s_bitcmp1_b32 s65, 23\n" \
  "s_cbranch_scc1 Lp12_%=\n" \
  "Lq12_%=:\n" \
  "v_mfma_f32_16x16x128_f8f6f4 a[172:175], v[144:151], v[88:95], a[172:175] cbsz:1 blgp:1\n" \
  "buffer_load_dwordx4 v[44:47], v163, s[36:39], s75 offen offset:3072\n" \
  "v_add_u32 v166, 4096, v165\n" \
  "s_cmp_gt_u32 s67, s68\n" \
  "s_cbranch_scc1 Lp13_%=\n" \
  "Lq13_%=:\n" \
  "s_waitcnt lgkmcnt(2)\n" \
  "v_mfma_f32_16x16x128_f8f6f4 a[176:179], v[152:159], v[64:71], a[176:179] cbsz:1 blgp:1\n" \
  "ds_read_b128 v[136:139], v161 offset:26624\n" \
  "ds_read_b128 v[140:143], v161 offset:27648\n" \
  "v_mfma_f32_16x16x128_f8f6f4 a[180:183], v[152:159], v[72:79], a[180:183] cbsz:1 blgp:1\n" \
  "buffer_load_dwordx4 v[40:43], v163, s[36:39], s75 offen offset:2048\n" \
  "v_mfma_f32_16x16x128_f8f6f4 a[184:187], v[152:159], v[80:87], a[184:187] cbsz:1 blgp:1\n" \
  "v_mul_f32 v192, s78, v192\n" \
  "v_mul_f32 v193, s78, v193\n" \
  "v_mul_f32 v194, s78, v194\n" \
  "v_mul_f32 v195, s78, v195\n" \
  "s_bitcmp1_b32 s65, 28\n" \
  "s_cbranch_scc1 Lp14_%=\n" \
  "buffer_store_dwordx4 v[192:195], v167, s[40:43], s50 offen offset:0 nt\n" \
  "Lq14_%=:\n" \
  "v_mfma_f32_16x16x128_f8f6f4 a[188:191], v[152:159], v[88:95], a[188:191] cbsz:1 blgp:1\n" \
  "buffer_load_dwordx4 v[48:51], v164, s[36:39], s75 offen offset:0\n" \
  "s_cmp_eq_u32 s74, 6\n" \
  "s_cbranch_scc1 Lsuf2_0_6_%=\n" \
  "Ls2_7_%=:\n" \
  "s_waitcnt lgkmcnt(2)\n" \
  "v_mfma_f32_16x16x128_f8f6f4 a[192:195], v[128:135], v[64:71], a[192:195] cbsz:1 blgp:1\n" \
  "s_add_u32 m0, s69, 57344\n" \
  "ds_read_b128 v[144:147], v161 offset:28672\n" \
  "ds_read_b128 v[148:151], v161 offset:29696\n" \
  "v_mfma_f32_16x16x128_f8f6f4 a[196:199], v[128:135], v[72:79], a[196:199] cbsz:1 blgp:1\n" \
  "buffer_load_dwordx4 v163, s[36:39], s62 offen lds\n" \
  "v_mfma_f32_16x16x128_f8f6f4 a[200:203], v[128:135], v[80:87], a[200:203] cbsz:1 blgp:1\n" \
  "v_mul_f32 v208, s78, v208\n" \
  "v_mul_f32 v209, s78, v209\n" \
  "v_mul_f32 v210, s78, v210\n" \
  "v_mul_f32 v211, s78, v211\n" \
  "s_bitcmp1_b32 s65, 28\n" \
  "s_cbranch_scc1 Lp15_%=\n" \
  "buffer_store_dwordx4 v[208:211], v167, s[40:43], s50 offen offset:64 nt\n" \
  "Lq15_%=:\n" \
  "v_mfma_f32_16x16x128_f8f6f4 a[204:207], v[128:135], v[88:95], a[204:207] cbsz:1 blgp:1\n" \
  "buffer_load_dwordx4 v[56:59], v164, s[36:39], s75 offen offset:2048\n" \
  "s_and_b32 s73, s66, 0xf00001ff\n" \
  "s_waitcnt lgkmcnt(2)\n" \
  "v_mfma_f32_16x16x128_f8f6f4 a[208:211], v[136:143], v[64:71], a[208:211] cbsz:1 blgp:1\n" \
  "ds_read_b128 v[152:155], v161 offset:30720\n" \
  "ds_read_b128 v[156:159], v161 offset:31744\n" \
  "v_mfma_f32_16x16x128_f8f6f4 a[212:215], v[136:143], v[72:79], a[212:215] cbsz:1 blgp:1\n" \
  "buffer_load_dwordx4 v[52:55], v164, s[36:39], s75 offen offset:1024\n" \
  "v_mfma_f32_16x16x128_f8f6f4 a[216:219], v[136:143], v[80:87], a[216:219] cbsz:1 blgp:1\n" \
  "s_bfe_u32 m0, s66, 0x40018\n" \
  "s_nop 0\n" \
  "s_movreld_b32 s56, s77\n" \
  "v_mfma_f32_16x16x128_f8f6f4 a[220:223], v[136:143], v[88:95], a[220:223] cbsz:1 blgp:1\n" \
  "buffer_load_dwordx4 v[60:63], v164, s[36:39], s75 offen offset:3072\n" \
  "Ls2_8_%=:\n" \
  "s_cmp_eq_u32 s73, s88\n" \
  "s_cbranch_scc1 Lf3_%=\n" \
  "Lm3_%=:\n" \
  "s_bitcmp1_b32 s66, 31\n" \
  "s_cbranch_scc1 Ldone_%=\n" \
  "s_and_b32 s98, s66, 15\n" \
  "s_bitcmp1_b32 s66, 10\n" \
  "s_cselect_b32 s74, 9, 0\n" \
  "s_add_u32 s98, s98, s74\n" \
  "s_mul_i32 s98, s98, Lpre3_0_1_%=-Lpre3_0_0_%=\n" \
  "s_add_u32 s98, s98, Lpre3_0_0_%=-Lpc_%=\n" \
  "s_add_u32 s98, s98, s94\n" \
  "s_addc_u32 s99, s95, 0\n" \
  "s_bfe_u32 s74, s66, 0x40004\n" \
  "s_waitcnt vmcnt(36) lgkmcnt(0)\n" \
  "s_barrier\n" \
  "s_bitcmp1_b32 s66, 8\n" \
  "s_cbranch_scc0 Lz3_%=\n" \
  "v_mfma_f32_16x16x128_f8f6f4 a[224:227], v[144:151], v[64:71], a[224:227] cbsz:1 blgp:1\n" \
  "s_add_u32 m0, s69, 94208\n" \
  "ds_read_b128 v[128:131], v161 offset:32768\n" \
  "ds_read_b128 v[132:135], v161 offset:33792\n" \
  "v_mfma_f32_16x16x128_f8f6f4 a[228:231], v[144:151], v[72:79], a[228:231] cbsz:1 blgp:1\n" \
  "buffer_load_dwordx4 v165, s[36:39], s63 offen lds\n" \
  "v_mfma_f32_16x16x128_f8f6f4 a[232:235], v[144:151], v[80:87], a[232:235] cbsz:1 blgp:1\n" \
  "v_mfma_f32_16x16x128_f8f6f4 a[236:239], v[144:151], v[88:95], a[236:239] cbsz:1 blgp:1\n" \
  "v_mfma_f32_16x16x128_f8f6f4 a[240:243], v[152:159], v[64:71], a[240:243] cbsz:1 blgp:1\n" \
  "ds_read_b128 v[136:139], v161 offset:34816\n" \
  "ds_read_b128 v[140:143], v161 offset:35840\n" \
  "v_mfma_f32_16x16x128_f8f6f4 a[244:247], v[152:159], v[72:79], a[244:247] cbsz:1 blgp:1\n" \
  "v_mfma_f32_16x16x128_f8f6f4 a[248:251], v[152:159], v[80:87], a[248:251] cbsz:1 blgp:1\n" \
  "v_mfma_f32_16x16x128_f8f6f4 a[252:255], v[152:159], v[88:95], a[252:255] cbsz:1 blgp:1\n" \
  "Lzr3_%=:\n" \
  "s_bitcmp1_b32 s66, 9\n" \
  "s_cbranch_scc1 Lpj3_%=\n" \
  "Ls3_1_%=:\n" \
  "s_waitcnt lgkmcnt(2)\n" \
  "v_mfma_f32_16x16x128_f8f6f4 a[0:3], v[128:135], v[96:103], a[0:3] cbsz:1 blgp:1\n" \
  "s_add_u32 m0, s69, 65536\n" \
  "ds_read_b128 v[144:147], v161 offset:36864\n" \
  "ds_read_b128 v[148:151], v161 offset:37888\n" \
  "v_mfma_f32_16x16x128_f8f6f4 a[4:7], v[128:135], v[104:111], a[4:7] cbsz:1 blgp:1\n" \
  "buffer_load_dwordx4 v165, s[36:39], s56 offen lds\n" \
  "v_mfma_f32_16x16x128_f8f6f4 a[8:11], v[128:135], v[112:119], a[8:11] cbsz:1 blgp:1\n" \
  "v_mfma_f32_16x16x128_f8f6f4 a[12:15], v[128:135], v[120:127], a[12:15] cbsz:1 blgp:1\n" \
  "s_waitcnt lgkmcnt(2)\n" \
  "v_mfma_f32_16x16x128_f8f6f4 a[16:19], v[136:143], v[96:103], a[16:19] cbsz:1 blgp:1\n" \
  "ds_read_b128 v[152:155], v161 offset:38912\n" \
  "ds_read_b128 v[156:159], v161 offset:39936\n" \
  "v_mfma_f32_16x16x128_f8f6f4 a[20:23], v[136:143], v[104:111], a[20:23] cbsz:1 blgp:1\n" \
  "v_mfma_f32_16x16x128_f8f6f4 a[24:27], v[136:143], v[112:119], a[24:27] cbsz:1 blgp:1\n" \
  "v_mfma_f32_16x16x128_f8f6f4 a[28:31], v[136:143], v[120:127], a[28:31] cbsz:1 blgp:1\n" \
  "s_cmp_eq_u32 s74, 1\n" \
  "s_cbranch_scc1 Lsuf3_0_1_%=\n" \
  "Ls3_2_%=:\n" \
  "s_waitcnt lgkmcnt(2)\n" \
  "v_mfma_f32_16x16x128_f8f6f4 a[32:35], v[144:151], v[96:103], a[32:35] cbsz:1 blgp:1\n" \
  "s_add_u32 m0, s69, 69632\n" \
  "ds_read_b128 v[128:131], v161 offset:40960\n" \
  "ds_read_b128 v[132:135], v161 offset:41984\n" \
  "v_mfma_f32_16x16x128_f8f6f4 a[36:39], v[144:151], v[104:111], a[36:39] cbsz:1 blgp:1\n" \
  "buffer_load_dwordx4 v165, s[36:39], s57 offen lds\n" \
  "v_mfma_f32_16x16x128_f8f6f4 a[40:43], v[144:151], v[112:119], a[40:43] cbsz:1 blgp:1\n" \
  "v_mfma_f32_16x16x128_f8f6f4 a[44:47], v[144:151], v[120:127], a[44:47] cbsz:1 blgp:1\n" \
  "s_waitcnt lgkmcnt(2)\n" \
  "v_mfma_f32_16x16x128_f8f6f4 a[48:51], v[152:159], v[96:103], a[48:51] cbsz:1 blgp:1\n" \
  "ds_read_b128 v[136:139], v161 offset:43008\n" \
  "ds_read_b128 v[140:143], v161 offset:44032\n" \
  "v_mfma_f32_16x16x128_f8f6f4 a[52:55], v[152:159], v[104:111], a[52:55] cbsz:1 blgp:1\n" \
  "v_mfma_f32_16x16x128_f8f6f4 a[56:59], v[152:159], v[112:119], a[56:59] cbsz:1 blgp:1\n" \
  "v_mfma_f32_16x16x128_f8f6f4 a[60:63], v[152:159], v[120:127], a[60:63] cbsz:1 blgp:1\n" \
  "s_cmp_eq_u32 s74, 2\n" \
  "s_cbranch_scc1 Lsuf3_0_2_%=\n" \
  "Ls3_3_%=:\n" \
  "s_waitcnt lgkmcnt(2)\n" \
  "v_mfma_f32_16x16x128_f8f6f4 a[64:67], v[128:135], v[96:103], a[64:67] cbsz:1 blgp:1\n" \
  "s_add_u32 m0, s69, 73728\n" \
  "ds_read_b128 v[144:147], v161 offset:45056\n" \
  "ds_read_b128 v[148:151], v161 offset:46080\n" \
  "v_mfma_f32_16x16x128_f8f6f4 a[68:71], v[128:135], v[104:111], a[68:71] cbsz:1 blgp:1\n" \
  "buffer_load_dwordx4 v165, s[36:39], s58 offen lds\n" \
  "v_mfma_f32_16x16x128_f8f6f4 a[72:75], v[128:135], v[112:119], a[72:75] cbsz:1 blgp:1\n" \
  "v_mfma_f32_16x16x128_f8f6f4 a[76:79], v[128:135], v[120:127], a[76:79] cbsz:1 blgp:1\n" \
  "s_waitcnt lgkmcnt(2)\n" \
  "v_mfma_f32_16x16x128_f8f6f4 a[80:83], v[136:143], v[96:103], a[80:83] cbsz:1 blgp:1\n" \
  "ds_read_b128 v[152:155], v161 offset:47104\n" \
  "ds_read_b128 v[156:159], v161 offset:48128\n" \
  "v_mfma_f32_16x16x128_f8f6f4 a[84:87], v[136:143], v[104:111], a[84:87] cbsz:1 blgp:1\n" \
  "v_mfma_f32_16x16x128_f8f6f4 a[88:91], v[136:143], v[112:119], a[88:91] cbsz:1 blgp:1\n" \
  "v_mfma_f32_16x16x128_f8f6f4 a[92:95], v[136:143], v[120:127], a[92:95] cbsz:1 blgp:1\n" \
  "s_cmp_eq_u32 s74, 3\n" \
  "s_cbranch_scc1 Lsuf3_0_3_%=\n" \
  "Ls3_4_%=:\n" \
  "s_waitcnt lgkmcnt(2)\n" \
  "v_mfma_f32_16x16x128_f8f6f4 a[96:99], v[144:151], v[96:103], a[96:99] cbsz:1 blgp:1\n" \
  "s_add_u32 m0, s69, 77824\n" \
  "ds_read_b128 v[128:131], v161 offset:49152\n" \
  "ds_read_b128 v[132:135], v161 offset:50176\n" \
  "v_mfma_f32_16x16x128_f8f6f4 a[100:103], v[144:151], v[104:111], a[100:103] cbsz:1 blgp:1\n" \
  "buffer_load_dwordx4 v165, s[36:39], s59 offen lds\n" \
  "v_mfma_f32_16x16x128_f8f6f4 a[104:107], v[144:151], v[112:119], a[104:107] cbsz:1 blgp:1\n" \
  "v_mfma_f32_16x16x128_f8f6f4 a[108:111], v[144:151], v[120:127], a[108:111] cbsz:1 blgp:1\n" \
  "s_waitcnt lgkmcnt(2)\n" \
  "v_mfma_f32_16x16x128_f8f6f4 a[112:115], v[152:159], v[96:103], a[112:115] cbsz:1 blgp:1\n" \
  "ds_read_b128 v[136:139], v161 offset:51200\n" \
  "ds_read_b128 v[140:143], v161 offset:52224\n" \
  "v_mfma_f32_16x16x128_f8f6f4 a[116:119], v[152:159], v[104:111], a[116:119] cbsz:1 blgp:1\n" \
  "v_mfma_f32_16x16x128_f8f6f4 a[120:123], v[152:159], v[112:119], a[120:123] cbsz:1 blgp:1\n" \
  "v_mfma_f32_16x16x128_f8f6f4 a[124:127], v[152:159], v[120:127], a[124:127] cbsz:1 blgp:1\n" \
  "s_cmp_eq_u32 s74, 4\n" \
  "s_cbranch_scc1 Lsuf3_0_4_%=\n" \
  "Ls3_5_%=:\n" \
  "s_waitcnt lgkmcnt(2)\n" \
  "v_mfma_f32_16x16x128_f8f6f4 a[128:131], v[128:135], v[96:103], a[128:131] cbsz:1 blgp:1\n" \
  "s_add_u32 m0, s69, 81920\n" \
  "ds_read_b128 v[144:147], v161 offset:53248\n" \
  "ds_read_b128 v[148:151], v161 offset:54272\n" \
  "v_mfma_f32_16x16x128_f8f6f4 a[132:135], v[128:135], v[104:111], a[132:135] cbsz:1 blgp:1\n" \
  "buffer_load_dwordx4 v165, s[36:39], s60 offen lds\n" \
  "v_mfma_f32_16x16x128_f8f6f4 a[136:139], v[128:135], v[112:119], a[136:139] cbsz:1 blgp:1\n" \
  "v_mfma_f32_16x16x128_f8f6f4 a[140:143], v[128:135], v[120:127], a[140:143] cbsz:1 blgp:1\n" \
  "buffer_load_dwordx4 v[68:71], v165, s[36:39], s75 offen offset:1024\n" \
  "s_waitcnt lgkmcnt(2)\n" \
  "v_mfma_f32_16x16x128_f8f6f4 a[144:147], v[136:143], v[96:103], a[144:147] cbsz:1 blgp:1\n" \
  "ds_read_b128 v[152:155], v161 offset:55296\n" \
  "ds_read_b128 v[156:159], v161 offset:56320\n" \
  "v_mfma_f32_16x16x128_f8f6f4 a[148:151], v[136:143], v[104:111], a[148:151] cbsz:1 blgp:1\n" \
  "buffer_load_dwordx4 v[64:67], v165, s[36:39], s75 offen offset:0\n" \
  "v_mfma_f32_16x16x128_f8f6f4 a[152:155], v[136:143], v[112:119], a[152:155] cbsz:1 blgp:1\n" \
  "v_mfma_f32_16x16x128_f8f6f4 a[156:159], v[136:143], v[120:127], a[156:159] cbsz:1 blgp:1\n" \
  "v_readlane_b32 s67, v175, s70\n" \
  "v_readlane_b32 s65, v178, s70\n" \
  "v_readlane_b32 s76, v169, s70\n" \
  "v_readlane_b32 s77, v172, s70\n" \
  "v_readlane_b32 s71, v181, s70\n" \
  "s_add_u32 s70, s70, 1\n" \
  "v_add_u32 v163, s76, v162\n" \
  "s_cmp_eq_u32 s74, 5\n" \
  "s_cbranch_scc1 Lsuf3_0_5_%=\n" \
  "Ls3_6_%=:\n" \
  "s_waitcnt lgkmcnt(2)\n" \
  "v_mfma_f32_16x16x128_f8f6f4 a[160:163], v[144:151], v[96:103], a[160:163] cbsz:1 blgp:1\n" \
  "s_add_u32 m0, s69, 86016\n" \
  "ds_read_b128 v[128:131], v161 offset:57344\n" \
  "ds_read_b128 v[132:135], v161 offset:58368\n" \
  "v_mfma_f32_16x16x128_f8f6f4 a[164:167], v[144:151], v[104:111], a[164:167] cbsz:1 blgp:1\n" \
  "buffer_load_dwordx4 v165, s[36:39], s61 offen lds\n" \
  "v_mfma_f32_16x16x128_f8f6f4 a[168:171], v[144:151], v[112:119], a[168:171] cbsz:1 blgp:1\n" \
  "v_mfma_f32_16x16x128_f8f6f4 a[172:175], v[144:151], v[120:127], a[172:175] cbsz:1 blgp:1\n" \
  "buffer_load_dwordx4 v[76:79], v165, s[36:39], s75 offen offset:3072\n" \
  "v_add_u32 v164, 4096, v163\n" \
  "s_cmp_gt_u32 s67, s68\n" \
  "s_cbranch_scc1 Lp16_%=\n" \
  "Lq16_%=:\n" \
  "s_waitcnt lgkmcnt(2)\n" \
  "v_mfma_f32_16x16x128_f8f6f4 a[176:179], v[152:159], v[96:103], a[176:179] cbsz:1 blgp:1\n" \
  "ds_read_b128 v[136:139], v161 offset:59392\n" \
  "ds_read_b128 v[140:143], v161 offset:60416\n" \
  "v_mfma_f32_16x16x128_f8f6f4 a[180:183], v[152:159], v[104:111], a[180:183] cbsz:1 blgp:1\n" \
  "buffer_load_dwordx4 v[72:75], v165, s[36:39], s75 offen offset:2048\n" \
  "v_mfma_f32_16x16x128_f8f6f4 a[184:187], v[152:159], v[112:119], a[184:187] cbsz:1 blgp:1\n" \
  "v_mul_f32 v196, s78, v196\n" \
  "v_mul_f32 v197, s78, v197\n" \
  "v_mul_f32 v198, s78, v198\n" \
  "v_mul_f32 v199, s78, v199\n" \
  "s_bitcmp1_b32 s66, 28\n" \
  "s_cbranch_scc1 Lp17_%=\n" \
  "buffer_store_dwordx4 v[196:199], v167, s[40:43], s51 offen offset:0 nt\n" \
  "Lq17_%=:\n" \
  "v_mfma_f32_16x16x128_f8f6f4 a[188:191], v[152:159], v[120:127], a[188:191] cbsz:1 blgp:1\n" \
  "buffer_load_dwordx4 v[80:83], v166, s[36:39], s75 offen offset:0\n" \
  "s_bfe_u32 s96, s65, 0x40014\n" \
  "s_mul_i32 s96, s96, Le0_1_%=-Le0_0_%=\n" \
  "s_add_u32 s96, s96, Le0_0_%=-Lpc_%=\n" \
  "s_add_u32 s96, s96, s94\n" \
  "s_addc_u32 s97, s95, 0\n" \
  "s_cmp_eq_u32 s74, 6\n" \
  "s_cbranch_scc1 Lsuf3_0_6_%=\n" \
  "Ls3_7_%=:\n" \
  "s_waitcnt lgkmcnt(2)\n" \
  "v_mfma_f32_16x16x128_f8f6f4 a[192:195], v[128:135], v[96:103], a[192:195] cbsz:1 blgp:1\n" \
  "s_add_u32 m0, s69, 90112\n" \
  "ds_read_b128 v[144:147], v161 offset:61440\n" \
  "ds_read_b128 v[148:151], v161 offset:62464\n" \
  "v_mfma_f32_16x16x128_f8f6f4 a[196:199], v[128:135], v[104:111], a[196:199] cbsz:1 blgp:1\n" \
  "buffer_load_dwordx4 v165, s[36:39], s62 offen lds\n" \
  "v_mfma_f32_16x16x128_f8f6f4 a[200:203], v[128:135], v[112:119], a[200:203] cbsz:1 blgp:1\n" \
  "v_mul_f32 v212, s78, v212\n" \
  "v_mul_f32 v213, s78, v213\n" \
  "v_mul_f32 v214, s78, v214\n" \
  "v_mul_f32 v215, s78, v215\n" \
  "s_bitcmp1_b32 s66, 28\n" \
  "s_cbranch_scc1 Lp18_%=\n" \
  "buffer_store_dwordx4 v[212:215], v167, s[40:43], s51 offen offset:64 nt\n" \
  "Lq18_%=:\n" \
  "v_mfma_f32_16x16x128_f8f6f4 a[204:207], v[128:135], v[120:127], a[204:207] cbsz:1 blgp:1\n" \
  "buffer_load_dwordx4 v[88:91], v166, s[36:39], s75 offen offset:2048\n" \
  "s_and_b32 s73, s65, 0xf00001ff\n" \
  "s_waitcnt lgkmcnt(2)\n" \
  "v_mfma_f32_16x16x128_f8f6f4 a[208:211], v[136:143], v[96:103], a[208:211] cbsz:1 blgp:1\n" \
  "ds_read_b128 v[152:155], v161 offset:63488\n" \
  "ds_read_b128 v[156:159], v161 offset:64512\n" \
  "v_mfma_f32_16x16x128_f8f6f4 a[212:215], v[136:143], v[104:111], a[212:215] cbsz:1 blgp:1\n" \
  "buffer_load_dwordx4 v[84:87], v166, s[36:39], s75 offen offset:1024\n" \
  "v_mfma_f32_16x16x128_f8f6f4 a[216:219], v[136:143], v[112:119], a[216:219] cbsz:1 blgp:1\n" \
  "s_bfe_u32 m0, s65, 0x40018\n" \
  "s_nop 0\n" \
  "s_movreld_b32 s56, s77\n" \
  "v_mfma_f32_16x16x128_f8f6f4 a[220:223], v[136:143], v[120:127], a[220:223] cbsz:1 blgp:1\n" \
  "buffer_load_dwordx4 v[92:95], v166, s[36:39], s75 offen offset:3072\n" \
  "Ls3_8_%=:\n" \
  "s_cmp_eq_u32 s73, s88\n" \
  "s_cbranch_scc0 Lm0_%=\n" \
  "s_setpc_b64 s[96:97]\n" \
  ".p2align 6\n" \
  "Le0_0_%=:\n" \
  "s_waitcnt vmcnt(36) lgkmcnt(0)\n" \
  "s_barrier\n" \
  "s_waitcnt lgkmcnt(2)\n" \
  "v_accvgpr_read_b32 v184, a224\n" \
  "v_accvgpr_read_b32 v185, a225\n" \
  "v_accvgpr_read_b32 v186, a226\n" \
  "v_accvgpr_read_b32 v187, a227\n" \
  "v_mfma_f32_16x16x128_f8f6f4 a[224:227], v[144:151], v[96:103], 0 cbsz:1 blgp:1\n" \
  "s_add_u32 m0, s69, 126976\n" \
  "ds_read_b128 v[128:131], v160 offset:0\n" \
  "ds_read_b128 v[132:135], v160 offset:1024\n" \
  "v_accvgpr_read_b32 v188, a228\n" \
  "v_accvgpr_read_b32 v189, a229\n" \
  "v_accvgpr_read_b32 v190, a230\n" \
  "v_accvgpr_read_b32 v191, a231\n" \
  "v_mfma_f32_16x16x128_f8f6f4 a[228:231], v[144:151], v[104:111], 0 cbsz:1 blgp:1\n" \
  "buffer_load_dwordx4 v163, s[36:39], s63 offen lds\n" \
  "v_accvgpr_read_b32 v192, a232\n" \
  "v_accvgpr_read_b32 v193, a233\n" \
  "v_accvgpr_read_b32 v194, a234\n" \
  "v_accvgpr_read_b32 v195, a235\n" \
  "v_mfma_f32_16x16x128_f8f6f4 a[232:235], v[144:151], v[112:119], 0 cbsz:1 blgp:1\n" \
  "v_readfirstlane_b32 s89, v218\n" \
  "v_accvgpr_read_b32 v196, a236\n" \
  "v_accvgpr_read_b32 v197, a237\n" \
  "v_accvgpr_read_b32 v198, a238\n" \
  "v_accvgpr_read_b32 v199, a239\n" \
  "v_mfma_f32_16x16x128_f8f6f4 a[236:239], v[144:151], v[120:127], 0 cbsz:1 blgp:1\n" \
  "s_bitcmp1_b32 s65, 18\n" \
  "s_cselect_b32 s42, s72, 0\n" \
  "s_mov_b32 s48, s71\n" \
  "s_waitcnt lgkmcnt(2)\n" \
  "v_accvgpr_read_b32 v200, a240\n" \
  "v_accvgpr_read_b32 v201, a241\n" \
  "v_accvgpr_read_b32 v202, a242\n" \
  "v_accvgpr_read_b32 v203, a243\n" \
  "v_mfma_f32_16x16x128_f8f6f4 a[240:243], v[152:159], v[96:103], 0 cbsz:1 blgp:1\n" \
  "ds_read_b128 v[136:139], v160 offset:2048\n" \
  "ds_read_b128 v[140:143], v160 offset:3072\n" \
  "v_accvgpr_read_b32 v204, a244\n" \
  "v_accvgpr_read_b32 v205, a245\n" \
  "v_accvgpr_read_b32 v206, a246\n" \
  "v_accvgpr_read_b32 v207, a247\n" \
  "v_mfma_f32_16x16x128_f8f6f4 a[244:247], v[152:159], v[104:111], 0 cbsz:1 blgp:1\n" \
  "v_accvgpr_read_b32 v208, a248\n" \
  "v_accvgpr_read_b32 v209, a249\n" \
  "v_accvgpr_read_b32 v210, a250\n" \
  "v_accvgpr_read_b32 v211, a251\n" \
  "v_mfma_f32_16x16x128_f8f6f4 a[248:251], v[152:159], v[112:119], 0 cbsz:1 blgp:1\n" \
  "v_accvgpr_read_b32 v212, a252\n" \
  "v_accvgpr_read_b32 v213, a253\n" \
  "v_accvgpr_read_b32 v214, a254\n" \
  "v_accvgpr_read_b32 v215, a255\n" \
  "v_mfma_f32_16x16x128_f8f6f4 a[252:255], v[152:159], v[120:127], 0 cbsz:1 blgp:1\n" \
  "s_add_u32 s49, s71, 1048576\n" \
  "s_add_u32 s50, s71, 2097152\n" \
  "s_add_u32 s51, s71, 3145728\n" \
  "s_waitcnt lgkmcnt(2)\n" \
  "v_mfma_f32_16x16x128_f8f6f4 a[0:3], v[128:135], v[0:7], a[0:3] cbsz:1 blgp:1\n" \
  "s_add_u32 m0, s69, 98304\n" \
  "ds_read_b128 v[144:147], v160 offset:4096\n" \
  "ds_read_b128 v[148:151], v160 offset:5120\n" \
  "v_mfma_f32_16x16x128_f8f6f4 a[4:7], v[128:135], v[8:15], a[4:7] cbsz:1 blgp:1\n" \
  "buffer_load_dwordx4 v163, s[36:39], s56 offen lds\n" \
  "v_mfma_f32_16x16x128_f8f6f4 a[8:11], v[128:135], v[16:23], a[8:11] cbsz:1 blgp:1\n" \
  "v_mfma_f32_16x16x128_f8f6f4 a[12:15], v[128:135], v[24:31], a[12:15] cbsz:1 blgp:1\n" \
  "s_cmp_eq_u32 s70, 64\n" \
  "s_cbranch_scc1 Lp19_%=\n" \
  "Lq19_%=:\n" \
  "s_waitcnt lgkmcnt(2)\n" \
  "v_mfma_f32_16x16x128_f8f6f4 a[16:19], v[136:143], v[0:7], a[16:19] cbsz:1 blgp:1\n" \
  "ds_read_b128 v[152:155], v160 offset:6144\n" \
  "ds_read_b128 v[156:159], v160 offset:7168\n" \
  "v_mfma_f32_16x16x128_f8f6f4 a[20:23], v[136:143], v[8:15], a[20:23] cbsz:1 blgp:1\n" \
  "buffer_load_dwordx4 v[96:99], v163, s[36:39], s75 offen offset:0\n" \
  "v_mfma_f32_16x16x128_f8f6f4 a[24:27], v[136:143], v[16:23], a[24:27] cbsz:1 blgp:1\n" \
  "s_max_u32 s68, s68, s89\n" \
  "v_mfma_f32_16x16x128_f8f6f4 a[28:31], v[136:143], v[24:31], a[28:31] cbsz:1 blgp:1\n" \
  "v_readlane_b32 s67, v175, s70\n" \
  "v_readlane_b32 s66, v178, s70\n" \
  "s_waitcnt lgkmcnt(2)\n" \
  "v_mfma_f32_16x16x128_f8f6f4 a[32:35], v[144:151], v[0:7], a[32:35] cbsz:1 blgp:1\n" \
  "s_add_u32 m0, s69, 102400\n" \
  "ds_read_b128 v[128:131], v160 offset:8192\n" \
  "ds_read_b128 v[132:135], v160 offset:9216\n" \
  "v_mfma_f32_16x16x128_f8f6f4 a[36:39], v[144:151], v[8:15], a[36:39] cbsz:1 blgp:1\n" \
  "buffer_load_dwordx4 v163, s[36:39], s57 offen lds\n" \
  "buffer_load_dwordx4 v[124:127], v164, s[36:39], s75 offen offset:3072\n" \
  "v_mfma_f32_16x16x128_f8f6f4 a[40:43], v[144:151], v[16:23], a[40:43] cbsz:1 blgp:1\n" \
  "v_mfma_f32_16x16x128_f8f6f4 a[44:47], v[144:151], v[24:31], a[44:47] cbsz:1 blgp:1\n" \
  "v_readlane_b32 s76, v169, s70\n" \
  "v_readlane_b32 s77, v172, s70\n" \
  "s_waitcnt lgkmcnt(2)\n" \
  "v_mfma_f32_16x16x128_f8f6f4 a[48:51], v[152:159], v[0:7], a[48:51] cbsz:1 blgp:1\n" \
  "ds_read_b128 v[136:139], v160 offset:10240\n" \
  "ds_read_b128 v[140:143], v160 offset:11264\n" \
  "v_mfma_f32_16x16x128_f8f6f4 a[52:55], v[152:159], v[8:15], a[52:55] cbsz:1 blgp:1\n" \
  "buffer_load_dwordx4 v[100:103], v163, s[36:39], s75 offen offset:1024\n" \
  "v_mfma_f32_16x16x128_f8f6f4 a[56:59], v[152:159], v[16:23], a[56:59] cbsz:1 blgp:1\n" \
  "v_mfma_f32_16x16x128_f8f6f4 a[60:63], v[152:159], v[24:31], a[60:63] cbsz:1 blgp:1\n" \
  "s_add_u32 s70, s70, 1\n" \
  "v_add_u32 v165, s76, v162\n" \
  "s_waitcnt lgkmcnt(2)\n" \
  "v_mfma_f32_16x16x128_f8f6f4 a[64:67], v[128:135], v[0:7], a[64:67] cbsz:1 blgp:1\n" \
  "s_add_u32 m0, s69, 106496\n" \
  "ds_read_b128 v[144:147], v160 offset:12288\n" \
  "ds_read_b128 v[148:151], v160 offset:13312\n" \
  "v_mfma_f32_16x16x128_f8f6f4 a[68:71], v[128:135], v[8:15], a[68:71] cbsz:1 blgp:1\n" \
  "buffer_load_dwordx4 v163, s[36:39], s58 offen lds\n" \
  "v_mfma_f32_16x16x128_f8f6f4 a[72:75], v[128:135], v[16:23], a[72:75] cbsz:1 blgp:1\n" \
  "v_mfma_f32_16x16x128_f8f6f4 a[76:79], v[128:135], v[24:31], a[76:79] cbsz:1 blgp:1\n" \
  "v_add_u32 v166, 4096, v165\n" \
  "s_cmp_gt_u32 s67, s68\n" \
  "s_cbranch_scc1 Lp20_%=\n" \
  "Lq20_%=:\n" \
  "s_waitcnt lgkmcnt(2)\n" \
  "v_mfma_f32_16x16x128_f8f6f4 a[80:83], v[136:143], v[0:7], a[80:83] cbsz:1 blgp:1\n" \
  "ds_read_b128 v[152:155], v160 offset:14336\n" \
  "ds_read_b128 v[156:159], v160 offset:15360\n" \
  "v_mfma_f32_16x16x128_f8f6f4 a[84:87], v[136:143], v[8:15], a[84:87] cbsz:1 blgp:1\n" \
  "buffer_load_dwordx4 v[104:107], v163, s[36:39], s75 offen offset:2048\n" \
  "v_mfma_f32_16x16x128_f8f6f4 a[88:91], v[136:143], v[16:23], a[88:91] cbsz:1 blgp:1\n" \
  "v_mfma_f32_16x16x128_f8f6f4 a[92:95], v[136:143], v[24:31], a[92:95] cbsz:1 blgp:1\n" \
  "s_waitcnt lgkmcnt(2)\n" \
  "v_mfma_f32_16x16x128_f8f6f4 a[96:99], v[144:151], v[0:7], a[96:99] cbsz:1 blgp:1\n" \
  "s_add_u32 m0, s69, 110592\n" \
  "ds_read_b128 v[128:131], v160 offset:16384\n" \
  "ds_read_b128 v[132:135], v160 offset:17408\n" \
  "v_mfma_f32_16x16x128_f8f6f4 a[100:103], v[144:151], v[8:15], a[100:103] cbsz:1 blgp:1\n" \
  "buffer_load_dwordx4 v163, s[36:39], s59 offen lds\n" \
  "v_mfma_f32_16x16x128_f8f6f4 a[104:107], v[144:151], v[16:23], a[104:107] cbsz:1 blgp:1\n" \
  "v_mfma_f32_16x16x128_f8f6f4 a[108:111], v[144:151], v[24:31], a[108:111] cbsz:1 blgp:1\n" \
  "s_waitcnt lgkmcnt(2)\n" \
  "v_mfma_f32_16x16x128_f8f6f4 a[112:115], v[152:159], v[0:7], a[112:115] cbsz:1 blgp:1\n" \
  "ds_read_b128 v[136:139], v160 offset:18432\n" \
  "ds_read_b128 v[140:143], v160 offset:19456\n" \
  "v_mfma_f32_16x16x128_f8f6f4 a[116:119], v[152:159], v[8:15], a[116:119] cbsz:1 blgp:1\n" \
  "buffer_load_dwordx4 v[108:111], v163, s[36:39], s75 offen offset:3072\n" \
  "v_mfma_f32_16x16x128_f8f6f4 a[120:123], v[152:159], v[16:23], a[120:123] cbsz:1 blgp:1\n" \
  "v_mfma_f32_16x16x128_f8f6f4 a[124:127], v[152:159], v[24:31], a[124:127] cbsz:1 blgp:1\n" \
  "s_waitcnt lgkmcnt(2)\n" \
  "v_mfma_f32_16x16x128_f8f6f4 a[128:131], v[128:135], v[0:7], a[128:131] cbsz:1 blgp:1\n" \
  "s_add_u32 m0, s69, 114688\n" \
  "ds_read_b128 v[144:147], v160 offset:20480\n" \
  "ds_read_b128 v[148:151], v160 offset:21504\n" \
  "v_mfma_f32_16x16x128_f8f6f4 a[132:135], v[128:135], v[8:15], a[132:135] cbsz:1 blgp:1\n" \
  "buffer_load_dwordx4 v163, s[36:39], s60 offen lds\n" \
  "v_mfma_f32_16x16x128_f8f6f4 a[136:139], v[128:135], v[16:23], a[136:139] cbsz:1 blgp:1\n" \
  "v_mfma_f32_16x16x128_f8f6f4 a[140:143], v[128:135], v[24:31], a[140:143] cbsz:1 blgp:1\n" \
  "s_waitcnt lgkmcnt(2)\n" \
  "v_mfma_f32_16x16x128_f8f6f4 a[144:147], v[136:143], v[0:7], a[144:147] cbsz:1 blgp:1\n" \
  "ds_read_b128 v[152:155], v160 offset:22528\n" \
  "ds_read_b128 v[156:159], v160 offset:23552\n" \
  "v_mfma_f32_16x16x128_f8f6f4 a[148:151], v[136:143], v[8:15], a[148:151] cbsz:1 blgp:1\n" \
  "buffer_load_dwordx4 v[112:115], v164, s[36:39], s75 offen offset:0\n" \
  "v_mfma_f32_16x16x128_f8f6f4 a[152:155], v[136:143], v[16:23], a[152:155] cbsz:1 blgp:1\n" \
  "v_mfma_f32_16x16x128_f8f6f4 a[156:159], v[136:143], v[24:31], a[156:159] cbsz:1 blgp:1\n" \
  "s_waitcnt lgkmcnt(2)\n" \
  "v_mfma_f32_16x16x128_f8f6f4 a[160:163], v[144:151], v[0:7], a[160:163] cbsz:1 blgp:1\n" \
  "s_add_u32 m0, s69, 118784\n" \
  "ds_read_b128 v[128:131], v160 offset:24576\n" \
  "ds_read_b128 v[132:135], v160 offset:25600\n" \
  "v_mfma_f32_16x16x128_f8f6f4 a[164:167], v[144:151], v[8:15], a[164:167] cbsz:1 blgp:1\n" \
  "buffer_load_dwordx4 v163, s[36:39], s61 offen lds\n" \
  "v_mfma_f32_16x16x128_f8f6f4 a[168:171], v[144:151], v[16:23], a[168:171] cbsz:1 blgp:1\n" \
  "v_mfma_f32_16x16x128_f8f6f4 a[172:175], v[144:151], v[24:31], a[172:175] cbsz:1 blgp:1\n" \
  "s_and_b32 s73, s66, 0xf00001ff\n" \
  "s_waitcnt lgkmcnt(2)\n" \
  "v_mfma_f32_16x16x128_f8f6f4 a[176:179], v[152:159], v[0:7], a[176:179] cbsz:1 blgp:1\n" \
  "ds_read_b128 v[136:139], v160 offset:26624\n" \
  "ds_read_b128 v[140:143], v160 offset:27648\n" \
  "v_mfma_f32_16x16x128_f8f6f4 a[180:183], v[152:159], v[8:15], a[180:183] cbsz:1 blgp:1\n" \
  "buffer_load_dwordx4 v[116:119], v164, s[36:39], s75 offen offset:1024\n" \
  "v_mfma_f32_16x16x128_f8f6f4 a[184:187], v[152:159], v[16:23], a[184:187] cbsz:1 blgp:1\n" \
  "v_mfma_f32_16x16x128_f8f6f4 a[188:191], v[152:159], v[24:31], a[188:191] cbsz:1 blgp:1\n" \
  "s_waitcnt lgkmcnt(2)\n" \
  "v_mfma_f32_16x16x128_f8f6f4 a[192:195], v[128:135], v[0:7], a[192:195] cbsz:1 blgp:1\n" \
  "s_add_u32 m0, s69, 122880\n" \
  "ds_read_b128 v[144:147], v160 offset:28672\n" \
  "ds_read_b128 v[148:151], v160 offset:29696\n" \
  "v_mfma_f32_16x16x128_f8f6f4 a[196:199], v[128:135], v[8:15], a[196:199] cbsz:1 blgp:1\n" \
  "buffer_load_dwordx4 v163, s[36:39], s62 offen lds\n" \
  "v_mfma_f32_16x16x128_f8f6f4 a[200:203], v[128:135], v[16:23], a[200:203] cbsz:1 blgp:1\n" \
  "v_mfma_f32_16x16x128_f8f6f4 a[204:207], v[128:135], v[24:31], a[204:207] cbsz:1 blgp:1\n" \
  "s_waitcnt lgkmcnt(2)\n" \
  "v_mfma_f32_16x16x128_f8f6f4 a[208:211], v[136:143], v[0:7], a[208:211] cbsz:1 blgp:1\n" \
  "v_mul_f32 v184, s78, v184\n" \
  "v_mul_f32 v185, s78, v185\n" \
  "v_mul_f32 v186, s78, v186\n" \
  "v_mul_f32 v187, s78, v187\n" \
  "ds_read_b128 v[152:155], v160 offset:30720\n" \
  "ds_read_b128 v[156:159], v160 offset:31744\n" \
  "v_mfma_f32_16x16x128_f8f6f4 a[212:215], v[136:143], v[8:15], a[212:215] cbsz:1 blgp:1\n" \
  "buffer_load_dwordx4 v[120:123], v164, s[36:39], s75 offen offset:2048\n" \
  "v_mfma_f32_16x16x128_f8f6f4 a[216:219], v[136:143], v[16:23], a[216:219] cbsz:1 blgp:1\n" \
  "buffer_store_dwordx4 v[184:187], v167, s[40:43], s48 offen offset:0 nt\n" \
  "v_mul_f32 v200, s78, v200\n" \
  "v_mul_f32 v201, s78, v201\n" \
  "v_mul_f32 v202, s78, v202\n" \
  "v_mul_f32 v203, s78, v203\n" \
  "v_mfma_f32_16x16x128_f8f6f4 a[220:223], v[136:143], v[24:31], a[220:223] cbsz:1 blgp:1\n" \
  "buffer_store_dwordx4 v[200:203], v167, s[40:43], s48 offen offset:64 nt\n" \
  "s_bfe_u32 m0, s66, 0x40018\n" \
  "s_nop 0\n" \
  "s_movreld_b32 s56, s77\n" \
  "s_cmp_eq_u32 s73, s88\n" \
  "s_cbranch_scc0 Lm1_%=\n" \
  "s_branch Lf1_%=\n" \
  ".p2align 6\n" \
  "Le0_1_%=:\n" \
  "s_waitcnt vmcnt(36) lgkmcnt(0)\n" \
  "s_barrier\n" \
  "s_waitcnt lgkmcnt(2)\n" \
  "v_mfma_f32_16x16x128_f8f6f4 a[224:227], v[144:151], v[96:103], a[224:227] cbsz:1 blgp:1\n" \
  "s_add_u32 m0, s69, 126976\n" \
  "ds_read_b128 v[128:131], v160 offset:0\n" \
  "ds_read_b128 v[132:135], v160 offset:1024\n" \
  "v_mfma_f32_16x16x128_f8f6f4 a[228:231], v[144:151], v[104:111], a[228:231] cbsz:1 blgp:1\n" \
  "buffer_load_dwordx4 v163, s[36:39], s63 offen lds\n" \
  "v_mfma_f32_16x16x128_f8f6f4 a[232:235], v[144:151], v[112:119], a[232:235] cbsz:1 blgp:1\n" \
  "v_readfirstlane_b32 s89, v218\n" \
  "v_mfma_f32_16x16x128_f8f6f4 a[236:239], v[144:151], v[120:127], a[236:239] cbsz:1 blgp:1\n" \
  "s_bitcmp1_b32 s65, 18\n" \
  "s_cselect_b32 s42, s72, 0\n" \
  "s_mov_b32 s48, s71\n" \
  "s_waitcnt lgkmcnt(2)\n" \
  "v_mfma_f32_16x16x128_f8f6f4 a[240:243], v[152:159], v[96:103], a[240:243] cbsz:1 blgp:1\n" \
  "ds_read_b128 v[136:139], v160 offset:2048\n" \
  "ds_read_b128 v[140:143], v160 offset:3072\n" \
  "v_mfma_f32_16x16x128_f8f6f4 a[244:247], v[152:159], v[104:111], a[244:247] cbsz:1 blgp:1\n" \
  "v_mfma_f32_16x16x128_f8f6f4 a[248:251], v[152:159], v[112:119], a[248:251] cbsz:1 blgp:1\n" \
  "v_mfma_f32_16x16x128_f8f6f4 a[252:255], v[152:159], v[120:127], a[252:255] cbsz:1 blgp:1\n" \
  "s_add_u32 s49, s71, 1048576\n" \
  "s_add_u32 s50, s71, 2097152\n" \
  "s_add_u32 s51, s71, 3145728\n" \
  "s_waitcnt lgkmcnt(2)\n" \
  "v_accvgpr_read_b32 v184, a0\n" \
  "v_accvgpr_read_b32 v185, a1\n" \
  "v_accvgpr_read_b32 v186, a2\n" \
  "v_accvgpr_read_b32 v187, a3\n" \
  "v_mfma_f32_16x16x128_f8f6f4 a[0:3], v[128:135], v[0:7], 0 cbsz:1 blgp:1\n" \
  "s_add_u32 m0, s69, 98304\n" \
  "ds_read_b128 v[144:147], v160 offset:4096\n" \
  "ds_read_b128 v[148:151], v160 offset:5120\n" \
  "v_accvgpr_read_b32 v188, a4\n" \
  "v_accvgpr_read_b32 v189, a5\n" \
  "v_accvgpr_read_b32 v190, a6\n" \
  "v_accvgpr_read_b32 v191, a7\n" \
  "v_mfma_f32_16x16x128_f8f6f4 a[4:7], v[128:135], v[8:15], 0 cbsz:1 blgp:1\n" \
  "buffer_load_dwordx4 v163, s[36:39], s56 offen lds\n" \
  "v_accvgpr_read_b32 v192, a8\n" \
  "v_accvgpr_read_b32 v193, a9\n" \
  "v_accvgpr_read_b32 v194, a10\n" \
  "v_accvgpr_read_b32 v195, a11\n" \
  "v_mfma_f32_16x16x128_f8f6f4 a[8:11], v[128:135], v[16:23], 0 cbsz:1 blgp:1\n" \
  "v_accvgpr_read_b32 v196, a12\n" \
  "v_accvgpr_read_b32 v197, a13\n" \
  "v_accvgpr_read_b32 v198, a14\n" \
  "v_accvgpr_read_b32 v199, a15\n" \
  "v_mfma_f32_16x16x128_f8f6f4 a[12:15], v[128:135], v[24:31], 0 cbsz:1 blgp:1\n" \
  "s_cmp_eq_u32 s70, 64\n" \
  "s_cbranch_scc1 Lp21_%=\n" \
  "Lq21_%=:\n" \
  "s_waitcnt lgkmcnt(2)\n" \
  "v_accvgpr_read_b32 v200, a16\n" \
  "v_accvgpr_read_b32 v201, a17\n" \
  "v_accvgpr_read_b32 v202, a18\n" \
  "v_accvgpr_read_b32 v203, a19\n" \
  "v_mfma_f32_16x16x128_f8f6f4 a[16:19], v[136:143], v[0:7], 0 cbsz:1 blgp:1\n" \
  "ds_read_b128 v[152:155], v160 offset:6144\n" \
  "ds_read_b128 v[156:159], v160 offset:7168\n" \
  "v_accvgpr_read_b32 v204, a20\n" \
  "v_accvgpr_read_b32 v205, a21\n" \
  "v_accvgpr_read_b32 v206, a22\n" \
  "v_accvgpr_read_b32 v207, a23\n" \
  "v_mfma_f32_16x16x128_f8f6f4 a[20:23], v[136:143], v[8:15], 0 cbsz:1 blgp:1\n" \
  "buffer_load_dwordx4 v[96:99], v163, s[36:39], s75 offen offset:0\n" \
  "v_accvgpr_read_b32 v208, a24\n" \
  "v_accvgpr_read_b32 v209, a25\n" \
  "v_accvgpr_read_b32 v210, a26\n" \
  "v_accvgpr_read_b32 v211, a27\n" \
  "v_mfma_f32_16x16x128_f8f6f4 a[24:27], v[136:143], v[16:23], 0 cbsz:1 blgp:1\n" \
  "s_max_u32 s68, s68, s89\n" \
  "v_accvgpr_read_b32 v212, a28\n" \
  "v_accvgpr_read_b32 v213, a29\n" \
  "v_accvgpr_read_b32 v214, a30\n" \
  "v_accvgpr_read_b32 v215, a31\n" \
  "v_mfma_f32_16x16x128_f8f6f4 a[28:31], v[136:143], v[24:31], 0 cbsz:1 blgp:1\n" \
  "v_readlane_b32 s67, v175, s70\n" \
  "v_readlane_b32 s66, v178, s70\n" \
  "s_waitcnt lgkmcnt(2)\n" \
  "v_mfma_f32_16x16x128_f8f6f4 a[32:35], v[144:151], v[0:7], a[32:35] cbsz:1 blgp:1\n" \
  "s_add_u32 m0, s69, 102400\n" \
  "ds_read_b128 v[128:131], v160 offset:8192\n" \
  "ds_read_b128 v[132:135], v160 offset:9216\n" \
  "v_mfma_f32_16x16x128_f8f6f4 a[36:39], v[144:151], v[8:15], a[36:39] cbsz:1 blgp:1\n" \
  "buffer_load_dwordx4 v163, s[36:39], s57 offen lds\n" \
  "buffer_load_dwordx4 v[124:127], v164, s[36:39], s75 offen offset:3072\n" \
  "v_mfma_f32_16x16x128_f8f6f4 a[40:43], v[144:151], v[16:23], a[40:43] cbsz:1 blgp:1\n" \
  "v_mfma_f32_16x16x128_f8f6f4 a[44:47], v[144:151], v[24:31], a[44:47] cbsz:1 blgp:1\n" \
  "v_readlane_b32 s76, v169, s70\n" \
  "v_readlane_b32 s77, v172, s70\n" \
  "s_waitcnt lgkmcnt(2)\n" \
  "v_mfma_f32_16x16x128_f8f6f4 a[48:51], v[152:159], v[0:7], a[48:51] cbsz:1 blgp:1\n" \
  "ds_read_b128 v[136:139], v160 offset:10240\n" \
  "ds_read_b128 v[140:143], v160 offset:11264\n" \
  "v_mfma_f32_16x16x128_f8f6f4 a[52:55], v[152:159], v[8:15], a[52:55] cbsz:1 blgp:1\n" \
  "buffer_load_dwordx4 v[100:103], v163, s[36:39], s75 offen offset:1024\n" \
  "v_mfma_f32_16x16x128_f8f6f4 a[56:59], v[152:159], v[16:23], a[56:59] cbsz:1 blgp:1\n" \
  "v_mfma_f32_16x16x128_f8f6f4 a[60:63], v[152:159], v[24:31], a[60:63] cbsz:1 blgp:1\n" \
  "s_add_u32 s70, s70, 1\n" \
  "v_add_u32 v165, s76, v162\n" \
  "s_waitcnt lgkmcnt(2)\n" \
  "v_mfma_f32_16x16x128_f8f6f4 a[64:67], v[128:135], v[0:7], a[64:67] cbsz:1 blgp:1\n" \
  "s_add_u32 m0, s69, 106496\n" \
  "ds_read_b128 v[144:147], v160 offset:12288\n" \
  "ds_read_b128 v[148:151], v160 offset:13312\n" \
  "v_mfma_f32_16x16x128_f8f6f4 a[68:71], v[128:135], v[8:15], a[68:71] cbsz:1 blgp:1\n" \
  "buffer_load_dwordx4 v163, s[36:39], s58 offen lds\n" \
  "v_mfma_f32_16x16x128_f8f6f4 a[72:75], v[128:135], v[16:23], a[72:75] cbsz:1 blgp:1\n" \
  "v_mfma_f32_16x16x128_f8f6f4 a[76:79], v[128:135], v[24:31], a[76:79] cbsz:1 blgp:1\n" \
  "v_add_u32 v166, 4096, v165\n" \
  "s_cmp_gt_u32 s67, s68\n" \
  "s_cbranch_scc1 Lp22_%=\n" \
  "Lq22_%=:\n" \
  "s_waitcnt lgkmcnt(2)\n" \
  "v_mfma_f32_16x16x128_f8f6f4 a[80:83], v[136:143], v[0:7], a[80:83] cbsz:1 blgp:1\n" \
  "ds_read_b128 v[152:155], v160 offset:14336\n" \
  "ds_read_b128 v[156:159], v160 offset:15360\n" \
  "v_mfma_f32_16x16x128_f8f6f4 a[84:87], v[136:143], v[8:15], a[84:87] cbsz:1 blgp:1\n" \
  "buffer_load_dwordx4 v[104:107], v163, s[36:39], s75 offen offset:2048\n" \
  "v_mfma_f32_16x16x128_f8f6f4 a[88:91], v[136:143], v[16:23], a[88:91] cbsz:1 blgp:1\n" \
  "v_mfma_f32_16x16x128_f8f6f4 a[92:95], v[136:143], v[24:31], a[92:95] cbsz:1 blgp:1\n" \
  "s_waitcnt lgkmcnt(2)\n" \
  "v_mfma_f32_16x16x128_f8f6f4 a[96:99], v[144:151], v[0:7], a[96:99] cbsz:1 blgp:1\n" \
  "s_add_u32 m0, s69, 110592\n" \
  "ds_read_b128 v[128:131], v160 offset:16384\n" \
  "ds_read_b128 v[132:135], v160 offset:17408\n" \
  "v_mfma_f32_16x16x128_f8f6f4 a[100:103], v[144:151], v[8:15], a[100:103] cbsz:1 blgp:1\n" \
  "buffer_load_dwordx4 v163, s[36:39], s59 offen lds\n" \
  "v_mfma_f32_16x16x128_f8f6f4 a[104:107], v[144:151], v[16:23], a[104:107] cbsz:1 blgp:1\n" \
  "v_mfma_f32_16x16x128_f8f6f4 a[108:111], v[144:151], v[24:31], a[108:111] cbsz:1 blgp:1\n" \
  "s_waitcnt lgkmcnt(2)\n" \
  "v_mfma_f32_16x16x128_f8f6f4 a[112:115], v[152:159], v[0:7], a[112:115] cbsz:1 blgp:1\n" \
  "ds_read_b128 v[136:139], v160 offset:18432\n" \
  "ds_read_b128 v[140:143], v160 offset:19456\n" \
  "v_mfma_f32_16x16x128_f8f6f4 a[116:119], v[152:159], v[8:15], a[116:119] cbsz:1 blgp:1\n" \
  "buffer_load_dwordx4 v[108:111], v163, s[36:39], s75 offen offset:3072\n" \
  "v_mfma_f32_16x16x128_f8f6f4 a[120:123], v[152:159], v[16:23], a[120:123] cbsz:1 blgp:1\n" \
  "v_mfma_f32_16x16x128_f8f6f4 a[124:127], v[152:159], v[24:31], a[124:127] cbsz:1 blgp:1\n" \
  "s_waitcnt lgkmcnt(2)\n" \
  "v_mfma_f32_16x16x128_f8f6f4 a[128:131], v[128:135], v[0:7], a[128:131] cbsz:1 blgp:1\n" \
  "s_add_u32 m0, s69, 114688\n" \
  "ds_read_b128 v[144:147], v160 offset:20480\n" \
  "ds_read_b128 v[148:151], v160 offset:21504\n" \
  "v_mfma_f32_16x16x128_f8f6f4 a[132:135], v[128:135], v[8:15], a[132:135] cbsz:1 blgp:1\n" \
  "buffer_load_dwordx4 v163, s[36:39], s60 offen lds\n" \
  "v_mfma_f32_16x16x128_f8f6f4 a[136:139], v[128:135], v[16:23], a[136:139] cbsz:1 blgp:1\n" \
  "v_mfma_f32_16x16x128_f8f6f4 a[140:143], v[128:135], v[24:31], a[140:143] cbsz:1 blgp:1\n" \
  "s_waitcnt lgkmcnt(2)\n" \
  "v_mfma_f32_16x16x128_f8f6f4 a[144:147], v[136:143], v[0:7], a[144:147] cbsz:1 blgp:1\n" \
  "ds_read_b128 v[152:155], v160 offset:22528\n" \
  "ds_read_b128 v[156:159], v160 offset:23552\n" \
  "v_mfma_f32_16x16x128_f8f6f4 a[148:151], v[136:143], v[8:15], a[148:151] cbsz:1 blgp:1\n" \
  "buffer_load_dwordx4 v[112:115], v164, s[36:39], s75 offen offset:0\n" \
  "v_mfma_f32_16x16x128_f8f6f4 a[152:155], v[136:143], v[16:23], a[152:155] cbsz:1 blgp:1\n" \
  "v_mfma_f32_16x16x128_f8f6f4 a[156:159], v[136:143], v[24:31], a[156:159] cbsz:1 blgp:1\n" \
  "s_waitcnt lgkmcnt(2)\n" \
  "v_mfma_f32_16x16x128_f8f6f4 a[160:163], v[144:151], v[0:7], a[160:163] cbsz:1 blgp:1\n" \
  "s_add_u32 m0, s69, 118784\n" \
  "ds_read_b128 v[128:131], v160 offset:24576\n" \
  "ds_read_b128 v[132:135], v160 offset:25600\n" \
  "v_mfma_f32_16x16x128_f8f6f4 a[164:167], v[144:151], v[8:15], a[164:167] cbsz:1 blgp:1\n" \
  "buffer_load_dwordx4 v163, s[36:39], s61 offen lds\n" \
  "v_mfma_f32_16x16x128_f8f6f4 a[168:171], v[144:151], v[16:23], a[168:171] cbsz:1 blgp:1\n" \
  "v_mfma_f32_16x16x128_f8f6f4 a[172:175], v[144:151], v[24:31], a[172:175] cbsz:1 blgp:1\n" \
  "s_and_b32 s73, s66, 0xf00001ff\n" \
  "s_waitcnt lgkmcnt(2)\n" \
  "v_mfma_f32_16x16x128_f8f6f4 a[176:179], v[152:159], v[0:7], a[176:179] cbsz:1 blgp:1\n" \
  "ds_read_b128 v[136:139], v160 offset:26624\n" \
  "ds_read_b128 v[140:143], v160 offset:27648\n" \
  "v_mfma_f32_16x16x128_f8f6f4 a[180:183], v[152:159], v[8:15], a[180:183] cbsz:1 blgp:1\n" \
  "buffer_load_dwordx4 v[116:119], v164, s[36:39], s75 offen offset:1024\n" \
  "v_mfma_f32_16x16x128_f8f6f4 a[184:187], v[152:159], v[16:23], a[184:187] cbsz:1 blgp:1\n" \
  "v_mfma_f32_16x16x128_f8f6f4 a[188:191], v[152:159], v[24:31], a[188:191] cbsz:1 blgp:1\n" \
  "s_waitcnt lgkmcnt(2)\n" \
  "v_mfma_f32_16x16x128_f8f6f4 a[192:195], v[128:135], v[0:7], a[192:195] cbsz:1 blgp:1\n" \
  "s_add_u32 m0, s69, 122880\n" \
  "ds_read_b128 v[144:147], v160 offset:28672\n" \
  "ds_read_b128 v[148:151], v160 offset:29696\n" \
  "v_mfma_f32_16x16x128_f8f6f4 a[196:199], v[128:135], v[8:15], a[196:199] cbsz:1 blgp:1\n" \
  "buffer_load_dwordx4 v163, s[36:39], s62 offen lds\n" \
  "v_mfma_f32_16x16x128_f8f6f4 a[200:203], v[128:135], v[16:23], a[200:203] cbsz:1 blgp:1\n" \
  "v_mfma_f32_16x16x128_f8f6f4 a[204:207], v[128:135], v[24:31], a[204:207] cbsz:1 blgp:1\n" \
  "s_waitcnt lgkmcnt(2)\n" \
  "v_mfma_f32_16x16x128_f8f6f4 a[208:211], v[136:143], v[0:7], a[208:211] cbsz:1 blgp:1\n" \
  "v_mul_f32 v184, s78, v184\n" \
  "v_mul_f32 v185, s78, v185\n" \
  "v_mul_f32 v186, s78, v186\n" \
  "v_mul_f32 v187, s78, v187\n" \
  "ds_read_b128 v[152:155], v160 offset:30720\n" \
  "ds_read_b128 v[156:159], v160 offset:31744\n" \
  "v_mfma_f32_16x16x128_f8f6f4 a[212:215], v[136:143], v[8:15], a[212:215] cbsz:1 blgp:1\n" \
  "buffer_load_dwordx4 v[120:123], v164, s[36:39], s75 offen offset:2048\n" \
  "v_mfma_f32_16x16x128_f8f6f4 a[216:219], v[136:143], v[16:23], a[216:219] cbsz:1 blgp:1\n" \
  "buffer_store_dwordx4 v[184:187], v167, s[40:43], s48 offen offset:0 nt\n" \
  "v_mul_f32 v200, s78, v200\n" \
  "v_mul_f32 v201, s78, v201\n" \
  "v_mul_f32 v202, s78, v202\n" \
  "v_mul_f32 v203, s78, v203\n" \
  "v_mfma_f32_16x16x128_f8f6f4 a[220:223], v[136:143], v[24:31], a[220:223] cbsz:1 blgp:1\n" \
  "buffer_store_dwordx4 v[200:203], v167, s[40:43], s48 offen offset:64 nt\n" \
  "s_bfe_u32 m0, s66, 0x40018\n" \
  "s_nop 0\n" \
  "s_movreld_b32 s56, s77\n" \
  "s_cmp_eq_u32 s73, s88\n" \
  "s_cbranch_scc0 Lm1_%=\n" \
  "s_branch Lf1_%=\n" \
  ".p2align 6\n" \
  "Le0_2_%=:\n" \
  "s_waitcnt vmcnt(36) lgkmcnt(0)\n" \
  "s_barrier\n" \
  "s_waitcnt lgkmcnt(2)\n" \
  "v_mfma_f32_16x16x128_f8f6f4 a[224:227], v[144:151], v[96:103], a[224:227] cbsz:1 blgp:1\n" \
  "s_add_u32 m0, s69, 126976\n" \
  "ds_read_b128 v[128:131], v160 offset:0\n" \
  "ds_read_b128 v[132:135], v160 offset:1024\n" \
  "v_mfma_f32_16x16x128_f8f6f4 a[228:231], v[144:151], v[104:111], a[228:231] cbsz:1 blgp:1\n" \
  "buffer_load_dwordx4 v163, s[36:39], s63 offen lds\n" \
  "v_mfma_f32_16x16x128_f8f6f4 a[232:235], v[144:151], v[112:119], a[232:235] cbsz:1 blgp:1\n" \
  "v_readfirstlane_b32 s89, v218\n" \
  "v_mfma_f32_16x16x128_f8f6f4 a[236:239], v[144:151], v[120:127], a[236:239] cbsz:1 blgp:1\n" \
  "s_bitcmp1_b32 s65, 18\n" \
  "s_cselect_b32 s42, s72, 0\n" \
  "s_mov_b32 s48, s71\n" \
  "s_waitcnt lgkmcnt(2)\n" \
  "v_mfma_f32_16x16x128_f8f6f4 a[240:243], v[152:159], v[96:103], a[240:243] cbsz:1 blgp:1\n" \
  "ds_read_b128 v[136:139], v160 offset:2048\n" \
  "ds_read_b128 v[140:143], v160 offset:3072\n" \
  "v_mfma_f32_16x16x128_f8f6f4 a[244:247], v[152:159], v[104:111], a[244:247] cbsz:1 blgp:1\n" \
  "v_mfma_f32_16x16x128_f8f6f4 a[248:251], v[152:159], v[112:119], a[248:251] cbsz:1 blgp:1\n" \
  "v_mfma_f32_16x16x128_f8f6f4 a[252:255], v[152:159], v[120:127], a[252:255] cbsz:1 blgp:1\n" \
  "s_add_u32 s49, s71, 1048576\n" \
  "s_add_u32 s50, s71, 2097152\n" \
  "s_add_u32 s51, s71, 3145728\n" \
  "s_waitcnt lgkmcnt(2)\n" \
  "v_mfma_f32_16x16x128_f8f6f4 a[0:3], v[128:135], v[0:7], a[0:3] cbsz:1 blgp:1\n" \
  "s_add_u32 m0, s69, 98304\n" \
  "ds_read_b128 v[144:147], v160 offset:4096\n" \
  "ds_read_b128 v[148:151], v160 offset:5120\n" \
  "v_mfma_f32_16x16x128_f8f6f4 a[4:7], v[128:135], v[8:15], a[4:7] cbsz:1 blgp:1\n" \
  "buffer_load_dwordx4 v163, s[36:39], s56 offen lds\n" \
  "v_mfma_f32_16x16x128_f8f6f4 a[8:11], v[128:135], v[16:23], a[8:11] cbsz:1 blgp:1\n" \
  "v_mfma_f32_16x16x128_f8f6f4 a[12:15], v[128:135], v[24:31], a[12:15] cbsz:1 blgp:1\n" \
  "s_cmp_eq_u32 s70, 64\n" \
  "s_cbranch_scc1 Lp23_%=\n" \
  "Lq23_%=:\n" \
  "s_waitcnt lgkmcnt(2)\n" \
  "v_mfma_f32_16x16x128_f8f6f4 a[16:19], v[136:143], v[0:7], a[16:19] cbsz:1 blgp:1\n" \
  "ds_read_b128 v[152:155], v160 offset:6144\n" \
  "ds_read_b128 v[156:159], v160 offset:7168\n" \
  "v_mfma_f32_16x16x128_f8f6f4 a[20:23], v[136:143], v[8:15], a[20:23] cbsz:1 blgp:1\n" \
  "buffer_load_dwordx4 v[96:99], v163, s[36:39], s75 offen offset:0\n" \
  "v_mfma_f32_16x16x128_f8f6f4 a[24:27], v[136:143], v[16:23], a[24:27] cbsz:1 blgp:1\n" \
  "s_max_u32 s68, s68, s89\n" \
  "v_mfma_f32_16x16x128_f8f6f4 a[28:31], v[136:143], v[24:31], a[28:31] cbsz:1 blgp:1\n" \
  "v_readlane_b32 s67, v175, s70\n" \
  "v_readlane_b32 s66, v178, s70\n" \
  "s_waitcnt lgkmcnt(2)\n" \
  "v_accvgpr_read_b32 v184, a32\n" \
  "v_accvgpr_read_b32 v185, a33\n" \
  "v_accvgpr_read_b32 v186, a34\n" \
  "v_accvgpr_read_b32 v187, a35\n" \
  "v_mfma_f32_16x16x128_f8f6f4 a[32:35], v[144:151], v[0:7], 0 cbsz:1 blgp:1\n" \
  "s_add_u32 m0, s69, 102400\n" \
  "ds_read_b128 v[128:131], v160 offset:8192\n" \
  "ds_read_b128 v[132:135], v160 offset:9216\n" \
  "v_accvgpr_read_b32 v188, a36\n" \
  "v_accvgpr_read_b32 v189, a37\n" \
  "v_accvgpr_read_b32 v190, a38\n" \
  "v_accvgpr_read_b32 v191, a39\n" \
  "v_mfma_f32_16x16x128_f8f6f4 a[36:39], v[144:151], v[8:15], 0 cbsz:1 blgp:1\n" \
  "buffer_load_dwordx4 v163, s[36:39], s57 offen lds\n" \
  "buffer_load_dwordx4 v[124:127], v164, s[36:39], s75 offen offset:3072\n" \
  "v_accvgpr_read_b32 v192, a40\n" \
  "v_accvgpr_read_b32 v193, a41\n" \
  "v_accvgpr_read_b32 v194, a42\n" \
  "v_accvgpr_read_b32 v195, a43\n" \
  "v_mfma_f32_16x16x128_f8f6f4 a[40:43], v[144:151], v[16:23], 0 cbsz:1 blgp:1\n" \
  "v_accvgpr_read_b32 v196, a44\n" \
  "v_accvgpr_read_b32 v197, a45\n" \
  "v_accvgpr_read_b32 v198, a46\n" \
  "v_accvgpr_read_b32 v199, a47\n" \
  "v_mfma_f32_16x16x128_f8f6f4 a[44:47], v[144:151], v[24:31], 0 cbsz:1 blgp:1\n" \
  "v_readlane_b32 s76, v169, s70\n" \
  "v_readlane_b32 s77, v172, s70\n" \
  "s_waitcnt lgkmcnt(2)\n" \
  "v_accvgpr_read_b32 v200, a48\n" \
  "v_accvgpr_read_b32 v201, a49\n" \
  "v_accvgpr_read_b32 v202, a50\n" \
  "v_accvgpr_read_b32 v203, a51\n" \
  "v_mfma_f32_16x16x128_f8f6f4 a[48:51], v[152:159], v[0:7], 0 cbsz:1 blgp:1\n" \
  "ds_read_b128 v[136:139], v160 offset:10240\n" \
  "ds_read_b128 v[140:143], v160 offset:11264\n" \
  "v_accvgpr_read_b32 v204, a52\n" \
  "v_accvgpr_read_b32 v205, a53\n" \
  "v_accvgpr_read_b32 v206, a54\n" \
  "v_accvgpr_read_b32 v207, a55\n" \
  "v_mfma_f32_16x16x128_f8f6f4 a[52:55], v[152:159], v[8:15], 0 cbsz:1 blgp:1\n" \
  "buffer_load_dwordx4 v[100:103], v163, s[36:39], s75 offen offset:1024\n" \
  "v_accvgpr_read_b32 v208, a56\n" \
  "v_accvgpr_read_b32 v209, a57\n" \
  "v_accvgpr_read_b32 v210, a58\n" \
  "v_accvgpr_read_b32 v211, a59\n" \
  "v_mfma_f32_16x16x128_f8f6f4 a[56:59], v[152:159], v[16:23], 0 cbsz:1 blgp:1\n" \
  "v_accvgpr_read_b32 v212, a60\n" \
  "v_accvgpr_read_b32 v213, a61\n" \
  "v_accvgpr_read_b32 v214, a62\n" \
  "v_accvgpr_read_b32 v215, a63\n" \
  "v_mfma_f32_16x16x128_f8f6f4 a[60:63], v[152:159], v[24:31], 0 cbsz:1 blgp:1\n" \
  "s_add_u32 s70, s70, 1\n" \
  "v_add_u32 v165, s76, v162\n" \
  "s_waitcnt lgkmcnt(2)\n" \
  "v_mfma_f32_16x16x128_f8f6f4 a[64:67], v[128:135], v[0:7], a[64:67] cbsz:1 blgp:1\n" \
  "s_add_u32 m0, s69, 106496\n" \
  "ds_read_b128 v[144:147], v160 offset:12288\n" \
  "ds_read_b128 v[148:151], v160 offset:13312\n" \
  "v_mfma_f32_16x16x128_f8f6f4 a[68:71], v[128:135], v[8:15], a[68:71] cbsz:1 blgp:1\n" \
  "buffer_load_dwordx4 v163, s[36:39], s58 offen lds\n" \
  "v_mfma_f32_16x16x128_f8f6f4 a[72:75], v[128:135], v[16:23], a[72:75] cbsz:1 blgp:1\n" \
  "v_mfma_f32_16x16x128_f8f6f4 a[76:79], v[128:135], v[24:31], a[76:79] cbsz:1 blgp:1\n" \
  "v_add_u32 v166, 4096, v165\n" \
  "s_cmp_gt_u32 s67, s68\n" \
  "s_cbranch_scc1 Lp24_%=\n" \
  "Lq24_%=:\n" \
  "s_waitcnt lgkmcnt(2)\n" \
  "v_mfma_f32_16x16x128_f8f6f4 a[80:83], v[136:143], v[0:7], a[80:83] cbsz:1 blgp:1\n" \
  "ds_read_b128 v[152:155], v160 offset:14336\n" \
  "ds_read_b128 v[156:159], v160 offset:15360\n" \
  "v_mfma_f32_16x16x128_f8f6f4 a[84:87], v[136:143], v[8:15], a[84:87] cbsz:1 blgp:1\n" \
  "buffer_load_dwordx4 v[104:107], v163, s[36:39], s75 offen offset:2048\n" \
  "v_mfma_f32_16x16x128_f8f6f4 a[88:91], v[136:143], v[16:23], a[88:91] cbsz:1 blgp:1\n" \
  "v_mfma_f32_16x16x128_f8f6f4 a[92:95], v[136:143], v[24:31], a[92:95] cbsz:1 blgp:1\n" \
  "s_waitcnt lgkmcnt(2)\n" \
  "v_mfma_f32_16x16x128_f8f6f4 a[96:99], v[144:151], v[0:7], a[96:99] cbsz:1 blgp:1\n" \
  "s_add_u32 m0, s69, 110592\n" \
  "ds_read_b128 v[128:131], v160 offset:16384\n" \
  "ds_read_b128 v[132:135], v160 offset:17408\n" \
  "v_mfma_f32_16x16x128_f8f6f4 a[100:103], v[144:151], v[8:15], a[100:103] cbsz:1 blgp:1\n" \
  "buffer_load_dwordx4 v163, s[36:39], s59 offen lds\n" \
  "v_mfma_f32_16x16x128_f8f6f4 a[104:107], v[144:151], v[16:23], a[104:107] cbsz:1 blgp:1\n" \
  "v_mfma_f32_16x16x128_f8f6f4 a[108:111], v[144:151], v[24:31], a[108:111] cbsz:1 blgp:1\n" \
  "s_waitcnt lgkmcnt(2)\n" \
  "v_mfma_f32_16x16x128_f8f6f4 a[112:115], v[152:159], v[0:7], a[112:115] cbsz:1 blgp:1\n" \
  "ds_read_b128 v[136:139], v160 offset:18432\n" \
  "ds_read_b128 v[140:143], v160 offset:19456\n" \
  "v_mfma_f32_16x16x128_f8f6f4 a[116:119], v[152:159], v[8:15], a[116:119] cbsz:1 blgp:1\n" \
  "buffer_load_dwordx4 v[108:111], v163, s[36:39], s75 offen offset:3072\n" \
  "v_mfma_f32_16x16x128_f8f6f4 a[120:123], v[152:159], v[16:23], a[120:123] cbsz:1 blgp:1\n" \
  "v_mfma_f32_16x16x128_f8f6f4 a[124:127], v[152:159], v[24:31], a[124:127] cbsz:1 blgp:1\n" \
  "s_waitcnt lgkmcnt(2)\n" \
  "v_mfma_f32_16x16x128_f8f6f4 a[128:131], v[128:135], v[0:7], a[128:131] cbsz:1 blgp:1\n" \
  "s_add_u32 m0, s69, 114688\n" \
  "ds_read_b128 v[144:147], v160 offset:20480\n" \
  "ds_read_b128 v[148:151], v160 offset:21504\n" \
  "v_mfma_f32_16x16x128_f8f6f4 a[132:135], v[128:135], v[8:15], a[132:135] cbsz:1 blgp:1\n" \
  "buffer_load_dwordx4 v163, s[36:39], s60 offen lds\n" \
  "v_mfma_f32_16x16x128_f8f6f4 a[136:139], v[128:135], v[16:23], a[136:139] cbsz:1 blgp:1\n" \
  "v_mfma_f32_16x16x128_f8f6f4 a[140:143], v[128:135], v[24:31], a[140:143] cbsz:1 blgp:1\n" \
  "s_waitcnt lgkmcnt(2)\n" \
  "v_mfma_f32_16x16x128_f8f6f4 a[144:147], v[136:143], v[0:7], a[144:147] cbsz:1 blgp:1\n" \
  "ds_read_b128 v[152:155], v160 offset:22528\n" \
  "ds_read_b128 v[156:159], v160 offset:23552\n" \
  "v_mfma_f32_16x16x128_f8f6f4 a[148:151], v[136:143], v[8:15], a[148:151] cbsz:1 blgp:1\n" \
  "buffer_load_dwordx4 v[112:115], v164, s[36:39], s75 offen offset:0\n" \
  "v_mfma_f32_16x16x128_f8f6f4 a[152:155], v[136:143], v[16:23], a[152:155] cbsz:1 blgp:1\n" \
  "v_mfma_f32_16x16x128_f8f6f4 a[156:159], v[136:143], v[24:31], a[156:159] cbsz:1 blgp:1\n" \
  "s_waitcnt lgkmcnt(2)\n" \
  "v_mfma_f32_16x16x128_f8f6f4 a[160:163], v[144:151], v[0:7], a[160:163] cbsz:1 blgp:1\n" \
  "s_add_u32 m0, s69, 118784\n" \
  "ds_read_b128 v[128:131], v160 offset:24576\n" \
  "ds_read_b128 v[132:135], v160 offset:25600\n" \
  "v_mfma_f32_16x16x128_f8f6f4 a[164:167], v[144:151], v[8:15], a[164:167] cbsz:1 blgp:1\n" \
  "buffer_load_dwordx4 v163, s[36:39], s61 offen lds\n" \
  "v_mfma_f32_16x16x128_f8f6f4 a[168:171], v[144:151], v[16:23], a[168:171] cbsz:1 blgp:1\n" \
  "v_mfma_f32_16x16x128_f8f6f4 a[172:175], v[144:151], v[24:31], a[172:175] cbsz:1 blgp:1\n" \
  "s_and_b32 s73, s66, 0xf00001ff\n" \
  "s_waitcnt lgkmcnt(2)\n" \
  "v_mfma_f32_16x16x128_f8f6f4 a[176:179], v[152:159], v[0:7], a[176:179] cbsz:1 blgp:1\n" \
  "ds_read_b128 v[136:139], v160 offset:26624\n" \
  "ds_read_b128 v[140:143], v160 offset:27648\n" \
  "v_mfma_f32_16x16x128_f8f6f4 a[180:183], v[152:159], v[8:15], a[180:183] cbsz:1 blgp:1\n" \
  "buffer_load_dwordx4 v[116:119], v164, s[36:39], s75 offen offset:1024\n" \
  "v_mfma_f32_16x16x128_f8f6f4 a[184:187], v[152:159], v[16:23], a[184:187] cbsz:1 blgp:1\n" \
  "v_mfma_f32_16x16x128_f8f6f4 a[188:191], v[152:159], v[24:31], a[188:191] cbsz:1 blgp:1\n" \
  "s_waitcnt lgkmcnt(2)\n" \
  "v_mfma_f32_16x16x128_f8f6f4 a[192:195], v[128:135], v[0:7], a[192:195] cbsz:1 blgp:1\n" \
  "s_add_u32 m0, s69, 122880\n" \
  "ds_read_b128 v[144:147], v160 offset:28672\n" \
  "ds_read_b128 v[148:151], v160 offset:29696\n" \
  "v_mfma_f32_16x16x128_f8f6f4 a[196:199], v[128:135], v[8:15], a[196:199] cbsz:1 blgp:1\n" \
  "buffer_load_dwordx4 v163, s[36:39], s62 offen lds\n" \
  "v_mfma_f32_16x16x128_f8f6f4 a[200:203], v[128:135], v[16:23], a[200:203] cbsz:1 blgp:1\n" \
  "v_mfma_f32_16x16x128_f8f6f4 a[204:207], v[128:135], v[24:31], a[204:207] cbsz:1 blgp:1\n" \
  "s_waitcnt lgkmcnt(2)\n" \
  "v_mfma_f32_16x16x128_f8f6f4 a[208:211], v[136:143], v[0:7], a[208:211] cbsz:1 blgp:1\n" \
  "v_mul_f32 v184, s78, v184\n" \
  "v_mul_f32 v185, s78, v185\n" \
  "v_mul_f32 v186, s78, v186\n" \
  "v_mul_f32 v187, s78, v187\n" \
  "ds_read_b128 v[152:155], v160 offset:30720\n" \
  "ds_read_b128 v[156:159], v160 offset:31744\n" \
  "v_mfma_f32_16x16x128_f8f6f4 a[212:215], v[136:143], v[8:15], a[212:215] cbsz:1 blgp:1\n" \
  "buffer_load_dwordx4 v[120:123], v164, s[36:39], s75 offen offset:2048\n" \
  "v_mfma_f32_16x16x128_f8f6f4 a[216:219], v[136:143], v[16:23], a[216:219] cbsz:1 blgp:1\n" \
  "buffer_store_dwordx4 v[184:187], v167, s[40:43], s48 offen offset:0 nt\n" \
  "v_mul_f32 v200, s78, v200\n" \
  "v_mul_f32 v201, s78, v201\n" \
  "v_mul_f32 v202, s78, v202\n" \
  "v_mul_f32 v203, s78, v203\n" \
  "v_mfma_f32_16x16x128_f8f6f4 a[220:223], v[136:143], v[24:31], a[220:223] cbsz:1 blgp:1\n" \
  "buffer_store_dwordx4 v[200:203], v167, s[40:43], s48 offen offset:64 nt\n" \
  "s_bfe_u32 m0, s66, 0x40018\n" \
  "s_nop 0\n" \
  "s_movreld_b32 s56, s77\n" \
  "s_cmp_eq_u32 s73, s88\n" \
  "s_cbranch_scc0 Lm1_%=\n" \
  "s_branch Lf1_%=\n" \
  ".p2align 6\n" \
  "Le0_3_%=:\n" \
  "s_waitcnt vmcnt(36) lgkmcnt(0)\n" \
  "s_barrier\n" \
  "s_waitcnt lgkmcnt(2)\n" \
  "v_mfma_f32_16x16x128_f8f6f4 a[224:227], v[144:151], v[96:103], a[224:227] cbsz:1 blgp:1\n" \
  "s_add_u32 m0, s69, 126976\n" \
  "ds_read_b128 v[128:131], v160 offset:0\n" \
  "ds_read_b128 v[132:135], v160 offset:1024\n" \
  "v_mfma_f32_16x16x128_f8f6f4 a[228:231], v[144:151], v[104:111], a[228:231] cbsz:1 blgp:1\n" \
  "buffer_load_dwordx4 v163, s[36:39], s63 offen lds\n" \
  "v_mfma_f32_16x16x128_f8f6f4 a[232:235], v[144:151], v[112:119], a[232:235] cbsz:1 blgp:1\n" \
  "v_readfirstlane_b32 s89, v218\n" \
  "v_mfma_f32_16x16x128_f8f6f4 a[236:239], v[144:151], v[120:127], a[236:239] cbsz:1 blgp:1\n" \
  "s_bitcmp1_b32 s65, 18\n" \
  "s_cselect_b32 s42, s72, 0\n" \
  "s_mov_b32 s48, s71\n" \
  "s_waitcnt lgkmcnt(2)\n" \
  "v_mfma_f32_16x16x128_f8f6f4 a[240:243], v[152:159], v[96:103], a[240:243] cbsz:1 blgp:1\n" \
  "ds_read_b128 v[136:139], v160 offset:2048\n" \
  "ds_read_b128 v[140:143], v160 offset:3072\n" \
  "v_mfma_f32_16x16x128_f8f6f4 a[244:247], v[152:159], v[104:111], a[244:247] cbsz:1 blgp:1\n" \
  "v_mfma_f32_16x16x128_f8f6f4 a[248:251], v[152:159], v[112:119], a[248:251] cbsz:1 blgp:1\n" \
  "v_mfma_f32_16x16x128_f8f6f4 a[252:255], v[152:159], v[120:127], a[252:255] cbsz:1 blgp:1\n" \
  "s_add_u32 s49, s71, 1048576\n" \
  "s_add_u32 s50, s71, 2097152\n" \
  "s_add_u32 s51, s71, 3145728\n" \
  "s_waitcnt lgkmcnt(2)\n" \
  "v_mfma_f32_16x16x128_f8f6f4 a[0:3], v[128:135], v[0:7], a[0:3] cbsz:1 blgp:1\n" \
  "s_add_u32 m0, s69, 98304\n" \
  "ds_read_b128 v[144:147], v160 offset:4096\n" \
  "ds_read_b128 v[148:151], v160 offset:5120\n" \
  "v_mfma_f32_16x16x128_f8f6f4 a[4:7], v[128:135], v[8:15], a[4:7] cbsz:1 blgp:1\n" \
  "buffer_load_dwordx4 v163, s[36:39], s56 offen lds\n" \
  "v_mfma_f32_16x16x128_f8f6f4 a[8:11], v[128:135], v[16:23], a[8:11] cbsz:1 blgp:1\n" \
  "v_mfma_f32_16x16x128_f8f6f4 a[12:15], v[128:135], v[24:31], a[12:15] cbsz:1 blgp:1\n" \
  "s_cmp_eq_u32 s70, 64\n" \
  "s_cbranch_scc1 Lp25_%=\n" \
  "Lq25_%=:\n" \
  "s_waitcnt lgkmcnt(2)\n" \
  "v_mfma_f32_16x16x128_f8f6f4 a[16:19], v[136:143], v[0:7], a[16:19] cbsz:1 blgp:1\n" \
  "ds_read_b128 v[152:155], v160 offset:6144\n" \
  "ds_read_b128 v[156:159], v160 offset:7168\n" \
  "v_mfma_f32_16x16x128_f8f6f4 a[20:23], v[136:143], v[8:15], a[20:23] cbsz:1 blgp:1\n" \
  "buffer_load_dwordx4 v[96:99], v163, s[36:39], s75 offen offset:0\n" \
  "v_mfma_f32_16x16x128_f8f6f4 a[24:27], v[136:143], v[16:23], a[24:27] cbsz:1 blgp:1\n" \
  "s_max_u32 s68, s68, s89\n" \
  "v_mfma_f32_16x16x128_f8f6f4 a[28:31], v[136:143], v[24:31], a[28:31] cbsz:1 blgp:1\n" \
  "v_readlane_b32 s67, v175, s70\n" \
  "v_readlane_b32 s66, v178, s70\n" \
  "s_waitcnt lgkmcnt(2)\n" \
  "v_mfma_f32_16x16x128_f8f6f4 a[32:35], v[144:151], v[0:7], a[32:35] cbsz:1 blgp:1\n" \
  "s_add_u32 m0, s69, 102400\n" \
  "ds_read_b128 v[128:131], v160 offset:8192\n" \
  "ds_read_b128 v[132:135], v160 offset:9216\n" \
  "v_mfma_f32_16x16x128_f8f6f4 a[36:39], v[144:151], v[8:15], a[36:39] cbsz:1 blgp:1\n" \
  "buffer_load_dwordx4 v163, s[36:39], s57 offen lds\n" \
  "buffer_load_dwordx4 v[124:127], v164, s[36:39], s75 offen offset:3072\n" \
  "v_mfma_f32_16x16x128_f8f6f4 a[40:43], v[144:151], v[16:23], a[40:43] cbsz:1 blgp:1\n" \
  "v_mfma_f32_16x16x128_f8f6f4 a[44:47], v[144:151], v[24:31], a[44:47] cbsz:1 blgp:1\n" \
  "v_readlane_b32 s76, v169, s70\n" \
  "v_readlane_b32 s77, v172, s70\n" \
  "s_waitcnt lgkmcnt(2)\n" \
  "v_mfma_f32_16x16x128_f8f6f4 a[48:51], v[152:159], v[0:7], a[48:51] cbsz:1 blgp:1\n" \
  "ds_read_b128 v[136:139], v160 offset:10240\n" \
  "ds_read_b128 v[140:143], v160 offset:11264\n" \
  "v_mfma_f32_16x16x128_f8f6f4 a[52:55], v[152:159], v[8:15], a[52:55] cbsz:1 blgp:1\n" \
  "buffer_load_dwordx4 v[100:103], v163, s[36:39], s75 offen offset:1024\n" \
  "v_mfma_f32_16x16x128_f8f6f4 a[56:59], v[152:159], v[16:23], a[56:59] cbsz:1 blgp:1\n" \
  "v_mfma_f32_16x16x128_f8f6f4 a[60:63], v[152:159], v[24:31], a[60:63] cbsz:1 blgp:1\n" \
  "s_add_u32 s70, s70, 1\n" \
  "v_add_u32 v165, s76, v162\n" \
  "s_waitcnt lgkmcnt(2)\n" \
  "v_accvgpr_read_b32 v184, a64\n" \
  "v_accvgpr_read_b32 v185, a65\n" \
  "v_accvgpr_read_b32 v186, a66\n" \
  "v_accvgpr_read_b32 v187, a67\n" \
  "v_mfma_f32_16x16x128_f8f6f4 a[64:67], v[128:135], v[0:7], 0 cbsz:1 blgp:1\n" \
  "s_add_u32 m0, s69, 106496\n" \
  "ds_read_b128 v[144:147], v160 offset:12288\n" \
  "ds_read_b128 v[148:151], v160 offset:13312\n" \
  "v_accvgpr_read_b32 v188, a68\n" \
  "v_accvgpr_read_b32 v189, a69\n" \
  "v_accvgpr_read_b32 v190, a70\n" \
  "v_accvgpr_read_b32 v191, a71\n" \
  "v_mfma_f32_16x16x128_f8f6f4 a[68:71], v[128:135], v[8:15], 0 cbsz:1 blgp:1\n" \
  "buffer_load_dwordx4 v163, s[36:39], s58 offen lds\n" \
  "v_accvgpr_read_b32 v192, a72\n" \
  "v_accvgpr_read_b32 v193, a73\n" \
  "v_accvgpr_read_b32 v194, a74\n" \
  "v_accvgpr_read_b32 v195, a75\n" \
  "v_mfma_f32_16x16x128_f8f6f4 a[72:75], v[128:135], v[16:23], 0 cbsz:1 blgp:1\n" \
  "v_accvgpr_read_b32 v196, a76\n" \
  "v_accvgpr_read_b32 v197, a77\n" \
  "v_accvgpr_read_b32 v198, a78\n" \
  "v_accvgpr_read_b32 v199, a79\n" \
  "v_mfma_f32_16x16x128_f8f6f4 a[76:79], v[128:135], v[24:31], 0 cbsz:1 blgp:1\n" \
  "v_add_u32 v166, 4096, v165\n" \
  "s_cmp_gt_u32 s67, s68\n" \
  "s_cbranch_scc1 Lp26_%=\n" \
  "Lq26_%=:\n" \
  "s_waitcnt lgkmcnt(2)\n" \
  "v_accvgpr_read_b32 v200, a80\n" \
  "v_accvgpr_read_b32 v201, a81\n" \
  "v_accvgpr_read_b32 v202, a82\n" \
  "v_accvgpr_read_b32 v203, a83\n" \
  "v_mfma_f32_16x16x128_f8f6f4 a[80:83], v[136:143], v[0:7], 0 cbsz:1 blgp:1\n" \
  "ds_read_b128 v[152:155], v160 offset:14336\n" \
  "ds_read_b128 v[156:159], v160 offset:15360\n" \
  "v_accvgpr_read_b32 v204, a84\n" \
  "v_accvgpr_read_b32 v205, a85\n" \
  "v_accvgpr_read_b32 v206, a86\n" \
  "v_accvgpr_read_b32 v207, a87\n" \
  "v_mfma_f32_16x16x128_f8f6f4 a[84:87], v[136:143], v[8:15], 0 cbsz:1 blgp:1\n" \
  "buffer_load_dwordx4 v[104:107], v163, s[36:39], s75 offen offset:2048\n" \
  "v_accvgpr_read_b32 v208, a88\n" \
  "v_accvgpr_read_b32 v209, a89\n" \
  "v_accvgpr_read_b32 v210, a90\n" \
  "v_accvgpr_read_b32 v211, a91\n" \
  "v_mfma_f32_16x16x128_f8f6f4 a[88:91], v[136:143], v[16:23], 0 cbsz:1 blgp:1\n" \
  "v_accvgpr_read_b32 v212, a92\n" \
  "v_accvgpr_read_b32 v213, a93\n" \
  "v_accvgpr_read_b32 v214, a94\n" \
  "v_accvgpr_read_b32 v215, a95\n" \
  "v_mfma_f32_16x16x128_f8f6f4 a[92:95], v[136:143], v[24:31], 0 cbsz:1 blgp:1\n" \
  "s_waitcnt lgkmcnt(2)\n" \
  "v_mfma_f32_16x16x128_f8f6f4 a[96:99], v[144:151], v[0:7], a[96:99] cbsz:1 blgp:1\n" \
  "s_add_u32 m0, s69, 110592\n" \
  "ds_read_b128 v[128:131], v160 offset:16384\n" \
  "ds_read_b128 v[132:135], v160 offset:17408\n" \
  "v_mfma_f32_16x16x128_f8f6f4 a[100:103], v[144:151], v[8:15], a[100:103] cbsz:1 blgp:1\n" \
  "buffer_load_dwordx4 v163, s[36:39], s59 offen lds\n" \
  "v_mfma_f32_16x16x128_f8f6f4 a[104:107], v[144:151], v[16:23], a[104:107] cbsz:1 blgp:1\n" \
  "v_mfma_f32_16x16x128_f8f6f4 a[108:111], v[144:151], v[24:31], a[108:111] cbsz:1 blgp:1\n" \
  "s_waitcnt lgkmcnt(2)\n" \
  "v_mfma_f32_16x16x128_f8f6f4 a[112:115], v[152:159], v[0:7], a[112:115] cbsz:1 blgp:1\n" \
  "ds_read_b128 v[136:139], v160 offset:18432\n" \
  "ds_read_b128 v[140:143], v160 offset:19456\n" \
  "v_mfma_f32_16x16x128_f8f6f4 a[116:119], v[152:159], v[8:15], a[116:119] cbsz:1 blgp:1\n" \
  "buffer_load_dwordx4 v[108:111], v163, s[36:39], s75 offen offset:3072\n" \
  "v_mfma_f32_16x16x128_f8f6f4 a[120:123], v[152:159], v[16:23], a[120:123] cbsz:1 blgp:1\n" \
  "v_mfma_f32_16x16x128_f8f6f4 a[124:127], v[152:159], v[24:31], a[124:127] cbsz:1 blgp:1\n" \
  "s_waitcnt lgkmcnt(2)\n" \
  "v_mfma_f32_16x16x128_f8f6f4 a[128:131], v[128:135], v[0:7], a[128:131] cbsz:1 blgp:1\n" \
  "s_add_u32 m0, s69, 114688\n" \
  "ds_read_b128 v[144:147], v160 offset:20480\n" \
  "ds_read_b128 v[148:151], v160 offset:21504\n" \
  "v_mfma_f32_16x16x128_f8f6f4 a[132:135], v[128:135], v[8:15], a[132:135] cbsz:1 blgp:1\n" \
  "buffer_load_dwordx4 v163, s[36:39], s60 offen lds\n" \
  "v_mfma_f32_16x16x128_f8f6f4 a[136:139], v[128:135], v[16:23], a[136:139] cbsz:1 blgp:1\n" \
  "v_mfma_f32_16x16x128_f8f6f4 a[140:143], v[128:135], v[24:31], a[140:143] cbsz:1 blgp:1\n" \
  "s_waitcnt lgkmcnt(2)\n" \
  "v_mfma_f32_16x16x128_f8f6f4 a[144:147], v[136:143], v[0:7], a[144:147] cbsz:1 blgp:1\n" \
  "ds_read_b128 v[152:155], v160 offset:22528\n" \
  "ds_read_b128 v[156:159], v160 offset:23552\n" \
  "v_mfma_f32_16x16x128_f8f6f4 a[148:151], v[136:143], v[8:15], a[148:151] cbsz:1 blgp:1\n" \
  "buffer_load_dwordx4 v[112:115], v164, s[36:39], s75 offen offset:0\n" \
  "v_mfma_f32_16x16x128_f8f6f4 a[152:155], v[136:143], v[16:23], a[152:155] cbsz:1 blgp:1\n" \
  "v_mfma_f32_16x16x128_f8f6f4 a[156:159], v[136:143], v[24:31], a[156:159] cbsz:1 blgp:1\n" \
  "s_waitcnt lgkmcnt(2)\n" \
  "v_mfma_f32_16x16x128_f8f6f4 a[160:163], v[144:151], v[0:7], a[160:163] cbsz:1 blgp:1\n" \
  "s_add_u32 m0, s69, 118784\n" \
  "ds_read_b128 v[128:131], v160 offset:24576\n" \
  "ds_read_b128 v[132:135], v160 offset:25600\n" \
  "v_mfma_f32_16x16x128_f8f6f4 a[164:167], v[144:151], v[8:15], a[164:167] cbsz:1 blgp:1\n" \
  "buffer_load_dwordx4 v163, s[36:39], s61 offen lds\n" \
  "v_mfma_f32_16x16x128_f8f6f4 a[168:171], v[144:151], v[16:23], a[168:171] cbsz:1 blgp:1\n" \
  "v_mfma_f32_16x16x128_f8f6f4 a[172:175], v[144:151], v[24:31], a[172:175] cbsz:1 blgp:1\n" \
  "s_and_b32 s73, s66, 0xf00001ff\n" \
  "s_waitcnt lgkmcnt(2)\n" \
  "v_mfma_f32_16x16x128_f8f6f4 a[176:179], v[152:159], v[0:7], a[176:179] cbsz:1 blgp:1\n" \
  "ds_read_b128 v[136:139], v160 offset:26624\n" \
  "ds_read_b128 v[140:143], v160 offset:27648\n" \
  "v_mfma_f32_16x16x128_f8f6f4 a[180:183], v[152:159], v[8:15], a[180:183] cbsz:1 blgp:1\n" \
  "buffer_load_dwordx4 v[116:119], v164, s[36:39], s75 offen offset:1024\n" \
  "v_mfma_f32_16x16x128_f8f6f4 a[184:187], v[152:159], v[16:23], a[184:187] cbsz:1 blgp:1\n" \
  "v_mfma_f32_16x16x128_f8f6f4 a[188:191], v[152:159], v[24:31], a[188:191] cbsz:1 blgp:1\n" \
  "s_waitcnt lgkmcnt(2)\n" \
  "v_mfma_f32_16x16x128_f8f6f4 a[192:195], v[128:135], v[0:7], a[192:195] cbsz:1 blgp:1\n" \
  "s_add_u32 m0, s69, 122880\n" \
  "ds_read_b128 v[144:147], v160 offset:28672\n" \
  "ds_read_b128 v[148:151], v160 offset:29696\n" \
  "v_mfma_f32_16x16x128_f8f6f4 a[196:199], v[128:135], v[8:15], a[196:199] cbsz:1 blgp:1\n" \
  "buffer_load_dwordx4 v163, s[36:39], s62 offen lds\n" \
  "v_mfma_f32_16x16x128_f8f6f4 a[200:203], v[128:135], v[16:23], a[200:203] cbsz:1 blgp:1\n" \
  "v_mfma_f32_16x16x128_f8f6f4 a[204:207], v[128:135], v[24:31], a[204:207] cbsz:1 blgp:1\n" \
  "s_waitcnt lgkmcnt(2)\n" \
  "v_mfma_f32_16x16x128_f8f6f4 a[208:211], v[136:143], v[0:7], a[208:211] cbsz:1 blgp:1\n" \
  "v_mul_f32 v184, s78, v184\n" \
  "v_mul_f32 v185, s78, v185\n" \
  "v_mul_f32 v186, s78, v186\n" \
  "v_mul_f32 v187, s78, v187\n" \
  "ds_read_b128 v[152:155], v160 offset:30720\n" \
  "ds_read_b128 v[156:159], v160 offset:31744\n" \
  "v_mfma_f32_16x16x128_f8f6f4 a[212:215], v[136:143], v[8:15], a[212:215] cbsz:1 blgp:1\n" \
  "buffer_load_dwordx4 v[120:123], v164, s[36:39], s75 offen offset:2048\n" \
  "v_mfma_f32_16x16x128_f8f6f4 a[216:219], v[136:143], v[16:23], a[216:219] cbsz:1 blgp:1\n" \
  "buffer_store_dwordx4 v[184:187], v167, s[40:43], s48 offen offset:0 nt\n" \
  "v_mul_f32 v200, s78, v200\n" \
  "v_mul_f32 v201, s78, v201\n" \
  "v_mul_f32 v202, s78, v202\n" \
  "v_mul_f32 v203, s78, v203\n" \
  "v_mfma_f32_16x16x128_f8f6f4 a[220:223], v[136:143], v[24:31], a[220:223] cbsz:1 blgp:1\n" \
  "buffer_store_dwordx4 v[200:203], v167, s[40:43], s48 offen offset:64 nt\n" \
  "s_bfe_u32 m0, s66, 0x40018\n" \
  "s_nop 0\n" \
  "s_movreld_b32 s56, s77\n" \
  "s_cmp_eq_u32 s73, s88\n" \
  "s_cbranch_scc0 Lm1_%=\n" \
  "s_branch Lf1_%=\n" \
  ".p2align 6\n" \
  "Le0_4_%=:\n" \
  "s_waitcnt vmcnt(36) lgkmcnt(0)\n" \
  "s_barrier\n" \
  "s_waitcnt lgkmcnt(2)\n" \
  "v_mfma_f32_16x16x128_f8f6f4 a[224:227], v[144:151], v[96:103], a[224:227] cbsz:1 blgp:1\n" \
  "s_add_u32 m0, s69, 126976\n" \
  "ds_read_b128 v[128:131], v160 offset:0\n" \
  "ds_read_b128 v[132:135], v160 offset:1024\n" \
  "v_mfma_f32_16x16x128_f8f6f4 a[228:231], v[144:151], v[104:111], a[228:231] cbsz:1 blgp:1\n" \
  "buffer_load_dwordx4 v163, s[36:39], s63 offen lds\n" \
  "v_mfma_f32_16x16x128_f8f6f4 a[232:235], v[144:151], v[112:119], a[232:235] cbsz:1 blgp:1\n" \
  "v_readfirstlane_b32 s89, v218\n" \
  "v_mfma_f32_16x16x128_f8f6f4 a[236:239], v[144:151], v[120:127], a[236:239] cbsz:1 blgp:1\n" \
  "s_bitcmp1_b32 s65, 18\n" \
  "s_cselect_b32 s42, s72, 0\n" \
  "s_mov_b32 s48, s71\n" \
  "s_waitcnt lgkmcnt(2)\n" \
  "v_mfma_f32_16x16x128_f8f6f4 a[240:243], v[152:159], v[96:103], a[240:243] cbsz:1 blgp:1\n" \
  "ds_read_b128 v[136:139], v160 offset:2048\n" \
  "ds_read_b128 v[140:143], v160 offset:3072\n" \
  "v_mfma_f32_16x16x128_f8f6f4 a[244:247], v[152:159], v[104:111], a[244:247] cbsz:1 blgp:1\n" \
  "v_mfma_f32_16x16x128_f8f6f4 a[248:251], v[152:159], v[112:119], a[248:251] cbsz:1 blgp:1\n" \
  "v_mfma_f32_16x16x128_f8f6f4 a[252:255], v[152:159], v[120:127], a[252:255] cbsz:1 blgp:1\n" \
  "s_add_u32 s49, s71, 1048576\n" \
  "s_add_u32 s50, s71, 2097152\n" \
  "s_add_u32 s51, s71, 3145728\n" \
  "s_waitcnt lgkmcnt(2)\n" \
  "v_mfma_f32_16x16x128_f8f6f4 a[0:3], v[128:135], v[0:7], a[0:3] cbsz:1 blgp:1\n" \
  "s_add_u32 m0, s69, 98304\n" \
  "ds_read_b128 v[144:147], v160 offset:4096\n" \
  "ds_read_b128 v[148:151], v160 offset:5120\n" \
  "v_mfma_f32_16x16x128_f8f6f4 a[4:7], v[128:135], v[8:15], a[4:7] cbsz:1 blgp:1\n" \
  "buffer_load_dwordx4 v163, s[36:39], s56 offen lds\n" \
  "v_mfma_f32_16x16x128_f8f6f4 a[8:11], v[128:135], v[16:23], a[8:11] cbsz:1 blgp:1\n" \
  "v_mfma_f32_16x16x128_f8f6f4 a[12:15], v[128:135], v[24:31], a[12:15] cbsz:1 blgp:1\n" \
  "s_cmp_eq_u32 s70, 64\n" \
  "s_cbranch_scc1 Lp27_%=\n" \
  "Lq27_%=:\n" \
  "s_waitcnt lgkmcnt(2)\n" \
  "v_mfma_f32_16x16x128_f8f6f4 a[16:19], v[136:143], v[0:7], a[16:19] cbsz:1 blgp:1\n" \
  "ds_read_b128 v[152:155], v160 offset:6144\n" \
  "ds_read_b128 v[156:159], v160 offset:7168\n" \
  "v_mfma_f32_16x16x128_f8f6f4 a[20:23], v[136:143], v[8:15], a[20:23] cbsz:1 blgp:1\n" \
  "buffer_load_dwordx4 v[96:99], v163, s[36:39], s75 offen offset:0\n" \
  "v_mfma_f32_16x16x128_f8f6f4 a[24:27], v[136:143], v[16:23], a[24:27] cbsz:1 blgp:1\n" \
  "s_max_u32 s68, s68, s89\n" \
  "v_mfma_f32_16x16x128_f8f6f4 a[28:31], v[136:143], v[24:31], a[28:31] cbsz:1 blgp:1\n" \
  "v_readlane_b32 s67, v175, s70\n" \
  "v_readlane_b32 s66, v178, s70\n" \
  "s_waitcnt lgkmcnt(2)\n" \
  "v_mfma_f32_16x16x128_f8f6f4 a[32:35], v[144:151], v[0:7], a[32:35] cbsz:1 blgp:1\n" \
  "s_add_u32 m0, s69, 102400\n" \
  "ds_read_b128 v[128:131], v160 offset:8192\n" \
  "ds_read_b128 v[132:135], v160 offset:9216\n" \
  "v_mfma_f32_16x16x128_f8f6f4 a[36:39], v[144:151], v[8:15], a[36:39] cbsz:1 blgp:1\n" \
  "buffer_load_dwordx4 v163, s[36:39], s57 offen lds\n" \
  "buffer_load_dwordx4 v[124:127], v164, s[36:39], s75 offen offset:3072\n" \
  "v_mfma_f32_16x16x128_f8f6f4 a[40:43], v[144:151], v[16:23], a[40:43] cbsz:1 blgp:1\n" \
  "v_mfma_f32_16x16x128_f8f6f4 a[44:47], v[144:151], v[24:31], a[44:47] cbsz:1 blgp:1\n" \
  "v_readlane_b32 s76, v169, s70\n" \
  "v_readlane_b32 s77, v172, s70\n" \
  "s_waitcnt lgkmcnt(2)\n" \
  "v_mfma_f32_16x16x128_f8f6f4 a[48:51], v[152:159], v[0:7], a[48:51] cbsz:1 blgp:1\n" \
  "ds_read_b128 v[136:139], v160 offset:10240\n" \
  "ds_read_b128 v[140:143], v160 offset:11264\n" \
  "v_mfma_f32_16x16x128_f8f6f4 a[52:55], v[152:159], v[8:15], a[52:55] cbsz:1 blgp:1\n" \
  "buffer_load_dwordx4 v[100:103], v163, s[36:39], s75 offen offset:1024\n" \
  "v_mfma_f32_16x16x128_f8f6f4 a[56:59], v[152:159], v[16:23], a[56:59] cbsz:1 blgp:1\n" \
  "v_mfma_f32_16x16x128_f8f6f4 a[60:63], v[152:159], v[24:31], a[60:63] cbsz:1 blgp:1\n" \
  "s_add_u32 s70, s70, 1\n" \
  "v_add_u32 v165, s76, v162\n" \
  "s_waitcnt lgkmcnt(2)\n" \
  "v_mfma_f32_16x16x128_f8f6f4 a[64:67], v[128:135], v[0:7], a[64:67] cbsz:1 blgp:1\n" \
  "s_add_u32 m0, s69, 106496\n" \
  "ds_read_b128 v[144:147], v160 offset:12288\n" \
  "ds_read_b128 v[148:151], v160 offset:13312\n" \
  "v_mfma_f32_16x16x128_f8f6f4 a[68:71], v[128:135], v[8:15], a[68:71] cbsz:1 blgp:1\n" \
  "buffer_load_dwordx4 v163, s[36:39], s58 offen lds\n" \
  "v_mfma_f32_16x16x128_f8f6f4 a[72:75], v[128:135], v[16:23], a[72:75] cbsz:1 blgp:1\n" \
  "v_mfma_f32_16x16x128_f8f6f4 a[76:79], v[128:135], v[24:31], a[76:79] cbsz:1 blgp:1\n" \
  "v_add_u32 v166, 4096, v165\n" \
  "s_cmp_gt_u32 s67, s68\n" \
  "s_cbranch_scc1 Lp28_%=\n" \
  "Lq28_%=:\n" \
  "s_waitcnt lgkmcnt(2)\n" \
  "v_mfma_f32_16x16x128_f8f6f4 a[80:83], v[136:143], v[0:7], a[80:83] cbsz:1 blgp:1\n" \
  "ds_read_b128 v[152:155], v160 offset:14336\n" \
  "ds_read_b128 v[156:159], v160 offset:15360\n" \
  "v_mfma_f32_16x16x128_f8f6f4 a[84:87], v[136:143], v[8:15], a[84:87] cbsz:1 blgp:1\n" \
  "buffer_load_dwordx4 v[104:107], v163, s[36:39], s75 offen offset:2048\n" \
  "v_mfma_f32_16x16x128_f8f6f4 a[88:91], v[136:143], v[16:23], a[88:91] cbsz:1 blgp:1\n" \
  "v_mfma_f32_16x16x128_f8f6f4 a[92:95], v[136:143], v[24:31], a[92:95] cbsz:1 blgp:1\n" \
  "s_waitcnt lgkmcnt(2)\n" \
  "v_accvgpr_read_b32 v184, a96\n" \
  "v_accvgpr_read_b32 v185, a97\n" \
  "v_accvgpr_read_b32 v186, a98\n" \
  "v_accvgpr_read_b32 v187, a99\n" \
  "v_mfma_f32_16x16x128_f8f6f4 a[96:99], v[144:151], v[0:7], 0 cbsz:1 blgp:1\n" \
  "s_add_u32 m0, s69, 110592\n" \
  "ds_read_b128 v[128:131], v160 offset:16384\n" \
  "ds_read_b128 v[132:135], v160 offset:17408\n" \
  "v_accvgpr_read_b32 v188, a100\n" \
  "v_accvgpr_read_b32 v189, a101\n" \
  "v_accvgpr_read_b32 v190, a102\n" \
  "v_accvgpr_read_b32 v191, a103\n" \
  "v_mfma_f32_16x16x128_f8f6f4 a[100:103], v[144:151], v[8:15], 0 cbsz:1 blgp:1\n" \
  "buffer_load_dwordx4 v163, s[36:39], s59 offen lds\n" \
  "v_accvgpr_read_b32 v192, a104\n" \
  "v_accvgpr_read_b32 v193, a105\n" \
  "v_accvgpr_read_b32 v194, a106\n" \
  "v_accvgpr_read_b32 v195, a107\n" \
  "v_mfma_f32_16x16x128_f8f6f4 a[104:107], v[144:151], v[16:23], 0 cbsz:1 blgp:1\n" \
  "v_accvgpr_read_b32 v196, a108\n" \
  "v_accvgpr_read_b32 v197, a109\n" \
  "v_accvgpr_read_b32 v198, a110\n" \
  "v_accvgpr_read_b32 v199, a111\n" \
  "v_mfma_f32_16x16x128_f8f6f4 a[108:111], v[144:151], v[24:31], 0 cbsz:1 blgp:1\n" \
  "s_waitcnt lgkmcnt(2)\n" \
  "v_accvgpr_read_b32 v200, a112\n" \
  "v_accvgpr_read_b32 v201, a113\n" \
  "v_accvgpr_read_b32 v202, a114\n" \
  "v_accvgpr_read_b32 v203, a115\n" \
  "v_mfma_f32_16x16x128_f8f6f4 a[112:115], v[152:159], v[0:7], 0 cbsz:1 blgp:1\n" \
  "ds_read_b128 v[136:139], v160 offset:18432\n" \
  "ds_read_b128 v[140:143], v160 offset:19456\n" \
  "v_accvgpr_read_b32 v204, a116\n" \
  "v_accvgpr_read_b32 v205, a117\n" \
  "v_accvgpr_read_b32 v206, a118\n" \
  "v_accvgpr_read_b32 v207, a119\n" \
  "v_mfma_f32_16x16x128_f8f6f4 a[116:119], v[152:159], v[8:15], 0 cbsz:1 blgp:1\n" \
  "buffer_load_dwordx4 v[108:111], v163, s[36:39], s75 offen offset:3072\n" \
  "v_accvgpr_read_b32 v208, a120\n" \
  "v_accvgpr_read_b32 v209, a121\n" \
  "v_accvgpr_read_b32 v210, a122\n" \
  "v_accvgpr_read_b32 v211, a123\n" \
  "v_mfma_f32_16x16x128_f8f6f4 a[120:123], v[152:159], v[16:23], 0 cbsz:1 blgp:1\n" \
  "v_accvgpr_read_b32 v212, a124\n" \
  "v_accvgpr_read_b32 v213, a125\n" \
  "v_accvgpr_read_b32 v214, a126\n" \
  "v_accvgpr_read_b32 v215, a127\n" \
  "v_mfma_f32_16x16x128_f8f6f4 a[124:127], v[152:159], v[24:31], 0 cbsz:1 blgp:1\n" \
  "s_waitcnt lgkmcnt(2)\n" \
  "v_mfma_f32_16x16x128_f8f6f4 a[128:131], v[128:135], v[0:7], a[128:131] cbsz:1 blgp:1\n" \
  "s_add_u32 m0, s69, 114688\n" \
  "ds_read_b128 v[144:147], v160 offset:20480\n" \
  "ds_read_b128 v[148:151], v160 offset:21504\n" \
  "v_mfma_f32_16x16x128_f8f6f4 a[132:135], v[128:135], v[8:15], a[132:135] cbsz:1 blgp:1\n" \
  "buffer_load_dwordx4 v163, s[36:39], s60 offen lds\n" \
  "v_mfma_f32_16x16x128_f8f6f4 a[136:139], v[128:135], v[16:23], a[136:139] cbsz:1 blgp:1\n" \
  "v_mfma_f32_16x16x128_f8f6f4 a[140:143], v[128:135], v[24:31], a[140:143] cbsz:1 blgp:1\n" \
  "s_waitcnt lgkmcnt(2)\n" \
  "v_mfma_f32_16x16x128_f8f6f4 a[144:147], v[136:143], v[0:7], a[144:147] cbsz:1 blgp:1\n" \
  "ds_read_b128 v[152:155], v160 offset:22528\n" \
  "ds_read_b128 v[156:159], v160 offset:23552\n" \
  "v_mfma_f32_16x16x128_f8f6f4 a[148:151], v[136:143], v[8:15], a[148:151] cbsz:1 blgp:1\n" \
  "buffer_load_dwordx4 v[112:115], v164, s[36:39], s75 offen offset:0\n" \
  "v_mfma_f32_16x16x128_f8f6f4 a[152:155], v[136:143], v[16:23], a[152:155] cbsz:1 blgp:1\n" \
  "v_mfma_f32_16x16x128_f8f6f4 a[156:159], v[136:143], v[24:31], a[156:159] cbsz:1 blgp:1\n" \
  "s_waitcnt lgkmcnt(2)\n" \
  "v_mfma_f32_16x16x128_f8f6f4 a[160:163], v[144:151], v[0:7], a[160:163] cbsz:1 blgp:1\n" \
  "s_add_u32 m0, s69, 118784\n" \
  "ds_read_b128 v[128:131], v160 offset:24576\n" \
  "ds_read_b128 v[132:135], v160 offset:25600\n" \
  "v_mfma_f32_16x16x128_f8f6f4 a[164:167], v[144:151], v[8:15], a[164:167] cbsz:1 blgp:1\n" \
  "buffer_load_dwordx4 v163, s[36:39], s61 offen lds\n" \
  "v_mfma_f32_16x16x128_f8f6f4 a[168:171], v[144:151], v[16:23], a[168:171] cbsz:1 blgp:1\n" \
  "v_mfma_f32_16x16x128_f8f6f4 a[172:175], v[144:151], v[24:31], a[172:175] cbsz:1 blgp:1\n" \
  "s_and_b32 s73, s66, 0xf00001ff\n" \
  "s_waitcnt lgkmcnt(2)\n" \
  "v_mfma_f32_16x16x128_f8f6f4 a[176:179], v[152:159], v[0:7], a[176:179] cbsz:1 blgp:1\n" \
  "ds_read_b128 v[136:139], v160 offset:26624\n" \
  "ds_read_b128 v[140:143], v160 offset:27648\n" \
  "v_mfma_f32_16x16x128_f8f6f4 a[180:183], v[152:159], v[8:15], a[180:183] cbsz:1 blgp:1\n" \
  "buffer_load_dwordx4 v[116:119], v164, s[36:39], s75 offen offset:1024\n" \
  "v_mfma_f32_16x16x128_f8f6f4 a[184:187], v[152:159], v[16:23], a[184:187] cbsz:1 blgp:1\n" \
  "v_mfma_f32_16x16x128_f8f6f4 a[188:191], v[152:159], v[24:31], a[188:191] cbsz:1 blgp:1\n" \
  "s_waitcnt lgkmcnt(2)\n" \
  "v_mfma_f32_16x16x128_f8f6f4 a[192:195], v[128:135], v[0:7], a[192:195] cbsz:1 blgp:1\n" \
  "s_add_u32 m0, s69, 122880\n" \
  "ds_read_b128 v[144:147], v160 offset:28672\n" \
  "ds_read_b128 v[148:151], v160 offset:29696\n" \
  "v_mfma_f32_16x16x128_f8f6f4 a[196:199], v[128:135], v[8:15], a[196:199] cbsz:1 blgp:1\n" \
  "buffer_load_dwordx4 v163, s[36:39], s62 offen lds\n" \
  "v_mfma_f32_16x16x128_f8f6f4 a[200:203], v[128:135], v[16:23], a[200:203] cbsz:1 blgp:1\n" \
  "v_mfma_f32_16x16x128_f8f6f4 a[204:207], v[128:135], v[24:31], a[204:207] cbsz:1 blgp:1\n" \
  "s_waitcnt lgkmcnt(2)\n" \
  "v_mfma_f32_16x16x128_f8f6f4 a[208:211], v[136:143], v[0:7], a[208:211] cbsz:1 blgp:1\n" \
  "v_mul_f32 v184, s78, v184\n" \
  "v_mul_f32 v185, s78, v185\n" \
  "v_mul_f32 v186, s78, v186\n" \
  "v_mul_f32 v187, s78, v187\n" \
  "ds_read_b128 v[152:155], v160 offset:30720\n" \
  "ds_read_b128 v[156:159], v160 offset:31744\n" \
  "v_mfma_f32_16x16x128_f8f6f4 a[212:215], v[136:143], v[8:15], a[212:215] cbsz:1 blgp:1\n" \
  "buffer_load_dwordx4 v[120:123], v164, s[36:39], s75 offen offset:2048\n" \
  "v_mfma_f32_16x16x128_f8f6f4 a[216:219], v[136:143], v[16:23], a[216:219] cbsz:1 blgp:1\n" \
  "buffer_store_dwordx4 v[184:187], v167, s[40:43], s48 offen offset:0 nt\n" \
  "v_mul_f32 v200, s78, v200\n" \
  "v_mul_f32 v201, s78, v201\n" \
  "v_mul_f32 v202, s78, v202\n" \
  "v_mul_f32 v203, s78, v203\n" \
  "v_mfma_f32_16x16x128_f8f6f4 a[220:223], v[136:143], v[24:31], a[220:223] cbsz:1 blgp:1\n" \
  "buffer_store_dwordx4 v[200:203], v167, s[40:43], s48 offen offset:64 nt\n" \
  "s_bfe_u32 m0, s66, 0x40018\n" \
  "s_nop 0\n" \
  "s_movreld_b32 s56, s77\n" \
  "s_cmp_eq_u32 s73, s88\n" \
  "s_cbranch_scc0 Lm1_%=\n" \
  "s_branch Lf1_%=\n" \
  ".p2align 6\n" \
  "Le0_5_%=:\n" \
  "s_waitcnt vmcnt(36) lgkmcnt(0)\n" \
  "s_barrier\n" \
  "s_waitcnt lgkmcnt(2)\n" \
  "v_mfma_f32_16x16x128_f8f6f4 a[224:227], v[144:151], v[96:103], a[224:227] cbsz:1 blgp:1\n" \
  "s_add_u32 m0, s69, 126976\n" \
  "ds_read_b128 v[128:131], v160 offset:0\n" \
  "ds_read_b128 v[132:135], v160 offset:1024\n" \
  "v_mfma_f32_16x16x128_f8f6f4 a[228:231], v[144:151], v[104:111], a[228:231] cbsz:1 blgp:1\n" \
  "buffer_load_dwordx4 v163, s[36:39], s63 offen lds\n" \
  "v_mfma_f32_16x16x128_f8f6f4 a[232:235], v[144:151], v[112:119], a[232:235] cbsz:1 blgp:1\n" \
  "v_readfirstlane_b32 s89, v218\n" \
  "v_mfma_f32_16x16x128_f8f6f4 a[236:239], v[144:151], v[120:127], a[236:239] cbsz:1 blgp:1\n" \
  "s_bitcmp1_b32 s65, 18\n" \
  "s_cselect_b32 s42, s72, 0\n" \
  "s_mov_b32 s48, s71\n" \
  "s_waitcnt lgkmcnt(2)\n" \
  "v_mfma_f32_16x16x128_f8f6f4 a[240:243], v[152:159], v[96:103], a[240:243] cbsz:1 blgp:1\n" \
  "ds_read_b128 v[136:139], v160 offset:2048\n" \
  "ds_read_b128 v[140:143], v160 offset:3072\n" \
  "v_mfma_f32_16x16x128_f8f6f4 a[244:247], v[152:159], v[104:111], a[244:247] cbsz:1 blgp:1\n" \
  "v_mfma_f32_16x16x128_f8f6f4 a[248:251], v[152:159], v[112:119], a[248:251] cbsz:1 blgp:1\n" \
  "v_mfma_f32_16x16x128_f8f6f4 a[252:255], v[152:159], v[120:127], a[252:255] cbsz:1 blgp:1\n" \
  "s_add_u32 s49, s71, 1048576\n" \
  "s_add_u32 s50, s71, 2097152\n" \
  "s_add_u32 s51, s71, 3145728\n" \
  "s_waitcnt lgkmcnt(2)\n" \
  "v_mfma_f32_16x16x128_f8f6f4 a[0:3], v[128:135], v[0:7], a[0:3] cbsz:1 blgp:1\n" \
  "s_add_u32 m0, s69, 98304\n" \
  "ds_read_b128 v[144:147], v160 offset:4096\n" \
  "ds_read_b128 v[148:151], v160 offset:5120\n" \
  "v_mfma_f32_16x16x128_f8f6f4 a[4:7], v[128:135], v[8:15], a[4:7] cbsz:1 blgp:1\n" \
  "buffer_load_dwordx4 v163, s[36:39], s56 offen lds\n" \
  "v_mfma_f32_16x16x128_f8f6f4 a[8:11], v[128:135], v[16:23], a[8:11] cbsz:1 blgp:1\n" \
  "v_mfma_f32_16x16x128_f8f6f4 a[12:15], v[128:135], v[24:31], a[12:15] cbsz:1 blgp:1\n" \
  "s_cmp_eq_u32 s70, 64\n" \
  "s_cbranch_scc1 Lp29_%=\n" \
  "Lq29_%=:\n" \
  "s_waitcnt lgkmcnt(2)\n" \
  "v_mfma_f32_16x16x128_f8f6f4 a[16:19], v[136:143], v[0:7], a[16:19] cbsz:1 blgp:1\n" \
  "ds_read_b128 v[152:155], v160 offset:6144\n" \
  "ds_read_b128 v[156:159], v160 offset:7168\n" \
  "v_mfma_f32_16x16x128_f8f6f4 a[20:23], v[136:143], v[8:15], a[20:23] cbsz:1 blgp:1\n" \
  "buffer_load_dwordx4 v[96:99], v163, s[36:39], s75 offen offset:0\n" \
  "v_mfma_f32_16x16x128_f8f6f4 a[24:27], v[136:143], v[16:23], a[24:27] cbsz:1 blgp:1\n" \
  "s_max_u32 s68, s68, s89\n" \
  "v_mfma_f32_16x16x128_f8f6f4 a[28:31], v[136:143], v[24:31], a[28:31] cbsz:1 blgp:1\n" \
  "v_readlane_b32 s67, v175, s70\n" \
  "v_readlane_b32 s66, v178, s70\n" \
  "s_waitcnt lgkmcnt(2)\n" \
  "v_mfma_f32_16x16x128_f8f6f4 a[32:35], v[144:151], v[0:7], a[32:35] cbsz:1 blgp:1\n" \
  "s_add_u32 m0, s69, 102400\n" \
  "ds_read_b128 v[128:131], v160 offset:8192\n" \
  "ds_read_b128 v[132:135], v160 offset:9216\n" \
  "v_mfma_f32_16x16x128_f8f6f4 a[36:39], v[144:151], v[8:15], a[36:39] cbsz:1 blgp:1\n" \
  "buffer_load_dwordx4 v163, s[36:39], s57 offen lds\n" \
  "buffer_load_dwordx4 v[124:127], v164, s[36:39], s75 offen offset:3072\n" \
  "v_mfma_f32_16x16x128_f8f6f4 a[40:43], v[144:151], v[16:23], a[40:43] cbsz:1 blgp:1\n" \
  "v_mfma_f32_16x16x128_f8f6f4 a[44:47], v[144:151], v[24:31], a[44:47] cbsz:1 blgp:1\n" \
  "v_readlane_b32 s76, v169, s70\n" \
  "v_readlane_b32 s77, v172, s70\n" \
  "s_waitcnt lgkmcnt(2)\n" \
  "v_mfma_f32_16x16x128_f8f6f4 a[48:51], v[152:159], v[0:7], a[48:51] cbsz:1 blgp:1\n" \
  "ds_read_b128 v[136:139], v160 offset:10240\n" \
  "ds_read_b128 v[140:143], v160 offset:11264\n" \
  "v_mfma_f32_16x16x128_f8f6f4 a[52:55], v[152:159], v[8:15], a[52:55] cbsz:1 blgp:1\n" \
  "buffer_load_dwordx4 v[100:103], v163, s[36:39], s75 offen offset:1024\n" \
  "v_mfma_f32_16x16x128_f8f6f4 a[56:59], v[152:159], v[16:23], a[56:59] cbsz:1 blgp:1\n" \
  "v_mfma_f32_16x16x128_f8f6f4 a[60:63], v[152:159], v[24:31], a[60:63] cbsz:1 blgp:1\n" \
  "s_add_u32 s70, s70, 1\n" \
  "v_add_u32 v165, s76, v162\n" \
  "s_waitcnt lgkmcnt(2)\n" \
  "v_mfma_f32_16x16x128_f8f6f4 a[64:67], v[128:135], v[0:7], a[64:67] cbsz:1 blgp:1\n" \
  "s_add_u32 m0, s69, 106496\n" \
  "ds_read_b128 v[144:147], v160 offset:12288\n" \
  "ds_read_b128 v[148:151], v160 offset:13312\n" \
  "v_mfma_f32_16x16x128_f8f6f4 a[68:71], v[128:135], v[8:15], a[68:71] cbsz:1 blgp:1\n" \
  "buffer_load_dwordx4 v163, s[36:39], s58 offen lds\n" \
  "v_mfma_f32_16x16x128_f8f6f4 a[72:75], v[128:135], v[16:23], a[72:75] cbsz:1 blgp:1\n" \
  "v_mfma_f32_16x16x128_f8f6f4 a[76:79], v[128:135], v[24:31], a[76:79] cbsz:1 blgp:1\n" \
  "v_add_u32 v166, 4096, v165\n" \
  "s_cmp_gt_u32 s67, s68\n" \
  "s_cbranch_scc1 Lp30_%=\n" \
  "Lq30_%=:\n" \
  "s_waitcnt lgkmcnt(2)\n" \
  "v_mfma_f32_16x16x128_f8f6f4 a[80:83], v[136:143], v[0:7], a[80:83] cbsz:1 blgp:1\n" \
  "ds_read_b128 v[152:155], v160 offset:14336\n" \
  "ds_read_b128 v[156:159], v160 offset:15360\n" \
  "v_mfma_f32_16x16x128_f8f6f4 a[84:87], v[136:143], v[8:15], a[84:87] cbsz:1 blgp:1\n" \
  "buffer_load_dwordx4 v[104:107], v163, s[36:39], s75 offen offset:2048\n" \
  "v_mfma_f32_16x16x128_f8f6f4 a[88:91], v[136:143], v[16:23], a[88:91] cbsz:1 blgp:1\n" \
  "v_mfma_f32_16x16x128_f8f6f4 a[92:95], v[136:143], v[24:31], a[92:95] cbsz:1 blgp:1\n" \
  "s_waitcnt lgkmcnt(2)\n" \
  "v_mfma_f32_16x16x128_f8f6f4 a[96:99], v[144:151], v[0:7], a[96:99] cbsz:1 blgp:1\n" \
  "s_add_u32 m0, s69, 110592\n" \
  "ds_read_b128 v[128:131], v160 offset:16384\n" \
  "ds_read_b128 v[132:135], v160 offset:17408\n" \
  "v_mfma_f32_16x16x128_f8f6f4 a[100:103], v[144:151], v[8:15], a[100:103] cbsz:1 blgp:1\n" \
  "buffer_load_dwordx4 v163, s[36:39], s59 offen lds\n" \
  "v_mfma_f32_16x16x128_f8f6f4 a[104:107], v[144:151], v[16:23], a[104:107] cbsz:1 blgp:1\n" \
  "v_mfma_f32_16x16x128_f8f6f4 a[108:111], v[144:151], v[24:31], a[108:111] cbsz:1 blgp:1\n" \
  "s_waitcnt lgkmcnt(2)\n" \
  "v_mfma_f32_16x16x128_f8f6f4 a[112:115], v[152:159], v[0:7], a[112:115] cbsz:1 blgp:1\n" \
  "ds_read_b128 v[136:139], v160 offset:18432\n" \
  "ds_read_b128 v[140:143], v160 offset:19456\n" \
  "v_mfma_f32_16x16x128_f8f6f4 a[116:119], v[152:159], v[8:15], a[116:119] cbsz:1 blgp:1\n" \
  "buffer_load_dwordx4 v[108:111], v163, s[36:39], s75 offen offset:3072\n" \
  "v_mfma_f32_16x16x128_f8f6f4 a[120:123], v[152:159], v[16:23], a[120:123] cbsz:1 blgp:1\n" \
  "v_mfma_f32_16x16x128_f8f6f4 a[124:127], v[152:159], v[24:31], a[124:127] cbsz:1 blgp:1\n" \
  "s_waitcnt lgkmcnt(2)\n" \
  "v_accvgpr_read_b32 v184, a128\n" \
  "v_accvgpr_read_b32 v185, a129\n" \
  "v_accvgpr_read_b32 v186, a130\n" \
  "v_accvgpr_read_b32 v187, a131\n" \
  "v_mfma_f32_16x16x128_f8f6f4 a[128:131], v[128:135], v[0:7], 0 cbsz:1 blgp:1\n" \
  "s_add_u32 m0, s69, 114688\n" \
  "ds_read_b128 v[144:147], v160 offset:20480\n" \
  "ds_read_b128 v[148:151], v160 offset:21504\n" \
  "v_accvgpr_read_b32 v188, a132\n" \
  "v_accvgpr_read_b32 v189, a133\n" \
  "v_accvgpr_read_b32 v190, a134\n" \
  "v_accvgpr_read_b32 v191, a135\n" \
  "v_mfma_f32_16x16x128_f8f6f4 a[132:135], v[128:135], v[8:15], 0 cbsz:1 blgp:1\n" \
  "buffer_load_dwordx4 v163, s[36:39], s60 offen lds\n" \
  "v_accvgpr_read_b32 v192, a136\n" \
  "v_accvgpr_read_b32 v193, a137\n" \
  "v_accvgpr_read_b32 v194, a138\n" \
  "v_accvgpr_read_b32 v195, a139\n" \
  "v_mfma_f32_16x16x128_f8f6f4 a[136:139], v[128:135], v[16:23], 0 cbsz:1 blgp:1\n" \
  "v_accvgpr_read_b32 v196, a140\n" \
  "v_accvgpr_read_b32 v197, a141\n" \
  "v_accvgpr_read_b32 v198, a142\n" \
  "v_accvgpr_read_b32 v199, a143\n" \
  "v_mfma_f32_16x16x128_f8f6f4 a[140:143], v[128:135], v[24:31], 0 cbsz:1 blgp:1\n" \
  "s_waitcnt lgkmcnt(2)\n" \
  "v_accvgpr_read_b32 v200, a144\n" \
  "v_accvgpr_read_b32 v201, a145\n" \
  "v_accvgpr_read_b32 v202, a146\n" \
  "v_accvgpr_read_b32 v203, a147\n" \
  "v_mfma_f32_16x16x128_f8f6f4 a[144:147], v[136:143], v[0:7], 0 cbsz:1 blgp:1\n" \
  "ds_read_b128 v[152:155], v160 offset:22528\n" \
  "ds_read_b128 v[156:159], v160 offset:23552\n" \
  "v_accvgpr_read_b32 v204, a148\n" \
  "v_accvgpr_read_b32 v205, a149\n" \
  "v_accvgpr_read_b32 v206, a150\n" \
  "v_accvgpr_read_b32 v207, a151\n" \
  "v_mfma_f32_16x16x128_f8f6f4 a[148:151], v[136:143], v[8:15], 0 cbsz:1 blgp:1\n" \
  "buffer_load_dwordx4 v[112:115], v164, s[36:39], s75 offen offset:0\n" \
  "v_accvgpr_read_b32 v208, a152\n" \
  "v_accvgpr_read_b32 v209, a153\n" \
  "v_accvgpr_read_b32 v210, a154\n" \
  "v_accvgpr_read_b32 v211, a155\n" \
  "v_mfma_f32_16x16x128_f8f6f4 a[152:155], v[136:143], v[16:23], 0 cbsz:1 blgp:1\n" \
  "v_accvgpr_read_b32 v212, a156\n" \
  "v_accvgpr_read_b32 v213, a157\n" \
  "v_accvgpr_read_b32 v214, a158\n" \
  "v_accvgpr_read_b32 v215, a159\n" \
  "v_mfma_f32_16x16x128_f8f6f4 a[156:159], v[136:143], v[24:31], 0 cbsz:1 blgp:1\n" \
  "s_waitcnt lgkmcnt(2)\n" \
  "v_mfma_f32_16x16x128_f8f6f4 a[160:163], v[144:151], v[0:7], a[160:163] cbsz:1 blgp:1\n" \
  "s_add_u32 m0, s69, 118784\n" \
  "ds_read_b128 v[128:131], v160 offset:24576\n" \
  "ds_read_b128 v[132:135], v160 offset:25600\n" \
  "v_mfma_f32_16x16x128_f8f6f4 a[164:167], v[144:151], v[8:15], a[164:167] cbsz:1 blgp:1\n" \
  "buffer_load_dwordx4 v163, s[36:39], s61 offen lds\n" \
  "v_mfma_f32_16x16x128_f8f6f4 a[168:171], v[144:151], v[16:23], a[168:171] cbsz:1 blgp:1\n" \
  "v_mfma_f32_16x16x128_f8f6f4 a[172:175], v[144:151], v[24:31], a[172:175] cbsz:1 blgp:1\n" \
  "s_and_b32 s73, s66, 0xf00001ff\n" \
  "s_waitcnt lgkmcnt(2)\n" \
  "v_mfma_f32_16x16x128_f8f6f4 a[176:179], v[152:159], v[0:7], a[176:179] cbsz:1 blgp:1\n" \
  "ds_read_b128 v[136:139], v160 offset:26624\n" \
  "ds_read_b128 v[140:143], v160 offset:27648\n" \
  "v_mfma_f32_16x16x128_f8f6f4 a[180:183], v[152:159], v[8:15], a[180:183] cbsz:1 blgp:1\n" \
  "buffer_load_dwordx4 v[116:119], v164, s[36:39], s75 offen offset:1024\n" \
  "v_mfma_f32_16x16x128_f8f6f4 a[184:187], v[152:159], v[16:23], a[184:187] cbsz:1 blgp:1\n" \
  "v_mfma_f32_16x16x128_f8f6f4 a[188:191], v[152:159], v[24:31], a[188:191] cbsz:1 blgp:1\n" \
  "s_waitcnt lgkmcnt(2)\n" \
  "v_mfma_f32_16x16x128_f8f6f4 a[192:195], v[128:135], v[0:7], a[192:195] cbsz:1 blgp:1\n" \
  "s_add_u32 m0, s69, 122880\n" \
  "ds_read_b128 v[144:147], v160 offset:28672\n" \
  "ds_read_b128 v[148:151], v160 offset:29696\n" \
  "v_mfma_f32_16x16x128_f8f6f4 a[196:199], v[128:135], v[8:15], a[196:199] cbsz:1 blgp:1\n" \
  "buffer_load_dwordx4 v163, s[36:39], s62 offen lds\n" \
  "v_mfma_f32_16x16x128_f8f6f4 a[200:203], v[128:135], v[16:23], a[200:203] cbsz:1 blgp:1\n" \
  "v_mfma_f32_16x16x128_f8f6f4 a[204:207], v[128:135], v[24:31], a[204:207] cbsz:1 blgp:1\n" \
  "s_waitcnt lgkmcnt(2)\n" \
  "v_mfma_f32_16x16x128_f8f6f4 a[208:211], v[136:143], v[0:7], a[208:211] cbsz:1 blgp:1\n" \
  "v_mul_f32 v184, s78, v184\n" \
  "v_mul_f32 v185, s78, v185\n" \
  "v_mul_f32 v186, s78, v186\n" \
  "v_mul_f32 v187, s78, v187\n" \
  "ds_read_b128 v[152:155], v160 offset:30720\n" \
  "ds_read_b128 v[156:159], v160 offset:31744\n" \
  "v_mfma_f32_16x16x128_f8f6f4 a[212:215], v[136:143], v[8:15], a[212:215] cbsz:1 blgp:1\n" \
  "buffer_load_dwordx4 v[120:123], v164, s[36:39], s75 offen offset:2048\n" \
  "v_mfma_f32_16x16x128_f8f6f4 a[216:219], v[136:143], v[16:23], a[216:219] cbsz:1 blgp:1\n" \
  "buffer_store_dwordx4 v[184:187], v167, s[40:43], s48 offen offset:0 nt\n" \
  "v_mul_f32 v200, s78, v200\n" \
  "v_mul_f32 v201, s78, v201\n" \
  "v_mul_f32 v202, s78, v202\n" \
  "v_mul_f32 v203, s78, v203\n" \
  "v_mfma_f32_16x16x128_f8f6f4 a[220:223], v[136:143], v[24:31], a[220:223] cbsz:1 blgp:1\n" \
  "buffer_store_dwordx4 v[200:203], v167, s[40:43], s48 offen offset:64 nt\n" \
  "s_bfe_u32 m0, s66, 0x40018\n" \
  "s_nop 0\n" \
  "s_movreld_b32 s56, s77\n" \
  "s_cmp_eq_u32 s73, s88\n" \
  "s_cbranch_scc0 Lm1_%=\n" \
  "s_branch Lf1_%=\n" \
  ".p2align 6\n" \
  "Le0_6_%=:\n" \
  "s_waitcnt vmcnt(36) lgkmcnt(0)\n" \
  "s_barrier\n" \
  "s_waitcnt lgkmcnt(2)\n" \
  "v_mfma_f32_16x16x128_f8f6f4 a[224:227], v[144:151], v[96:103], a[224:227] cbsz:1 blgp:1\n" \
  "s_add_u32 m0, s69, 126976\n" \
  "ds_read_b128 v[128:131], v160 offset:0\n" \
  "ds_read_b128 v[132:135], v160 offset:1024\n" \
  "v_mfma_f32_16x16x128_f8f6f4 a[228:231], v[144:151], v[104:111], a[228:231] cbsz:1 blgp:1\n" \
  "buffer_load_dwordx4 v163, s[36:39], s63 offen lds\n" \
  "v_mfma_f32_16x16x128_f8f6f4 a[232:235], v[144:151], v[112:119], a[232:235] cbsz:1 blgp:1\n" \
  "v_readfirstlane_b32 s89, v218\n" \
  "v_mfma_f32_16x16x128_f8f6f4 a[236:239], v[144:151], v[120:127], a[236:239] cbsz:1 blgp:1\n" \
  "s_bitcmp1_b32 s65, 18\n" \
  "s_cselect_b32 s42, s72, 0\n" \
  "s_mov_b32 s48, s71\n" \
  "s_waitcnt lgkmcnt(2)\n" \
  "v_mfma_f32_16x16x128_f8f6f4 a[240:243], v[152:159], v[96:103], a[240:243] cbsz:1 blgp:1\n" \
  "ds_read_b128 v[136:139], v160 offset:2048\n" \
  "ds_read_b128 v[140:143], v160 offset:3072\n" \
  "v_mfma_f32_16x16x128_f8f6f4 a[244:247], v[152:159], v[104:111], a[244:247] cbsz:1 blgp:1\n" \
  "v_mfma_f32_16x16x128_f8f6f4 a[248:251], v[152:159], v[112:119], a[248:251] cbsz:1 blgp:1\n" \
  "v_mfma_f32_16x16x128_f8f6f4 a[252:255], v[152:159], v[120:127], a[252:255] cbsz:1 blgp:1\n" \
  "s_add_u32 s49, s71, 1048576\n" \
  "s_add_u32 s50, s71, 2097152\n" \
  "s_add_u32 s51, s71, 3145728\n" \
  "s_waitcnt lgkmcnt(2)\n" \
  "v_mfma_f32_16x16x128_f8f6f4 a[0:3], v[128:135], v[0:7], a[0:3] cbsz:1 blgp:1\n" \
  "s_add_u32 m0, s69, 98304\n" \
  "ds_read_b128 v[144:147], v160 offset:4096\n" \
  "ds_read_b128 v[148:151], v160 offset:5120\n" \
  "v_mfma_f32_16x16x128_f8f6f4 a[4:7], v[128:135], v[8:15], a[4:7] cbsz:1 blgp:1\n" \
  "buffer_load_dwordx4 v163, s[36:39], s56 offen lds\n" \
  "v_mfma_f32_16x16x128_f8f6f4 a[8:11], v[128:135], v[16:23], a[8:11] cbsz:1 blgp:1\n" \
  "v_mfma_f32_16x16x128_f8f6f4 a[12:15], v[128:135], v[24:31], a[12:15] cbsz:1 blgp:1\n" \
  "s_cmp_eq_u32 s70, 64\n" \
  "s_cbranch_scc1 Lp31_%=\n" \
  "Lq31_%=:\n" \
  "s_waitcnt lgkmcnt(2)\n" \
  "v_mfma_f32_16x16x128_f8f6f4 a[16:19], v[136:143], v[0:7], a[16:19] cbsz:1 blgp:1\n" \
  "ds_read_b128 v[152:155], v160 offset:6144\n" \
  "ds_read_b128 v[156:159], v160 offset:7168\n" \
  "v_mfma_f32_16x16x128_f8f6f4 a[20:23], v[136:143], v[8:15], a[20:23] cbsz:1 blgp:1\n" \
  "buffer_load_dwordx4 v[96:99], v163, s[36:39], s75 offen offset:0\n" \
  "v_mfma_f32_16x16x128_f8f6f4 a[24:27], v[136:143], v[16:23], a[24:27] cbsz:1 blgp:1\n" \
  "s_max_u32 s68, s68, s89\n" \
  "v_mfma_f32_16x16x128_f8f6f4 a[28:31], v[136:143], v[24:31], a[28:31] cbsz:1 blgp:1\n" \
  "v_readlane_b32 s67, v175, s70\n" \
  "v_readlane_b32 s66, v178, s70\n" \
  "s_waitcnt lgkmcnt(2)\n" \
  "v_mfma_f32_16x16x128_f8f6f4 a[32:35], v[144:151], v[0:7], a[32:35] cbsz:1 blgp:1\n" \
  "s_add_u32 m0, s69, 102400\n" \
  "ds_read_b128 v[128:131], v160 offset:8192\n" \
  "ds_read_b128 v[132:135], v160 offset:9216\n" \
  "v_mfma_f32_16x16x128_f8f6f4 a[36:39], v[144:151], v[8:15], a[36:39] cbsz:1 blgp:1\n" \
  "buffer_load_dwordx4 v163, s[36:39], s57 offen lds\n" \
  "buffer_load_dwordx4 v[124:127], v164, s[36:39], s75 offen offset:3072\n" \
  "v_mfma_f32_16x16x128_f8f6f4 a[40:43], v[144:151], v[16:23], a[40:43] cbsz:1 blgp:1\n" \
  "v_mfma_f32_16x16x128_f8f6f4 a[44:47], v[144:151], v[24:31], a[44:47] cbsz:1 blgp:1\n" \
  "v_readlane_b32 s76, v169, s70\n" \
  "v_readlane_b32 s77, v172, s70\n" \
  "s_waitcnt lgkmcnt(2)\n" \
  "v_mfma_f32_16x16x128_f8f6f4 a[48:51], v[152:159], v[0:7], a[48:51] cbsz:1 blgp:1\n" \
  "ds_read_b128 v[136:139], v160 offset:10240\n" \
  "ds_read_b128 v[140:143], v160 offset:11264\n" \
  "v_mfma_f32_16x16x128_f8f6f4 a[52:55], v[152:159], v[8:15], a[52:55] cbsz:1 blgp:1\n" \
  "buffer_load_dwordx4 v[100:103], v163, s[36:39], s75 offen offset:1024\n" \
  "v_mfma_f32_16x16x128_f8f6f4 a[56:59], v[152:159], v[16:23], a[56:59] cbsz:1 blgp:1\n" \
  "v_mfma_f32_16x16x128_f8f6f4 a[60:63], v[152:159], v[24:31], a[60:63] cbsz:1 blgp:1\n" \
  "s_add_u32 s70, s70, 1\n" \
  "v_add_u32 v165, s76, v162\n" \
  "s_waitcnt lgkmcnt(2)\n" \
  "v_mfma_f32_16x16x128_f8f6f4 a[64:67], v[128:135], v[0:7], a[64:67] cbsz:1 blgp:1\n" \
  "s_add_u32 m0, s69, 106496\n" \
  "ds_read_b128 v[144:147], v160 offset:12288\n" \
  "ds_read_b128 v[148:151], v160 offset:13312\n" \
  "v_mfma_f32_16x16x128_f8f6f4 a[68:71], v[128:135], v[8:15], a[68:71] cbsz:1 blgp:1\n" \
  "buffer_load_dwordx4 v163, s[36:39], s58 offen lds\n" \
  "v_mfma_f32_16x16x128_f8f6f4 a[72:75], v[128:135], v[16:23], a[72:75] cbsz:1 blgp:1\n" \
  "v_mfma_f32_16x16x128_f8f6f4 a[76:79], v[128:135], v[24:31], a[76:79] cbsz:1 blgp:1\n" \
  "v_add_u32 v166, 4096, v165\n" \
  "s_cmp_gt_u32 s67, s68\n" \
  "s_cbranch_scc1 Lp32_%=\n" \
  "Lq32_%=:\n" \
  "s_waitcnt lgkmcnt(2)\n" \
  "v_mfma_f32_16x16x128_f8f6f4 a[80:83], v[136:143], v[0:7], a[80:83] cbsz:1 blgp:1\n" \
  "ds_read_b128 v[152:155], v160 offset:14336\n" \
  "ds_read_b128 v[156:159], v160 offset:15360\n" \
  "v_mfma_f32_16x16x128_f8f6f4 a[84:87], v[136:143], v[8:15], a[84:87] cbsz:1 blgp:1\n" \
  "buffer_load_dwordx4 v[104:107], v163, s[36:39], s75 offen offset:2048\n" \
  "v_mfma_f32_16x16x128_f8f6f4 a[88:91], v[136:143], v[16:23], a[88:91] cbsz:1 blgp:1\n" \
  "v_mfma_f32_16x16x128_f8f6f4 a[92:95], v[136:143], v[24:31], a[92:95] cbsz:1 blgp:1\n" \
  "s_waitcnt lgkmcnt(2)\n" \
  "v_mfma_f32_16x16x128_f8f6f4 a[96:99], v[144:151], v[0:7], a[96:99] cbsz:1 blgp:1\n" \
  "s_add_u32 m0, s69, 110592\n" \
  "ds_read_b128 v[128:131], v160 offset:16384\n" \
  "ds_read_b128 v[132:135], v160 offset:17408\n" \
  "v_mfma_f32_16x16x128_f8f6f4 a[100:103], v[144:151], v[8:15], a[100:103] cbsz:1 blgp:1\n" \
  "buffer_load_dwordx4 v163, s[36:39], s59 offen lds\n" \
  "v_mfma_f32_16x16x128_f8f6f4 a[104:107], v[144:151], v[16:23], a[104:107] cbsz:1 blgp:1\n" \
  "v_mfma_f32_16x16x128_f8f6f4 a[108:111], v[144:151], v[24:31], a[108:111] cbsz:1 blgp:1\n" \
  "s_waitcnt lgkmcnt(2)\n" \
  "v_mfma_f32_16x16x128_f8f6f4 a[112:115], v[152:159], v[0:7], a[112:115] cbsz:1 blgp:1\n" \
  "ds_read_b128 v[136:139], v160 offset:18432\n" \
  "ds_read_b128 v[140:143], v160 offset:19456\n" \
  "v_mfma_f32_16x16x128_f8f6f4 a[116:119], v[152:159], v[8:15], a[116:119] cbsz:1 blgp:1\n" \
  "buffer_load_dwordx4 v[108:111], v163, s[36:39], s75 offen offset:3072\n" \
  "v_mfma_f32_16x16x128_f8f6f4 a[120:123], v[152:159], v[16:23], a[120:123] cbsz:1 blgp:1\n" \
  "v_mfma_f32_16x16x128_f8f6f4 a[124:127], v[152:159], v[24:31], a[124:127] cbsz:1 blgp:1\n" \
  "s_waitcnt lgkmcnt(2)\n" \
  "v_mfma_f32_16x16x128_f8f6f4 a[128:131], v[128:135], v[0:7], a[128:131] cbsz:1 blgp:1\n" \
  "s_add_u32 m0, s69, 114688\n" \
  "ds_read_b128 v[144:147], v160 offset:20480\n" \
  "ds_read_b128 v[148:151], v160 offset:21504\n" \
  "v_mfma_f32_16x16x128_f8f6f4 a[132:135], v[128:135], v[8:15], a[132:135] cbsz:1 blgp:1\n" \
  "buffer_load_dwordx4 v163, s[36:39], s60 offen lds\n" \
  "v_mfma_f32_16x16x128_f8f6f4 a[136:139], v[128:135], v[16:23], a[136:139] cbsz:1 blgp:1\n" \
  "v_mfma_f32_16x16x128_f8f6f4 a[140:143], v[128:135], v[24:31], a[140:143] cbsz:1 blgp:1\n" \
  "s_waitcnt lgkmcnt(2)\n" \
  "v_mfma_f32_16x16x128_f8f6f4 a[144:147], v[136:143], v[0:7], a[144:147] cbsz:1 blgp:1\n" \
  "ds_read_b128 v[152:155], v160 offset:22528\n" \
  "ds_read_b128 v[156:159], v160 offset:23552\n" \
  "v_mfma_f32_16x16x128_f8f6f4 a[148:151], v[136:143], v[8:15], a[148:151] cbsz:1 blgp:1\n" \
  "buffer_load_dwordx4 v[112:115], v164, s[36:39], s75 offen offset:0\n" \
  "v_mfma_f32_16x16x128_f8f6f4 a[152:155], v[136:143], v[16:23], a[152:155] cbsz:1 blgp:1\n" \
  "v_mfma_f32_16x16x128_f8f6f4 a[156:159], v[136:143], v[24:31], a[156:159] cbsz:1 blgp:1\n" \
  "s_waitcnt lgkmcnt(2)\n" \
  "v_accvgpr_read_b32 v184, a160\n" \
  "v_accvgpr_read_b32 v185, a161\n" \
  "v_accvgpr_read_b32 v186, a162\n" \
  "v_accvgpr_read_b32 v187, a163\n" \
  "v_mfma_f32_16x16x128_f8f6f4 a[160:163], v[144:151], v[0:7], 0 cbsz:1 blgp:1\n" \
  "s_add_u32 m0, s69, 118784\n" \
  "ds_read_b128 v[128:131], v160 offset:24576\n" \
  "ds_read_b128 v[132:135], v160 offset:25600\n" \
  "v_accvgpr_read_b32 v188, a164\n" \
  "v_accvgpr_read_b32 v189, a165\n" \
  "v_accvgpr_read_b32 v190, a166\n" \
  "v_accvgpr_read_b32 v191, a167\n" \
  "v_mfma_f32_16x16x128_f8f6f4 a[164:167], v[144:151], v[8:15], 0 cbsz:1 blgp:1\n" \
  "buffer_load_dwordx4 v163, s[36:39], s61 offen lds\n" \
  "v_accvgpr_read_b32 v192, a168\n" \
  "v_accvgpr_read_b32 v193, a169\n" \
  "v_accvgpr_read_b32 v194, a170\n" \
  "v_accvgpr_read_b32 v195, a171\n" \
  "v_mfma_f32_16x16x128_f8f6f4 a[168:171], v[144:151], v[16:23], 0 cbsz:1 blgp:1\n" \
  "v_accvgpr_read_b32 v196, a172\n" \
  "v_accvgpr_read_b32 v197, a173\n" \
  "v_accvgpr_read_b32 v198, a174\n" \
  "v_accvgpr_read_b32 v199, a175\n" \
  "v_mfma_f32_16x16x128_f8f6f4 a[172:175], v[144:151], v[24:31], 0 cbsz:1 blgp:1\n" \
  "s_and_b32 s73, s66, 0xf00001ff\n" \
  "s_waitcnt lgkmcnt(2)\n" \
  "v_accvgpr_read_b32 v200, a176\n" \
  "v_accvgpr_read_b32 v201, a177\n" \
  "v_accvgpr_read_b32 v202, a178\n" \
  "v_accvgpr_read_b32 v203, a179\n" \
  "v_mfma_f32_16x16x128_f8f6f4 a[176:179], v[152:159], v[0:7], 0 cbsz:1 blgp:1\n" \
  "ds_read_b128 v[136:139], v160 offset:26624\n" \
  "ds_read_b128 v[140:143], v160 offset:27648\n" \
  "v_accvgpr_read_b32 v204, a180\n" \
  "v_accvgpr_read_b32 v205, a181\n" \
  "v_accvgpr_read_b32 v206, a182\n" \
  "v_accvgpr_read_b32 v207, a183\n" \
  "v_mfma_f32_16x16x128_f8f6f4 a[180:183], v[152:159], v[8:15], 0 cbsz:1 blgp:1\n" \
  "buffer_load_dwordx4 v[116:119], v164, s[36:39], s75 offen offset:1024\n" \
  "v_accvgpr_read_b32 v208, a184\n" \
  "v_accvgpr_read_b32 v209, a185\n" \
  "v_accvgpr_read_b32 v210, a186\n" \
  "v_accvgpr_read_b32 v211, a187\n" \
  "v_mfma_f32_16x16x128_f8f6f4 a[184:187], v[152:159], v[16:23], 0 cbsz:1 blgp:1\n" \
  "v_accvgpr_read_b32 v212, a188\n" \
  "v_accvgpr_read_b32 v213, a189\n" \
  "v_accvgpr_read_b32 v214, a190\n" \
  "v_accvgpr_read_b32 v215, a191\n" \
  "v_mfma_f32_16x16x128_f8f6f4 a[188:191], v[152:159], v[24:31], 0 cbsz:1 blgp:1\n" \
  "s_waitcnt lgkmcnt(2)\n" \
  "v_mfma_f32_16x16x128_f8f6f4 a[192:195], v[128:135], v[0:7], a[192:195] cbsz:1 blgp:1\n" \
  "s_add_u32 m0, s69, 122880\n" \
  "ds_read_b128 v[144:147], v160 offset:28672\n" \
  "ds_read_b128 v[148:151], v160 offset:29696\n" \
  "v_mfma_f32_16x16x128_f8f6f4 a[196:199], v[128:135], v[8:15], a[196:199] cbsz:1 blgp:1\n" \
  "buffer_load_dwordx4 v163, s[36:39], s62 offen lds\n" \
  "v_mfma_f32_16x16x128_f8f6f4 a[200:203], v[128:135], v[16:23], a[200:203] cbsz:1 blgp:1\n" \
  "v_mfma_f32_16x16x128_f8f6f4 a[204:207], v[128:135], v[24:31], a[204:207] cbsz:1 blgp:1\n" \
  "s_waitcnt lgkmcnt(2)\n" \
  "v_mfma_f32_16x16x128_f8f6f4 a[208:211], v[136:143], v[0:7], a[208:211] cbsz:1 blgp:1\n" \
  "v_mul_f32 v184, s78, v184\n" \
  "v_mul_f32 v185, s78, v185\n" \
  "v_mul_f32 v186, s78, v186\n" \
  "v_mul_f32 v187, s78, v187\n" \
  "ds_read_b128 v[152:155], v160 offset:30720\n" \
  "ds_read_b128 v[156:159], v160 offset:31744\n" \
  "v_mfma_f32_16x16x128_f8f6f4 a[212:215], v[136:143], v[8:15], a[212:215] cbsz:1 blgp:1\n" \
  "buffer_load_dwordx4 v[120:123], v164, s[36:39], s75 offen offset:2048\n" \
  "v_mfma_f32_16x16x128_f8f6f4 a[216:219], v[136:143], v[16:23], a[216:219] cbsz:1 blgp:1\n" \
  "buffer_store_dwordx4 v[184:187], v167, s[40:43], s48 offen offset:0 nt\n" \
  "v_mul_f32 v200, s78, v200\n" \
  "v_mul_f32 v201, s78, v201\n" \
  "v_mul_f32 v202, s78, v202\n" \
  "v_mul_f32 v203, s78, v203\n" \
  "v_mfma_f32_16x16x128_f8f6f4 a[220:223], v[136:143], v[24:31], a[220:223] cbsz:1 blgp:1\n" \
  "buffer_store_dwordx4 v[200:203], v167, s[40:43], s48 offen offset:64 nt\n" \
  "s_bfe_u32 m0, s66, 0x40018\n" \
  "s_nop 0\n" \
  "s_movreld_b32 s56, s77\n" \
  "s_cmp_eq_u32 s73, s88\n" \
  "s_cbranch_scc0 Lm1_%=\n" \
  "s_branch Lf1_%=\n" \
  ".p2align 6\n" \
  "Le0_7_%=:\n" \
  "s_waitcnt vmcnt(36) lgkmcnt(0)\n" \
  "s_barrier\n" \
  "s_waitcnt lgkmcnt(2)\n" \
  "v_mfma_f32_16x16x128_f8f6f4 a[224:227], v[144:151], v[96:103], a[224:227] cbsz:1 blgp:1\n" \
  "s_add_u32 m0, s69, 126976\n" \
  "ds_read_b128 v[128:131], v160 offset:0\n" \
  "ds_read_b128 v[132:135], v160 offset:1024\n" \
  "v_mfma_f32_16x16x128_f8f6f4 a[228:231], v[144:151], v[104:111], a[228:231] cbsz:1 blgp:1\n" \
  "buffer_load_dwordx4 v163, s[36:39], s63 offen lds\n" \
  "v_mfma_f32_16x16x128_f8f6f4 a[232:235], v[144:151], v[112:119], a[232:235] cbsz:1 blgp:1\n" \
  "v_readfirstlane_b32 s89, v218\n" \
  "v_mfma_f32_16x16x128_f8f6f4 a[236:239], v[144:151], v[120:127], a[236:239] cbsz:1 blgp:1\n" \
  "s_bitcmp1_b32 s65, 18\n" \
  "s_cselect_b32 s42, s72, 0\n" \
  "s_mov_b32 s48, s71\n" \
  "s_waitcnt lgkmcnt(2)\n" \
  "v_mfma_f32_16x16x128_f8f6f4 a[240:243], v[152:159], v[96:103], a[240:243] cbsz:1 blgp:1\n" \
  "ds_read_b128 v[136:139], v160 offset:2048\n" \
  "ds_read_b128 v[140:143], v160 offset:3072\n" \
  "v_mfma_f32_16x16x128_f8f6f4 a[244:247], v[152:159], v[104:111], a[244:247] cbsz:1 blgp:1\n" \
  "v_mfma_f32_16x16x128_f8f6f4 a[248:251], v[152:159], v[112:119], a[248:251] cbsz:1 blgp:1\n" \
  "v_mfma_f32_16x16x128_f8f6f4 a[252:255], v[152:159], v[120:127], a[252:255] cbsz:1 blgp:1\n" \
  "s_add_u32 s49, s71, 1048576\n" \
  "s_add_u32 s50, s71, 2097152\n" \
  "s_add_u32 s51, s71, 3145728\n" \
  "s_waitcnt lgkmcnt(2)\n" \
  "v_mfma_f32_16x16x128_f8f6f4 a[0:3], v[128:135], v[0:7], a[0:3] cbsz:1 blgp:1\n" \
  "s_add_u32 m0, s69, 98304\n" \
  "ds_read_b128 v[144:147], v160 offset:4096\n" \
  "ds_read_b128 v[148:151], v160 offset:5120\n" \
  "v_mfma_f32_16x16x128_f8f6f4 a[4:7], v[128:135], v[8:15], a[4:7] cbsz:1 blgp:1\n" \
  "buffer_load_dwordx4 v163, s[36:39], s56 offen lds\n" \
  "v_mfma_f32_16x16x128_f8f6f4 a[8:11], v[128:135], v[16:23], a[8:11] cbsz:1 blgp:1\n" \
  "v_mfma_f32_16x16x128_f8f6f4 a[12:15], v[128:135], v[24:31], a[12:15] cbsz:1 blgp:1\n" \
  "s_cmp_eq_u32 s70, 64\n" \
  "s_cbranch_scc1 Lp33_%=\n" \
  "Lq33_%=:\n" \
  "s_waitcnt lgkmcnt(2)\n" \
  "v_mfma_f32_16x16x128_f8f6f4 a[16:19], v[136:143], v[0:7], a[16:19] cbsz:1 blgp:1\n" \
  "ds_read_b128 v[152:155], v160 offset:6144\n" \
  "ds_read_b128 v[156:159], v160 offset:7168\n" \
  "v_mfma_f32_16x16x128_f8f6f4 a[20:23], v[136:143], v[8:15], a[20:23] cbsz:1 blgp:1\n" \
  "buffer_load_dwordx4 v[96:99], v163, s[36:39], s75 offen offset:0\n" \
  "v_mfma_f32_16x16x128_f8f6f4 a[24:27], v[136:143], v[16:23], a[24:27] cbsz:1 blgp:1\n" \
  "s_max_u32 s68, s68, s89\n" \
  "v_mfma_f32_16x16x128_f8f6f4 a[28:31], v[136:143], v[24:31], a[28:31] cbsz:1 blgp:1\n" \
  "v_readlane_b32 s67, v175, s70\n" \
  "v_readlane_b32 s66, v178, s70\n" \
  "s_waitcnt lgkmcnt(2)\n" \
  "v_mfma_f32_16x16x128_f8f6f4 a[32:35], v[144:151], v[0:7], a[32:35] cbsz:1 blgp:1\n" \
  "s_add_u32 m0, s69, 102400\n" \
  "ds_read_b128 v[128:131], v160 offset:8192\n" \
  "ds_read_b128 v[132:135], v160 offset:9216\n" \
  "v_mfma_f32_16x16x128_f8f6f4 a[36:39], v[144:151], v[8:15], a[36:39] cbsz:1 blgp:1\n" \
  "buffer_load_dwordx4 v163, s[36:39], s57 offen lds\n" \
  "buffer_load_dwordx4 v[124:127], v164, s[36:39], s75 offen offset:3072\n" \
  "v_mfma_f32_16x16x128_f8f6f4 a[40:43], v[144:151], v[16:23], a[40:43] cbsz:1 blgp:1\n" \
  "v_mfma_f32_16x16x128_f8f6f4 a[44:47], v[144:151], v[24:31], a[44:47] cbsz:1 blgp:1\n" \
  "v_readlane_b32 s76, v169, s70\n" \
  "v_readlane_b32 s77, v172, s70\n" \
  "s_waitcnt lgkmcnt(2)\n" \
  "v_mfma_f32_16x16x128_f8f6f4 a[48:51], v[152:159], v[0:7], a[48:51] cbsz:1 blgp:1\n" \
  "ds_read_b128 v[136:139], v160 offset:10240\n" \
  "ds_read_b128 v[140:143], v160 offset:11264\n" \
  "v_mfma_f32_16x16x128_f8f6f4 a[52:55], v[152:159], v[8:15], a[52:55] cbsz:1 blgp:1\n" \
  "buffer_load_dwordx4 v[100:103], v163, s[36:39], s75 offen offset:1024\n" \
  "v_mfma_f32_16x16x128_f8f6f4 a[56:59], v[152:159], v[16:23], a[56:59] cbsz:1 blgp:1\n" \
  "v_mfma_f32_16x16x128_f8f6f4 a[60:63], v[152:159], v[24:31], a[60:63] cbsz:1 blgp:1\n" \
  "s_add_u32 s70, s70, 1\n" \
  "v_add_u32 v165, s76, v162\n" \
  "s_waitcnt lgkmcnt(2)\n" \
  "v_mfma_f32_16x16x128_f8f6f4 a[64:67], v[128:135], v[0:7], a[64:67] cbsz:1 blgp:1\n" \
  "s_add_u32 m0, s69, 106496\n" \
  "ds_read_b128 v[144:147], v160 offset:12288\n" \
  "ds_read_b128 v[148:151], v160 offset:13312\n" \
  "v_mfma_f32_16x16x128_f8f6f4 a[68:71], v[128:135], v[8:15], a[68:71] cbsz:1 blgp:1\n" \
  "buffer_load_dwordx4 v163, s[36:39], s58 offen lds\n" \
  "v_mfma_f32_16x16x128_f8f6f4 a[72:75], v[128:135], v[16:23], a[72:75] cbsz:1 blgp:1\n" \
  "v_mfma_f32_16x16x128_f8f6f4 a[76:79], v[128:135], v[24:31], a[76:79] cbsz:1 blgp:1\n" \
  "v_add_u32 v166, 4096, v165\n" \
  "s_cmp_gt_u32 s67, s68\n" \
  "s_cbranch_scc1 Lp34_%=\n" \
  "Lq34_%=:\n" \
  "s_waitcnt lgkmcnt(2)\n" \
  "v_mfma_f32_16x16x128_f8f6f4 a[80:83], v[136:143], v[0:7], a[80:83] cbsz:1 blgp:1\n" \
  "ds_read_b128 v[152:155], v160 offset:14336\n" \
  "ds_read_b128 v[156:159], v160 offset:15360\n" \
  "v_mfma_f32_16x16x128_f8f6f4 a[84:87], v[136:143], v[8:15], a[84:87] cbsz:1 blgp:1\n" \
  "buffer_load_dwordx4 v[104:107], v163, s[36:39], s75 offen offset:2048\n" \
  "v_mfma_f32_16x16x128_f8f6f4 a[88:91], v[136:143], v[16:23], a[88:91] cbsz:1 blgp:1\n" \
  "v_mfma_f32_16x16x128_f8f6f4 a[92:95], v[136:143], v[24:31], a[92:95] cbsz:1 blgp:1\n" \
  "s_waitcnt lgkmcnt(2)\n" \
  "v_mfma_f32_16x16x128_f8f6f4 a[96:99], v[144:151], v[0:7], a[96:99] cbsz:1 blgp:1\n" \
  "s_add_u32 m0, s69, 110592\n" \
  "ds_read_b128 v[128:131], v160 offset:16384\n" \
  "ds_read_b128 v[132:135], v160 offset:17408\n" \
  "v_mfma_f32_16x16x128_f8f6f4 a[100:103], v[144:151], v[8:15], a[100:103] cbsz:1 blgp:1\n" \
  "buffer_load_dwordx4 v163, s[36:39], s59 offen lds\n" \
  "v_mfma_f32_16x16x128_f8f6f4 a[104:107], v[144:151], v[16:23], a[104:107] cbsz:1 blgp:1\n" \
  "v_mfma_f32_16x16x128_f8f6f4 a[108:111], v[144:151], v[24:31], a[108:111] cbsz:1 blgp:1\n" \
  "s_waitcnt lgkmcnt(2)\n" \
  "v_mfma_f32_16x16x128_f8f6f4 a[112:115], v[152:159], v[0:7], a[112:115] cbsz:1 blgp:1\n" \
  "ds_read_b128 v[136:139], v160 offset:18432\n" \
  "ds_read_b128 v[140:143], v160 offset:19456\n" \
  "v_mfma_f32_16x16x128_f8f6f4 a[116:119], v[152:159], v[8:15], a[116:119] cbsz:1 blgp:1\n" \
  "buffer_load_dwordx4 v[108:111], v163, s[36:39], s75 offen offset:3072\n" \
  "v_mfma_f32_16x16x128_f8f6f4 a[120:123], v[152:159], v[16:23], a[120:123] cbsz:1 blgp:1\n" \
  "v_mfma_f32_16x16x128_f8f6f4 a[124:127], v[152:159], v[24:31], a[124:127] cbsz:1 blgp:1\n" \
  "s_waitcnt lgkmcnt(2)\n" \
  "v_mfma_f32_16x16x128_f8f6f4 a[128:131], v[128:135], v[0:7], a[128:131] cbsz:1 blgp:1\n" \
  "s_add_u32 m0, s69, 114688\n" \
  "ds_read_b128 v[144:147], v160 offset:20480\n" \
  "ds_read_b128 v[148:151], v160 offset:21504\n" \
  "v_mfma_f32_16x16x128_f8f6f4 a[132:135], v[128:135], v[8:15], a[132:135] cbsz:1 blgp:1\n" \
  "buffer_load_dwordx4 v163, s[36:39], s60 offen lds\n" \
  "v_mfma_f32_16x16x128_f8f6f4 a[136:139], v[128:135], v[16:23], a[136:139] cbsz:1 blgp:1\n" \
  "v_mfma_f32_16x16x128_f8f6f4 a[140:143], v[128:135], v[24:31], a[140:143] cbsz:1 blgp:1\n" \
  "s_waitcnt lgkmcnt(2)\n" \
  "v_mfma_f32_16x16x128_f8f6f4 a[144:147], v[136:143], v[0:7], a[144:147] cbsz:1 blgp:1\n" \
  "ds_read_b128 v[152:155], v160 offset:22528\n" \
  "ds_read_b128 v[156:159], v160 offset:23552\n" \
  "v_mfma_f32_16x16x128_f8f6f4 a[148:151], v[136:143], v[8:15], a[148:151] cbsz:1 blgp:1\n" \
  "buffer_load_dwordx4 v[112:115], v164, s[36:39], s75 offen offset:0\n" \
  "v_mfma_f32_16x16x128_f8f6f4 a[152:155], v[136:143], v[16:23], a[152:155] cbsz:1 blgp:1\n" \
  "v_mfma_f32_16x16x128_f8f6f4 a[156:159], v[136:143], v[24:31], a[156:159] cbsz:1 blgp:1\n" \
  "s_waitcnt lgkmcnt(2)\n" \
  "v_mfma_f32_16x16x128_f8f6f4 a[160:163], v[144:151], v[0:7], a[160:163] cbsz:1 blgp:1\n" \
  "s_add_u32 m0, s69, 118784\n" \
  "ds_read_b128 v[128:131], v160 offset:24576\n" \
  "ds_read_b128 v[132:135], v160 offset:25600\n" \
  "v_mfma_f32_16x16x128_f8f6f4 a[164:167], v[144:151], v[8:15], a[164:167] cbsz:1 blgp:1\n" \
  "buffer_load_dwordx4 v163, s[36:39], s61 offen lds\n" \
  "v_mfma_f32_16x16x128_f8f6f4 a[168:171], v[144:151], v[16:23], a[168:171] cbsz:1 blgp:1\n" \
  "v_mfma_f32_16x16x128_f8f6f4 a[172:175], v[144:151], v[24:31], a[172:175] cbsz:1 blgp:1\n" \
  "s_and_b32 s73, s66, 0xf00001ff\n" \
  "s_waitcnt lgkmcnt(2)\n" \
  "v_mfma_f32_16x16x128_f8f6f4 a[176:179], v[152:159], v[0:7], a[176:179] cbsz:1 blgp:1\n" \
  "ds_read_b128 v[136:139], v160 offset:26624\n" \
  "ds_read_b128 v[140:143], v160 offset:27648\n" \
  "v_mfma_f32_16x16x128_f8f6f4 a[180:183], v[152:159], v[8:15], a[180:183] cbsz:1 blgp:1\n" \
  "buffer_load_dwordx4 v[116:119], v164, s[36:39], s75 offen offset:1024\n" \
  "v_mfma_f32_16x16x128_f8f6f4 a[184:187], v[152:159], v[16:23], a[184:187] cbsz:1 blgp:1\n" \
  "v_mfma_f32_16x16x128_f8f6f4 a[188:191], v[152:159], v[24:31], a[188:191] cbsz:1 blgp:1\n" \
  "s_waitcnt lgkmcnt(2)\n" \
  "v_accvgpr_read_b32 v184, a192\n" \
  "v_accvgpr_read_b32 v185, a193\n" \
  "v_accvgpr_read_b32 v186, a194\n" \
  "v_accvgpr_read_b32 v187, a195\n" \
  "v_mfma_f32_16x16x128_f8f6f4 a[192:195], v[128:135], v[0:7], 0 cbsz:1 blgp:1\n" \
  "s_add_u32 m0, s69, 122880\n" \
  "ds_read_b128 v[144:147], v160 offset:28672\n" \
  "ds_read_b128 v[148:151], v160 offset:29696\n" \
  "v_accvgpr_read_b32 v188, a196\n" \
  "v_accvgpr_read_b32 v189, a197\n" \
  "v_accvgpr_read_b32 v190, a198\n" \
  "v_accvgpr_read_b32 v191, a199\n" \
  "v_mfma_f32_16x16x128_f8f6f4 a[196:199], v[128:135], v[8:15], 0 cbsz:1 blgp:1\n" \
  "buffer_load_dwordx4 v163, s[36:39], s62 offen lds\n" \
  "v_accvgpr_read_b32 v192, a200\n" \
  "v_accvgpr_read_b32 v193, a201\n" \
  "v_accvgpr_read_b32 v194, a202\n" \
  "v_accvgpr_read_b32 v195, a203\n" \
  "v_mfma_f32_16x16x128_f8f6f4 a[200:203], v[128:135], v[16:23], 0 cbsz:1 blgp:1\n" \
  "v_accvgpr_read_b32 v196, a204\n" \
  "v_accvgpr_read_b32 v197, a205\n" \
  "v_accvgpr_read_b32 v198, a206\n" \
  "v_accvgpr_read_b32 v199, a207\n" \
  "v_mfma_f32_16x16x128_f8f6f4 a[204:207], v[128:135], v[24:31], 0 cbsz:1 blgp:1\n" \
  "s_waitcnt lgkmcnt(2)\n" \
  "v_accvgpr_read_b32 v200, a208\n" \
  "v_accvgpr_read_b32 v201, a209\n" \
  "v_accvgpr_read_b32 v202, a210\n" \
  "v_accvgpr_read_b32 v203, a211\n" \
  "v_mfma_f32_16x16x128_f8f6f4 a[208:211], v[136:143], v[0:7], 0 cbsz:1 blgp:1\n" \
  "v_mul_f32 v184, s78, v184\n" \
  "v_mul_f32 v185, s78, v185\n" \
  "v_mul_f32 v186, s78, v186\n" \
  "v_mul_f32 v187, s78, v187\n" \
  "ds_read_b128 v[152:155], v160 offset:30720\n" \
  "ds_read_b128 v[156:159], v160 offset:31744\n" \
  "v_accvgpr_read_b32 v204, a212\n" \
  "v_accvgpr_read_b32 v205, a213\n" \
  "v_accvgpr_read_b32 v206, a214\n" \
  "v_accvgpr_read_b32 v207, a215\n" \
  "v_mfma_f32_16x16x128_f8f6f4 a[212:215], v[136:143], v[8:15], 0 cbsz:1 blgp:1\n" \
  "buffer_load_dwordx4 v[120:123], v164, s[36:39], s75 offen offset:2048\n" \
  "v_accvgpr_read_b32 v208, a216\n" \
  "v_accvgpr_read_b32 v209, a217\n" \
  "v_accvgpr_read_b32 v210, a218\n" \
  "v_accvgpr_read_b32 v211, a219\n" \
  "v_mfma_f32_16x16x128_f8f6f4 a[216:219], v[136:143], v[16:23], 0 cbsz:1 blgp:1\n" \
  "buffer_store_dwordx4 v[184:187], v167, s[40:43], s48 offen offset:0 nt\n" \
  "v_mul_f32 v200, s78, v200\n" \
  "v_mul_f32 v201, s78, v201\n" \
  "v_mul_f32 v202, s78, v202\n" \
  "v_mul_f32 v203, s78, v203\n" \
  "v_accvgpr_read_b32 v212, a220\n" \
  "v_accvgpr_read_b32 v213, a221\n" \
  "v_accvgpr_read_b32 v214, a222\n" \
  "v_accvgpr_read_b32 v215, a223\n" \
  "v_mfma_f32_16x16x128_f8f6f4 a[220:223], v[136:143], v[24:31], 0 cbsz:1 blgp:1\n" \
  "buffer_store_dwordx4 v[200:203], v167, s[40:43], s48 offen offset:64 nt\n" \
  "s_bfe_u32 m0, s66, 0x40018\n" \
  "s_nop 0\n" \
  "s_movreld_b32 s56, s77\n" \
  "s_cmp_eq_u32 s73, s88\n" \
  "s_cbranch_scc0 Lm1_%=\n" \
  "s_branch Lf1_%=\n" \
  ".p2align 6\n" \
  "Lf1_%=:\n" \
  "s_waitcnt vmcnt(36) lgkmcnt(0)\n" \
  "s_barrier\n" \
  "v_mfma_f32_16x16x128_f8f6f4 a[224:227], v[144:151], v[0:7], a[224:227] cbsz:1 blgp:1\n" \
  "s_add_u32 m0, s69, 28672\n" \
  "ds_read_b128 v[128:131], v160 offset:32768\n" \
  "ds_read_b128 v[132:135], v160 offset:33792\n" \
  "v_mfma_f32_16x16x128_f8f6f4 a[228:231], v[144:151], v[8:15], a[228:231] cbsz:1 blgp:1\n" \
  "buffer_load_dwordx4 v165, s[36:39], s63 offen lds\n" \
  "v_mfma_f32_16x16x128_f8f6f4 a[232:235], v[144:151], v[16:23], a[232:235] cbsz:1 blgp:1\n" \
  "v_mfma_f32_16x16x128_f8f6f4 a[236:239], v[144:151], v[24:31], a[236:239] cbsz:1 blgp:1\n" \
  "v_mfma_f32_16x16x128_f8f6f4 a[240:243], v[152:159], v[0:7], a[240:243] cbsz:1 blgp:1\n" \
  "ds_read_b128 v[136:139], v160 offset:34816\n" \
  "ds_read_b128 v[140:143], v160 offset:35840\n" \
  "v_mfma_f32_16x16x128_f8f6f4 a[244:247], v[152:159], v[8:15], a[244:247] cbsz:1 blgp:1\n" \
  "v_mfma_f32_16x16x128_f8f6f4 a[248:251], v[152:159], v[16:23], a[248:251] cbsz:1 blgp:1\n" \
  "v_mfma_f32_16x16x128_f8f6f4 a[252:255], v[152:159], v[24:31], a[252:255] cbsz:1 blgp:1\n" \
  "s_waitcnt lgkmcnt(2)\n" \
  "v_mfma_f32_16x16x128_f8f6f4 a[0:3], v[128:135], v[32:39], a[0:3] cbsz:1 blgp:1\n" \
  "s_add_u32 m0, s69, 0\n" \
  "ds_read_b128 v[144:147], v160 offset:36864\n" \
  "ds_read_b128 v[148:151], v160 offset:37888\n" \
  "v_mfma_f32_16x16x128_f8f6f4 a[4:7], v[128:135], v[40:47], a[4:7] cbsz:1 blgp:1\n" \
  "buffer_load_dwordx4 v165, s[36:39], s56 offen lds\n" \
  "v_mfma_f32_16x16x128_f8f6f4 a[8:11], v[128:135], v[48:55], a[8:11] cbsz:1 blgp:1\n" \
  "v_mfma_f32_16x16x128_f8f6f4 a[12:15], v[128:135], v[56:63], a[12:15] cbsz:1 blgp:1\n" \
  "s_waitcnt lgkmcnt(2)\n" \
  "v_mfma_f32_16x16x128_f8f6f4 a[16:19], v[136:143], v[32:39], a[16:19] cbsz:1 blgp:1\n" \
  "ds_read_b128 v[152:155], v160 offset:38912\n" \
  "ds_read_b128 v[156:159], v160 offset:39936\n" \
  "v_mfma_f32_16x16x128_f8f6f4 a[20:23], v[136:143], v[40:47], a[20:23] cbsz:1 blgp:1\n" \
  "buffer_load_dwordx4 v[0:3], v165, s[36:39], s75 offen offset:0\n" \
  "v_mfma_f32_16x16x128_f8f6f4 a[24:27], v[136:143], v[48:55], a[24:27] cbsz:1 blgp:1\n" \
  "v_mfma_f32_16x16x128_f8f6f4 a[28:31], v[136:143], v[56:63], a[28:31] cbsz:1 blgp:1\n" \
  "v_readlane_b32 s67, v175, s70\n" \
  "v_readlane_b32 s65, v178, s70\n" \
  "s_waitcnt lgkmcnt(2)\n" \
  "v_mfma_f32_16x16x128_f8f6f4 a[32:35], v[144:151], v[32:39], a[32:35] cbsz:1 blgp:1\n" \
  "s_add_u32 m0, s69, 4096\n" \
  "ds_read_b128 v[128:131], v160 offset:40960\n" \
  "ds_read_b128 v[132:135], v160 offset:41984\n" \
  "v_mfma_f32_16x16x128_f8f6f4 a[36:39], v[144:151], v[40:47], a[36:39] cbsz:1 blgp:1\n" \
  "buffer_load_dwordx4 v165, s[36:39], s57 offen lds\n" \
  "buffer_load_dwordx4 v[28:31], v166, s[36:39], s75 offen offset:3072\n" \
  "v_mfma_f32_16x16x128_f8f6f4 a[40:43], v[144:151], v[48:55], a[40:43] cbsz:1 blgp:1\n" \
  "buffer_load_dword v218, off, s[44:47], 0 sc1\n" \
  "v_mfma_f32_16x16x128_f8f6f4 a[44:47], v[144:151], v[56:63], a[44:47] cbsz:1 blgp:1\n" \
  "v_readlane_b32 s76, v169, s70\n" \
  "v_readlane_b32 s77, v172, s70\n" \
  "s_waitcnt lgkmcnt(2)\n" \
  "v_mfma_f32_16x16x128_f8f6f4 a[48:51], v[152:159], v[32:39], a[48:51] cbsz:1 blgp:1\n" \
  "ds_read_b128 v[136:139], v160 offset:43008\n" \
  "ds_read_b128 v[140:143], v160 offset:44032\n" \
  "v_mfma_f32_16x16x128_f8f6f4 a[52:55], v[152:159], v[40:47], a[52:55] cbsz:1 blgp:1\n" \
  "buffer_load_dwordx4 v[4:7], v165, s[36:39], s75 offen offset:1024\n" \
  "v_mfma_f32_16x16x128_f8f6f4 a[56:59], v[152:159], v[48:55], a[56:59] cbsz:1 blgp:1\n" \
  "v_mfma_f32_16x16x128_f8f6f4 a[60:63], v[152:159], v[56:63], a[60:63] cbsz:1 blgp:1\n" \
  "s_add_u32 s70, s70, 1\n" \
  "v_add_u32 v163, s76, v162\n" \
  "s_waitcnt lgkmcnt(2)\n" \
  "v_mfma_f32_16x16x128_f8f6f4 a[64:67], v[128:135], v[32:39], a[64:67] cbsz:1 blgp:1\n" \
  "s_add_u32 m0, s69, 8192\n" \
  "ds_read_b128 v[144:147], v160 offset:45056\n" \
  "ds_read_b128 v[148:151], v160 offset:46080\n" \
  "v_mfma_f32_16x16x128_f8f6f4 a[68:71], v[128:135], v[40:47], a[68:71] cbsz:1 blgp:1\n" \
  "buffer_load_dwordx4 v165, s[36:39], s58 offen lds\n" \
  "v_mfma_f32_16x16x128_f8f6f4 a[72:75], v[128:135], v[48:55], a[72:75] cbsz:1 blgp:1\n" \
  "v_mfma_f32_16x16x128_f8f6f4 a[76:79], v[128:135], v[56:63], a[76:79] cbsz:1 blgp:1\n" \
  "v_add_u32 v164, 4096, v163\n" \
  "s_cmp_gt_u32 s67, s68\n" \
  "s_cbranch_scc1 Lp35_%=\n" \
  "Lq35_%=:\n" \
  "s_waitcnt lgkmcnt(2)\n" \
  "v_mfma_f32_16x16x128_f8f6f4 a[80:83], v[136:143], v[32:39], a[80:83] cbsz:1 blgp:1\n" \
  "ds_read_b128 v[152:155], v160 offset:47104\n" \
  "ds_read_b128 v[156:159], v160 offset:48128\n" \
  "v_mfma_f32_16x16x128_f8f6f4 a[84:87], v[136:143], v[40:47], a[84:87] cbsz:1 blgp:1\n" \
  "buffer_load_dwordx4 v[8:11], v165, s[36:39], s75 offen offset:2048\n" \
  "v_mfma_f32_16x16x128_f8f6f4 a[88:91], v[136:143], v[48:55], a[88:91] cbsz:1 blgp:1\n" \
  "v_mfma_f32_16x16x128_f8f6f4 a[92:95], v[136:143], v[56:63], a[92:95] cbsz:1 blgp:1\n" \
  "s_waitcnt lgkmcnt(2)\n" \
  "v_mfma_f32_16x16x128_f8f6f4 a[96:99], v[144:151], v[32:39], a[96:99] cbsz:1 blgp:1\n" \
  "v_mul_f32 v188, s78, v188\n" \
  "v_mul_f32 v189, s78, v189\n" \
  "v_mul_f32 v190, s78, v190\n" \
  "v_mul_f32 v191, s78, v191\n" \
  "s_add_u32 m0, s69, 12288\n" \
  "ds_read_b128 v[128:131], v160 offset:49152\n" \
  "ds_read_b128 v[132:135], v160 offset:50176\n" \
  "v_mfma_f32_16x16x128_f8f6f4 a[100:103], v[144:151], v[40:47], a[100:103] cbsz:1 blgp:1\n" \
  "buffer_load_dwordx4 v165, s[36:39], s59 offen lds\n" \
  "v_mfma_f32_16x16x128_f8f6f4 a[104:107], v[144:151], v[48:55], a[104:107] cbsz:1 blgp:1\n" \
  "buffer_store_dwordx4 v[188:191], v167, s[40:43], s49 offen offset:0 nt\n" \
  "v_mfma_f32_16x16x128_f8f6f4 a[108:111], v[144:151], v[56:63], a[108:111] cbsz:1 blgp:1\n" \
  "s_waitcnt lgkmcnt(2)\n" \
  "v_mfma_f32_16x16x128_f8f6f4 a[112:115], v[152:159], v[32:39], a[112:115] cbsz:1 blgp:1\n" \
  "ds_read_b128 v[136:139], v160 offset:51200\n" \
  "ds_read_b128 v[140:143], v160 offset:52224\n" \
  "v_mfma_f32_16x16x128_f8f6f4 a[116:119], v[152:159], v[40:47], a[116:119] cbsz:1 blgp:1\n" \
  "buffer_load_dwordx4 v[12:15], v165, s[36:39], s75 offen offset:3072\n" \
  "v_mfma_f32_16x16x128_f8f6f4 a[120:123], v[152:159], v[48:55], a[120:123] cbsz:1 blgp:1\n" \
  "v_mfma_f32_16x16x128_f8f6f4 a[124:127], v[152:159], v[56:63], a[124:127] cbsz:1 blgp:1\n" \
  "s_waitcnt lgkmcnt(2)\n" \
  "v_mfma_f32_16x16x128_f8f6f4 a[128:131], v[128:135], v[32:39], a[128:131] cbsz:1 blgp:1\n" \
  "s_add_u32 m0, s69, 16384\n" \
  "ds_read_b128 v[144:147], v160 offset:53248\n" \
  "ds_read_b128 v[148:151], v160 offset:54272\n" \
  "v_mfma_f32_16x16x128_f8f6f4 a[132:135], v[128:135], v[40:47], a[132:135] cbsz:1 blgp:1\n" \
  "buffer_load_dwordx4 v165, s[36:39], s60 offen lds\n" \
  "v_mfma_f32_16x16x128_f8f6f4 a[136:139], v[128:135], v[48:55], a[136:139] cbsz:1 blgp:1\n" \
  "v_mfma_f32_16x16x128_f8f6f4 a[140:143], v[128:135], v[56:63], a[140:143] cbsz:1 blgp:1\n" \
  "s_waitcnt lgkmcnt(2)\n" \
  "v_mfma_f32_16x16x128_f8f6f4 a[144:147], v[136:143], v[32:39], a[144:147] cbsz:1 blgp:1\n" \
  "ds_read_b128 v[152:155], v160 offset:55296\n" \
  "ds_read_b128 v[156:159], v160 offset:56320\n" \
  "v_mfma_f32_16x16x128_f8f6f4 a[148:151], v[136:143], v[40:47], a[148:151] cbsz:1 blgp:1\n" \
  "buffer_load_dwordx4 v[16:19], v166, s[36:39], s75 offen offset:0\n" \
  "v_mfma_f32_16x16x128_f8f6f4 a[152:155], v[136:143], v[48:55], a[152:155] cbsz:1 blgp:1\n" \
  "v_mfma_f32_16x16x128_f8f6f4 a[156:159], v[136:143], v[56:63], a[156:159] cbsz:1 blgp:1\n" \
  "s_waitcnt lgkmcnt(2)\n" \
  "v_mfma_f32_16x16x128_f8f6f4 a[160:163], v[144:151], v[32:39], a[160:163] cbsz:1 blgp:1\n" \
  "v_mul_f32 v204, s78, v204\n" \
  "v_mul_f32 v205, s78, v205\n" \
  "v_mul_f32 v206, s78, v206\n" \
  "v_mul_f32 v207, s78, v207\n" \
  "s_add_u32 m0, s69, 20480\n" \
  "ds_read_b128 v[128:131], v160 offset:57344\n" \
  "ds_read_b128 v[132:135], v160 offset:58368\n" \
  "v_mfma_f32_16x16x128_f8f6f4 a[164:167], v[144:151], v[40:47], a[164:167] cbsz:1 blgp:1\n" \
  "buffer_load_dwordx4 v165, s[36:39], s61 offen lds\n" \
  "v_mfma_f32_16x16x128_f8f6f4 a[168:171], v[144:151], v[48:55], a[168:171] cbsz:1 blgp:1\n" \
  "buffer_store_dwordx4 v[204:207], v167, s[40:43], s49 offen offset:64 nt\n" \
  "v_mfma_f32_16x16x128_f8f6f4 a[172:175], v[144:151], v[56:63], a[172:175] cbsz:1 blgp:1\n" \
  "s_and_b32 s73, s65, 0xf00001ff\n" \
  "s_waitcnt lgkmcnt(2)\n" \
  "v_mfma_f32_16x16x128_f8f6f4 a[176:179], v[152:159], v[32:39], a[176:179] cbsz:1 blgp:1\n" \
  "ds_read_b128 v[136:139], v160 offset:59392\n" \
  "ds_read_b128 v[140:143], v160 offset:60416\n" \
  "v_mfma_f32_16x16x128_f8f6f4 a[180:183], v[152:159], v[40:47], a[180:183] cbsz:1 blgp:1\n" \
  "buffer_load_dwordx4 v[20:23], v166, s[36:39], s75 offen offset:1024\n" \
  "v_mfma_f32_16x16x128_f8f6f4 a[184:187], v[152:159], v[48:55], a[184:187] cbsz:1 blgp:1\n" \
  "v_mfma_f32_16x16x128_f8f6f4 a[188:191], v[152:159], v[56:63], a[188:191] cbsz:1 blgp:1\n" \
  "s_waitcnt lgkmcnt(2)\n" \
  "v_mfma_f32_16x16x128_f8f6f4 a[192:195], v[128:135], v[32:39], a[192:195] cbsz:1 blgp:1\n" \
  "s_add_u32 m0, s69, 24576\n" \
  "ds_read_b128 v[144:147], v160 offset:61440\n" \
  "ds_read_b128 v[148:151], v160 offset:62464\n" \
  "v_mfma_f32_16x16x128_f8f6f4 a[196:199], v[128:135], v[40:47], a[196:199] cbsz:1 blgp:1\n" \
  "buffer_load_dwordx4 v165, s[36:39], s62 offen lds\n" \
  "v_mfma_f32_16x16x128_f8f6f4 a[200:203], v[128:135], v[48:55], a[200:203] cbsz:1 blgp:1\n" \
  "v_mfma_f32_16x16x128_f8f6f4 a[204:207], v[128:135], v[56:63], a[204:207] cbsz:1 blgp:1\n" \
  "s_waitcnt lgkmcnt(2)\n" \
  "v_mfma_f32_16x16x128_f8f6f4 a[208:211], v[136:143], v[32:39], a[208:211] cbsz:1 blgp:1\n" \
  "ds_read_b128 v[152:155], v160 offset:63488\n" \
  "ds_read_b128 v[156:159], v160 offset:64512\n" \
  "v_mfma_f32_16x16x128_f8f6f4 a[212:215], v[136:143], v[40:47], a[212:215] cbsz:1 blgp:1\n" \
  "buffer_load_dwordx4 v[24:27], v166, s[36:39], s75 offen offset:2048\n" \
  "v_mfma_f32_16x16x128_f8f6f4 a[216:219], v[136:143], v[48:55], a[216:219] cbsz:1 blgp:1\n" \
  "s_bfe_u32 m0, s65, 0x40018\n" \
  "s_nop 0\n" \
  "s_movreld_b32 s56, s77\n" \
  "v_mfma_f32_16x16x128_f8f6f4 a[220:223], v[136:143], v[56:63], a[220:223] cbsz:1 blgp:1\n" \
  "s_cmp_eq_u32 s73, s88\n" \
  "s_cbranch_scc0 Lm2_%=\n" \
  "Lf2_%=:\n" \
  "s_waitcnt vmcnt(36) lgkmcnt(0)\n" \
  "s_barrier\n" \
  "v_mfma_f32_16x16x128_f8f6f4 a[224:227], v[144:151], v[32:39], a[224:227] cbsz:1 blgp:1\n" \
  "s_add_u32 m0, s69, 61440\n" \
  "ds_read_b128 v[128:131], v161 offset:0\n" \
  "ds_read_b128 v[132:135], v161 offset:1024\n" \
  "v_mfma_f32_16x16x128_f8f6f4 a[228:231], v[144:151], v[40:47], a[228:231] cbsz:1 blgp:1\n" \
  "buffer_load_dwordx4 v163, s[36:39], s63 offen lds\n" \
  "v_mfma_f32_16x16x128_f8f6f4 a[232:235], v[144:151], v[48:55], a[232:235] cbsz:1 blgp:1\n" \
  "v_mfma_f32_16x16x128_f8f6f4 a[236:239], v[144:151], v[56:63], a[236:239] cbsz:1 blgp:1\n" \
  "v_mfma_f32_16x16x128_f8f6f4 a[240:243], v[152:159], v[32:39], a[240:243] cbsz:1 blgp:1\n" \
  "ds_read_b128 v[136:139], v161 offset:2048\n" \
  "ds_read_b128 v[140:143], v161 offset:3072\n" \
  "v_mfma_f32_16x16x128_f8f6f4 a[244:247], v[152:159], v[40:47], a[244:247] cbsz:1 blgp:1\n" \
  "v_mfma_f32_16x16x128_f8f6f4 a[248:251], v[152:159], v[48:55], a[248:251] cbsz:1 blgp:1\n" \
  "v_mfma_f32_16x16x128_f8f6f4 a[252:255], v[152:159], v[56:63], a[252:255] cbsz:1 blgp:1\n" \
  "s_waitcnt lgkmcnt(2)\n" \
  "v_mfma_f32_16x16x128_f8f6f4 a[0:3], v[128:135], v[64:71], a[0:3] cbsz:1 blgp:1\n" \
  "s_add_u32 m0, s69, 32768\n" \
  "ds_read_b128 v[144:147], v161 offset:4096\n" \
  "ds_read_b128 v[148:151], v161 offset:5120\n" \
  "v_mfma_f32_16x16x128_f8f6f4 a[4:7], v[128:135], v[72:79], a[4:7] cbsz:1 blgp:1\n" \
  "buffer_load_dwordx4 v163, s[36:39], s56 offen lds\n" \
  "v_mfma_f32_16x16x128_f8f6f4 a[8:11], v[128:135], v[80:87], a[8:11] cbsz:1 blgp:1\n" \
  "v_mfma_f32_16x16x128_f8f6f4 a[12:15], v[128:135], v[88:95], a[12:15] cbsz:1 blgp:1\n" \
  "s_waitcnt lgkmcnt(2)\n" \
  "v_mfma_f32_16x16x128_f8f6f4 a[16:19], v[136:143], v[64:71], a[16:19] cbsz:1 blgp:1\n" \
  "ds_read_b128 v[152:155], v161 offset:6144\n" \
  "ds_read_b128 v[156:159], v161 offset:7168\n" \
  "v_mfma_f32_16x16x128_f8f6f4 a[20:23], v[136:143], v[72:79], a[20:23] cbsz:1 blgp:1\n" \
  "buffer_load_dwordx4 v[32:35], v163, s[36:39], s75 offen offset:0\n" \
  "v_mfma_f32_16x16x128_f8f6f4 a[24:27], v[136:143], v[80:87], a[24:27] cbsz:1 blgp:1\n" \
  "v_mfma_f32_16x16x128_f8f6f4 a[28:31], v[136:143], v[88:95], a[28:31] cbsz:1 blgp:1\n" \
  "v_readlane_b32 s67, v175, s70\n" \
  "v_readlane_b32 s66, v178, s70\n" \
  "s_waitcnt lgkmcnt(2)\n" \
  "v_mfma_f32_16x16x128_f8f6f4 a[32:35], v[144:151], v[64:71], a[32:35] cbsz:1 blgp:1\n" \
  "s_add_u32 m0, s69, 36864\n" \
  "ds_read_b128 v[128:131], v161 offset:8192\n" \
  "ds_read_b128 v[132:135], v161 offset:9216\n" \
  "v_mfma_f32_16x16x128_f8f6f4 a[36:39], v[144:151], v[72:79], a[36:39] cbsz:1 blgp:1\n" \
  "buffer_load_dwordx4 v163, s[36:39], s57 offen lds\n" \
  "buffer_load_dwordx4 v[60:63], v164, s[36:39], s75 offen offset:3072\n" \
  "v_mfma_f32_16x16x128_f8f6f4 a[40:43], v[144:151], v[80:87], a[40:43] cbsz:1 blgp:1\n" \
  "v_mfma_f32_16x16x128_f8f6f4 a[44:47], v[144:151], v[88:95], a[44:47] cbsz:1 blgp:1\n" \
  "v_readlane_b32 s76, v169, s70\n" \
  "v_readlane_b32 s77, v172, s70\n" \
  "s_waitcnt lgkmcnt(2)\n" \
  "v_mfma_f32_16x16x128_f8f6f4 a[48:51], v[152:159], v[64:71], a[48:51] cbsz:1 blgp:1\n" \
  "ds_read_b128 v[136:139], v161 offset:10240\n" \
  "ds_read_b128 v[140:143], v161 offset:11264\n" \
  "v_mfma_f32_16x16x128_f8f6f4 a[52:55], v[152:159], v[72:79], a[52:55] cbsz:1 blgp:1\n" \
  "buffer_load_dwordx4 v[36:39], v163, s[36:39], s75 offen offset:1024\n" \
  "v_mfma_f32_16x16x128_f8f6f4 a[56:59], v[152:159], v[80:87], a[56:59] cbsz:1 blgp:1\n" \
  "v_mfma_f32_16x16x128_f8f6f4 a[60:63], v[152:159], v[88:95], a[60:63] cbsz:1 blgp:1\n" \
  "s_add_u32 s70, s70, 1\n" \
  "v_add_u32 v165, s76, v162\n" \
  "s_waitcnt lgkmcnt(2)\n" \
  "v_mfma_f32_16x16x128_f8f6f4 a[64:67], v[128:135], v[64:71], a[64:67] cbsz:1 blgp:1\n" \
  "s_add_u32 m0, s69, 40960\n" \
  "ds_read_b128 v[144:147], v161 offset:12288\n" \
  "ds_read_b128 v[148:151], v161 offset:13312\n" \
  "v_mfma_f32_16x16x128_f8f6f4 a[68:71], v[128:135], v[72:79], a[68:71] cbsz:1 blgp:1\n" \
  "buffer_load_dwordx4 v163, s[36:39], s58 offen lds\n" \
  "v_mfma_f32_16x16x128_f8f6f4 a[72:75], v[128:135], v[80:87], a[72:75] cbsz:1 blgp:1\n" \
  "v_mfma_f32_16x16x128_f8f6f4 a[76:79], v[128:135], v[88:95], a[76:79] cbsz:1 blgp:1\n" \
  "v_add_u32 v166, 4096, v165\n" \
  "s_cmp_gt_u32 s67, s68\n" \
  "s_cbranch_scc1 Lp36_%=\n" \
  "Lq36_%=:\n" \
  "s_waitcnt lgkmcnt(2)\n" \
  "v_mfma_f32_16x16x128_f8f6f4 a[80:83], v[136:143], v[64:71], a[80:83] cbsz:1 blgp:1\n" \
  "ds_read_b128 v[152:155], v161 offset:14336\n" \
  "ds_read_b128 v[156:159], v161 offset:15360\n" \
  "v_mfma_f32_16x16x128_f8f6f4 a[84:87], v[136:143], v[72:79], a[84:87] cbsz:1 blgp:1\n" \
  "buffer_load_dwordx4 v[40:43], v163, s[36:39], s75 offen offset:2048\n" \
  "v_mfma_f32_16x16x128_f8f6f4 a[88:91], v[136:143], v[80:87], a[88:91] cbsz:1 blgp:1\n" \
  "v_mfma_f32_16x16x128_f8f6f4 a[92:95], v[136:143], v[88:95], a[92:95] cbsz:1 blgp:1\n" \
  "s_waitcnt lgkmcnt(2)\n" \
  "v_mfma_f32_16x16x128_f8f6f4 a[96:99], v[144:151], v[64:71], a[96:99] cbsz:1 blgp:1\n" \
  "v_mul_f32 v192, s78, v192\n" \
  "v_mul_f32 v193, s78, v193\n" \
  "v_mul_f32 v194, s78, v194\n" \
  "v_mul_f32 v195, s78, v195\n" \
  "s_add_u32 m0, s69, 45056\n" \
  "ds_read_b128 v[128:131], v161 offset:16384\n" \
  "ds_read_b128 v[132:135], v161 offset:17408\n" \
  "v_mfma_f32_16x16x128_f8f6f4 a[100:103], v[144:151], v[72:79], a[100:103] cbsz:1 blgp:1\n" \
  "buffer_load_dwordx4 v163, s[36:39], s59 offen lds\n" \
  "v_mfma_f32_16x16x128_f8f6f4 a[104:107], v[144:151], v[80:87], a[104:107] cbsz:1 blgp:1\n" \
  "buffer_store_dwordx4 v[192:195], v167, s[40:43], s50 offen offset:0 nt\n" \
  "v_mfma_f32_16x16x128_f8f6f4 a[108:111], v[144:151], v[88:95], a[108:111] cbsz:1 blgp:1\n" \
  "s_waitcnt lgkmcnt(2)\n" \
  "v_mfma_f32_16x16x128_f8f6f4 a[112:115], v[152:159], v[64:71], a[112:115] cbsz:1 blgp:1\n" \
  "ds_read_b128 v[136:139], v161 offset:18432\n" \
  "ds_read_b128 v[140:143], v161 offset:19456\n" \
  "v_mfma_f32_16x16x128_f8f6f4 a[116:119], v[152:159], v[72:79], a[116:119] cbsz:1 blgp:1\n" \
  "buffer_load_dwordx4 v[44:47], v163, s[36:39], s75 offen offset:3072\n" \
  "v_mfma_f32_16x16x128_f8f6f4 a[120:123], v[152:159], v[80:87], a[120:123] cbsz:1 blgp:1\n" \
  "v_mfma_f32_16x16x128_f8f6f4 a[124:127], v[152:159], v[88:95], a[124:127] cbsz:1 blgp:1\n" \
  "s_waitcnt lgkmcnt(2)\n" \
  "v_mfma_f32_16x16x128_f8f6f4 a[128:131], v[128:135], v[64:71], a[128:131] cbsz:1 blgp:1\n" \
  "s_add_u32 m0, s69, 49152\n" \
  "ds_read_b128 v[144:147], v161 offset:20480\n" \
  "ds_read_b128 v[148:151], v161 offset:21504\n" \
  "v_mfma_f32_16x16x128_f8f6f4 a[132:135], v[128:135], v[72:79], a[132:135] cbsz:1 blgp:1\n" \
  "buffer_load_dwordx4 v163, s[36:39], s60 offen lds\n" \
  "v_mfma_f32_16x16x128_f8f6f4 a[136:139], v[128:135], v[80:87], a[136:139] cbsz:1 blgp:1\n" \
  "v_mfma_f32_16x16x128_f8f6f4 a[140:143], v[128:135], v[88:95], a[140:143] cbsz:1 blgp:1\n" \
  "s_waitcnt lgkmcnt(2)\n" \
  "v_mfma_f32_16x16x128_f8f6f4 a[144:147], v[136:143], v[64:71], a[144:147] cbsz:1 blgp:1\n" \
  "ds_read_b128 v[152:155], v161 offset:22528\n" \
  "ds_read_b128 v[156:159], v161 offset:23552\n" \
  "v_mfma_f32_16x16x128_f8f6f4 a[148:151], v[136:143], v[72:79], a[148:151] cbsz:1 blgp:1\n" \
  "buffer_load_dwordx4 v[48:51], v164, s[36:39], s75 offen offset:0\n" \
  "v_mfma_f32_16x16x128_f8f6f4 a[152:155], v[136:143], v[80:87], a[152:155] cbsz:1 blgp:1\n" \
  "v_mfma_f32_16x16x128_f8f6f4 a[156:159], v[136:143], v[88:95], a[156:159] cbsz:1 blgp:1\n" \
  "s_waitcnt lgkmcnt(2)\n" \
  "v_mfma_f32_16x16x128_f8f6f4 a[160:163], v[144:151], v[64:71], a[160:163] cbsz:1 blgp:1\n" \
  "v_mul_f32 v208, s78, v208\n" \
  "v_mul_f32 v209, s78, v209\n" \
  "v_mul_f32 v210, s78, v210\n" \
  "v_mul_f32 v211, s78, v211\n" \
  "s_add_u32 m0, s69, 53248\n" \
  "ds_read_b128 v[128:131], v161 offset:24576\n" \
  "ds_read_b128 v[132:135], v161 offset:25600\n" \
  "v_mfma_f32_16x16x128_f8f6f4 a[164:167], v[144:151], v[72:79], a[164:167] cbsz:1 blgp:1\n" \
  "buffer_load_dwordx4 v163, s[36:39], s61 offen lds\n" \
  "v_mfma_f32_16x16x128_f8f6f4 a[168:171], v[144:151], v[80:87], a[168:171] cbsz:1 blgp:1\n" \
  "buffer_store_dwordx4 v[208:211], v167, s[40:43], s50 offen offset:64 nt\n" \
  "v_mfma_f32_16x16x128_f8f6f4 a[172:175], v[144:151], v[88:95], a[172:175] cbsz:1 blgp:1\n" \
  "s_and_b32 s73, s66, 0xf00001ff\n" \
  "s_waitcnt lgkmcnt(2)\n" \
  "v_mfma_f32_16x16x128_f8f6f4 a[176:179], v[152:159], v[64:71], a[176:179] cbsz:1 blgp:1\n" \
  "ds_read_b128 v[136:139], v161 offset:26624\n" \
  "ds_read_b128 v[140:143], v161 offset:27648\n" \
  "v_mfma_f32_16x16x128_f8f6f4 a[180:183], v[152:159], v[72:79], a[180:183] cbsz:1 blgp:1\n" \
  "buffer_load_dwordx4 v[52:55], v164, s[36:39], s75 offen offset:1024\n" \
  "v_mfma_f32_16x16x128_f8f6f4 a[184:187], v[152:159], v[80:87], a[184:187] cbsz:1 blgp:1\n" \
  "v_mfma_f32_16x16x128_f8f6f4 a[188:191], v[152:159], v[88:95], a[188:191] cbsz:1 blgp:1\n" \
  "s_waitcnt lgkmcnt(2)\n" \
  "v_mfma_f32_16x16x128_f8f6f4 a[192:195], v[128:135], v[64:71], a[192:195] cbsz:1 blgp:1\n" \
  "s_add_u32 m0, s69, 57344\n" \
  "ds_read_b128 v[144:147], v161 offset:28672\n" \
  "ds_read_b128 v[148:151], v161 offset:29696\n" \
  "v_mfma_f32_16x16x128_f8f6f4 a[196:199], v[128:135], v[72:79], a[196:199] cbsz:1 blgp:1\n" \
  "buffer_load_dwordx4 v163, s[36:39], s62 offen lds\n" \
  "v_mfma_f32_16x16x128_f8f6f4 a[200:203], v[128:135], v[80:87], a[200:203] cbsz:1 blgp:1\n" \
  "v_mfma_f32_16x16x128_f8f6f4 a[204:207], v[128:135], v[88:95], a[204:207] cbsz:1 blgp:1\n" \
  "s_waitcnt lgkmcnt(2)\n" \
  "v_mfma_f32_16x16x128_f8f6f4 a[208:211], v[136:143], v[64:71], a[208:211] cbsz:1 blgp:1\n" \
  "ds_read_b128 v[152:155], v161 offset:30720\n" \
  "ds_read_b128 v[156:159], v161 offset:31744\n" \
  "v_mfma_f32_16x16x128_f8f6f4 a[212:215], v[136:143], v[72:79], a[212:215] cbsz:1 blgp:1\n" \
  "buffer_load_dwordx4 v[56:59], v164, s[36:39], s75 offen offset:2048\n" \
  "v_mfma_f32_16x16x128_f8f6f4 a[216:219], v[136:143], v[80:87], a[216:219] cbsz:1 blgp:1\n" \
  "s_bfe_u32 m0, s66, 0x40018\n" \
  "s_nop 0\n" \
  "s_movreld_b32 s56, s77\n" \
  "v_mfma_f32_16x16x128_f8f6f4 a[220:223], v[136:143], v[88:95], a[220:223] cbsz:1 blgp:1\n" \
  "s_cmp_eq_u32 s73, s88\n" \
  "s_cbranch_scc0 Lm3_%=\n" \
  "Lf3_%=:\n" \
  "s_waitcnt vmcnt(36) lgkmcnt(0)\n" \
  "s_barrier\n" \
  "v_mfma_f32_16x16x128_f8f6f4 a[224:227], v[144:151], v[64:71], a[224:227] cbsz:1 blgp:1\n" \
  "s_add_u32 m0, s69, 94208\n" \
  "ds_read_b128 v[128:131], v161 offset:32768\n" \
  "ds_read_b128 v[132:135], v161 offset:33792\n" \
  "v_mfma_f32_16x16x128_f8f6f4 a[228:231], v[144:151], v[72:79], a[228:231] cbsz:1 blgp:1\n" \
  "buffer_load_dwordx4 v165, s[36:39], s63 offen lds\n" \
  "v_mfma_f32_16x16x128_f8f6f4 a[232:235], v[144:151], v[80:87], a[232:235] cbsz:1 blgp:1\n" \
  "v_mfma_f32_16x16x128_f8f6f4 a[236:239], v[144:151], v[88:95], a[236:239] cbsz:1 blgp:1\n" \
  "v_mfma_f32_16x16x128_f8f6f4 a[240:243], v[152:159], v[64:71], a[240:243] cbsz:1 blgp:1\n" \
  "ds_read_b128 v[136:139], v161 offset:34816\n" \
  "ds_read_b128 v[140:143], v161 offset:35840\n" \
  "v_mfma_f32_16x16x128_f8f6f4 a[244:247], v[152:159], v[72:79], a[244:247] cbsz:1 blgp:1\n" \
  "v_mfma_f32_16x16x128_f8f6f4 a[248:251], v[152:159], v[80:87], a[248:251] cbsz:1 blgp:1\n" \
  "v_mfma_f32_16x16x128_f8f6f4 a[252:255], v[152:159], v[88:95], a[252:255] cbsz:1 blgp:1\n" \
  "s_waitcnt lgkmcnt(2)\n" \
  "v_mfma_f32_16x16x128_f8f6f4 a[0:3], v[128:135], v[96:103], a[0:3] cbsz:1 blgp:1\n" \
  "s_add_u32 m0, s69, 65536\n" \
  "ds_read_b128 v[144:147], v161 offset:36864\n" \
  "ds_read_b128 v[148:151], v161 offset:37888\n" \
  "v_mfma_f32_16x16x128_f8f6f4 a[4:7], v[128:135], v[104:111], a[4:7] cbsz:1 blgp:1\n" \
  "buffer_load_dwordx4 v165, s[36:39], s56 offen lds\n" \
  "v_mfma_f32_16x16x128_f8f6f4 a[8:11], v[128:135], v[112:119], a[8:11] cbsz:1 blgp:1\n" \
  "v_mfma_f32_16x16x128_f8f6f4 a[12:15], v[128:135], v[120:127], a[12:15] cbsz:1 blgp:1\n" \
  "s_waitcnt lgkmcnt(2)\n" \
  "v_mfma_f32_16x16x128_f8f6f4 a[16:19], v[136:143], v[96:103], a[16:19] cbsz:1 blgp:1\n" \
  "ds_read_b128 v[152:155], v161 offset:38912\n" \
  "ds_read_b128 v[156:159], v161 offset:39936\n" \
  "v_mfma_f32_16x16x128_f8f6f4 a[20:23], v[136:143], v[104:111], a[20:23] cbsz:1 blgp:1\n" \
  "buffer_load_dwordx4 v[64:67], v165, s[36:39], s75 offen offset:0\n" \
  "v_mfma_f32_16x16x128_f8f6f4 a[24:27], v[136:143], v[112:119], a[24:27] cbsz:1 blgp:1\n" \
  "v_mfma_f32_16x16x128_f8f6f4 a[28:31], v[136:143], v[120:127], a[28:31] cbsz:1 blgp:1\n" \
  "v_readlane_b32 s67, v175, s70\n" \
  "v_readlane_b32 s65, v178, s70\n" \
  "s_waitcnt lgkmcnt(2)\n" \
  "v_mfma_f32_16x16x128_f8f6f4 a[32:35], v[144:151], v[96:103], a[32:35] cbsz:1 blgp:1\n" \
  "s_add_u32 m0, s69, 69632\n" \
  "ds_read_b128 v[128:131], v161 offset:40960\n" \
  "ds_read_b128 v[132:135], v161 offset:41984\n" \
  "v_mfma_f32_16x16x128_f8f6f4 a[36:39], v[144:151], v[104:111], a[36:39] cbsz:1 blgp:1\n" \
  "buffer_load_dwordx4 v165, s[36:39], s57 offen lds\n" \
  "buffer_load_dwordx4 v[92:95], v166, s[36:39], s75 offen offset:3072\n" \
  "v_mfma_f32_16x16x128_f8f6f4 a[40:43], v[144:151], v[112:119], a[40:43] cbsz:1 blgp:1\n" \
  "v_mfma_f32_16x16x128_f8f6f4 a[44:47], v[144:151], v[120:127], a[44:47] cbsz:1 blgp:1\n" \
  "v_readlane_b32 s76, v169, s70\n" \
  "v_readlane_b32 s77, v172, s70\n" \
  "s_waitcnt lgkmcnt(2)\n" \
  "v_mfma_f32_16x16x128_f8f6f4 a[48:51], v[152:159], v[96:103], a[48:51] cbsz:1 blgp:1\n" \
  "ds_read_b128 v[136:139], v161 offset:43008\n" \
  "ds_read_b128 v[140:143], v161 offset:44032\n" \
  "v_mfma_f32_16x16x128_f8f6f4 a[52:55], v[152:159], v[104:111], a[52:55] cbsz:1 blgp:1\n" \
  "buffer_load_dwordx4 v[68:71], v165, s[36:39], s75 offen offset:1024\n" \
  "v_mfma_f32_16x16x128_f8f6f4 a[56:59], v[152:159], v[112:119], a[56:59] cbsz:1 blgp:1\n" \
  "v_mfma_f32_16x16x128_f8f6f4 a[60:63], v[152:159], v[120:127], a[60:63] cbsz:1 blgp:1\n" \
  "v_readlane_b32 s71, v181, s70\n" \
  "s_add_u32 s70, s70, 1\n" \
  "v_add_u32 v163, s76, v162\n" \
  "s_waitcnt lgkmcnt(2)\n" \
  "v_mfma_f32_16x16x128_f8f6f4 a[64:67], v[128:135], v[96:103], a[64:67] cbsz:1 blgp:1\n" \
  "s_add_u32 m0, s69, 73728\n" \
  "ds_read_b128 v[144:147], v161 offset:45056\n" \
  "ds_read_b128 v[148:151], v161 offset:46080\n" \
  "v_mfma_f32_16x16x128_f8f6f4 a[68:71], v[128:135], v[104:111], a[68:71] cbsz:1 blgp:1\n" \
  "buffer_load_dwordx4 v165, s[36:39], s58 offen lds\n" \
  "v_mfma_f32_16x16x128_f8f6f4 a[72:75], v[128:135], v[112:119], a[72:75] cbsz:1 blgp:1\n" \
  "v_mfma_f32_16x16x128_f8f6f4 a[76:79], v[128:135], v[120:127], a[76:79] cbsz:1 blgp:1\n" \
  "v_add_u32 v164, 4096, v163\n" \
  "s_cmp_gt_u32 s67, s68\n" \
  "s_cbranch_scc1 Lp37_%=\n" \
  "Lq37_%=:\n" \
  "s_waitcnt lgkmcnt(2)\n" \
  "v_mfma_f32_16x16x128_f8f6f4 a[80:83], v[136:143], v[96:103], a[80:83] cbsz:1 blgp:1\n" \
  "ds_read_b128 v[152:155], v161 offset:47104\n" \
  "ds_read_b128 v[156:159], v161 offset:48128\n" \
  "v_mfma_f32_16x16x128_f8f6f4 a[84:87], v[136:143], v[104:111], a[84:87] cbsz:1 blgp:1\n" \
  "buffer_load_dwordx4 v[72:75], v165, s[36:39], s75 offen offset:2048\n" \
  "v_mfma_f32_16x16x128_f8f6f4 a[88:91], v[136:143], v[112:119], a[88:91] cbsz:1 blgp:1\n" \
  "v_mfma_f32_16x16x128_f8f6f4 a[92:95], v[136:143], v[120:127], a[92:95] cbsz:1 blgp:1\n" \
  "s_waitcnt lgkmcnt(2)\n" \
  "v_mfma_f32_16x16x128_f8f6f4 a[96:99], v[144:151], v[96:103], a[96:99] cbsz:1 blgp:1\n" \
  "v_mul_f32 v196, s78, v196\n" \
  "v_mul_f32 v197, s78, v197\n" \
  "v_mul_f32 v198, s78, v198\n" \
  "v_mul_f32 v199, s78, v199\n" \
  "s_add_u32 m0, s69, 77824\n" \
  "ds_read_b128 v[128:131], v161 offset:49152\n" \
  "ds_read_b128 v[132:135], v161 offset:50176\n" \
  "v_mfma_f32_16x16x128_f8f6f4 a[100:103], v[144:151], v[104:111], a[100:103] cbsz:1 blgp:1\n" \
  "buffer_load_dwordx4 v165, s[36:39], s59 offen lds\n" \
  "v_mfma_f32_16x16x128_f8f6f4 a[104:107], v[144:151], v[112:119], a[104:107] cbsz:1 blgp:1\n" \
  "buffer_store_dwordx4 v[196:199], v167, s[40:43], s51 offen offset:0 nt\n" \
  "v_mfma_f32_16x16x128_f8f6f4 a[108:111], v[144:151], v[120:127], a[108:111] cbsz:1 blgp:1\n" \
  "s_waitcnt lgkmcnt(2)\n" \
  "v_mfma_f32_16x16x128_f8f6f4 a[112:115], v[152:159], v[96:103], a[112:115] cbsz:1 blgp:1\n" \
  "ds_read_b128 v[136:139], v161 offset:51200\n" \
  "ds_read_b128 v[140:143], v161 offset:52224\n" \
  "v_mfma_f32_16x16x128_f8f6f4 a[116:119], v[152:159], v[104:111], a[116:119] cbsz:1 blgp:1\n" \
  "buffer_load_dwordx4 v[76:79], v165, s[36:39], s75 offen offset:3072\n" \
  "v_mfma_f32_16x16x128_f8f6f4 a[120:123], v[152:159], v[112:119], a[120:123] cbsz:1 blgp:1\n" \
  "v_mfma_f32_16x16x128_f8f6f4 a[124:127], v[152:159], v[120:127], a[124:127] cbsz:1 blgp:1\n" \
  "s_bfe_u32 s96, s65, 0x40014\n" \
  "s_mul_i32 s96, s96, Le0_1_%=-Le0_0_%=\n" \
  "s_add_u32 s96, s96, Le0_0_%=-Lpc_%=\n" \
  "s_waitcnt lgkmcnt(2)\n" \
  "v_mfma_f32_16x16x128_f8f6f4 a[128:131], v[128:135], v[96:103], a[128:131] cbsz:1 blgp:1\n" \
  "s_add_u32 m0, s69, 81920\n" \
  "ds_read_b128 v[144:147], v161 offset:53248\n" \
  "ds_read_b128 v[148:151], v161 offset:54272\n" \
  "v_mfma_f32_16x16x128_f8f6f4 a[132:135], v[128:135], v[104:111], a[132:135] cbsz:1 blgp:1\n" \
  "buffer_load_dwordx4 v165, s[36:39], s60 offen lds\n" \
  "v_mfma_f32_16x16x128_f8f6f4 a[136:139], v[128:135], v[112:119], a[136:139] cbsz:1 blgp:1\n" \
  "v_mfma_f32_16x16x128_f8f6f4 a[140:143], v[128:135], v[120:127], a[140:143] cbsz:1 blgp:1\n" \
  "s_add_u32 s96, s96, s94\n" \
  "s_addc_u32 s97, s95, 0\n" \
  "s_waitcnt lgkmcnt(2)\n" \
  "v_mfma_f32_16x16x128_f8f6f4 a[144:147], v[136:143], v[96:103], a[144:147] cbsz:1 blgp:1\n" \
  "ds_read_b128 v[152:155], v161 offset:55296\n" \
  "ds_read_b128 v[156:159], v161 offset:56320\n" \
  "v_mfma_f32_16x16x128_f8f6f4 a[148:151], v[136:143], v[104:111], a[148:151] cbsz:1 blgp:1\n" \
  "buffer_load_dwordx4 v[80:83], v166, s[36:39], s75 offen offset:0\n" \
  "v_mfma_f32_16x16x128_f8f6f4 a[152:155], v[136:143], v[112:119], a[152:155] cbsz:1 blgp:1\n" \
  "v_mfma_f32_16x16x128_f8f6f4 a[156:159], v[136:143], v[120:127], a[156:159] cbsz:1 blgp:1\n" \
  "s_waitcnt lgkmcnt(2)\n" \
  "v_mfma_f32_16x16x128_f8f6f4 a[160:163], v[144:151], v[96:103], a[160:163] cbsz:1 blgp:1\n" \
  "v_mul_f32 v212, s78, v212\n" \
  "v_mul_f32 v213, s78, v213\n" \
  "v_mul_f32 v214, s78, v214\n" \
  "v_mul_f32 v215, s78, v215\n" \
  "s_add_u32 m0, s69, 86016\n" \
  "ds_read_b128 v[128:131], v161 offset:57344\n" \
  "ds_read_b128 v[132:135], v161 offset:58368\n" \
  "v_mfma_f32_16x16x128_f8f6f4 a[164:167], v[144:151], v[104:111], a[164:167] cbsz:1 blgp:1\n" \
  "buffer_load_dwordx4 v165, s[36:39], s61 offen lds\n" \
  "v_mfma_f32_16x16x128_f8f6f4 a[168:171], v[144:151], v[112:119], a[168:171] cbsz:1 blgp:1\n" \
  "buffer_store_dwordx4 v[212:215], v167, s[40:43], s51 offen offset:64 nt\n" \
  "v_mfma_f32_16x16x128_f8f6f4 a[172:175], v[144:151], v[120:127], a[172:175] cbsz:1 blgp:1\n" \
  "s_and_b32 s73, s65, 0xf00001ff\n" \
  "s_waitcnt lgkmcnt(2)\n" \
  "v_mfma_f32_16x16x128_f8f6f4 a[176:179], v[152:159], v[96:103], a[176:179] cbsz:1 blgp:1\n" \
  "ds_read_b128 v[136:139], v161 offset:59392\n" \
  "ds_read_b128 v[140:143], v161 offset:60416\n" \
  "v_mfma_f32_16x16x128_f8f6f4 a[180:183], v[152:159], v[104:111], a[180:183] cbsz:1 blgp:1\n" \
  "buffer_load_dwordx4 v[84:87], v166, s[36:39], s75 offen offset:1024\n" \
  "v_mfma_f32_16x16x128_f8f6f4 a[184:187], v[152:159], v[112:119], a[184:187] cbsz:1 blgp:1\n" \
  "v_mfma_f32_16x16x128_f8f6f4 a[188:191], v[152:159], v[120:127], a[188:191] cbsz:1 blgp:1\n" \
  "s_waitcnt lgkmcnt(2)\n" \
  "v_mfma_f32_16x16x128_f8f6f4 a[192:195], v[128:135], v[96:103], a[192:195] cbsz:1 blgp:1\n" \
  "s_add_u32 m0, s69, 90112\n" \
  "ds_read_b128 v[144:147], v161 offset:61440\n" \
  "ds_read_b128 v[148:151], v161 offset:62464\n" \
  "v_mfma_f32_16x16x128_f8f6f4 a[196:199], v[128:135], v[104:111], a[196:199] cbsz:1 blgp:1\n" \
  "buffer_load_dwordx4 v165, s[36:39], s62 offen lds\n" \
  "v_mfma_f32_16x16x128_f8f6f4 a[200:203], v[128:135], v[112:119], a[200:203] cbsz:1 blgp:1\n" \
  "v_mfma_f32_16x16x128_f8f6f4 a[204:207], v[128:135], v[120:127], a[204:207] cbsz:1 blgp:1\n" \
  "s_waitcnt lgkmcnt(2)\n" \
  "v_mfma_f32_16x16x128_f8f6f4 a[208:211], v[136:143], v[96:103], a[208:211] cbsz:1 blgp:1\n" \
  "ds_read_b128 v[152:155], v161 offset:63488\n" \
  "ds_read_b128 v[156:159], v161 offset:64512\n" \
  "v_mfma_f32_16x16x128_f8f6f4 a[212:215], v[136:143], v[104:111], a[212:215] cbsz:1 blgp:1\n" \
  "buffer_load_dwordx4 v[88:91], v166, s[36:39], s75 offen offset:2048\n" \
  "v_mfma_f32_16x16x128_f8f6f4 a[216:219], v[136:143], v[112:119], a[216:219] cbsz:1 blgp:1\n" \
  "s_bfe_u32 m0, s65, 0x40018\n" \
  "s_nop 0\n" \
  "s_movreld_b32 s56, s77\n" \
  "v_mfma_f32_16x16x128_f8f6f4 a[220:223], v[136:143], v[120:127], a[220:223] cbsz:1 blgp:1\n" \
  "s_cmp_eq_u32 s73, s88\n" \
  "s_cbranch_scc0 Lm0_%=\n" \
  "s_setpc_b64 s[96:97]\n" \
  "Lrd_%=:\n" \
  "s_waitcnt vmcnt(36) lgkmcnt(0)\n" \
  "s_barrier\n" \
  "ds_read_b128 v[128:131], v160 offset:0\n" \
  "ds_read_b128 v[132:135], v160 offset:1024\n" \
  "ds_read_b128 v[136:139], v160 offset:2048\n" \
  "ds_read_b128 v[140:143], v160 offset:3072\n" \
  "s_mov_b32 s42, s72\n" \
  "s_mov_b32 s48, s71\n" \
  "s_add_u32 s49, s71, 1048576\n" \
  "s_add_u32 s50, s71, 2097152\n" \
  "s_add_u32 s51, s71, 3145728\n" \
  "s_cmp_eq_u32 s70, 64\n" \
  "s_cbranch_scc1 Lp38_%=\n" \
  "Lq38_%=:\n" \
  "v_accvgpr_read_b32 v184, a224\n" \
  "v_accvgpr_read_b32 v185, a225\n" \
  "v_accvgpr_read_b32 v186, a226\n" \
  "v_accvgpr_read_b32 v187, a227\n" \
  "v_accvgpr_read_b32 v200, a240\n" \
  "v_accvgpr_read_b32 v201, a241\n" \
  "v_accvgpr_read_b32 v202, a242\n" \
  "v_accvgpr_read_b32 v203, a243\n" \
  "v_accvgpr_read_b32 v188, a228\n" \
  "v_accvgpr_read_b32 v189, a229\n" \
  "v_accvgpr_read_b32 v190, a230\n" \
  "v_accvgpr_read_b32 v191, a231\n" \
  "v_accvgpr_read_b32 v204, a244\n" \
  "v_accvgpr_read_b32 v205, a245\n" \
  "v_accvgpr_read_b32 v206, a246\n" \
  "v_accvgpr_read_b32 v207, a247\n" \
  "s_waitcnt lgkmcnt(2)\n" \
  "v_mfma_f32_16x16x128_f8f6f4 a[0:3], v[128:135], v[0:7], a[0:3] cbsz:1 blgp:1\n" \
  "ds_read_b128 v[144:147], v160 offset:4096\n" \
  "ds_read_b128 v[148:151], v160 offset:5120\n" \
  "v_mfma_f32_16x16x128_f8f6f4 a[4:7], v[128:135], v[8:15], a[4:7] cbsz:1 blgp:1\n" \
  "v_accvgpr_read_b32 v192, a232\n" \
  "v_accvgpr_read_b32 v193, a233\n" \
  "v_accvgpr_read_b32 v194, a234\n" \
  "v_accvgpr_read_b32 v195, a235\n" \
  "v_mfma_f32_16x16x128_f8f6f4 a[8:11], v[128:135], v[16:23], a[8:11] cbsz:1 blgp:1\n" \
  "v_accvgpr_read_b32 v208, a248\n" \
  "v_accvgpr_read_b32 v209, a249\n" \
  "v_accvgpr_read_b32 v210, a250\n" \
  "v_accvgpr_read_b32 v211, a251\n" \
  "v_mfma_f32_16x16x128_f8f6f4 a[12:15], v[128:135], v[24:31], a[12:15] cbsz:1 blgp:1\n" \
  "v_accvgpr_read_b32 v196, a236\n" \
  "v_accvgpr_read_b32 v197, a237\n" \
  "v_accvgpr_read_b32 v198, a238\n" \
  "v_accvgpr_read_b32 v199, a239\n" \
  "s_waitcnt lgkmcnt(2)\n" \
  "v_mfma_f32_16x16x128_f8f6f4 a[16:19], v[136:143], v[0:7], a[16:19] cbsz:1 blgp:1\n" \
  "ds_read_b128 v[152:155], v160 offset:6144\n" \
  "ds_read_b128 v[156:159], v160 offset:7168\n" \
  "v_mfma_f32_16x16x128_f8f6f4 a[20:23], v[136:143], v[8:15], a[20:23] cbsz:1 blgp:1\n" \
  "v_accvgpr_read_b32 v212, a252\n" \
  "v_accvgpr_read_b32 v213, a253\n" \
  "v_accvgpr_read_b32 v214, a254\n" \
  "v_accvgpr_read_b32 v215, a255\n" \
  "s_add_u32 m0, s69, 98304\n" \
  "s_nop 0\n" \
  "buffer_load_dwordx4 v163, s[36:39], s56 offen lds\n" \
  "v_mfma_f32_16x16x128_f8f6f4 a[24:27], v[136:143], v[16:23], a[24:27] cbsz:1 blgp:1\n" \
  "s_add_u32 m0, s69, 102400\n" \
  "s_nop 0\n" \
  "buffer_load_dwordx4 v163, s[36:39], s57 offen lds\n" \
  "v_mfma_f32_16x16x128_f8f6f4 a[28:31], v[136:143], v[24:31], a[28:31] cbsz:1 blgp:1\n" \
  "v_readlane_b32 s67, v175, s70\n" \
  "v_readlane_b32 s76, v169, s70\n" \
  "s_waitcnt lgkmcnt(2)\n" \
  "v_mfma_f32_16x16x128_f8f6f4 a[32:35], v[144:151], v[0:7], a[32:35] cbsz:1 blgp:1\n" \
  "ds_read_b128 v[128:131], v160 offset:8192\n" \
  "ds_read_b128 v[132:135], v160 offset:9216\n" \
  "v_mfma_f32_16x16x128_f8f6f4 a[36:39], v[144:151], v[8:15], a[36:39] cbsz:1 blgp:1\n" \
  "s_add_u32 m0, s69, 106496\n" \
  "s_nop 0\n" \
  "buffer_load_dwordx4 v163, s[36:39], s58 offen lds\n" \
  "v_mfma_f32_16x16x128_f8f6f4 a[40:43], v[144:151], v[16:23], a[40:43] cbsz:1 blgp:1\n" \
  "v_mfma_f32_16x16x128_f8f6f4 a[44:47], v[144:151], v[24:31], a[44:47] cbsz:1 blgp:1\n" \
  "s_add_u32 m0, s69, 110592\n" \
  "s_nop 0\n" \
  "buffer_load_dwordx4 v163, s[36:39], s59 offen lds\n" \
  "s_waitcnt lgkmcnt(2)\n" \
  "v_mfma_f32_16x16x128_f8f6f4 a[48:51], v[152:159], v[0:7], a[48:51] cbsz:1 blgp:1\n" \
  "ds_read_b128 v[136:139], v160 offset:10240\n" \
  "ds_read_b128 v[140:143], v160 offset:11264\n" \
  "v_mfma_f32_16x16x128_f8f6f4 a[52:55], v[152:159], v[8:15], a[52:55] cbsz:1 blgp:1\n" \
  "s_add_u32 s70, s70, 1\n" \
  "s_nop 0\n" \
  "v_add_u32 v165, s76, v162\n" \
  "v_mfma_f32_16x16x128_f8f6f4 a[56:59], v[152:159], v[16:23], a[56:59] cbsz:1 blgp:1\n" \
  "s_add_u32 m0, s69, 114688\n" \
  "s_nop 0\n" \
  "buffer_load_dwordx4 v163, s[36:39], s60 offen lds\n" \
  "v_mfma_f32_16x16x128_f8f6f4 a[60:63], v[152:159], v[24:31], a[60:63] cbsz:1 blgp:1\n" \
  "s_waitcnt lgkmcnt(2)\n" \
  "v_mfma_f32_16x16x128_f8f6f4 a[64:67], v[128:135], v[0:7], a[64:67] cbsz:1 blgp:1\n" \
  "ds_read_b128 v[144:147], v160 offset:12288\n" \
  "ds_read_b128 v[148:151], v160 offset:13312\n" \
  "v_mfma_f32_16x16x128_f8f6f4 a[68:71], v[128:135], v[8:15], a[68:71] cbsz:1 blgp:1\n" \
  "s_add_u32 m0, s69, 118784\n" \
  "s_nop 0\n" \
  "buffer_load_dwordx4 v163, s[36:39], s61 offen lds\n" \
  "v_mfma_f32_16x16x128_f8f6f4 a[72:75], v[128:135], v[16:23], a[72:75] cbsz:1 blgp:1\n" \
  "v_mfma_f32_16x16x128_f8f6f4 a[76:79], v[128:135], v[24:31], a[76:79] cbsz:1 blgp:1\n" \
  "s_add_u32 m0, s69, 122880\n" \
  "s_nop 0\n" \
  "buffer_load_dwordx4 v163, s[36:39], s62 offen lds\n" \
  "s_waitcnt lgkmcnt(2)\n" \
  "v_mfma_f32_16x16x128_f8f6f4 a[80:83], v[136:143], v[0:7], a[80:83] cbsz:1 blgp:1\n" \
  "ds_read_b128 v[152:155], v160 offset:14336\n" \
  "ds_read_b128 v[156:159], v160 offset:15360\n" \
  "v_mfma_f32_16x16x128_f8f6f4 a[84:87], v[136:143], v[8:15], a[84:87] cbsz:1 blgp:1\n" \
  "v_add_u32 v166, 4096, v165\n" \
  "s_cmp_gt_u32 s67, s68\n" \
  "s_cbranch_scc1 Lp39_%=\n" \
  "Lq39_%=:\n" \
  "v_mfma_f32_16x16x128_f8f6f4 a[88:91], v[136:143], v[16:23], a[88:91] cbsz:1 blgp:1\n" \
  "buffer_load_dwordx4 v[96:99], v163, s[36:39], s75 offen offset:0\n" \
  "v_mfma_f32_16x16x128_f8f6f4 a[92:95], v[136:143], v[24:31], a[92:95] cbsz:1 blgp:1\n" \
  "s_waitcnt lgkmcnt(2)\n" \
  "v_mfma_f32_16x16x128_f8f6f4 a[96:99], v[144:151], v[0:7], a[96:99] cbsz:1 blgp:1\n" \
  "ds_read_b128 v[128:131], v160 offset:16384\n" \
  "ds_read_b128 v[132:135], v160 offset:17408\n" \
  "v_mfma_f32_16x16x128_f8f6f4 a[100:103], v[144:151], v[8:15], a[100:103] cbsz:1 blgp:1\n" \
  "buffer_load_dwordx4 v[100:103], v163, s[36:39], s75 offen offset:1024\n" \
  "v_mfma_f32_16x16x128_f8f6f4 a[104:107], v[144:151], v[16:23], a[104:107] cbsz:1 blgp:1\n" \
  "buffer_load_dwordx4 v[104:107], v163, s[36:39], s75 offen offset:2048\n" \
  "v_mfma_f32_16x16x128_f8f6f4 a[108:111], v[144:151], v[24:31], a[108:111] cbsz:1 blgp:1\n" \
  "s_waitcnt lgkmcnt(2)\n" \
  "v_mfma_f32_16x16x128_f8f6f4 a[112:115], v[152:159], v[0:7], a[112:115] cbsz:1 blgp:1\n" \
  "ds_read_b128 v[136:139], v160 offset:18432\n" \
  "ds_read_b128 v[140:143], v160 offset:19456\n" \
  "v_mfma_f32_16x16x128_f8f6f4 a[116:119], v[152:159], v[8:15], a[116:119] cbsz:1 blgp:1\n" \
  "buffer_load_dwordx4 v[108:111], v163, s[36:39], s75 offen offset:3072\n" \
  "v_mfma_f32_16x16x128_f8f6f4 a[120:123], v[152:159], v[16:23], a[120:123] cbsz:1 blgp:1\n" \
  "v_mfma_f32_16x16x128_f8f6f4 a[124:127], v[152:159], v[24:31], a[124:127] cbsz:1 blgp:1\n" \
  "buffer_load_dwordx4 v[112:115], v164, s[36:39], s75 offen offset:0\n" \
  "s_waitcnt lgkmcnt(2)\n" \
  "v_mfma_f32_16x16x128_f8f6f4 a[128:131], v[128:135], v[0:7], a[128:131] cbsz:1 blgp:1\n" \
  "ds_read_b128 v[144:147], v160 offset:20480\n" \
  "ds_read_b128 v[148:151], v160 offset:21504\n" \
  "v_mfma_f32_16x16x128_f8f6f4 a[132:135], v[128:135], v[8:15], a[132:135] cbsz:1 blgp:1\n" \
  "buffer_load_dwordx4 v[116:119], v164, s[36:39], s75 offen offset:1024\n" \
  "v_mfma_f32_16x16x128_f8f6f4 a[136:139], v[128:135], v[16:23], a[136:139] cbsz:1 blgp:1\n" \
  "v_mfma_f32_16x16x128_f8f6f4 a[140:143], v[128:135], v[24:31], a[140:143] cbsz:1 blgp:1\n" \
  "v_mul_f32 v184, s78, v184\n" \
  "v_mul_f32 v185, s78, v185\n" \
  "v_mul_f32 v186, s78, v186\n" \
  "v_mul_f32 v187, s78, v187\n" \
  "s_waitcnt lgkmcnt(2)\n" \
  "v_mfma_f32_16x16x128_f8f6f4 a[144:147], v[136:143], v[0:7], a[144:147] cbsz:1 blgp:1\n" \
  "ds_read_b128 v[152:155], v160 offset:22528\n" \
  "ds_read_b128 v[156:159], v160 offset:23552\n" \
  "v_mfma_f32_16x16x128_f8f6f4 a[148:151], v[136:143], v[8:15], a[148:151] cbsz:1 blgp:1\n" \
  "buffer_store_dwordx4 v[184:187], v167, s[40:43], s48 offen offset:0 nt\n" \
  "v_mfma_f32_16x16x128_f8f6f4 a[152:155], v[136:143], v[16:23], a[152:155] cbsz:1 blgp:1\n" \
  "v_mfma_f32_16x16x128_f8f6f4 a[156:159], v[136:143], v[24:31], a[156:159] cbsz:1 blgp:1\n" \
  "buffer_load_dwordx4 v[120:123], v164, s[36:39], s75 offen offset:2048\n" \
  "s_waitcnt lgkmcnt(2)\n" \
  "v_mfma_f32_16x16x128_f8f6f4 a[160:163], v[144:151], v[0:7], a[160:163] cbsz:1 blgp:1\n" \
  "ds_read_b128 v[128:131], v160 offset:24576\n" \
  "ds_read_b128 v[132:135], v160 offset:25600\n" \
  "v_mfma_f32_16x16x128_f8f6f4 a[164:167], v[144:151], v[8:15], a[164:167] cbsz:1 blgp:1\n" \
  "v_mul_f32 v200, s78, v200\n" \
  "v_mul_f32 v201, s78, v201\n" \
  "v_mul_f32 v202, s78, v202\n" \
  "v_mul_f32 v203, s78, v203\n" \
  "v_mfma_f32_16x16x128_f8f6f4 a[168:171], v[144:151], v[16:23], a[168:171] cbsz:1 blgp:1\n" \
  "buffer_store_dwordx4 v[200:203], v167, s[40:43], s48 offen offset:64 nt\n" \
  "v_mfma_f32_16x16x128_f8f6f4 a[172:175], v[144:151], v[24:31], a[172:175] cbsz:1 blgp:1\n" \
  "s_waitcnt lgkmcnt(2)\n" \
  "v_mfma_f32_16x16x128_f8f6f4 a[176:179], v[152:159], v[0:7], a[176:179] cbsz:1 blgp:1\n" \
  "ds_read_b128 v[136:139], v160 offset:26624\n" \
  "ds_read_b128 v[140:143], v160 offset:27648\n" \
  "v_mfma_f32_16x16x128_f8f6f4 a[180:183], v[152:159], v[8:15], a[180:183] cbsz:1 blgp:1\n" \
  "buffer_load_dwordx4 v[124:127], v164, s[36:39], s75 offen offset:3072\n" \
  "v_mfma_f32_16x16x128_f8f6f4 a[184:187], v[152:159], v[16:23], a[184:187] cbsz:1 blgp:1\n" \
  "v_mfma_f32_16x16x128_f8f6f4 a[188:191], v[152:159], v[24:31], a[188:191] cbsz:1 blgp:1\n" \
  "s_waitcnt vmcnt(35) lgkmcnt(0)\n" \
  "s_barrier\n" \
  "v_mfma_f32_16x16x128_f8f6f4 a[192:195], v[128:135], v[0:7], a[192:195] cbsz:1 blgp:1\n" \
  "ds_read_b128 v[144:147], v160 offset:32768\n" \
  "ds_read_b128 v[148:151], v160 offset:33792\n" \
  "v_mfma_f32_16x16x128_f8f6f4 a[196:199], v[128:135], v[8:15], a[196:199] cbsz:1 blgp:1\n" \
  "s_add_u32 m0, s69, 4096\n" \
  "s_nop 0\n" \
  "buffer_load_dwordx4 v165, s[36:39], s57 offen lds\n" \
  "v_mfma_f32_16x16x128_f8f6f4 a[200:203], v[128:135], v[16:23], a[200:203] cbsz:1 blgp:1\n" \
  "s_add_u32 m0, s69, 8192\n" \
  "s_nop 0\n" \
  "buffer_load_dwordx4 v165, s[36:39], s58 offen lds\n" \
  "v_mfma_f32_16x16x128_f8f6f4 a[204:207], v[128:135], v[24:31], a[204:207] cbsz:1 blgp:1\n" \
  "v_mfma_f32_16x16x128_f8f6f4 a[208:211], v[136:143], v[0:7], a[208:211] cbsz:1 blgp:1\n" \
  "ds_read_b128 v[152:155], v160 offset:34816\n" \
  "ds_read_b128 v[156:159], v160 offset:35840\n" \
  "v_mfma_f32_16x16x128_f8f6f4 a[212:215], v[136:143], v[8:15], a[212:215] cbsz:1 blgp:1\n" \
  "v_readlane_b32 s67, v175, s70\n" \
  "v_readlane_b32 s76, v169, s70\n" \
  "v_mfma_f32_16x16x128_f8f6f4 a[216:219], v[136:143], v[16:23], a[216:219] cbsz:1 blgp:1\n" \
  "s_add_u32 m0, s69, 12288\n" \
  "s_nop 0\n" \
  "buffer_load_dwordx4 v165, s[36:39], s59 offen lds\n" \
  "v_mfma_f32_16x16x128_f8f6f4 a[220:223], v[136:143], v[24:31], a[220:223] cbsz:1 blgp:1\n" \
  "s_waitcnt lgkmcnt(2)\n" \
  "v_mfma_f32_16x16x128_f8f6f4 a[0:3], v[144:151], v[32:39], a[0:3] cbsz:1 blgp:1\n" \
  "ds_read_b128 v[128:131], v160 offset:36864\n" \
  "ds_read_b128 v[132:135], v160 offset:37888\n" \
  "v_mfma_f32_16x16x128_f8f6f4 a[4:7], v[144:151], v[40:47], a[4:7] cbsz:1 blgp:1\n" \
  "s_add_u32 m0, s69, 16384\n" \
  "s_nop 0\n" \
  "buffer_load_dwordx4 v165, s[36:39], s60 offen lds\n" \
  "v_mfma_f32_16x16x128_f8f6f4 a[8:11], v[144:151], v[48:55], a[8:11] cbsz:1 blgp:1\n" \
  "v_mfma_f32_16x16x128_f8f6f4 a[12:15], v[144:151], v[56:63], a[12:15] cbsz:1 blgp:1\n" \
  "s_waitcnt lgkmcnt(2)\n" \
  "v_mfma_f32_16x16x128_f8f6f4 a[16:19], v[152:159], v[32:39], a[16:19] cbsz:1 blgp:1\n" \
  "ds_read_b128 v[136:139], v160 offset:38912\n" \
  "ds_read_b128 v[140:143], v160 offset:39936\n" \
  "v_mfma_f32_16x16x128_f8f6f4 a[20:23], v[152:159], v[40:47], a[20:23] cbsz:1 blgp:1\n" \
  "s_add_u32 s70, s70, 1\n" \
  "s_nop 0\n" \
  "v_add_u32 v163, s76, v162\n" \
  "v_mfma_f32_16x16x128_f8f6f4 a[24:27], v[152:159], v[48:55], a[24:27] cbsz:1 blgp:1\n" \
  "s_add_u32 m0, s69, 20480\n" \
  "s_nop 0\n" \
  "buffer_load_dwordx4 v165, s[36:39], s61 offen lds\n" \
  "v_mfma_f32_16x16x128_f8f6f4 a[28:31], v[152:159], v[56:63], a[28:31] cbsz:1 blgp:1\n" \
  "s_waitcnt lgkmcnt(2)\n" \
  "v_mfma_f32_16x16x128_f8f6f4 a[32:35], v[128:135], v[32:39], a[32:35] cbsz:1 blgp:1\n" \
  "ds_read_b128 v[144:147], v160 offset:40960\n" \
  "ds_read_b128 v[148:151], v160 offset:41984\n" \
  "v_mfma_f32_16x16x128_f8f6f4 a[36:39], v[128:135], v[40:47], a[36:39] cbsz:1 blgp:1\n" \
  "s_add_u32 m0, s69, 24576\n" \
  "s_nop 0\n" \
  "buffer_load_dwordx4 v165, s[36:39], s62 offen lds\n" \
  "v_mfma_f32_16x16x128_f8f6f4 a[40:43], v[128:135], v[48:55], a[40:43] cbsz:1 blgp:1\n" \
  "v_mfma_f32_16x16x128_f8f6f4 a[44:47], v[128:135], v[56:63], a[44:47] cbsz:1 blgp:1\n" \
  "s_waitcnt lgkmcnt(2)\n" \
  "v_mfma_f32_16x16x128_f8f6f4 a[48:51], v[136:143], v[32:39], a[48:51] cbsz:1 blgp:1\n" \
  "ds_read_b128 v[152:155], v160 offset:43008\n" \
  "ds_read_b128 v[156:159], v160 offset:44032\n" \
  "v_mfma_f32_16x16x128_f8f6f4 a[52:55], v[136:143], v[40:47], a[52:55] cbsz:1 blgp:1\n" \
  "v_add_u32 v164, 4096, v163\n" \
  "s_cmp_gt_u32 s67, s68\n" \
  "s_cbranch_scc1 Lp40_%=\n" \
  "Lq40_%=:\n" \
  "buffer_load_dwordx4 v[0:3], v165, s[36:39], s75 offen offset:0\n" \
  "v_mfma_f32_16x16x128_f8f6f4 a[56:59], v[136:143], v[48:55], a[56:59] cbsz:1 blgp:1\n" \
  "v_mfma_f32_16x16x128_f8f6f4 a[60:63], v[136:143], v[56:63], a[60:63] cbsz:1 blgp:1\n" \
  "s_waitcnt lgkmcnt(2)\n" \
  "v_mfma_f32_16x16x128_f8f6f4 a[64:67], v[144:151], v[32:39], a[64:67] cbsz:1 blgp:1\n" \
  "ds_read_b128 v[128:131], v160 offset:45056\n" \
  "ds_read_b128 v[132:135], v160 offset:46080\n" \
  "v_mfma_f32_16x16x128_f8f6f4 a[68:71], v[144:151], v[40:47], a[68:71] cbsz:1 blgp:1\n" \
  "buffer_load_dwordx4 v[4:7], v165, s[36:39], s75 offen offset:1024\n" \
  "v_mfma_f32_16x16x128_f8f6f4 a[72:75], v[144:151], v[48:55], a[72:75] cbsz:1 blgp:1\n" \
  "v_mfma_f32_16x16x128_f8f6f4 a[76:79], v[144:151], v[56:63], a[76:79] cbsz:1 blgp:1\n" \
  "v_mul_f32 v188, s78, v188\n" \
  "v_mul_f32 v189, s78, v189\n" \
  "v_mul_f32 v190, s78, v190\n" \
  "v_mul_f32 v191, s78, v191\n" \
  "s_waitcnt lgkmcnt(2)\n" \
  "v_mfma_f32_16x16x128_f8f6f4 a[80:83], v[152:159], v[32:39], a[80:83] cbsz:1 blgp:1\n" \
  "ds_read_b128 v[136:139], v160 offset:47104\n" \
  "ds_read_b128 v[140:143], v160 offset:48128\n" \
  "v_mfma_f32_16x16x128_f8f6f4 a[84:87], v[152:159], v[40:47], a[84:87] cbsz:1 blgp:1\n" \
  "buffer_store_dwordx4 v[188:191], v167, s[40:43], s49 offen offset:0 nt\n" \
  "v_mfma_f32_16x16x128_f8f6f4 a[88:91], v[152:159], v[48:55], a[88:91] cbsz:1 blgp:1\n" \
  "v_mfma_f32_16x16x128_f8f6f4 a[92:95], v[152:159], v[56:63], a[92:95] cbsz:1 blgp:1\n" \
  "s_waitcnt lgkmcnt(2)\n" \
  "v_mfma_f32_16x16x128_f8f6f4 a[96:99], v[128:135], v[32:39], a[96:99] cbsz:1 blgp:1\n" \
  "ds_read_b128 v[144:147], v160 offset:49152\n" \
  "ds_read_b128 v[148:151], v160 offset:50176\n" \
  "v_mfma_f32_16x16x128_f8f6f4 a[100:103], v[128:135], v[40:47], a[100:103] cbsz:1 blgp:1\n" \
  "buffer_load_dwordx4 v[8:11], v165, s[36:39], s75 offen offset:2048\n" \
  "v_mfma_f32_16x16x128_f8f6f4 a[104:107], v[128:135], v[48:55], a[104:107] cbsz:1 blgp:1\n" \
  "v_mfma_f32_16x16x128_f8f6f4 a[108:111], v[128:135], v[56:63], a[108:111] cbsz:1 blgp:1\n" \
  "buffer_load_dwordx4 v[12:15], v165, s[36:39], s75 offen offset:3072\n" \
  "s_waitcnt lgkmcnt(2)\n" \
  "v_mfma_f32_16x16x128_f8f6f4 a[112:115], v[136:143], v[32:39], a[112:115] cbsz:1 blgp:1\n" \
  "ds_read_b128 v[152:155], v160 offset:51200\n" \
  "ds_read_b128 v[156:159], v160 offset:52224\n" \
  "v_mfma_f32_16x16x128_f8f6f4 a[116:119], v[136:143], v[40:47], a[116:119] cbsz:1 blgp:1\n" \
  "v_mfma_f32_16x16x128_f8f6f4 a[120:123], v[136:143], v[48:55], a[120:123] cbsz:1 blgp:1\n" \
  "buffer_load_dwordx4 v[16:19], v166, s[36:39], s75 offen offset:0\n" \
  "v_mfma_f32_16x16x128_f8f6f4 a[124:127], v[136:143], v[56:63], a[124:127] cbsz:1 blgp:1\n" \
  "s_waitcnt lgkmcnt(2)\n" \
  "v_mfma_f32_16x16x128_f8f6f4 a[128:131], v[144:151], v[32:39], a[128:131] cbsz:1 blgp:1\n" \
  "ds_read_b128 v[128:131], v160 offset:53248\n" \
  "ds_read_b128 v[132:135], v160 offset:54272\n" \
  "v_mfma_f32_16x16x128_f8f6f4 a[132:135], v[144:151], v[40:47], a[132:135] cbsz:1 blgp:1\n" \
  "v_mul_f32 v204, s78, v204\n" \
  "v_mul_f32 v205, s78, v205\n" \
  "v_mul_f32 v206, s78, v206\n" \
  "v_mul_f32 v207, s78, v207\n" \
  "v_mfma_f32_16x16x128_f8f6f4 a[136:139], v[144:151], v[48:55], a[136:139] cbsz:1 blgp:1\n" \
  "buffer_store_dwordx4 v[204:207], v167, s[40:43], s49 offen offset:64 nt\n" \
  "v_mfma_f32_16x16x128_f8f6f4 a[140:143], v[144:151], v[56:63], a[140:143] cbsz:1 blgp:1\n" \
  "s_waitcnt lgkmcnt(2)\n" \
  "v_mfma_f32_16x16x128_f8f6f4 a[144:147], v[152:159], v[32:39], a[144:147] cbsz:1 blgp:1\n" \
  "ds_read_b128 v[136:139], v160 offset:55296\n" \
  "ds_read_b128 v[140:143], v160 offset:56320\n" \
  "v_mfma_f32_16x16x128_f8f6f4 a[148:151], v[152:159], v[40:47], a[148:151] cbsz:1 blgp:1\n" \
  "buffer_load_dwordx4 v[20:23], v166, s[36:39], s75 offen offset:1024\n" \
  "v_mfma_f32_16x16x128_f8f6f4 a[152:155], v[152:159], v[48:55], a[152:155] cbsz:1 blgp:1\n" \
  "v_mfma_f32_16x16x128_f8f6f4 a[156:159], v[152:159], v[56:63], a[156:159] cbsz:1 blgp:1\n" \
  "s_waitcnt lgkmcnt(2)\n" \
  "v_mfma_f32_16x16x128_f8f6f4 a[160:163], v[128:135], v[32:39], a[160:163] cbsz:1 blgp:1\n" \
  "ds_read_b128 v[144:147], v160 offset:57344\n" \
  "ds_read_b128 v[148:151], v160 offset:58368\n" \
  "v_mfma_f32_16x16x128_f8f6f4 a[164:167], v[128:135], v[40:47], a[164:167] cbsz:1 blgp:1\n" \
  "buffer_load_dwordx4 v[24:27], v166, s[36:39], s75 offen offset:2048\n" \
  "v_mfma_f32_16x16x128_f8f6f4 a[168:171], v[128:135], v[48:55], a[168:171] cbsz:1 blgp:1\n" \
  "v_mfma_f32_16x16x128_f8f6f4 a[172:175], v[128:135], v[56:63], a[172:175] cbsz:1 blgp:1\n" \
  "buffer_load_dwordx4 v[28:31], v166, s[36:39], s75 offen offset:3072\n" \
  "s_waitcnt lgkmcnt(2)\n" \
  "v_mfma_f32_16x16x128_f8f6f4 a[176:179], v[136:143], v[32:39], a[176:179] cbsz:1 blgp:1\n" \
  "ds_read_b128 v[152:155], v160 offset:59392\n" \
  "ds_read_b128 v[156:159], v160 offset:60416\n" \
  "v_mfma_f32_16x16x128_f8f6f4 a[180:183], v[136:143], v[40:47], a[180:183] cbsz:1 blgp:1\n" \
  "v_mfma_f32_16x16x128_f8f6f4 a[184:187], v[136:143], v[48:55], a[184:187] cbsz:1 blgp:1\n" \
  "v_mfma_f32_16x16x128_f8f6f4 a[188:191], v[136:143], v[56:63], a[188:191] cbsz:1 blgp:1\n" \
  "s_waitcnt vmcnt(33) lgkmcnt(0)\n" \
  "s_barrier\n" \
  "v_mfma_f32_16x16x128_f8f6f4 a[192:195], v[144:151], v[32:39], a[192:195] cbsz:1 blgp:1\n" \
  "ds_read_b128 v[128:131], v161 offset:0\n" \
  "ds_read_b128 v[132:135], v161 offset:1024\n" \
  "v_mfma_f32_16x16x128_f8f6f4 a[196:199], v[144:151], v[40:47], a[196:199] cbsz:1 blgp:1\n" \
  "s_add_u32 m0, s69, 36864\n" \
  "s_nop 0\n" \
  "buffer_load_dwordx4 v163, s[36:39], s57 offen lds\n" \
  "v_mfma_f32_16x16x128_f8f6f4 a[200:203], v[144:151], v[48:55], a[200:203] cbsz:1 blgp:1\n" \
  "s_add_u32 m0, s69, 40960\n" \
  "s_nop 0\n" \
  "buffer_load_dwordx4 v163, s[36:39], s58 offen lds\n" \
  "v_mfma_f32_16x16x128_f8f6f4 a[204:207], v[144:151], v[56:63], a[204:207] cbsz:1 blgp:1\n" \
  "v_mfma_f32_16x16x128_f8f6f4 a[208:211], v[152:159], v[32:39], a[208:211] cbsz:1 blgp:1\n" \
  "ds_read_b128 v[136:139], v161 offset:2048\n" \
  "ds_read_b128 v[140:143], v161 offset:3072\n" \
  "v_mfma_f32_16x16x128_f8f6f4 a[212:215], v[152:159], v[40:47], a[212:215] cbsz:1 blgp:1\n" \
  "v_readlane_b32 s67, v175, s70\n" \
  "v_readlane_b32 s76, v169, s70\n" \
  "v_mfma_f32_16x16x128_f8f6f4 a[216:219], v[152:159], v[48:55], a[216:219] cbsz:1 blgp:1\n" \
  "s_add_u32 m0, s69, 45056\n" \
  "s_nop 0\n" \
  "buffer_load_dwordx4 v163, s[36:39], s59 offen lds\n" \
  "v_mfma_f32_16x16x128_f8f6f4 a[220:223], v[152:159], v[56:63], a[220:223] cbsz:1 blgp:1\n" \
  "s_waitcnt lgkmcnt(2)\n" \
  "v_mfma_f32_16x16x128_f8f6f4 a[0:3], v[128:135], v[64:71], a[0:3] cbsz:1 blgp:1\n" \
  "ds_read_b128 v[144:147], v161 offset:4096\n" \
  "ds_read_b128 v[148:151], v161 offset:5120\n" \
  "v_mfma_f32_16x16x128_f8f6f4 a[4:7], v[128:135], v[72:79], a[4:7] cbsz:1 blgp:1\n" \
  "s_add_u32 m0, s69, 49152\n" \
  "s_nop 0\n" \
  "buffer_load_dwordx4 v163, s[36:39], s60 offen lds\n" \
  "v_mfma_f32_16x16x128_f8f6f4 a[8:11], v[128:135], v[80:87], a[8:11] cbsz:1 blgp:1\n" \
  "v_mfma_f32_16x16x128_f8f6f4 a[12:15], v[128:135], v[88:95], a[12:15] cbsz:1 blgp:1\n" \
  "s_waitcnt lgkmcnt(2)\n" \
  "v_mfma_f32_16x16x128_f8f6f4 a[16:19], v[136:143], v[64:71], a[16:19] cbsz:1 blgp:1\n" \
  "ds_read_b128 v[152:155], v161 offset:6144\n" \
  "ds_read_b128 v[156:159], v161 offset:7168\n" \
  "v_mfma_f32_16x16x128_f8f6f4 a[20:23], v[136:143], v[72:79], a[20:23] cbsz:1 blgp:1\n" \
  "s_add_u32 s70, s70, 1\n" \
  "s_nop 0\n" \
  "v_add_u32 v165, s76, v162\n" \
  "v_mfma_f32_16x16x128_f8f6f4 a[24:27], v[136:143], v[80:87], a[24:27] cbsz:1 blgp:1\n" \
  "s_add_u32 m0, s69, 53248\n" \
  "s_nop 0\n" \
  "buffer_load_dwordx4 v163, s[36:39], s61 offen lds\n" \
  "v_mfma_f32_16x16x128_f8f6f4 a[28:31], v[136:143], v[88:95], a[28:31] cbsz:1 blgp:1\n" \
  "s_waitcnt lgkmcnt(2)\n" \
  "v_mfma_f32_16x16x128_f8f6f4 a[32:35], v[144:151], v[64:71], a[32:35] cbsz:1 blgp:1\n" \
  "ds_read_b128 v[128:131], v161 offset:8192\n" \
  "ds_read_b128 v[132:135], v161 offset:9216\n" \
  "v_mfma_f32_16x16x128_f8f6f4 a[36:39], v[144:151], v[72:79], a[36:39] cbsz:1 blgp:1\n" \
  "s_add_u32 m0, s69, 57344\n" \
  "s_nop 0\n" \
  "buffer_load_dwordx4 v163, s[36:39], s62 offen lds\n" \
  "v_mfma_f32_16x16x128_f8f6f4 a[40:43], v[144:151], v[80:87], a[40:43] cbsz:1 blgp:1\n" \
  "v_mfma_f32_16x16x128_f8f6f4 a[44:47], v[144:151], v[88:95], a[44:47] cbsz:1 blgp:1\n" \
  "s_waitcnt lgkmcnt(2)\n" \
  "v_mfma_f32_16x16x128_f8f6f4 a[48:51], v[152:159], v[64:71], a[48:51] cbsz:1 blgp:1\n" \
  "ds_read_b128 v[136:139], v161 offset:10240\n" \
  "ds_read_b128 v[140:143], v161 offset:11264\n" \
  "v_mfma_f32_16x16x128_f8f6f4 a[52:55], v[152:159], v[72:79], a[52:55] cbsz:1 blgp:1\n" \
  "v_add_u32 v166, 4096, v165\n" \
  "s_cmp_gt_u32 s67, s68\n" \
  "s_cbranch_scc1 Lp41_%=\n" \
  "Lq41_%=:\n" \
  "buffer_load_dwordx4 v[32:35], v163, s[36:39], s75 offen offset:0\n" \
  "v_mfma_f32_16x16x128_f8f6f4 a[56:59], v[152:159], v[80:87], a[56:59] cbsz:1 blgp:1\n" \
  "v_mfma_f32_16x16x128_f8f6f4 a[60:63], v[152:159], v[88:95], a[60:63] cbsz:1 blgp:1\n" \
  "s_waitcnt lgkmcnt(2)\n" \
  "v_mfma_f32_16x16x128_f8f6f4 a[64:67], v[128:135], v[64:71], a[64:67] cbsz:1 blgp:1\n" \
  "ds_read_b128 v[144:147], v161 offset:12288\n" \
  "ds_read_b128 v[148:151], v161 offset:13312\n" \
  "v_mfma_f32_16x16x128_f8f6f4 a[68:71], v[128:135], v[72:79], a[68:71] cbsz:1 blgp:1\n" \
  "buffer_load_dwordx4 v[36:39], v163, s[36:39], s75 offen offset:1024\n" \
  "v_mfma_f32_16x16x128_f8f6f4 a[72:75], v[128:135], v[80:87], a[72:75] cbsz:1 blgp:1\n" \
  "v_mfma_f32_16x16x128_f8f6f4 a[76:79], v[128:135], v[88:95], a[76:79] cbsz:1 blgp:1\n" \
  "v_mul_f32 v192, s78, v192\n" \
  "v_mul_f32 v193, s78, v193\n" \
  "v_mul_f32 v194, s78, v194\n" \
  "v_mul_f32 v195, s78, v195\n" \
  "s_waitcnt lgkmcnt(2)\n" \
  "v_mfma_f32_16x16x128_f8f6f4 a[80:83], v[136:143], v[64:71], a[80:83] cbsz:1 blgp:1\n" \
  "ds_read_b128 v[152:155], v161 offset:14336\n" \
  "ds_read_b128 v[156:159], v161 offset:15360\n" \
  "v_mfma_f32_16x16x128_f8f6f4 a[84:87], v[136:143], v[72:79], a[84:87] cbsz:1 blgp:1\n" \
  "buffer_store_dwordx4 v[192:195], v167, s[40:43], s50 offen offset:0 nt\n" \
  "v_mfma_f32_16x16x128_f8f6f4 a[88:91], v[136:143], v[80:87], a[88:91] cbsz:1 blgp:1\n" \
  "v_mfma_f32_16x16x128_f8f6f4 a[92:95], v[136:143], v[88:95], a[92:95] cbsz:1 blgp:1\n" \
  "s_waitcnt lgkmcnt(2)\n" \
  "v_mfma_f32_16x16x128_f8f6f4 a[96:99], v[144:151], v[64:71], a[96:99] cbsz:1 blgp:1\n" \
  "ds_read_b128 v[128:131], v161 offset:16384\n" \
  "ds_read_b128 v[132:135], v161 offset:17408\n" \
  "v_mfma_f32_16x16x128_f8f6f4 a[100:103], v[144:151], v[72:79], a[100:103] cbsz:1 blgp:1\n" \
  "buffer_load_dwordx4 v[40:43], v163, s[36:39], s75 offen offset:2048\n" \
  "v_mfma_f32_16x16x128_f8f6f4 a[104:107], v[144:151], v[80:87], a[104:107] cbsz:1 blgp:1\n" \
  "v_mfma_f32_16x16x128_f8f6f4 a[108:111], v[144:151], v[88:95], a[108:111] cbsz:1 blgp:1\n" \
  "buffer_load_dwordx4 v[44:47], v163, s[36:39], s75 offen offset:3072\n" \
  "s_waitcnt lgkmcnt(2)\n" \
  "v_mfma_f32_16x16x128_f8f6f4 a[112:115], v[152:159], v[64:71], a[112:115] cbsz:1 blgp:1\n" \
  "ds_read_b128 v[136:139], v161 offset:18432\n" \
  "ds_read_b128 v[140:143], v161 offset:19456\n" \
  "v_mfma_f32_16x16x128_f8f6f4 a[116:119], v[152:159], v[72:79], a[116:119] cbsz:1 blgp:1\n" \
  "v_mfma_f32_16x16x128_f8f6f4 a[120:123], v[152:159], v[80:87], a[120:123] cbsz:1 blgp:1\n" \
  "buffer_load_dwordx4 v[48:51], v164, s[36:39], s75 offen offset:0\n" \
  "v_mfma_f32_16x16x128_f8f6f4 a[124:127], v[152:159], v[88:95], a[124:127] cbsz:1 blgp:1\n" \
  "s_waitcnt lgkmcnt(2)\n" \
  "v_mfma_f32_16x16x128_f8f6f4 a[128:131], v[128:135], v[64:71], a[128:131] cbsz:1 blgp:1\n" \
  "ds_read_b128 v[144:147], v161 offset:20480\n" \
  "ds_read_b128 v[148:151], v161 offset:21504\n" \
  "v_mfma_f32_16x16x128_f8f6f4 a[132:135], v[128:135], v[72:79], a[132:135] cbsz:1 blgp:1\n" \
  "v_mul_f32 v208, s78, v208\n" \
  "v_mul_f32 v209, s78, v209\n" \
  "v_mul_f32 v210, s78, v210\n" \
  "v_mul_f32 v211, s78, v211\n" \
  "v_mfma_f32_16x16x128_f8f6f4 a[136:139], v[128:135], v[80:87], a[136:139] cbsz:1 blgp:1\n" \
  "buffer_store_dwordx4 v[208:211], v167, s[40:43], s50 offen offset:64 nt\n" \
  "v_mfma_f32_16x16x128_f8f6f4 a[140:143], v[128:135], v[88:95], a[140:143] cbsz:1 blgp:1\n" \
  "s_waitcnt lgkmcnt(2)\n" \
  "v_mfma_f32_16x16x128_f8f6f4 a[144:147], v[136:143], v[64:71], a[144:147] cbsz:1 blgp:1\n" \
  "ds_read_b128 v[152:155], v161 offset:22528\n" \
  "ds_read_b128 v[156:159], v161 offset:23552\n" \
  "v_mfma_f32_16x16x128_f8f6f4 a[148:151], v[136:143], v[72:79], a[148:151] cbsz:1 blgp:1\n" \
  "buffer_load_dwordx4 v[52:55], v164, s[36:39], s75 offen offset:1024\n" \
  "v_mfma_f32_16x16x128_f8f6f4 a[152:155], v[136:143], v[80:87], a[152:155] cbsz:1 blgp:1\n" \
  "v_mfma_f32_16x16x128_f8f6f4 a[156:159], v[136:143], v[88:95], a[156:159] cbsz:1 blgp:1\n" \
  "s_waitcnt lgkmcnt(2)\n" \
  "v_mfma_f32_16x16x128_f8f6f4 a[160:163], v[144:151], v[64:71], a[160:163] cbsz:1 blgp:1\n" \
  "ds_read_b128 v[128:131], v161 offset:24576\n" \
  "ds_read_b128 v[132:135], v161 offset:25600\n" \
  "v_mfma_f32_16x16x128_f8f6f4 a[164:167], v[144:151], v[72:79], a[164:167] cbsz:1 blgp:1\n" \
  "buffer_load_dwordx4 v[56:59], v164, s[36:39], s75 offen offset:2048\n" \
  "v_mfma_f32_16x16x128_f8f6f4 a[168:171], v[144:151], v[80:87], a[168:171] cbsz:1 blgp:1\n" \
  "v_mfma_f32_16x16x128_f8f6f4 a[172:175], v[144:151], v[88:95], a[172:175] cbsz:1 blgp:1\n" \
  "buffer_load_dwordx4 v[60:63], v164, s[36:39], s75 offen offset:3072\n" \
  "s_waitcnt lgkmcnt(2)\n" \
  "v_mfma_f32_16x16x128_f8f6f4 a[176:179], v[152:159], v[64:71], a[176:179] cbsz:1 blgp:1\n" \
  "ds_read_b128 v[136:139], v161 offset:26624\n" \
  "ds_read_b128 v[140:143], v161 offset:27648\n" \
  "v_mfma_f32_16x16x128_f8f6f4 a[180:183], v[152:159], v[72:79], a[180:183] cbsz:1 blgp:1\n" \
  "v_mfma_f32_16x16x128_f8f6f4 a[184:187], v[152:159], v[80:87], a[184:187] cbsz:1 blgp:1\n" \
  "v_mfma_f32_16x16x128_f8f6f4 a[188:191], v[152:159], v[88:95], a[188:191] cbsz:1 blgp:1\n" \
  "s_waitcnt vmcnt(32) lgkmcnt(0)\n" \
  "s_barrier\n" \
  "v_mfma_f32_16x16x128_f8f6f4 a[192:195], v[128:135], v[64:71], a[192:195] cbsz:1 blgp:1\n" \
  "ds_read_b128 v[144:147], v161 offset:32768\n" \
  "ds_read_b128 v[148:151], v161 offset:33792\n" \
  "v_mfma_f32_16x16x128_f8f6f4 a[196:199], v[128:135], v[72:79], a[196:199] cbsz:1 blgp:1\n" \
  "s_add_u32 m0, s69, 69632\n" \
  "s_nop 0\n" \
  "buffer_load_dwordx4 v165, s[36:39], s57 offen lds\n" \
  "v_mfma_f32_16x16x128_f8f6f4 a[200:203], v[128:135], v[80:87], a[200:203] cbsz:1 blgp:1\n" \
  "s_add_u32 m0, s69, 73728\n" \
  "s_nop 0\n" \
  "buffer_load_dwordx4 v165, s[36:39], s58 offen lds\n" \
  "v_mfma_f32_16x16x128_f8f6f4 a[204:207], v[128:135], v[88:95], a[204:207] cbsz:1 blgp:1\n" \
  "v_mfma_f32_16x16x128_f8f6f4 a[208:211], v[136:143], v[64:71], a[208:211] cbsz:1 blgp:1\n" \
  "ds_read_b128 v[152:155], v161 offset:34816\n" \
  "ds_read_b128 v[156:159], v161 offset:35840\n" \
  "v_mfma_f32_16x16x128_f8f6f4 a[212:215], v[136:143], v[72:79], a[212:215] cbsz:1 blgp:1\n" \
  "v_readlane_b32 s67, v175, s70\n" \
  "v_readlane_b32 s76, v169, s70\n" \
  "v_readlane_b32 s71, v181, s70\n" \
  "v_mfma_f32_16x16x128_f8f6f4 a[216:219], v[136:143], v[80:87], a[216:219] cbsz:1 blgp:1\n" \
  "s_add_u32 m0, s69, 77824\n" \
  "s_nop 0\n" \
  "buffer_load_dwordx4 v165, s[36:39], s59 offen lds\n" \
  "v_mfma_f32_16x16x128_f8f6f4 a[220:223], v[136:143], v[88:95], a[220:223] cbsz:1 blgp:1\n" \
  "s_waitcnt lgkmcnt(2)\n" \
  "v_mfma_f32_16x16x128_f8f6f4 a[0:3], v[144:151], v[96:103], a[0:3] cbsz:1 blgp:1\n" \
  "ds_read_b128 v[128:131], v161 offset:36864\n" \
  "ds_read_b128 v[132:135], v161 offset:37888\n" \
  "v_mfma_f32_16x16x128_f8f6f4 a[4:7], v[144:151], v[104:111], a[4:7] cbsz:1 blgp:1\n" \
  "s_add_u32 m0, s69, 81920\n" \
  "s_nop 0\n" \
  "buffer_load_dwordx4 v165, s[36:39], s60 offen lds\n" \
  "v_mfma_f32_16x16x128_f8f6f4 a[8:11], v[144:151], v[112:119], a[8:11] cbsz:1 blgp:1\n" \
  "v_mfma_f32_16x16x128_f8f6f4 a[12:15], v[144:151], v[120:127], a[12:15] cbsz:1 blgp:1\n" \
  "s_waitcnt lgkmcnt(2)\n" \
  "v_mfma_f32_16x16x128_f8f6f4 a[16:19], v[152:159], v[96:103], a[16:19] cbsz:1 blgp:1\n" \
  "ds_read_b128 v[136:139], v161 offset:38912\n" \
  "ds_read_b128 v[140:143], v161 offset:39936\n" \
  "v_mfma_f32_16x16x128_f8f6f4 a[20:23], v[152:159], v[104:111], a[20:23] cbsz:1 blgp:1\n" \
  "s_add_u32 s70, s70, 1\n" \
  "s_nop 0\n" \
  "v_add_u32 v163, s76, v162\n" \
  "v_mfma_f32_16x16x128_f8f6f4 a[24:27], v[152:159], v[112:119], a[24:27] cbsz:1 blgp:1\n" \
  "s_add_u32 m0, s69, 86016\n" \
  "s_nop 0\n" \
  "buffer_load_dwordx4 v165, s[36:39], s61 offen lds\n" \
  "v_mfma_f32_16x16x128_f8f6f4 a[28:31], v[152:159], v[120:127], a[28:31] cbsz:1 blgp:1\n" \
  "s_waitcnt lgkmcnt(2)\n" \
  "v_mfma_f32_16x16x128_f8f6f4 a[32:35], v[128:135], v[96:103], a[32:35] cbsz:1 blgp:1\n" \
  "ds_read_b128 v[144:147], v161 offset:40960\n" \
  "ds_read_b128 v[148:151], v161 offset:41984\n" \
  "v_mfma_f32_16x16x128_f8f6f4 a[36:39], v[128:135], v[104:111], a[36:39] cbsz:1 blgp:1\n" \
  "s_add_u32 m0, s69, 90112\n" \
  "s_nop 0\n" \
  "buffer_load_dwordx4 v165, s[36:39], s62 offen lds\n" \
  "v_mfma_f32_16x16x128_f8f6f4 a[40:43], v[128:135], v[112:119], a[40:43] cbsz:1 blgp:1\n" \
  "v_mfma_f32_16x16x128_f8f6f4 a[44:47], v[128:135], v[120:127], a[44:47] cbsz:1 blgp:1\n" \
  "s_waitcnt lgkmcnt(2)\n" \
  "v_mfma_f32_16x16x128_f8f6f4 a[48:51], v[136:143], v[96:103], a[48:51] cbsz:1 blgp:1\n" \
  "ds_read_b128 v[152:155], v161 offset:43008\n" \
  "ds_read_b128 v[156:159], v161 offset:44032\n" \
  "v_mfma_f32_16x16x128_f8f6f4 a[52:55], v[136:143], v[104:111], a[52:55] cbsz:1 blgp:1\n" \
  "v_add_u32 v164, 4096, v163\n" \
  "s_cmp_gt_u32 s67, s68\n" \
  "s_cbranch_scc1 Lp42_%=\n" \
  "Lq42_%=:\n" \
  "buffer_load_dwordx4 v[64:67], v165, s[36:39], s75 offen offset:0\n" \
  "v_mfma_f32_16x16x128_f8f6f4 a[56:59], v[136:143], v[112:119], a[56:59] cbsz:1 blgp:1\n" \
  "v_mfma_f32_16x16x128_f8f6f4 a[60:63], v[136:143], v[120:127], a[60:63] cbsz:1 blgp:1\n" \
  "s_waitcnt lgkmcnt(2)\n" \
  "v_mfma_f32_16x16x128_f8f6f4 a[64:67], v[144:151], v[96:103], a[64:67] cbsz:1 blgp:1\n" \
  "ds_read_b128 v[128:131], v161 offset:45056\n" \
  "ds_read_b128 v[132:135], v161 offset:46080\n" \
  "v_mfma_f32_16x16x128_f8f6f4 a[68:71], v[144:151], v[104:111], a[68:71] cbsz:1 blgp:1\n" \
  "buffer_load_dwordx4 v[68:71], v165, s[36:39], s75 offen offset:1024\n" \
  "v_mfma_f32_16x16x128_f8f6f4 a[72:75], v[144:151], v[112:119], a[72:75] cbsz:1 blgp:1\n" \
  "v_mfma_f32_16x16x128_f8f6f4 a[76:79], v[144:151], v[120:127], a[76:79] cbsz:1 blgp:1\n" \
  "v_mul_f32 v196, s78, v196\n" \
  "v_mul_f32 v197, s78, v197\n" \
  "v_mul_f32 v198, s78, v198\n" \
  "v_mul_f32 v199, s78, v199\n" \
  "s_waitcnt lgkmcnt(2)\n" \
  "v_mfma_f32_16x16x128_f8f6f4 a[80:83], v[152:159], v[96:103], a[80:83] cbsz:1 blgp:1\n" \
  "ds_read_b128 v[136:139], v161 offset:47104\n" \
  "ds_read_b128 v[140:143], v161 offset:48128\n" \
  "v_mfma_f32_16x16x128_f8f6f4 a[84:87], v[152:159], v[104:111], a[84:87] cbsz:1 blgp:1\n" \
  "buffer_store_dwordx4 v[196:199], v167, s[40:43], s51 offen offset:0 nt\n" \
  "v_mfma_f32_16x16x128_f8f6f4 a[88:91], v[152:159], v[112:119], a[88:91] cbsz:1 blgp:1\n" \
  "v_mfma_f32_16x16x128_f8f6f4 a[92:95], v[152:159], v[120:127], a[92:95] cbsz:1 blgp:1\n" \
  "s_waitcnt lgkmcnt(2)\n" \
  "v_mfma_f32_16x16x128_f8f6f4 a[96:99], v[128:135], v[96:103], a[96:99] cbsz:1 blgp:1\n" \
  "ds_read_b128 v[144:147], v161 offset:49152\n" \
  "ds_read_b128 v[148:151], v161 offset:50176\n" \
  "v_mfma_f32_16x16x128_f8f6f4 a[100:103], v[128:135], v[104:111], a[100:103] cbsz:1 blgp:1\n" \
  "buffer_load_dwordx4 v[72:75], v165, s[36:39], s75 offen offset:2048\n" \
  "v_mfma_f32_16x16x128_f8f6f4 a[104:107], v[128:135], v[112:119], a[104:107] cbsz:1 blgp:1\n" \
  "v_mfma_f32_16x16x128_f8f6f4 a[108:111], v[128:135], v[120:127], a[108:111] cbsz:1 blgp:1\n" \
  "buffer_load_dwordx4 v[76:79], v165, s[36:39], s75 offen offset:3072\n" \
  "s_waitcnt lgkmcnt(2)\n" \
  "v_mfma_f32_16x16x128_f8f6f4 a[112:115], v[136:143], v[96:103], a[112:115] cbsz:1 blgp:1\n" \
  "ds_read_b128 v[152:155], v161 offset:51200\n" \
  "ds_read_b128 v[156:159], v161 offset:52224\n" \
  "v_mfma_f32_16x16x128_f8f6f4 a[116:119], v[136:143], v[104:111], a[116:119] cbsz:1 blgp:1\n" \
  "v_mfma_f32_16x16x128_f8f6f4 a[120:123], v[136:143], v[112:119], a[120:123] cbsz:1 blgp:1\n" \
  "buffer_load_dwordx4 v[80:83], v166, s[36:39], s75 offen offset:0\n" \
  "v_mfma_f32_16x16x128_f8f6f4 a[124:127], v[136:143], v[120:127], a[124:127] cbsz:1 blgp:1\n" \
  "s_waitcnt lgkmcnt(2)\n" \
  "v_mfma_f32_16x16x128_f8f6f4 a[128:131], v[144:151], v[96:103], a[128:131] cbsz:1 blgp:1\n" \
  "ds_read_b128 v[128:131], v161 offset:53248\n" \
  "ds_read_b128 v[132:135], v161 offset:54272\n" \
  "v_mfma_f32_16x16x128_f8f6f4 a[132:135], v[144:151], v[104:111], a[132:135] cbsz:1 blgp:1\n" \
  "v_mul_f32 v212, s78, v212\n" \
  "v_mul_f32 v213, s78, v213\n" \
  "v_mul_f32 v214, s78, v214\n" \
  "v_mul_f32 v215, s78, v215\n" \
  "v_mfma_f32_16x16x128_f8f6f4 a[136:139], v[144:151], v[112:119], a[136:139] cbsz:1 blgp:1\n" \
  "buffer_store_dwordx4 v[212:215], v167, s[40:43], s51 offen offset:64 nt\n" \
  "v_mfma_f32_16x16x128_f8f6f4 a[140:143], v[144:151], v[120:127], a[140:143] cbsz:1 blgp:1\n" \
  "s_waitcnt lgkmcnt(2)\n" \
  "v_mfma_f32_16x16x128_f8f6f4 a[144:147], v[152:159], v[96:103], a[144:147] cbsz:1 blgp:1\n" \
  "ds_read_b128 v[136:139], v161 offset:55296\n" \
  "ds_read_b128 v[140:143], v161 offset:56320\n" \
  "v_mfma_f32_16x16x128_f8f6f4 a[148:151], v[152:159], v[104:111], a[148:151] cbsz:1 blgp:1\n" \
  "buffer_load_dwordx4 v[84:87], v166, s[36:39], s75 offen offset:1024\n" \
  "v_mfma_f32_16x16x128_f8f6f4 a[152:155], v[152:159], v[112:119], a[152:155] cbsz:1 blgp:1\n" \
  "v_mfma_f32_16x16x128_f8f6f4 a[156:159], v[152:159], v[120:127], a[156:159] cbsz:1 blgp:1\n" \
  "s_waitcnt lgkmcnt(2)\n" \
  "v_mfma_f32_16x16x128_f8f6f4 a[160:163], v[128:135], v[96:103], a[160:163] cbsz:1 blgp:1\n" \
  "ds_read_b128 v[144:147], v161 offset:57344\n" \
  "ds_read_b128 v[148:151], v161 offset:58368\n" \
  "v_mfma_f32_16x16x128_f8f6f4 a[164:167], v[128:135], v[104:111], a[164:167] cbsz:1 blgp:1\n" \
  "buffer_load_dwordx4 v[88:91], v166, s[36:39], s75 offen offset:2048\n" \
  "v_mfma_f32_16x16x128_f8f6f4 a[168:171], v[128:135], v[112:119], a[168:171] cbsz:1 blgp:1\n" \
  "v_mfma_f32_16x16x128_f8f6f4 a[172:175], v[128:135], v[120:127], a[172:175] cbsz:1 blgp:1\n" \
  "buffer_load_dwordx4 v[92:95], v166, s[36:39], s75 offen offset:3072\n" \
  "s_waitcnt lgkmcnt(2)\n" \
  "v_mfma_f32_16x16x128_f8f6f4 a[176:179], v[136:143], v[96:103], a[176:179] cbsz:1 blgp:1\n" \
  "ds_read_b128 v[152:155], v161 offset:59392\n" \
  "ds_read_b128 v[156:159], v161 offset:60416\n" \
  "v_mfma_f32_16x16x128_f8f6f4 a[180:183], v[136:143], v[104:111], a[180:183] cbsz:1 blgp:1\n" \
  "v_mfma_f32_16x16x128_f8f6f4 a[184:187], v[136:143], v[112:119], a[184:187] cbsz:1 blgp:1\n" \
  "v_mfma_f32_16x16x128_f8f6f4 a[188:191], v[136:143], v[120:127], a[188:191] cbsz:1 blgp:1\n" \
  "s_waitcnt vmcnt(32) lgkmcnt(0)\n" \
  "s_barrier\n" \
  "v_mfma_f32_16x16x128_f8f6f4 a[192:195], v[144:151], v[96:103], a[192:195] cbsz:1 blgp:1\n" \
  "ds_read_b128 v[128:131], v160 offset:4096\n" \
  "ds_read_b128 v[132:135], v160 offset:5120\n" \
  "v_mfma_f32_16x16x128_f8f6f4 a[196:199], v[144:151], v[104:111], a[196:199] cbsz:1 blgp:1\n" \
  "s_mov_b32 s42, s72\n" \
  "s_mov_b32 s48, s71\n" \
  "s_add_u32 s49, s71, 1048576\n" \
  "s_add_u32 s50, s71, 2097152\n" \
  "v_mfma_f32_16x16x128_f8f6f4 a[200:203], v[144:151], v[112:119], a[200:203] cbsz:1 blgp:1\n" \
  "s_add_u32 s51, s71, 3145728\n" \
  "s_cmp_eq_u32 s70, 64\n" \
  "s_cbranch_scc1 Lp43_%=\n" \
  "Lq43_%=:\n" \
  "v_mfma_f32_16x16x128_f8f6f4 a[204:207], v[144:151], v[120:127], a[204:207] cbsz:1 blgp:1\n" \
  "v_accvgpr_read_b32 v184, a0\n" \
  "v_accvgpr_read_b32 v185, a1\n" \
  "v_accvgpr_read_b32 v186, a2\n" \
  "v_accvgpr_read_b32 v187, a3\n" \
  "v_mfma_f32_16x16x128_f8f6f4 a[208:211], v[152:159], v[96:103], a[208:211] cbsz:1 blgp:1\n" \
  "ds_read_b128 v[136:139], v160 offset:6144\n" \
  "ds_read_b128 v[140:143], v160 offset:7168\n" \
  "v_mfma_f32_16x16x128_f8f6f4 a[212:215], v[152:159], v[104:111], a[212:215] cbsz:1 blgp:1\n" \
  "v_accvgpr_read_b32 v200, a16\n" \
  "v_accvgpr_read_b32 v201, a17\n" \
  "v_accvgpr_read_b32 v202, a18\n" \
  "v_accvgpr_read_b32 v203, a19\n" \
  "v_mfma_f32_16x16x128_f8f6f4 a[216:219], v[152:159], v[112:119], a[216:219] cbsz:1 blgp:1\n" \
  "v_accvgpr_read_b32 v188, a4\n" \
  "v_accvgpr_read_b32 v189, a5\n" \
  "v_accvgpr_read_b32 v190, a6\n" \
  "v_accvgpr_read_b32 v191, a7\n" \
  "v_mfma_f32_16x16x128_f8f6f4 a[220:223], v[152:159], v[120:127], a[220:223] cbsz:1 blgp:1\n" \
  "v_accvgpr_read_b32 v204, a20\n" \
  "v_accvgpr_read_b32 v205, a21\n" \
  "v_accvgpr_read_b32 v206, a22\n" \
  "v_accvgpr_read_b32 v207, a23\n" \
  "s_waitcnt lgkmcnt(2)\n" \
  "v_mfma_f32_16x16x128_f8f6f4 a[32:35], v[128:135], v[0:7], a[32:35] cbsz:1 blgp:1\n" \
  "ds_read_b128 v[144:147], v160 offset:8192\n" \
  "ds_read_b128 v[148:151], v160 offset:9216\n" \
  "v_mfma_f32_16x16x128_f8f6f4 a[36:39], v[128:135], v[8:15], a[36:39] cbsz:1 blgp:1\n" \
  "v_accvgpr_read_b32 v192, a8\n" \
  "v_accvgpr_read_b32 v193, a9\n" \
  "v_accvgpr_read_b32 v194, a10\n" \
  "v_accvgpr_read_b32 v195, a11\n" \
  "v_mfma_f32_16x16x128_f8f6f4 a[40:43], v[128:135], v[16:23], a[40:43] cbsz:1 blgp:1\n" \
  "v_accvgpr_read_b32 v208, a24\n" \
  "v_accvgpr_read_b32 v209, a25\n" \
  "v_accvgpr_read_b32 v210, a26\n" \
  "v_accvgpr_read_b32 v211, a27\n" \
  "v_mfma_f32_16x16x128_f8f6f4 a[44:47], v[128:135], v[24:31], a[44:47] cbsz:1 blgp:1\n" \
  "v_accvgpr_read_b32 v196, a12\n" \
  "v_accvgpr_read_b32 v197, a13\n" \
  "v_accvgpr_read_b32 v198, a14\n" \
  "v_accvgpr_read_b32 v199, a15\n" \
  "s_waitcnt lgkmcnt(2)\n" \
  "v_mfma_f32_16x16x128_f8f6f4 a[48:51], v[136:143], v[0:7], a[48:51] cbsz:1 blgp:1\n" \
  "ds_read_b128 v[152:155], v160 offset:10240\n" \
  "ds_read_b128 v[156:159], v160 offset:11264\n" \
  "v_mfma_f32_16x16x128_f8f6f4 a[52:55], v[136:143], v[8:15], a[52:55] cbsz:1 blgp:1\n" \
  "v_accvgpr_read_b32 v212, a28\n" \
  "v_accvgpr_read_b32 v213, a29\n" \
  "v_accvgpr_read_b32 v214, a30\n" \
  "v_accvgpr_read_b32 v215, a31\n" \
  "s_add_u32 m0, s69, 102400\n" \
  "s_nop 0\n" \
  "buffer_load_dwordx4 v163, s[36:39], s57 offen lds\n" \
  "v_mfma_f32_16x16x128_f8f6f4 a[56:59], v[136:143], v[16:23], a[56:59] cbsz:1 blgp:1\n" \
  "s_add_u32 m0, s69, 106496\n" \
  "s_nop 0\n" \
  "buffer_load_dwordx4 v163, s[36:39], s58 offen lds\n" \
  "v_readlane_b32 s67, v175, s70\n" \
  "v_readlane_b32 s76, v169, s70\n" \
  "v_mfma_f32_16x16x128_f8f6f4 a[60:63], v[136:143], v[24:31], a[60:63] cbsz:1 blgp:1\n" \
  "s_add_u32 m0, s69, 110592\n" \
  "s_nop 0\n" \
  "buffer_load_dwordx4 v163, s[36:39], s59 offen lds\n" \
  "s_waitcnt lgkmcnt(2)\n" \
  "v_mfma_f32_16x16x128_f8f6f4 a[64:67], v[144:151], v[0:7], a[64:67] cbsz:1 blgp:1\n" \
  "ds_read_b128 v[128:131], v160 offset:12288\n" \
  "ds_read_b128 v[132:135], v160 offset:13312\n" \
  "v_mfma_f32_16x16x128_f8f6f4 a[68:71], v[144:151], v[8:15], a[68:71] cbsz:1 blgp:1\n" \
  "s_add_u32 m0, s69, 114688\n" \
  "s_nop 0\n" \
  "buffer_load_dwordx4 v163, s[36:39], s60 offen lds\n" \
  "v_mfma_f32_16x16x128_f8f6f4 a[72:75], v[144:151], v[16:23], a[72:75] cbsz:1 blgp:1\n" \
  "s_add_u32 s70, s70, 1\n" \
  "s_nop 0\n" \
  "v_add_u32 v165, s76, v162\n" \
  "v_mfma_f32_16x16x128_f8f6f4 a[76:79], v[144:151], v[24:31], a[76:79] cbsz:1 blgp:1\n" \
  "s_add_u32 m0, s69, 118784\n" \
  "s_nop 0\n" \
  "buffer_load_dwordx4 v163, s[36:39], s61 offen lds\n" \
  "s_waitcnt lgkmcnt(2)\n" \
  "v_mfma_f32_16x16x128_f8f6f4 a[80:83], v[152:159], v[0:7], a[80:83] cbsz:1 blgp:1\n" \
  "ds_read_b128 v[136:139], v160 offset:14336\n" \
  "ds_read_b128 v[140:143], v160 offset:15360\n" \
  "v_mfma_f32_16x16x128_f8f6f4 a[84:87], v[152:159], v[8:15], a[84:87] cbsz:1 blgp:1\n" \
  "s_add_u32 m0, s69, 122880\n" \
  "s_nop 0\n" \
  "buffer_load_dwordx4 v163, s[36:39], s62 offen lds\n" \
  "v_mfma_f32_16x16x128_f8f6f4 a[88:91], v[152:159], v[16:23], a[88:91] cbsz:1 blgp:1\n" \
  "v_mfma_f32_16x16x128_f8f6f4 a[92:95], v[152:159], v[24:31], a[92:95] cbsz:1 blgp:1\n" \
  "v_add_u32 v166, 4096, v165\n" \
  "s_cmp_gt_u32 s67, s68\n" \
  "s_cbranch_scc1 Lp44_%=\n" \
  "Lq44_%=:\n" \
  "s_waitcnt lgkmcnt(2)\n" \
  "v_mfma_f32_16x16x128_f8f6f4 a[96:99], v[128:135], v[0:7], a[96:99] cbsz:1 blgp:1\n" \
  "ds_read_b128 v[144:147], v160 offset:16384\n" \
  "ds_read_b128 v[148:151], v160 offset:17408\n" \
  "v_mfma_f32_16x16x128_f8f6f4 a[100:103], v[128:135], v[8:15], a[100:103] cbsz:1 blgp:1\n" \
  "buffer_load_dwordx4 v[96:99], v163, s[36:39], s75 offen offset:0\n" \
  "v_mfma_f32_16x16x128_f8f6f4 a[104:107], v[128:135], v[16:23], a[104:107] cbsz:1 blgp:1\n" \
  "buffer_load_dwordx4 v[100:103], v163, s[36:39], s75 offen offset:1024\n" \
  "v_mfma_f32_16x16x128_f8f6f4 a[108:111], v[128:135], v[24:31], a[108:111] cbsz:1 blgp:1\n" \
  "s_waitcnt lgkmcnt(2)\n" \
  "v_mfma_f32_16x16x128_f8f6f4 a[112:115], v[136:143], v[0:7], a[112:115] cbsz:1 blgp:1\n" \
  "ds_read_b128 v[152:155], v160 offset:18432\n" \
  "ds_read_b128 v[156:159], v160 offset:19456\n" \
  "v_mfma_f32_16x16x128_f8f6f4 a[116:119], v[136:143], v[8:15], a[116:119] cbsz:1 blgp:1\n" \
  "buffer_load_dwordx4 v[104:107], v163, s[36:39], s75 offen offset:2048\n" \
  "v_mfma_f32_16x16x128_f8f6f4 a[120:123], v[136:143], v[16:23], a[120:123] cbsz:1 blgp:1\n" \
  "buffer_load_dwordx4 v[108:111], v163, s[36:39], s75 offen offset:3072\n" \
  "v_mfma_f32_16x16x128_f8f6f4 a[124:127], v[136:143], v[24:31], a[124:127] cbsz:1 blgp:1\n" \
  "s_waitcnt lgkmcnt(2)\n" \
  "v_mfma_f32_16x16x128_f8f6f4 a[128:131], v[144:151], v[0:7], a[128:131] cbsz:1 blgp:1\n" \
  "ds_read_b128 v[128:131], v160 offset:20480\n" \
  "ds_read_b128 v[132:135], v160 offset:21504\n" \
  "v_mfma_f32_16x16x128_f8f6f4 a[132:135], v[144:151], v[8:15], a[132:135] cbsz:1 blgp:1\n" \
  "buffer_load_dwordx4 v[112:115], v164, s[36:39], s75 offen offset:0\n" \
  "v_mfma_f32_16x16x128_f8f6f4 a[136:139], v[144:151], v[16:23], a[136:139] cbsz:1 blgp:1\n" \
  "buffer_load_dwordx4 v[116:119], v164, s[36:39], s75 offen offset:1024\n" \
  "v_mfma_f32_16x16x128_f8f6f4 a[140:143], v[144:151], v[24:31], a[140:143] cbsz:1 blgp:1\n" \
  "s_waitcnt lgkmcnt(2)\n" \
  "v_mfma_f32_16x16x128_f8f6f4 a[144:147], v[152:159], v[0:7], a[144:147] cbsz:1 blgp:1\n" \
  "ds_read_b128 v[136:139], v160 offset:22528\n" \
  "ds_read_b128 v[140:143], v160 offset:23552\n" \
  "v_mfma_f32_16x16x128_f8f6f4 a[148:151], v[152:159], v[8:15], a[148:151] cbsz:1 blgp:1\n" \
  "v_mul_f32 v184, s78, v184\n" \
  "v_mul_f32 v185, s78, v185\n" \
  "v_mul_f32 v186, s78, v186\n" \
  "v_mul_f32 v187, s78, v187\n" \
  "buffer_store_dwordx4 v[184:187], v167, s[40:43], s48 offen offset:0 nt\n" \
  "v_mfma_f32_16x16x128_f8f6f4 a[152:155], v[152:159], v[16:23], a[152:155] cbsz:1 blgp:1\n" \
  "v_mfma_f32_16x16x128_f8f6f4 a[156:159], v[152:159], v[24:31], a[156:159] cbsz:1 blgp:1\n" \
  "buffer_load_dwordx4 v[120:123], v164, s[36:39], s75 offen offset:2048\n" \
  "s_waitcnt lgkmcnt(2)\n" \
  "v_mfma_f32_16x16x128_f8f6f4 a[160:163], v[128:135], v[0:7], a[160:163] cbsz:1 blgp:1\n" \
  "ds_read_b128 v[144:147], v160 offset:24576\n" \
  "ds_read_b128 v[148:151], v160 offset:25600\n" \
  "v_mfma_f32_16x16x128_f8f6f4 a[164:167], v[128:135], v[8:15], a[164:167] cbsz:1 blgp:1\n" \
  "v_mul_f32 v200, s78, v200\n" \
  "v_mul_f32 v201, s78, v201\n" \
  "v_mul_f32 v202, s78, v202\n" \
  "v_mul_f32 v203, s78, v203\n" \
  "v_mfma_f32_16x16x128_f8f6f4 a[168:171], v[128:135], v[16:23], a[168:171] cbsz:1 blgp:1\n" \
  "buffer_store_dwordx4 v[200:203], v167, s[40:43], s48 offen offset:64 nt\n" \
  "v_mfma_f32_16x16x128_f8f6f4 a[172:175], v[128:135], v[24:31], a[172:175] cbsz:1 blgp:1\n" \
  "s_waitcnt lgkmcnt(2)\n" \
  "v_mfma_f32_16x16x128_f8f6f4 a[176:179], v[136:143], v[0:7], a[176:179] cbsz:1 blgp:1\n" \
  "ds_read_b128 v[152:155], v160 offset:26624\n" \
  "ds_read_b128 v[156:159], v160 offset:27648\n" \
  "v_mfma_f32_16x16x128_f8f6f4 a[180:183], v[136:143], v[8:15], a[180:183] cbsz:1 blgp:1\n" \
  "buffer_load_dwordx4 v[124:127], v164, s[36:39], s75 offen offset:3072\n" \
  "v_mfma_f32_16x16x128_f8f6f4 a[184:187], v[136:143], v[16:23], a[184:187] cbsz:1 blgp:1\n" \
  "v_mfma_f32_16x16x128_f8f6f4 a[188:191], v[136:143], v[24:31], a[188:191] cbsz:1 blgp:1\n" \
  "s_waitcnt vmcnt(32) lgkmcnt(0)\n" \
  "s_barrier\n" \
  "v_mfma_f32_16x16x128_f8f6f4 a[192:195], v[144:151], v[0:7], a[192:195] cbsz:1 blgp:1\n" \
  "ds_read_b128 v[128:131], v160 offset:36864\n" \
  "ds_read_b128 v[132:135], v160 offset:37888\n" \
  "v_mfma_f32_16x16x128_f8f6f4 a[196:199], v[144:151], v[8:15], a[196:199] cbsz:1 blgp:1\n" \
  "s_add_u32 m0, s69, 8192\n" \
  "s_nop 0\n" \
  "buffer_load_dwordx4 v165, s[36:39], s58 offen lds\n" \
  "v_mfma_f32_16x16x128_f8f6f4 a[200:203], v[144:151], v[16:23], a[200:203] cbsz:1 blgp:1\n" \
  "s_add_u32 m0, s69, 12288\n" \
  "s_nop 0\n" \
  "buffer_load_dwordx4 v165, s[36:39], s59 offen lds\n" \
  "v_mfma_f32_16x16x128_f8f6f4 a[204:207], v[144:151], v[24:31], a[204:207] cbsz:1 blgp:1\n" \
  "v_mfma_f32_16x16x128_f8f6f4 a[208:211], v[152:159], v[0:7], a[208:211] cbsz:1 blgp:1\n" \
  "ds_read_b128 v[136:139], v160 offset:38912\n" \
  "ds_read_b128 v[140:143], v160 offset:39936\n" \
  "v_mfma_f32_16x16x128_f8f6f4 a[212:215], v[152:159], v[8:15], a[212:215] cbsz:1 blgp:1\n" \
  "v_readlane_b32 s67, v175, s70\n" \
  "v_readlane_b32 s76, v169, s70\n" \
  "s_add_u32 m0, s69, 16384\n" \
  "s_nop 0\n" \
  "buffer_load_dwordx4 v165, s[36:39], s60 offen lds\n" \
  "v_mfma_f32_16x16x128_f8f6f4 a[216:219], v[152:159], v[16:23], a[216:219] cbsz:1 blgp:1\n" \
  "v_mfma_f32_16x16x128_f8f6f4 a[220:223], v[152:159], v[24:31], a[220:223] cbsz:1 blgp:1\n" \
  "s_waitcnt lgkmcnt(2)\n" \
  "v_mfma_f32_16x16x128_f8f6f4 a[32:35], v[128:135], v[32:39], a[32:35] cbsz:1 blgp:1\n" \
  "ds_read_b128 v[144:147], v160 offset:40960\n" \
  "ds_read_b128 v[148:151], v160 offset:41984\n" \
  "v_mfma_f32_16x16x128_f8f6f4 a[36:39], v[128:135], v[40:47], a[36:39] cbsz:1 blgp:1\n" \
  "s_add_u32 m0, s69, 20480\n" \
  "s_nop 0\n" \
  "buffer_load_dwordx4 v165, s[36:39], s61 offen lds\n" \
  "v_mfma_f32_16x16x128_f8f6f4 a[40:43], v[128:135], v[48:55], a[40:43] cbsz:1 blgp:1\n" \
  "v_mfma_f32_16x16x128_f8f6f4 a[44:47], v[128:135], v[56:63], a[44:47] cbsz:1 blgp:1\n" \
  "s_add_u32 s70, s70, 1\n" \
  "s_nop 0\n" \
  "v_add_u32 v163, s76, v162\n" \
  "s_waitcnt lgkmcnt(2)\n" \
  "v_mfma_f32_16x16x128_f8f6f4 a[48:51], v[136:143], v[32:39], a[48:51] cbsz:1 blgp:1\n" \
  "ds_read_b128 v[152:155], v160 offset:43008\n" \
  "ds_read_b128 v[156:159], v160 offset:44032\n" \
  "v_mfma_f32_16x16x128_f8f6f4 a[52:55], v[136:143], v[40:47], a[52:55] cbsz:1 blgp:1\n" \
  "s_add_u32 m0, s69, 24576\n" \
  "s_nop 0\n" \
  "buffer_load_dwordx4 v165, s[36:39], s62 offen lds\n" \
  "v_mfma_f32_16x16x128_f8f6f4 a[56:59], v[136:143], v[48:55], a[56:59] cbsz:1 blgp:1\n" \
  "v_mfma_f32_16x16x128_f8f6f4 a[60:63], v[136:143], v[56:63], a[60:63] cbsz:1 blgp:1\n" \
  "v_add_u32 v164, 4096, v163\n" \
  "s_cmp_gt_u32 s67, s68\n" \
  "s_cbranch_scc1 Lp45_%=\n" \
  "Lq45_%=:\n" \
  "s_waitcnt lgkmcnt(2)\n" \
  "v_mfma_f32_16x16x128_f8f6f4 a[64:67], v[144:151], v[32:39], a[64:67] cbsz:1 blgp:1\n" \
  "ds_read_b128 v[128:131], v160 offset:45056\n" \
  "ds_read_b128 v[132:135], v160 offset:46080\n" \
  "v_mfma_f32_16x16x128_f8f6f4 a[68:71], v[144:151], v[40:47], a[68:71] cbsz:1 blgp:1\n" \
  "buffer_load_dwordx4 v[0:3], v165, s[36:39], s75 offen offset:0\n" \
  "v_mfma_f32_16x16x128_f8f6f4 a[72:75], v[144:151], v[48:55], a[72:75] cbsz:1 blgp:1\n" \
  "v_mfma_f32_16x16x128_f8f6f4 a[76:79], v[144:151], v[56:63], a[76:79] cbsz:1 blgp:1\n" \
  "buffer_load_dwordx4 v[4:7], v165, s[36:39], s75 offen offset:1024\n" \
  "s_waitcnt lgkmcnt(2)\n" \
  "v_mfma_f32_16x16x128_f8f6f4 a[80:83], v[152:159], v[32:39], a[80:83] cbsz:1 blgp:1\n" \
  "ds_read_b128 v[136:139], v160 offset:47104\n" \
  "ds_read_b128 v[140:143], v160 offset:48128\n" \
  "v_mfma_f32_16x16x128_f8f6f4 a[84:87], v[152:159], v[40:47], a[84:87] cbsz:1 blgp:1\n" \
  "v_mfma_f32_16x16x128_f8f6f4 a[88:91], v[152:159], v[48:55], a[88:91] cbsz:1 blgp:1\n" \
  "v_mul_f32 v188, s78, v188\n" \
  "v_mul_f32 v189, s78, v189\n" \
  "v_mul_f32 v190, s78, v190\n" \
  "v_mul_f32 v191, s78, v191\n" \
  "v_mfma_f32_16x16x128_f8f6f4 a[92:95], v[152:159], v[56:63], a[92:95] cbsz:1 blgp:1\n" \
  "buffer_store_dwordx4 v[188:191], v167, s[40:43], s49 offen offset:0 nt\n" \
  "s_waitcnt lgkmcnt(2)\n" \
  "v_mfma_f32_16x16x128_f8f6f4 a[96:99], v[128:135], v[32:39], a[96:99] cbsz:1 blgp:1\n" \
  "ds_read_b128 v[144:147], v160 offset:49152\n" \
  "ds_read_b128 v[148:151], v160 offset:50176\n" \
  "v_mfma_f32_16x16x128_f8f6f4 a[100:103], v[128:135], v[40:47], a[100:103] cbsz:1 blgp:1\n" \
  "v_mfma_f32_16x16x128_f8f6f4 a[104:107], v[128:135], v[48:55], a[104:107] cbsz:1 blgp:1\n" \
  "buffer_load_dwordx4 v[8:11], v165, s[36:39], s75 offen offset:2048\n" \
  "v_mfma_f32_16x16x128_f8f6f4 a[108:111], v[128:135], v[56:63], a[108:111] cbsz:1 blgp:1\n" \
  "s_waitcnt lgkmcnt(2)\n" \
  "v_mfma_f32_16x16x128_f8f6f4 a[112:115], v[136:143], v[32:39], a[112:115] cbsz:1 blgp:1\n" \
  "ds_read_b128 v[152:155], v160 offset:51200\n" \
  "ds_read_b128 v[156:159], v160 offset:52224\n" \
  "v_mfma_f32_16x16x128_f8f6f4 a[116:119], v[136:143], v[40:47], a[116:119] cbsz:1 blgp:1\n" \
  "buffer_load_dwordx4 v[12:15], v165, s[36:39], s75 offen offset:3072\n" \
  "v_mfma_f32_16x16x128_f8f6f4 a[120:123], v[136:143], v[48:55], a[120:123] cbsz:1 blgp:1\n" \
  "v_mfma_f32_16x16x128_f8f6f4 a[124:127], v[136:143], v[56:63], a[124:127] cbsz:1 blgp:1\n" \
  "buffer_load_dwordx4 v[16:19], v166, s[36:39], s75 offen offset:0\n" \
  "s_waitcnt lgkmcnt(2)\n" \
  "v_mfma_f32_16x16x128_f8f6f4 a[128:131], v[144:151], v[32:39], a[128:131] cbsz:1 blgp:1\n" \
  "ds_read_b128 v[128:131], v160 offset:53248\n" \
  "ds_read_b128 v[132:135], v160 offset:54272\n" \
  "v_mfma_f32_16x16x128_f8f6f4 a[132:135], v[144:151], v[40:47], a[132:135] cbsz:1 blgp:1\n" \
  "v_mfma_f32_16x16x128_f8f6f4 a[136:139], v[144:151], v[48:55], a[136:139] cbsz:1 blgp:1\n" \
  "v_mul_f32 v204, s78, v204\n" \
  "v_mul_f32 v205, s78, v205\n" \
  "v_mul_f32 v206, s78, v206\n" \
  "v_mul_f32 v207, s78, v207\n" \
  "v_mfma_f32_16x16x128_f8f6f4 a[140:143], v[144:151], v[56:63], a[140:143] cbsz:1 blgp:1\n" \
  "s_waitcnt lgkmcnt(2)\n" \
  "v_mfma_f32_16x16x128_f8f6f4 a[144:147], v[152:159], v[32:39], a[144:147] cbsz:1 blgp:1\n" \
  "ds_read_b128 v[136:139], v160 offset:55296\n" \
  "ds_read_b128 v[140:143], v160 offset:56320\n" \
  "v_mfma_f32_16x16x128_f8f6f4 a[148:151], v[152:159], v[40:47], a[148:151] cbsz:1 blgp:1\n" \
  "buffer_store_dwordx4 v[204:207], v167, s[40:43], s49 offen offset:64 nt\n" \
  "v_mfma_f32_16x16x128_f8f6f4 a[152:155], v[152:159], v[48:55], a[152:155] cbsz:1 blgp:1\n" \
  "buffer_load_dwordx4 v[20:23], v166, s[36:39], s75 offen offset:1024\n" \
  "v_mfma_f32_16x16x128_f8f6f4 a[156:159], v[152:159], v[56:63], a[156:159] cbsz:1 blgp:1\n" \
  "s_waitcnt lgkmcnt(2)\n" \
  "v_mfma_f32_16x16x128_f8f6f4 a[160:163], v[128:135], v[32:39], a[160:163] cbsz:1 blgp:1\n" \
  "ds_read_b128 v[144:147], v160 offset:57344\n" \
  "ds_read_b128 v[148:151], v160 offset:58368\n" \
  "v_mfma_f32_16x16x128_f8f6f4 a[164:167], v[128:135], v[40:47], a[164:167] cbsz:1 blgp:1\n" \
  "buffer_load_dwordx4 v[24:27], v166, s[36:39], s75 offen offset:2048\n" \
  "v_mfma_f32_16x16x128_f8f6f4 a[168:171], v[128:135], v[48:55], a[168:171] cbsz:1 blgp:1\n" \
  "v_mfma_f32_16x16x128_f8f6f4 a[172:175], v[128:135], v[56:63], a[172:175] cbsz:1 blgp:1\n" \
  "s_waitcnt lgkmcnt(2)\n" \
  "v_mfma_f32_16x16x128_f8f6f4 a[176:179], v[136:143], v[32:39], a[176:179] cbsz:1 blgp:1\n" \
  "ds_read_b128 v[152:155], v160 offset:59392\n" \
  "ds_read_b128 v[156:159], v160 offset:60416\n" \
  "v_mfma_f32_16x16x128_f8f6f4 a[180:183], v[136:143], v[40:47], a[180:183] cbsz:1 blgp:1\n" \
  "buffer_load_dwordx4 v[28:31], v166, s[36:39], s75 offen offset:3072\n" \
  "v_mfma_f32_16x16x128_f8f6f4 a[184:187], v[136:143], v[48:55], a[184:187] cbsz:1 blgp:1\n" \
  "v_mfma_f32_16x16x128_f8f6f4 a[188:191], v[136:143], v[56:63], a[188:191] cbsz:1 blgp:1\n" \
  "s_waitcnt vmcnt(31) lgkmcnt(0)\n" \
  "s_barrier\n" \
  "v_mfma_f32_16x16x128_f8f6f4 a[192:195], v[144:151], v[32:39], a[192:195] cbsz:1 blgp:1\n" \
  "ds_read_b128 v[128:131], v161 offset:4096\n" \
  "ds_read_b128 v[132:135], v161 offset:5120\n" \
  "v_mfma_f32_16x16x128_f8f6f4 a[196:199], v[144:151], v[40:47], a[196:199] cbsz:1 blgp:1\n" \
  "s_add_u32 m0, s69, 40960\n" \
  "s_nop 0\n" \
  "buffer_load_dwordx4 v163, s[36:39], s58 offen lds\n" \
  "v_mfma_f32_16x16x128_f8f6f4 a[200:203], v[144:151], v[48:55], a[200:203] cbsz:1 blgp:1\n" \
  "s_add_u32 m0, s69, 45056\n" \
  "s_nop 0\n" \
  "buffer_load_dwordx4 v163, s[36:39], s59 offen lds\n" \
  "v_mfma_f32_16x16x128_f8f6f4 a[204:207], v[144:151], v[56:63], a[204:207] cbsz:1 blgp:1\n" \
  "v_mfma_f32_16x16x128_f8f6f4 a[208:211], v[152:159], v[32:39], a[208:211] cbsz:1 blgp:1\n" \
  "ds_read_b128 v[136:139], v161 offset:6144\n" \
  "ds_read_b128 v[140:143], v161 offset:7168\n" \
  "v_mfma_f32_16x16x128_f8f6f4 a[212:215], v[152:159], v[40:47], a[212:215] cbsz:1 blgp:1\n" \
  "v_readlane_b32 s67, v175, s70\n" \
  "v_readlane_b32 s76, v169, s70\n" \
  "s_add_u32 m0, s69, 49152\n" \
  "s_nop 0\n" \
  "buffer_load_dwordx4 v163, s[36:39], s60 offen lds\n" \
  "v_mfma_f32_16x16x128_f8f6f4 a[216:219], v[152:159], v[48:55], a[216:219] cbsz:1 blgp:1\n" \
  "v_mfma_f32_16x16x128_f8f6f4 a[220:223], v[152:159], v[56:63], a[220:223] cbsz:1 blgp:1\n" \
  "s_waitcnt lgkmcnt(2)\n" \
  "v_mfma_f32_16x16x128_f8f6f4 a[32:35], v[128:135], v[64:71], a[32:35] cbsz:1 blgp:1\n" \
  "ds_read_b128 v[144:147], v161 offset:8192\n" \
  "ds_read_b128 v[148:151], v161 offset:9216\n" \
  "v_mfma_f32_16x16x128_f8f6f4 a[36:39], v[128:135], v[72:79], a[36:39] cbsz:1 blgp:1\n" \
  "s_add_u32 m0, s69, 53248\n" \
  "s_nop 0\n" \
  "buffer_load_dwordx4 v163, s[36:39], s61 offen lds\n" \
  "v_mfma_f32_16x16x128_f8f6f4 a[40:43], v[128:135], v[80:87], a[40:43] cbsz:1 blgp:1\n" \
  "v_mfma_f32_16x16x128_f8f6f4 a[44:47], v[128:135], v[88:95], a[44:47] cbsz:1 blgp:1\n" \
  "s_add_u32 s70, s70, 1\n" \
  "s_nop 0\n" \
  "v_add_u32 v165, s76, v162\n" \
  "s_waitcnt lgkmcnt(2)\n" \
  "v_mfma_f32_16x16x128_f8f6f4 a[48:51], v[136:143], v[64:71], a[48:51] cbsz:1 blgp:1\n" \
  "ds_read_b128 v[152:155], v161 offset:10240\n" \
  "ds_read_b128 v[156:159], v161 offset:11264\n" \
  "v_mfma_f32_16x16x128_f8f6f4 a[52:55], v[136:143], v[72:79], a[52:55] cbsz:1 blgp:1\n" \
  "s_add_u32 m0, s69, 57344\n" \
  "s_nop 0\n" \
  "buffer_load_dwordx4 v163, s[36:39], s62 offen lds\n" \
  "v_mfma_f32_16x16x128_f8f6f4 a[56:59], v[136:143], v[80:87], a[56:59] cbsz:1 blgp:1\n" \
  "v_mfma_f32_16x16x128_f8f6f4 a[60:63], v[136:143], v[88:95], a[60:63] cbsz:1 blgp:1\n" \
  "v_add_u32 v166, 4096, v165\n" \
  "s_cmp_gt_u32 s67, s68\n" \
  "s_cbranch_scc1 Lp46_%=\n" \
  "Lq46_%=:\n" \
  "s_waitcnt lgkmcnt(2)\n" \
  "v_mfma_f32_16x16x128_f8f6f4 a[64:67], v[144:151], v[64:71], a[64:67] cbsz:1 blgp:1\n" \
  "ds_read_b128 v[128:131], v161 offset:12288\n" \
  "ds_read_b128 v[132:135], v161 offset:13312\n" \
  "v_mfma_f32_16x16x128_f8f6f4 a[68:71], v[144:151], v[72:79], a[68:71] cbsz:1 blgp:1\n" \
  "buffer_load_dwordx4 v[32:35], v163, s[36:39], s75 offen offset:0\n" \
  "v_mfma_f32_16x16x128_f8f6f4 a[72:75], v[144:151], v[80:87], a[72:75] cbsz:1 blgp:1\n" \
  "v_mfma_f32_16x16x128_f8f6f4 a[76:79], v[144:151], v[88:95], a[76:79] cbsz:1 blgp:1\n" \
  "buffer_load_dwordx4 v[36:39], v163, s[36:39], s75 offen offset:1024\n" \
  "s_waitcnt lgkmcnt(2)\n" \
  "v_mfma_f32_16x16x128_f8f6f4 a[80:83], v[152:159], v[64:71], a[80:83] cbsz:1 blgp:1\n" \
  "ds_read_b128 v[136:139], v161 offset:14336\n" \
  "ds_read_b128 v[140:143], v161 offset:15360\n" \
  "v_mfma_f32_16x16x128_f8f6f4 a[84:87], v[152:159], v[72:79], a[84:87] cbsz:1 blgp:1\n" \
  "v_mfma_f32_16x16x128_f8f6f4 a[88:91], v[152:159], v[80:87], a[88:91] cbsz:1 blgp:1\n" \
  "v_mul_f32 v192, s78, v192\n" \
  "v_mul_f32 v193, s78, v193\n" \
  "v_mul_f32 v194, s78, v194\n" \
  "v_mul_f32 v195, s78, v195\n" \
  "v_mfma_f32_16x16x128_f8f6f4 a[92:95], v[152:159], v[88:95], a[92:95] cbsz:1 blgp:1\n" \
  "buffer_store_dwordx4 v[192:195], v167, s[40:43], s50 offen offset:0 nt\n" \
  "s_waitcnt lgkmcnt(2)\n" \
  "v_mfma_f32_16x16x128_f8f6f4 a[96:99], v[128:135], v[64:71], a[96:99] cbsz:1 blgp:1\n" \
  "ds_read_b128 v[144:147], v161 offset:16384\n" \
  "ds_read_b128 v[148:151], v161 offset:17408\n" \
  "v_mfma_f32_16x16x128_f8f6f4 a[100:103], v[128:135], v[72:79], a[100:103] cbsz:1 blgp:1\n" \
  "v_mfma_f32_16x16x128_f8f6f4 a[104:107], v[128:135], v[80:87], a[104:107] cbsz:1 blgp:1\n" \
  "buffer_load_dwordx4 v[40:43], v163, s[36:39], s75 offen offset:2048\n" \
  "v_mfma_f32_16x16x128_f8f6f4 a[108:111], v[128:135], v[88:95], a[108:111] cbsz:1 blgp:1\n" \
  "s_waitcnt lgkmcnt(2)\n" \
  "v_mfma_f32_16x16x128_f8f6f4 a[112:115], v[136:143], v[64:71], a[112:115] cbsz:1 blgp:1\n" \
  "ds_read_b128 v[152:155], v161 offset:18432\n" \
  "ds_read_b128 v[156:159], v161 offset:19456\n" \
  "v_mfma_f32_16x16x128_f8f6f4 a[116:119], v[136:143], v[72:79], a[116:119] cbsz:1 blgp:1\n" \
  "buffer_load_dwordx4 v[44:47], v163, s[36:39], s75 offen offset:3072\n" \
  "v_mfma_f32_16x16x128_f8f6f4 a[120:123], v[136:143], v[80:87], a[120:123] cbsz:1 blgp:1\n" \
  "v_mfma_f32_16x16x128_f8f6f4 a[124:127], v[136:143], v[88:95], a[124:127] cbsz:1 blgp:1\n" \
  "buffer_load_dwordx4 v[48:51], v164, s[36:39], s75 offen offset:0\n" \
  "s_waitcnt lgkmcnt(2)\n" \
  "v_mfma_f32_16x16x128_f8f6f4 a[128:131], v[144:151], v[64:71], a[128:131] cbsz:1 blgp:1\n" \
  "ds_read_b128 v[128:131], v161 offset:20480\n" \
  "ds_read_b128 v[132:135], v161 offset:21504\n" \
  "v_mfma_f32_16x16x128_f8f6f4 a[132:135], v[144:151], v[72:79], a[132:135] cbsz:1 blgp:1\n" \
  "v_mfma_f32_16x16x128_f8f6f4 a[136:139], v[144:151], v[80:87], a[136:139] cbsz:1 blgp:1\n" \
  "v_mul_f32 v208, s78, v208\n" \
  "v_mul_f32 v209, s78, v209\n" \
  "v_mul_f32 v210, s78, v210\n" \
  "v_mul_f32 v211, s78, v211\n" \
  "v_mfma_f32_16x16x128_f8f6f4 a[140:143], v[144:151], v[88:95], a[140:143] cbsz:1 blgp:1\n" \
  "s_waitcnt lgkmcnt(2)\n" \
  "v_mfma_f32_16x16x128_f8f6f4 a[144:147], v[152:159], v[64:71], a[144:147] cbsz:1 blgp:1\n" \
  "ds_read_b128 v[136:139], v161 offset:22528\n" \
  "ds_read_b128 v[140:143], v161 offset:23552\n" \
  "v_mfma_f32_16x16x128_f8f6f4 a[148:151], v[152:159], v[72:79], a[148:151] cbsz:1 blgp:1\n" \
  "buffer_store_dwordx4 v[208:211], v167, s[40:43], s50 offen offset:64 nt\n" \
  "v_mfma_f32_16x16x128_f8f6f4 a[152:155], v[152:159], v[80:87], a[152:155] cbsz:1 blgp:1\n" \
  "buffer_load_dwordx4 v[52:55], v164, s[36:39], s75 offen offset:1024\n" \
  "v_mfma_f32_16x16x128_f8f6f4 a[156:159], v[152:159], v[88:95], a[156:159] cbsz:1 blgp:1\n" \
  "s_waitcnt lgkmcnt(2)\n" \
  "v_mfma_f32_16x16x128_f8f6f4 a[160:163], v[128:135], v[64:71], a[160:163] cbsz:1 blgp:1\n" \
  "ds_read_b128 v[144:147], v161 offset:24576\n" \
  "ds_read_b128 v[148:151], v161 offset:25600\n" \
  "v_mfma_f32_16x16x128_f8f6f4 a[164:167], v[128:135], v[72:79], a[164:167] cbsz:1 blgp:1\n" \
  "buffer_load_dwordx4 v[56:59], v164, s[36:39], s75 offen offset:2048\n" \
  "v_mfma_f32_16x16x128_f8f6f4 a[168:171], v[128:135], v[80:87], a[168:171] cbsz:1 blgp:1\n" \
  "v_mfma_f32_16x16x128_f8f6f4 a[172:175], v[128:135], v[88:95], a[172:175] cbsz:1 blgp:1\n" \
  "s_waitcnt lgkmcnt(2)\n" \
  "v_mfma_f32_16x16x128_f8f6f4 a[176:179], v[136:143], v[64:71], a[176:179] cbsz:1 blgp:1\n" \
  "ds_read_b128 v[152:155], v161 offset:26624\n" \
  "ds_read_b128 v[156:159], v161 offset:27648\n" \
  "v_mfma_f32_16x16x128_f8f6f4 a[180:183], v[136:143], v[72:79], a[180:183] cbsz:1 blgp:1\n" \
  "buffer_load_dwordx4 v[60:63], v164, s[36:39], s75 offen offset:3072\n" \
  "v_mfma_f32_16x16x128_f8f6f4 a[184:187], v[136:143], v[80:87], a[184:187] cbsz:1 blgp:1\n" \
  "v_mfma_f32_16x16x128_f8f6f4 a[188:191], v[136:143], v[88:95], a[188:191] cbsz:1 blgp:1\n" \
  "s_waitcnt vmcnt(30) lgkmcnt(0)\n" \
  "s_barrier\n" \
  "v_mfma_f32_16x16x128_f8f6f4 a[192:195], v[144:151], v[64:71], a[192:195] cbsz:1 blgp:1\n" \
  "ds_read_b128 v[128:131], v161 offset:36864\n" \
  "ds_read_b128 v[132:135], v161 offset:37888\n" \
  "v_mfma_f32_16x16x128_f8f6f4 a[196:199], v[144:151], v[72:79], a[196:199] cbsz:1 blgp:1\n" \
  "s_add_u32 m0, s69, 73728\n" \
  "s_nop 0\n" \
  "buffer_load_dwordx4 v165, s[36:39], s58 offen lds\n" \
  "v_mfma_f32_16x16x128_f8f6f4 a[200:203], v[144:151], v[80:87], a[200:203] cbsz:1 blgp:1\n" \
  "s_add_u32 m0, s69, 77824\n" \
  "s_nop 0\n" \
  "buffer_load_dwordx4 v165, s[36:39], s59 offen lds\n" \
  "v_mfma_f32_16x16x128_f8f6f4 a[204:207], v[144:151], v[88:95], a[204:207] cbsz:1 blgp:1\n" \
  "v_mfma_f32_16x16x128_f8f6f4 a[208:211], v[152:159], v[64:71], a[208:211] cbsz:1 blgp:1\n" \
  "ds_read_b128 v[136:139], v161 offset:38912\n" \
  "ds_read_b128 v[140:143], v161 offset:39936\n" \
  "v_mfma_f32_16x16x128_f8f6f4 a[212:215], v[152:159], v[72:79], a[212:215] cbsz:1 blgp:1\n" \
  "v_readlane_b32 s67, v175, s70\n" \
  "v_readlane_b32 s76, v169, s70\n" \
  "v_readlane_b32 s71, v181, s70\n" \
  "s_add_u32 m0, s69, 81920\n" \
  "s_nop 0\n" \
  "buffer_load_dwordx4 v165, s[36:39], s60 offen lds\n" \
  "v_mfma_f32_16x16x128_f8f6f4 a[216:219], v[152:159], v[80:87], a[216:219] cbsz:1 blgp:1\n" \
  "v_mfma_f32_16x16x128_f8f6f4 a[220:223], v[152:159], v[88:95], a[220:223] cbsz:1 blgp:1\n" \
  "s_waitcnt lgkmcnt(2)\n" \
  "v_mfma_f32_16x16x128_f8f6f4 a[32:35], v[128:135], v[96:103], a[32:35] cbsz:1 blgp:1\n" \
  "ds_read_b128 v[144:147], v161 offset:40960\n" \
  "ds_read_b128 v[148:151], v161 offset:41984\n" \
  "v_mfma_f32_16x16x128_f8f6f4 a[36:39], v[128:135], v[104:111], a[36:39] cbsz:1 blgp:1\n" \
  "s_add_u32 m0, s69, 86016\n" \
  "s_nop 0\n" \
  "buffer_load_dwordx4 v165, s[36:39], s61 offen lds\n" \
  "v_mfma_f32_16x16x128_f8f6f4 a[40:43], v[128:135], v[112:119], a[40:43] cbsz:1 blgp:1\n" \
  "v_mfma_f32_16x16x128_f8f6f4 a[44:47], v[128:135], v[120:127], a[44:47] cbsz:1 blgp:1\n" \
  "s_add_u32 s70, s70, 1\n" \
  "s_nop 0\n" \
  "v_add_u32 v163, s76, v162\n" \
  "s_waitcnt lgkmcnt(2)\n" \
  "v_mfma_f32_16x16x128_f8f6f4 a[48:51], v[136:143], v[96:103], a[48:51] cbsz:1 blgp:1\n" \
  "ds_read_b128 v[152:155], v161 offset:43008\n" \
  "ds_read_b128 v[156:159], v161 offset:44032\n" \
  "v_mfma_f32_16x16x128_f8f6f4 a[52:55], v[136:143], v[104:111], a[52:55] cbsz:1 blgp:1\n" \
  "s_add_u32 m0, s69, 90112\n" \
  "s_nop 0\n" \
  "buffer_load_dwordx4 v165, s[36:39], s62 offen lds\n" \
  "v_mfma_f32_16x16x128_f8f6f4 a[56:59], v[136:143], v[112:119], a[56:59] cbsz:1 blgp:1\n" \
  "v_mfma_f32_16x16x128_f8f6f4 a[60:63], v[136:143], v[120:127], a[60:63] cbsz:1 blgp:1\n" \
  "v_add_u32 v164, 4096, v163\n" \
  "s_cmp_gt_u32 s67, s68\n" \
  "s_cbranch_scc1 Lp47_%=\n" \
  "Lq47_%=:\n" \
  "s_waitcnt lgkmcnt(2)\n" \
  "v_mfma_f32_16x16x128_f8f6f4 a[64:67], v[144:151], v[96:103], a[64:67] cbsz:1 blgp:1\n" \
  "ds_read_b128 v[128:131], v161 offset:45056\n" \
  "ds_read_b128 v[132:135], v161 offset:46080\n" \
  "v_mfma_f32_16x16x128_f8f6f4 a[68:71], v[144:151], v[104:111], a[68:71] cbsz:1 blgp:1\n" \
  "buffer_load_dwordx4 v[64:67], v165, s[36:39], s75 offen offset:0\n" \
  "v_mfma_f32_16x16x128_f8f6f4 a[72:75], v[144:151], v[112:119], a[72:75] cbsz:1 blgp:1\n" \
  "v_mfma_f32_16x16x128_f8f6f4 a[76:79], v[144:151], v[120:127], a[76:79] cbsz:1 blgp:1\n" \
  "buffer_load_dwordx4 v[68:71], v165, s[36:39], s75 offen offset:1024\n" \
  "s_waitcnt lgkmcnt(2)\n" \
  "v_mfma_f32_16x16x128_f8f6f4 a[80:83], v[152:159], v[96:103], a[80:83] cbsz:1 blgp:1\n" \
  "ds_read_b128 v[136:139], v161 offset:47104\n" \
  "ds_read_b128 v[140:143], v161 offset:48128\n" \
  "v_mfma_f32_16x16x128_f8f6f4 a[84:87], v[152:159], v[104:111], a[84:87] cbsz:1 blgp:1\n" \
  "v_mfma_f32_16x16x128_f8f6f4 a[88:91], v[152:159], v[112:119], a[88:91] cbsz:1 blgp:1\n" \
  "v_mul_f32 v196, s78, v196\n" \
  "v_mul_f32 v197, s78, v197\n" \
  "v_mul_f32 v198, s78, v198\n" \
  "v_mul_f32 v199, s78, v199\n" \
  "v_mfma_f32_16x16x128_f8f6f4 a[92:95], v[152:159], v[120:127], a[92:95] cbsz:1 blgp:1\n" \
  "buffer_store_dwordx4 v[196:199], v167, s[40:43], s51 offen offset:0 nt\n" \
  "s_waitcnt lgkmcnt(2)\n" \
  "v_mfma_f32_16x16x128_f8f6f4 a[96:99], v[128:135], v[96:103], a[96:99] cbsz:1 blgp:1\n" \
  "ds_read_b128 v[144:147], v161 offset:49152\n" \
  "ds_read_b128 v[148:151], v161 offset:50176\n" \
  "v_mfma_f32_16x16x128_f8f6f4 a[100:103], v[128:135], v[104:111], a[100:103] cbsz:1 blgp:1\n" \
  "v_mfma_f32_16x16x128_f8f6f4 a[104:107], v[128:135], v[112:119], a[104:107] cbsz:1 blgp:1\n" \
  "buffer_load_dwordx4 v[72:75], v165, s[36:39], s75 offen offset:2048\n" \
  "v_mfma_f32_16x16x128_f8f6f4 a[108:111], v[128:135], v[120:127], a[108:111] cbsz:1 blgp:1\n" \
  "s_waitcnt lgkmcnt(2)\n" \
  "v_mfma_f32_16x16x128_f8f6f4 a[112:115], v[136:143], v[96:103], a[112:115] cbsz:1 blgp:1\n" \
  "ds_read_b128 v[152:155], v161 offset:51200\n" \
  "ds_read_b128 v[156:159], v161 offset:52224\n" \
  "v_mfma_f32_16x16x128_f8f6f4 a[116:119], v[136:143], v[104:111], a[116:119] cbsz:1 blgp:1\n" \
  "buffer_load_dwordx4 v[76:79], v165, s[36:39], s75 offen offset:3072\n" \
  "v_mfma_f32_16x16x128_f8f6f4 a[120:123], v[136:143], v[112:119], a[120:123] cbsz:1 blgp:1\n" \
  "v_mfma_f32_16x16x128_f8f6f4 a[124:127], v[136:143], v[120:127], a[124:127] cbsz:1 blgp:1\n" \
  "buffer_load_dwordx4 v[80:83], v166, s[36:39], s75 offen offset:0\n" \
  "s_waitcnt lgkmcnt(2)\n" \
  "v_mfma_f32_16x16x128_f8f6f4 a[128:131], v[144:151], v[96:103], a[128:131] cbsz:1 blgp:1\n" \
  "ds_read_b128 v[128:131], v161 offset:53248\n" \
  "ds_read_b128 v[132:135], v161 offset:54272\n" \
  "v_mfma_f32_16x16x128_f8f6f4 a[132:135], v[144:151], v[104:111], a[132:135] cbsz:1 blgp:1\n" \
  "v_mfma_f32_16x16x128_f8f6f4 a[136:139], v[144:151], v[112:119], a[136:139] cbsz:1 blgp:1\n" \
  "v_mul_f32 v212, s78, v212\n" \
  "v_mul_f32 v213, s78, v213\n" \
  "v_mul_f32 v214, s78, v214\n" \
  "v_mul_f32 v215, s78, v215\n" \
  "v_mfma_f32_16x16x128_f8f6f4 a[140:143], v[144:151], v[120:127], a[140:143] cbsz:1 blgp:1\n" \
  "s_waitcnt lgkmcnt(2)\n" \
  "v_mfma_f32_16x16x128_f8f6f4 a[144:147], v[152:159], v[96:103], a[144:147] cbsz:1 blgp:1\n" \
  "ds_read_b128 v[136:139], v161 offset:55296\n" \
  "ds_read_b128 v[140:143], v161 offset:56320\n" \
  "v_mfma_f32_16x16x128_f8f6f4 a[148:151], v[152:159], v[104:111], a[148:151] cbsz:1 blgp:1\n" \
  "buffer_store_dwordx4 v[212:215], v167, s[40:43], s51 offen offset:64 nt\n" \
  "v_mfma_f32_16x16x128_f8f6f4 a[152:155], v[152:159], v[112:119], a[152:155] cbsz:1 blgp:1\n" \
  "buffer_load_dwordx4 v[84:87], v166, s[36:39], s75 offen offset:1024\n" \
  "v_mfma_f32_16x16x128_f8f6f4 a[156:159], v[152:159], v[120:127], a[156:159] cbsz:1 blgp:1\n" \
  "s_waitcnt lgkmcnt(2)\n" \
  "v_mfma_f32_16x16x128_f8f6f4 a[160:163], v[128:135], v[96:103], a[160:163] cbsz:1 blgp:1\n" \
  "ds_read_b128 v[144:147], v161 offset:57344\n" \
  "ds_read_b128 v[148:151], v161 offset:58368\n" \
  "v_mfma_f32_16x16x128_f8f6f4 a[164:167], v[128:135], v[104:111], a[164:167] cbsz:1 blgp:1\n" \
  "buffer_load_dwordx4 v[88:91], v166, s[36:39], s75 offen offset:2048\n" \
  "v_mfma_f32_16x16x128_f8f6f4 a[168:171], v[128:135], v[112:119], a[168:171] cbsz:1 blgp:1\n" \
  "v_mfma_f32_16x16x128_f8f6f4 a[172:175], v[128:135], v[120:127], a[172:175] cbsz:1 blgp:1\n" \
  "s_waitcnt lgkmcnt(2)\n" \
  "v_mfma_f32_16x16x128_f8f6f4 a[176:179], v[136:143], v[96:103], a[176:179] cbsz:1 blgp:1\n" \
  "ds_read_b128 v[152:155], v161 offset:59392\n" \
  "ds_read_b128 v[156:159], v161 offset:60416\n" \
  "v_mfma_f32_16x16x128_f8f6f4 a[180:183], v[136:143], v[104:111], a[180:183] cbsz:1 blgp:1\n" \
  "buffer_load_dwordx4 v[92:95], v166, s[36:39], s75 offen offset:3072\n" \
  "v_mfma_f32_16x16x128_f8f6f4 a[184:187], v[136:143], v[112:119], a[184:187] cbsz:1 blgp:1\n" \
  "v_mfma_f32_16x16x128_f8f6f4 a[188:191], v[136:143], v[120:127], a[188:191] cbsz:1 blgp:1\n" \
  "s_waitcnt vmcnt(30) lgkmcnt(0)\n" \
  "s_barrier\n" \
  "v_mfma_f32_16x16x128_f8f6f4 a[192:195], v[144:151], v[96:103], a[192:195] cbsz:1 blgp:1\n" \
  "ds_read_b128 v[128:131], v160 offset:8192\n" \
  "ds_read_b128 v[132:135], v160 offset:9216\n" \
  "v_mfma_f32_16x16x128_f8f6f4 a[196:199], v[144:151], v[104:111], a[196:199] cbsz:1 blgp:1\n" \
  "s_mov_b32 s42, s72\n" \
  "s_mov_b32 s48, s71\n" \
  "s_add_u32 s49, s71, 1048576\n" \
  "s_add_u32 s50, s71, 2097152\n" \
  "v_mfma_f32_16x16x128_f8f6f4 a[200:203], v[144:151], v[112:119], a[200:203] cbsz:1 blgp:1\n" \
  "s_add_u32 s51, s71, 3145728\n" \
  "s_cmp_eq_u32 s70, 64\n" \
  "s_cbranch_scc1 Lp48_%=\n" \
  "Lq48_%=:\n" \
  "v_mfma_f32_16x16x128_f8f6f4 a[204:207], v[144:151], v[120:127], a[204:207] cbsz:1 blgp:1\n" \
  "v_accvgpr_read_b32 v184, a32\n" \
  "v_accvgpr_read_b32 v185, a33\n" \
  "v_accvgpr_read_b32 v186, a34\n" \
  "v_accvgpr_read_b32 v187, a35\n" \
  "v_mfma_f32_16x16x128_f8f6f4 a[208:211], v[152:159], v[96:103], a[208:211] cbsz:1 blgp:1\n" \
  "ds_read_b128 v[136:139], v160 offset:10240\n" \
  "ds_read_b128 v[140:143], v160 offset:11264\n" \
  "v_mfma_f32_16x16x128_f8f6f4 a[212:215], v[152:159], v[104:111], a[212:215] cbsz:1 blgp:1\n" \
  "v_accvgpr_read_b32 v200, a48\n" \
  "v_accvgpr_read_b32 v201, a49\n" \
  "v_accvgpr_read_b32 v202, a50\n" \
  "v_accvgpr_read_b32 v203, a51\n" \
  "v_mfma_f32_16x16x128_f8f6f4 a[216:219], v[152:159], v[112:119], a[216:219] cbsz:1 blgp:1\n" \
  "v_accvgpr_read_b32 v188, a36\n" \
  "v_accvgpr_read_b32 v189, a37\n" \
  "v_accvgpr_read_b32 v190, a38\n" \
  "v_accvgpr_read_b32 v191, a39\n" \
  "v_mfma_f32_16x16x128_f8f6f4 a[220:223], v[152:159], v[120:127], a[220:223] cbsz:1 blgp:1\n" \
  "v_accvgpr_read_b32 v204, a52\n" \
  "v_accvgpr_read_b32 v205, a53\n" \
  "v_accvgpr_read_b32 v206, a54\n" \
  "v_accvgpr_read_b32 v207, a55\n" \
  "s_waitcnt lgkmcnt(2)\n" \
  "v_mfma_f32_16x16x128_f8f6f4 a[64:67], v[128:135], v[0:7], a[64:67] cbsz:1 blgp:1\n" \
  "ds_read_b128 v[144:147], v160 offset:12288\n" \
  "ds_read_b128 v[148:151], v160 offset:13312\n" \
  "v_mfma_f32_16x16x128_f8f6f4 a[68:71], v[128:135], v[8:15], a[68:71] cbsz:1 blgp:1\n" \
  "v_accvgpr_read_b32 v192, a40\n" \
  "v_accvgpr_read_b32 v193, a41\n" \
  "v_accvgpr_read_b32 v194, a42\n" \
  "v_accvgpr_read_b32 v195, a43\n" \
  "v_mfma_f32_16x16x128_f8f6f4 a[72:75], v[128:135], v[16:23], a[72:75] cbsz:1 blgp:1\n" \
  "v_accvgpr_read_b32 v208, a56\n" \
  "v_accvgpr_read_b32 v209, a57\n" \
  "v_accvgpr_read_b32 v210, a58\n" \
  "v_accvgpr_read_b32 v211, a59\n" \
  "v_mfma_f32_16x16x128_f8f6f4 a[76:79], v[128:135], v[24:31], a[76:79] cbsz:1 blgp:1\n" \
  "v_accvgpr_read_b32 v196, a44\n" \
  "v_accvgpr_read_b32 v197, a45\n" \
  "v_accvgpr_read_b32 v198, a46\n" \
  "v_accvgpr_read_b32 v199, a47\n" \
  "s_waitcnt lgkmcnt(2)\n" \
  "v_mfma_f32_16x16x128_f8f6f4 a[80:83], v[136:143], v[0:7], a[80:83] cbsz:1 blgp:1\n" \
  "ds_read_b128 v[152:155], v160 offset:14336\n" \
  "ds_read_b128 v[156:159], v160 offset:15360\n" \
  "v_mfma_f32_16x16x128_f8f6f4 a[84:87], v[136:143], v[8:15], a[84:87] cbsz:1 blgp:1\n" \
  "v_accvgpr_read_b32 v212, a60\n" \
  "v_accvgpr_read_b32 v213, a61\n" \
  "v_accvgpr_read_b32 v214, a62\n" \
  "v_accvgpr_read_b32 v215, a63\n" \
  "s_add_u32 m0, s69, 106496\n" \
  "s_nop 0\n" \
  "buffer_load_dwordx4 v163, s[36:39], s58 offen lds\n" \
  "v_mfma_f32_16x16x128_f8f6f4 a[88:91], v[136:143], v[16:23], a[88:91] cbsz:1 blgp:1\n" \
  "s_add_u32 m0, s69, 110592\n" \
  "s_nop 0\n" \
  "buffer_load_dwordx4 v163, s[36:39], s59 offen lds\n" \
  "v_readlane_b32 s67, v175, s70\n" \
  "v_readlane_b32 s76, v169, s70\n" \
  "v_mfma_f32_16x16x128_f8f6f4 a[92:95], v[136:143], v[24:31], a[92:95] cbsz:1 blgp:1\n" \
  "s_add_u32 m0, s69, 114688\n" \
  "s_nop 0\n" \
  "buffer_load_dwordx4 v163, s[36:39], s60 offen lds\n" \
  "s_waitcnt lgkmcnt(2)\n" \
  "v_mfma_f32_16x16x128_f8f6f4 a[96:99], v[144:151], v[0:7], a[96:99] cbsz:1 blgp:1\n" \
  "ds_read_b128 v[128:131], v160 offset:16384\n" \
  "ds_read_b128 v[132:135], v160 offset:17408\n" \
  "v_mfma_f32_16x16x128_f8f6f4 a[100:103], v[144:151], v[8:15], a[100:103] cbsz:1 blgp:1\n" \
  "s_add_u32 m0, s69, 118784\n" \
  "s_nop 0\n" \
  "buffer_load_dwordx4 v163, s[36:39], s61 offen lds\n" \
  "s_add_u32 s70, s70, 1\n" \
  "s_nop 0\n" \
  "v_add_u32 v165, s76, v162\n" \
  "v_mfma_f32_16x16x128_f8f6f4 a[104:107], v[144:151], v[16:23], a[104:107] cbsz:1 blgp:1\n" \
  "s_add_u32 m0, s69, 122880\n" \
  "s_nop 0\n" \
  "buffer_load_dwordx4 v163, s[36:39], s62 offen lds\n" \
  "v_add_u32 v166, 4096, v165\n" \
  "s_cmp_gt_u32 s67, s68\n" \
  "s_cbranch_scc1 Lp49_%=\n" \
  "Lq49_%=:\n" \
  "v_mfma_f32_16x16x128_f8f6f4 a[108:111], v[144:151], v[24:31], a[108:111] cbsz:1 blgp:1\n" \
  "buffer_load_dwordx4 v[96:99], v163, s[36:39], s75 offen offset:0\n" \
  "s_waitcnt lgkmcnt(2)\n" \
  "v_mfma_f32_16x16x128_f8f6f4 a[112:115], v[152:159], v[0:7], a[112:115] cbsz:1 blgp:1\n" \
  "ds_read_b128 v[136:139], v160 offset:18432\n" \
  "ds_read_b128 v[140:143], v160 offset:19456\n" \
  "v_mfma_f32_16x16x128_f8f6f4 a[116:119], v[152:159], v[8:15], a[116:119] cbsz:1 blgp:1\n" \
  "buffer_load_dwordx4 v[100:103], v163, s[36:39], s75 offen offset:1024\n" \
  "v_mfma_f32_16x16x128_f8f6f4 a[120:123], v[152:159], v[16:23], a[120:123] cbsz:1 blgp:1\n" \
  "v_mfma_f32_16x16x128_f8f6f4 a[124:127], v[152:159], v[24:31], a[124:127] cbsz:1 blgp:1\n" \
  "buffer_load_dwordx4 v[104:107], v163, s[36:39], s75 offen offset:2048\n" \
  "s_waitcnt lgkmcnt(2)\n" \
  "v_mfma_f32_16x16x128_f8f6f4 a[128:131], v[128:135], v[0:7], a[128:131] cbsz:1 blgp:1\n" \
  "ds_read_b128 v[144:147], v160 offset:20480\n" \
  "ds_read_b128 v[148:151], v160 offset:21504\n" \
  "v_mfma_f32_16x16x128_f8f6f4 a[132:135], v[128:135], v[8:15], a[132:135] cbsz:1 blgp:1\n" \
  "buffer_load_dwordx4 v[108:111], v163, s[36:39], s75 offen offset:3072\n" \
  "v_mfma_f32_16x16x128_f8f6f4 a[136:139], v[128:135], v[16:23], a[136:139] cbsz:1 blgp:1\n" \
  "buffer_load_dwordx4 v[112:115], v164, s[36:39], s75 offen offset:0\n" \
  "v_mfma_f32_16x16x128_f8f6f4 a[140:143], v[128:135], v[24:31], a[140:143] cbsz:1 blgp:1\n" \
  "s_waitcnt lgkmcnt(2)\n" \
  "v_mfma_f32_16x16x128_f8f6f4 a[144:147], v[136:143], v[0:7], a[144:147] cbsz:1 blgp:1\n" \
  "ds_read_b128 v[152:155], v160 offset:22528\n" \
  "ds_read_b128 v[156:159], v160 offset:23552\n" \
  "v_mfma_f32_16x16x128_f8f6f4 a[148:151], v[136:143], v[8:15], a[148:151] cbsz:1 blgp:1\n" \
  "buffer_load_dwordx4 v[116:119], v164, s[36:39], s75 offen offset:1024\n" \
  "v_mfma_f32_16x16x128_f8f6f4 a[152:155], v[136:143], v[16:23], a[152:155] cbsz:1 blgp:1\n" \
  "v_mul_f32 v184, s78, v184\n" \
  "v_mul_f32 v185, s78, v185\n" \
  "v_mul_f32 v186, s78, v186\n" \
  "v_mul_f32 v187, s78, v187\n" \
  "v_mfma_f32_16x16x128_f8f6f4 a[156:159], v[136:143], v[24:31], a[156:159] cbsz:1 blgp:1\n" \
  "buffer_store_dwordx4 v[184:187], v167, s[40:43], s48 offen offset:0 nt\n" \
  "s_waitcnt lgkmcnt(2)\n" \
  "v_mfma_f32_16x16x128_f8f6f4 a[160:163], v[144:151], v[0:7], a[160:163] cbsz:1 blgp:1\n" \
  "ds_read_b128 v[128:131], v160 offset:24576\n" \
  "ds_read_b128 v[132:135], v160 offset:25600\n" \
  "v_mfma_f32_16x16x128_f8f6f4 a[164:167], v[144:151], v[8:15], a[164:167] cbsz:1 blgp:1\n" \
  "buffer_load_dwordx4 v[120:123], v164, s[36:39], s75 offen offset:2048\n" \
  "v_mfma_f32_16x16x128_f8f6f4 a[168:171], v[144:151], v[16:23], a[168:171] cbsz:1 blgp:1\n" \
  "v_mul_f32 v200, s78, v200\n" \
  "v_mul_f32 v201, s78, v201\n" \
  "v_mul_f32 v202, s78, v202\n" \
  "v_mul_f32 v203, s78, v203\n" \
  "v_mfma_f32_16x16x128_f8f6f4 a[172:175], v[144:151], v[24:31], a[172:175] cbsz:1 blgp:1\n" \
  "buffer_store_dwordx4 v[200:203], v167, s[40:43], s48 offen offset:64 nt\n" \
  "s_waitcnt lgkmcnt(2)\n" \
  "v_mfma_f32_16x16x128_f8f6f4 a[176:179], v[152:159], v[0:7], a[176:179] cbsz:1 blgp:1\n" \
  "ds_read_b128 v[136:139], v160 offset:26624\n" \
  "ds_read_b128 v[140:143], v160 offset:27648\n" \
  "v_mfma_f32_16x16x128_f8f6f4 a[180:183], v[152:159], v[8:15], a[180:183] cbsz:1 blgp:1\n" \
  "buffer_load_dwordx4 v[124:127], v164, s[36:39], s75 offen offset:3072\n" \
  "v_mfma_f32_16x16x128_f8f6f4 a[184:187], v[152:159], v[16:23], a[184:187] cbsz:1 blgp:1\n" \
  "v_mfma_f32_16x16x128_f8f6f4 a[188:191], v[152:159], v[24:31], a[188:191] cbsz:1 blgp:1\n" \
  "s_waitcnt vmcnt(30) lgkmcnt(0)\n" \
  "s_barrier\n" \
  "v_mfma_f32_16x16x128_f8f6f4 a[192:195], v[128:135], v[0:7], a[192:195] cbsz:1 blgp:1\n" \
  "ds_read_b128 v[144:147], v160 offset:40960\n" \
  "ds_read_b128 v[148:151], v160 offset:41984\n" \
  "v_mfma_f32_16x16x128_f8f6f4 a[196:199], v[128:135], v[8:15], a[196:199] cbsz:1 blgp:1\n" \
  "s_add_u32 m0, s69, 12288\n" \
  "s_nop 0\n" \
  "buffer_load_dwordx4 v165, s[36:39], s59 offen lds\n" \
  "v_mfma_f32_16x16x128_f8f6f4 a[200:203], v[128:135], v[16:23], a[200:203] cbsz:1 blgp:1\n" \
  "s_add_u32 m0, s69, 16384\n" \
  "s_nop 0\n" \
  "buffer_load_dwordx4 v165, s[36:39], s60 offen lds\n" \
  "v_mfma_f32_16x16x128_f8f6f4 a[204:207], v[128:135], v[24:31], a[204:207] cbsz:1 blgp:1\n" \
  "v_readlane_b32 s67, v175, s70\n" \
  "v_readlane_b32 s76, v169, s70\n" \
  "v_mfma_f32_16x16x128_f8f6f4 a[208:211], v[136:143], v[0:7], a[208:211] cbsz:1 blgp:1\n" \
  "ds_read_b128 v[152:155], v160 offset:43008\n" \
  "ds_read_b128 v[156:159], v160 offset:44032\n" \
  "v_mfma_f32_16x16x128_f8f6f4 a[212:215], v[136:143], v[8:15], a[212:215] cbsz:1 blgp:1\n" \
  "s_add_u32 m0, s69, 20480\n" \
  "s_nop 0\n" \
  "buffer_load_dwordx4 v165, s[36:39], s61 offen lds\n" \
  "v_mfma_f32_16x16x128_f8f6f4 a[216:219], v[136:143], v[16:23], a[216:219] cbsz:1 blgp:1\n" \
  "v_mfma_f32_16x16x128_f8f6f4 a[220:223], v[136:143], v[24:31], a[220:223] cbsz:1 blgp:1\n" \
  "s_add_u32 m0, s69, 24576\n" \
  "s_nop 0\n" \
  "buffer_load_dwordx4 v165, s[36:39], s62 offen lds\n" \
  "s_waitcnt lgkmcnt(2)\n" \
  "v_mfma_f32_16x16x128_f8f6f4 a[64:67], v[144:151], v[32:39], a[64:67] cbsz:1 blgp:1\n" \
  "ds_read_b128 v[128:131], v160 offset:45056\n" \
  "ds_read_b128 v[132:135], v160 offset:46080\n" \
  "v_mfma_f32_16x16x128_f8f6f4 a[68:71], v[144:151], v[40:47], a[68:71] cbsz:1 blgp:1\n" \
  "s_add_u32 s70, s70, 1\n" \
  "s_nop 0\n" \
  "v_add_u32 v163, s76, v162\n" \
  "v_mfma_f32_16x16x128_f8f6f4 a[72:75], v[144:151], v[48:55], a[72:75] cbsz:1 blgp:1\n" \
  "v_mfma_f32_16x16x128_f8f6f4 a[76:79], v[144:151], v[56:63], a[76:79] cbsz:1 blgp:1\n" \
  "v_add_u32 v164, 4096, v163\n" \
  "s_cmp_gt_u32 s67, s68\n" \
  "s_cbranch_scc1 Lp50_%=\n" \
  "Lq50_%=:\n" \
  "s_waitcnt lgkmcnt(2)\n" \
  "v_mfma_f32_16x16x128_f8f6f4 a[80:83], v[152:159], v[32:39], a[80:83] cbsz:1 blgp:1\n" \
  "ds_read_b128 v[136:139], v160 offset:47104\n" \
  "ds_read_b128 v[140:143], v160 offset:48128\n" \
  "v_mfma_f32_16x16x128_f8f6f4 a[84:87], v[152:159], v[40:47], a[84:87] cbsz:1 blgp:1\n" \
  "buffer_load_dwordx4 v[0:3], v165, s[36:39], s75 offen offset:0\n" \
  "v_mfma_f32_16x16x128_f8f6f4 a[88:91], v[152:159], v[48:55], a[88:91] cbsz:1 blgp:1\n" \
  "buffer_load_dwordx4 v[4:7], v165, s[36:39], s75 offen offset:1024\n" \
  "v_mfma_f32_16x16x128_f8f6f4 a[92:95], v[152:159], v[56:63], a[92:95] cbsz:1 blgp:1\n" \
  "s_waitcnt lgkmcnt(2)\n" \
  "v_mfma_f32_16x16x128_f8f6f4 a[96:99], v[128:135], v[32:39], a[96:99] cbsz:1 blgp:1\n" \
  "ds_read_b128 v[144:147], v160 offset:49152\n" \
  "ds_read_b128 v[148:151], v160 offset:50176\n" \
  "v_mfma_f32_16x16x128_f8f6f4 a[100:103], v[128:135], v[40:47], a[100:103] cbsz:1 blgp:1\n" \
  "v_mul_f32 v188, s78, v188\n" \
  "v_mul_f32 v189, s78, v189\n" \
  "v_mul_f32 v190, s78, v190\n" \
  "v_mul_f32 v191, s78, v191\n" \
  "v_mfma_f32_16x16x128_f8f6f4 a[104:107], v[128:135], v[48:55], a[104:107] cbsz:1 blgp:1\n" \
  "buffer_store_dwordx4 v[188:191], v167, s[40:43], s49 offen offset:0 nt\n" \
  "v_mfma_f32_16x16x128_f8f6f4 a[108:111], v[128:135], v[56:63], a[108:111] cbsz:1 blgp:1\n" \
  "s_waitcnt lgkmcnt(2)\n" \
  "v_mfma_f32_16x16x128_f8f6f4 a[112:115], v[136:143], v[32:39], a[112:115] cbsz:1 blgp:1\n" \
  "ds_read_b128 v[152:155], v160 offset:51200\n" \
  "ds_read_b128 v[156:159], v160 offset:52224\n" \
  "v_mfma_f32_16x16x128_f8f6f4 a[116:119], v[136:143], v[40:47], a[116:119] cbsz:1 blgp:1\n" \
  "buffer_load_dwordx4 v[8:11], v165, s[36:39], s75 offen offset:2048\n" \
  "v_mfma_f32_16x16x128_f8f6f4 a[120:123], v[136:143], v[48:55], a[120:123] cbsz:1 blgp:1\n" \
  "v_mfma_f32_16x16x128_f8f6f4 a[124:127], v[136:143], v[56:63], a[124:127] cbsz:1 blgp:1\n" \
  "buffer_load_dwordx4 v[12:15], v165, s[36:39], s75 offen offset:3072\n" \
  "s_waitcnt lgkmcnt(2)\n" \
  "v_mfma_f32_16x16x128_f8f6f4 a[128:131], v[144:151], v[32:39], a[128:131] cbsz:1 blgp:1\n" \
  "ds_read_b128 v[128:131], v160 offset:53248\n" \
  "ds_read_b128 v[132:135], v160 offset:54272\n" \
  "v_mfma_f32_16x16x128_f8f6f4 a[132:135], v[144:151], v[40:47], a[132:135] cbsz:1 blgp:1\n" \
  "buffer_load_dwordx4 v[16:19], v166, s[36:39], s75 offen offset:0\n" \
  "v_mfma_f32_16x16x128_f8f6f4 a[136:139], v[144:151], v[48:55], a[136:139] cbsz:1 blgp:1\n" \
  "v_mfma_f32_16x16x128_f8f6f4 a[140:143], v[144:151], v[56:63], a[140:143] cbsz:1 blgp:1\n" \
  "s_waitcnt lgkmcnt(2)\n" \
  "v_mfma_f32_16x16x128_f8f6f4 a[144:147], v[152:159], v[32:39], a[144:147] cbsz:1 blgp:1\n" \
  "ds_read_b128 v[136:139], v160 offset:55296\n" \
  "ds_read_b128 v[140:143], v160 offset:56320\n" \
  "v_mfma_f32_16x16x128_f8f6f4 a[148:151], v[152:159], v[40:47], a[148:151] cbsz:1 blgp:1\n" \
  "v_mul_f32 v204, s78, v204\n" \
  "v_mul_f32 v205, s78, v205\n" \
  "v_mul_f32 v206, s78, v206\n" \
  "v_mul_f32 v207, s78, v207\n" \
  "buffer_store_dwordx4 v[204:207], v167, s[40:43], s49 offen offset:64 nt\n" \
  "v_mfma_f32_16x16x128_f8f6f4 a[152:155], v[152:159], v[48:55], a[152:155] cbsz:1 blgp:1\n" \
  "v_mfma_f32_16x16x128_f8f6f4 a[156:159], v[152:159], v[56:63], a[156:159] cbsz:1 blgp:1\n" \
  "buffer_load_dwordx4 v[20:23], v166, s[36:39], s75 offen offset:1024\n" \
  "s_waitcnt lgkmcnt(2)\n" \
  "v_mfma_f32_16x16x128_f8f6f4 a[160:163], v[128:135], v[32:39], a[160:163] cbsz:1 blgp:1\n" \
  "ds_read_b128 v[144:147], v160 offset:57344\n" \
  "ds_read_b128 v[148:151], v160 offset:58368\n" \
  "v_mfma_f32_16x16x128_f8f6f4 a[164:167], v[128:135], v[40:47], a[164:167] cbsz:1 blgp:1\n" \
  "v_mfma_f32_16x16x128_f8f6f4 a[168:171], v[128:135], v[48:55], a[168:171] cbsz:1 blgp:1\n" \
  "buffer_load_dwordx4 v[24:27], v166, s[36:39], s75 offen offset:2048\n" \
  "v_mfma_f32_16x16x128_f8f6f4 a[172:175], v[128:135], v[56:63], a[172:175] cbsz:1 blgp:1\n" \
  "s_waitcnt lgkmcnt(2)\n" \
  "v_mfma_f32_16x16x128_f8f6f4 a[176:179], v[136:143], v[32:39], a[176:179] cbsz:1 blgp:1\n" \
  "ds_read_b128 v[152:155], v160 offset:59392\n" \
  "ds_read_b128 v[156:159], v160 offset:60416\n" \
  "v_mfma_f32_16x16x128_f8f6f4 a[180:183], v[136:143], v[40:47], a[180:183] cbsz:1 blgp:1\n" \
  "buffer_load_dwordx4 v[28:31], v166, s[36:39], s75 offen offset:3072\n" \
  "v_mfma_f32_16x16x128_f8f6f4 a[184:187], v[136:143], v[48:55], a[184:187] cbsz:1 blgp:1\n" \
  "v_mfma_f32_16x16x128_f8f6f4 a[188:191], v[136:143], v[56:63], a[188:191] cbsz:1 blgp:1\n" \
  "s_waitcnt vmcnt(29) lgkmcnt(0)\n" \
  "s_barrier\n" \
  "v_mfma_f32_16x16x128_f8f6f4 a[192:195], v[144:151], v[32:39], a[192:195] cbsz:1 blgp:1\n" \
  "ds_read_b128 v[128:131], v161 offset:8192\n" \
  "ds_read_b128 v[132:135], v161 offset:9216\n" \
  "v_mfma_f32_16x16x128_f8f6f4 a[196:199], v[144:151], v[40:47], a[196:199] cbsz:1 blgp:1\n" \
  "s_add_u32 m0, s69, 45056\n" \
  "s_nop 0\n" \
  "buffer_load_dwordx4 v163, s[36:39], s59 offen lds\n" \
  "v_mfma_f32_16x16x128_f8f6f4 a[200:203], v[144:151], v[48:55], a[200:203] cbsz:1 blgp:1\n" \
  "s_add_u32 m0, s69, 49152\n" \
  "s_nop 0\n" \
  "buffer_load_dwordx4 v163, s[36:39], s60 offen lds\n" \
  "v_mfma_f32_16x16x128_f8f6f4 a[204:207], v[144:151], v[56:63], a[204:207] cbsz:1 blgp:1\n" \
  "v_readlane_b32 s67, v175, s70\n" \
  "v_readlane_b32 s76, v169, s70\n" \
  "v_mfma_f32_16x16x128_f8f6f4 a[208:211], v[152:159], v[32:39], a[208:211] cbsz:1 blgp:1\n" \
  "ds_read_b128 v[136:139], v161 offset:10240\n" \
  "ds_read_b128 v[140:143], v161 offset:11264\n" \
  "v_mfma_f32_16x16x128_f8f6f4 a[212:215], v[152:159], v[40:47], a[212:215] cbsz:1 blgp:1\n" \
  "s_add_u32 m0, s69, 53248\n" \
  "s_nop 0\n" \
  "buffer_load_dwordx4 v163, s[36:39], s61 offen lds\n" \
  "v_mfma_f32_16x16x128_f8f6f4 a[216:219], v[152:159], v[48:55], a[216:219] cbsz:1 blgp:1\n" \
  "v_mfma_f32_16x16x128_f8f6f4 a[220:223], v[152:159], v[56:63], a[220:223] cbsz:1 blgp:1\n" \
  "s_add_u32 m0, s69, 57344\n" \
  "s_nop 0\n" \
  "buffer_load_dwordx4 v163, s[36:39], s62 offen lds\n" \
  "s_waitcnt lgkmcnt(2)\n" \
  "v_mfma_f32_16x16x128_f8f6f4 a[64:67], v[128:135], v[64:71], a[64:67] cbsz:1 blgp:1\n" \
  "ds_read_b128 v[144:147], v161 offset:12288\n" \
  "ds_read_b128 v[148:151], v161 offset:13312\n" \
  "v_mfma_f32_16x16x128_f8f6f4 a[68:71], v[128:135], v[72:79], a[68:71] cbsz:1 blgp:1\n" \
  "s_add_u32 s70, s70, 1\n" \
  "s_nop 0\n" \
  "v_add_u32 v165, s76, v162\n" \
  "v_mfma_f32_16x16x128_f8f6f4 a[72:75], v[128:135], v[80:87], a[72:75] cbsz:1 blgp:1\n" \
  "v_mfma_f32_16x16x128_f8f6f4 a[76:79], v[128:135], v[88:95], a[76:79] cbsz:1 blgp:1\n" \
  "v_add_u32 v166, 4096, v165\n" \
  "s_cmp_gt_u32 s67, s68\n" \
  "s_cbranch_scc1 Lp51_%=\n" \
  "Lq51_%=:\n" \
  "s_waitcnt lgkmcnt(2)\n" \
  "v_mfma_f32_16x16x128_f8f6f4 a[80:83], v[136:143], v[64:71], a[80:83] cbsz:1 blgp:1\n" \
  "ds_read_b128 v[152:155], v161 offset:14336\n" \
  "ds_read_b128 v[156:159], v161 offset:15360\n" \
  "v_mfma_f32_16x16x128_f8f6f4 a[84:87], v[136:143], v[72:79], a[84:87] cbsz:1 blgp:1\n" \
  "buffer_load_dwordx4 v[32:35], v163, s[36:39], s75 offen offset:0\n" \
  "v_mfma_f32_16x16x128_f8f6f4 a[88:91], v[136:143], v[80:87], a[88:91] cbsz:1 blgp:1\n" \
  "buffer_load_dwordx4 v[36:39], v163, s[36:39], s75 offen offset:1024\n" \
  "v_mfma_f32_16x16x128_f8f6f4 a[92:95], v[136:143], v[88:95], a[92:95] cbsz:1 blgp:1\n" \
  "s_waitcnt lgkmcnt(2)\n" \
  "v_mfma_f32_16x16x128_f8f6f4 a[96:99], v[144:151], v[64:71], a[96:99] cbsz:1 blgp:1\n" \
  "ds_read_b128 v[128:131], v161 offset:16384\n" \
  "ds_read_b128 v[132:135], v161 offset:17408\n" \
  "v_mfma_f32_16x16x128_f8f6f4 a[100:103], v[144:151], v[72:79], a[100:103] cbsz:1 blgp:1\n" \
  "v_mul_f32 v192, s78, v192\n" \
  "v_mul_f32 v193, s78, v193\n" \
  "v_mul_f32 v194, s78, v194\n" \
  "v_mul_f32 v195, s78, v195\n" \
  "v_mfma_f32_16x16x128_f8f6f4 a[104:107], v[144:151], v[80:87], a[104:107] cbsz:1 blgp:1\n" \
  "buffer_store_dwordx4 v[192:195], v167, s[40:43], s50 offen offset:0 nt\n" \
  "v_mfma_f32_16x16x128_f8f6f4 a[108:111], v[144:151], v[88:95], a[108:111] cbsz:1 blgp:1\n" \
  "s_waitcnt lgkmcnt(2)\n" \
  "v_mfma_f32_16x16x128_f8f6f4 a[112:115], v[152:159], v[64:71], a[112:115] cbsz:1 blgp:1\n" \
  "ds_read_b128 v[136:139], v161 offset:18432\n" \
  "ds_read_b128 v[140:143], v161 offset:19456\n" \
  "v_mfma_f32_16x16x128_f8f6f4 a[116:119], v[152:159], v[72:79], a[116:119] cbsz:1 blgp:1\n" \
  "buffer_load_dwordx4 v[40:43], v163, s[36:39], s75 offen offset:2048\n" \
  "v_mfma_f32_16x16x128_f8f6f4 a[120:123], v[152:159], v[80:87], a[120:123] cbsz:1 blgp:1\n" \
  "v_mfma_f32_16x16x128_f8f6f4 a[124:127], v[152:159], v[88:95], a[124:127] cbsz:1 blgp:1\n" \
  "buffer_load_dwordx4 v[44:47], v163, s[36:39], s75 offen offset:3072\n" \
  "s_waitcnt lgkmcnt(2)\n" \
  "v_mfma_f32_16x16x128_f8f6f4 a[128:131], v[128:135], v[64:71], a[128:131] cbsz:1 blgp:1\n" \
  "ds_read_b128 v[144:147], v161 offset:20480\n" \
  "ds_read_b128 v[148:151], v161 offset:21504\n" \
  "v_mfma_f32_16x16x128_f8f6f4 a[132:135], v[128:135], v[72:79], a[132:135] cbsz:1 blgp:1\n" \
  "buffer_load_dwordx4 v[48:51], v164, s[36:39], s75 offen offset:0\n" \
  "v_mfma_f32_16x16x128_f8f6f4 a[136:139], v[128:135], v[80:87], a[136:139] cbsz:1 blgp:1\n" \
  "v_mfma_f32_16x16x128_f8f6f4 a[140:143], v[128:135], v[88:95], a[140:143] cbsz:1 blgp:1\n" \
  "s_waitcnt lgkmcnt(2)\n" \
  "v_mfma_f32_16x16x128_f8f6f4 a[144:147], v[136:143], v[64:71], a[144:147] cbsz:1 blgp:1\n" \
  "ds_read_b128 v[152:155], v161 offset:22528\n" \
  "ds_read_b128 v[156:159], v161 offset:23552\n" \
  "v_mfma_f32_16x16x128_f8f6f4 a[148:151], v[136:143], v[72:79], a[148:151] cbsz:1 blgp:1\n" \
  "v_mul_f32 v208, s78, v208\n" \
  "v_mul_f32 v209, s78, v209\n" \
  "v_mul_f32 v210, s78, v210\n" \
  "v_mul_f32 v211, s78, v211\n" \
  "buffer_store_dwordx4 v[208:211], v167, s[40:43], s50 offen offset:64 nt\n" \
  "v_mfma_f32_16x16x128_f8f6f4 a[152:155], v[136:143], v[80:87], a[152:155] cbsz:1 blgp:1\n" \
  "v_mfma_f32_16x16x128_f8f6f4 a[156:159], v[136:143], v[88:95], a[156:159] cbsz:1 blgp:1\n" \
  "buffer_load_dwordx4 v[52:55], v164, s[36:39], s75 offen offset:1024\n" \
  "s_waitcnt lgkmcnt(2)\n" \
  "v_mfma_f32_16x16x128_f8f6f4 a[160:163], v[144:151], v[64:71], a[160:163] cbsz:1 blgp:1\n" \
  "ds_read_b128 v[128:131], v161 offset:24576\n" \
  "ds_read_b128 v[132:135], v161 offset:25600\n" \
  "v_mfma_f32_16x16x128_f8f6f4 a[164:167], v[144:151], v[72:79], a[164:167] cbsz:1 blgp:1\n" \
  "v_mfma_f32_16x16x128_f8f6f4 a[168:171], v[144:151], v[80:87], a[168:171] cbsz:1 blgp:1\n" \
  "buffer_load_dwordx4 v[56:59], v164, s[36:39], s75 offen offset:2048\n" \
  "v_mfma_f32_16x16x128_f8f6f4 a[172:175], v[144:151], v[88:95], a[172:175] cbsz:1 blgp:1\n" \
  "s_waitcnt lgkmcnt(2)\n" \
  "v_mfma_f32_16x16x128_f8f6f4 a[176:179], v[152:159], v[64:71], a[176:179] cbsz:1 blgp:1\n" \
  "ds_read_b128 v[136:139], v161 offset:26624\n" \
  "ds_read_b128 v[140:143], v161 offset:27648\n" \
  "v_mfma_f32_16x16x128_f8f6f4 a[180:183], v[152:159], v[72:79], a[180:183] cbsz:1 blgp:1\n" \
  "buffer_load_dwordx4 v[60:63], v164, s[36:39], s75 offen offset:3072\n" \
  "v_mfma_f32_16x16x128_f8f6f4 a[184:187], v[152:159], v[80:87], a[184:187] cbsz:1 blgp:1\n" \
  "v_mfma_f32_16x16x128_f8f6f4 a[188:191], v[152:159], v[88:95], a[188:191] cbsz:1 blgp:1\n" \
  "s_waitcnt vmcnt(28) lgkmcnt(0)\n" \
  "s_barrier\n" \
  "v_mfma_f32_16x16x128_f8f6f4 a[192:195], v[128:135], v[64:71], a[192:195] cbsz:1 blgp:1\n" \
  "ds_read_b128 v[144:147], v161 offset:40960\n" \
  "ds_read_b128 v[148:151], v161 offset:41984\n" \
  "v_mfma_f32_16x16x128_f8f6f4 a[196:199], v[128:135], v[72:79], a[196:199] cbsz:1 blgp:1\n" \
  "s_add_u32 m0, s69, 77824\n" \
  "s_nop 0\n" \
  "buffer_load_dwordx4 v165, s[36:39], s59 offen lds\n" \
  "v_mfma_f32_16x16x128_f8f6f4 a[200:203], v[128:135], v[80:87], a[200:203] cbsz:1 blgp:1\n" \
  "s_add_u32 m0, s69, 81920\n" \
  "s_nop 0\n" \
  "buffer_load_dwordx4 v165, s[36:39], s60 offen lds\n" \
  "v_mfma_f32_16x16x128_f8f6f4 a[204:207], v[128:135], v[88:95], a[204:207] cbsz:1 blgp:1\n" \
  "v_readlane_b32 s67, v175, s70\n" \
  "v_readlane_b32 s76, v169, s70\n" \
  "v_readlane_b32 s71, v181, s70\n" \
  "v_mfma_f32_16x16x128_f8f6f4 a[208:211], v[136:143], v[64:71], a[208:211] cbsz:1 blgp:1\n" \
  "ds_read_b128 v[152:155], v161 offset:43008\n" \
  "ds_read_b128 v[156:159], v161 offset:44032\n" \
  "v_mfma_f32_16x16x128_f8f6f4 a[212:215], v[136:143], v[72:79], a[212:215] cbsz:1 blgp:1\n" \
  "s_add_u32 m0, s69, 86016\n" \
  "s_nop 0\n" \
  "buffer_load_dwordx4 v165, s[36:39], s61 offen lds\n" \
  "v_mfma_f32_16x16x128_f8f6f4 a[216:219], v[136:143], v[80:87], a[216:219] cbsz:1 blgp:1\n" \
  "v_mfma_f32_16x16x128_f8f6f4 a[220:223], v[136:143], v[88:95], a[220:223] cbsz:1 blgp:1\n" \
  "s_add_u32 m0, s69, 90112\n" \
  "s_nop 0\n" \
  "buffer_load_dwordx4 v165, s[36:39], s62 offen lds\n" \
  "s_waitcnt lgkmcnt(2)\n" \
  "v_mfma_f32_16x16x128_f8f6f4 a[64:67], v[144:151], v[96:103], a[64:67] cbsz:1 blgp:1\n" \
  "ds_read_b128 v[128:131], v161 offset:45056\n" \
  "ds_read_b128 v[132:135], v161 offset:46080\n" \
  "v_mfma_f32_16x16x128_f8f6f4 a[68:71], v[144:151], v[104:111], a[68:71] cbsz:1 blgp:1\n" \
  "s_add_u32 s70, s70, 1\n" \
  "s_nop 0\n" \
  "v_add_u32 v163, s76, v162\n" \
  "v_mfma_f32_16x16x128_f8f6f4 a[72:75], v[144:151], v[112:119], a[72:75] cbsz:1 blgp:1\n" \
  "v_mfma_f32_16x16x128_f8f6f4 a[76:79], v[144:151], v[120:127], a[76:79] cbsz:1 blgp:1\n" \
  "v_add_u32 v164, 4096, v163\n" \
  "s_cmp_gt_u32 s67, s68\n" \
  "s_cbranch_scc1 Lp52_%=\n" \
  "Lq52_%=:\n" \
  "s_waitcnt lgkmcnt(2)\n" \
  "v_mfma_f32_16x16x128_f8f6f4 a[80:83], v[152:159], v[96:103], a[80:83] cbsz:1 blgp:1\n" \
  "ds_read_b128 v[136:139], v161 offset:47104\n" \
  "ds_read_b128 v[140:143], v161 offset:48128\n" \
  "v_mfma_f32_16x16x128_f8f6f4 a[84:87], v[152:159], v[104:111], a[84:87] cbsz:1 blgp:1\n" \
  "buffer_load_dwordx4 v[64:67], v165, s[36:39], s75 offen offset:0\n" \
  "v_mfma_f32_16x16x128_f8f6f4 a[88:91], v[152:159], v[112:119], a[88:91] cbsz:1 blgp:1\n" \
  "buffer_load_dwordx4 v[68:71], v165, s[36:39], s75 offen offset:1024\n" \
  "v_mfma_f32_16x16x128_f8f6f4 a[92:95], v[152:159], v[120:127], a[92:95] cbsz:1 blgp:1\n" \
  "s_waitcnt lgkmcnt(2)\n" \
  "v_mfma_f32_16x16x128_f8f6f4 a[96:99], v[128:135], v[96:103], a[96:99] cbsz:1 blgp:1\n" \
  "ds_read_b128 v[144:147], v161 offset:49152\n" \
  "ds_read_b128 v[148:151], v161 offset:50176\n" \
  "v_mfma_f32_16x16x128_f8f6f4 a[100:103], v[128:135], v[104:111], a[100:103] cbsz:1 blgp:1\n" \
  "v_mul_f32 v196, s78, v196\n" \
  "v_mul_f32 v197, s78, v197\n" \
  "v_mul_f32 v198, s78, v198\n" \
  "v_mul_f32 v199, s78, v199\n" \
  "v_mfma_f32_16x16x128_f8f6f4 a[104:107], v[128:135], v[112:119], a[104:107] cbsz:1 blgp:1\n" \
  "buffer_store_dwordx4 v[196:199], v167, s[40:43], s51 offen offset:0 nt\n" \
  "v_mfma_f32_16x16x128_f8f6f4 a[108:111], v[128:135], v[120:127], a[108:111] cbsz:1 blgp:1\n" \
  "s_waitcnt lgkmcnt(2)\n" \
  "v_mfma_f32_16x16x128_f8f6f4 a[112:115], v[136:143], v[96:103], a[112:115] cbsz:1 blgp:1\n" \
  "ds_read_b128 v[152:155], v161 offset:51200\n" \
  "ds_read_b128 v[156:159], v161 offset:52224\n" \
  "v_mfma_f32_16x16x128_f8f6f4 a[116:119], v[136:143], v[104:111], a[116:119] cbsz:1 blgp:1\n" \
  "buffer_load_dwordx4 v[72:75], v165, s[36:39], s75 offen offset:2048\n" \
  "v_mfma_f32_16x16x128_f8f6f4 a[120:123], v[136:143], v[112:119], a[120:123] cbsz:1 blgp:1\n" \
  "v_mfma_f32_16x16x128_f8f6f4 a[124:127], v[136:143], v[120:127], a[124:127] cbsz:1 blgp:1\n" \
  "buffer_load_dwordx4 v[76:79], v165, s[36:39], s75 offen offset:3072\n" \
  "s_waitcnt lgkmcnt(2)\n" \
  "v_mfma_f32_16x16x128_f8f6f4 a[128:131], v[144:151], v[96:103], a[128:131] cbsz:1 blgp:1\n" \
  "ds_read_b128 v[128:131], v161 offset:53248\n" \
  "ds_read_b128 v[132:135], v161 offset:54272\n" \
  "v_mfma_f32_16x16x128_f8f6f4 a[132:135], v[144:151], v[104:111], a[132:135] cbsz:1 blgp:1\n" \
  "buffer_load_dwordx4 v[80:83], v166, s[36:39], s75 offen offset:0\n" \
  "v_mfma_f32_16x16x128_f8f6f4 a[136:139], v[144:151], v[112:119], a[136:139] cbsz:1 blgp:1\n" \
  "v_mfma_f32_16x16x128_f8f6f4 a[140:143], v[144:151], v[120:127], a[140:143] cbsz:1 blgp:1\n" \
  "s_waitcnt lgkmcnt(2)\n" \
  "v_mfma_f32_16x16x128_f8f6f4 a[144:147], v[152:159], v[96:103], a[144:147] cbsz:1 blgp:1\n" \
  "ds_read_b128 v[136:139], v161 offset:55296\n" \
  "ds_read_b128 v[140:143], v161 offset:56320\n" \
  "v_mfma_f32_16x16x128_f8f6f4 a[148:151], v[152:159], v[104:111], a[148:151] cbsz:1 blgp:1\n" \
  "v_mul_f32 v212, s78, v212\n" \
  "v_mul_f32 v213, s78, v213\n" \
  "v_mul_f32 v214, s78, v214\n" \
  "v_mul_f32 v215, s78, v215\n" \
  "buffer_store_dwordx4 v[212:215], v167, s[40:43], s51 offen offset:64 nt\n" \
  "v_mfma_f32_16x16x128_f8f6f4 a[152:155], v[152:159], v[112:119], a[152:155] cbsz:1 blgp:1\n" \
  "v_mfma_f32_16x16x128_f8f6f4 a[156:159], v[152:159], v[120:127], a[156:159] cbsz:1 blgp:1\n" \
  "buffer_load_dwordx4 v[84:87], v166, s[36:39], s75 offen offset:1024\n" \
  "s_waitcnt lgkmcnt(2)\n" \
  "v_mfma_f32_16x16x128_f8f6f4 a[160:163], v[128:135], v[96:103], a[160:163] cbsz:1 blgp:1\n" \
  "ds_read_b128 v[144:147], v161 offset:57344\n" \
  "ds_read_b128 v[148:151], v161 offset:58368\n" \
  "v_mfma_f32_16x16x128_f8f6f4 a[164:167], v[128:135], v[104:111], a[164:167] cbsz:1 blgp:1\n" \
  "v_mfma_f32_16x16x128_f8f6f4 a[168:171], v[128:135], v[112:119], a[168:171] cbsz:1 blgp:1\n" \
  "buffer_load_dwordx4 v[88:91], v166, s[36:39], s75 offen offset:2048\n" \
  "v_mfma_f32_16x16x128_f8f6f4 a[172:175], v[128:135], v[120:127], a[172:175] cbsz:1 blgp:1\n" \
  "s_waitcnt lgkmcnt(2)\n" \
  "v_mfma_f32_16x16x128_f8f6f4 a[176:179], v[136:143], v[96:103], a[176:179] cbsz:1 blgp:1\n" \
  "ds_read_b128 v[152:155], v161 offset:59392\n" \
  "ds_read_b128 v[156:159], v161 offset:60416\n" \
  "v_mfma_f32_16x16x128_f8f6f4 a[180:183], v[136:143], v[104:111], a[180:183] cbsz:1 blgp:1\n" \
  "buffer_load_dwordx4 v[92:95], v166, s[36:39], s75 offen offset:3072\n" \
  "v_mfma_f32_16x16x128_f8f6f4 a[184:187], v[136:143], v[112:119], a[184:187] cbsz:1 blgp:1\n" \
  "v_mfma_f32_16x16x128_f8f6f4 a[188:191], v[136:143], v[120:127], a[188:191] cbsz:1 blgp:1\n" \
  "s_waitcnt vmcnt(28) lgkmcnt(0)\n" \
  "s_barrier\n" \
  "v_mfma_f32_16x16x128_f8f6f4 a[192:195], v[144:151], v[96:103], a[192:195] cbsz:1 blgp:1\n" \
  "ds_read_b128 v[128:131], v160 offset:12288\n" \
  "ds_read_b128 v[132:135], v160 offset:13312\n" \
  "v_mfma_f32_16x16x128_f8f6f4 a[196:199], v[144:151], v[104:111], a[196:199] cbsz:1 blgp:1\n" \
  "s_mov_b32 s42, s72\n" \
  "s_mov_b32 s48, s71\n" \
  "s_add_u32 s49, s71, 1048576\n" \
  "s_add_u32 s50, s71, 2097152\n" \
  "v_mfma_f32_16x16x128_f8f6f4 a[200:203], v[144:151], v[112:119], a[200:203] cbsz:1 blgp:1\n" \
  "s_add_u32 s51, s71, 3145728\n" \
  "s_cmp_eq_u32 s70, 64\n" \
  "s_cbranch_scc1 Lp53_%=\n" \
  "Lq53_%=:\n" \
  "v_mfma_f32_16x16x128_f8f6f4 a[204:207], v[144:151], v[120:127], a[204:207] cbsz:1 blgp:1\n" \
  "v_accvgpr_read_b32 v184, a64\n" \
  "v_accvgpr_read_b32 v185, a65\n" \
  "v_accvgpr_read_b32 v186, a66\n" \
  "v_accvgpr_read_b32 v187, a67\n" \
  "v_mfma_f32_16x16x128_f8f6f4 a[208:211], v[152:159], v[96:103], a[208:211] cbsz:1 blgp:1\n" \
  "ds_read_b128 v[136:139], v160 offset:14336\n" \
  "ds_read_b128 v[140:143], v160 offset:15360\n" \
  "v_mfma_f32_16x16x128_f8f6f4 a[212:215], v[152:159], v[104:111], a[212:215] cbsz:1 blgp:1\n" \
  "v_accvgpr_read_b32 v200, a80\n" \
  "v_accvgpr_read_b32 v201, a81\n" \
  "v_accvgpr_read_b32 v202, a82\n" \
  "v_accvgpr_read_b32 v203, a83\n" \
  "v_mfma_f32_16x16x128_f8f6f4 a[216:219], v[152:159], v[112:119], a[216:219] cbsz:1 blgp:1\n" \
  "v_accvgpr_read_b32 v188, a68\n" \
  "v_accvgpr_read_b32 v189, a69\n" \
  "v_accvgpr_read_b32 v190, a70\n" \
  "v_accvgpr_read_b32 v191, a71\n" \
  "v_mfma_f32_16x16x128_f8f6f4 a[220:223], v[152:159], v[120:127], a[220:223] cbsz:1 blgp:1\n" \
  "v_accvgpr_read_b32 v204, a84\n" \
  "v_accvgpr_read_b32 v205, a85\n" \
  "v_accvgpr_read_b32 v206, a86\n" \
  "v_accvgpr_read_b32 v207, a87\n" \
  "s_waitcnt lgkmcnt(2)\n" \
  "v_mfma_f32_16x16x128_f8f6f4 a[96:99], v[128:135], v[0:7], a[96:99] cbsz:1 blgp:1\n" \
  "ds_read_b128 v[144:147], v160 offset:16384\n" \
  "ds_read_b128 v[148:151], v160 offset:17408\n" \
  "v_mfma_f32_16x16x128_f8f6f4 a[100:103], v[128:135], v[8:15], a[100:103] cbsz:1 blgp:1\n" \
  "v_accvgpr_read_b32 v192, a72\n" \
  "v_accvgpr_read_b32 v193, a73\n" \
  "v_accvgpr_read_b32 v194, a74\n" \
  "v_accvgpr_read_b32 v195, a75\n" \
  "v_mfma_f32_16x16x128_f8f6f4 a[104:107], v[128:135], v[16:23], a[104:107] cbsz:1 blgp:1\n" \
  "v_accvgpr_read_b32 v208, a88\n" \
  "v_accvgpr_read_b32 v209, a89\n" \
  "v_accvgpr_read_b32 v210, a90\n" \
  "v_accvgpr_read_b32 v211, a91\n" \
  "v_mfma_f32_16x16x128_f8f6f4 a[108:111], v[128:135], v[24:31], a[108:111] cbsz:1 blgp:1\n" \
  "v_accvgpr_read_b32 v196, a76\n" \
  "v_accvgpr_read_b32 v197, a77\n" \
  "v_accvgpr_read_b32 v198, a78\n" \
  "v_accvgpr_read_b32 v199, a79\n" \
  "s_waitcnt lgkmcnt(2)\n" \
  "v_mfma_f32_16x16x128_f8f6f4 a[112:115], v[136:143], v[0:7], a[112:115] cbsz:1 blgp:1\n" \
  "ds_read_b128 v[152:155], v160 offset:18432\n" \
  "ds_read_b128 v[156:159], v160 offset:19456\n" \
  "v_mfma_f32_16x16x128_f8f6f4 a[116:119], v[136:143], v[8:15], a[116:119] cbsz:1 blgp:1\n" \
  "v_accvgpr_read_b32 v212, a92\n" \
  "v_accvgpr_read_b32 v213, a93\n" \
  "v_accvgpr_read_b32 v214, a94\n" \
  "v_accvgpr_read_b32 v215, a95\n" \
  "s_add_u32 m0, s69, 110592\n" \
  "s_nop 0\n" \
  "buffer_load_dwordx4 v163, s[36:39], s59 offen lds\n" \
  "v_mfma_f32_16x16x128_f8f6f4 a[120:123], v[136:143], v[16:23], a[120:123] cbsz:1 blgp:1\n" \
  "s_add_u32 m0, s69, 114688\n" \
  "s_nop 0\n" \
  "buffer_load_dwordx4 v163, s[36:39], s60 offen lds\n" \
  "v_readlane_b32 s67, v175, s70\n" \
  "v_readlane_b32 s76, v169, s70\n" \
  "v_mfma_f32_16x16x128_f8f6f4 a[124:127], v[136:143], v[24:31], a[124:127] cbsz:1 blgp:1\n" \
  "s_add_u32 m0, s69, 118784\n" \
  "s_nop 0\n" \
  "buffer_load_dwordx4 v163, s[36:39], s61 offen lds\n" \
  "s_waitcnt lgkmcnt(2)\n" \
  "v_mfma_f32_16x16x128_f8f6f4 a[128:131], v[144:151], v[0:7], a[128:131] cbsz:1 blgp:1\n" \
  "ds_read_b128 v[128:131], v160 offset:20480\n" \
  "ds_read_b128 v[132:135], v160 offset:21504\n" \
  "v_mfma_f32_16x16x128_f8f6f4 a[132:135], v[144:151], v[8:15], a[132:135] cbsz:1 blgp:1\n" \
  "s_add_u32 m0, s69, 122880\n" \
  "s_nop 0\n" \
  "buffer_load_dwordx4 v163, s[36:39], s62 offen lds\n" \
  "s_add_u32 s70, s70, 1\n" \
  "s_nop 0\n" \
  "v_add_u32 v165, s76, v162\n" \
  "v_add_u32 v166, 4096, v165\n" \
  "s_cmp_gt_u32 s67, s68\n" \
  "s_cbranch_scc1 Lp54_%=\n" \
  "Lq54_%=:\n" \
  "v_mfma_f32_16x16x128_f8f6f4 a[136:139], v[144:151], v[16:23], a[136:139] cbsz:1 blgp:1\n" \
  "buffer_load_dwordx4 v[96:99], v163, s[36:39], s75 offen offset:0\n" \
  "v_mfma_f32_16x16x128_f8f6f4 a[140:143], v[144:151], v[24:31], a[140:143] cbsz:1 blgp:1\n" \
  "buffer_load_dwordx4 v[100:103], v163, s[36:39], s75 offen offset:1024\n" \
  "s_waitcnt lgkmcnt(2)\n" \
  "v_mfma_f32_16x16x128_f8f6f4 a[144:147], v[152:159], v[0:7], a[144:147] cbsz:1 blgp:1\n" \
  "ds_read_b128 v[136:139], v160 offset:22528\n" \
  "ds_read_b128 v[140:143], v160 offset:23552\n" \
  "v_mfma_f32_16x16x128_f8f6f4 a[148:151], v[152:159], v[8:15], a[148:151] cbsz:1 blgp:1\n" \
  "buffer_load_dwordx4 v[104:107], v163, s[36:39], s75 offen offset:2048\n" \
  "v_mfma_f32_16x16x128_f8f6f4 a[152:155], v[152:159], v[16:23], a[152:155] cbsz:1 blgp:1\n" \
  "buffer_load_dwordx4 v[108:111], v163, s[36:39], s75 offen offset:3072\n" \
  "v_mfma_f32_16x16x128_f8f6f4 a[156:159], v[152:159], v[24:31], a[156:159] cbsz:1 blgp:1\n" \
  "buffer_load_dwordx4 v[112:115], v164, s[36:39], s75 offen offset:0\n" \
  "s_waitcnt lgkmcnt(2)\n" \
  "v_mfma_f32_16x16x128_f8f6f4 a[160:163], v[128:135], v[0:7], a[160:163] cbsz:1 blgp:1\n" \
  "ds_read_b128 v[144:147], v160 offset:24576\n" \
  "ds_read_b128 v[148:151], v160 offset:25600\n" \
  "v_mfma_f32_16x16x128_f8f6f4 a[164:167], v[128:135], v[8:15], a[164:167] cbsz:1 blgp:1\n" \
  "buffer_load_dwordx4 v[116:119], v164, s[36:39], s75 offen offset:1024\n" \
  "v_mul_f32 v184, s78, v184\n" \
  "v_mul_f32 v185, s78, v185\n" \
  "v_mul_f32 v186, s78, v186\n" \
  "v_mul_f32 v187, s78, v187\n" \
  "v_mfma_f32_16x16x128_f8f6f4 a[168:171], v[128:135], v[16:23], a[168:171] cbsz:1 blgp:1\n" \
  "buffer_store_dwordx4 v[184:187], v167, s[40:43], s48 offen offset:0 nt\n" \
  "v_mfma_f32_16x16x128_f8f6f4 a[172:175], v[128:135], v[24:31], a[172:175] cbsz:1 blgp:1\n" \
  "buffer_load_dwordx4 v[120:123], v164, s[36:39], s75 offen offset:2048\n" \
  "v_mul_f32 v200, s78, v200\n" \
  "v_mul_f32 v201, s78, v201\n" \
  "v_mul_f32 v202, s78, v202\n" \
  "v_mul_f32 v203, s78, v203\n" \
  "s_waitcnt lgkmcnt(2)\n" \
  "v_mfma_f32_16x16x128_f8f6f4 a[176:179], v[136:143], v[0:7], a[176:179] cbsz:1 blgp:1\n" \
  "ds_read_b128 v[152:155], v160 offset:26624\n" \
  "ds_read_b128 v[156:159], v160 offset:27648\n" \
  "v_mfma_f32_16x16x128_f8f6f4 a[180:183], v[136:143], v[8:15], a[180:183] cbsz:1 blgp:1\n" \
  "buffer_store_dwordx4 v[200:203], v167, s[40:43], s48 offen offset:64 nt\n" \
  "v_mfma_f32_16x16x128_f8f6f4 a[184:187], v[136:143], v[16:23], a[184:187] cbsz:1 blgp:1\n" \
  "buffer_load_dwordx4 v[124:127], v164, s[36:39], s75 offen offset:3072\n" \
  "v_mfma_f32_16x16x128_f8f6f4 a[188:191], v[136:143], v[24:31], a[188:191] cbsz:1 blgp:1\n" \
  "s_waitcnt vmcnt(28) lgkmcnt(0)\n" \
  "s_barrier\n" \
  "v_mfma_f32_16x16x128_f8f6f4 a[192:195], v[144:151], v[0:7], a[192:195] cbsz:1 blgp:1\n" \
  "ds_read_b128 v[128:131], v160 offset:45056\n" \
  "ds_read_b128 v[132:135], v160 offset:46080\n" \
  "v_mfma_f32_16x16x128_f8f6f4 a[196:199], v[144:151], v[8:15], a[196:199] cbsz:1 blgp:1\n" \
  "s_add_u32 m0, s69, 16384\n" \
  "s_nop 0\n" \
  "buffer_load_dwordx4 v165, s[36:39], s60 offen lds\n" \
  "v_mfma_f32_16x16x128_f8f6f4 a[200:203], v[144:151], v[16:23], a[200:203] cbsz:1 blgp:1\n" \
  "s_add_u32 m0, s69, 20480\n" \
  "s_nop 0\n" \
  "buffer_load_dwordx4 v165, s[36:39], s61 offen lds\n" \
  "v_mfma_f32_16x16x128_f8f6f4 a[204:207], v[144:151], v[24:31], a[204:207] cbsz:1 blgp:1\n" \
  "v_readlane_b32 s67, v175, s70\n" \
  "v_readlane_b32 s76, v169, s70\n" \
  "v_mfma_f32_16x16x128_f8f6f4 a[208:211], v[152:159], v[0:7], a[208:211] cbsz:1 blgp:1\n" \
  "ds_read_b128 v[136:139], v160 offset:47104\n" \
  "ds_read_b128 v[140:143], v160 offset:48128\n" \
  "v_mfma_f32_16x16x128_f8f6f4 a[212:215], v[152:159], v[8:15], a[212:215] cbsz:1 blgp:1\n" \
  "s_add_u32 m0, s69, 24576\n" \
  "s_nop 0\n" \
  "buffer_load_dwordx4 v165, s[36:39], s62 offen lds\n" \
  "v_mfma_f32_16x16x128_f8f6f4 a[216:219], v[152:159], v[16:23], a[216:219] cbsz:1 blgp:1\n" \
  "s_add_u32 s70, s70, 1\n" \
  "s_nop 0\n" \
  "v_add_u32 v163, s76, v162\n" \
  "v_mfma_f32_16x16x128_f8f6f4 a[220:223], v[152:159], v[24:31], a[220:223] cbsz:1 blgp:1\n" \
  "v_add_u32 v164, 4096, v163\n" \
  "s_cmp_gt_u32 s67, s68\n" \
  "s_cbranch_scc1 Lp55_%=\n" \
  "Lq55_%=:\n" \
  "s_waitcnt lgkmcnt(2)\n" \
  "v_mfma_f32_16x16x128_f8f6f4 a[96:99], v[128:135], v[32:39], a[96:99] cbsz:1 blgp:1\n" \
  "ds_read_b128 v[144:147], v160 offset:49152\n" \
  "ds_read_b128 v[148:151], v160 offset:50176\n" \
  "v_mfma_f32_16x16x128_f8f6f4 a[100:103], v[128:135], v[40:47], a[100:103] cbsz:1 blgp:1\n" \
  "buffer_load_dwordx4 v[0:3], v165, s[36:39], s75 offen offset:0\n" \
  "v_mfma_f32_16x16x128_f8f6f4 a[104:107], v[128:135], v[48:55], a[104:107] cbsz:1 blgp:1\n" \
  "buffer_load_dwordx4 v[4:7], v165, s[36:39], s75 offen offset:1024\n" \
  "v_mfma_f32_16x16x128_f8f6f4 a[108:111], v[128:135], v[56:63], a[108:111] cbsz:1 blgp:1\n" \
  "s_waitcnt lgkmcnt(2)\n" \
  "v_mfma_f32_16x16x128_f8f6f4 a[112:115], v[136:143], v[32:39], a[112:115] cbsz:1 blgp:1\n" \
  "ds_read_b128 v[152:155], v160 offset:51200\n" \
  "ds_read_b128 v[156:159], v160 offset:52224\n" \
  "v_mfma_f32_16x16x128_f8f6f4 a[116:119], v[136:143], v[40:47], a[116:119] cbsz:1 blgp:1\n" \
  "v_mul_f32 v188, s78, v188\n" \
  "v_mul_f32 v189, s78, v189\n" \
  "v_mul_f32 v190, s78, v190\n" \
  "v_mul_f32 v191, s78, v191\n" \
  "buffer_store_dwordx4 v[188:191], v167, s[40:43], s49 offen offset:0 nt\n" \
  "v_mfma_f32_16x16x128_f8f6f4 a[120:123], v[136:143], v[48:55], a[120:123] cbsz:1 blgp:1\n" \
  "v_mfma_f32_16x16x128_f8f6f4 a[124:127], v[136:143], v[56:63], a[124:127] cbsz:1 blgp:1\n" \
  "buffer_load_dwordx4 v[8:11], v165, s[36:39], s75 offen offset:2048\n" \
  "s_waitcnt lgkmcnt(2)\n" \
  "v_mfma_f32_16x16x128_f8f6f4 a[128:131], v[144:151], v[32:39], a[128:131] cbsz:1 blgp:1\n" \
  "ds_read_b128 v[128:131], v160 offset:53248\n" \
  "ds_read_b128 v[132:135], v160 offset:54272\n" \
  "v_mfma_f32_16x16x128_f8f6f4 a[132:135], v[144:151], v[40:47], a[132:135] cbsz:1 blgp:1\n" \
  "buffer_load_dwordx4 v[12:15], v165, s[36:39], s75 offen offset:3072\n" \
  "v_mfma_f32_16x16x128_f8f6f4 a[136:139], v[144:151], v[48:55], a[136:139] cbsz:1 blgp:1\n" \
  "v_mfma_f32_16x16x128_f8f6f4 a[140:143], v[144:151], v[56:63], a[140:143] cbsz:1 blgp:1\n" \
  "buffer_load_dwordx4 v[16:19], v166, s[36:39], s75 offen offset:0\n" \
  "s_waitcnt lgkmcnt(2)\n" \
  "v_mfma_f32_16x16x128_f8f6f4 a[144:147], v[152:159], v[32:39], a[144:147] cbsz:1 blgp:1\n" \
  "ds_read_b128 v[136:139], v160 offset:55296\n" \
  "ds_read_b128 v[140:143], v160 offset:56320\n" \
  "v_mfma_f32_16x16x128_f8f6f4 a[148:151], v[152:159], v[40:47], a[148:151] cbsz:1 blgp:1\n" \
  "v_mul_f32 v204, s78, v204\n" \
  "v_mul_f32 v205, s78, v205\n" \
  "v_mul_f32 v206, s78, v206\n" \
  "v_mul_f32 v207, s78, v207\n" \
  "v_mfma_f32_16x16x128_f8f6f4 a[152:155], v[152:159], v[48:55], a[152:155] cbsz:1 blgp:1\n" \
  "buffer_store_dwordx4 v[204:207], v167, s[40:43], s49 offen offset:64 nt\n" \
  "v_mfma_f32_16x16x128_f8f6f4 a[156:159], v[152:159], v[56:63], a[156:159] cbsz:1 blgp:1\n" \
  "s_waitcnt lgkmcnt(2)\n" \
  "v_mfma_f32_16x16x128_f8f6f4 a[160:163], v[128:135], v[32:39], a[160:163] cbsz:1 blgp:1\n" \
  "ds_read_b128 v[144:147], v160 offset:57344\n" \
  "ds_read_b128 v[148:151], v160 offset:58368\n" \
  "v_mfma_f32_16x16x128_f8f6f4 a[164:167], v[128:135], v[40:47], a[164:167] cbsz:1 blgp:1\n" \
  "buffer_load_dwordx4 v[20:23], v166, s[36:39], s75 offen offset:1024\n" \
  "v_mfma_f32_16x16x128_f8f6f4 a[168:171], v[128:135], v[48:55], a[168:171] cbsz:1 blgp:1\n" \
  "buffer_load_dwordx4 v[24:27], v166, s[36:39], s75 offen offset:2048\n" \
  "v_mfma_f32_16x16x128_f8f6f4 a[172:175], v[128:135], v[56:63], a[172:175] cbsz:1 blgp:1\n" \
  "s_waitcnt lgkmcnt(2)\n" \
  "v_mfma_f32_16x16x128_f8f6f4 a[176:179], v[136:143], v[32:39], a[176:179] cbsz:1 blgp:1\n" \
  "ds_read_b128 v[152:155], v160 offset:59392\n" \
  "ds_read_b128 v[156:159], v160 offset:60416\n" \
  "v_mfma_f32_16x16x128_f8f6f4 a[180:183], v[136:143], v[40:47], a[180:183] cbsz:1 blgp:1\n" \
  "buffer_load_dwordx4 v[28:31], v166, s[36:39], s75 offen offset:3072\n" \
  "v_mfma_f32_16x16x128_f8f6f4 a[184:187], v[136:143], v[48:55], a[184:187] cbsz:1 blgp:1\n" \
  "v_mfma_f32_16x16x128_f8f6f4 a[188:191], v[136:143], v[56:63], a[188:191] cbsz:1 blgp:1\n" \
  "s_waitcnt vmcnt(27) lgkmcnt(0)\n" \
  "s_barrier\n" \
  "v_mfma_f32_16x16x128_f8f6f4 a[192:195], v[144:151], v[32:39], a[192:195] cbsz:1 blgp:1\n" \
  "ds_read_b128 v[128:131], v161 offset:12288\n" \
  "ds_read_b128 v[132:135], v161 offset:13312\n" \
  "v_mfma_f32_16x16x128_f8f6f4 a[196:199], v[144:151], v[40:47], a[196:199] cbsz:1 blgp:1\n" \
  "s_add_u32 m0, s69, 49152\n" \
  "s_nop 0\n" \
  "buffer_load_dwordx4 v163, s[36:39], s60 offen lds\n" \
  "v_mfma_f32_16x16x128_f8f6f4 a[200:203], v[144:151], v[48:55], a[200:203] cbsz:1 blgp:1\n" \
  "s_add_u32 m0, s69, 53248\n" \
  "s_nop 0\n" \
  "buffer_load_dwordx4 v163, s[36:39], s61 offen lds\n" \
  "v_mfma_f32_16x16x128_f8f6f4 a[204:207], v[144:151], v[56:63], a[204:207] cbsz:1 blgp:1\n" \
  "v_readlane_b32 s67, v175, s70\n" \
  "v_readlane_b32 s76, v169, s70\n" \
  "v_mfma_f32_16x16x128_f8f6f4 a[208:211], v[152:159], v[32:39], a[208:211] cbsz:1 blgp:1\n" \
  "ds_read_b128 v[136:139], v161 offset:14336\n" \
  "ds_read_b128 v[140:143], v161 offset:15360\n" \
  "v_mfma_f32_16x16x128_f8f6f4 a[212:215], v[152:159], v[40:47], a[212:215] cbsz:1 blgp:1\n" \
  "s_add_u32 m0, s69, 57344\n" \
  "s_nop 0\n" \
  "buffer_load_dwordx4 v163, s[36:39], s62 offen lds\n" \
  "v_mfma_f32_16x16x128_f8f6f4 a[216:219], v[152:159], v[48:55], a[216:219] cbsz:1 blgp:1\n" \
  "s_add_u32 s70, s70, 1\n" \
  "s_nop 0\n" \
  "v_add_u32 v165, s76, v162\n" \
  "v_mfma_f32_16x16x128_f8f6f4 a[220:223], v[152:159], v[56:63], a[220:223] cbsz:1 blgp:1\n" \
  "v_add_u32 v166, 4096, v165\n" \
  "s_cmp_gt_u32 s67, s68\n" \
  "s_cbranch_scc1 Lp56_%=\n" \
  "Lq56_%=:\n" \
  "s_waitcnt lgkmcnt(2)\n" \
  "v_mfma_f32_16x16x128_f8f6f4 a[96:99], v[128:135], v[64:71], a[96:99] cbsz:1 blgp:1\n" \
  "ds_read_b128 v[144:147], v161 offset:16384\n" \
  "ds_read_b128 v[148:151], v161 offset:17408\n" \
  "v_mfma_f32_16x16x128_f8f6f4 a[100:103], v[128:135], v[72:79], a[100:103] cbsz:1 blgp:1\n" \
  "buffer_load_dwordx4 v[32:35], v163, s[36:39], s75 offen offset:0\n" \
  "v_mfma_f32_16x16x128_f8f6f4 a[104:107], v[128:135], v[80:87], a[104:107] cbsz:1 blgp:1\n" \
  "buffer_load_dwordx4 v[36:39], v163, s[36:39], s75 offen offset:1024\n" \
  "v_mfma_f32_16x16x128_f8f6f4 a[108:111], v[128:135], v[88:95], a[108:111] cbsz:1 blgp:1\n" \
  "s_waitcnt lgkmcnt(2)\n" \
  "v_mfma_f32_16x16x128_f8f6f4 a[112:115], v[136:143], v[64:71], a[112:115] cbsz:1 blgp:1\n" \
  "ds_read_b128 v[152:155], v161 offset:18432\n" \
  "ds_read_b128 v[156:159], v161 offset:19456\n" \
  "v_mfma_f32_16x16x128_f8f6f4 a[116:119], v[136:143], v[72:79], a[116:119] cbsz:1 blgp:1\n" \
  "v_mul_f32 v192, s78, v192\n" \
  "v_mul_f32 v193, s78, v193\n" \
  "v_mul_f32 v194, s78, v194\n" \
  "v_mul_f32 v195, s78, v195\n" \
  "buffer_store_dwordx4 v[192:195], v167, s[40:43], s50 offen offset:0 nt\n" \
  "v_mfma_f32_16x16x128_f8f6f4 a[120:123], v[136:143], v[80:87], a[120:123] cbsz:1 blgp:1\n" \
  "v_mfma_f32_16x16x128_f8f6f4 a[124:127], v[136:143], v[88:95], a[124:127] cbsz:1 blgp:1\n" \
  "buffer_load_dwordx4 v[40:43], v163, s[36:39], s75 offen offset:2048\n" \
  "s_waitcnt lgkmcnt(2)\n" \
  "v_mfma_f32_16x16x128_f8f6f4 a[128:131], v[144:151], v[64:71], a[128:131] cbsz:1 blgp:1\n" \
  "ds_read_b128 v[128:131], v161 offset:20480\n" \
  "ds_read_b128 v[132:135], v161 offset:21504\n" \
  "v_mfma_f32_16x16x128_f8f6f4 a[132:135], v[144:151], v[72:79], a[132:135] cbsz:1 blgp:1\n" \
  "buffer_load_dwordx4 v[44:47], v163, s[36:39], s75 offen offset:3072\n" \
  "v_mfma_f32_16x16x128_f8f6f4 a[136:139], v[144:151], v[80:87], a[136:139] cbsz:1 blgp:1\n" \
  "v_mfma_f32_16x16x128_f8f6f4 a[140:143], v[144:151], v[88:95], a[140:143] cbsz:1 blgp:1\n" \
  "buffer_load_dwordx4 v[48:51], v164, s[36:39], s75 offen offset:0\n" \
  "s_waitcnt lgkmcnt(2)\n" \
  "v_mfma_f32_16x16x128_f8f6f4 a[144:147], v[152:159], v[64:71], a[144:147] cbsz:1 blgp:1\n" \
  "ds_read_b128 v[136:139], v161 offset:22528\n" \
  "ds_read_b128 v[140:143], v161 offset:23552\n" \
  "v_mfma_f32_16x16x128_f8f6f4 a[148:151], v[152:159], v[72:79], a[148:151] cbsz:1 blgp:1\n" \
  "v_mul_f32 v208, s78, v208\n" \
  "v_mul_f32 v209, s78, v209\n" \
  "v_mul_f32 v210, s78, v210\n" \
  "v_mul_f32 v211, s78, v211\n" \
  "v_mfma_f32_16x16x128_f8f6f4 a[152:155], v[152:159], v[80:87], a[152:155] cbsz:1 blgp:1\n" \
  "buffer_store_dwordx4 v[208:211], v167, s[40:43], s50 offen offset:64 nt\n" \
  "v_mfma_f32_16x16x128_f8f6f4 a[156:159], v[152:159], v[88:95], a[156:159] cbsz:1 blgp:1\n" \
  "s_waitcnt lgkmcnt(2)\n" \
  "v_mfma_f32_16x16x128_f8f6f4 a[160:163], v[128:135], v[64:71], a[160:163] cbsz:1 blgp:1\n" \
  "ds_read_b128 v[144:147], v161 offset:24576\n" \
  "ds_read_b128 v[148:151], v161 offset:25600\n" \
  "v_mfma_f32_16x16x128_f8f6f4 a[164:167], v[128:135], v[72:79], a[164:167] cbsz:1 blgp:1\n" \
  "buffer_load_dwordx4 v[52:55], v164, s[36:39], s75 offen offset:1024\n" \
  "v_mfma_f32_16x16x128_f8f6f4 a[168:171], v[128:135], v[80:87], a[168:171] cbsz:1 blgp:1\n" \
  "buffer_load_dwordx4 v[56:59], v164, s[36:39], s75 offen offset:2048\n" \
  "v_mfma_f32_16x16x128_f8f6f4 a[172:175], v[128:135], v[88:95], a[172:175] cbsz:1 blgp:1\n" \
  "s_waitcnt lgkmcnt(2)\n" \
  "v_mfma_f32_16x16x128_f8f6f4 a[176:179], v[136:143], v[64:71], a[176:179] cbsz:1 blgp:1\n" \
  "ds_read_b128 v[152:155], v161 offset:26624\n" \
  "ds_read_b128 v[156:159], v161 offset:27648\n" \
  "v_mfma_f32_16x16x128_f8f6f4 a[180:183], v[136:143], v[72:79], a[180:183] cbsz:1 blgp:1\n" \
  "buffer_load_dwordx4 v[60:63], v164, s[36:39], s75 offen offset:3072\n" \
  "v_mfma_f32_16x16x128_f8f6f4 a[184:187], v[136:143], v[80:87], a[184:187] cbsz:1 blgp:1\n" \
  "v_mfma_f32_16x16x128_f8f6f4 a[188:191], v[136:143], v[88:95], a[188:191] cbsz:1 blgp:1\n" \
  "s_waitcnt vmcnt(26) lgkmcnt(0)\n" \
  "s_barrier\n" \
  "v_mfma_f32_16x16x128_f8f6f4 a[192:195], v[144:151], v[64:71], a[192:195] cbsz:1 blgp:1\n" \
  "ds_read_b128 v[128:131], v161 offset:45056\n" \
  "ds_read_b128 v[132:135], v161 offset:46080\n" \
  "v_mfma_f32_16x16x128_f8f6f4 a[196:199], v[144:151], v[72:79], a[196:199] cbsz:1 blgp:1\n" \
  "s_add_u32 m0, s69, 81920\n" \
  "s_nop 0\n" \
  "buffer_load_dwordx4 v165, s[36:39], s60 offen lds\n" \
  "v_mfma_f32_16x16x128_f8f6f4 a[200:203], v[144:151], v[80:87], a[200:203] cbsz:1 blgp:1\n" \
  "s_add_u32 m0, s69, 86016\n" \
  "s_nop 0\n" \
  "buffer_load_dwordx4 v165, s[36:39], s61 offen lds\n" \
  "v_mfma_f32_16x16x128_f8f6f4 a[204:207], v[144:151], v[88:95], a[204:207] cbsz:1 blgp:1\n" \
  "v_readlane_b32 s67, v175, s70\n" \
  "v_readlane_b32 s76, v169, s70\n" \
  "v_readlane_b32 s71, v181, s70\n" \
  "v_mfma_f32_16x16x128_f8f6f4 a[208:211], v[152:159], v[64:71], a[208:211] cbsz:1 blgp:1\n" \
  "ds_read_b128 v[136:139], v161 offset:47104\n" \
  "ds_read_b128 v[140:143], v161 offset:48128\n" \
  "v_mfma_f32_16x16x128_f8f6f4 a[212:215], v[152:159], v[72:79], a[212:215] cbsz:1 blgp:1\n" \
  "s_add_u32 m0, s69, 90112\n" \
  "s_nop 0\n" \
  "buffer_load_dwordx4 v165, s[36:39], s62 offen lds\n" \
  "v_mfma_f32_16x16x128_f8f6f4 a[216:219], v[152:159], v[80:87], a[216:219] cbsz:1 blgp:1\n" \
  "s_add_u32 s70, s70, 1\n" \
  "s_nop 0\n" \
  "v_add_u32 v163, s76, v162\n" \
  "v_mfma_f32_16x16x128_f8f6f4 a[220:223], v[152:159], v[88:95], a[220:223] cbsz:1 blgp:1\n" \
  "v_add_u32 v164, 4096, v163\n" \
  "s_cmp_gt_u32 s67, s68\n" \
  "s_cbranch_scc1 Lp57_%=\n" \
  "Lq57_%=:\n" \
  "s_waitcnt lgkmcnt(2)\n" \
  "v_mfma_f32_16x16x128_f8f6f4 a[96:99], v[128:135], v[96:103], a[96:99] cbsz:1 blgp:1\n" \
  "ds_read_b128 v[144:147], v161 offset:49152\n" \
  "ds_read_b128 v[148:151], v161 offset:50176\n" \
  "v_mfma_f32_16x16x128_f8f6f4 a[100:103], v[128:135], v[104:111], a[100:103] cbsz:1 blgp:1\n" \
  "buffer_load_dwordx4 v[64:67], v165, s[36:39], s75 offen offset:0\n" \
  "v_mfma_f32_16x16x128_f8f6f4 a[104:107], v[128:135], v[112:119], a[104:107] cbsz:1 blgp:1\n" \
  "buffer_load_dwordx4 v[68:71], v165, s[36:39], s75 offen offset:1024\n" \
  "v_mfma_f32_16x16x128_f8f6f4 a[108:111], v[128:135], v[120:127], a[108:111] cbsz:1 blgp:1\n" \
  "s_waitcnt lgkmcnt(2)\n" \
  "v_mfma_f32_16x16x128_f8f6f4 a[112:115], v[136:143], v[96:103], a[112:115] cbsz:1 blgp:1\n" \
  "ds_read_b128 v[152:155], v161 offset:51200\n" \
  "ds_read_b128 v[156:159], v161 offset:52224\n" \
  "v_mfma_f32_16x16x128_f8f6f4 a[116:119], v[136:143], v[104:111], a[116:119] cbsz:1 blgp:1\n" \
  "v_mul_f32 v196, s78, v196\n" \
  "v_mul_f32 v197, s78, v197\n" \
  "v_mul_f32 v198, s78, v198\n" \
  "v_mul_f32 v199, s78, v199\n" \
  "buffer_store_dwordx4 v[196:199], v167, s[40:43], s51 offen offset:0 nt\n" \
  "v_mfma_f32_16x16x128_f8f6f4 a[120:123], v[136:143], v[112:119], a[120:123] cbsz:1 blgp:1\n" \
  "v_mfma_f32_16x16x128_f8f6f4 a[124:127], v[136:143], v[120:127], a[124:127] cbsz:1 blgp:1\n" \
  "buffer_load_dwordx4 v[72:75], v165, s[36:39], s75 offen offset:2048\n" \
  "s_waitcnt lgkmcnt(2)\n" \
  "v_mfma_f32_16x16x128_f8f6f4 a[128:131], v[144:151], v[96:103], a[128:131] cbsz:1 blgp:1\n" \
  "ds_read_b128 v[128:131], v161 offset:53248\n" \
  "ds_read_b128 v[132:135], v161 offset:54272\n" \
  "v_mfma_f32_16x16x128_f8f6f4 a[132:135], v[144:151], v[104:111], a[132:135] cbsz:1 blgp:1\n" \
  "buffer_load_dwordx4 v[76:79], v165, s[36:39], s75 offen offset:3072\n" \
  "v_mfma_f32_16x16x128_f8f6f4 a[136:139], v[144:151], v[112:119], a[136:139] cbsz:1 blgp:1\n" \
  "v_mfma_f32_16x16x128_f8f6f4 a[140:143], v[144:151], v[120:127], a[140:143] cbsz:1 blgp:1\n" \
  "buffer_load_dwordx4 v[80:83], v166, s[36:39], s75 offen offset:0\n" \
  "s_waitcnt lgkmcnt(2)\n" \
  "v_mfma_f32_16x16x128_f8f6f4 a[144:147], v[152:159], v[96:103], a[144:147] cbsz:1 blgp:1\n" \
  "ds_read_b128 v[136:139], v161 offset:55296\n" \
  "ds_read_b128 v[140:143], v161 offset:56320\n" \
  "v_mfma_f32_16x16x128_f8f6f4 a[148:151], v[152:159], v[104:111], a[148:151] cbsz:1 blgp:1\n" \
  "v_mul_f32 v212, s78, v212\n" \
  "v_mul_f32 v213, s78, v213\n" \
  "v_mul_f32 v214, s78, v214\n" \
  "v_mul_f32 v215, s78, v215\n" \
  "v_mfma_f32_16x16x128_f8f6f4 a[152:155], v[152:159], v[112:119], a[152:155] cbsz:1 blgp:1\n" \
  "buffer_store_dwordx4 v[212:215], v167, s[40:43], s51 offen offset:64 nt\n" \
  "v_mfma_f32_16x16x128_f8f6f4 a[156:159], v[152:159], v[120:127], a[156:159] cbsz:1 blgp:1\n" \
  "s_waitcnt lgkmcnt(2)\n" \
  "v_mfma_f32_16x16x128_f8f6f4 a[160:163], v[128:135], v[96:103], a[160:163] cbsz:1 blgp:1\n" \
  "ds_read_b128 v[144:147], v161 offset:57344\n" \
  "ds_read_b128 v[148:151], v161 offset:58368\n" \
  "v_mfma_f32_16x16x128_f8f6f4 a[164:167], v[128:135], v[104:111], a[164:167] cbsz:1 blgp:1\n" \
  "buffer_load_dwordx4 v[84:87], v166, s[36:39], s75 offen offset:1024\n" \
  "v_mfma_f32_16x16x128_f8f6f4 a[168:171], v[128:135], v[112:119], a[168:171] cbsz:1 blgp:1\n" \
  "buffer_load_dwordx4 v[88:91], v166, s[36:39], s75 offen offset:2048\n" \
  "v_mfma_f32_16x16x128_f8f6f4 a[172:175], v[128:135], v[120:127], a[172:175] cbsz:1 blgp:1\n" \
  "s_waitcnt lgkmcnt(2)\n" \
  "v_mfma_f32_16x16x128_f8f6f4 a[176:179], v[136:143], v[96:103], a[176:179] cbsz:1 blgp:1\n" \
  "ds_read_b128 v[152:155], v161 offset:59392\n" \
  "ds_read_b128 v[156:159], v161 offset:60416\n" \
  "v_mfma_f32_16x16x128_f8f6f4 a[180:183], v[136:143], v[104:111], a[180:183] cbsz:1 blgp:1\n" \
  "buffer_load_dwordx4 v[92:95], v166, s[36:39], s75 offen offset:3072\n" \
  "v_mfma_f32_16x16x128_f8f6f4 a[184:187], v[136:143], v[112:119], a[184:187] cbsz:1 blgp:1\n" \
  "v_mfma_f32_16x16x128_f8f6f4 a[188:191], v[136:143], v[120:127], a[188:191] cbsz:1 blgp:1\n" \
  "s_waitcnt vmcnt(26) lgkmcnt(0)\n" \
  "s_barrier\n" \
  "v_mfma_f32_16x16x128_f8f6f4 a[192:195], v[144:151], v[96:103], a[192:195] cbsz:1 blgp:1\n" \
  "ds_read_b128 v[128:131], v160 offset:16384\n" \
  "ds_read_b128 v[132:135], v160 offset:17408\n" \
  "v_mfma_f32_16x16x128_f8f6f4 a[196:199], v[144:151], v[104:111], a[196:199] cbsz:1 blgp:1\n" \
  "s_mov_b32 s42, s72\n" \
  "s_mov_b32 s48, s71\n" \
  "s_add_u32 s49, s71, 1048576\n" \
  "s_add_u32 s50, s71, 2097152\n" \
  "v_mfma_f32_16x16x128_f8f6f4 a[200:203], v[144:151], v[112:119], a[200:203] cbsz:1 blgp:1\n" \
  "s_add_u32 s51, s71, 3145728\n" \
  "s_cmp_eq_u32 s70, 64\n" \
  "s_cbranch_scc1 Lp58_%=\n" \
  "Lq58_%=:\n" \
  "v_mfma_f32_16x16x128_f8f6f4 a[204:207], v[144:151], v[120:127], a[204:207] cbsz:1 blgp:1\n" \
  "v_accvgpr_read_b32 v184, a96\n" \
  "v_accvgpr_read_b32 v185, a97\n" \
  "v_accvgpr_read_b32 v186, a98\n" \
  "v_accvgpr_read_b32 v187, a99\n" \
  "v_mfma_f32_16x16x128_f8f6f4 a[208:211], v[152:159], v[96:103], a[208:211] cbsz:1 blgp:1\n" \
  "ds_read_b128 v[136:139], v160 offset:18432\n" \
  "ds_read_b128 v[140:143], v160 offset:19456\n" \
  "v_mfma_f32_16x16x128_f8f6f4 a[212:215], v[152:159], v[104:111], a[212:215] cbsz:1 blgp:1\n" \
  "v_accvgpr_read_b32 v200, a112\n" \
  "v_accvgpr_read_b32 v201, a113\n" \
  "v_accvgpr_read_b32 v202, a114\n" \
  "v_accvgpr_read_b32 v203, a115\n" \
  "v_mfma_f32_16x16x128_f8f6f4 a[216:219], v[152:159], v[112:119], a[216:219] cbsz:1 blgp:1\n" \
  "v_accvgpr_read_b32 v188, a100\n" \
  "v_accvgpr_read_b32 v189, a101\n" \
  "v_accvgpr_read_b32 v190, a102\n" \
  "v_accvgpr_read_b32 v191, a103\n" \
  "v_mfma_f32_16x16x128_f8f6f4 a[220:223], v[152:159], v[120:127], a[220:223] cbsz:1 blgp:1\n" \
  "v_accvgpr_read_b32 v204, a116\n" \
  "v_accvgpr_read_b32 v205, a117\n" \
  "v_accvgpr_read_b32 v206, a118\n" \
  "v_accvgpr_read_b32 v207, a119\n" \
  "s_waitcnt lgkmcnt(2)\n" \
  "v_mfma_f32_16x16x128_f8f6f4 a[128:131], v[128:135], v[0:7], a[128:131] cbsz:1 blgp:1\n" \
  "ds_read_b128 v[144:147], v160 offset:20480\n" \
  "ds_read_b128 v[148:151], v160 offset:21504\n" \
  "v_mfma_f32_16x16x128_f8f6f4 a[132:135], v[128:135], v[8:15], a[132:135] cbsz:1 blgp:1\n" \
  "v_accvgpr_read_b32 v192, a104\n" \
  "v_accvgpr_read_b32 v193, a105\n" \
  "v_accvgpr_read_b32 v194, a106\n" \
  "v_accvgpr_read_b32 v195, a107\n" \
  "v_mfma_f32_16x16x128_f8f6f4 a[136:139], v[128:135], v[16:23], a[136:139] cbsz:1 blgp:1\n" \
  "v_accvgpr_read_b32 v208, a120\n" \
  "v_accvgpr_read_b32 v209, a121\n" \
  "v_accvgpr_read_b32 v210, a122\n" \
  "v_accvgpr_read_b32 v211, a123\n" \
  "v_mfma_f32_16x16x128_f8f6f4 a[140:143], v[128:135], v[24:31], a[140:143] cbsz:1 blgp:1\n" \
  "v_accvgpr_read_b32 v196, a108\n" \
  "v_accvgpr_read_b32 v197, a109\n" \
  "v_accvgpr_read_b32 v198, a110\n" \
  "v_accvgpr_read_b32 v199, a111\n" \
  "s_waitcnt lgkmcnt(2)\n" \
  "v_mfma_f32_16x16x128_f8f6f4 a[144:147], v[136:143], v[0:7], a[144:147] cbsz:1 blgp:1\n" \
  "ds_read_b128 v[152:155], v160 offset:22528\n" \
  "ds_read_b128 v[156:159], v160 offset:23552\n" \
  "v_mfma_f32_16x16x128_f8f6f4 a[148:151], v[136:143], v[8:15], a[148:151] cbsz:1 blgp:1\n" \
  "v_accvgpr_read_b32 v212, a124\n" \
  "v_accvgpr_read_b32 v213, a125\n" \
  "v_accvgpr_read_b32 v214, a126\n" \
  "v_accvgpr_read_b32 v215, a127\n" \
  "s_add_u32 m0, s69, 114688\n" \
  "s_nop 0\n" \
  "buffer_load_dwordx4 v163, s[36:39], s60 offen lds\n" \
  "v_mfma_f32_16x16x128_f8f6f4 a[152:155], v[136:143], v[16:23], a[152:155] cbsz:1 blgp:1\n" \
  "s_add_u32 m0, s69, 118784\n" \
  "s_nop 0\n" \
  "buffer_load_dwordx4 v163, s[36:39], s61 offen lds\n" \
  "v_readlane_b32 s67, v175, s70\n" \
  "v_readlane_b32 s76, v169, s70\n" \
  "v_mfma_f32_16x16x128_f8f6f4 a[156:159], v[136:143], v[24:31], a[156:159] cbsz:1 blgp:1\n" \
  "s_add_u32 m0, s69, 122880\n" \
  "s_nop 0\n" \
  "buffer_load_dwordx4 v163, s[36:39], s62 offen lds\n" \
  "s_add_u32 s70, s70, 1\n" \
  "s_nop 0\n" \
  "v_add_u32 v165, s76, v162\n" \
  "v_add_u32 v166, 4096, v165\n" \
  "s_cmp_gt_u32 s67, s68\n" \
  "s_cbranch_scc1 Lp59_%=\n" \
  "Lq59_%=:\n" \
  "s_waitcnt lgkmcnt(2)\n" \
  "v_mfma_f32_16x16x128_f8f6f4 a[160:163], v[144:151], v[0:7], a[160:163] cbsz:1 blgp:1\n" \
  "ds_read_b128 v[128:131], v160 offset:24576\n" \
  "ds_read_b128 v[132:135], v160 offset:25600\n" \
  "v_mfma_f32_16x16x128_f8f6f4 a[164:167], v[144:151], v[8:15], a[164:167] cbsz:1 blgp:1\n" \
  "buffer_load_dwordx4 v[96:99], v163, s[36:39], s75 offen offset:0\n" \
  "v_mfma_f32_16x16x128_f8f6f4 a[168:171], v[144:151], v[16:23], a[168:171] cbsz:1 blgp:1\n" \
  "buffer_load_dwordx4 v[100:103], v163, s[36:39], s75 offen offset:1024\n" \
  "v_mfma_f32_16x16x128_f8f6f4 a[172:175], v[144:151], v[24:31], a[172:175] cbsz:1 blgp:1\n" \
  "buffer_load_dwordx4 v[104:107], v163, s[36:39], s75 offen offset:2048\n" \
  "s_waitcnt lgkmcnt(2)\n" \
  "v_mfma_f32_16x16x128_f8f6f4 a[176:179], v[152:159], v[0:7], a[176:179] cbsz:1 blgp:1\n" \
  "ds_read_b128 v[136:139], v160 offset:26624\n" \
  "ds_read_b128 v[140:143], v160 offset:27648\n" \
  "v_mfma_f32_16x16x128_f8f6f4 a[180:183], v[152:159], v[8:15], a[180:183] cbsz:1 blgp:1\n" \
  "buffer_load_dwordx4 v[108:111], v163, s[36:39], s75 offen offset:3072\n" \
  "v_mfma_f32_16x16x128_f8f6f4 a[184:187], v[152:159], v[16:23], a[184:187] cbsz:1 blgp:1\n" \
  "buffer_load_dwordx4 v[112:115], v164, s[36:39], s75 offen offset:0\n" \
  "v_mfma_f32_16x16x128_f8f6f4 a[188:191], v[152:159], v[24:31], a[188:191] cbsz:1 blgp:1\n" \
  "buffer_load_dwordx4 v[116:119], v164, s[36:39], s75 offen offset:1024\n" \
  "v_mul_f32 v184, s78, v184\n" \
  "v_mul_f32 v185, s78, v185\n" \
  "v_mul_f32 v186, s78, v186\n" \
  "v_mul_f32 v187, s78, v187\n" \
  "buffer_store_dwordx4 v[184:187], v167, s[40:43], s48 offen offset:0 nt\n" \
  "buffer_load_dwordx4 v[120:123], v164, s[36:39], s75 offen offset:2048\n" \
  "v_mul_f32 v200, s78, v200\n" \
  "v_mul_f32 v201, s78, v201\n" \
  "v_mul_f32 v202, s78, v202\n" \
  "v_mul_f32 v203, s78, v203\n" \
  "buffer_store_dwordx4 v[200:203], v167, s[40:43], s48 offen offset:64 nt\n" \
  "buffer_load_dwordx4 v[124:127], v164, s[36:39], s75 offen offset:3072\n" \
  "s_waitcnt vmcnt(26) lgkmcnt(0)\n" \
  "s_barrier\n" \
  "v_mfma_f32_16x16x128_f8f6f4 a[192:195], v[128:135], v[0:7], a[192:195] cbsz:1 blgp:1\n" \
  "ds_read_b128 v[144:147], v160 offset:49152\n" \
  "ds_read_b128 v[148:151], v160 offset:50176\n" \
  "v_mfma_f32_16x16x128_f8f6f4 a[196:199], v[128:135], v[8:15], a[196:199] cbsz:1 blgp:1\n" \
  "s_add_u32 m0, s69, 20480\n" \
  "s_nop 0\n" \
  "buffer_load_dwordx4 v165, s[36:39], s61 offen lds\n" \
  "v_mfma_f32_16x16x128_f8f6f4 a[200:203], v[128:135], v[16:23], a[200:203] cbsz:1 blgp:1\n" \
  "s_add_u32 m0, s69, 24576\n" \
  "s_nop 0\n" \
  "buffer_load_dwordx4 v165, s[36:39], s62 offen lds\n" \
  "v_readlane_b32 s67, v175, s70\n" \
  "v_readlane_b32 s76, v169, s70\n" \
  "v_mfma_f32_16x16x128_f8f6f4 a[204:207], v[128:135], v[24:31], a[204:207] cbsz:1 blgp:1\n" \
  "s_add_u32 s70, s70, 1\n" \
  "s_nop 0\n" \
  "v_add_u32 v163, s76, v162\n" \
  "v_add_u32 v164, 4096, v163\n" \
  "s_cmp_gt_u32 s67, s68\n" \
  "s_cbranch_scc1 Lp60_%=\n" \
  "Lq60_%=:\n" \
  "v_mfma_f32_16x16x128_f8f6f4 a[208:211], v[136:143], v[0:7], a[208:211] cbsz:1 blgp:1\n" \
  "ds_read_b128 v[152:155], v160 offset:51200\n" \
  "ds_read_b128 v[156:159], v160 offset:52224\n" \
  "v_mfma_f32_16x16x128_f8f6f4 a[212:215], v[136:143], v[8:15], a[212:215] cbsz:1 blgp:1\n" \
  "v_mfma_f32_16x16x128_f8f6f4 a[216:219], v[136:143], v[16:23], a[216:219] cbsz:1 blgp:1\n" \
  "v_mfma_f32_16x16x128_f8f6f4 a[220:223], v[136:143], v[24:31], a[220:223] cbsz:1 blgp:1\n" \
  "buffer_load_dwordx4 v[0:3], v165, s[36:39], s75 offen offset:0\n" \
  "s_waitcnt lgkmcnt(2)\n" \
  "v_mfma_f32_16x16x128_f8f6f4 a[128:131], v[144:151], v[32:39], a[128:131] cbsz:1 blgp:1\n" \
  "ds_read_b128 v[128:131], v160 offset:53248\n" \
  "ds_read_b128 v[132:135], v160 offset:54272\n" \
  "v_mfma_f32_16x16x128_f8f6f4 a[132:135], v[144:151], v[40:47], a[132:135] cbsz:1 blgp:1\n" \
  "buffer_load_dwordx4 v[4:7], v165, s[36:39], s75 offen offset:1024\n" \
  "v_mul_f32 v188, s78, v188\n" \
  "v_mul_f32 v189, s78, v189\n" \
  "v_mul_f32 v190, s78, v190\n" \
  "v_mul_f32 v191, s78, v191\n" \
  "v_mfma_f32_16x16x128_f8f6f4 a[136:139], v[144:151], v[48:55], a[136:139] cbsz:1 blgp:1\n" \
  "buffer_store_dwordx4 v[188:191], v167, s[40:43], s49 offen offset:0 nt\n" \
  "v_mfma_f32_16x16x128_f8f6f4 a[140:143], v[144:151], v[56:63], a[140:143] cbsz:1 blgp:1\n" \
  "buffer_load_dwordx4 v[8:11], v165, s[36:39], s75 offen offset:2048\n" \
  "s_waitcnt lgkmcnt(2)\n" \
  "v_mfma_f32_16x16x128_f8f6f4 a[144:147], v[152:159], v[32:39], a[144:147] cbsz:1 blgp:1\n" \
  "ds_read_b128 v[136:139], v160 offset:55296\n" \
  "ds_read_b128 v[140:143], v160 offset:56320\n" \
  "v_mfma_f32_16x16x128_f8f6f4 a[148:151], v[152:159], v[40:47], a[148:151] cbsz:1 blgp:1\n" \
  "buffer_load_dwordx4 v[12:15], v165, s[36:39], s75 offen offset:3072\n" \
  "v_mfma_f32_16x16x128_f8f6f4 a[152:155], v[152:159], v[48:55], a[152:155] cbsz:1 blgp:1\n" \
  "buffer_load_dwordx4 v[16:19], v166, s[36:39], s75 offen offset:0\n" \
  "v_mfma_f32_16x16x128_f8f6f4 a[156:159], v[152:159], v[56:63], a[156:159] cbsz:1 blgp:1\n" \
  "v_mul_f32 v204, s78, v204\n" \
  "v_mul_f32 v205, s78, v205\n" \
  "v_mul_f32 v206, s78, v206\n" \
  "v_mul_f32 v207, s78, v207\n" \
  "s_waitcnt lgkmcnt(2)\n" \
  "v_mfma_f32_16x16x128_f8f6f4 a[160:163], v[128:135], v[32:39], a[160:163] cbsz:1 blgp:1\n" \
  "ds_read_b128 v[144:147], v160 offset:57344\n" \
  "ds_read_b128 v[148:151], v160 offset:58368\n" \
  "v_mfma_f32_16x16x128_f8f6f4 a[164:167], v[128:135], v[40:47], a[164:167] cbsz:1 blgp:1\n" \
  "buffer_store_dwordx4 v[204:207], v167, s[40:43], s49 offen offset:64 nt\n" \
  "v_mfma_f32_16x16x128_f8f6f4 a[168:171], v[128:135], v[48:55], a[168:171] cbsz:1 blgp:1\n" \
  "buffer_load_dwordx4 v[20:23], v166, s[36:39], s75 offen offset:1024\n" \
  "v_mfma_f32_16x16x128_f8f6f4 a[172:175], v[128:135], v[56:63], a[172:175] cbsz:1 blgp:1\n" \
  "buffer_load_dwordx4 v[24:27], v166, s[36:39], s75 offen offset:2048\n" \
  "s_waitcnt lgkmcnt(2)\n" \
  "v_mfma_f32_16x16x128_f8f6f4 a[176:179], v[136:143], v[32:39], a[176:179] cbsz:1 blgp:1\n" \
  "ds_read_b128 v[152:155], v160 offset:59392\n" \
  "ds_read_b128 v[156:159], v160 offset:60416\n" \
  "v_mfma_f32_16x16x128_f8f6f4 a[180:183], v[136:143], v[40:47], a[180:183] cbsz:1 blgp:1\n" \
  "buffer_load_dwordx4 v[28:31], v166, s[36:39], s75 offen offset:3072\n" \
  "v_mfma_f32_16x16x128_f8f6f4 a[184:187], v[136:143], v[48:55], a[184:187] cbsz:1 blgp:1\n" \
  "v_mfma_f32_16x16x128_f8f6f4 a[188:191], v[136:143], v[56:63], a[188:191] cbsz:1 blgp:1\n" \
  "s_waitcnt vmcnt(25) lgkmcnt(0)\n" \
  "s_barrier\n" \
  "v_mfma_f32_16x16x128_f8f6f4 a[192:195], v[144:151], v[32:39], a[192:195] cbsz:1 blgp:1\n" \
  "ds_read_b128 v[128:131], v161 offset:16384\n" \
  "ds_read_b128 v[132:135], v161 offset:17408\n" \
  "v_mfma_f32_16x16x128_f8f6f4 a[196:199], v[144:151], v[40:47], a[196:199] cbsz:1 blgp:1\n" \
  "s_add_u32 m0, s69, 53248\n" \
  "s_nop 0\n" \
  "buffer_load_dwordx4 v163, s[36:39], s61 offen lds\n" \
  "v_mfma_f32_16x16x128_f8f6f4 a[200:203], v[144:151], v[48:55], a[200:203] cbsz:1 blgp:1\n" \
  "s_add_u32 m0, s69, 57344\n" \
  "s_nop 0\n" \
  "buffer_load_dwordx4 v163, s[36:39], s62 offen lds\n" \
  "v_readlane_b32 s67, v175, s70\n" \
  "v_readlane_b32 s76, v169, s70\n" \
  "v_mfma_f32_16x16x128_f8f6f4 a[204:207], v[144:151], v[56:63], a[204:207] cbsz:1 blgp:1\n" \
  "s_add_u32 s70, s70, 1\n" \
  "s_nop 0\n" \
  "v_add_u32 v165, s76, v162\n" \
  "v_add_u32 v166, 4096, v165\n" \
  "s_cmp_gt_u32 s67, s68\n" \
  "s_cbranch_scc1 Lp61_%=\n" \
  "Lq61_%=:\n" \
  "v_mfma_f32_16x16x128_f8f6f4 a[208:211], v[152:159], v[32:39], a[208:211] cbsz:1 blgp:1\n" \
  "ds_read_b128 v[136:139], v161 offset:18432\n" \
  "ds_read_b128 v[140:143], v161 offset:19456\n" \
  "v_mfma_f32_16x16x128_f8f6f4 a[212:215], v[152:159], v[40:47], a[212:215] cbsz:1 blgp:1\n" \
  "v_mfma_f32_16x16x128_f8f6f4 a[216:219], v[152:159], v[48:55], a[216:219] cbsz:1 blgp:1\n" \
  "v_mfma_f32_16x16x128_f8f6f4 a[220:223], v[152:159], v[56:63], a[220:223] cbsz:1 blgp:1\n" \
  "buffer_load_dwordx4 v[32:35], v163, s[36:39], s75 offen offset:0\n" \
  "s_waitcnt lgkmcnt(2)\n" \
  "v_mfma_f32_16x16x128_f8f6f4 a[128:131], v[128:135], v[64:71], a[128:131] cbsz:1 blgp:1\n" \
  "ds_read_b128 v[144:147], v161 offset:20480\n" \
  "ds_read_b128 v[148:151], v161 offset:21504\n" \
  "v_mfma_f32_16x16x128_f8f6f4 a[132:135], v[128:135], v[72:79], a[132:135] cbsz:1 blgp:1\n" \
  "buffer_load_dwordx4 v[36:39], v163, s[36:39], s75 offen offset:1024\n" \
  "v_mul_f32 v192, s78, v192\n" \
  "v_mul_f32 v193, s78, v193\n" \
  "v_mul_f32 v194, s78, v194\n" \
  "v_mul_f32 v195, s78, v195\n" \
  "v_mfma_f32_16x16x128_f8f6f4 a[136:139], v[128:135], v[80:87], a[136:139] cbsz:1 blgp:1\n" \
  "buffer_store_dwordx4 v[192:195], v167, s[40:43], s50 offen offset:0 nt\n" \
  "v_mfma_f32_16x16x128_f8f6f4 a[140:143], v[128:135], v[88:95], a[140:143] cbsz:1 blgp:1\n" \
  "buffer_load_dwordx4 v[40:43], v163, s[36:39], s75 offen offset:2048\n" \
  "s_waitcnt lgkmcnt(2)\n" \
  "v_mfma_f32_16x16x128_f8f6f4 a[144:147], v[136:143], v[64:71], a[144:147] cbsz:1 blgp:1\n" \
  "ds_read_b128 v[152:155], v161 offset:22528\n" \
  "ds_read_b128 v[156:159], v161 offset:23552\n" \
  "v_mfma_f32_16x16x128_f8f6f4 a[148:151], v[136:143], v[72:79], a[148:151] cbsz:1 blgp:1\n" \
  "buffer_load_dwordx4 v[44:47], v163, s[36:39], s75 offen offset:3072\n" \
  "v_mfma_f32_16x16x128_f8f6f4 a[152:155], v[136:143], v[80:87], a[152:155] cbsz:1 blgp:1\n" \
  "buffer_load_dwordx4 v[48:51], v164, s[36:39], s75 offen offset:0\n" \
  "v_mfma_f32_16x16x128_f8f6f4 a[156:159], v[136:143], v[88:95], a[156:159] cbsz:1 blgp:1\n" \
  "v_mul_f32 v208, s78, v208\n" \
  "v_mul_f32 v209, s78, v209\n" \
  "v_mul_f32 v210, s78, v210\n" \
  "v_mul_f32 v211, s78, v211\n" \
  "s_waitcnt lgkmcnt(2)\n" \
  "v_mfma_f32_16x16x128_f8f6f4 a[160:163], v[144:151], v[64:71], a[160:163] cbsz:1 blgp:1\n" \
  "ds_read_b128 v[128:131], v161 offset:24576\n" \
  "ds_read_b128 v[132:135], v161 offset:25600\n" \
  "v_mfma_f32_16x16x128_f8f6f4 a[164:167], v[144:151], v[72:79], a[164:167] cbsz:1 blgp:1\n" \
  "buffer_store_dwordx4 v[208:211], v167, s[40:43], s50 offen offset:64 nt\n" \
  "v_mfma_f32_16x16x128_f8f6f4 a[168:171], v[144:151], v[80:87], a[168:171] cbsz:1 blgp:1\n" \
  "buffer_load_dwordx4 v[52:55], v164, s[36:39], s75 offen offset:1024\n" \
  "v_mfma_f32_16x16x128_f8f6f4 a[172:175], v[144:151], v[88:95], a[172:175] cbsz:1 blgp:1\n" \
  "buffer_load_dwordx4 v[56:59], v164, s[36:39], s75 offen offset:2048\n" \
  "s_waitcnt lgkmcnt(2)\n" \
  "v_mfma_f32_16x16x128_f8f6f4 a[176:179], v[152:159], v[64:71], a[176:179] cbsz:1 blgp:1\n" \
  "ds_read_b128 v[136:139], v161 offset:26624\n" \
  "ds_read_b128 v[140:143], v161 offset:27648\n" \
  "v_mfma_f32_16x16x128_f8f6f4 a[180:183], v[152:159], v[72:79], a[180:183] cbsz:1 blgp:1\n" \
  "buffer_load_dwordx4 v[60:63], v164, s[36:39], s75 offen offset:3072\n" \
  "v_mfma_f32_16x16x128_f8f6f4 a[184:187], v[152:159], v[80:87], a[184:187] cbsz:1 blgp:1\n" \
  "v_mfma_f32_16x16x128_f8f6f4 a[188:191], v[152:159], v[88:95], a[188:191] cbsz:1 blgp:1\n" \
  "s_waitcnt vmcnt(24) lgkmcnt(0)\n" \
  "s_barrier\n" \
  "v_mfma_f32_16x16x128_f8f6f4 a[192:195], v[128:135], v[64:71], a[192:195] cbsz:1 blgp:1\n" \
  "ds_read_b128 v[144:147], v161 offset:49152\n" \
  "ds_read_b128 v[148:151], v161 offset:50176\n" \
  "v_mfma_f32_16x16x128_f8f6f4 a[196:199], v[128:135], v[72:79], a[196:199] cbsz:1 blgp:1\n" \
  "s_add_u32 m0, s69, 86016\n" \
  "s_nop 0\n" \
  "buffer_load_dwordx4 v165, s[36:39], s61 offen lds\n" \
  "v_mfma_f32_16x16x128_f8f6f4 a[200:203], v[128:135], v[80:87], a[200:203] cbsz:1 blgp:1\n" \
  "s_add_u32 m0, s69, 90112\n" \
  "s_nop 0\n" \
  "buffer_load_dwordx4 v165, s[36:39], s62 offen lds\n" \
  "v_readlane_b32 s67, v175, s70\n" \
  "v_readlane_b32 s76, v169, s70\n" \
  "v_readlane_b32 s71, v181, s70\n" \
  "v_mfma_f32_16x16x128_f8f6f4 a[204:207], v[128:135], v[88:95], a[204:207] cbsz:1 blgp:1\n" \
  "s_add_u32 s70, s70, 1\n" \
  "s_nop 0\n" \
  "v_add_u32 v163, s76, v162\n" \
  "v_add_u32 v164, 4096, v163\n" \
  "s_cmp_gt_u32 s67, s68\n" \
  "s_cbranch_scc1 Lp62_%=\n" \
  "Lq62_%=:\n" \
  "v_mfma_f32_16x16x128_f8f6f4 a[208:211], v[136:143], v[64:71], a[208:211] cbsz:1 blgp:1\n" \
  "ds_read_b128 v[152:155], v161 offset:51200\n" \
  "ds_read_b128 v[156:159], v161 offset:52224\n" \
  "v_mfma_f32_16x16x128_f8f6f4 a[212:215], v[136:143], v[72:79], a[212:215] cbsz:1 blgp:1\n" \
  "v_mfma_f32_16x16x128_f8f6f4 a[216:219], v[136:143], v[80:87], a[216:219] cbsz:1 blgp:1\n" \
  "v_mfma_f32_16x16x128_f8f6f4 a[220:223], v[136:143], v[88:95], a[220:223] cbsz:1 blgp:1\n" \
  "buffer_load_dwordx4 v[64:67], v165, s[36:39], s75 offen offset:0\n" \
  "s_waitcnt lgkmcnt(2)\n" \
  "v_mfma_f32_16x16x128_f8f6f4 a[128:131], v[144:151], v[96:103], a[128:131] cbsz:1 blgp:1\n" \
  "ds_read_b128 v[128:131], v161 offset:53248\n" \
  "ds_read_b128 v[132:135], v161 offset:54272\n" \
  "v_mfma_f32_16x16x128_f8f6f4 a[132:135], v[144:151], v[104:111], a[132:135] cbsz:1 blgp:1\n" \
  "buffer_load_dwordx4 v[68:71], v165, s[36:39], s75 offen offset:1024\n" \
  "v_mul_f32 v196, s78, v196\n" \
  "v_mul_f32 v197, s78, v197\n" \
  "v_mul_f32 v198, s78, v198\n" \
  "v_mul_f32 v199, s78, v199\n" \
  "v_mfma_f32_16x16x128_f8f6f4 a[136:139], v[144:151], v[112:119], a[136:139] cbsz:1 blgp:1\n" \
  "buffer_store_dwordx4 v[196:199], v167, s[40:43], s51 offen offset:0 nt\n" \
  "v_mfma_f32_16x16x128_f8f6f4 a[140:143], v[144:151], v[120:127], a[140:143] cbsz:1 blgp:1\n" \
  "buffer_load_dwordx4 v[72:75], v165, s[36:39], s75 offen offset:2048\n" \
  "s_waitcnt lgkmcnt(2)\n" \
  "v_mfma_f32_16x16x128_f8f6f4 a[144:147], v[152:159], v[96:103], a[144:147] cbsz:1 blgp:1\n" \
  "ds_read_b128 v[136:139], v161 offset:55296\n" \
  "ds_read_b128 v[140:143], v161 offset:56320\n" \
  "v_mfma_f32_16x16x128_f8f6f4 a[148:151], v[152:159], v[104:111], a[148:151] cbsz:1 blgp:1\n" \
  "buffer_load_dwordx4 v[76:79], v165, s[36:39], s75 offen offset:3072\n" \
  "v_mfma_f32_16x16x128_f8f6f4 a[152:155], v[152:159], v[112:119], a[152:155] cbsz:1 blgp:1\n" \
  "buffer_load_dwordx4 v[80:83], v166, s[36:39], s75 offen offset:0\n" \
  "v_mfma_f32_16x16x128_f8f6f4 a[156:159], v[152:159], v[120:127], a[156:159] cbsz:1 blgp:1\n" \
  "v_mul_f32 v212, s78, v212\n" \
  "v_mul_f32 v213, s78, v213\n" \
  "v_mul_f32 v214, s78, v214\n" \
  "v_mul_f32 v215, s78, v215\n" \
  "s_waitcnt lgkmcnt(2)\n" \
  "v_mfma_f32_16x16x128_f8f6f4 a[160:163], v[128:135], v[96:103], a[160:163] cbsz:1 blgp:1\n" \
  "ds_read_b128 v[144:147], v161 offset:57344\n" \
  "ds_read_b128 v[148:151], v161 offset:58368\n" \
  "v_mfma_f32_16x16x128_f8f6f4 a[164:167], v[128:135], v[104:111], a[164:167] cbsz:1 blgp:1\n" \
  "buffer_store_dwordx4 v[212:215], v167, s[40:43], s51 offen offset:64 nt\n" \
  "v_mfma_f32_16x16x128_f8f6f4 a[168:171], v[128:135], v[112:119], a[168:171] cbsz:1 blgp:1\n" \
  "buffer_load_dwordx4 v[84:87], v166, s[36:39], s75 offen offset:1024\n" \
  "v_mfma_f32_16x16x128_f8f6f4 a[172:175], v[128:135], v[120:127], a[172:175] cbsz:1 blgp:1\n" \
  "buffer_load_dwordx4 v[88:91], v166, s[36:39], s75 offen offset:2048\n" \
  "s_waitcnt lgkmcnt(2)\n" \
  "v_mfma_f32_16x16x128_f8f6f4 a[176:179], v[136:143], v[96:103], a[176:179] cbsz:1 blgp:1\n" \
  "ds_read_b128 v[152:155], v161 offset:59392\n" \
  "ds_read_b128 v[156:159], v161 offset:60416\n" \
  "v_mfma_f32_16x16x128_f8f6f4 a[180:183], v[136:143], v[104:111], a[180:183] cbsz:1 blgp:1\n" \
  "buffer_load_dwordx4 v[92:95], v166, s[36:39], s75 offen offset:3072\n" \
  "v_mfma_f32_16x16x128_f8f6f4 a[184:187], v[136:143], v[112:119], a[184:187] cbsz:1 blgp:1\n" \
  "v_mfma_f32_16x16x128_f8f6f4 a[188:191], v[136:143], v[120:127], a[188:191] cbsz:1 blgp:1\n" \
  "s_waitcnt vmcnt(24) lgkmcnt(0)\n" \
  "s_barrier\n" \
  "v_mfma_f32_16x16x128_f8f6f4 a[192:195], v[144:151], v[96:103], a[192:195] cbsz:1 blgp:1\n" \
  "ds_read_b128 v[128:131], v160 offset:20480\n" \
  "ds_read_b128 v[132:135], v160 offset:21504\n" \
  "v_mfma_f32_16x16x128_f8f6f4 a[196:199], v[144:151], v[104:111], a[196:199] cbsz:1 blgp:1\n" \
  "s_mov_b32 s42, s72\n" \
  "s_mov_b32 s48, s71\n" \
  "s_add_u32 s49, s71, 1048576\n" \
  "s_add_u32 s50, s71, 2097152\n" \
  "v_mfma_f32_16x16x128_f8f6f4 a[200:203], v[144:151], v[112:119], a[200:203] cbsz:1 blgp:1\n" \
  "s_add_u32 s51, s71, 3145728\n" \
  "s_cmp_eq_u32 s70, 64\n" \
  "s_cbranch_scc1 Lp63_%=\n" \
  "Lq63_%=:\n" \
  "v_mfma_f32_16x16x128_f8f6f4 a[204:207], v[144:151], v[120:127], a[204:207] cbsz:1 blgp:1\n" \
  "v_accvgpr_read_b32 v184, a128\n" \
  "v_accvgpr_read_b32 v185, a129\n" \
  "v_accvgpr_read_b32 v186, a130\n" \
  "v_accvgpr_read_b32 v187, a131\n" \
  "v_mfma_f32_16x16x128_f8f6f4 a[208:211], v[152:159], v[96:103], a[208:211] cbsz:1 blgp:1\n" \
  "ds_read_b128 v[136:139], v160 offset:22528\n" \
  "ds_read_b128 v[140:143], v160 offset:23552\n" \
  "v_mfma_f32_16x16x128_f8f6f4 a[212:215], v[152:159], v[104:111], a[212:215] cbsz:1 blgp:1\n" \
  "v_accvgpr_read_b32 v200, a144\n" \
  "v_accvgpr_read_b32 v201, a145\n" \
  "v_accvgpr_read_b32 v202, a146\n" \
  "v_accvgpr_read_b32 v203, a147\n" \
  "v_mfma_f32_16x16x128_f8f6f4 a[216:219], v[152:159], v[112:119], a[216:219] cbsz:1 blgp:1\n" \
  "v_accvgpr_read_b32 v188, a132\n" \
  "v_accvgpr_read_b32 v189, a133\n" \
  "v_accvgpr_read_b32 v190, a134\n" \
  "v_accvgpr_read_b32 v191, a135\n" \
  "v_mfma_f32_16x16x128_f8f6f4 a[220:223], v[152:159], v[120:127], a[220:223] cbsz:1 blgp:1\n" \
  "v_accvgpr_read_b32 v204, a148\n" \
  "v_accvgpr_read_b32 v205, a149\n" \
  "v_accvgpr_read_b32 v206, a150\n" \
  "v_accvgpr_read_b32 v207, a151\n" \
  "s_waitcnt lgkmcnt(2)\n" \
  "v_mfma_f32_16x16x128_f8f6f4 a[160:163], v[128:135], v[0:7], a[160:163] cbsz:1 blgp:1\n" \
  "ds_read_b128 v[144:147], v160 offset:24576\n" \
  "ds_read_b128 v[148:151], v160 offset:25600\n" \
  "v_mfma_f32_16x16x128_f8f6f4 a[164:167], v[128:135], v[8:15], a[164:167] cbsz:1 blgp:1\n" \
  "v_accvgpr_read_b32 v192, a136\n" \
  "v_accvgpr_read_b32 v193, a137\n" \
  "v_accvgpr_read_b32 v194, a138\n" \
  "v_accvgpr_read_b32 v195, a139\n" \
  "v_mfma_f32_16x16x128_f8f6f4 a[168:171], v[128:135], v[16:23], a[168:171] cbsz:1 blgp:1\n" \
  "v_accvgpr_read_b32 v208, a152\n" \
  "v_accvgpr_read_b32 v209, a153\n" \
  "v_accvgpr_read_b32 v210, a154\n" \
  "v_accvgpr_read_b32 v211, a155\n" \
  "v_mfma_f32_16x16x128_f8f6f4 a[172:175], v[128:135], v[24:31], a[172:175] cbsz:1 blgp:1\n" \
  "v_accvgpr_read_b32 v196, a140\n" \
  "v_accvgpr_read_b32 v197, a141\n" \
  "v_accvgpr_read_b32 v198, a142\n" \
  "v_accvgpr_read_b32 v199, a143\n" \
  "s_waitcnt lgkmcnt(2)\n" \
  "v_mfma_f32_16x16x128_f8f6f4 a[176:179], v[136:143], v[0:7], a[176:179] cbsz:1 blgp:1\n" \
  "ds_read_b128 v[152:155], v160 offset:26624\n" \
  "ds_read_b128 v[156:159], v160 offset:27648\n" \
  "v_mfma_f32_16x16x128_f8f6f4 a[180:183], v[136:143], v[8:15], a[180:183] cbsz:1 blgp:1\n" \
  "v_accvgpr_read_b32 v212, a156\n" \
  "v_accvgpr_read_b32 v213, a157\n" \
  "v_accvgpr_read_b32 v214, a158\n" \
  "v_accvgpr_read_b32 v215, a159\n" \
  "s_add_u32 m0, s69, 118784\n" \
  "s_nop 0\n" \
  "buffer_load_dwordx4 v163, s[36:39], s61 offen lds\n" \
  "v_mfma_f32_16x16x128_f8f6f4 a[184:187], v[136:143], v[16:23], a[184:187] cbsz:1 blgp:1\n" \
  "s_add_u32 m0, s69, 122880\n" \
  "s_nop 0\n" \
  "buffer_load_dwordx4 v163, s[36:39], s62 offen lds\n" \
  "v_readlane_b32 s67, v175, s70\n" \
  "v_readlane_b32 s76, v169, s70\n" \
  "s_add_u32 s70, s70, 1\n" \
  "s_nop 0\n" \
  "v_add_u32 v165, s76, v162\n" \
  "v_mfma_f32_16x16x128_f8f6f4 a[188:191], v[136:143], v[24:31], a[188:191] cbsz:1 blgp:1\n" \
  "v_add_u32 v166, 4096, v165\n" \
  "s_cmp_gt_u32 s67, s68\n" \
  "s_cbranch_scc1 Lp64_%=\n" \
  "Lq64_%=:\n" \
  "buffer_load_dwordx4 v[96:99], v163, s[36:39], s75 offen offset:0\n" \
  "buffer_load_dwordx4 v[100:103], v163, s[36:39], s75 offen offset:1024\n" \
  "buffer_load_dwordx4 v[104:107], v163, s[36:39], s75 offen offset:2048\n" \
  "buffer_load_dwordx4 v[108:111], v163, s[36:39], s75 offen offset:3072\n" \
  "buffer_load_dwordx4 v[112:115], v164, s[36:39], s75 offen offset:0\n" \
  "buffer_load_dwordx4 v[116:119], v164, s[36:39], s75 offen offset:1024\n" \
  "v_mul_f32 v184, s78, v184\n" \
  "v_mul_f32 v185, s78, v185\n" \
  "v_mul_f32 v186, s78, v186\n" \
  "v_mul_f32 v187, s78, v187\n" \
  "buffer_store_dwordx4 v[184:187], v167, s[40:43], s48 offen offset:0 nt\n" \
  "buffer_load_dwordx4 v[120:123], v164, s[36:39], s75 offen offset:2048\n" \
  "v_mul_f32 v200, s78, v200\n" \
  "v_mul_f32 v201, s78, v201\n" \
  "v_mul_f32 v202, s78, v202\n" \
  "v_mul_f32 v203, s78, v203\n" \
  "buffer_store_dwordx4 v[200:203], v167, s[40:43], s48 offen offset:64 nt\n" \
  "buffer_load_dwordx4 v[124:127], v164, s[36:39], s75 offen offset:3072\n" \
  "s_waitcnt vmcnt(24) lgkmcnt(0)\n" \
  "s_barrier\n" \
  "v_mfma_f32_16x16x128_f8f6f4 a[192:195], v[144:151], v[0:7], a[192:195] cbsz:1 blgp:1\n" \
  "ds_read_b128 v[128:131], v160 offset:53248\n" \
  "ds_read_b128 v[132:135], v160 offset:54272\n" \
  "v_mfma_f32_16x16x128_f8f6f4 a[196:199], v[144:151], v[8:15], a[196:199] cbsz:1 blgp:1\n" \
  "s_add_u32 m0, s69, 24576\n" \
  "s_nop 0\n" \
  "buffer_load_dwordx4 v165, s[36:39], s62 offen lds\n" \
  "v_readlane_b32 s67, v175, s70\n" \
  "v_readlane_b32 s76, v169, s70\n" \
  "s_add_u32 s70, s70, 1\n" \
  "s_nop 0\n" \
  "v_add_u32 v163, s76, v162\n" \
  "v_mfma_f32_16x16x128_f8f6f4 a[200:203], v[144:151], v[16:23], a[200:203] cbsz:1 blgp:1\n" \
  "v_add_u32 v164, 4096, v163\n" \
  "s_cmp_gt_u32 s67, s68\n" \
  "s_cbranch_scc1 Lp65_%=\n" \
  "Lq65_%=:\n" \
  "v_mfma_f32_16x16x128_f8f6f4 a[204:207], v[144:151], v[24:31], a[204:207] cbsz:1 blgp:1\n" \
  "v_mfma_f32_16x16x128_f8f6f4 a[208:211], v[152:159], v[0:7], a[208:211] cbsz:1 blgp:1\n" \
  "ds_read_b128 v[136:139], v160 offset:55296\n" \
  "ds_read_b128 v[140:143], v160 offset:56320\n" \
  "v_mfma_f32_16x16x128_f8f6f4 a[212:215], v[152:159], v[8:15], a[212:215] cbsz:1 blgp:1\n" \
  "v_mfma_f32_16x16x128_f8f6f4 a[216:219], v[152:159], v[16:23], a[216:219] cbsz:1 blgp:1\n" \
  "v_mfma_f32_16x16x128_f8f6f4 a[220:223], v[152:159], v[24:31], a[220:223] cbsz:1 blgp:1\n" \
  "buffer_load_dwordx4 v[0:3], v165, s[36:39], s75 offen offset:0\n" \
  "s_waitcnt lgkmcnt(2)\n" \
  "v_mfma_f32_16x16x128_f8f6f4 a[160:163], v[128:135], v[32:39], a[160:163] cbsz:1 blgp:1\n" \
  "ds_read_b128 v[144:147], v160 offset:57344\n" \
  "ds_read_b128 v[148:151], v160 offset:58368\n" \
  "v_mfma_f32_16x16x128_f8f6f4 a[164:167], v[128:135], v[40:47], a[164:167] cbsz:1 blgp:1\n" \
  "buffer_load_dwordx4 v[4:7], v165, s[36:39], s75 offen offset:1024\n" \
  "v_mul_f32 v188, s78, v188\n" \
  "v_mul_f32 v189, s78, v189\n" \
  "v_mul_f32 v190, s78, v190\n" \
  "v_mul_f32 v191, s78, v191\n" \
  "v_mfma_f32_16x16x128_f8f6f4 a[168:171], v[128:135], v[48:55], a[168:171] cbsz:1 blgp:1\n" \
  "buffer_store_dwordx4 v[188:191], v167, s[40:43], s49 offen offset:0 nt\n" \
  "v_mfma_f32_16x16x128_f8f6f4 a[172:175], v[128:135], v[56:63], a[172:175] cbsz:1 blgp:1\n" \
  "buffer_load_dwordx4 v[8:11], v165, s[36:39], s75 offen offset:2048\n" \
  "s_waitcnt lgkmcnt(2)\n" \
  "v_mfma_f32_16x16x128_f8f6f4 a[176:179], v[136:143], v[32:39], a[176:179] cbsz:1 blgp:1\n" \
  "ds_read_b128 v[152:155], v160 offset:59392\n" \
  "ds_read_b128 v[156:159], v160 offset:60416\n" \
  "v_mfma_f32_16x16x128_f8f6f4 a[180:183], v[136:143], v[40:47], a[180:183] cbsz:1 blgp:1\n" \
  "buffer_load_dwordx4 v[12:15], v165, s[36:39], s75 offen offset:3072\n" \
  "v_mfma_f32_16x16x128_f8f6f4 a[184:187], v[136:143], v[48:55], a[184:187] cbsz:1 blgp:1\n" \
  "buffer_load_dwordx4 v[16:19], v166, s[36:39], s75 offen offset:0\n" \
  "v_mul_f32 v204, s78, v204\n" \
  "v_mul_f32 v205, s78, v205\n" \
  "v_mul_f32 v206, s78, v206\n" \
  "v_mul_f32 v207, s78, v207\n" \
  "v_mfma_f32_16x16x128_f8f6f4 a[188:191], v[136:143], v[56:63], a[188:191] cbsz:1 blgp:1\n" \
  "buffer_store_dwordx4 v[204:207], v167, s[40:43], s49 offen offset:64 nt\n" \
  "buffer_load_dwordx4 v[20:23], v166, s[36:39], s75 offen offset:1024\n" \
  "buffer_load_dwordx4 v[24:27], v166, s[36:39], s75 offen offset:2048\n" \
  "buffer_load_dwordx4 v[28:31], v166, s[36:39], s75 offen offset:3072\n" \
  "s_waitcnt vmcnt(23) lgkmcnt(0)\n" \
  "s_barrier\n" \
  "v_mfma_f32_16x16x128_f8f6f4 a[192:195], v[144:151], v[32:39], a[192:195] cbsz:1 blgp:1\n" \
  "ds_read_b128 v[128:131], v161 offset:20480\n" \
  "ds_read_b128 v[132:135], v161 offset:21504\n" \
  "v_mfma_f32_16x16x128_f8f6f4 a[196:199], v[144:151], v[40:47], a[196:199] cbsz:1 blgp:1\n" \
  "s_add_u32 m0, s69, 57344\n" \
  "s_nop 0\n" \
  "buffer_load_dwordx4 v163, s[36:39], s62 offen lds\n" \
  "v_readlane_b32 s67, v175, s70\n" \
  "v_readlane_b32 s76, v169, s70\n" \
  "s_add_u32 s70, s70, 1\n" \
  "s_nop 0\n" \
  "v_add_u32 v165, s76, v162\n" \
  "v_mfma_f32_16x16x128_f8f6f4 a[200:203], v[144:151], v[48:55], a[200:203] cbsz:1 blgp:1\n" \
  "v_add_u32 v166, 4096, v165\n" \
  "s_cmp_gt_u32 s67, s68\n" \
  "s_cbranch_scc1 Lp66_%=\n" \
  "Lq66_%=:\n" \
  "v_mfma_f32_16x16x128_f8f6f4 a[204:207], v[144:151], v[56:63], a[204:207] cbsz:1 blgp:1\n" \
  "v_mfma_f32_16x16x128_f8f6f4 a[208:211], v[152:159], v[32:39], a[208:211] cbsz:1 blgp:1\n" \
  "ds_read_b128 v[136:139], v161 offset:22528\n" \
  "ds_read_b128 v[140:143], v161 offset:23552\n" \
  "v_mfma_f32_16x16x128_f8f6f4 a[212:215], v[152:159], v[40:47], a[212:215] cbsz:1 blgp:1\n" \
  "v_mfma_f32_16x16x128_f8f6f4 a[216:219], v[152:159], v[48:55], a[216:219] cbsz:1 blgp:1\n" \
  "v_mfma_f32_16x16x128_f8f6f4 a[220:223], v[152:159], v[56:63], a[220:223] cbsz:1 blgp:1\n" \
  "buffer_load_dwordx4 v[32:35], v163, s[36:39], s75 offen offset:0\n" \
  "s_waitcnt lgkmcnt(2)\n" \
  "v_mfma_f32_16x16x128_f8f6f4 a[160:163], v[128:135], v[64:71], a[160:163] cbsz:1 blgp:1\n" \
  "ds_read_b128 v[144:147], v161 offset:24576\n" \
  "ds_read_b128 v[148:151], v161 offset:25600\n" \
  "v_mfma_f32_16x16x128_f8f6f4 a[164:167], v[128:135], v[72:79], a[164:167] cbsz:1 blgp:1\n" \
  "buffer_load_dwordx4 v[36:39], v163, s[36:39], s75 offen offset:1024\n" \
  "v_mul_f32 v192, s78, v192\n" \
  "v_mul_f32 v193, s78, v193\n" \
  "v_mul_f32 v194, s78, v194\n" \
  "v_mul_f32 v195, s78, v195\n" \
  "v_mfma_f32_16x16x128_f8f6f4 a[168:171], v[128:135], v[80:87], a[168:171] cbsz:1 blgp:1\n" \
  "buffer_store_dwordx4 v[192:195], v167, s[40:43], s50 offen offset:0 nt\n" \
  "v_mfma_f32_16x16x128_f8f6f4 a[172:175], v[128:135], v[88:95], a[172:175] cbsz:1 blgp:1\n" \
  "buffer_load_dwordx4 v[40:43], v163, s[36:39], s75 offen offset:2048\n" \
  "s_waitcnt lgkmcnt(2)\n" \
  "v_mfma_f32_16x16x128_f8f6f4 a[176:179], v[136:143], v[64:71], a[176:179] cbsz:1 blgp:1\n" \
  "ds_read_b128 v[152:155], v161 offset:26624\n" \
  "ds_read_b128 v[156:159], v161 offset:27648\n" \
  "v_mfma_f32_16x16x128_f8f6f4 a[180:183], v[136:143], v[72:79], a[180:183] cbsz:1 blgp:1\n" \
  "buffer_load_dwordx4 v[44:47], v163, s[36:39], s75 offen offset:3072\n" \
  "v_mfma_f32_16x16x128_f8f6f4 a[184:187], v[136:143], v[80:87], a[184:187] cbsz:1 blgp:1\n" \
  "buffer_load_dwordx4 v[48:51], v164, s[36:39], s75 offen offset:0\n" \
  "v_mul_f32 v208, s78, v208\n" \
  "v_mul_f32 v209, s78, v209\n" \
  "v_mul_f32 v210, s78, v210\n" \
  "v_mul_f32 v211, s78, v211\n" \
  "v_mfma_f32_16x16x128_f8f6f4 a[188:191], v[136:143], v[88:95], a[188:191] cbsz:1 blgp:1\n" \
  "buffer_store_dwordx4 v[208:211], v167, s[40:43], s50 offen offset:64 nt\n" \
  "buffer_load_dwordx4 v[52:55], v164, s[36:39], s75 offen offset:1024\n" \
  "buffer_load_dwordx4 v[56:59], v164, s[36:39], s75 offen offset:2048\n" \
  "buffer_load_dwordx4 v[60:63], v164, s[36:39], s75 offen offset:3072\n" \
  "s_waitcnt vmcnt(22) lgkmcnt(0)\n" \
  "s_barrier\n" \
  "v_mfma_f32_16x16x128_f8f6f4 a[192:195], v[144:151], v[64:71], a[192:195] cbsz:1 blgp:1\n" \
  "ds_read_b128 v[128:131], v161 offset:53248\n" \
  "ds_read_b128 v[132:135], v161 offset:54272\n" \
  "v_mfma_f32_16x16x128_f8f6f4 a[196:199], v[144:151], v[72:79], a[196:199] cbsz:1 blgp:1\n" \
  "s_add_u32 m0, s69, 90112\n" \
  "s_nop 0\n" \
  "buffer_load_dwordx4 v165, s[36:39], s62 offen lds\n" \
  "v_readlane_b32 s67, v175, s70\n" \
  "v_readlane_b32 s76, v169, s70\n" \
  "v_readlane_b32 s71, v181, s70\n" \
  "v_mfma_f32_16x16x128_f8f6f4 a[200:203], v[144:151], v[80:87], a[200:203] cbsz:1 blgp:1\n" \
  "s_add_u32 s70, s70, 1\n" \
  "s_nop 0\n" \
  "v_add_u32 v163, s76, v162\n" \
  "v_add_u32 v164, 4096, v163\n" \
  "s_cmp_gt_u32 s67, s68\n" \
  "s_cbranch_scc1 Lp67_%=\n" \
  "Lq67_%=:\n" \
  "v_mfma_f32_16x16x128_f8f6f4 a[204:207], v[144:151], v[88:95], a[204:207] cbsz:1 blgp:1\n" \
  "v_mfma_f32_16x16x128_f8f6f4 a[208:211], v[152:159], v[64:71], a[208:211] cbsz:1 blgp:1\n" \
  "ds_read_b128 v[136:139], v161 offset:55296\n" \
  "ds_read_b128 v[140:143], v161 offset:56320\n" \
  "v_mfma_f32_16x16x128_f8f6f4 a[212:215], v[152:159], v[72:79], a[212:215] cbsz:1 blgp:1\n" \
  "v_mfma_f32_16x16x128_f8f6f4 a[216:219], v[152:159], v[80:87], a[216:219] cbsz:1 blgp:1\n" \
  "v_mfma_f32_16x16x128_f8f6f4 a[220:223], v[152:159], v[88:95], a[220:223] cbsz:1 blgp:1\n" \
  "buffer_load_dwordx4 v[64:67], v165, s[36:39], s75 offen offset:0\n" \
  "s_waitcnt lgkmcnt(2)\n" \
  "v_mfma_f32_16x16x128_f8f6f4 a[160:163], v[128:135], v[96:103], a[160:163] cbsz:1 blgp:1\n" \
  "ds_read_b128 v[144:147], v161 offset:57344\n" \
  "ds_read_b128 v[148:151], v161 offset:58368\n" \
  "v_mfma_f32_16x16x128_f8f6f4 a[164:167], v[128:135], v[104:111], a[164:167] cbsz:1 blgp:1\n" \
  "buffer_load_dwordx4 v[68:71], v165, s[36:39], s75 offen offset:1024\n" \
  "v_mul_f32 v196, s78, v196\n" \
  "v_mul_f32 v197, s78, v197\n" \
  "v_mul_f32 v198, s78, v198\n" \
  "v_mul_f32 v199, s78, v199\n" \
  "v_mfma_f32_16x16x128_f8f6f4 a[168:171], v[128:135], v[112:119], a[168:171] cbsz:1 blgp:1\n" \
  "buffer_store_dwordx4 v[196:199], v167, s[40:43], s51 offen offset:0 nt\n" \
  "v_mfma_f32_16x16x128_f8f6f4 a[172:175], v[128:135], v[120:127], a[172:175] cbsz:1 blgp:1\n" \
  "buffer_load_dwordx4 v[72:75], v165, s[36:39], s75 offen offset:2048\n" \
  "s_waitcnt lgkmcnt(2)\n" \
  "v_mfma_f32_16x16x128_f8f6f4 a[176:179], v[136:143], v[96:103], a[176:179] cbsz:1 blgp:1\n" \
  "ds_read_b128 v[152:155], v161 offset:59392\n" \
  "ds_read_b128 v[156:159], v161 offset:60416\n" \
  "v_mfma_f32_16x16x128_f8f6f4 a[180:183], v[136:143], v[104:111], a[180:183] cbsz:1 blgp:1\n" \
  "buffer_load_dwordx4 v[76:79], v165, s[36:39], s75 offen offset:3072\n" \
  "v_mfma_f32_16x16x128_f8f6f4 a[184:187], v[136:143], v[112:119], a[184:187] cbsz:1 blgp:1\n" \
  "buffer_load_dwordx4 v[80:83], v166, s[36:39], s75 offen offset:0\n" \
  "v_mul_f32 v212, s78, v212\n" \
  "v_mul_f32 v213, s78, v213\n" \
  "v_mul_f32 v214, s78, v214\n" \
  "v_mul_f32 v215, s78, v215\n" \
  "v_mfma_f32_16x16x128_f8f6f4 a[188:191], v[136:143], v[120:127], a[188:191] cbsz:1 blgp:1\n" \
  "buffer_store_dwordx4 v[212:215], v167, s[40:43], s51 offen offset:64 nt\n" \
  "buffer_load_dwordx4 v[84:87], v166, s[36:39], s75 offen offset:1024\n" \
  "buffer_load_dwordx4 v[88:91], v166, s[36:39], s75 offen offset:2048\n" \
  "buffer_load_dwordx4 v[92:95], v166, s[36:39], s75 offen offset:3072\n" \
  "s_waitcnt vmcnt(22) lgkmcnt(0)\n" \
  "s_barrier\n" \
  "v_mfma_f32_16x16x128_f8f6f4 a[192:195], v[144:151], v[96:103], a[192:195] cbsz:1 blgp:1\n" \
  "ds_read_b128 v[128:131], v160 offset:24576\n" \
  "ds_read_b128 v[132:135], v160 offset:25600\n" \
  "v_mfma_f32_16x16x128_f8f6f4 a[196:199], v[144:151], v[104:111], a[196:199] cbsz:1 blgp:1\n" \
  "s_mov_b32 s42, s72\n" \
  "s_mov_b32 s48, s71\n" \
  "s_add_u32 s49, s71, 1048576\n" \
  "s_add_u32 s50, s71, 2097152\n" \
  "v_mfma_f32_16x16x128_f8f6f4 a[200:203], v[144:151], v[112:119], a[200:203] cbsz:1 blgp:1\n" \
  "s_add_u32 s51, s71, 3145728\n" \
  "s_cmp_eq_u32 s70, 64\n" \
  "s_cbranch_scc1 Lp68_%=\n" \
  "Lq68_%=:\n" \
  "v_mfma_f32_16x16x128_f8f6f4 a[204:207], v[144:151], v[120:127], a[204:207] cbsz:1 blgp:1\n" \
  "v_accvgpr_read_b32 v184, a160\n" \
  "v_accvgpr_read_b32 v185, a161\n" \
  "v_accvgpr_read_b32 v186, a162\n" \
  "v_accvgpr_read_b32 v187, a163\n" \
  "v_mfma_f32_16x16x128_f8f6f4 a[208:211], v[152:159], v[96:103], a[208:211] cbsz:1 blgp:1\n" \
  "ds_read_b128 v[136:139], v160 offset:26624\n" \
  "ds_read_b128 v[140:143], v160 offset:27648\n" \
  "v_mfma_f32_16x16x128_f8f6f4 a[212:215], v[152:159], v[104:111], a[212:215] cbsz:1 blgp:1\n" \
  "v_accvgpr_read_b32 v200, a176\n" \
  "v_accvgpr_read_b32 v201, a177\n" \
  "v_accvgpr_read_b32 v202, a178\n" \
  "v_accvgpr_read_b32 v203, a179\n" \
  "v_mfma_f32_16x16x128_f8f6f4 a[216:219], v[152:159], v[112:119], a[216:219] cbsz:1 blgp:1\n" \
  "v_accvgpr_read_b32 v188, a164\n" \
  "v_accvgpr_read_b32 v189, a165\n" \
  "v_accvgpr_read_b32 v190, a166\n" \
  "v_accvgpr_read_b32 v191, a167\n" \
  "v_mfma_f32_16x16x128_f8f6f4 a[220:223], v[152:159], v[120:127], a[220:223] cbsz:1 blgp:1\n" \
  "v_accvgpr_read_b32 v204, a180\n" \
  "v_accvgpr_read_b32 v205, a181\n" \
  "v_accvgpr_read_b32 v206, a182\n" \
  "v_accvgpr_read_b32 v207, a183\n" \
  "v_accvgpr_read_b32 v192, a168\n" \
  "v_accvgpr_read_b32 v193, a169\n" \
  "v_accvgpr_read_b32 v194, a170\n" \
  "v_accvgpr_read_b32 v195, a171\n" \
  "v_accvgpr_read_b32 v208, a184\n" \
  "v_accvgpr_read_b32 v209, a185\n" \
  "v_accvgpr_read_b32 v210, a186\n" \
  "v_accvgpr_read_b32 v211, a187\n" \
  "v_accvgpr_read_b32 v196, a172\n" \
  "v_accvgpr_read_b32 v197, a173\n" \
  "v_accvgpr_read_b32 v198, a174\n" \
  "v_accvgpr_read_b32 v199, a175\n" \
  "v_accvgpr_read_b32 v212, a188\n" \
  "v_accvgpr_read_b32 v213, a189\n" \
  "v_accvgpr_read_b32 v214, a190\n" \
  "v_accvgpr_read_b32 v215, a191\n" \
  "s_add_u32 m0, s69, 122880\n" \
  "s_nop 0\n" \
  "buffer_load_dwordx4 v163, s[36:39], s62 offen lds\n" \
  "v_readlane_b32 s67, v175, s70\n" \
  "v_readlane_b32 s76, v169, s70\n" \
  "s_add_u32 s70, s70, 1\n" \
  "s_nop 0\n" \
  "v_add_u32 v165, s76, v162\n" \
  "v_add_u32 v166, 4096, v165\n" \
  "s_cmp_gt_u32 s67, s68\n" \
  "s_cbranch_scc1 Lp69_%=\n" \
  "Lq69_%=:\n" \
  "buffer_load_dwordx4 v[96:99], v163, s[36:39], s75 offen offset:0\n" \
  "buffer_load_dwordx4 v[100:103], v163, s[36:39], s75 offen offset:1024\n" \
  "buffer_load_dwordx4 v[104:107], v163, s[36:39], s75 offen offset:2048\n" \
  "buffer_load_dwordx4 v[108:111], v163, s[36:39], s75 offen offset:3072\n" \
  "buffer_load_dwordx4 v[112:115], v164, s[36:39], s75 offen offset:0\n" \
  "buffer_load_dwordx4 v[116:119], v164, s[36:39], s75 offen offset:1024\n" \
  "v_mul_f32 v184, s78, v184\n" \
  "v_mul_f32 v185, s78, v185\n" \
  "v_mul_f32 v186, s78, v186\n" \
  "v_mul_f32 v187, s78, v187\n" \
  "buffer_store_dwordx4 v[184:187], v167, s[40:43], s48 offen offset:0 nt\n" \
  "buffer_load_dwordx4 v[120:123], v164, s[36:39], s75 offen offset:2048\n" \
  "v_mul_f32 v200, s78, v200\n" \
  "v_mul_f32 v201, s78, v201\n" \
  "v_mul_f32 v202, s78, v202\n" \
  "v_mul_f32 v203, s78, v203\n" \
  "buffer_store_dwordx4 v[200:203], v167, s[40:43], s48 offen offset:64 nt\n" \
  "buffer_load_dwordx4 v[124:127], v164, s[36:39], s75 offen offset:3072\n" \
  "s_waitcnt vmcnt(22) lgkmcnt(0)\n" \
  "s_barrier\n" \
  "v_mfma_f32_16x16x128_f8f6f4 a[192:195], v[128:135], v[0:7], a[192:195] cbsz:1 blgp:1\n" \
  "ds_read_b128 v[144:147], v160 offset:57344\n" \
  "ds_read_b128 v[148:151], v160 offset:58368\n" \
  "v_mfma_f32_16x16x128_f8f6f4 a[196:199], v[128:135], v[8:15], a[196:199] cbsz:1 blgp:1\n" \
  "v_readlane_b32 s67, v175, s70\n" \
  "v_readlane_b32 s76, v169, s70\n" \
  "s_add_u32 s70, s70, 1\n" \
  "s_nop 0\n" \
  "v_add_u32 v163, s76, v162\n" \
  "v_mfma_f32_16x16x128_f8f6f4 a[200:203], v[128:135], v[16:23], a[200:203] cbsz:1 blgp:1\n" \
  "v_add_u32 v164, 4096, v163\n" \
  "s_cmp_gt_u32 s67, s68\n" \
  "s_cbranch_scc1 Lp70_%=\n" \
  "Lq70_%=:\n" \
  "v_mfma_f32_16x16x128_f8f6f4 a[204:207], v[128:135], v[24:31], a[204:207] cbsz:1 blgp:1\n" \
  "v_mul_f32 v188, s78, v188\n" \
  "v_mul_f32 v189, s78, v189\n" \
  "v_mul_f32 v190, s78, v190\n" \
  "v_mul_f32 v191, s78, v191\n" \
  "buffer_store_dwordx4 v[188:191], v167, s[40:43], s49 offen offset:0 nt\n" \
  "v_mfma_f32_16x16x128_f8f6f4 a[208:211], v[136:143], v[0:7], a[208:211] cbsz:1 blgp:1\n" \
  "ds_read_b128 v[152:155], v160 offset:59392\n" \
  "ds_read_b128 v[156:159], v160 offset:60416\n" \
  "v_mfma_f32_16x16x128_f8f6f4 a[212:215], v[136:143], v[8:15], a[212:215] cbsz:1 blgp:1\n" \
  "v_mul_f32 v204, s78, v204\n" \
  "v_mul_f32 v205, s78, v205\n" \
  "v_mul_f32 v206, s78, v206\n" \
  "v_mul_f32 v207, s78, v207\n" \
  "buffer_store_dwordx4 v[204:207], v167, s[40:43], s49 offen offset:64 nt\n" \
  "v_mfma_f32_16x16x128_f8f6f4 a[216:219], v[136:143], v[16:23], a[216:219] cbsz:1 blgp:1\n" \
  "v_mfma_f32_16x16x128_f8f6f4 a[220:223], v[136:143], v[24:31], a[220:223] cbsz:1 blgp:1\n" \
  "s_waitcnt vmcnt(13) lgkmcnt(0)\n" \
  "s_barrier\n" \
  "v_mfma_f32_16x16x128_f8f6f4 a[192:195], v[144:151], v[32:39], a[192:195] cbsz:1 blgp:1\n" \
  "ds_read_b128 v[128:131], v161 offset:24576\n" \
  "ds_read_b128 v[132:135], v161 offset:25600\n" \
  "v_mfma_f32_16x16x128_f8f6f4 a[196:199], v[144:151], v[40:47], a[196:199] cbsz:1 blgp:1\n" \
  "v_readlane_b32 s67, v175, s70\n" \
  "v_readlane_b32 s76, v169, s70\n" \
  "s_add_u32 s70, s70, 1\n" \
  "s_nop 0\n" \
  "v_add_u32 v165, s76, v162\n" \
  "v_mfma_f32_16x16x128_f8f6f4 a[200:203], v[144:151], v[48:55], a[200:203] cbsz:1 blgp:1\n" \
  "v_add_u32 v166, 4096, v165\n" \
  "s_cmp_gt_u32 s67, s68\n" \
  "s_cbranch_scc1 Lp71_%=\n" \
  "Lq71_%=:\n" \
  "v_mfma_f32_16x16x128_f8f6f4 a[204:207], v[144:151], v[56:63], a[204:207] cbsz:1 blgp:1\n" \
  "v_mul_f32 v192, s78, v192\n" \
  "v_mul_f32 v193, s78, v193\n" \
  "v_mul_f32 v194, s78, v194\n" \
  "v_mul_f32 v195, s78, v195\n" \
  "buffer_store_dwordx4 v[192:195], v167, s[40:43], s50 offen offset:0 nt\n" \
  "v_mfma_f32_16x16x128_f8f6f4 a[208:211], v[152:159], v[32:39], a[208:211] cbsz:1 blgp:1\n" \
  "ds_read_b128 v[136:139], v161 offset:26624\n" \
  "ds_read_b128 v[140:143], v161 offset:27648\n" \
  "v_mfma_f32_16x16x128_f8f6f4 a[212:215], v[152:159], v[40:47], a[212:215] cbsz:1 blgp:1\n" \
  "v_mul_f32 v208, s78, v208\n" \
  "v_mul_f32 v209, s78, v209\n" \
  "v_mul_f32 v210, s78, v210\n" \
  "v_mul_f32 v211, s78, v211\n" \
  "buffer_store_dwordx4 v[208:211], v167, s[40:43], s50 offen offset:64 nt\n" \
  "v_mfma_f32_16x16x128_f8f6f4 a[216:219], v[152:159], v[48:55], a[216:219] cbsz:1 blgp:1\n" \
  "v_mfma_f32_16x16x128_f8f6f4 a[220:223], v[152:159], v[56:63], a[220:223] cbsz:1 blgp:1\n" \
  "s_waitcnt vmcnt(4) lgkmcnt(0)\n" \
  "s_barrier\n" \
  "v_mfma_f32_16x16x128_f8f6f4 a[192:195], v[128:135], v[64:71], a[192:195] cbsz:1 blgp:1\n" \
  "ds_read_b128 v[144:147], v161 offset:57344\n" \
  "ds_read_b128 v[148:151], v161 offset:58368\n" \
  "v_mfma_f32_16x16x128_f8f6f4 a[196:199], v[128:135], v[72:79], a[196:199] cbsz:1 blgp:1\n" \
  "v_mul_f32 v196, s78, v196\n" \
  "v_mul_f32 v197, s78, v197\n" \
  "v_mul_f32 v198, s78, v198\n" \
  "v_mul_f32 v199, s78, v199\n" \
  "buffer_store_dwordx4 v[196:199], v167, s[40:43], s51 offen offset:0 nt\n" \
  "v_mfma_f32_16x16x128_f8f6f4 a[200:203], v[128:135], v[80:87], a[200:203] cbsz:1 blgp:1\n" \
  "v_mfma_f32_16x16x128_f8f6f4 a[204:207], v[128:135], v[88:95], a[204:207] cbsz:1 blgp:1\n" \
  "v_mul_f32 v212, s78, v212\n" \
  "v_mul_f32 v213, s78, v213\n" \
  "v_mul_f32 v214, s78, v214\n" \
  "v_mul_f32 v215, s78, v215\n" \
  "v_mfma_f32_16x16x128_f8f6f4 a[208:211], v[136:143], v[64:71], a[208:211] cbsz:1 blgp:1\n" \
  "ds_read_b128 v[152:155], v161 offset:59392\n" \
  "ds_read_b128 v[156:159], v161 offset:60416\n" \
  "v_mfma_f32_16x16x128_f8f6f4 a[212:215], v[136:143], v[72:79], a[212:215] cbsz:1 blgp:1\n" \
  "buffer_store_dwordx4 v[212:215], v167, s[40:43], s51 offen offset:64 nt\n" \
  "v_mfma_f32_16x16x128_f8f6f4 a[216:219], v[136:143], v[80:87], a[216:219] cbsz:1 blgp:1\n" \
  "v_mfma_f32_16x16x128_f8f6f4 a[220:223], v[136:143], v[88:95], a[220:223] cbsz:1 blgp:1\n" \
  "s_waitcnt vmcnt(4) lgkmcnt(0)\n" \
  "v_mfma_f32_16x16x128_f8f6f4 a[192:195], v[144:151], v[96:103], a[192:195] cbsz:1 blgp:1\n" \
  "v_mfma_f32_16x16x128_f8f6f4 a[196:199], v[144:151], v[104:111], a[196:199] cbsz:1 blgp:1\n" \
  "v_mfma_f32_16x16x128_f8f6f4 a[200:203], v[144:151], v[112:119], a[200:203] cbsz:1 blgp:1\n" \
  "v_mfma_f32_16x16x128_f8f6f4 a[204:207], v[144:151], v[120:127], a[204:207] cbsz:1 blgp:1\n" \
  "v_mfma_f32_16x16x128_f8f6f4 a[208:211], v[152:159], v[96:103], a[208:211] cbsz:1 blgp:1\n" \
  "v_mfma_f32_16x16x128_f8f6f4 a[212:215], v[152:159], v[104:111], a[212:215] cbsz:1 blgp:1\n" \
  "v_mfma_f32_16x16x128_f8f6f4 a[216:219], v[152:159], v[112:119], a[216:219] cbsz:1 blgp:1\n" \
  "v_mfma_f32_16x16x128_f8f6f4 a[220:223], v[152:159], v[120:127], a[220:223] cbsz:1 blgp:1\n" \
  "global_load_dword v218, %[tmoo], s[80:81] sc1\n" \
  "v_readlane_b32 s71, v181, s70\n" \
  "s_nop 7\n" \
  "s_nop 3\n" \
  "s_mov_b32 s48, s71\n" \
  "s_add_u32 s49, s71, 1048576\n" \
  "s_add_u32 s50, s71, 2097152\n" \
  "s_add_u32 s51, s71, 3145728\n" \
  "v_accvgpr_read_b32 v184, a192\n" \
  "v_accvgpr_read_b32 v185, a193\n" \
  "v_accvgpr_read_b32 v186, a194\n" \
  "v_accvgpr_read_b32 v187, a195\n" \
  "v_mul_f32 v184, s78, v184\n" \
  "v_mul_f32 v185, s78, v185\n" \
  "v_mul_f32 v186, s78, v186\n" \
  "v_mul_f32 v187, s78, v187\n" \
  "buffer_store_dwordx4 v[184:187], v167, s[40:43], s48 offen offset:0 nt\n" \
  "v_accvgpr_read_b32 v200, a208\n" \
  "v_accvgpr_read_b32 v201, a209\n" \
  "v_accvgpr_read_b32 v202, a210\n" \
  "v_accvgpr_read_b32 v203, a211\n" \
  "v_mul_f32 v200, s78, v200\n" \
  "v_mul_f32 v201, s78, v201\n" \
  "v_mul_f32 v202, s78, v202\n" \
  "v_mul_f32 v203, s78, v203\n" \
  "buffer_store_dwordx4 v[200:203], v167, s[40:43], s48 offen offset:64 nt\n" \
  "v_accvgpr_read_b32 v188, a196\n" \
  "v_accvgpr_read_b32 v189, a197\n" \
  "v_accvgpr_read_b32 v190, a198\n" \
  "v_accvgpr_read_b32 v191, a199\n" \
  "v_mul_f32 v188, s78, v188\n" \
  "v_mul_f32 v189, s78, v189\n" \
  "v_mul_f32 v190, s78, v190\n" \
  "v_mul_f32 v191, s78, v191\n" \
  "buffer_store_dwordx4 v[188:191], v167, s[40:43], s49 offen offset:0 nt\n" \
  "v_accvgpr_read_b32 v204, a212\n" \
  "v_accvgpr_read_b32 v205, a213\n" \
  "v_accvgpr_read_b32 v206, a214\n" \
  "v_accvgpr_read_b32 v207, a215\n" \
  "v_mul_f32 v204, s78, v204\n" \
  "v_mul_f32 v205, s78, v205\n" \
  "v_mul_f32 v206, s78, v206\n" \
  "v_mul_f32 v207, s78, v207\n" \
  "buffer_store_dwordx4 v[204:207], v167, s[40:43], s49 offen offset:64 nt\n" \
  "v_accvgpr_read_b32 v192, a200\n" \
  "v_accvgpr_read_b32 v193, a201\n" \
  "v_accvgpr_read_b32 v194, a202\n" \
  "v_accvgpr_read_b32 v195, a203\n" \
  "v_mul_f32 v192, s78, v192\n" \
  "v_mul_f32 v193, s78, v193\n" \
  "v_mul_f32 v194, s78, v194\n" \
  "v_mul_f32 v195, s78, v195\n" \
  "buffer_store_dwordx4 v[192:195], v167, s[40:43], s50 offen offset:0 nt\n" \
  "v_accvgpr_read_b32 v208, a216\n" \
  "v_accvgpr_read_b32 v209, a217\n" \
  "v_accvgpr_read_b32 v210, a218\n" \
  "v_accvgpr_read_b32 v211, a219\n" \
  "v_mul_f32 v208, s78, v208\n" \
  "v_mul_f32 v209, s78, v209\n" \
  "v_mul_f32 v210, s78, v210\n" \
  "v_mul_f32 v211, s78, v211\n" \
  "buffer_store_dwordx4 v[208:211], v167, s[40:43], s50 offen offset:64 nt\n" \
  "v_accvgpr_read_b32 v196, a204\n" \
  "v_accvgpr_read_b32 v197, a205\n" \
  "v_accvgpr_read_b32 v198, a206\n" \
  "v_accvgpr_read_b32 v199, a207\n" \
  "v_mul_f32 v196, s78, v196\n" \
  "v_mul_f32 v197, s78, v197\n" \
  "v_mul_f32 v198, s78, v198\n" \
  "v_mul_f32 v199, s78, v199\n" \
  "buffer_store_dwordx4 v[196:199], v167, s[40:43], s51 offen offset:0 nt\n" \
  "v_accvgpr_read_b32 v212, a220\n" \
  "v_accvgpr_read_b32 v213, a221\n" \
  "v_accvgpr_read_b32 v214, a222\n" \
  "v_accvgpr_read_b32 v215, a223\n" \
  "v_mul_f32 v212, s78, v212\n" \
  "v_mul_f32 v213, s78, v213\n" \
  "v_mul_f32 v214, s78, v214\n" \
  "v_mul_f32 v215, s78, v215\n" \
  "buffer_store_dwordx4 v[212:215], v167, s[40:43], s51 offen offset:64 nt\n" \
  "s_branch Ldone_%=\n" \
  "Lme0_%=:\n" \
  "s_swappc_b64 s[92:93], s[96:97]\n" \
  "s_branch Lmer0_%=\n" \
  "Lz0_%=:\n" \
  "ds_read_b128 v[128:131], v160 offset:0\n" \
  "ds_read_b128 v[132:135], v160 offset:1024\n" \
  "ds_read_b128 v[136:139], v160 offset:2048\n" \
  "ds_read_b128 v[140:143], v160 offset:3072\n" \
  "s_add_u32 m0, s69, 126976\n" \
  "s_nop 0\n" \
  "buffer_load_dwordx4 v163, s[36:39], s63 offen lds\n" \
  "s_branch Lzr0_%=\n" \
  "Lpj0_%=:\n" \
  "s_setpc_b64 s[98:99]\n" \
  "Lsuf0_0_1_%=:\n" \
  "s_bitcmp1_b32 s65, 10\n" \
  "s_cbranch_scc1 Lsuf0_1_1_%=\n" \
  "ds_read_b128 v[144:147], v160 offset:28672\n" \
  "ds_read_b128 v[148:151], v160 offset:29696\n" \
  "ds_read_b128 v[152:155], v160 offset:30720\n" \
  "ds_read_b128 v[156:159], v160 offset:31744\n" \
  "s_add_u32 m0, s69, 102400\n" \
  "s_nop 0\n" \
  "buffer_load_dwordx4 v163, s[36:39], s57 offen lds\n" \
  "s_add_u32 m0, s69, 106496\n" \
  "s_nop 0\n" \
  "buffer_load_dwordx4 v163, s[36:39], s58 offen lds\n" \
  "s_add_u32 m0, s69, 110592\n" \
  "s_nop 0\n" \
  "buffer_load_dwordx4 v163, s[36:39], s59 offen lds\n" \
  "s_add_u32 m0, s69, 114688\n" \
  "s_nop 0\n" \
  "buffer_load_dwordx4 v163, s[36:39], s60 offen lds\n" \
  "buffer_load_dwordx4 v[100:103], v163, s[36:39], s75 offen offset:1024\n" \
  "v_readfirstlane_b32 s89, v218\n" \
  "s_nop 1\n" \
  "s_bitcmp1_b32 s65, 18\n" \
  "s_cselect_b32 s42, s72, 0\n" \
  "s_mov_b32 s48, s71\n" \
  "s_bitcmp1_b32 s65, 28\n" \
  "s_cselect_b32 s73, 3, 0x100\n" \
  "s_lshl_b32 s73, s73, 12\n" \
  "s_add_u32 s49, s48, s73\n" \
  "s_add_u32 s50, s49, s73\n" \
  "s_add_u32 s51, s50, s73\n" \
  "s_cmp_eq_u32 s70, 64\n" \
  "s_cbranch_scc1 Lp72_%=\n" \
  "Lq72_%=:\n" \
  "buffer_load_dwordx4 v[96:99], v163, s[36:39], s75 offen offset:0\n" \
  "s_max_u32 s68, s68, s89\n" \
  "v_readlane_b32 s67, v175, s70\n" \
  "v_readlane_b32 s66, v178, s70\n" \
  "v_readlane_b32 s76, v169, s70\n" \
  "v_readlane_b32 s77, v172, s70\n" \
  "s_add_u32 s70, s70, 1\n" \
  "v_add_u32 v165, s76, v162\n" \
  "s_add_u32 m0, s69, 118784\n" \
  "s_nop 0\n" \
  "buffer_load_dwordx4 v163, s[36:39], s61 offen lds\n" \
  "buffer_load_dwordx4 v[108:111], v163, s[36:39], s75 offen offset:3072\n" \
  "v_add_u32 v166, 4096, v165\n" \
  "s_cmp_gt_u32 s67, s68\n" \
  "s_cbranch_scc1 Lp73_%=\n" \
  "Lq73_%=:\n" \
  "buffer_load_dwordx4 v[104:107], v163, s[36:39], s75 offen offset:2048\n" \
  "buffer_load_dwordx4 v[112:115], v164, s[36:39], s75 offen offset:0\n" \
  "s_add_u32 m0, s69, 122880\n" \
  "s_nop 0\n" \
  "buffer_load_dwordx4 v163, s[36:39], s62 offen lds\n" \
  "buffer_load_dwordx4 v[120:123], v164, s[36:39], s75 offen offset:2048\n" \
  "s_and_b32 s73, s66, 0xf00001ff\n" \
  "buffer_load_dwordx4 v[116:119], v164, s[36:39], s75 offen offset:1024\n" \
  "buffer_load_dwordx4 v[124:127], v164, s[36:39], s75 offen offset:3072\n" \
  "v_mul_f32 v184, s78, v184\n" \
  "v_mul_f32 v185, s78, v185\n" \
  "v_mul_f32 v186, s78, v186\n" \
  "v_mul_f32 v187, s78, v187\n" \
  "s_bitcmp1_b32 s65, 28\n" \
  "s_cbranch_scc1 Lp74_%=\n" \
  "buffer_store_dwordx4 v[184:187], v167, s[40:43], s48 offen offset:0 nt\n" \
  "Lq74_%=:\n" \
  "v_mul_f32 v200, s78, v200\n" \
  "v_mul_f32 v201, s78, v201\n" \
  "v_mul_f32 v202, s78, v202\n" \
  "v_mul_f32 v203, s78, v203\n" \
  "s_bitcmp1_b32 s65, 28\n" \
  "s_cbranch_scc1 Lp75_%=\n" \
  "buffer_store_dwordx4 v[200:203], v167, s[40:43], s48 offen offset:64 nt\n" \
  "Lq75_%=:\n" \
  "s_bfe_u32 m0, s66, 0x40018\n" \
  "s_nop 0\n" \
  "s_movreld_b32 s56, s77\n" \
  "s_branch Ls0_8_%=\n" \
  "Lsuf0_0_2_%=:\n" \
  "s_bitcmp1_b32 s65, 10\n" \
  "s_cbranch_scc1 Lsuf0_1_2_%=\n" \
  "ds_read_b128 v[144:147], v160 offset:28672\n" \
  "ds_read_b128 v[148:151], v160 offset:29696\n" \
  "ds_read_b128 v[152:155], v160 offset:30720\n" \
  "ds_read_b128 v[156:159], v160 offset:31744\n" \
  "s_add_u32 m0, s69, 106496\n" \
  "s_nop 0\n" \
  "buffer_load_dwordx4 v163, s[36:39], s58 offen lds\n" \
  "s_add_u32 m0, s69, 110592\n" \
  "s_nop 0\n" \
  "buffer_load_dwordx4 v163, s[36:39], s59 offen lds\n" \
  "s_add_u32 m0, s69, 114688\n" \
  "s_nop 0\n" \
  "buffer_load_dwordx4 v163, s[36:39], s60 offen lds\n" \
  "buffer_load_dwordx4 v[100:103], v163, s[36:39], s75 offen offset:1024\n" \
  "v_readfirstlane_b32 s89, v218\n" \
  "s_nop 1\n" \
  "s_bitcmp1_b32 s65, 18\n" \
  "s_cselect_b32 s42, s72, 0\n" \
  "s_mov_b32 s48, s71\n" \
  "s_bitcmp1_b32 s65, 28\n" \
  "s_cselect_b32 s73, 3, 0x100\n" \
  "s_lshl_b32 s73, s73, 12\n" \
  "s_add_u32 s49, s48, s73\n" \
  "s_add_u32 s50, s49, s73\n" \
  "s_add_u32 s51, s50, s73\n" \
  "s_cmp_eq_u32 s70, 64\n" \
  "s_cbranch_scc1 Lp76_%=\n" \
  "Lq76_%=:\n" \
  "buffer_load_dwordx4 v[96:99], v163, s[36:39], s75 offen offset:0\n" \
  "s_max_u32 s68, s68, s89\n" \
  "v_readlane_b32 s67, v175, s70\n" \
  "v_readlane_b32 s66, v178, s70\n" \
  "v_readlane_b32 s76, v169, s70\n" \
  "v_readlane_b32 s77, v172, s70\n" \
  "s_add_u32 s70, s70, 1\n" \
  "v_add_u32 v165, s76, v162\n" \
  "s_add_u32 m0, s69, 118784\n" \
  "s_nop 0\n" \
  "buffer_load_dwordx4 v163, s[36:39], s61 offen lds\n" \
  "buffer_load_dwordx4 v[108:111], v163, s[36:39], s75 offen offset:3072\n" \
  "v_add_u32 v166, 4096, v165\n" \
  "s_cmp_gt_u32 s67, s68\n" \
  "s_cbranch_scc1 Lp77_%=\n" \
  "Lq77_%=:\n" \
  "buffer_load_dwordx4 v[104:107], v163, s[36:39], s75 offen offset:2048\n" \
  "buffer_load_dwordx4 v[112:115], v164, s[36:39], s75 offen offset:0\n" \
  "s_add_u32 m0, s69, 122880\n" \
  "s_nop 0\n" \
  "buffer_load_dwordx4 v163, s[36:39], s62 offen lds\n" \
  "buffer_load_dwordx4 v[120:123], v164, s[36:39], s75 offen offset:2048\n" \
  "s_and_b32 s73, s66, 0xf00001ff\n" \
  "buffer_load_dwordx4 v[116:119], v164, s[36:39], s75 offen offset:1024\n" \
  "buffer_load_dwordx4 v[124:127], v164, s[36:39], s75 offen offset:3072\n" \
  "v_mul_f32 v184, s78, v184\n" \
  "v_mul_f32 v185, s78, v185\n" \
  "v_mul_f32 v186, s78, v186\n" \
  "v_mul_f32 v187, s78, v187\n" \
  "s_bitcmp1_b32 s65, 28\n" \
  "s_cbranch_scc1 Lp78_%=\n" \
  "buffer_store_dwordx4 v[184:187], v167, s[40:43], s48 offen offset:0 nt\n" \
  "Lq78_%=:\n" \
  "v_mul_f32 v200, s78, v200\n" \
  "v_mul_f32 v201, s78, v201\n" \
  "v_mul_f32 v202, s78, v202\n" \
  "v_mul_f32 v203, s78, v203\n" \
  "s_bitcmp1_b32 s65, 28\n" \
  "s_cbranch_scc1 Lp79_%=\n" \
  "buffer_store_dwordx4 v[200:203], v167, s[40:43], s48 offen offset:64 nt\n" \
  "Lq79_%=:\n" \
  "s_bfe_u32 m0, s66, 0x40018\n" \
  "s_nop 0\n" \
  "s_movreld_b32 s56, s77\n" \
  "s_branch Ls0_8_%=\n" \
  "Lsuf0_0_3_%=:\n" \
  "s_bitcmp1_b32 s65, 10\n" \
  "s_cbranch_scc1 Lsuf0_1_3_%=\n" \
  "ds_read_b128 v[144:147], v160 offset:28672\n" \
  "ds_read_b128 v[148:151], v160 offset:29696\n" \
  "ds_read_b128 v[152:155], v160 offset:30720\n" \
  "ds_read_b128 v[156:159], v160 offset:31744\n" \
  "s_add_u32 m0, s69, 110592\n" \
  "s_nop 0\n" \
  "buffer_load_dwordx4 v163, s[36:39], s59 offen lds\n" \
  "s_add_u32 m0, s69, 114688\n" \
  "s_nop 0\n" \
  "buffer_load_dwordx4 v163, s[36:39], s60 offen lds\n" \
  "buffer_load_dwordx4 v[100:103], v163, s[36:39], s75 offen offset:1024\n" \
  "v_readfirstlane_b32 s89, v218\n" \
  "s_nop 1\n" \
  "s_bitcmp1_b32 s65, 18\n" \
  "s_cselect_b32 s42, s72, 0\n" \
  "s_mov_b32 s48, s71\n" \
  "s_bitcmp1_b32 s65, 28\n" \
  "s_cselect_b32 s73, 3, 0x100\n" \
  "s_lshl_b32 s73, s73, 12\n" \
  "s_add_u32 s49, s48, s73\n" \
  "s_add_u32 s50, s49, s73\n" \
  "s_add_u32 s51, s50, s73\n" \
  "s_cmp_eq_u32 s70, 64\n" \
  "s_cbranch_scc1 Lp80_%=\n" \
  "Lq80_%=:\n" \
  "buffer_load_dwordx4 v[96:99], v163, s[36:39], s75 offen offset:0\n" \
  "s_max_u32 s68, s68, s89\n" \
  "v_readlane_b32 s67, v175, s70\n" \
  "v_readlane_b32 s66, v178, s70\n" \
  "v_readlane_b32 s76, v169, s70\n" \
  "v_readlane_b32 s77, v172, s70\n" \
  "s_add_u32 s70, s70, 1\n" \
  "v_add_u32 v165, s76, v162\n" \
  "s_add_u32 m0, s69, 118784\n" \
  "s_nop 0\n" \
  "buffer_load_dwordx4 v163, s[36:39], s61 offen lds\n" \
  "buffer_load_dwordx4 v[108:111], v163, s[36:39], s75 offen offset:3072\n" \
  "v_add_u32 v166, 4096, v165\n" \
  "s_cmp_gt_u32 s67, s68\n" \
  "s_cbranch_scc1 Lp81_%=\n" \
  "Lq81_%=:\n" \
  "buffer_load_dwordx4 v[104:107], v163, s[36:39], s75 offen offset:2048\n" \
  "buffer_load_dwordx4 v[112:115], v164, s[36:39], s75 offen offset:0\n" \
  "s_add_u32 m0, s69, 122880\n" \
  "s_nop 0\n" \
  "buffer_load_dwordx4 v163, s[36:39], s62 offen lds\n" \
  "buffer_load_dwordx4 v[120:123], v164, s[36:39], s75 offen offset:2048\n" \
  "s_and_b32 s73, s66, 0xf00001ff\n" \
  "buffer_load_dwordx4 v[116:119], v164, s[36:39], s75 offen offset:1024\n" \
  "buffer_load_dwordx4 v[124:127], v164, s[36:39], s75 offen offset:3072\n" \
  "v_mul_f32 v184, s78, v184\n" \
  "v_mul_f32 v185, s78, v185\n" \
  "v_mul_f32 v186, s78, v186\n" \
  "v_mul_f32 v187, s78, v187\n" \
  "s_bitcmp1_b32 s65, 28\n" \
  "s_cbranch_scc1 Lp82_%=\n" \
  "buffer_store_dwordx4 v[184:187], v167, s[40:43], s48 offen offset:0 nt\n" \
  "Lq82_%=:\n" \
  "v_mul_f32 v200, s78, v200\n" \
  "v_mul_f32 v201, s78, v201\n" \
  "v_mul_f32 v202, s78, v202\n" \
  "v_mul_f32 v203, s78, v203\n" \
  "s_bitcmp1_b32 s65, 28\n" \
  "s_cbranch_scc1 Lp83_%=\n" \
  "buffer_store_dwordx4 v[200:203], v167, s[40:43], s48 offen offset:64 nt\n" \
  "Lq83_%=:\n" \
  "s_bfe_u32 m0, s66, 0x40018\n" \
  "s_nop 0\n" \
  "s_movreld_b32 s56, s77\n" \
  "s_branch Ls0_8_%=\n" \
  "Lsuf0_0_4_%=:\n" \
  "s_bitcmp1_b32 s65, 10\n" \
  "s_cbranch_scc1 Lsuf0_1_4_%=\n" \
  "ds_read_b128 v[144:147], v160 offset:28672\n" \
  "ds_read_b128 v[148:151], v160 offset:29696\n" \
  "ds_read_b128 v[152:155], v160 offset:30720\n" \
  "ds_read_b128 v[156:159], v160 offset:31744\n" \
  "s_add_u32 m0, s69, 114688\n" \
  "s_nop 0\n" \
  "buffer_load_dwordx4 v163, s[36:39], s60 offen lds\n" \
  "buffer_load_dwordx4 v[100:103], v163, s[36:39], s75 offen offset:1024\n" \
  "v_readfirstlane_b32 s89, v218\n" \
  "s_nop 1\n" \
  "s_bitcmp1_b32 s65, 18\n" \
  "s_cselect_b32 s42, s72, 0\n" \
  "s_mov_b32 s48, s71\n" \
  "s_bitcmp1_b32 s65, 28\n" \
  "s_cselect_b32 s73, 3, 0x100\n" \
  "s_lshl_b32 s73, s73, 12\n" \
  "s_add_u32 s49, s48, s73\n" \
  "s_add_u32 s50, s49, s73\n" \
  "s_add_u32 s51, s50, s73\n" \
  "s_cmp_eq_u32 s70, 64\n" \
  "s_cbranch_scc1 Lp84_%=\n" \
  "Lq84_%=:\n" \
  "buffer_load_dwordx4 v[96:99], v163, s[36:39], s75 offen offset:0\n" \
  "s_max_u32 s68, s68, s89\n" \
  "v_readlane_b32 s67, v175, s70\n" \
  "v_readlane_b32 s66, v178, s70\n" \
  "v_readlane_b32 s76, v169, s70\n" \
  "v_readlane_b32 s77, v172, s70\n" \
  "s_add_u32 s70, s70, 1\n" \
  "v_add_u32 v165, s76, v162\n" \
  "s_add_u32 m0, s69, 118784\n" \
  "s_nop 0\n" \
  "buffer_load_dwordx4 v163, s[36:39], s61 offen lds\n" \
  "buffer_load_dwordx4 v[108:111], v163, s[36:39], s75 offen offset:3072\n" \
  "v_add_u32 v166, 4096, v165\n" \
  "s_cmp_gt_u32 s67, s68\n" \
  "s_cbranch_scc1 Lp85_%=\n" \
  "Lq85_%=:\n" \
  "buffer_load_dwordx4 v[104:107], v163, s[36:39], s75 offen offset:2048\n" \
  "buffer_load_dwordx4 v[112:115], v164, s[36:39], s75 offen offset:0\n" \
  "s_add_u32 m0, s69, 122880\n" \
  "s_nop 0\n" \
  "buffer_load_dwordx4 v163, s[36:39], s62 offen lds\n" \
  "buffer_load_dwordx4 v[120:123], v164, s[36:39], s75 offen offset:2048\n" \
  "s_and_b32 s73, s66, 0xf00001ff\n" \
  "buffer_load_dwordx4 v[116:119], v164, s[36:39], s75 offen offset:1024\n" \
  "buffer_load_dwordx4 v[124:127], v164, s[36:39], s75 offen offset:3072\n" \
  "v_mul_f32 v184, s78, v184\n" \
  "v_mul_f32 v185, s78, v185\n" \
  "v_mul_f32 v186, s78, v186\n" \
  "v_mul_f32 v187, s78, v187\n" \
  "s_bitcmp1_b32 s65, 28\n" \
  "s_cbranch_scc1 Lp86_%=\n" \
  "buffer_store_dwordx4 v[184:187], v167, s[40:43], s48 offen offset:0 nt\n" \
  "Lq86_%=:\n" \
  "v_mul_f32 v200, s78, v200\n" \
  "v_mul_f32 v201, s78, v201\n" \
  "v_mul_f32 v202, s78, v202\n" \
  "v_mul_f32 v203, s78, v203\n" \
  "s_bitcmp1_b32 s65, 28\n" \
  "s_cbranch_scc1 Lp87_%=\n" \
  "buffer_store_dwordx4 v[200:203], v167, s[40:43], s48 offen offset:64 nt\n" \
  "Lq87_%=:\n" \
  "s_bfe_u32 m0, s66, 0x40018\n" \
  "s_nop 0\n" \
  "s_movreld_b32 s56, s77\n" \
  "s_branch Ls0_8_%=\n" \
  "Lsuf0_0_5_%=:\n" \
  "s_bitcmp1_b32 s65, 10\n" \
  "s_cbranch_scc1 Lsuf0_1_5_%=\n" \
  "ds_read_b128 v[144:147], v160 offset:28672\n" \
  "ds_read_b128 v[148:151], v160 offset:29696\n" \
  "ds_read_b128 v[152:155], v160 offset:30720\n" \
  "ds_read_b128 v[156:159], v160 offset:31744\n" \
  "s_add_u32 m0, s69, 118784\n" \
  "s_nop 0\n" \
  "buffer_load_dwordx4 v163, s[36:39], s61 offen lds\n" \
  "buffer_load_dwordx4 v[108:111], v163, s[36:39], s75 offen offset:3072\n" \
  "v_add_u32 v166, 4096, v165\n" \
  "s_cmp_gt_u32 s67, s68\n" \
  "s_cbranch_scc1 Lp88_%=\n" \
  "Lq88_%=:\n" \
  "buffer_load_dwordx4 v[104:107], v163, s[36:39], s75 offen offset:2048\n" \
  "buffer_load_dwordx4 v[112:115], v164, s[36:39], s75 offen offset:0\n" \
  "s_add_u32 m0, s69, 122880\n" \
  "s_nop 0\n" \
  "buffer_load_dwordx4 v163, s[36:39], s62 offen lds\n" \
  "buffer_load_dwordx4 v[120:123], v164, s[36:39], s75 offen offset:2048\n" \
  "s_and_b32 s73, s66, 0xf00001ff\n" \
  "buffer_load_dwordx4 v[116:119], v164, s[36:39], s75 offen offset:1024\n" \
  "buffer_load_dwordx4 v[124:127], v164, s[36:39], s75 offen offset:3072\n" \
  "v_mul_f32 v184, s78, v184\n" \
  "v_mul_f32 v185, s78, v185\n" \
  "v_mul_f32 v186, s78, v186\n" \
  "v_mul_f32 v187, s78, v187\n" \
  "s_bitcmp1_b32 s65, 28\n" \
  "s_cbranch_scc1 Lp89_%=\n" \
  "buffer_store_dwordx4 v[184:187], v167, s[40:43], s48 offen offset:0 nt\n" \
  "Lq89_%=:\n" \
  "v_mul_f32 v200, s78, v200\n" \
  "v_mul_f32 v201, s78, v201\n" \
  "v_mul_f32 v202, s78, v202\n" \
  "v_mul_f32 v203, s78, v203\n" \
  "s_bitcmp1_b32 s65, 28\n" \
  "s_cbranch_scc1 Lp90_%=\n" \
  "buffer_store_dwordx4 v[200:203], v167, s[40:43], s48 offen offset:64 nt\n" \
  "Lq90_%=:\n" \
  "s_bfe_u32 m0, s66, 0x40018\n" \
  "s_nop 0\n" \
  "s_movreld_b32 s56, s77\n" \
  "s_branch Ls0_8_%=\n" \
  "Lsuf0_0_6_%=:\n" \
  "s_bitcmp1_b32 s65, 10\n" \
  "s_cbranch_scc1 Lsuf0_1_6_%=\n" \
  "ds_read_b128 v[144:147], v160 offset:28672\n" \
  "ds_read_b128 v[148:151], v160 offset:29696\n" \
  "ds_read_b128 v[152:155], v160 offset:30720\n" \
  "ds_read_b128 v[156:159], v160 offset:31744\n" \
  "s_add_u32 m0, s69, 122880\n" \
  "s_nop 0\n" \
  "buffer_load_dwordx4 v163, s[36:39], s62 offen lds\n" \
  "buffer_load_dwordx4 v[120:123], v164, s[36:39], s75 offen offset:2048\n" \
  "s_and_b32 s73, s66, 0xf00001ff\n" \
  "buffer_load_dwordx4 v[116:119], v164, s[36:39], s75 offen offset:1024\n" \
  "buffer_load_dwordx4 v[124:127], v164, s[36:39], s75 offen offset:3072\n" \
  "v_mul_f32 v184, s78, v184\n" \
  "v_mul_f32 v185, s78, v185\n" \
  "v_mul_f32 v186, s78, v186\n" \
  "v_mul_f32 v187, s78, v187\n" \
  "s_bitcmp1_b32 s65, 28\n" \
  "s_cbranch_scc1 Lp91_%=\n" \
  "buffer_store_dwordx4 v[184:187], v167, s[40:43], s48 offen offset:0 nt\n" \
  "Lq91_%=:\n" \
  "v_mul_f32 v200, s78, v200\n" \
  "v_mul_f32 v201, s78, v201\n" \
  "v_mul_f32 v202, s78, v202\n" \
  "v_mul_f32 v203, s78, v203\n" \
  "s_bitcmp1_b32 s65, 28\n" \
  "s_cbranch_scc1 Lp92_%=\n" \
  "buffer_store_dwordx4 v[200:203], v167, s[40:43], s48 offen offset:64 nt\n" \
  "Lq92_%=:\n" \
  "s_bfe_u32 m0, s66, 0x40018\n" \
  "s_nop 0\n" \
  "s_movreld_b32 s56, s77\n" \
  "s_branch Ls0_8_%=\n" \
  "Lsuf0_1_1_%=:\n" \
  "buffer_load_dwordx4 v[100:103], v163, s[36:39], s75 offen offset:1024\n" \
  "v_readfirstlane_b32 s89, v218\n" \
  "s_nop 1\n" \
  "s_bitcmp1_b32 s65, 18\n" \
  "s_cselect_b32 s42, s72, 0\n" \
  "s_mov_b32 s48, s71\n" \
  "s_bitcmp1_b32 s65, 28\n" \
  "s_cselect_b32 s73, 3, 0x100\n" \
  "s_lshl_b32 s73, s73, 12\n" \
  "s_add_u32 s49, s48, s73\n" \
  "s_add_u32 s50, s49, s73\n" \
  "s_add_u32 s51, s50, s73\n" \
  "s_cmp_eq_u32 s70, 64\n" \
  "s_cbranch_scc1 Lp93_%=\n" \
  "Lq93_%=:\n" \
  "buffer_load_dwordx4 v[96:99], v163, s[36:39], s75 offen offset:0\n" \
  "s_max_u32 s68, s68, s89\n" \
  "v_readlane_b32 s67, v175, s70\n" \
  "v_readlane_b32 s66, v178, s70\n" \
  "v_readlane_b32 s76, v169, s70\n" \
  "v_readlane_b32 s77, v172, s70\n" \
  "s_add_u32 s70, s70, 1\n" \
  "v_add_u32 v165, s76, v162\n" \
  "buffer_load_dwordx4 v[108:111], v163, s[36:39], s75 offen offset:3072\n" \
  "v_add_u32 v166, 4096, v165\n" \
  "s_cmp_gt_u32 s67, s68\n" \
  "s_cbranch_scc1 Lp94_%=\n" \
  "Lq94_%=:\n" \
  "buffer_load_dwordx4 v[104:107], v163, s[36:39], s75 offen offset:2048\n" \
  "buffer_load_dwordx4 v[112:115], v164, s[36:39], s75 offen offset:0\n" \
  "buffer_load_dwordx4 v[120:123], v164, s[36:39], s75 offen offset:2048\n" \
  "s_and_b32 s73, s66, 0xf00001ff\n" \
  "buffer_load_dwordx4 v[116:119], v164, s[36:39], s75 offen offset:1024\n" \
  "buffer_load_dwordx4 v[124:127], v164, s[36:39], s75 offen offset:3072\n" \
  "v_mul_f32 v184, s78, v184\n" \
  "v_mul_f32 v185, s78, v185\n" \
  "v_mul_f32 v186, s78, v186\n" \
  "v_mul_f32 v187, s78, v187\n" \
  "s_bitcmp1_b32 s65, 28\n" \
  "s_cbranch_scc1 Lp95_%=\n" \
  "buffer_store_dwordx4 v[184:187], v167, s[40:43], s48 offen offset:0 nt\n" \
  "Lq95_%=:\n" \
  "v_mul_f32 v200, s78, v200\n" \
  "v_mul_f32 v201, s78, v201\n" \
  "v_mul_f32 v202, s78, v202\n" \
  "v_mul_f32 v203, s78, v203\n" \
  "s_bitcmp1_b32 s65, 28\n" \
  "s_cbranch_scc1 Lp96_%=\n" \
  "buffer_store_dwordx4 v[200:203], v167, s[40:43], s48 offen offset:64 nt\n" \
  "Lq96_%=:\n" \
  "s_bfe_u32 m0, s66, 0x40018\n" \
  "s_nop 0\n" \
  "s_movreld_b32 s56, s77\n" \
  "s_branch Ls0_8_%=\n" \
  "Lsuf0_1_2_%=:\n" \
  "buffer_load_dwordx4 v[100:103], v163, s[36:39], s75 offen offset:1024\n" \
  "v_readfirstlane_b32 s89, v218\n" \
  "s_nop 1\n" \
  "s_bitcmp1_b32 s65, 18\n" \
  "s_cselect_b32 s42, s72, 0\n" \
  "s_mov_b32 s48, s71\n" \
  "s_bitcmp1_b32 s65, 28\n" \
  "s_cselect_b32 s73, 3, 0x100\n" \
  "s_lshl_b32 s73, s73, 12\n" \
  "s_add_u32 s49, s48, s73\n" \
  "s_add_u32 s50, s49, s73\n" \
  "s_add_u32 s51, s50, s73\n" \
  "s_cmp_eq_u32 s70, 64\n" \
  "s_cbranch_scc1 Lp97_%=\n" \
  "Lq97_%=:\n" \
  "buffer_load_dwordx4 v[96:99], v163, s[36:39], s75 offen offset:0\n" \
  "s_max_u32 s68, s68, s89\n" \
  "v_readlane_b32 s67, v175, s70\n" \
  "v_readlane_b32 s66, v178, s70\n" \
  "v_readlane_b32 s76, v169, s70\n" \
  "v_readlane_b32 s77, v172, s70\n" \
  "s_add_u32 s70, s70, 1\n" \
  "v_add_u32 v165, s76, v162\n" \
  "buffer_load_dwordx4 v[108:111], v163, s[36:39], s75 offen offset:3072\n" \
  "v_add_u32 v166, 4096, v165\n" \
  "s_cmp_gt_u32 s67, s68\n" \
  "s_cbranch_scc1 Lp98_%=\n" \
  "Lq98_%=:\n" \
  "buffer_load_dwordx4 v[104:107], v163, s[36:39], s75 offen offset:2048\n" \
  "buffer_load_dwordx4 v[112:115], v164, s[36:39], s75 offen offset:0\n" \
  "buffer_load_dwordx4 v[120:123], v164, s[36:39], s75 offen offset:2048\n" \
  "s_and_b32 s73, s66, 0xf00001ff\n" \
  "buffer_load_dwordx4 v[116:119], v164, s[36:39], s75 offen offset:1024\n" \
  "buffer_load_dwordx4 v[124:127], v164, s[36:39], s75 offen offset:3072\n" \
  "v_mul_f32 v184, s78, v184\n" \
  "v_mul_f32 v185, s78, v185\n" \
  "v_mul_f32 v186, s78, v186\n" \
  "v_mul_f32 v187, s78, v187\n" \
  "s_bitcmp1_b32 s65, 28\n" \
  "s_cbranch_scc1 Lp99_%=\n" \
  "buffer_store_dwordx4 v[184:187], v167, s[40:43], s48 offen offset:0 nt\n" \
  "Lq99_%=:\n" \
  "v_mul_f32 v200, s78, v200\n" \
  "v_mul_f32 v201, s78, v201\n" \
  "v_mul_f32 v202, s78, v202\n" \
  "v_mul_f32 v203, s78, v203\n" \
  "s_bitcmp1_b32 s65, 28\n" \
  "s_cbranch_scc1 Lp100_%=\n" \
  "buffer_store_dwordx4 v[200:203], v167, s[40:43], s48 offen offset:64 nt\n" \
  "Lq100_%=:\n" \
  "s_bfe_u32 m0, s66, 0x40018\n" \
  "s_nop 0\n" \
  "s_movreld_b32 s56, s77\n" \
  "s_branch Ls0_8_%=\n" \
  "Lsuf0_1_3_%=:\n" \
  "buffer_load_dwordx4 v[100:103], v163, s[36:39], s75 offen offset:1024\n" \
  "v_readfirstlane_b32 s89, v218\n" \
  "s_nop 1\n" \
  "s_bitcmp1_b32 s65, 18\n" \
  "s_cselect_b32 s42, s72, 0\n" \
  "s_mov_b32 s48, s71\n" \
  "s_bitcmp1_b32 s65, 28\n" \
  "s_cselect_b32 s73, 3, 0x100\n" \
  "s_lshl_b32 s73, s73, 12\n" \
  "s_add_u32 s49, s48, s73\n" \
  "s_add_u32 s50, s49, s73\n" \
  "s_add_u32 s51, s50, s73\n" \
  "s_cmp_eq_u32 s70, 64\n" \
  "s_cbranch_scc1 Lp101_%=\n" \
  "Lq101_%=:\n" \
  "buffer_load_dwordx4 v[96:99], v163, s[36:39], s75 offen offset:0\n" \
  "s_max_u32 s68, s68, s89\n" \
  "v_readlane_b32 s67, v175, s70\n" \
  "v_readlane_b32 s66, v178, s70\n" \
  "v_readlane_b32 s76, v169, s70\n" \
  "v_readlane_b32 s77, v172, s70\n" \
  "s_add_u32 s70, s70, 1\n" \
  "v_add_u32 v165, s76, v162\n" \
  "buffer_load_dwordx4 v[108:111], v163, s[36:39], s75 offen offset:3072\n" \
  "v_add_u32 v166, 4096, v165\n" \
  "s_cmp_gt_u32 s67, s68\n" \
  "s_cbranch_scc1 Lp102_%=\n" \
  "Lq102_%=:\n" \
  "buffer_load_dwordx4 v[104:107], v163, s[36:39], s75 offen offset:2048\n" \
  "buffer_load_dwordx4 v[112:115], v164, s[36:39], s75 offen offset:0\n" \
  "buffer_load_dwordx4 v[120:123], v164, s[36:39], s75 offen offset:2048\n" \
  "s_and_b32 s73, s66, 0xf00001ff\n" \
  "buffer_load_dwordx4 v[116:119], v164, s[36:39], s75 offen offset:1024\n" \
  "buffer_load_dwordx4 v[124:127], v164, s[36:39], s75 offen offset:3072\n" \
  "v_mul_f32 v184, s78, v184\n" \
  "v_mul_f32 v185, s78, v185\n" \
  "v_mul_f32 v186, s78, v186\n" \
  "v_mul_f32 v187, s78, v187\n" \
  "s_bitcmp1_b32 s65, 28\n" \
  "s_cbranch_scc1 Lp103_%=\n" \
  "buffer_store_dwordx4 v[184:187], v167, s[40:43], s48 offen offset:0 nt\n" \
  "Lq103_%=:\n" \
  "v_mul_f32 v200, s78, v200\n" \
  "v_mul_f32 v201, s78, v201\n" \
  "v_mul_f32 v202, s78, v202\n" \
  "v_mul_f32 v203, s78, v203\n" \
  "s_bitcmp1_b32 s65, 28\n" \
  "s_cbranch_scc1 Lp104_%=\n" \
  "buffer_store_dwordx4 v[200:203], v167, s[40:43], s48 offen offset:64 nt\n" \
  "Lq104_%=:\n" \
  "s_bfe_u32 m0, s66, 0x40018\n" \
  "s_nop 0\n" \
  "s_movreld_b32 s56, s77\n" \
  "s_branch Ls0_8_%=\n" \
  "Lsuf0_1_4_%=:\n" \
  "buffer_load_dwordx4 v[100:103], v163, s[36:39], s75 offen offset:1024\n" \
  "v_readfirstlane_b32 s89, v218\n" \
  "s_nop 1\n" \
  "s_bitcmp1_b32 s65, 18\n" \
  "s_cselect_b32 s42, s72, 0\n" \
  "s_mov_b32 s48, s71\n" \
  "s_bitcmp1_b32 s65, 28\n" \
  "s_cselect_b32 s73, 3, 0x100\n" \
  "s_lshl_b32 s73, s73, 12\n" \
  "s_add_u32 s49, s48, s73\n" \
  "s_add_u32 s50, s49, s73\n" \
  "s_add_u32 s51, s50, s73\n" \
  "s_cmp_eq_u32 s70, 64\n" \
  "s_cbranch_scc1 Lp105_%=\n" \
  "Lq105_%=:\n" \
  "buffer_load_dwordx4 v[96:99], v163, s[36:39], s75 offen offset:0\n" \
  "s_max_u32 s68, s68, s89\n" \
  "v_readlane_b32 s67, v175, s70\n" \
  "v_readlane_b32 s66, v178, s70\n" \
  "v_readlane_b32 s76, v169, s70\n" \
  "v_readlane_b32 s77, v172, s70\n" \
  "s_add_u32 s70, s70, 1\n" \
  "v_add_u32 v165, s76, v162\n" \
  "buffer_load_dwordx4 v[108:111], v163, s[36:39], s75 offen offset:3072\n" \
  "v_add_u32 v166, 4096, v165\n" \
  "s_cmp_gt_u32 s67, s68\n" \
  "s_cbranch_scc1 Lp106_%=\n" \
  "Lq106_%=:\n" \
  "buffer_load_dwordx4 v[104:107], v163, s[36:39], s75 offen offset:2048\n" \
  "buffer_load_dwordx4 v[112:115], v164, s[36:39], s75 offen offset:0\n" \
  "buffer_load_dwordx4 v[120:123], v164, s[36:39], s75 offen offset:2048\n" \
  "s_and_b32 s73, s66, 0xf00001ff\n" \
  "buffer_load_dwordx4 v[116:119], v164, s[36:39], s75 offen offset:1024\n" \
  "buffer_load_dwordx4 v[124:127], v164, s[36:39], s75 offen offset:3072\n" \
  "v_mul_f32 v184, s78, v184\n" \
  "v_mul_f32 v185, s78, v185\n" \
  "v_mul_f32 v186, s78, v186\n" \
  "v_mul_f32 v187, s78, v187\n" \
  "s_bitcmp1_b32 s65, 28\n" \
  "s_cbranch_scc1 Lp107_%=\n" \
  "buffer_store_dwordx4 v[184:187], v167, s[40:43], s48 offen offset:0 nt\n" \
  "Lq107_%=:\n" \
  "v_mul_f32 v200, s78, v200\n" \
  "v_mul_f32 v201, s78, v201\n" \
  "v_mul_f32 v202, s78, v202\n" \
  "v_mul_f32 v203, s78, v203\n" \
  "s_bitcmp1_b32 s65, 28\n" \
  "s_cbranch_scc1 Lp108_%=\n" \
  "buffer_store_dwordx4 v[200:203], v167, s[40:43], s48 offen offset:64 nt\n" \
  "Lq108_%=:\n" \
  "s_bfe_u32 m0, s66, 0x40018\n" \
  "s_nop 0\n" \
  "s_movreld_b32 s56, s77\n" \
  "s_branch Ls0_8_%=\n" \
  "Lsuf0_1_5_%=:\n" \
  "buffer_load_dwordx4 v[108:111], v163, s[36:39], s75 offen offset:3072\n" \
  "v_add_u32 v166, 4096, v165\n" \
  "s_cmp_gt_u32 s67, s68\n" \
  "s_cbranch_scc1 Lp109_%=\n" \
  "Lq109_%=:\n" \
  "buffer_load_dwordx4 v[104:107], v163, s[36:39], s75 offen offset:2048\n" \
  "buffer_load_dwordx4 v[112:115], v164, s[36:39], s75 offen offset:0\n" \
  "buffer_load_dwordx4 v[120:123], v164, s[36:39], s75 offen offset:2048\n" \
  "s_and_b32 s73, s66, 0xf00001ff\n" \
  "buffer_load_dwordx4 v[116:119], v164, s[36:39], s75 offen offset:1024\n" \
  "buffer_load_dwordx4 v[124:127], v164, s[36:39], s75 offen offset:3072\n" \
  "v_mul_f32 v184, s78, v184\n" \
  "v_mul_f32 v185, s78, v185\n" \
  "v_mul_f32 v186, s78, v186\n" \
  "v_mul_f32 v187, s78, v187\n" \
  "s_bitcmp1_b32 s65, 28\n" \
  "s_cbranch_scc1 Lp110_%=\n" \
  "buffer_store_dwordx4 v[184:187], v167, s[40:43], s48 offen offset:0 nt\n" \
  "Lq110_%=:\n" \
  "v_mul_f32 v200, s78, v200\n" \
  "v_mul_f32 v201, s78, v201\n" \
  "v_mul_f32 v202, s78, v202\n" \
  "v_mul_f32 v203, s78, v203\n" \
  "s_bitcmp1_b32 s65, 28\n" \
  "s_cbranch_scc1 Lp111_%=\n" \
  "buffer_store_dwordx4 v[200:203], v167, s[40:43], s48 offen offset:64 nt\n" \
  "Lq111_%=:\n" \
  "s_bfe_u32 m0, s66, 0x40018\n" \
  "s_nop 0\n" \
  "s_movreld_b32 s56, s77\n" \
  "s_branch Ls0_8_%=\n" \
  "Lsuf0_1_6_%=:\n" \
  "buffer_load_dwordx4 v[120:123], v164, s[36:39], s75 offen offset:2048\n" \
  "s_and_b32 s73, s66, 0xf00001ff\n" \
  "buffer_load_dwordx4 v[116:119], v164, s[36:39], s75 offen offset:1024\n" \
  "buffer_load_dwordx4 v[124:127], v164, s[36:39], s75 offen offset:3072\n" \
  "v_mul_f32 v184, s78, v184\n" \
  "v_mul_f32 v185, s78, v185\n" \
  "v_mul_f32 v186, s78, v186\n" \
  "v_mul_f32 v187, s78, v187\n" \
  "s_bitcmp1_b32 s65, 28\n" \
  "s_cbranch_scc1 Lp112_%=\n" \
  "buffer_store_dwordx4 v[184:187], v167, s[40:43], s48 offen offset:0 nt\n" \
  "Lq112_%=:\n" \
  "v_mul_f32 v200, s78, v200\n" \
  "v_mul_f32 v201, s78, v201\n" \
  "v_mul_f32 v202, s78, v202\n" \
  "v_mul_f32 v203, s78, v203\n" \
  "s_bitcmp1_b32 s65, 28\n" \
  "s_cbranch_scc1 Lp113_%=\n" \
  "buffer_store_dwordx4 v[200:203], v167, s[40:43], s48 offen offset:64 nt\n" \
  "Lq113_%=:\n" \
  "s_bfe_u32 m0, s66, 0x40018\n" \
  "s_nop 0\n" \
  "s_movreld_b32 s56, s77\n" \
  "s_branch Ls0_8_%=\n" \
  ".p2align 6\n" \
  "Lpre0_0_0_%=:\n" \
  "s_branch Ls0_1_%=\n" \
  ".org Lpre0_0_0_%=+704\n" \
  "Lpre0_0_1_%=:\n" \
  "s_branch Ls0_1_%=\n" \
  ".org Lpre0_0_0_%=+1408\n" \
  "Lpre0_0_2_%=:\n" \
  "ds_read_b128 v[144:147], v160 offset:4096\n" \
  "ds_read_b128 v[148:151], v160 offset:5120\n" \
  "ds_read_b128 v[152:155], v160 offset:6144\n" \
  "ds_read_b128 v[156:159], v160 offset:7168\n" \
  "s_add_u32 m0, s69, 98304\n" \
  "s_nop 0\n" \
  "buffer_load_dwordx4 v163, s[36:39], s56 offen lds\n" \
  "s_branch Ls0_2_%=\n" \
  ".org Lpre0_0_0_%=+2112\n" \
  "Lpre0_0_3_%=:\n" \
  "ds_read_b128 v[128:131], v160 offset:8192\n" \
  "ds_read_b128 v[132:135], v160 offset:9216\n" \
  "ds_read_b128 v[136:139], v160 offset:10240\n" \
  "ds_read_b128 v[140:143], v160 offset:11264\n" \
  "s_add_u32 m0, s69, 98304\n" \
  "s_nop 0\n" \
  "buffer_load_dwordx4 v163, s[36:39], s56 offen lds\n" \
  "s_add_u32 m0, s69, 102400\n" \
  "s_nop 0\n" \
  "buffer_load_dwordx4 v163, s[36:39], s57 offen lds\n" \
  "s_branch Ls0_3_%=\n" \
  ".org Lpre0_0_0_%=+2816\n" \
  "Lpre0_0_4_%=:\n" \
  "ds_read_b128 v[144:147], v160 offset:12288\n" \
  "ds_read_b128 v[148:151], v160 offset:13312\n" \
  "ds_read_b128 v[152:155], v160 offset:14336\n" \
  "ds_read_b128 v[156:159], v160 offset:15360\n" \
  "s_add_u32 m0, s69, 98304\n" \
  "s_nop 0\n" \
  "buffer_load_dwordx4 v163, s[36:39], s56 offen lds\n" \
  "s_add_u32 m0, s69, 102400\n" \
  "s_nop 0\n" \
  "buffer_load_dwordx4 v163, s[36:39], s57 offen lds\n" \
  "s_add_u32 m0, s69, 106496\n" \
  "s_nop 0\n" \
  "buffer_load_dwordx4 v163, s[36:39], s58 offen lds\n" \
  "s_branch Ls0_4_%=\n" \
  ".org Lpre0_0_0_%=+3520\n" \
  "Lpre0_0_5_%=:\n" \
  "ds_read_b128 v[128:131], v160 offset:16384\n" \
  "ds_read_b128 v[132:135], v160 offset:17408\n" \
  "ds_read_b128 v[136:139], v160 offset:18432\n" \
  "ds_read_b128 v[140:143], v160 offset:19456\n" \
  "s_add_u32 m0, s69, 98304\n" \
  "s_nop 0\n" \
  "buffer_load_dwordx4 v163, s[36:39], s56 offen lds\n" \
  "s_add_u32 m0, s69, 102400\n" \
  "s_nop 0\n" \
  "buffer_load_dwordx4 v163, s[36:39], s57 offen lds\n" \
  "s_add_u32 m0, s69, 106496\n" \
  "s_nop 0\n" \
  "buffer_load_dwordx4 v163, s[36:39], s58 offen lds\n" \
  "s_add_u32 m0, s69, 110592\n" \
  "s_nop 0\n" \
  "buffer_load_dwordx4 v163, s[36:39], s59 offen lds\n" \
  "s_branch Ls0_5_%=\n" \
  ".org Lpre0_0_0_%=+4224\n" \
  "Lpre0_0_6_%=:\n" \
  "ds_read_b128 v[144:147], v160 offset:20480\n" \
  "ds_read_b128 v[148:151], v160 offset:21504\n" \
  "ds_read_b128 v[152:155], v160 offset:22528\n" \
  "ds_read_b128 v[156:159], v160 offset:23552\n" \
  "s_add_u32 m0, s69, 98304\n" \
  "s_nop 0\n" \
  "buffer_load_dwordx4 v163, s[36:39], s56 offen lds\n" \
  "s_add_u32 m0, s69, 102400\n" \
  "s_nop 0\n" \
  "buffer_load_dwordx4 v163, s[36:39], s57 offen lds\n" \
  "s_add_u32 m0, s69, 106496\n" \
  "s_nop 0\n" \
  "buffer_load_dwordx4 v163, s[36:39], s58 offen lds\n" \
  "s_add_u32 m0, s69, 110592\n" \
  "s_nop 0\n" \
  "buffer_load_dwordx4 v163, s[36:39], s59 offen lds\n" \
  "s_add_u32 m0, s69, 114688\n" \
  "s_nop 0\n" \
  "buffer_load_dwordx4 v163, s[36:39], s60 offen lds\n" \
  "buffer_load_dwordx4 v[100:103], v163, s[36:39], s75 offen offset:1024\n" \
  "v_readfirstlane_b32 s89, v218\n" \
  "s_nop 1\n" \
  "s_bitcmp1_b32 s65, 18\n" \
  "s_cselect_b32 s42, s72, 0\n" \
  "s_mov_b32 s48, s71\n" \
  "s_bitcmp1_b32 s65, 28\n" \
  "s_cselect_b32 s73, 3, 0x100\n" \
  "s_lshl_b32 s73, s73, 12\n" \
  "s_add_u32 s49, s48, s73\n" \
  "s_add_u32 s50, s49, s73\n" \
  "s_add_u32 s51, s50, s73\n" \
  "s_cmp_eq_u32 s70, 64\n" \
  "s_cbranch_scc1 Lp114_%=\n" \
  "Lq114_%=:\n" \
  "buffer_load_dwordx4 v[96:99], v163, s[36:39], s75 offen offset:0\n" \
  "s_max_u32 s68, s68, s89\n" \
  "v_readlane_b32 s67, v175, s70\n" \
  "v_readlane_b32 s66, v178, s70\n" \
  "v_readlane_b32 s76, v169, s70\n" \
  "v_readlane_b32 s77, v172, s70\n" \
  "s_add_u32 s70, s70, 1\n" \
  "v_add_u32 v165, s76, v162\n" \
  "s_branch Ls0_6_%=\n" \
  ".org Lpre0_0_0_%=+4928\n" \
  "Lpre0_0_7_%=:\n" \
  "ds_read_b128 v[128:131], v160 offset:24576\n" \
  "ds_read_b128 v[132:135], v160 offset:25600\n" \
  "ds_read_b128 v[136:139], v160 offset:26624\n" \
  "ds_read_b128 v[140:143], v160 offset:27648\n" \
  "s_add_u32 m0, s69, 98304\n" \
  "s_nop 0\n" \
  "buffer_load_dwordx4 v163, s[36:39], s56 offen lds\n" \
  "s_add_u32 m0, s69, 102400\n" \
  "s_nop 0\n" \
  "buffer_load_dwordx4 v163, s[36:39], s57 offen lds\n" \
  "s_add_u32 m0, s69, 106496\n" \
  "s_nop 0\n" \
  "buffer_load_dwordx4 v163, s[36:39], s58 offen lds\n" \
  "s_add_u32 m0, s69, 110592\n" \
  "s_nop 0\n" \
  "buffer_load_dwordx4 v163, s[36:39], s59 offen lds\n" \
  "s_add_u32 m0, s69, 114688\n" \
  "s_nop 0\n" \
  "buffer_load_dwordx4 v163, s[36:39], s60 offen lds\n" \
  "buffer_load_dwordx4 v[100:103], v163, s[36:39], s75 offen offset:1024\n" \
  "v_readfirstlane_b32 s89, v218\n" \
  "s_nop 1\n" \
  "s_bitcmp1_b32 s65, 18\n" \
  "s_cselect_b32 s42, s72, 0\n" \
  "s_mov_b32 s48, s71\n" \
  "s_bitcmp1_b32 s65, 28\n" \
  "s_cselect_b32 s73, 3, 0x100\n" \
  "s_lshl_b32 s73, s73, 12\n" \
  "s_add_u32 s49, s48, s73\n" \
  "s_add_u32 s50, s49, s73\n" \
  "s_add_u32 s51, s50, s73\n" \
  "s_cmp_eq_u32 s70, 64\n" \
  "s_cbranch_scc1 Lp115_%=\n" \
  "Lq115_%=:\n" \
  "buffer_load_dwordx4 v[96:99], v163, s[36:39], s75 offen offset:0\n" \
  "s_max_u32 s68, s68, s89\n" \
  "v_readlane_b32 s67, v175, s70\n" \
  "v_readlane_b32 s66, v178, s70\n" \
  "v_readlane_b32 s76, v169, s70\n" \
  "v_readlane_b32 s77, v172, s70\n" \
  "s_add_u32 s70, s70, 1\n" \
  "v_add_u32 v165, s76, v162\n" \
  "s_add_u32 m0, s69, 118784\n" \
  "s_nop 0\n" \
  "buffer_load_dwordx4 v163, s[36:39], s61 offen lds\n" \
  "buffer_load_dwordx4 v[108:111], v163, s[36:39], s75 offen offset:3072\n" \
  "v_add_u32 v166, 4096, v165\n" \
  "s_cmp_gt_u32 s67, s68\n" \
  "s_cbranch_scc1 Lp116_%=\n" \
  "Lq116_%=:\n" \
  "buffer_load_dwordx4 v[104:107], v163, s[36:39], s75 offen offset:2048\n" \
  "buffer_load_dwordx4 v[112:115], v164, s[36:39], s75 offen offset:0\n" \
  "s_branch Ls0_7_%=\n" \
  ".org Lpre0_0_0_%=+5632\n" \
  "Lpre0_0_8_%=:\n" \
  "ds_read_b128 v[144:147], v160 offset:28672\n" \
  "ds_read_b128 v[148:151], v160 offset:29696\n" \
  "ds_read_b128 v[152:155], v160 offset:30720\n" \
  "ds_read_b128 v[156:159], v160 offset:31744\n" \
  "s_add_u32 m0, s69, 98304\n" \
  "s_nop 0\n" \
  "buffer_load_dwordx4 v163, s[36:39], s56 offen lds\n" \
  "s_add_u32 m0, s69, 102400\n" \
  "s_nop 0\n" \
  "buffer_load_dwordx4 v163, s[36:39], s57 offen lds\n" \
  "s_add_u32 m0, s69, 106496\n" \
  "s_nop 0\n" \
  "buffer_load_dwordx4 v163, s[36:39], s58 offen lds\n" \
  "s_add_u32 m0, s69, 110592\n" \
  "s_nop 0\n" \
  "buffer_load_dwordx4 v163, s[36:39], s59 offen lds\n" \
  "s_add_u32 m0, s69, 114688\n" \
  "s_nop 0\n" \
  "buffer_load_dwordx4 v163, s[36:39], s60 offen lds\n" \
  "buffer_load_dwordx4 v[100:103], v163, s[36:39], s75 offen offset:1024\n" \
  "v_readfirstlane_b32 s89, v218\n" \
  "s_nop 1\n" \
  "s_bitcmp1_b32 s65, 18\n" \
  "s_cselect_b32 s42, s72, 0\n" \
  "s_mov_b32 s48, s71\n" \
  "s_bitcmp1_b32 s65, 28\n" \
  "s_cselect_b32 s73, 3, 0x100\n" \
  "s_lshl_b32 s73, s73, 12\n" \
  "s_add_u32 s49, s48, s73\n" \
  "s_add_u32 s50, s49, s73\n" \
  "s_add_u32 s51, s50, s73\n" \
  "s_cmp_eq_u32 s70, 64\n" \
  "s_cbranch_scc1 Lp117_%=\n" \
  "Lq117_%=:\n" \
  "buffer_load_dwordx4 v[96:99], v163, s[36:39], s75 offen offset:0\n" \
  "s_max_u32 s68, s68, s89\n" \
  "v_readlane_b32 s67, v175, s70\n" \
  "v_readlane_b32 s66, v178, s70\n" \
  "v_readlane_b32 s76, v169, s70\n" \
  "v_readlane_b32 s77, v172, s70\n" \
  "s_add_u32 s70, s70, 1\n" \
  "v_add_u32 v165, s76, v162\n" \
  "s_add_u32 m0, s69, 118784\n" \
  "s_nop 0\n" \
  "buffer_load_dwordx4 v163, s[36:39], s61 offen lds\n" \
  "buffer_load_dwordx4 v[108:111], v163, s[36:39], s75 offen offset:3072\n" \
  "v_add_u32 v166, 4096, v165\n" \
  "s_cmp_gt_u32 s67, s68\n" \
  "s_cbranch_scc1 Lp118_%=\n" \
  "Lq118_%=:\n" \
  "buffer_load_dwordx4 v[104:107], v163, s[36:39], s75 offen offset:2048\n" \
  "buffer_load_dwordx4 v[112:115], v164, s[36:39], s75 offen offset:0\n" \
  "s_add_u32 m0, s69, 122880\n" \
  "s_nop 0\n" \
  "buffer_load_dwordx4 v163, s[36:39], s62 offen lds\n" \
  "buffer_load_dwordx4 v[120:123], v164, s[36:39], s75 offen offset:2048\n" \
  "s_and_b32 s73, s66, 0xf00001ff\n" \
  "buffer_load_dwordx4 v[116:119], v164, s[36:39], s75 offen offset:1024\n" \
  "buffer_load_dwordx4 v[124:127], v164, s[36:39], s75 offen offset:3072\n" \
  "v_mul_f32 v184, s78, v184\n" \
  "v_mul_f32 v185, s78, v185\n" \
  "v_mul_f32 v186, s78, v186\n" \
  "v_mul_f32 v187, s78, v187\n" \
  "s_bitcmp1_b32 s65, 28\n" \
  "s_cbranch_scc1 Lp119_%=\n" \
  "buffer_store_dwordx4 v[184:187], v167, s[40:43], s48 offen offset:0 nt\n" \
  "Lq119_%=:\n" \
  "v_mul_f32 v200, s78, v200\n" \
  "v_mul_f32 v201, s78, v201\n" \
  "v_mul_f32 v202, s78, v202\n" \
  "v_mul_f32 v203, s78, v203\n" \
  "s_bitcmp1_b32 s65, 28\n" \
  "s_cbranch_scc1 Lp120_%=\n" \
  "buffer_store_dwordx4 v[200:203], v167, s[40:43], s48 offen offset:64 nt\n" \
  "Lq120_%=:\n" \
  "s_bfe_u32 m0, s66, 0x40018\n" \
  "s_nop 0\n" \
  "s_movreld_b32 s56, s77\n" \
  "s_branch Ls0_8_%=\n" \
  ".org Lpre0_0_0_%=+6336\n" \
  "Lpre0_1_0_%=:\n" \
  "s_branch Ls0_1_%=\n" \
  ".org Lpre0_0_0_%=+7040\n" \
  "Lpre0_1_1_%=:\n" \
  "s_branch Ls0_1_%=\n" \
  ".org Lpre0_0_0_%=+7744\n" \
  "Lpre0_1_2_%=:\n" \
  "ds_read_b128 v[144:147], v160 offset:4096\n" \
  "ds_read_b128 v[148:151], v160 offset:5120\n" \
  "ds_read_b128 v[152:155], v160 offset:6144\n" \
  "ds_read_b128 v[156:159], v160 offset:7168\n" \
  "s_branch Ls0_2_%=\n" \
  ".org Lpre0_0_0_%=+8448\n" \
  "Lpre0_1_3_%=:\n" \
  "ds_read_b128 v[128:131], v160 offset:8192\n" \
  "ds_read_b128 v[132:135], v160 offset:9216\n" \
  "ds_read_b128 v[136:139], v160 offset:10240\n" \
  "ds_read_b128 v[140:143], v160 offset:11264\n" \
  "s_branch Ls0_3_%=\n" \
  ".org Lpre0_0_0_%=+9152\n" \
  "Lpre0_1_4_%=:\n" \
  "ds_read_b128 v[144:147], v160 offset:12288\n" \
  "ds_read_b128 v[148:151], v160 offset:13312\n" \
  "ds_read_b128 v[152:155], v160 offset:14336\n" \
  "ds_read_b128 v[156:159], v160 offset:15360\n" \
  "s_branch Ls0_4_%=\n" \
  ".org Lpre0_0_0_%=+9856\n" \
  "Lpre0_1_5_%=:\n" \
  "ds_read_b128 v[128:131], v160 offset:16384\n" \
  "ds_read_b128 v[132:135], v160 offset:17408\n" \
  "ds_read_b128 v[136:139], v160 offset:18432\n" \
  "ds_read_b128 v[140:143], v160 offset:19456\n" \
  "s_branch Ls0_5_%=\n" \
  ".org Lpre0_0_0_%=+10560\n" \
  "Lpre0_1_6_%=:\n" \
  "ds_read_b128 v[144:147], v160 offset:20480\n" \
  "ds_read_b128 v[148:151], v160 offset:21504\n" \
  "ds_read_b128 v[152:155], v160 offset:22528\n" \
  "ds_read_b128 v[156:159], v160 offset:23552\n" \
  "buffer_load_dwordx4 v[100:103], v163, s[36:39], s75 offen offset:1024\n" \
  "v_readfirstlane_b32 s89, v218\n" \
  "s_nop 1\n" \
  "s_bitcmp1_b32 s65, 18\n" \
  "s_cselect_b32 s42, s72, 0\n" \
  "s_mov_b32 s48, s71\n" \
  "s_bitcmp1_b32 s65, 28\n" \
  "s_cselect_b32 s73, 3, 0x100\n" \
  "s_lshl_b32 s73, s73, 12\n" \
  "s_add_u32 s49, s48, s73\n" \
  "s_add_u32 s50, s49, s73\n" \
  "s_add_u32 s51, s50, s73\n" \
  "s_cmp_eq_u32 s70, 64\n" \
  "s_cbranch_scc1 Lp121_%=\n" \
  "Lq121_%=:\n" \
  "buffer_load_dwordx4 v[96:99], v163, s[36:39], s75 offen offset:0\n" \
  "s_max_u32 s68, s68, s89\n" \
  "v_readlane_b32 s67, v175, s70\n" \
  "v_readlane_b32 s66, v178, s70\n" \
  "v_readlane_b32 s76, v169, s70\n" \
  "v_readlane_b32 s77, v172, s70\n" \
  "s_add_u32 s70, s70, 1\n" \
  "v_add_u32 v165, s76, v162\n" \
  "s_branch Ls0_6_%=\n" \
  ".org Lpre0_0_0_%=+11264\n" \
  "Lpre0_1_7_%=:\n" \
  "ds_read_b128 v[128:131], v160 offset:24576\n" \
  "ds_read_b128 v[132:135], v160 offset:25600\n" \
  "ds_read_b128 v[136:139], v160 offset:26624\n" \
  "ds_read_b128 v[140:143], v160 offset:27648\n" \
  "buffer_load_dwordx4 v[100:103], v163, s[36:39], s75 offen offset:1024\n" \
  "v_readfirstlane_b32 s89, v218\n" \
  "s_nop 1\n" \
  "s_bitcmp1_b32 s65, 18\n" \
  "s_cselect_b32 s42, s72, 0\n" \
  "s_mov_b32 s48, s71\n" \
  "s_bitcmp1_b32 s65, 28\n" \
  "s_cselect_b32 s73, 3, 0x100\n" \
  "s_lshl_b32 s73, s73, 12\n" \
  "s_add_u32 s49, s48, s73\n" \
  "s_add_u32 s50, s49, s73\n" \
  "s_add_u32 s51, s50, s73\n" \
  "s_cmp_eq_u32 s70, 64\n" \
  "s_cbranch_scc1 Lp122_%=\n" \
  "Lq122_%=:\n" \
  "buffer_load_dwordx4 v[96:99], v163, s[36:39], s75 offen offset:0\n" \
  "s_max_u32 s68, s68, s89\n" \
  "v_readlane_b32 s67, v175, s70\n" \
  "v_readlane_b32 s66, v178, s70\n" \
  "v_readlane_b32 s76, v169, s70\n" \
  "v_readlane_b32 s77, v172, s70\n" \
  "s_add_u32 s70, s70, 1\n" \
  "v_add_u32 v165, s76, v162\n" \
  "buffer_load_dwordx4 v[108:111], v163, s[36:39], s75 offen offset:3072\n" \
  "v_add_u32 v166, 4096, v165\n" \
  "s_cmp_gt_u32 s67, s68\n" \
  "s_cbranch_scc1 Lp123_%=\n" \
  "Lq123_%=:\n" \
  "buffer_load_dwordx4 v[104:107], v163, s[36:39], s75 offen offset:2048\n" \
  "buffer_load_dwordx4 v[112:115], v164, s[36:39], s75 offen offset:0\n" \
  "s_branch Ls0_7_%=\n" \
  ".org Lpre0_0_0_%=+11968\n" \
  "Lpre0_1_8_%=:\n" \
  "ds_read_b128 v[144:147], v160 offset:28672\n" \
  "ds_read_b128 v[148:151], v160 offset:29696\n" \
  "ds_read_b128 v[152:155], v160 offset:30720\n" \
  "ds_read_b128 v[156:159], v160 offset:31744\n" \
  "buffer_load_dwordx4 v[100:103], v163, s[36:39], s75 offen offset:1024\n" \
  "v_readfirstlane_b32 s89, v218\n" \
  "s_nop 1\n" \
  "s_bitcmp1_b32 s65, 18\n" \
  "s_cselect_b32 s42, s72, 0\n" \
  "s_mov_b32 s48, s71\n" \
  "s_bitcmp1_b32 s65, 28\n" \
  "s_cselect_b32 s73, 3, 0x100\n" \
  "s_lshl_b32 s73, s73, 12\n" \
  "s_add_u32 s49, s48, s73\n" \
  "s_add_u32 s50, s49, s73\n" \
  "s_add_u32 s51, s50, s73\n" \
  "s_cmp_eq_u32 s70, 64\n" \
  "s_cbranch_scc1 Lp124_%=\n" \
  "Lq124_%=:\n" \
  "buffer_load_dwordx4 v[96:99], v163, s[36:39], s75 offen offset:0\n" \
  "s_max_u32 s68, s68, s89\n" \
  "v_readlane_b32 s67, v175, s70\n" \
  "v_readlane_b32 s66, v178, s70\n" \
  "v_readlane_b32 s76, v169, s70\n" \
  "v_readlane_b32 s77, v172, s70\n" \
  "s_add_u32 s70, s70, 1\n" \
  "v_add_u32 v165, s76, v162\n" \
  "buffer_load_dwordx4 v[108:111], v163, s[36:39], s75 offen offset:3072\n" \
  "v_add_u32 v166, 4096, v165\n" \
  "s_cmp_gt_u32 s67, s68\n" \
  "s_cbranch_scc1 Lp125_%=\n" \
  "Lq125_%=:\n" \
  "buffer_load_dwordx4 v[104:107], v163, s[36:39], s75 offen offset:2048\n" \
  "buffer_load_dwordx4 v[112:115], v164, s[36:39], s75 offen offset:0\n" \
  "buffer_load_dwordx4 v[120:123], v164, s[36:39], s75 offen offset:2048\n" \
  "s_and_b32 s73, s66, 0xf00001ff\n" \
  "buffer_load_dwordx4 v[116:119], v164, s[36:39], s75 offen offset:1024\n" \
  "buffer_load_dwordx4 v[124:127], v164, s[36:39], s75 offen offset:3072\n" \
  "v_mul_f32 v184, s78, v184\n" \
  "v_mul_f32 v185, s78, v185\n" \
  "v_mul_f32 v186, s78, v186\n" \
  "v_mul_f32 v187, s78, v187\n" \
  "s_bitcmp1_b32 s65, 28\n" \
  "s_cbranch_scc1 Lp126_%=\n" \
  "buffer_store_dwordx4 v[184:187], v167, s[40:43], s48 offen offset:0 nt\n" \
  "Lq126_%=:\n" \
  "v_mul_f32 v200, s78, v200\n" \
  "v_mul_f32 v201, s78, v201\n" \
  "v_mul_f32 v202, s78, v202\n" \
  "v_mul_f32 v203, s78, v203\n" \
  "s_bitcmp1_b32 s65, 28\n" \
  "s_cbranch_scc1 Lp127_%=\n" \
  "buffer_store_dwordx4 v[200:203], v167, s[40:43], s48 offen offset:64 nt\n" \
  "Lq127_%=:\n" \
  "s_bfe_u32 m0, s66, 0x40018\n" \
  "s_nop 0\n" \
  "s_movreld_b32 s56, s77\n" \
  "s_branch Ls0_8_%=\n" \
  ".p2align 6\n" \
  "Lz1_%=:\n" \
  "ds_read_b128 v[128:131], v160 offset:32768\n" \
  "ds_read_b128 v[132:135], v160 offset:33792\n" \
  "ds_read_b128 v[136:139], v160 offset:34816\n" \
  "ds_read_b128 v[140:143], v160 offset:35840\n" \
  "s_add_u32 m0, s69, 28672\n" \
  "s_nop 0\n" \
  "buffer_load_dwordx4 v165, s[36:39], s63 offen lds\n" \
  "s_branch Lzr1_%=\n" \
  "Lpj1_%=:\n" \
  "s_setpc_b64 s[98:99]\n" \
  "Lsuf1_0_1_%=:\n" \
  "s_bitcmp1_b32 s66, 10\n" \
  "s_cbranch_scc1 Lsuf1_1_1_%=\n" \
  "ds_read_b128 v[144:147], v160 offset:61440\n" \
  "ds_read_b128 v[148:151], v160 offset:62464\n" \
  "ds_read_b128 v[152:155], v160 offset:63488\n" \
  "ds_read_b128 v[156:159], v160 offset:64512\n" \
  "s_add_u32 m0, s69, 4096\n" \
  "s_nop 0\n" \
  "buffer_load_dwordx4 v165, s[36:39], s57 offen lds\n" \
  "s_add_u32 m0, s69, 8192\n" \
  "s_nop 0\n" \
  "buffer_load_dwordx4 v165, s[36:39], s58 offen lds\n" \
  "s_add_u32 m0, s69, 12288\n" \
  "s_nop 0\n" \
  "buffer_load_dwordx4 v165, s[36:39], s59 offen lds\n" \
  "s_add_u32 m0, s69, 16384\n" \
  "s_nop 0\n" \
  "buffer_load_dwordx4 v165, s[36:39], s60 offen lds\n" \
  "buffer_load_dwordx4 v[4:7], v165, s[36:39], s75 offen offset:1024\n" \
  "buffer_load_dwordx4 v[0:3], v165, s[36:39], s75 offen offset:0\n" \
  "buffer_load_dword v218, off, s[44:47], 0 sc1\n" \
  "v_readlane_b32 s67, v175, s70\n" \
  "v_readlane_b32 s65, v178, s70\n" \
  "v_readlane_b32 s76, v169, s70\n" \
  "v_readlane_b32 s77, v172, s70\n" \
  "s_add_u32 s70, s70, 1\n" \
  "v_add_u32 v163, s76, v162\n" \
  "s_add_u32 m0, s69, 20480\n" \
  "s_nop 0\n" \
  "buffer_load_dwordx4 v165, s[36:39], s61 offen lds\n" \
  "buffer_load_dwordx4 v[12:15], v165, s[36:39], s75 offen offset:3072\n" \
  "v_add_u32 v164, 4096, v163\n" \
  "s_cmp_gt_u32 s67, s68\n" \
  "s_cbranch_scc1 Lp128_%=\n" \
  "Lq128_%=:\n" \
  "buffer_load_dwordx4 v[8:11], v165, s[36:39], s75 offen offset:2048\n" \
  "buffer_load_dwordx4 v[16:19], v166, s[36:39], s75 offen offset:0\n" \
  "v_mul_f32 v188, s78, v188\n" \
  "v_mul_f32 v189, s78, v189\n" \
  "v_mul_f32 v190, s78, v190\n" \
  "v_mul_f32 v191, s78, v191\n" \
  "s_bitcmp1_b32 s66, 28\n" \
  "s_cbranch_scc1 Lp129_%=\n" \
  "buffer_store_dwordx4 v[188:191], v167, s[40:43], s49 offen offset:0 nt\n" \
  "Lq129_%=:\n" \
  "s_add_u32 m0, s69, 24576\n" \
  "s_nop 0\n" \
  "buffer_load_dwordx4 v165, s[36:39], s62 offen lds\n" \
  "buffer_load_dwordx4 v[24:27], v166, s[36:39], s75 offen offset:2048\n" \
  "v_mul_f32 v204, s78, v204\n" \
  "v_mul_f32 v205, s78, v205\n" \
  "v_mul_f32 v206, s78, v206\n" \
  "v_mul_f32 v207, s78, v207\n" \
  "s_bitcmp1_b32 s66, 28\n" \
  "s_cbranch_scc1 Lp130_%=\n" \
  "buffer_store_dwordx4 v[204:207], v167, s[40:43], s49 offen offset:64 nt\n" \
  "Lq130_%=:\n" \
  "s_and_b32 s73, s65, 0xf00001ff\n" \
  "buffer_load_dwordx4 v[20:23], v166, s[36:39], s75 offen offset:1024\n" \
  "buffer_load_dwordx4 v[28:31], v166, s[36:39], s75 offen offset:3072\n" \
  "s_bfe_u32 m0, s65, 0x40018\n" \
  "s_nop 0\n" \
  "s_movreld_b32 s56, s77\n" \
  "s_branch Ls1_8_%=\n" \
  "Lsuf1_0_2_%=:\n" \
  "s_bitcmp1_b32 s66, 10\n" \
  "s_cbranch_scc1 Lsuf1_1_2_%=\n" \
  "ds_read_b128 v[144:147], v160 offset:61440\n" \
  "ds_read_b128 v[148:151], v160 offset:62464\n" \
  "ds_read_b128 v[152:155], v160 offset:63488\n" \
  "ds_read_b128 v[156:159], v160 offset:64512\n" \
  "s_add_u32 m0, s69, 8192\n" \
  "s_nop 0\n" \
  "buffer_load_dwordx4 v165, s[36:39], s58 offen lds\n" \
  "s_add_u32 m0, s69, 12288\n" \
  "s_nop 0\n" \
  "buffer_load_dwordx4 v165, s[36:39], s59 offen lds\n" \
  "s_add_u32 m0, s69, 16384\n" \
  "s_nop 0\n" \
  "buffer_load_dwordx4 v165, s[36:39], s60 offen lds\n" \
  "buffer_load_dwordx4 v[4:7], v165, s[36:39], s75 offen offset:1024\n" \
  "buffer_load_dwordx4 v[0:3], v165, s[36:39], s75 offen offset:0\n" \
  "buffer_load_dword v218, off, s[44:47], 0 sc1\n" \
  "v_readlane_b32 s67, v175, s70\n" \
  "v_readlane_b32 s65, v178, s70\n" \
  "v_readlane_b32 s76, v169, s70\n" \
  "v_readlane_b32 s77, v172, s70\n" \
  "s_add_u32 s70, s70, 1\n" \
  "v_add_u32 v163, s76, v162\n" \
  "s_add_u32 m0, s69, 20480\n" \
  "s_nop 0\n" \
  "buffer_load_dwordx4 v165, s[36:39], s61 offen lds\n" \
  "buffer_load_dwordx4 v[12:15], v165, s[36:39], s75 offen offset:3072\n" \
  "v_add_u32 v164, 4096, v163\n" \
  "s_cmp_gt_u32 s67, s68\n" \
  "s_cbranch_scc1 Lp131_%=\n" \
  "Lq131_%=:\n" \
  "buffer_load_dwordx4 v[8:11], v165, s[36:39], s75 offen offset:2048\n" \
  "buffer_load_dwordx4 v[16:19], v166, s[36:39], s75 offen offset:0\n" \
  "v_mul_f32 v188, s78, v188\n" \
  "v_mul_f32 v189, s78, v189\n" \
  "v_mul_f32 v190, s78, v190\n" \
  "v_mul_f32 v191, s78, v191\n" \
  "s_bitcmp1_b32 s66, 28\n" \
  "s_cbranch_scc1 Lp132_%=\n" \
  "buffer_store_dwordx4 v[188:191], v167, s[40:43], s49 offen offset:0 nt\n" \
  "Lq132_%=:\n" \
  "s_add_u32 m0, s69, 24576\n" \
  "s_nop 0\n" \
  "buffer_load_dwordx4 v165, s[36:39], s62 offen lds\n" \
  "buffer_load_dwordx4 v[24:27], v166, s[36:39], s75 offen offset:2048\n" \
  "v_mul_f32 v204, s78, v204\n" \
  "v_mul_f32 v205, s78, v205\n" \
  "v_mul_f32 v206, s78, v206\n" \
  "v_mul_f32 v207, s78, v207\n" \
  "s_bitcmp1_b32 s66, 28\n" \
  "s_cbranch_scc1 Lp133_%=\n" \
  "buffer_store_dwordx4 v[204:207], v167, s[40:43], s49 offen offset:64 nt\n" \
  "Lq133_%=:\n" \
  "s_and_b32 s73, s65, 0xf00001ff\n" \
  "buffer_load_dwordx4 v[20:23], v166, s[36:39], s75 offen offset:1024\n" \
  "buffer_load_dwordx4 v[28:31], v166, s[36:39], s75 offen offset:3072\n" \
  "s_bfe_u32 m0, s65, 0x40018\n" \
  "s_nop 0\n" \
  "s_movreld_b32 s56, s77\n" \
  "s_branch Ls1_8_%=\n" \
  "Lsuf1_0_3_%=:\n" \
  "s_bitcmp1_b32 s66, 10\n" \
  "s_cbranch_scc1 Lsuf1_1_3_%=\n" \
  "ds_read_b128 v[144:147], v160 offset:61440\n" \
  "ds_read_b128 v[148:151], v160 offset:62464\n" \
  "ds_read_b128 v[152:155], v160 offset:63488\n" \
  "ds_read_b128 v[156:159], v160 offset:64512\n" \
  "s_add_u32 m0, s69, 12288\n" \
  "s_nop 0\n" \
  "buffer_load_dwordx4 v165, s[36:39], s59 offen lds\n" \
  "s_add_u32 m0, s69, 16384\n" \
  "s_nop 0\n" \
  "buffer_load_dwordx4 v165, s[36:39], s60 offen lds\n" \
  "buffer_load_dwordx4 v[4:7], v165, s[36:39], s75 offen offset:1024\n" \
  "buffer_load_dwordx4 v[0:3], v165, s[36:39], s75 offen offset:0\n" \
  "buffer_load_dword v218, off, s[44:47], 0 sc1\n" \
  "v_readlane_b32 s67, v175, s70\n" \
  "v_readlane_b32 s65, v178, s70\n" \
  "v_readlane_b32 s76, v169, s70\n" \
  "v_readlane_b32 s77, v172, s70\n" \
  "s_add_u32 s70, s70, 1\n" \
  "v_add_u32 v163, s76, v162\n" \
  "s_add_u32 m0, s69, 20480\n" \
  "s_nop 0\n" \
  "buffer_load_dwordx4 v165, s[36:39], s61 offen lds\n" \
  "buffer_load_dwordx4 v[12:15], v165, s[36:39], s75 offen offset:3072\n" \
  "v_add_u32 v164, 4096, v163\n" \
  "s_cmp_gt_u32 s67, s68\n" \
  "s_cbranch_scc1 Lp134_%=\n" \
  "Lq134_%=:\n" \
  "buffer_load_dwordx4 v[8:11], v165, s[36:39], s75 offen offset:2048\n" \
  "buffer_load_dwordx4 v[16:19], v166, s[36:39], s75 offen offset:0\n" \
  "v_mul_f32 v188, s78, v188\n" \
  "v_mul_f32 v189, s78, v189\n" \
  "v_mul_f32 v190, s78, v190\n" \
  "v_mul_f32 v191, s78, v191\n" \
  "s_bitcmp1_b32 s66, 28\n" \
  "s_cbranch_scc1 Lp135_%=\n" \
  "buffer_store_dwordx4 v[188:191], v167, s[40:43], s49 offen offset:0 nt\n" \
  "Lq135_%=:\n" \
  "s_add_u32 m0, s69, 24576\n" \
  "s_nop 0\n" \
  "buffer_load_dwordx4 v165, s[36:39], s62 offen lds\n" \
  "buffer_load_dwordx4 v[24:27], v166, s[36:39], s75 offen offset:2048\n" \
  "v_mul_f32 v204, s78, v204\n" \
  "v_mul_f32 v205, s78, v205\n" \
  "v_mul_f32 v206, s78, v206\n" \
  "v_mul_f32 v207, s78, v207\n" \
  "s_bitcmp1_b32 s66, 28\n" \
  "s_cbranch_scc1 Lp136_%=\n" \
  "buffer_store_dwordx4 v[204:207], v167, s[40:43], s49 offen offset:64 nt\n" \
  "Lq136_%=:\n" \
  "s_and_b32 s73, s65, 0xf00001ff\n" \
  "buffer_load_dwordx4 v[20:23], v166, s[36:39], s75 offen offset:1024\n" \
  "buffer_load_dwordx4 v[28:31], v166, s[36:39], s75 offen offset:3072\n" \
  "s_bfe_u32 m0, s65, 0x40018\n" \
  "s_nop 0\n" \
  "s_movreld_b32 s56, s77\n" \
  "s_branch Ls1_8_%=\n" \
  "Lsuf1_0_4_%=:\n" \
  "s_bitcmp1_b32 s66, 10\n" \
  "s_cbranch_scc1 Lsuf1_1_4_%=\n" \
  "ds_read_b128 v[144:147], v160 offset:61440\n" \
  "ds_read_b128 v[148:151], v160 offset:62464\n" \
  "ds_read_b128 v[152:155], v160 offset:63488\n" \
  "ds_read_b128 v[156:159], v160 offset:64512\n" \
  "s_add_u32 m0, s69, 16384\n" \
  "s_nop 0\n" \
  "buffer_load_dwordx4 v165, s[36:39], s60 offen lds\n" \
  "buffer_load_dwordx4 v[4:7], v165, s[36:39], s75 offen offset:1024\n" \
  "buffer_load_dwordx4 v[0:3], v165, s[36:39], s75 offen offset:0\n" \
  "buffer_load_dword v218, off, s[44:47], 0 sc1\n" \
  "v_readlane_b32 s67, v175, s70\n" \
  "v_readlane_b32 s65, v178, s70\n" \
  "v_readlane_b32 s76, v169, s70\n" \
  "v_readlane_b32 s77, v172, s70\n" \
  "s_add_u32 s70, s70, 1\n" \
  "v_add_u32 v163, s76, v162\n" \
  "s_add_u32 m0, s69, 20480\n" \
  "s_nop 0\n" \
  "buffer_load_dwordx4 v165, s[36:39], s61 offen lds\n" \
  "buffer_load_dwordx4 v[12:15], v165, s[36:39], s75 offen offset:3072\n" \
  "v_add_u32 v164, 4096, v163\n" \
  "s_cmp_gt_u32 s67, s68\n" \
  "s_cbranch_scc1 Lp137_%=\n" \
  "Lq137_%=:\n" \
  "buffer_load_dwordx4 v[8:11], v165, s[36:39], s75 offen offset:2048\n" \
  "buffer_load_dwordx4 v[16:19], v166, s[36:39], s75 offen offset:0\n" \
  "v_mul_f32 v188, s78, v188\n" \
  "v_mul_f32 v189, s78, v189\n" \
  "v_mul_f32 v190, s78, v190\n" \
  "v_mul_f32 v191, s78, v191\n" \
  "s_bitcmp1_b32 s66, 28\n" \
  "s_cbranch_scc1 Lp138_%=\n" \
  "buffer_store_dwordx4 v[188:191], v167, s[40:43], s49 offen offset:0 nt\n" \
  "Lq138_%=:\n" \
  "s_add_u32 m0, s69, 24576\n" \
  "s_nop 0\n" \
  "buffer_load_dwordx4 v165, s[36:39], s62 offen lds\n" \
  "buffer_load_dwordx4 v[24:27], v166, s[36:39], s75 offen offset:2048\n" \
  "v_mul_f32 v204, s78, v204\n" \
  "v_mul_f32 v205, s78, v205\n" \
  "v_mul_f32 v206, s78, v206\n" \
  "v_mul_f32 v207, s78, v207\n" \
  "s_bitcmp1_b32 s66, 28\n" \
  "s_cbranch_scc1 Lp139_%=\n" \
  "buffer_store_dwordx4 v[204:207], v167, s[40:43], s49 offen offset:64 nt\n" \
  "Lq139_%=:\n" \
  "s_and_b32 s73, s65, 0xf00001ff\n" \
  "buffer_load_dwordx4 v[20:23], v166, s[36:39], s75 offen offset:1024\n" \
  "buffer_load_dwordx4 v[28:31], v166, s[36:39], s75 offen offset:3072\n" \
  "s_bfe_u32 m0, s65, 0x40018\n" \
  "s_nop 0\n" \
  "s_movreld_b32 s56, s77\n" \
  "s_branch Ls1_8_%=\n" \
  "Lsuf1_0_5_%=:\n" \
  "s_bitcmp1_b32 s66, 10\n" \
  "s_cbranch_scc1 Lsuf1_1_5_%=\n" \
  "ds_read_b128 v[144:147], v160 offset:61440\n" \
  "ds_read_b128 v[148:151], v160 offset:62464\n" \
  "ds_read_b128 v[152:155], v160 offset:63488\n" \
  "ds_read_b128 v[156:159], v160 offset:64512\n" \
  "s_add_u32 m0, s69, 20480\n" \
  "s_nop 0\n" \
  "buffer_load_dwordx4 v165, s[36:39], s61 offen lds\n" \
  "buffer_load_dwordx4 v[12:15], v165, s[36:39], s75 offen offset:3072\n" \
  "v_add_u32 v164, 4096, v163\n" \
  "s_cmp_gt_u32 s67, s68\n" \
  "s_cbranch_scc1 Lp140_%=\n" \
  "Lq140_%=:\n" \
  "buffer_load_dwordx4 v[8:11], v165, s[36:39], s75 offen offset:2048\n" \
  "buffer_load_dwordx4 v[16:19], v166, s[36:39], s75 offen offset:0\n" \
  "v_mul_f32 v188, s78, v188\n" \
  "v_mul_f32 v189, s78, v189\n" \
  "v_mul_f32 v190, s78, v190\n" \
  "v_mul_f32 v191, s78, v191\n" \
  "s_bitcmp1_b32 s66, 28\n" \
  "s_cbranch_scc1 Lp141_%=\n" \
  "buffer_store_dwordx4 v[188:191], v167, s[40:43], s49 offen offset:0 nt\n" \
  "Lq141_%=:\n" \
  "s_add_u32 m0, s69, 24576\n" \
  "s_nop 0\n" \
  "buffer_load_dwordx4 v165, s[36:39], s62 offen lds\n" \
  "buffer_load_dwordx4 v[24:27], v166, s[36:39], s75 offen offset:2048\n" \
  "v_mul_f32 v204, s78, v204\n" \
  "v_mul_f32 v205, s78, v205\n" \
  "v_mul_f32 v206, s78, v206\n" \
  "v_mul_f32 v207, s78, v207\n" \
  "s_bitcmp1_b32 s66, 28\n" \
  "s_cbranch_scc1 Lp142_%=\n" \
  "buffer_store_dwordx4 v[204:207], v167, s[40:43], s49 offen offset:64 nt\n" \
  "Lq142_%=:\n" \
  "s_and_b32 s73, s65, 0xf00001ff\n" \
  "buffer_load_dwordx4 v[20:23], v166, s[36:39], s75 offen offset:1024\n" \
  "buffer_load_dwordx4 v[28:31], v166, s[36:39], s75 offen offset:3072\n" \
  "s_bfe_u32 m0, s65, 0x40018\n" \
  "s_nop 0\n" \
  "s_movreld_b32 s56, s77\n" \
  "s_branch Ls1_8_%=\n" \
  "Lsuf1_0_6_%=:\n" \
  "s_bitcmp1_b32 s66, 10\n" \
  "s_cbranch_scc1 Lsuf1_1_6_%=\n" \
  "ds_read_b128 v[144:147], v160 offset:61440\n" \
  "ds_read_b128 v[148:151], v160 offset:62464\n" \
  "ds_read_b128 v[152:155], v160 offset:63488\n" \
  "ds_read_b128 v[156:159], v160 offset:64512\n" \
  "s_add_u32 m0, s69, 24576\n" \
  "s_nop 0\n" \
  "buffer_load_dwordx4 v165, s[36:39], s62 offen lds\n" \
  "buffer_load_dwordx4 v[24:27], v166, s[36:39], s75 offen offset:2048\n" \
  "v_mul_f32 v204, s78, v204\n" \
  "v_mul_f32 v205, s78, v205\n" \
  "v_mul_f32 v206, s78, v206\n" \
  "v_mul_f32 v207, s78, v207\n" \
  "s_bitcmp1_b32 s66, 28\n" \
  "s_cbranch_scc1 Lp143_%=\n" \
  "buffer_store_dwordx4 v[204:207], v167, s[40:43], s49 offen offset:64 nt\n" \
  "Lq143_%=:\n" \
  "s_and_b32 s73, s65, 0xf00001ff\n" \
  "buffer_load_dwordx4 v[20:23], v166, s[36:39], s75 offen offset:1024\n" \
  "buffer_load_dwordx4 v[28:31], v166, s[36:39], s75 offen offset:3072\n" \
  "s_bfe_u32 m0, s65, 0x40018\n" \
  "s_nop 0\n" \
  "s_movreld_b32 s56, s77\n" \
  "s_branch Ls1_8_%=\n" \
  "Lsuf1_1_1_%=:\n" \
  "buffer_load_dwordx4 v[4:7], v165, s[36:39], s75 offen offset:1024\n" \
  "buffer_load_dwordx4 v[0:3], v165, s[36:39], s75 offen offset:0\n" \
  "buffer_load_dword v218, off, s[44:47], 0 sc1\n" \
  "v_readlane_b32 s67, v175, s70\n" \
  "v_readlane_b32 s65, v178, s70\n" \
  "v_readlane_b32 s76, v169, s70\n" \
  "v_readlane_b32 s77, v172, s70\n" \
  "s_add_u32 s70, s70, 1\n" \
  "v_add_u32 v163, s76, v162\n" \
  "buffer_load_dwordx4 v[12:15], v165, s[36:39], s75 offen offset:3072\n" \
  "v_add_u32 v164, 4096, v163\n" \
  "s_cmp_gt_u32 s67, s68\n" \
  "s_cbranch_scc1 Lp144_%=\n" \
  "Lq144_%=:\n" \
  "buffer_load_dwordx4 v[8:11], v165, s[36:39], s75 offen offset:2048\n" \
  "buffer_load_dwordx4 v[16:19], v166, s[36:39], s75 offen offset:0\n" \
  "v_mul_f32 v188, s78, v188\n" \
  "v_mul_f32 v189, s78, v189\n" \
  "v_mul_f32 v190, s78, v190\n" \
  "v_mul_f32 v191, s78, v191\n" \
  "s_bitcmp1_b32 s66, 28\n" \
  "s_cbranch_scc1 Lp145_%=\n" \
  "buffer_store_dwordx4 v[188:191], v167, s[40:43], s49 offen offset:0 nt\n" \
  "Lq145_%=:\n" \
  "buffer_load_dwordx4 v[24:27], v166, s[36:39], s75 offen offset:2048\n" \
  "v_mul_f32 v204, s78, v204\n" \
  "v_mul_f32 v205, s78, v205\n" \
  "v_mul_f32 v206, s78, v206\n" \
  "v_mul_f32 v207, s78, v207\n" \
  "s_bitcmp1_b32 s66, 28\n" \
  "s_cbranch_scc1 Lp146_%=\n" \
  "buffer_store_dwordx4 v[204:207], v167, s[40:43], s49 offen offset:64 nt\n" \
  "Lq146_%=:\n" \
  "s_and_b32 s73, s65, 0xf00001ff\n" \
  "buffer_load_dwordx4 v[20:23], v166, s[36:39], s75 offen offset:1024\n" \
  "buffer_load_dwordx4 v[28:31], v166, s[36:39], s75 offen offset:3072\n" \
  "s_bfe_u32 m0, s65, 0x40018\n" \
  "s_nop 0\n" \
  "s_movreld_b32 s56, s77\n" \
  "s_branch Ls1_8_%=\n" \
  "Lsuf1_1_2_%=:\n" \
  "buffer_load_dwordx4 v[4:7], v165, s[36:39], s75 offen offset:1024\n" \
  "buffer_load_dwordx4 v[0:3], v165, s[36:39], s75 offen offset:0\n" \
  "buffer_load_dword v218, off, s[44:47], 0 sc1\n" \
  "v_readlane_b32 s67, v175, s70\n" \
  "v_readlane_b32 s65, v178, s70\n" \
  "v_readlane_b32 s76, v169, s70\n" \
  "v_readlane_b32 s77, v172, s70\n" \
  "s_add_u32 s70, s70, 1\n" \
  "v_add_u32 v163, s76, v162\n" \
  "buffer_load_dwordx4 v[12:15], v165, s[36:39], s75 offen offset:3072\n" \
  "v_add_u32 v164, 4096, v163\n" \
  "s_cmp_gt_u32 s67, s68\n" \
  "s_cbranch_scc1 Lp147_%=\n" \
  "Lq147_%=:\n" \
  "buffer_load_dwordx4 v[8:11], v165, s[36:39], s75 offen offset:2048\n" \
  "buffer_load_dwordx4 v[16:19], v166, s[36:39], s75 offen offset:0\n" \
  "v_mul_f32 v188, s78, v188\n" \
  "v_mul_f32 v189, s78, v189\n" \
  "v_mul_f32 v190, s78, v190\n" \
  "v_mul_f32 v191, s78, v191\n" \
  "s_bitcmp1_b32 s66, 28\n" \
  "s_cbranch_scc1 Lp148_%=\n" \
  "buffer_store_dwordx4 v[188:191], v167, s[40:43], s49 offen offset:0 nt\n" \
  "Lq148_%=:\n" \
  "buffer_load_dwordx4 v[24:27], v166, s[36:39], s75 offen offset:2048\n" \
  "v_mul_f32 v204, s78, v204\n" \
  "v_mul_f32 v205, s78, v205\n" \
  "v_mul_f32 v206, s78, v206\n" \
  "v_mul_f32 v207, s78, v207\n" \
  "s_bitcmp1_b32 s66, 28\n" \
  "s_cbranch_scc1 Lp149_%=\n" \
  "buffer_store_dwordx4 v[204:207], v167, s[40:43], s49 offen offset:64 nt\n" \
  "Lq149_%=:\n" \
  "s_and_b32 s73, s65, 0xf00001ff\n" \
  "buffer_load_dwordx4 v[20:23], v166, s[36:39], s75 offen offset:1024\n" \
  "buffer_load_dwordx4 v[28:31], v166, s[36:39], s75 offen offset:3072\n" \
  "s_bfe_u32 m0, s65, 0x40018\n" \
  "s_nop 0\n" \
  "s_movreld_b32 s56, s77\n" \
  "s_branch Ls1_8_%=\n" \
  "Lsuf1_1_3_%=:\n" \
  "buffer_load_dwordx4 v[4:7], v165, s[36:39], s75 offen offset:1024\n" \
  "buffer_load_dwordx4 v[0:3], v165, s[36:39], s75 offen offset:0\n" \
  "buffer_load_dword v218, off, s[44:47], 0 sc1\n" \
  "v_readlane_b32 s67, v175, s70\n" \
  "v_readlane_b32 s65, v178, s70\n" \
  "v_readlane_b32 s76, v169, s70\n" \
  "v_readlane_b32 s77, v172, s70\n" \
  "s_add_u32 s70, s70, 1\n" \
  "v_add_u32 v163, s76, v162\n" \
  "buffer_load_dwordx4 v[12:15], v165, s[36:39], s75 offen offset:3072\n" \
  "v_add_u32 v164, 4096, v163\n" \
  "s_cmp_gt_u32 s67, s68\n" \
  "s_cbranch_scc1 Lp150_%=\n" \
  "Lq150_%=:\n" \
  "buffer_load_dwordx4 v[8:11], v165, s[36:39], s75 offen offset:2048\n" \
  "buffer_load_dwordx4 v[16:19], v166, s[36:39], s75 offen offset:0\n" \
  "v_mul_f32 v188, s78, v188\n" \
  "v_mul_f32 v189, s78, v189\n" \
  "v_mul_f32 v190, s78, v190\n" \
  "v_mul_f32 v191, s78, v191\n" \
  "s_bitcmp1_b32 s66, 28\n" \
  "s_cbranch_scc1 Lp151_%=\n" \
  "buffer_store_dwordx4 v[188:191], v167, s[40:43], s49 offen offset:0 nt\n" \
  "Lq151_%=:\n" \
  "buffer_load_dwordx4 v[24:27], v166, s[36:39], s75 offen offset:2048\n" \
  "v_mul_f32 v204, s78, v204\n" \
  "v_mul_f32 v205, s78, v205\n" \
  "v_mul_f32 v206, s78, v206\n" \
  "v_mul_f32 v207, s78, v207\n" \
  "s_bitcmp1_b32 s66, 28\n" \
  "s_cbranch_scc1 Lp152_%=\n" \
  "buffer_store_dwordx4 v[204:207], v167, s[40:43], s49 offen offset:64 nt\n" \
  "Lq152_%=:\n" \
  "s_and_b32 s73, s65, 0xf00001ff\n" \
  "buffer_load_dwordx4 v[20:23], v166, s[36:39], s75 offen offset:1024\n" \
  "buffer_load_dwordx4 v[28:31], v166, s[36:39], s75 offen offset:3072\n" \
  "s_bfe_u32 m0, s65, 0x40018\n" \
  "s_nop 0\n" \
  "s_movreld_b32 s56, s77\n" \
  "s_branch Ls1_8_%=\n" \
  "Lsuf1_1_4_%=:\n" \
  "buffer_load_dwordx4 v[4:7], v165, s[36:39], s75 offen offset:1024\n" \
  "buffer_load_dwordx4 v[0:3], v165, s[36:39], s75 offen offset:0\n" \
  "buffer_load_dword v218, off, s[44:47], 0 sc1\n" \
  "v_readlane_b32 s67, v175, s70\n" \
  "v_readlane_b32 s65, v178, s70\n" \
  "v_readlane_b32 s76, v169, s70\n" \
  "v_readlane_b32 s77, v172, s70\n" \
  "s_add_u32 s70, s70, 1\n" \
  "v_add_u32 v163, s76, v162\n" \
  "buffer_load_dwordx4 v[12:15], v165, s[36:39], s75 offen offset:3072\n" \
  "v_add_u32 v164, 4096, v163\n" \
  "s_cmp_gt_u32 s67, s68\n" \
  "s_cbranch_scc1 Lp153_%=\n" \
  "Lq153_%=:\n" \
  "buffer_load_dwordx4 v[8:11], v165, s[36:39], s75 offen offset:2048\n" \
  "buffer_load_dwordx4 v[16:19], v166, s[36:39], s75 offen offset:0\n" \
  "v_mul_f32 v188, s78, v188\n" \
  "v_mul_f32 v189, s78, v189\n" \
  "v_mul_f32 v190, s78, v190\n" \
  "v_mul_f32 v191, s78, v191\n" \
  "s_bitcmp1_b32 s66, 28\n" \
  "s_cbranch_scc1 Lp154_%=\n" \
  "buffer_store_dwordx4 v[188:191], v167, s[40:43], s49 offen offset:0 nt\n" \
  "Lq154_%=:\n" \
  "buffer_load_dwordx4 v[24:27], v166, s[36:39], s75 offen offset:2048\n" \
  "v_mul_f32 v204, s78, v204\n" \
  "v_mul_f32 v205, s78, v205\n" \
  "v_mul_f32 v206, s78, v206\n" \
  "v_mul_f32 v207, s78, v207\n" \
  "s_bitcmp1_b32 s66, 28\n" \
  "s_cbranch_scc1 Lp155_%=\n" \
  "buffer_store_dwordx4 v[204:207], v167, s[40:43], s49 offen offset:64 nt\n" \
  "Lq155_%=:\n" \
  "s_and_b32 s73, s65, 0xf00001ff\n" \
  "buffer_load_dwordx4 v[20:23], v166, s[36:39], s75 offen offset:1024\n" \
  "buffer_load_dwordx4 v[28:31], v166, s[36:39], s75 offen offset:3072\n" \
  "s_bfe_u32 m0, s65, 0x40018\n" \
  "s_nop 0\n" \
  "s_movreld_b32 s56, s77\n" \
  "s_branch Ls1_8_%=\n" \
  "Lsuf1_1_5_%=:\n" \
  "buffer_load_dwordx4 v[12:15], v165, s[36:39], s75 offen offset:3072\n" \
  "v_add_u32 v164, 4096, v163\n" \
  "s_cmp_gt_u32 s67, s68\n" \
  "s_cbranch_scc1 Lp156_%=\n" \
  "Lq156_%=:\n" \
  "buffer_load_dwordx4 v[8:11], v165, s[36:39], s75 offen offset:2048\n" \
  "buffer_load_dwordx4 v[16:19], v166, s[36:39], s75 offen offset:0\n" \
  "v_mul_f32 v188, s78, v188\n" \
  "v_mul_f32 v189, s78, v189\n" \
  "v_mul_f32 v190, s78, v190\n" \
  "v_mul_f32 v191, s78, v191\n" \
  "s_bitcmp1_b32 s66, 28\n" \
  "s_cbranch_scc1 Lp157_%=\n" \
  "buffer_store_dwordx4 v[188:191], v167, s[40:43], s49 offen offset:0 nt\n" \
  "Lq157_%=:\n" \
  "buffer_load_dwordx4 v[24:27], v166, s[36:39], s75 offen offset:2048\n" \
  "v_mul_f32 v204, s78, v204\n" \
  "v_mul_f32 v205, s78, v205\n" \
  "v_mul_f32 v206, s78, v206\n" \
  "v_mul_f32 v207, s78, v207\n" \
  "s_bitcmp1_b32 s66, 28\n" \
  "s_cbranch_scc1 Lp158_%=\n" \
  "buffer_store_dwordx4 v[204:207], v167, s[40:43], s49 offen offset:64 nt\n" \
  "Lq158_%=:\n" \
  "s_and_b32 s73, s65, 0xf00001ff\n" \
  "buffer_load_dwordx4 v[20:23], v166, s[36:39], s75 offen offset:1024\n" \
  "buffer_load_dwordx4 v[28:31], v166, s[36:39], s75 offen offset:3072\n" \
  "s_bfe_u32 m0, s65, 0x40018\n" \
  "s_nop 0\n" \
  "s_movreld_b32 s56, s77\n" \
  "s_branch Ls1_8_%=\n" \
  "Lsuf1_1_6_%=:\n" \
  "buffer_load_dwordx4 v[24:27], v166, s[36:39], s75 offen offset:2048\n" \
  "v_mul_f32 v204, s78, v204\n" \
  "v_mul_f32 v205, s78, v205\n" \
  "v_mul_f32 v206, s78, v206\n" \
  "v_mul_f32 v207, s78, v207\n" \
  "s_bitcmp1_b32 s66, 28\n" \
  "s_cbranch_scc1 Lp159_%=\n" \
  "buffer_store_dwordx4 v[204:207], v167, s[40:43], s49 offen offset:64 nt\n" \
  "Lq159_%=:\n" \
  "s_and_b32 s73, s65, 0xf00001ff\n" \
  "buffer_load_dwordx4 v[20:23], v166, s[36:39], s75 offen offset:1024\n" \
  "buffer_load_dwordx4 v[28:31], v166, s[36:39], s75 offen offset:3072\n" \
  "s_bfe_u32 m0, s65, 0x40018\n" \
  "s_nop 0\n" \
  "s_movreld_b32 s56, s77\n" \
  "s_branch Ls1_8_%=\n" \
  ".p2align 6\n" \
  "Lpre1_0_0_%=:\n" \
  "s_branch Ls1_1_%=\n" \
  ".org Lpre1_0_0_%=+704\n" \
  "Lpre1_0_1_%=:\n" \
  "s_branch Ls1_1_%=\n" \
  ".org Lpre1_0_0_%=+1408\n" \
  "Lpre1_0_2_%=:\n" \
  "ds_read_b128 v[144:147], v160 offset:36864\n" \
  "ds_read_b128 v[148:151], v160 offset:37888\n" \
  "ds_read_b128 v[152:155], v160 offset:38912\n" \
  "ds_read_b128 v[156:159], v160 offset:39936\n" \
  "s_add_u32 m0, s69, 0\n" \
  "s_nop 0\n" \
  "buffer_load_dwordx4 v165, s[36:39], s56 offen lds\n" \
  "s_branch Ls1_2_%=\n" \
  ".org Lpre1_0_0_%=+2112\n" \
  "Lpre1_0_3_%=:\n" \
  "ds_read_b128 v[128:131], v160 offset:40960\n" \
  "ds_read_b128 v[132:135], v160 offset:41984\n" \
  "ds_read_b128 v[136:139], v160 offset:43008\n" \
  "ds_read_b128 v[140:143], v160 offset:44032\n" \
  "s_add_u32 m0, s69, 0\n" \
  "s_nop 0\n" \
  "buffer_load_dwordx4 v165, s[36:39], s56 offen lds\n" \
  "s_add_u32 m0, s69, 4096\n" \
  "s_nop 0\n" \
  "buffer_load_dwordx4 v165, s[36:39], s57 offen lds\n" \
  "s_branch Ls1_3_%=\n" \
  ".org Lpre1_0_0_%=+2816\n" \
  "Lpre1_0_4_%=:\n" \
  "ds_read_b128 v[144:147], v160 offset:45056\n" \
  "ds_read_b128 v[148:151], v160 offset:46080\n" \
  "ds_read_b128 v[152:155], v160 offset:47104\n" \
  "ds_read_b128 v[156:159], v160 offset:48128\n" \
  "s_add_u32 m0, s69, 0\n" \
  "s_nop 0\n" \
  "buffer_load_dwordx4 v165, s[36:39], s56 offen lds\n" \
  "s_add_u32 m0, s69, 4096\n" \
  "s_nop 0\n" \
  "buffer_load_dwordx4 v165, s[36:39], s57 offen lds\n" \
  "s_add_u32 m0, s69, 8192\n" \
  "s_nop 0\n" \
  "buffer_load_dwordx4 v165, s[36:39], s58 offen lds\n" \
  "s_branch Ls1_4_%=\n" \
  ".org Lpre1_0_0_%=+3520\n" \
  "Lpre1_0_5_%=:\n" \
  "ds_read_b128 v[128:131], v160 offset:49152\n" \
  "ds_read_b128 v[132:135], v160 offset:50176\n" \
  "ds_read_b128 v[136:139], v160 offset:51200\n" \
  "ds_read_b128 v[140:143], v160 offset:52224\n" \
  "s_add_u32 m0, s69, 0\n" \
  "s_nop 0\n" \
  "buffer_load_dwordx4 v165, s[36:39], s56 offen lds\n" \
  "s_add_u32 m0, s69, 4096\n" \
  "s_nop 0\n" \
  "buffer_load_dwordx4 v165, s[36:39], s57 offen lds\n" \
  "s_add_u32 m0, s69, 8192\n" \
  "s_nop 0\n" \
  "buffer_load_dwordx4 v165, s[36:39], s58 offen lds\n" \
  "s_add_u32 m0, s69, 12288\n" \
  "s_nop 0\n" \
  "buffer_load_dwordx4 v165, s[36:39], s59 offen lds\n" \
  "s_branch Ls1_5_%=\n" \
  ".org Lpre1_0_0_%=+4224\n" \
  "Lpre1_0_6_%=:\n" \
  "ds_read_b128 v[144:147], v160 offset:53248\n" \
  "ds_read_b128 v[148:151], v160 offset:54272\n" \
  "ds_read_b128 v[152:155], v160 offset:55296\n" \
  "ds_read_b128 v[156:159], v160 offset:56320\n" \
  "s_add_u32 m0, s69, 0\n" \
  "s_nop 0\n" \
  "buffer_load_dwordx4 v165, s[36:39], s56 offen lds\n" \
  "s_add_u32 m0, s69, 4096\n" \
  "s_nop 0\n" \
  "buffer_load_dwordx4 v165, s[36:39], s57 offen lds\n" \
  "s_add_u32 m0, s69, 8192\n" \
  "s_nop 0\n" \
  "buffer_load_dwordx4 v165, s[36:39], s58 offen lds\n" \
  "s_add_u32 m0, s69, 12288\n" \
  "s_nop 0\n" \
  "buffer_load_dwordx4 v165, s[36:39], s59 offen lds\n" \
  "s_add_u32 m0, s69, 16384\n" \
  "s_nop 0\n" \
  "buffer_load_dwordx4 v165, s[36:39], s60 offen lds\n" \
  "buffer_load_dwordx4 v[4:7], v165, s[36:39], s75 offen offset:1024\n" \
  "buffer_load_dwordx4 v[0:3], v165, s[36:39], s75 offen offset:0\n" \
  "buffer_load_dword v218, off, s[44:47], 0 sc1\n" \
  "v_readlane_b32 s67, v175, s70\n" \
  "v_readlane_b32 s65, v178, s70\n" \
  "v_readlane_b32 s76, v169, s70\n" \
  "v_readlane_b32 s77, v172, s70\n" \
  "s_add_u32 s70, s70, 1\n" \
  "v_add_u32 v163, s76, v162\n" \
  "s_branch Ls1_6_%=\n" \
  ".org Lpre1_0_0_%=+4928\n" \
  "Lpre1_0_7_%=:\n" \
  "ds_read_b128 v[128:131], v160 offset:57344\n" \
  "ds_read_b128 v[132:135], v160 offset:58368\n" \
  "ds_read_b128 v[136:139], v160 offset:59392\n" \
  "ds_read_b128 v[140:143], v160 offset:60416\n" \
  "s_add_u32 m0, s69, 0\n" \
  "s_nop 0\n" \
  "buffer_load_dwordx4 v165, s[36:39], s56 offen lds\n" \
  "s_add_u32 m0, s69, 4096\n" \
  "s_nop 0\n" \
  "buffer_load_dwordx4 v165, s[36:39], s57 offen lds\n" \
  "s_add_u32 m0, s69, 8192\n" \
  "s_nop 0\n" \
  "buffer_load_dwordx4 v165, s[36:39], s58 offen lds\n" \
  "s_add_u32 m0, s69, 12288\n" \
  "s_nop 0\n" \
  "buffer_load_dwordx4 v165, s[36:39], s59 offen lds\n" \
  "s_add_u32 m0, s69, 16384\n" \
  "s_nop 0\n" \
  "buffer_load_dwordx4 v165, s[36:39], s60 offen lds\n" \
  "buffer_load_dwordx4 v[4:7], v165, s[36:39], s75 offen offset:1024\n" \
  "buffer_load_dwordx4 v[0:3], v165, s[36:39], s75 offen offset:0\n" \
  "buffer_load_dword v218, off, s[44:47], 0 sc1\n" \
  "v_readlane_b32 s67, v175, s70\n" \
  "v_readlane_b32 s65, v178, s70\n" \
  "v_readlane_b32 s76, v169, s70\n" \
  "v_readlane_b32 s77, v172, s70\n" \
  "s_add_u32 s70, s70, 1\n" \
  "v_add_u32 v163, s76, v162\n" \
  "s_add_u32 m0, s69, 20480\n" \
  "s_nop 0\n" \
  "buffer_load_dwordx4 v165, s[36:39], s61 offen lds\n" \
  "buffer_load_dwordx4 v[12:15], v165, s[36:39], s75 offen offset:3072\n" \
  "v_add_u32 v164, 4096, v163\n" \
  "s_cmp_gt_u32 s67, s68\n" \
  "s_cbranch_scc1 Lp160_%=\n" \
  "Lq160_%=:\n" \
  "buffer_load_dwordx4 v[8:11], v165, s[36:39], s75 offen offset:2048\n" \
  "buffer_load_dwordx4 v[16:19], v166, s[36:39], s75 offen offset:0\n" \
  "v_mul_f32 v188, s78, v188\n" \
  "v_mul_f32 v189, s78, v189\n" \
  "v_mul_f32 v190, s78, v190\n" \
  "v_mul_f32 v191, s78, v191\n" \
  "s_bitcmp1_b32 s66, 28\n" \
  "s_cbranch_scc1 Lp161_%=\n" \
  "buffer_store_dwordx4 v[188:191], v167, s[40:43], s49 offen offset:0 nt\n" \
  "Lq161_%=:\n" \
  "s_branch Ls1_7_%=\n" \
  ".org Lpre1_0_0_%=+5632\n" \
  "Lpre1_0_8_%=:\n" \
  "ds_read_b128 v[144:147], v160 offset:61440\n" \
  "ds_read_b128 v[148:151], v160 offset:62464\n" \
  "ds_read_b128 v[152:155], v160 offset:63488\n" \
  "ds_read_b128 v[156:159], v160 offset:64512\n" \
  "s_add_u32 m0, s69, 0\n" \
  "s_nop 0\n" \
  "buffer_load_dwordx4 v165, s[36:39], s56 offen lds\n" \
  "s_add_u32 m0, s69, 4096\n" \
  "s_nop 0\n" \
  "buffer_load_dwordx4 v165, s[36:39], s57 offen lds\n" \
  "s_add_u32 m0, s69, 8192\n" \
  "s_nop 0\n" \
  "buffer_load_dwordx4 v165, s[36:39], s58 offen lds\n" \
  "s_add_u32 m0, s69, 12288\n" \
  "s_nop 0\n" \
  "buffer_load_dwordx4 v165, s[36:39], s59 offen lds\n" \
  "s_add_u32 m0, s69, 16384\n" \
  "s_nop 0\n" \
  "buffer_load_dwordx4 v165, s[36:39], s60 offen lds\n" \
  "buffer_load_dwordx4 v[4:7], v165, s[36:39], s75 offen offset:1024\n" \
  "buffer_load_dwordx4 v[0:3], v165, s[36:39], s75 offen offset:0\n" \
  "buffer_load_dword v218, off, s[44:47], 0 sc1\n" \
  "v_readlane_b32 s67, v175, s70\n" \
  "v_readlane_b32 s65, v178, s70\n" \
  "v_readlane_b32 s76, v169, s70\n" \
  "v_readlane_b32 s77, v172, s70\n" \
  "s_add_u32 s70, s70, 1\n" \
  "v_add_u32 v163, s76, v162\n" \
  "s_add_u32 m0, s69, 20480\n" \
  "s_nop 0\n" \
  "buffer_load_dwordx4 v165, s[36:39], s61 offen lds\n" \
  "buffer_load_dwordx4 v[12:15], v165, s[36:39], s75 offen offset:3072\n" \
  "v_add_u32 v164, 4096, v163\n" \
  "s_cmp_gt_u32 s67, s68\n" \
  "s_cbranch_scc1 Lp162_%=\n" \
  "Lq162_%=:\n" \
  "buffer_load_dwordx4 v[8:11], v165, s[36:39], s75 offen offset:2048\n" \
  "buffer_load_dwordx4 v[16:19], v166, s[36:39], s75 offen offset:0\n" \
  "v_mul_f32 v188, s78, v188\n" \
  "v_mul_f32 v189, s78, v189\n" \
  "v_mul_f32 v190, s78, v190\n" \
  "v_mul_f32 v191, s78, v191\n" \
  "s_bitcmp1_b32 s66, 28\n" \
  "s_cbranch_scc1 Lp163_%=\n" \
  "buffer_store_dwordx4 v[188:191], v167, s[40:43], s49 offen offset:0 nt\n" \
  "Lq163_%=:\n" \
  "s_add_u32 m0, s69, 24576\n" \
  "s_nop 0\n" \
  "buffer_load_dwordx4 v165, s[36:39], s62 offen lds\n" \
  "buffer_load_dwordx4 v[24:27], v166, s[36:39], s75 offen offset:2048\n" \
  "v_mul_f32 v204, s78, v204\n" \
  "v_mul_f32 v205, s78, v205\n" \
  "v_mul_f32 v206, s78, v206\n" \
  "v_mul_f32 v207, s78, v207\n" \
  "s_bitcmp1_b32 s66, 28\n" \
  "s_cbranch_scc1 Lp164_%=\n" \
  "buffer_store_dwordx4 v[204:207], v167, s[40:43], s49 offen offset:64 nt\n" \
  "Lq164_%=:\n" \
  "s_and_b32 s73, s65, 0xf00001ff\n" \
  "buffer_load_dwordx4 v[20:23], v166, s[36:39], s75 offen offset:1024\n" \
  "buffer_load_dwordx4 v[28:31], v166, s[36:39], s75 offen offset:3072\n" \
  "s_bfe_u32 m0, s65, 0x40018\n" \
  "s_nop 0\n" \
  "s_movreld_b32 s56, s77\n" \
  "s_branch Ls1_8_%=\n" \
  ".org Lpre1_0_0_%=+6336\n" \
  "Lpre1_1_0_%=:\n" \
  "s_branch Ls1_1_%=\n" \
  ".org Lpre1_0_0_%=+7040\n" \
  "Lpre1_1_1_%=:\n" \
  "s_branch Ls1_1_%=\n" \
  ".org Lpre1_0_0_%=+7744\n" \
  "Lpre1_1_2_%=:\n" \
  "ds_read_b128 v[144:147], v160 offset:36864\n" \
  "ds_read_b128 v[148:151], v160 offset:37888\n" \
  "ds_read_b128 v[152:155], v160 offset:38912\n" \
  "ds_read_b128 v[156:159], v160 offset:39936\n" \
  "s_branch Ls1_2_%=\n" \
  ".org Lpre1_0_0_%=+8448\n" \
  "Lpre1_1_3_%=:\n" \
  "ds_read_b128 v[128:131], v160 offset:40960\n" \
  "ds_read_b128 v[132:135], v160 offset:41984\n" \
  "ds_read_b128 v[136:139], v160 offset:43008\n" \
  "ds_read_b128 v[140:143], v160 offset:44032\n" \
  "s_branch Ls1_3_%=\n" \
  ".org Lpre1_0_0_%=+9152\n" \
  "Lpre1_1_4_%=:\n" \
  "ds_read_b128 v[144:147], v160 offset:45056\n" \
  "ds_read_b128 v[148:151], v160 offset:46080\n" \
  "ds_read_b128 v[152:155], v160 offset:47104\n" \
  "ds_read_b128 v[156:159], v160 offset:48128\n" \
  "s_branch Ls1_4_%=\n" \
  ".org Lpre1_0_0_%=+9856\n" \
  "Lpre1_1_5_%=:\n" \
  "ds_read_b128 v[128:131], v160 offset:49152\n" \
  "ds_read_b128 v[132:135], v160 offset:50176\n" \
  "ds_read_b128 v[136:139], v160 offset:51200\n" \
  "ds_read_b128 v[140:143], v160 offset:52224\n" \
  "s_branch Ls1_5_%=\n" \
  ".org Lpre1_0_0_%=+10560\n" \
  "Lpre1_1_6_%=:\n" \
  "ds_read_b128 v[144:147], v160 offset:53248\n" \
  "ds_read_b128 v[148:151], v160 offset:54272\n" \
  "ds_read_b128 v[152:155], v160 offset:55296\n" \
  "ds_read_b128 v[156:159], v160 offset:56320\n" \
  "buffer_load_dwordx4 v[4:7], v165, s[36:39], s75 offen offset:1024\n" \
  "buffer_load_dwordx4 v[0:3], v165, s[36:39], s75 offen offset:0\n" \
  "buffer_load_dword v218, off, s[44:47], 0 sc1\n" \
  "v_readlane_b32 s67, v175, s70\n" \
  "v_readlane_b32 s65, v178, s70\n" \
  "v_readlane_b32 s76, v169, s70\n" \
  "v_readlane_b32 s77, v172, s70\n" \
  "s_add_u32 s70, s70, 1\n" \
  "v_add_u32 v163, s76, v162\n" \
  "s_branch Ls1_6_%=\n" \
  ".org Lpre1_0_0_%=+11264\n" \
  "Lpre1_1_7_%=:\n" \
  "ds_read_b128 v[128:131], v160 offset:57344\n" \
  "ds_read_b128 v[132:135], v160 offset:58368\n" \
  "ds_read_b128 v[136:139], v160 offset:59392\n" \
  "ds_read_b128 v[140:143], v160 offset:60416\n" \
  "buffer_load_dwordx4 v[4:7], v165, s[36:39], s75 offen offset:1024\n" \
  "buffer_load_dwordx4 v[0:3], v165, s[36:39], s75 offen offset:0\n" \
  "buffer_load_dword v218, off, s[44:47], 0 sc1\n" \
  "v_readlane_b32 s67, v175, s70\n" \
  "v_readlane_b32 s65, v178, s70\n" \
  "v_readlane_b32 s76, v169, s70\n" \
  "v_readlane_b32 s77, v172, s70\n" \
  "s_add_u32 s70, s70, 1\n" \
  "v_add_u32 v163, s76, v162\n" \
  "buffer_load_dwordx4 v[12:15], v165, s[36:39], s75 offen offset:3072\n" \
  "v_add_u32 v164, 4096, v163\n" \
  "s_cmp_gt_u32 s67, s68\n" \
  "s_cbranch_scc1 Lp165_%=\n" \
  "Lq165_%=:\n" \
  "buffer_load_dwordx4 v[8:11], v165, s[36:39], s75 offen offset:2048\n" \
  "buffer_load_dwordx4 v[16:19], v166, s[36:39], s75 offen offset:0\n" \
  "v_mul_f32 v188, s78, v188\n" \
  "v_mul_f32 v189, s78, v189\n" \
  "v_mul_f32 v190, s78, v190\n" \
  "v_mul_f32 v191, s78, v191\n" \
  "s_bitcmp1_b32 s66, 28\n" \
  "s_cbranch_scc1 Lp166_%=\n" \
  "buffer_store_dwordx4 v[188:191], v167, s[40:43], s49 offen offset:0 nt\n" \
  "Lq166_%=:\n" \
  "s_branch Ls1_7_%=\n" \
  ".org Lpre1_0_0_%=+11968\n" \
  "Lpre1_1_8_%=:\n" \
  "ds_read_b128 v[144:147], v160 offset:61440\n" \
  "ds_read_b128 v[148:151], v160 offset:62464\n" \
  "ds_read_b128 v[152:155], v160 offset:63488\n" \
  "ds_read_b128 v[156:159], v160 offset:64512\n" \
  "buffer_load_dwordx4 v[4:7], v165, s[36:39], s75 offen offset:1024\n" \
  "buffer_load_dwordx4 v[0:3], v165, s[36:39], s75 offen offset:0\n" \
  "buffer_load_dword v218, off, s[44:47], 0 sc1\n" \
  "v_readlane_b32 s67, v175, s70\n" \
  "v_readlane_b32 s65, v178, s70\n" \
  "v_readlane_b32 s76, v169, s70\n" \
  "v_readlane_b32 s77, v172, s70\n" \
  "s_add_u32 s70, s70, 1\n" \
  "v_add_u32 v163, s76, v162\n" \
  "buffer_load_dwordx4 v[12:15], v165, s[36:39], s75 offen offset:3072\n" \
  "v_add_u32 v164, 4096, v163\n" \
  "s_cmp_gt_u32 s67, s68\n" \
  "s_cbranch_scc1 Lp167_%=\n" \
  "Lq167_%=:\n" \
  "buffer_load_dwordx4 v[8:11], v165, s[36:39], s75 offen offset:2048\n" \
  "buffer_load_dwordx4 v[16:19], v166, s[36:39], s75 offen offset:0\n" \
  "v_mul_f32 v188, s78, v188\n" \
  "v_mul_f32 v189, s78, v189\n" \
  "v_mul_f32 v190, s78, v190\n" \
  "v_mul_f32 v191, s78, v191\n" \
  "s_bitcmp1_b32 s66, 28\n" \
  "s_cbranch_scc1 Lp168_%=\n" \
  "buffer_store_dwordx4 v[188:191], v167, s[40:43], s49 offen offset:0 nt\n" \
  "Lq168_%=:\n" \
  "buffer_load_dwordx4 v[24:27], v166, s[36:39], s75 offen offset:2048\n" \
  "v_mul_f32 v204, s78, v204\n" \
  "v_mul_f32 v205, s78, v205\n" \
  "v_mul_f32 v206, s78, v206\n" \
  "v_mul_f32 v207, s78, v207\n" \
  "s_bitcmp1_b32 s66, 28\n" \
  "s_cbranch_scc1 Lp169_%=\n" \
  "buffer_store_dwordx4 v[204:207], v167, s[40:43], s49 offen offset:64 nt\n" \
  "Lq169_%=:\n" \
  "s_and_b32 s73, s65, 0xf00001ff\n" \
  "buffer_load_dwordx4 v[20:23], v166, s[36:39], s75 offen offset:1024\n" \
  "buffer_load_dwordx4 v[28:31], v166, s[36:39], s75 offen offset:3072\n" \
  "s_bfe_u32 m0, s65, 0x40018\n" \
  "s_nop 0\n" \
  "s_movreld_b32 s56, s77\n" \
  "s_branch Ls1_8_%=\n" \
  ".p2align 6\n" \
  "Lz2_%=:\n" \
  "ds_read_b128 v[128:131], v161 offset:0\n" \
  "ds_read_b128 v[132:135], v161 offset:1024\n" \
  "ds_read_b128 v[136:139], v161 offset:2048\n" \
  "ds_read_b128 v[140:143], v161 offset:3072\n" \
  "s_add_u32 m0, s69, 61440\n" \
  "s_nop 0\n" \
  "buffer_load_dwordx4 v163, s[36:39], s63 offen lds\n" \
  "s_branch Lzr2_%=\n" \
  "Lpj2_%=:\n" \
  "s_setpc_b64 s[98:99]\n" \
  "Lsuf2_0_1_%=:\n" \
  "s_bitcmp1_b32 s65, 10\n" \
  "s_cbranch_scc1 Lsuf2_1_1_%=\n" \
  "ds_read_b128 v[144:147], v161 offset:28672\n" \
  "ds_read_b128 v[148:151], v161 offset:29696\n" \
  "ds_read_b128 v[152:155], v161 offset:30720\n" \
  "ds_read_b128 v[156:159], v161 offset:31744\n" \
  "s_add_u32 m0, s69, 36864\n" \
  "s_nop 0\n" \
  "buffer_load_dwordx4 v163, s[36:39], s57 offen lds\n" \
  "s_add_u32 m0, s69, 40960\n" \
  "s_nop 0\n" \
  "buffer_load_dwordx4 v163, s[36:39], s58 offen lds\n" \
  "s_add_u32 m0, s69, 45056\n" \
  "s_nop 0\n" \
  "buffer_load_dwordx4 v163, s[36:39], s59 offen lds\n" \
  "s_add_u32 m0, s69, 49152\n" \
  "s_nop 0\n" \
  "buffer_load_dwordx4 v163, s[36:39], s60 offen lds\n" \
  "buffer_load_dwordx4 v[36:39], v163, s[36:39], s75 offen offset:1024\n" \
  "buffer_load_dwordx4 v[32:35], v163, s[36:39], s75 offen offset:0\n" \
  "v_readlane_b32 s67, v175, s70\n" \
  "v_readlane_b32 s66, v178, s70\n" \
  "v_readlane_b32 s76, v169, s70\n" \
  "v_readlane_b32 s77, v172, s70\n" \
  "s_add_u32 s70, s70, 1\n" \
  "v_add_u32 v165, s76, v162\n" \
  "s_add_u32 m0, s69, 53248\n" \
  "s_nop 0\n" \
  "buffer_load_dwordx4 v163, s[36:39], s61 offen lds\n" \
  "buffer_load_dwordx4 v[44:47], v163, s[36:39], s75 offen offset:3072\n" \
  "s_bitcmp1_b32 s65, 23\n" \
  "s_cbranch_scc1 Lp170_%=\n" \
  "Lq170_%=:\n" \
  "v_add_u32 v166, 4096, v165\n" \
  "s_cmp_gt_u32 s67, s68\n" \
  "s_cbranch_scc1 Lp171_%=\n" \
  "Lq171_%=:\n" \
  "buffer_load_dwordx4 v[40:43], v163, s[36:39], s75 offen offset:2048\n" \
  "buffer_load_dwordx4 v[48:51], v164, s[36:39], s75 offen offset:0\n" \
  "v_mul_f32 v192, s78, v192\n" \
  "v_mul_f32 v193, s78, v193\n" \
  "v_mul_f32 v194, s78, v194\n" \
  "v_mul_f32 v195, s78, v195\n" \
  "s_bitcmp1_b32 s65, 28\n" \
  "s_cbranch_scc1 Lp172_%=\n" \
  "buffer_store_dwordx4 v[192:195], v167, s[40:43], s50 offen offset:0 nt\n" \
  "Lq172_%=:\n" \
  "s_add_u32 m0, s69, 57344\n" \
  "s_nop 0\n" \
  "buffer_load_dwordx4 v163, s[36:39], s62 offen lds\n" \
  "buffer_load_dwordx4 v[56:59], v164, s[36:39], s75 offen offset:2048\n" \
  "v_mul_f32 v208, s78, v208\n" \
  "v_mul_f32 v209, s78, v209\n" \
  "v_mul_f32 v210, s78, v210\n" \
  "v_mul_f32 v211, s78, v211\n" \
  "s_bitcmp1_b32 s65, 28\n" \
  "s_cbranch_scc1 Lp173_%=\n" \
  "buffer_store_dwordx4 v[208:211], v167, s[40:43], s50 offen offset:64 nt\n" \
  "Lq173_%=:\n" \
  "s_and_b32 s73, s66, 0xf00001ff\n" \
  "buffer_load_dwordx4 v[52:55], v164, s[36:39], s75 offen offset:1024\n" \
  "buffer_load_dwordx4 v[60:63], v164, s[36:39], s75 offen offset:3072\n" \
  "s_bfe_u32 m0, s66, 0x40018\n" \
  "s_nop 0\n" \
  "s_movreld_b32 s56, s77\n" \
  "s_branch Ls2_8_%=\n" \
  "Lsuf2_0_2_%=:\n" \
  "s_bitcmp1_b32 s65, 10\n" \
  "s_cbranch_scc1 Lsuf2_1_2_%=\n" \
  "ds_read_b128 v[144:147], v161 offset:28672\n" \
  "ds_read_b128 v[148:151], v161 offset:29696\n" \
  "ds_read_b128 v[152:155], v161 offset:30720\n" \
  "ds_read_b128 v[156:159], v161 offset:31744\n" \
  "s_add_u32 m0, s69, 40960\n" \
  "s_nop 0\n" \
  "buffer_load_dwordx4 v163, s[36:39], s58 offen lds\n" \
  "s_add_u32 m0, s69, 45056\n" \
  "s_nop 0\n" \
  "buffer_load_dwordx4 v163, s[36:39], s59 offen lds\n" \
  "s_add_u32 m0, s69, 49152\n" \
  "s_nop 0\n" \
  "buffer_load_dwordx4 v163, s[36:39], s60 offen lds\n" \
  "buffer_load_dwordx4 v[36:39], v163, s[36:39], s75 offen offset:1024\n" \
  "buffer_load_dwordx4 v[32:35], v163, s[36:39], s75 offen offset:0\n" \
  "v_readlane_b32 s67, v175, s70\n" \
  "v_readlane_b32 s66, v178, s70\n" \
  "v_readlane_b32 s76, v169, s70\n" \
  "v_readlane_b32 s77, v172, s70\n" \
  "s_add_u32 s70, s70, 1\n" \
  "v_add_u32 v165, s76, v162\n" \
  "s_add_u32 m0, s69, 53248\n" \
  "s_nop 0\n" \
  "buffer_load_dwordx4 v163, s[36:39], s61 offen lds\n" \
  "buffer_load_dwordx4 v[44:47], v163, s[36:39], s75 offen offset:3072\n" \
  "s_bitcmp1_b32 s65, 23\n" \
  "s_cbranch_scc1 Lp174_%=\n" \
  "Lq174_%=:\n" \
  "v_add_u32 v166, 4096, v165\n" \
  "s_cmp_gt_u32 s67, s68\n" \
  "s_cbranch_scc1 Lp175_%=\n" \
  "Lq175_%=:\n" \
  "buffer_load_dwordx4 v[40:43], v163, s[36:39], s75 offen offset:2048\n" \
  "buffer_load_dwordx4 v[48:51], v164, s[36:39], s75 offen offset:0\n" \
  "v_mul_f32 v192, s78, v192\n" \
  "v_mul_f32 v193, s78, v193\n" \
  "v_mul_f32 v194, s78, v194\n" \
  "v_mul_f32 v195, s78, v195\n" \
  "s_bitcmp1_b32 s65, 28\n" \
  "s_cbranch_scc1 Lp176_%=\n" \
  "buffer_store_dwordx4 v[192:195], v167, s[40:43], s50 offen offset:0 nt\n" \
  "Lq176_%=:\n" \
  "s_add_u32 m0, s69, 57344\n" \
  "s_nop 0\n" \
  "buffer_load_dwordx4 v163, s[36:39], s62 offen lds\n" \
  "buffer_load_dwordx4 v[56:59], v164, s[36:39], s75 offen offset:2048\n" \
  "v_mul_f32 v208, s78, v208\n" \
  "v_mul_f32 v209, s78, v209\n" \
  "v_mul_f32 v210, s78, v210\n" \
  "v_mul_f32 v211, s78, v211\n" \
  "s_bitcmp1_b32 s65, 28\n" \
  "s_cbranch_scc1 Lp177_%=\n" \
  "buffer_store_dwordx4 v[208:211], v167, s[40:43], s50 offen offset:64 nt\n" \
  "Lq177_%=:\n" \
  "s_and_b32 s73, s66, 0xf00001ff\n" \
  "buffer_load_dwordx4 v[52:55], v164, s[36:39], s75 offen offset:1024\n" \
  "buffer_load_dwordx4 v[60:63], v164, s[36:39], s75 offen offset:3072\n" \
  "s_bfe_u32 m0, s66, 0x40018\n" \
  "s_nop 0\n" \
  "s_movreld_b32 s56, s77\n" \
  "s_branch Ls2_8_%=\n" \
  "Lsuf2_0_3_%=:\n" \
  "s_bitcmp1_b32 s65, 10\n" \
  "s_cbranch_scc1 Lsuf2_1_3_%=\n" \
  "ds_read_b128 v[144:147], v161 offset:28672\n" \
  "ds_read_b128 v[148:151], v161 offset:29696\n" \
  "ds_read_b128 v[152:155], v161 offset:30720\n" \
  "ds_read_b128 v[156:159], v161 offset:31744\n" \
  "s_add_u32 m0, s69, 45056\n" \
  "s_nop 0\n" \
  "buffer_load_dwordx4 v163, s[36:39], s59 offen lds\n" \
  "s_add_u32 m0, s69, 49152\n" \
  "s_nop 0\n" \
  "buffer_load_dwordx4 v163, s[36:39], s60 offen lds\n" \
  "buffer_load_dwordx4 v[36:39], v163, s[36:39], s75 offen offset:1024\n" \
  "buffer_load_dwordx4 v[32:35], v163, s[36:39], s75 offen offset:0\n" \
  "v_readlane_b32 s67, v175, s70\n" \
  "v_readlane_b32 s66, v178, s70\n" \
  "v_readlane_b32 s76, v169, s70\n" \
  "v_readlane_b32 s77, v172, s70\n" \
  "s_add_u32 s70, s70, 1\n" \
  "v_add_u32 v165, s76, v162\n" \
  "s_add_u32 m0, s69, 53248\n" \
  "s_nop 0\n" \
  "buffer_load_dwordx4 v163, s[36:39], s61 offen lds\n" \
  "buffer_load_dwordx4 v[44:47], v163, s[36:39], s75 offen offset:3072\n" \
  "s_bitcmp1_b32 s65, 23\n" \
  "s_cbranch_scc1 Lp178_%=\n" \
  "Lq178_%=:\n" \
  "v_add_u32 v166, 4096, v165\n" \
  "s_cmp_gt_u32 s67, s68\n" \
  "s_cbranch_scc1 Lp179_%=\n" \
  "Lq179_%=:\n" \
  "buffer_load_dwordx4 v[40:43], v163, s[36:39], s75 offen offset:2048\n" \
  "buffer_load_dwordx4 v[48:51], v164, s[36:39], s75 offen offset:0\n" \
  "v_mul_f32 v192, s78, v192\n" \
  "v_mul_f32 v193, s78, v193\n" \
  "v_mul_f32 v194, s78, v194\n" \
  "v_mul_f32 v195, s78, v195\n" \
  "s_bitcmp1_b32 s65, 28\n" \
  "s_cbranch_scc1 Lp180_%=\n" \
  "buffer_store_dwordx4 v[192:195], v167, s[40:43], s50 offen offset:0 nt\n" \
  "Lq180_%=:\n" \
  "s_add_u32 m0, s69, 57344\n" \
  "s_nop 0\n" \
  "buffer_load_dwordx4 v163, s[36:39], s62 offen lds\n" \
  "buffer_load_dwordx4 v[56:59], v164, s[36:39], s75 offen offset:2048\n" \
  "v_mul_f32 v208, s78, v208\n" \
  "v_mul_f32 v209, s78, v209\n" \
  "v_mul_f32 v210, s78, v210\n" \
  "v_mul_f32 v211, s78, v211\n" \
  "s_bitcmp1_b32 s65, 28\n" \
  "s_cbranch_scc1 Lp181_%=\n" \
  "buffer_store_dwordx4 v[208:211], v167, s[40:43], s50 offen offset:64 nt\n" \
  "Lq181_%=:\n" \
  "s_and_b32 s73, s66, 0xf00001ff\n" \
  "buffer_load_dwordx4 v[52:55], v164, s[36:39], s75 offen offset:1024\n" \
  "buffer_load_dwordx4 v[60:63], v164, s[36:39], s75 offen offset:3072\n" \
  "s_bfe_u32 m0, s66, 0x40018\n" \
  "s_nop 0\n" \
  "s_movreld_b32 s56, s77\n" \
  "s_branch Ls2_8_%=\n" \
  "Lsuf2_0_4_%=:\n" \
  "s_bitcmp1_b32 s65, 10\n" \
  "s_cbranch_scc1 Lsuf2_1_4_%=\n" \
  "ds_read_b128 v[144:147], v161 offset:28672\n" \
  "ds_read_b128 v[148:151], v161 offset:29696\n" \
  "ds_read_b128 v[152:155], v161 offset:30720\n" \
  "ds_read_b128 v[156:159], v161 offset:31744\n" \
  "s_add_u32 m0, s69, 49152\n" \
  "s_nop 0\n" \
  "buffer_load_dwordx4 v163, s[36:39], s60 offen lds\n" \
  "buffer_load_dwordx4 v[36:39], v163, s[36:39], s75 offen offset:1024\n" \
  "buffer_load_dwordx4 v[32:35], v163, s[36:39], s75 offen offset:0\n" \
  "v_readlane_b32 s67, v175, s70\n" \
  "v_readlane_b32 s66, v178, s70\n" \
  "v_readlane_b32 s76, v169, s70\n" \
  "v_readlane_b32 s77, v172, s70\n" \
  "s_add_u32 s70, s70, 1\n" \
  "v_add_u32 v165, s76, v162\n" \
  "s_add_u32 m0, s69, 53248\n" \
  "s_nop 0\n" \
  "buffer_load_dwordx4 v163, s[36:39], s61 offen lds\n" \
  "buffer_load_dwordx4 v[44:47], v163, s[36:39], s75 offen offset:3072\n" \
  "s_bitcmp1_b32 s65, 23\n" \
  "s_cbranch_scc1 Lp182_%=\n" \
  "Lq182_%=:\n" \
  "v_add_u32 v166, 4096, v165\n" \
  "s_cmp_gt_u32 s67, s68\n" \
  "s_cbranch_scc1 Lp183_%=\n" \
  "Lq183_%=:\n" \
  "buffer_load_dwordx4 v[40:43], v163, s[36:39], s75 offen offset:2048\n" \
  "buffer_load_dwordx4 v[48:51], v164, s[36:39], s75 offen offset:0\n" \
  "v_mul_f32 v192, s78, v192\n" \
  "v_mul_f32 v193, s78, v193\n" \
  "v_mul_f32 v194, s78, v194\n" \
  "v_mul_f32 v195, s78, v195\n" \
  "s_bitcmp1_b32 s65, 28\n" \
  "s_cbranch_scc1 Lp184_%=\n" \
  "buffer_store_dwordx4 v[192:195], v167, s[40:43], s50 offen offset:0 nt\n" \
  "Lq184_%=:\n" \
  "s_add_u32 m0, s69, 57344\n" \
  "s_nop 0\n" \
  "buffer_load_dwordx4 v163, s[36:39], s62 offen lds\n" \
  "buffer_load_dwordx4 v[56:59], v164, s[36:39], s75 offen offset:2048\n" \
  "v_mul_f32 v208, s78, v208\n" \
  "v_mul_f32 v209, s78, v209\n" \
  "v_mul_f32 v210, s78, v210\n" \
  "v_mul_f32 v211, s78, v211\n" \
  "s_bitcmp1_b32 s65, 28\n" \
  "s_cbranch_scc1 Lp185_%=\n" \
  "buffer_store_dwordx4 v[208:211], v167, s[40:43], s50 offen offset:64 nt\n" \
  "Lq185_%=:\n" \
  "s_and_b32 s73, s66, 0xf00001ff\n" \
  "buffer_load_dwordx4 v[52:55], v164, s[36:39], s75 offen offset:1024\n" \
  "buffer_load_dwordx4 v[60:63], v164, s[36:39], s75 offen offset:3072\n" \
  "s_bfe_u32 m0, s66, 0x40018\n" \
  "s_nop 0\n" \
  "s_movreld_b32 s56, s77\n" \
  "s_branch Ls2_8_%=\n" \
  "Lsuf2_0_5_%=:\n" \
  "s_bitcmp1_b32 s65, 10\n" \
  "s_cbranch_scc1 Lsuf2_1_5_%=\n" \
  "ds_read_b128 v[144:147], v161 offset:28672\n" \
  "ds_read_b128 v[148:151], v161 offset:29696\n" \
  "ds_read_b128 v[152:155], v161 offset:30720\n" \
  "ds_read_b128 v[156:159], v161 offset:31744\n" \
  "s_add_u32 m0, s69, 53248\n" \
  "s_nop 0\n" \
  "buffer_load_dwordx4 v163, s[36:39], s61 offen lds\n" \
  "buffer_load_dwordx4 v[44:47], v163, s[36:39], s75 offen offset:3072\n" \
  "s_bitcmp1_b32 s65, 23\n" \
  "s_cbranch_scc1 Lp186_%=\n" \
  "Lq186_%=:\n" \
  "v_add_u32 v166, 4096, v165\n" \
  "s_cmp_gt_u32 s67, s68\n" \
  "s_cbranch_scc1 Lp187_%=\n" \
  "Lq187_%=:\n" \
  "buffer_load_dwordx4 v[40:43], v163, s[36:39], s75 offen offset:2048\n" \
  "buffer_load_dwordx4 v[48:51], v164, s[36:39], s75 offen offset:0\n" \
  "v_mul_f32 v192, s78, v192\n" \
  "v_mul_f32 v193, s78, v193\n" \
  "v_mul_f32 v194, s78, v194\n" \
  "v_mul_f32 v195, s78, v195\n" \
  "s_bitcmp1_b32 s65, 28\n" \
  "s_cbranch_scc1 Lp188_%=\n" \
  "buffer_store_dwordx4 v[192:195], v167, s[40:43], s50 offen offset:0 nt\n" \
  "Lq188_%=:\n" \
  "s_add_u32 m0, s69, 57344\n" \
  "s_nop 0\n" \
  "buffer_load_dwordx4 v163, s[36:39], s62 offen lds\n" \
  "buffer_load_dwordx4 v[56:59], v164, s[36:39], s75 offen offset:2048\n" \
  "v_mul_f32 v208, s78, v208\n" \
  "v_mul_f32 v209, s78, v209\n" \
  "v_mul_f32 v210, s78, v210\n" \
  "v_mul_f32 v211, s78, v211\n" \
  "s_bitcmp1_b32 s65, 28\n" \
  "s_cbranch_scc1 Lp189_%=\n" \
  "buffer_store_dwordx4 v[208:211], v167, s[40:43], s50 offen offset:64 nt\n" \
  "Lq189_%=:\n" \
  "s_and_b32 s73, s66, 0xf00001ff\n" \
  "buffer_load_dwordx4 v[52:55], v164, s[36:39], s75 offen offset:1024\n" \
  "buffer_load_dwordx4 v[60:63], v164, s[36:39], s75 offen offset:3072\n" \
  "s_bfe_u32 m0, s66, 0x40018\n" \
  "s_nop 0\n" \
  "s_movreld_b32 s56, s77\n" \
  "s_branch Ls2_8_%=\n" \
  "Lsuf2_0_6_%=:\n" \
  "s_bitcmp1_b32 s65, 10\n" \
  "s_cbranch_scc1 Lsuf2_1_6_%=\n" \
  "ds_read_b128 v[144:147], v161 offset:28672\n" \
  "ds_read_b128 v[148:151], v161 offset:29696\n" \
  "ds_read_b128 v[152:155], v161 offset:30720\n" \
  "ds_read_b128 v[156:159], v161 offset:31744\n" \
  "s_add_u32 m0, s69, 57344\n" \
  "s_nop 0\n" \
  "buffer_load_dwordx4 v163, s[36:39], s62 offen lds\n" \
  "buffer_load_dwordx4 v[56:59], v164, s[36:39], s75 offen offset:2048\n" \
  "v_mul_f32 v208, s78, v208\n" \
  "v_mul_f32 v209, s78, v209\n" \
  "v_mul_f32 v210, s78, v210\n" \
  "v_mul_f32 v211, s78, v211\n" \
  "s_bitcmp1_b32 s65, 28\n" \
  "s_cbranch_scc1 Lp190_%=\n" \
  "buffer_store_dwordx4 v[208:211], v167, s[40:43], s50 offen offset:64 nt\n" \
  "Lq190_%=:\n" \
  "s_and_b32 s73, s66, 0xf00001ff\n" \
  "buffer_load_dwordx4 v[52:55], v164, s[36:39], s75 offen offset:1024\n" \
  "buffer_load_dwordx4 v[60:63], v164, s[36:39], s75 offen offset:3072\n" \
  "s_bfe_u32 m0, s66, 0x40018\n" \
  "s_nop 0\n" \
  "s_movreld_b32 s56, s77\n" \
  "s_branch Ls2_8_%=\n" \
  "Lsuf2_1_1_%=:\n" \
  "buffer_load_dwordx4 v[36:39], v163, s[36:39], s75 offen offset:1024\n" \
  "buffer_load_dwordx4 v[32:35], v163, s[36:39], s75 offen offset:0\n" \
  "v_readlane_b32 s67, v175, s70\n" \
  "v_readlane_b32 s66, v178, s70\n" \
  "v_readlane_b32 s76, v169, s70\n" \
  "v_readlane_b32 s77, v172, s70\n" \
  "s_add_u32 s70, s70, 1\n" \
  "v_add_u32 v165, s76, v162\n" \
  "buffer_load_dwordx4 v[44:47], v163, s[36:39], s75 offen offset:3072\n" \
  "s_bitcmp1_b32 s65, 23\n" \
  "s_cbranch_scc1 Lp191_%=\n" \
  "Lq191_%=:\n" \
  "v_add_u32 v166, 4096, v165\n" \
  "s_cmp_gt_u32 s67, s68\n" \
  "s_cbranch_scc1 Lp192_%=\n" \
  "Lq192_%=:\n" \
  "buffer_load_dwordx4 v[40:43], v163, s[36:39], s75 offen offset:2048\n" \
  "buffer_load_dwordx4 v[48:51], v164, s[36:39], s75 offen offset:0\n" \
  "v_mul_f32 v192, s78, v192\n" \
  "v_mul_f32 v193, s78, v193\n" \
  "v_mul_f32 v194, s78, v194\n" \
  "v_mul_f32 v195, s78, v195\n" \
  "s_bitcmp1_b32 s65, 28\n" \
  "s_cbranch_scc1 Lp193_%=\n" \
  "buffer_store_dwordx4 v[192:195], v167, s[40:43], s50 offen offset:0 nt\n" \
  "Lq193_%=:\n" \
  "buffer_load_dwordx4 v[56:59], v164, s[36:39], s75 offen offset:2048\n" \
  "v_mul_f32 v208, s78, v208\n" \
  "v_mul_f32 v209, s78, v209\n" \
  "v_mul_f32 v210, s78, v210\n" \
  "v_mul_f32 v211, s78, v211\n" \
  "s_bitcmp1_b32 s65, 28\n" \
  "s_cbranch_scc1 Lp194_%=\n" \
  "buffer_store_dwordx4 v[208:211], v167, s[40:43], s50 offen offset:64 nt\n" \
  "Lq194_%=:\n" \
  "s_and_b32 s73, s66, 0xf00001ff\n" \
  "buffer_load_dwordx4 v[52:55], v164, s[36:39], s75 offen offset:1024\n" \
  "buffer_load_dwordx4 v[60:63], v164, s[36:39], s75 offen offset:3072\n" \
  "s_bfe_u32 m0, s66, 0x40018\n" \
  "s_nop 0\n" \
  "s_movreld_b32 s56, s77\n" \
  "s_branch Ls2_8_%=\n" \
  "Lsuf2_1_2_%=:\n" \
  "buffer_load_dwordx4 v[36:39], v163, s[36:39], s75 offen offset:1024\n" \
  "buffer_load_dwordx4 v[32:35], v163, s[36:39], s75 offen offset:0\n" \
  "v_readlane_b32 s67, v175, s70\n" \
  "v_readlane_b32 s66, v178, s70\n" \
  "v_readlane_b32 s76, v169, s70\n" \
  "v_readlane_b32 s77, v172, s70\n" \
  "s_add_u32 s70, s70, 1\n" \
  "v_add_u32 v165, s76, v162\n" \
  "buffer_load_dwordx4 v[44:47], v163, s[36:39], s75 offen offset:3072\n" \
  "s_bitcmp1_b32 s65, 23\n" \
  "s_cbranch_scc1 Lp195_%=\n" \
  "Lq195_%=:\n" \
  "v_add_u32 v166, 4096, v165\n" \
  "s_cmp_gt_u32 s67, s68\n" \
  "s_cbranch_scc1 Lp196_%=\n" \
  "Lq196_%=:\n" \
  "buffer_load_dwordx4 v[40:43], v163, s[36:39], s75 offen offset:2048\n" \
  "buffer_load_dwordx4 v[48:51], v164, s[36:39], s75 offen offset:0\n" \
  "v_mul_f32 v192, s78, v192\n" \
  "v_mul_f32 v193, s78, v193\n" \
  "v_mul_f32 v194, s78, v194\n" \
  "v_mul_f32 v195, s78, v195\n" \
  "s_bitcmp1_b32 s65, 28\n" \
  "s_cbranch_scc1 Lp197_%=\n" \
  "buffer_store_dwordx4 v[192:195], v167, s[40:43], s50 offen offset:0 nt\n" \
  "Lq197_%=:\n" \
  "buffer_load_dwordx4 v[56:59], v164, s[36:39], s75 offen offset:2048\n" \
  "v_mul_f32 v208, s78, v208\n" \
  "v_mul_f32 v209, s78, v209\n" \
  "v_mul_f32 v210, s78, v210\n" \
  "v_mul_f32 v211, s78, v211\n" \
  "s_bitcmp1_b32 s65, 28\n" \
  "s_cbranch_scc1 Lp198_%=\n" \
  "buffer_store_dwordx4 v[208:211], v167, s[40:43], s50 offen offset:64 nt\n" \
  "Lq198_%=:\n" \
  "s_and_b32 s73, s66, 0xf00001ff\n" \
  "buffer_load_dwordx4 v[52:55], v164, s[36:39], s75 offen offset:1024\n" \
  "buffer_load_dwordx4 v[60:63], v164, s[36:39], s75 offen offset:3072\n" \
  "s_bfe_u32 m0, s66, 0x40018\n" \
  "s_nop 0\n" \
  "s_movreld_b32 s56, s77\n" \
  "s_branch Ls2_8_%=\n" \
  "Lsuf2_1_3_%=:\n" \
  "buffer_load_dwordx4 v[36:39], v163, s[36:39], s75 offen offset:1024\n" \
  "buffer_load_dwordx4 v[32:35], v163, s[36:39], s75 offen offset:0\n" \
  "v_readlane_b32 s67, v175, s70\n" \
  "v_readlane_b32 s66, v178, s70\n" \
  "v_readlane_b32 s76, v169, s70\n" \
  "v_readlane_b32 s77, v172, s70\n" \
  "s_add_u32 s70, s70, 1\n" \
  "v_add_u32 v165, s76, v162\n" \
  "buffer_load_dwordx4 v[44:47], v163, s[36:39], s75 offen offset:3072\n" \
  "s_bitcmp1_b32 s65, 23\n" \
  "s_cbranch_scc1 Lp199_%=\n" \
  "Lq199_%=:\n" \
  "v_add_u32 v166, 4096, v165\n" \
  "s_cmp_gt_u32 s67, s68\n" \
  "s_cbranch_scc1 Lp200_%=\n" \
  "Lq200_%=:\n" \
  "buffer_load_dwordx4 v[40:43], v163, s[36:39], s75 offen offset:2048\n" \
  "buffer_load_dwordx4 v[48:51], v164, s[36:39], s75 offen offset:0\n" \
  "v_mul_f32 v192, s78, v192\n" \
  "v_mul_f32 v193, s78, v193\n" \
  "v_mul_f32 v194, s78, v194\n" \
  "v_mul_f32 v195, s78, v195\n" \
  "s_bitcmp1_b32 s65, 28\n" \
  "s_cbranch_scc1 Lp201_%=\n" \
  "buffer_store_dwordx4 v[192:195], v167, s[40:43], s50 offen offset:0 nt\n" \
  "Lq201_%=:\n" \
  "buffer_load_dwordx4 v[56:59], v164, s[36:39], s75 offen offset:2048\n" \
  "v_mul_f32 v208, s78, v208\n" \
  "v_mul_f32 v209, s78, v209\n" \
  "v_mul_f32 v210, s78, v210\n" \
  "v_mul_f32 v211, s78, v211\n" \
  "s_bitcmp1_b32 s65, 28\n" \
  "s_cbranch_scc1 Lp202_%=\n" \
  "buffer_store_dwordx4 v[208:211], v167, s[40:43], s50 offen offset:64 nt\n" \
  "Lq202_%=:\n" \
  "s_and_b32 s73, s66, 0xf00001ff\n" \
  "buffer_load_dwordx4 v[52:55], v164, s[36:39], s75 offen offset:1024\n" \
  "buffer_load_dwordx4 v[60:63], v164, s[36:39], s75 offen offset:3072\n" \
  "s_bfe_u32 m0, s66, 0x40018\n" \
  "s_nop 0\n" \
  "s_movreld_b32 s56, s77\n" \
  "s_branch Ls2_8_%=\n" \
  "Lsuf2_1_4_%=:\n" \
  "buffer_load_dwordx4 v[36:39], v163, s[36:39], s75 offen offset:1024\n" \
  "buffer_load_dwordx4 v[32:35], v163, s[36:39], s75 offen offset:0\n" \
  "v_readlane_b32 s67, v175, s70\n" \
  "v_readlane_b32 s66, v178, s70\n" \
  "v_readlane_b32 s76, v169, s70\n" \
  "v_readlane_b32 s77, v172, s70\n" \
  "s_add_u32 s70, s70, 1\n" \
  "v_add_u32 v165, s76, v162\n" \
  "buffer_load_dwordx4 v[44:47], v163, s[36:39], s75 offen offset:3072\n" \
  "s_bitcmp1_b32 s65, 23\n" \
  "s_cbranch_scc1 Lp203_%=\n" \
  "Lq203_%=:\n" \
  "v_add_u32 v166, 4096, v165\n" \
  "s_cmp_gt_u32 s67, s68\n" \
  "s_cbranch_scc1 Lp204_%=\n" \
  "Lq204_%=:\n" \
  "buffer_load_dwordx4 v[40:43], v163, s[36:39], s75 offen offset:2048\n" \
  "buffer_load_dwordx4 v[48:51], v164, s[36:39], s75 offen offset:0\n" \
  "v_mul_f32 v192, s78, v192\n" \
  "v_mul_f32 v193, s78, v193\n" \
  "v_mul_f32 v194, s78, v194\n" \
  "v_mul_f32 v195, s78, v195\n" \
  "s_bitcmp1_b32 s65, 28\n" \
  "s_cbranch_scc1 Lp205_%=\n" \
  "buffer_store_dwordx4 v[192:195], v167, s[40:43], s50 offen offset:0 nt\n" \
  "Lq205_%=:\n" \
  "buffer_load_dwordx4 v[56:59], v164, s[36:39], s75 offen offset:2048\n" \
  "v_mul_f32 v208, s78, v208\n" \
  "v_mul_f32 v209, s78, v209\n" \
  "v_mul_f32 v210, s78, v210\n" \
  "v_mul_f32 v211, s78, v211\n" \
  "s_bitcmp1_b32 s65, 28\n" \
  "s_cbranch_scc1 Lp206_%=\n" \
  "buffer_store_dwordx4 v[208:211], v167, s[40:43], s50 offen offset:64 nt\n" \
  "Lq206_%=:\n" \
  "s_and_b32 s73, s66, 0xf00001ff\n" \
  "buffer_load_dwordx4 v[52:55], v164, s[36:39], s75 offen offset:1024\n" \
  "buffer_load_dwordx4 v[60:63], v164, s[36:39], s75 offen offset:3072\n" \
  "s_bfe_u32 m0, s66, 0x40018\n" \
  "s_nop 0\n" \
  "s_movreld_b32 s56, s77\n" \
  "s_branch Ls2_8_%=\n" \
  "Lsuf2_1_5_%=:\n" \
  "buffer_load_dwordx4 v[44:47], v163, s[36:39], s75 offen offset:3072\n" \
  "s_bitcmp1_b32 s65, 23\n" \
  "s_cbranch_scc1 Lp207_%=\n" \
  "Lq207_%=:\n" \
  "v_add_u32 v166, 4096, v165\n" \
  "s_cmp_gt_u32 s67, s68\n" \
  "s_cbranch_scc1 Lp208_%=\n" \
  "Lq208_%=:\n" \
  "buffer_load_dwordx4 v[40:43], v163, s[36:39], s75 offen offset:2048\n" \
  "buffer_load_dwordx4 v[48:51], v164, s[36:39], s75 offen offset:0\n" \
  "v_mul_f32 v192, s78, v192\n" \
  "v_mul_f32 v193, s78, v193\n" \
  "v_mul_f32 v194, s78, v194\n" \
  "v_mul_f32 v195, s78, v195\n" \
  "s_bitcmp1_b32 s65, 28\n" \
  "s_cbranch_scc1 Lp209_%=\n" \
  "buffer_store_dwordx4 v[192:195], v167, s[40:43], s50 offen offset:0 nt\n" \
  "Lq209_%=:\n" \
  "buffer_load_dwordx4 v[56:59], v164, s[36:39], s75 offen offset:2048\n" \
  "v_mul_f32 v208, s78, v208\n" \
  "v_mul_f32 v209, s78, v209\n" \
  "v_mul_f32 v210, s78, v210\n" \
  "v_mul_f32 v211, s78, v211\n" \
  "s_bitcmp1_b32 s65, 28\n" \
  "s_cbranch_scc1 Lp210_%=\n" \
  "buffer_store_dwordx4 v[208:211], v167, s[40:43], s50 offen offset:64 nt\n" \
  "Lq210_%=:\n" \
  "s_and_b32 s73, s66, 0xf00001ff\n" \
  "buffer_load_dwordx4 v[52:55], v164, s[36:39], s75 offen offset:1024\n" \
  "buffer_load_dwordx4 v[60:63], v164, s[36:39], s75 offen offset:3072\n" \
  "s_bfe_u32 m0, s66, 0x40018\n" \
  "s_nop 0\n" \
  "s_movreld_b32 s56, s77\n" \
  "s_branch Ls2_8_%=\n" \
  "Lsuf2_1_6_%=:\n" \
  "buffer_load_dwordx4 v[56:59], v164, s[36:39], s75 offen offset:2048\n" \
  "v_mul_f32 v208, s78, v208\n" \
  "v_mul_f32 v209, s78, v209\n" \
  "v_mul_f32 v210, s78, v210\n" \
  "v_mul_f32 v211, s78, v211\n" \
  "s_bitcmp1_b32 s65, 28\n" \
  "s_cbranch_scc1 Lp211_%=\n" \
  "buffer_store_dwordx4 v[208:211], v167, s[40:43], s50 offen offset:64 nt\n" \
  "Lq211_%=:\n" \
  "s_and_b32 s73, s66, 0xf00001ff\n" \
  "buffer_load_dwordx4 v[52:55], v164, s[36:39], s75 offen offset:1024\n" \
  "buffer_load_dwordx4 v[60:63], v164, s[36:39], s75 offen offset:3072\n" \
  "s_bfe_u32 m0, s66, 0x40018\n" \
  "s_nop 0\n" \
  "s_movreld_b32 s56, s77\n" \
  "s_branch Ls2_8_%=\n" \
  ".p2align 6\n" \
  "Lpre2_0_0_%=:\n" \
  "s_branch Ls2_1_%=\n" \
  ".org Lpre2_0_0_%=+704\n" \
  "Lpre2_0_1_%=:\n" \
  "s_branch Ls2_1_%=\n" \
  ".org Lpre2_0_0_%=+1408\n" \
  "Lpre2_0_2_%=:\n" \
  "ds_read_b128 v[144:147], v161 offset:4096\n" \
  "ds_read_b128 v[148:151], v161 offset:5120\n" \
  "ds_read_b128 v[152:155], v161 offset:6144\n" \
  "ds_read_b128 v[156:159], v161 offset:7168\n" \
  "s_add_u32 m0, s69, 32768\n" \
  "s_nop 0\n" \
  "buffer_load_dwordx4 v163, s[36:39], s56 offen lds\n" \
  "s_branch Ls2_2_%=\n" \
  ".org Lpre2_0_0_%=+2112\n" \
  "Lpre2_0_3_%=:\n" \
  "ds_read_b128 v[128:131], v161 offset:8192\n" \
  "ds_read_b128 v[132:135], v161 offset:9216\n" \
  "ds_read_b128 v[136:139], v161 offset:10240\n" \
  "ds_read_b128 v[140:143], v161 offset:11264\n" \
  "s_add_u32 m0, s69, 32768\n" \
  "s_nop 0\n" \
  "buffer_load_dwordx4 v163, s[36:39], s56 offen lds\n" \
  "s_add_u32 m0, s69, 36864\n" \
  "s_nop 0\n" \
  "buffer_load_dwordx4 v163, s[36:39], s57 offen lds\n" \
  "s_branch Ls2_3_%=\n" \
  ".org Lpre2_0_0_%=+2816\n" \
  "Lpre2_0_4_%=:\n" \
  "ds_read_b128 v[144:147], v161 offset:12288\n" \
  "ds_read_b128 v[148:151], v161 offset:13312\n" \
  "ds_read_b128 v[152:155], v161 offset:14336\n" \
  "ds_read_b128 v[156:159], v161 offset:15360\n" \
  "s_add_u32 m0, s69, 32768\n" \
  "s_nop 0\n" \
  "buffer_load_dwordx4 v163, s[36:39], s56 offen lds\n" \
  "s_add_u32 m0, s69, 36864\n" \
  "s_nop 0\n" \
  "buffer_load_dwordx4 v163, s[36:39], s57 offen lds\n" \
  "s_add_u32 m0, s69, 40960\n" \
  "s_nop 0\n" \
  "buffer_load_dwordx4 v163, s[36:39], s58 offen lds\n" \
  "s_branch Ls2_4_%=\n" \
  ".org Lpre2_0_0_%=+3520\n" \
  "Lpre2_0_5_%=:\n" \
  "ds_read_b128 v[128:131], v161 offset:16384\n" \
  "ds_read_b128 v[132:135], v161 offset:17408\n" \
  "ds_read_b128 v[136:139], v161 offset:18432\n" \
  "ds_read_b128 v[140:143], v161 offset:19456\n" \
  "s_add_u32 m0, s69, 32768\n" \
  "s_nop 0\n" \
  "buffer_load_dwordx4 v163, s[36:39], s56 offen lds\n" \
  "s_add_u32 m0, s69, 36864\n" \
  "s_nop 0\n" \
  "buffer_load_dwordx4 v163, s[36:39], s57 offen lds\n" \
  "s_add_u32 m0, s69, 40960\n" \
  "s_nop 0\n" \
  "buffer_load_dwordx4 v163, s[36:39], s58 offen lds\n" \
  "s_add_u32 m0, s69, 45056\n" \
  "s_nop 0\n" \
  "buffer_load_dwordx4 v163, s[36:39], s59 offen lds\n" \
  "s_branch Ls2_5_%=\n" \
  ".org Lpre2_0_0_%=+4224\n" \
  "Lpre2_0_6_%=:\n" \
  "ds_read_b128 v[144:147], v161 offset:20480\n" \
  "ds_read_b128 v[148:151], v161 offset:21504\n" \
  "ds_read_b128 v[152:155], v161 offset:22528\n" \
  "ds_read_b128 v[156:159], v161 offset:23552\n" \
  "s_add_u32 m0, s69, 32768\n" \
  "s_nop 0\n" \
  "buffer_load_dwordx4 v163, s[36:39], s56 offen lds\n" \
  "s_add_u32 m0, s69, 36864\n" \
  "s_nop 0\n" \
  "buffer_load_dwordx4 v163, s[36:39], s57 offen lds\n" \
  "s_add_u32 m0, s69, 40960\n" \
  "s_nop 0\n" \
  "buffer_load_dwordx4 v163, s[36:39], s58 offen lds\n" \
  "s_add_u32 m0, s69, 45056\n" \
  "s_nop 0\n" \
  "buffer_load_dwordx4 v163, s[36:39], s59 offen lds\n" \
  "s_add_u32 m0, s69, 49152\n" \
  "s_nop 0\n" \
  "buffer_load_dwordx4 v163, s[36:39], s60 offen lds\n" \
  "buffer_load_dwordx4 v[36:39], v163, s[36:39], s75 offen offset:1024\n" \
  "buffer_load_dwordx4 v[32:35], v163, s[36:39], s75 offen offset:0\n" \
  "v_readlane_b32 s67, v175, s70\n" \
  "v_readlane_b32 s66, v178, s70\n" \
  "v_readlane_b32 s76, v169, s70\n" \
  "v_readlane_b32 s77, v172, s70\n" \
  "s_add_u32 s70, s70, 1\n" \
  "v_add_u32 v165, s76, v162\n" \
  "s_branch Ls2_6_%=\n" \
  ".org Lpre2_0_0_%=+4928\n" \
  "Lpre2_0_7_%=:\n" \
  "ds_read_b128 v[128:131], v161 offset:24576\n" \
  "ds_read_b128 v[132:135], v161 offset:25600\n" \
  "ds_read_b128 v[136:139], v161 offset:26624\n" \
  "ds_read_b128 v[140:143], v161 offset:27648\n" \
  "s_add_u32 m0, s69, 32768\n" \
  "s_nop 0\n" \
  "buffer_load_dwordx4 v163, s[36:39], s56 offen lds\n" \
  "s_add_u32 m0, s69, 36864\n" \
  "s_nop 0\n" \
  "buffer_load_dwordx4 v163, s[36:39], s57 offen lds\n" \
  "s_add_u32 m0, s69, 40960\n" \
  "s_nop 0\n" \
  "buffer_load_dwordx4 v163, s[36:39], s58 offen lds\n" \
  "s_add_u32 m0, s69, 45056\n" \
  "s_nop 0\n" \
  "buffer_load_dwordx4 v163, s[36:39], s59 offen lds\n" \
  "s_add_u32 m0, s69, 49152\n" \
  "s_nop 0\n" \
  "buffer_load_dwordx4 v163, s[36:39], s60 offen lds\n" \
  "buffer_load_dwordx4 v[36:39], v163, s[36:39], s75 offen offset:1024\n" \
  "buffer_load_dwordx4 v[32:35], v163, s[36:39], s75 offen offset:0\n" \
  "v_readlane_b32 s67, v175, s70\n" \
  "v_readlane_b32 s66, v178, s70\n" \
  "v_readlane_b32 s76, v169, s70\n" \
  "v_readlane_b32 s77, v172, s70\n" \
  "s_add_u32 s70, s70, 1\n" \
  "v_add_u32 v165, s76, v162\n" \
  "s_add_u32 m0, s69, 53248\n" \
  "s_nop 0\n" \
  "buffer_load_dwordx4 v163, s[36:39], s61 offen lds\n" \
  "buffer_load_dwordx4 v[44:47], v163, s[36:39], s75 offen offset:3072\n" \
  "s_bitcmp1_b32 s65, 23\n" \
  "s_cbranch_scc1 Lp212_%=\n" \
  "Lq212_%=:\n" \
  "v_add_u32 v166, 4096, v165\n" \
  "s_cmp_gt_u32 s67, s68\n" \
  "s_cbranch_scc1 Lp213_%=\n" \
  "Lq213_%=:\n" \
  "buffer_load_dwordx4 v[40:43], v163, s[36:39], s75 offen offset:2048\n" \
  "buffer_load_dwordx4 v[48:51], v164, s[36:39], s75 offen offset:0\n" \
  "v_mul_f32 v192, s78, v192\n" \
  "v_mul_f32 v193, s78, v193\n" \
  "v_mul_f32 v194, s78, v194\n" \
  "v_mul_f32 v195, s78, v195\n" \
  "s_bitcmp1_b32 s65, 28\n" \
  "s_cbranch_scc1 Lp214_%=\n" \
  "buffer_store_dwordx4 v[192:195], v167, s[40:43], s50 offen offset:0 nt\n" \
  "Lq214_%=:\n" \
  "s_branch Ls2_7_%=\n" \
  ".org Lpre2_0_0_%=+5632\n" \
  "Lpre2_0_8_%=:\n" \
  "ds_read_b128 v[144:147], v161 offset:28672\n" \
  "ds_read_b128 v[148:151], v161 offset:29696\n" \
  "ds_read_b128 v[152:155], v161 offset:30720\n" \
  "ds_read_b128 v[156:159], v161 offset:31744\n" \
  "s_add_u32 m0, s69, 32768\n" \
  "s_nop 0\n" \
  "buffer_load_dwordx4 v163, s[36:39], s56 offen lds\n" \
  "s_add_u32 m0, s69, 36864\n" \
  "s_nop 0\n" \
  "buffer_load_dwordx4 v163, s[36:39], s57 offen lds\n" \
  "s_add_u32 m0, s69, 40960\n" \
  "s_nop 0\n" \
  "buffer_load_dwordx4 v163, s[36:39], s58 offen lds\n" \
  "s_add_u32 m0, s69, 45056\n" \
  "s_nop 0\n" \
  "buffer_load_dwordx4 v163, s[36:39], s59 offen lds\n" \
  "s_add_u32 m0, s69, 49152\n" \
  "s_nop 0\n" \
  "buffer_load_dwordx4 v163, s[36:39], s60 offen lds\n" \
  "buffer_load_dwordx4 v[36:39], v163, s[36:39], s75 offen offset:1024\n" \
  "buffer_load_dwordx4 v[32:35], v163, s[36:39], s75 offen offset:0\n" \
  "v_readlane_b32 s67, v175, s70\n" \
  "v_readlane_b32 s66, v178, s70\n" \
  "v_readlane_b32 s76, v169, s70\n" \
  "v_readlane_b32 s77, v172, s70\n" \
  "s_add_u32 s70, s70, 1\n" \
  "v_add_u32 v165, s76, v162\n" \
  "s_add_u32 m0, s69, 53248\n" \
  "s_nop 0\n" \
  "buffer_load_dwordx4 v163, s[36:39], s61 offen lds\n" \
  "buffer_load_dwordx4 v[44:47], v163, s[36:39], s75 offen offset:3072\n" \
  "s_bitcmp1_b32 s65, 23\n" \
  "s_cbranch_scc1 Lp215_%=\n" \
  "Lq215_%=:\n" \
  "v_add_u32 v166, 4096, v165\n" \
  "s_cmp_gt_u32 s67, s68\n" \
  "s_cbranch_scc1 Lp216_%=\n" \
  "Lq216_%=:\n" \
  "buffer_load_dwordx4 v[40:43], v163, s[36:39], s75 offen offset:2048\n" \
  "buffer_load_dwordx4 v[48:51], v164, s[36:39], s75 offen offset:0\n" \
  "v_mul_f32 v192, s78, v192\n" \
  "v_mul_f32 v193, s78, v193\n" \
  "v_mul_f32 v194, s78, v194\n" \
  "v_mul_f32 v195, s78, v195\n" \
  "s_bitcmp1_b32 s65, 28\n" \
  "s_cbranch_scc1 Lp217_%=\n" \
  "buffer_store_dwordx4 v[192:195], v167, s[40:43], s50 offen offset:0 nt\n" \
  "Lq217_%=:\n" \
  "s_add_u32 m0, s69, 57344\n" \
  "s_nop 0\n" \
  "buffer_load_dwordx4 v163, s[36:39], s62 offen lds\n" \
  "buffer_load_dwordx4 v[56:59], v164, s[36:39], s75 offen offset:2048\n" \
  "v_mul_f32 v208, s78, v208\n" \
  "v_mul_f32 v209, s78, v209\n" \
  "v_mul_f32 v210, s78, v210\n" \
  "v_mul_f32 v211, s78, v211\n" \
  "s_bitcmp1_b32 s65, 28\n" \
  "s_cbranch_scc1 Lp218_%=\n" \
  "buffer_store_dwordx4 v[208:211], v167, s[40:43], s50 offen offset:64 nt\n" \
  "Lq218_%=:\n" \
  "s_and_b32 s73, s66, 0xf00001ff\n" \
  "buffer_load_dwordx4 v[52:55], v164, s[36:39], s75 offen offset:1024\n" \
  "buffer_load_dwordx4 v[60:63], v164, s[36:39], s75 offen offset:3072\n" \
  "s_bfe_u32 m0, s66, 0x40018\n" \
  "s_nop 0\n" \
  "s_movreld_b32 s56, s77\n" \
  "s_branch Ls2_8_%=\n" \
  ".org Lpre2_0_0_%=+6336\n" \
  "Lpre2_1_0_%=:\n" \
  "s_branch Ls2_1_%=\n" \
  ".org Lpre2_0_0_%=+7040\n" \
  "Lpre2_1_1_%=:\n" \
  "s_branch Ls2_1_%=\n" \
  ".org Lpre2_0_0_%=+7744\n" \
  "Lpre2_1_2_%=:\n" \
  "ds_read_b128 v[144:147], v161 offset:4096\n" \
  "ds_read_b128 v[148:151], v161 offset:5120\n" \
  "ds_read_b128 v[152:155], v161 offset:6144\n" \
  "ds_read_b128 v[156:159], v161 offset:7168\n" \
  "s_branch Ls2_2_%=\n" \
  ".org Lpre2_0_0_%=+8448\n" \
  "Lpre2_1_3_%=:\n" \
  "ds_read_b128 v[128:131], v161 offset:8192\n" \
  "ds_read_b128 v[132:135], v161 offset:9216\n" \
  "ds_read_b128 v[136:139], v161 offset:10240\n" \
  "ds_read_b128 v[140:143], v161 offset:11264\n" \
  "s_branch Ls2_3_%=\n" \
  ".org Lpre2_0_0_%=+9152\n" \
  "Lpre2_1_4_%=:\n" \
  "ds_read_b128 v[144:147], v161 offset:12288\n" \
  "ds_read_b128 v[148:151], v161 offset:13312\n" \
  "ds_read_b128 v[152:155], v161 offset:14336\n" \
  "ds_read_b128 v[156:159], v161 offset:15360\n" \
  "s_branch Ls2_4_%=\n" \
  ".org Lpre2_0_0_%=+9856\n" \
  "Lpre2_1_5_%=:\n" \
  "ds_read_b128 v[128:131], v161 offset:16384\n" \
  "ds_read_b128 v[132:135], v161 offset:17408\n" \
  "ds_read_b128 v[136:139], v161 offset:18432\n" \
  "ds_read_b128 v[140:143], v161 offset:19456\n" \
  "s_branch Ls2_5_%=\n" \
  ".org Lpre2_0_0_%=+10560\n" \
  "Lpre2_1_6_%=:\n" \
  "ds_read_b128 v[144:147], v161 offset:20480\n" \
  "ds_read_b128 v[148:151], v161 offset:21504\n" \
  "ds_read_b128 v[152:155], v161 offset:22528\n" \
  "ds_read_b128 v[156:159], v161 offset:23552\n" \
  "buffer_load_dwordx4 v[36:39], v163, s[36:39], s75 offen offset:1024\n" \
  "buffer_load_dwordx4 v[32:35], v163, s[36:39], s75 offen offset:0\n" \
  "v_readlane_b32 s67, v175, s70\n" \
  "v_readlane_b32 s66, v178, s70\n" \
  "v_readlane_b32 s76, v169, s70\n" \
  "v_readlane_b32 s77, v172, s70\n" \
  "s_add_u32 s70, s70, 1\n" \
  "v_add_u32 v165, s76, v162\n" \
  "s_branch Ls2_6_%=\n" \
  ".org Lpre2_0_0_%=+11264\n" \
  "Lpre2_1_7_%=:\n" \
  "ds_read_b128 v[128:131], v161 offset:24576\n" \
  "ds_read_b128 v[132:135], v161 offset:25600\n" \
  "ds_read_b128 v[136:139], v161 offset:26624\n" \
  "ds_read_b128 v[140:143], v161 offset:27648\n" \
  "buffer_load_dwordx4 v[36:39], v163, s[36:39], s75 offen offset:1024\n" \
  "buffer_load_dwordx4 v[32:35], v163, s[36:39], s75 offen offset:0\n" \
  "v_readlane_b32 s67, v175, s70\n" \
  "v_readlane_b32 s66, v178, s70\n" \
  "v_readlane_b32 s76, v169, s70\n" \
  "v_readlane_b32 s77, v172, s70\n" \
  "s_add_u32 s70, s70, 1\n" \
  "v_add_u32 v165, s76, v162\n" \
  "buffer_load_dwordx4 v[44:47], v163, s[36:39], s75 offen offset:3072\n" \
  "s_bitcmp1_b32 s65, 23\n" \
  "s_cbranch_scc1 Lp219_%=\n" \
  "Lq219_%=:\n" \
  "v_add_u32 v166, 4096, v165\n" \
  "s_cmp_gt_u32 s67, s68\n" \
  "s_cbranch_scc1 Lp220_%=\n" \
  "Lq220_%=:\n" \
  "buffer_load_dwordx4 v[40:43], v163, s[36:39], s75 offen offset:2048\n" \
  "buffer_load_dwordx4 v[48:51], v164, s[36:39], s75 offen offset:0\n" \
  "v_mul_f32 v192, s78, v192\n" \
  "v_mul_f32 v193, s78, v193\n" \
  "v_mul_f32 v194, s78, v194\n" \
  "v_mul_f32 v195, s78, v195\n" \
  "s_bitcmp1_b32 s65, 28\n" \
  "s_cbranch_scc1 Lp221_%=\n" \
  "buffer_store_dwordx4 v[192:195], v167, s[40:43], s50 offen offset:0 nt\n" \
  "Lq221_%=:\n" \
  "s_branch Ls2_7_%=\n" \
  ".org Lpre2_0_0_%=+11968\n" \
  "Lpre2_1_8_%=:\n" \
  "ds_read_b128 v[144:147], v161 offset:28672\n" \
  "ds_read_b128 v[148:151], v161 offset:29696\n" \
  "ds_read_b128 v[152:155], v161 offset:30720\n" \
  "ds_read_b128 v[156:159], v161 offset:31744\n" \
  "buffer_load_dwordx4 v[36:39], v163, s[36:39], s75 offen offset:1024\n" \
  "buffer_load_dwordx4 v[32:35], v163, s[36:39], s75 offen offset:0\n" \
  "v_readlane_b32 s67, v175, s70\n" \
  "v_readlane_b32 s66, v178, s70\n" \
  "v_readlane_b32 s76, v169, s70\n" \
  "v_readlane_b32 s77, v172, s70\n" \
  "s_add_u32 s70, s70, 1\n" \
  "v_add_u32 v165, s76, v162\n" \
  "buffer_load_dwordx4 v[44:47], v163, s[36:39], s75 offen offset:3072\n" \
  "s_bitcmp1_b32 s65, 23\n" \
  "s_cbranch_scc1 Lp222_%=\n" \
  "Lq222_%=:\n" \
  "v_add_u32 v166, 4096, v165\n" \
  "s_cmp_gt_u32 s67, s68\n" \
  "s_cbranch_scc1 Lp223_%=\n" \
  "Lq223_%=:\n" \
  "buffer_load_dwordx4 v[40:43], v163, s[36:39], s75 offen offset:2048\n" \
  "buffer_load_dwordx4 v[48:51], v164, s[36:39], s75 offen offset:0\n" \
  "v_mul_f32 v192, s78, v192\n" \
  "v_mul_f32 v193, s78, v193\n" \
  "v_mul_f32 v194, s78, v194\n" \
  "v_mul_f32 v195, s78, v195\n" \
  "s_bitcmp1_b32 s65, 28\n" \
  "s_cbranch_scc1 Lp224_%=\n" \
  "buffer_store_dwordx4 v[192:195], v167, s[40:43], s50 offen offset:0 nt\n" \
  "Lq224_%=:\n" \
  "buffer_load_dwordx4 v[56:59], v164, s[36:39], s75 offen offset:2048\n" \
  "v_mul_f32 v208, s78, v208\n" \
  "v_mul_f32 v209, s78, v209\n" \
  "v_mul_f32 v210, s78, v210\n" \
  "v_mul_f32 v211, s78, v211\n" \
  "s_bitcmp1_b32 s65, 28\n" \
  "s_cbranch_scc1 Lp225_%=\n" \
  "buffer_store_dwordx4 v[208:211], v167, s[40:43], s50 offen offset:64 nt\n" \
  "Lq225_%=:\n" \
  "s_and_b32 s73, s66, 0xf00001ff\n" \
  "buffer_load_dwordx4 v[52:55], v164, s[36:39], s75 offen offset:1024\n" \
  "buffer_load_dwordx4 v[60:63], v164, s[36:39], s75 offen offset:3072\n" \
  "s_bfe_u32 m0, s66, 0x40018\n" \
  "s_nop 0\n" \
  "s_movreld_b32 s56, s77\n" \
  "s_branch Ls2_8_%=\n" \
  ".p2align 6\n" \
  "Lz3_%=:\n" \
  "ds_read_b128 v[128:131], v161 offset:32768\n" \
  "ds_read_b128 v[132:135], v161 offset:33792\n" \
  "ds_read_b128 v[136:139], v161 offset:34816\n" \
  "ds_read_b128 v[140:143], v161 offset:35840\n" \
  "s_add_u32 m0, s69, 94208\n" \
  "s_nop 0\n" \
  "buffer_load_dwordx4 v165, s[36:39], s63 offen lds\n" \
  "s_branch Lzr3_%=\n" \
  "Lpj3_%=:\n" \
  "s_setpc_b64 s[98:99]\n" \
  "Lsuf3_0_1_%=:\n" \
  "s_bitcmp1_b32 s66, 10\n" \
  "s_cbranch_scc1 Lsuf3_1_1_%=\n" \
  "ds_read_b128 v[144:147], v161 offset:61440\n" \
  "ds_read_b128 v[148:151], v161 offset:62464\n" \
  "ds_read_b128 v[152:155], v161 offset:63488\n" \
  "ds_read_b128 v[156:159], v161 offset:64512\n" \
  "s_add_u32 m0, s69, 69632\n" \
  "s_nop 0\n" \
  "buffer_load_dwordx4 v165, s[36:39], s57 offen lds\n" \
  "s_add_u32 m0, s69, 73728\n" \
  "s_nop 0\n" \
  "buffer_load_dwordx4 v165, s[36:39], s58 offen lds\n" \
  "s_add_u32 m0, s69, 77824\n" \
  "s_nop 0\n" \
  "buffer_load_dwordx4 v165, s[36:39], s59 offen lds\n" \
  "s_add_u32 m0, s69, 81920\n" \
  "s_nop 0\n" \
  "buffer_load_dwordx4 v165, s[36:39], s60 offen lds\n" \
  "buffer_load_dwordx4 v[68:71], v165, s[36:39], s75 offen offset:1024\n" \
  "buffer_load_dwordx4 v[64:67], v165, s[36:39], s75 offen offset:0\n" \
  "v_readlane_b32 s67, v175, s70\n" \
  "v_readlane_b32 s65, v178, s70\n" \
  "v_readlane_b32 s76, v169, s70\n" \
  "v_readlane_b32 s77, v172, s70\n" \
  "v_readlane_b32 s71, v181, s70\n" \
  "s_add_u32 s70, s70, 1\n" \
  "v_add_u32 v163, s76, v162\n" \
  "s_add_u32 m0, s69, 86016\n" \
  "s_nop 0\n" \
  "buffer_load_dwordx4 v165, s[36:39], s61 offen lds\n" \
  "buffer_load_dwordx4 v[76:79], v165, s[36:39], s75 offen offset:3072\n" \
  "v_add_u32 v164, 4096, v163\n" \
  "s_cmp_gt_u32 s67, s68\n" \
  "s_cbranch_scc1 Lp226_%=\n" \
  "Lq226_%=:\n" \
  "buffer_load_dwordx4 v[72:75], v165, s[36:39], s75 offen offset:2048\n" \
  "buffer_load_dwordx4 v[80:83], v166, s[36:39], s75 offen offset:0\n" \
  "v_mul_f32 v196, s78, v196\n" \
  "v_mul_f32 v197, s78, v197\n" \
  "v_mul_f32 v198, s78, v198\n" \
  "v_mul_f32 v199, s78, v199\n" \
  "s_bitcmp1_b32 s66, 28\n" \
  "s_cbranch_scc1 Lp227_%=\n" \
  "buffer_store_dwordx4 v[196:199], v167, s[40:43], s51 offen offset:0 nt\n" \
  "Lq227_%=:\n" \
  "s_bfe_u32 s96, s65, 0x40014\n" \
  "s_mul_i32 s96, s96, Le0_1_%=-Le0_0_%=\n" \
  "s_add_u32 s96, s96, Le0_0_%=-Lpc_%=\n" \
  "s_add_u32 s96, s96, s94\n" \
  "s_addc_u32 s97, s95, 0\n" \
  "s_add_u32 m0, s69, 90112\n" \
  "s_nop 0\n" \
  "buffer_load_dwordx4 v165, s[36:39], s62 offen lds\n" \
  "buffer_load_dwordx4 v[88:91], v166, s[36:39], s75 offen offset:2048\n" \
  "v_mul_f32 v212, s78, v212\n" \
  "v_mul_f32 v213, s78, v213\n" \
  "v_mul_f32 v214, s78, v214\n" \
  "v_mul_f32 v215, s78, v215\n" \
  "s_bitcmp1_b32 s66, 28\n" \
  "s_cbranch_scc1 Lp228_%=\n" \
  "buffer_store_dwordx4 v[212:215], v167, s[40:43], s51 offen offset:64 nt\n" \
  "Lq228_%=:\n" \
  "s_and_b32 s73, s65, 0xf00001ff\n" \
  "buffer_load_dwordx4 v[84:87], v166, s[36:39], s75 offen offset:1024\n" \
  "buffer_load_dwordx4 v[92:95], v166, s[36:39], s75 offen offset:3072\n" \
  "s_bfe_u32 m0, s65, 0x40018\n" \
  "s_nop 0\n" \
  "s_movreld_b32 s56, s77\n" \
  "s_branch Ls3_8_%=\n" \
  "Lsuf3_0_2_%=:\n" \
  "s_bitcmp1_b32 s66, 10\n" \
  "s_cbranch_scc1 Lsuf3_1_2_%=\n" \
  "ds_read_b128 v[144:147], v161 offset:61440\n" \
  "ds_read_b128 v[148:151], v161 offset:62464\n" \
  "ds_read_b128 v[152:155], v161 offset:63488\n" \
  "ds_read_b128 v[156:159], v161 offset:64512\n" \
  "s_add_u32 m0, s69, 73728\n" \
  "s_nop 0\n" \
  "buffer_load_dwordx4 v165, s[36:39], s58 offen lds\n" \
  "s_add_u32 m0, s69, 77824\n" \
  "s_nop 0\n" \
  "buffer_load_dwordx4 v165, s[36:39], s59 offen lds\n" \
  "s_add_u32 m0, s69, 81920\n" \
  "s_nop 0\n" \
  "buffer_load_dwordx4 v165, s[36:39], s60 offen lds\n" \
  "buffer_load_dwordx4 v[68:71], v165, s[36:39], s75 offen offset:1024\n" \
  "buffer_load_dwordx4 v[64:67], v165, s[36:39], s75 offen offset:0\n" \
  "v_readlane_b32 s67, v175, s70\n" \
  "v_readlane_b32 s65, v178, s70\n" \
  "v_readlane_b32 s76, v169, s70\n" \
  "v_readlane_b32 s77, v172, s70\n" \
  "v_readlane_b32 s71, v181, s70\n" \
  "s_add_u32 s70, s70, 1\n" \
  "v_add_u32 v163, s76, v162\n" \
  "s_add_u32 m0, s69, 86016\n" \
  "s_nop 0\n" \
  "buffer_load_dwordx4 v165, s[36:39], s61 offen lds\n" \
  "buffer_load_dwordx4 v[76:79], v165, s[36:39], s75 offen offset:3072\n" \
  "v_add_u32 v164, 4096, v163\n" \
  "s_cmp_gt_u32 s67, s68\n" \
  "s_cbranch_scc1 Lp229_%=\n" \
  "Lq229_%=:\n" \
  "buffer_load_dwordx4 v[72:75], v165, s[36:39], s75 offen offset:2048\n" \
  "buffer_load_dwordx4 v[80:83], v166, s[36:39], s75 offen offset:0\n" \
  "v_mul_f32 v196, s78, v196\n" \
  "v_mul_f32 v197, s78, v197\n" \
  "v_mul_f32 v198, s78, v198\n" \
  "v_mul_f32 v199, s78, v199\n" \
  "s_bitcmp1_b32 s66, 28\n" \
  "s_cbranch_scc1 Lp230_%=\n" \
  "buffer_store_dwordx4 v[196:199], v167, s[40:43], s51 offen offset:0 nt\n" \
  "Lq230_%=:\n" \
  "s_bfe_u32 s96, s65, 0x40014\n" \
  "s_mul_i32 s96, s96, Le0_1_%=-Le0_0_%=\n" \
  "s_add_u32 s96, s96, Le0_0_%=-Lpc_%=\n" \
  "s_add_u32 s96, s96, s94\n" \
  "s_addc_u32 s97, s95, 0\n" \
  "s_add_u32 m0, s69, 90112\n" \
  "s_nop 0\n" \
  "buffer_load_dwordx4 v165, s[36:39], s62 offen lds\n" \
  "buffer_load_dwordx4 v[88:91], v166, s[36:39], s75 offen offset:2048\n" \
  "v_mul_f32 v212, s78, v212\n" \
  "v_mul_f32 v213, s78, v213\n" \
  "v_mul_f32 v214, s78, v214\n" \
  "v_mul_f32 v215, s78, v215\n" \
  "s_bitcmp1_b32 s66, 28\n" \
  "s_cbranch_scc1 Lp231_%=\n" \
  "buffer_store_dwordx4 v[212:215], v167, s[40:43], s51 offen offset:64 nt\n" \
  "Lq231_%=:\n" \
  "s_and_b32 s73, s65, 0xf00001ff\n" \
  "buffer_load_dwordx4 v[84:87], v166, s[36:39], s75 offen offset:1024\n" \
  "buffer_load_dwordx4 v[92:95], v166, s[36:39], s75 offen offset:3072\n" \
  "s_bfe_u32 m0, s65, 0x40018\n" \
  "s_nop 0\n" \
  "s_movreld_b32 s56, s77\n" \
  "s_branch Ls3_8_%=\n" \
  "Lsuf3_0_3_%=:\n" \
  "s_bitcmp1_b32 s66, 10\n" \
  "s_cbranch_scc1 Lsuf3_1_3_%=\n" \
  "ds_read_b128 v[144:147], v161 offset:61440\n" \
  "ds_read_b128 v[148:151], v161 offset:62464\n" \
  "ds_read_b128 v[152:155], v161 offset:63488\n" \
  "ds_read_b128 v[156:159], v161 offset:64512\n" \
  "s_add_u32 m0, s69, 77824\n" \
  "s_nop 0\n" \
  "buffer_load_dwordx4 v165, s[36:39], s59 offen lds\n" \
  "s_add_u32 m0, s69, 81920\n" \
  "s_nop 0\n" \
  "buffer_load_dwordx4 v165, s[36:39], s60 offen lds\n" \
  "buffer_load_dwordx4 v[68:71], v165, s[36:39], s75 offen offset:1024\n" \
  "buffer_load_dwordx4 v[64:67], v165, s[36:39], s75 offen offset:0\n" \
  "v_readlane_b32 s67, v175, s70\n" \
  "v_readlane_b32 s65, v178, s70\n" \
  "v_readlane_b32 s76, v169, s70\n" \
  "v_readlane_b32 s77, v172, s70\n" \
  "v_readlane_b32 s71, v181, s70\n" \
  "s_add_u32 s70, s70, 1\n" \
  "v_add_u32 v163, s76, v162\n" \
  "s_add_u32 m0, s69, 86016\n" \
  "s_nop 0\n" \
  "buffer_load_dwordx4 v165, s[36:39], s61 offen lds\n" \
  "buffer_load_dwordx4 v[76:79], v165, s[36:39], s75 offen offset:3072\n" \
  "v_add_u32 v164, 4096, v163\n" \
  "s_cmp_gt_u32 s67, s68\n" \
  "s_cbranch_scc1 Lp232_%=\n" \
  "Lq232_%=:\n" \
  "buffer_load_dwordx4 v[72:75], v165, s[36:39], s75 offen offset:2048\n" \
  "buffer_load_dwordx4 v[80:83], v166, s[36:39], s75 offen offset:0\n" \
  "v_mul_f32 v196, s78, v196\n" \
  "v_mul_f32 v197, s78, v197\n" \
  "v_mul_f32 v198, s78, v198\n" \
  "v_mul_f32 v199, s78, v199\n" \
  "s_bitcmp1_b32 s66, 28\n" \
  "s_cbranch_scc1 Lp233_%=\n" \
  "buffer_store_dwordx4 v[196:199], v167, s[40:43], s51 offen offset:0 nt\n" \
  "Lq233_%=:\n" \
  "s_bfe_u32 s96, s65, 0x40014\n" \
  "s_mul_i32 s96, s96, Le0_1_%=-Le0_0_%=\n" \
  "s_add_u32 s96, s96, Le0_0_%=-Lpc_%=\n" \
  "s_add_u32 s96, s96, s94\n" \
  "s_addc_u32 s97, s95, 0\n" \
  "s_add_u32 m0, s69, 90112\n" \
  "s_nop 0\n" \
  "buffer_load_dwordx4 v165, s[36:39], s62 offen lds\n" \
  "buffer_load_dwordx4 v[88:91], v166, s[36:39], s75 offen offset:2048\n" \
  "v_mul_f32 v212, s78, v212\n" \
  "v_mul_f32 v213, s78, v213\n" \
  "v_mul_f32 v214, s78, v214\n" \
  "v_mul_f32 v215, s78, v215\n" \
  "s_bitcmp1_b32 s66, 28\n" \
  "s_cbranch_scc1 Lp234_%=\n" \
  "buffer_store_dwordx4 v[212:215], v167, s[40:43], s51 offen offset:64 nt\n" \
  "Lq234_%=:\n" \
  "s_and_b32 s73, s65, 0xf00001ff\n" \
  "buffer_load_dwordx4 v[84:87], v166, s[36:39], s75 offen offset:1024\n" \
  "buffer_load_dwordx4 v[92:95], v166, s[36:39], s75 offen offset:3072\n" \
  "s_bfe_u32 m0, s65, 0x40018\n" \
  "s_nop 0\n" \
  "s_movreld_b32 s56, s77\n" \
  "s_branch Ls3_8_%=\n" \
  "Lsuf3_0_4_%=:\n" \
  "s_bitcmp1_b32 s66, 10\n" \
  "s_cbranch_scc1 Lsuf3_1_4_%=\n" \
  "ds_read_b128 v[144:147], v161 offset:61440\n" \
  "ds_read_b128 v[148:151], v161 offset:62464\n" \
  "ds_read_b128 v[152:155], v161 offset:63488\n" \
  "ds_read_b128 v[156:159], v161 offset:64512\n" \
  "s_add_u32 m0, s69, 81920\n" \
  "s_nop 0\n" \
  "buffer_load_dwordx4 v165, s[36:39], s60 offen lds\n" \
  "buffer_load_dwordx4 v[68:71], v165, s[36:39], s75 offen offset:1024\n" \
  "buffer_load_dwordx4 v[64:67], v165, s[36:39], s75 offen offset:0\n" \
  "v_readlane_b32 s67, v175, s70\n" \
  "v_readlane_b32 s65, v178, s70\n" \
  "v_readlane_b32 s76, v169, s70\n" \
  "v_readlane_b32 s77, v172, s70\n" \
  "v_readlane_b32 s71, v181, s70\n" \
  "s_add_u32 s70, s70, 1\n" \
  "v_add_u32 v163, s76, v162\n" \
  "s_add_u32 m0, s69, 86016\n" \
  "s_nop 0\n" \
  "buffer_load_dwordx4 v165, s[36:39], s61 offen lds\n" \
  "buffer_load_dwordx4 v[76:79], v165, s[36:39], s75 offen offset:3072\n" \
  "v_add_u32 v164, 4096, v163\n" \
  "s_cmp_gt_u32 s67, s68\n" \
  "s_cbranch_scc1 Lp235_%=\n" \
  "Lq235_%=:\n" \
  "buffer_load_dwordx4 v[72:75], v165, s[36:39], s75 offen offset:2048\n" \
  "buffer_load_dwordx4 v[80:83], v166, s[36:39], s75 offen offset:0\n" \
  "v_mul_f32 v196, s78, v196\n" \
  "v_mul_f32 v197, s78, v197\n" \
  "v_mul_f32 v198, s78, v198\n" \
  "v_mul_f32 v199, s78, v199\n" \
  "s_bitcmp1_b32 s66, 28\n" \
  "s_cbranch_scc1 Lp236_%=\n" \
  "buffer_store_dwordx4 v[196:199], v167, s[40:43], s51 offen offset:0 nt\n" \
  "Lq236_%=:\n" \
  "s_bfe_u32 s96, s65, 0x40014\n" \
  "s_mul_i32 s96, s96, Le0_1_%=-Le0_0_%=\n" \
  "s_add_u32 s96, s96, Le0_0_%=-Lpc_%=\n" \
  "s_add_u32 s96, s96, s94\n" \
  "s_addc_u32 s97, s95, 0\n" \
  "s_add_u32 m0, s69, 90112\n" \
  "s_nop 0\n" \
  "buffer_load_dwordx4 v165, s[36:39], s62 offen lds\n" \
  "buffer_load_dwordx4 v[88:91], v166, s[36:39], s75 offen offset:2048\n" \
  "v_mul_f32 v212, s78, v212\n" \
  "v_mul_f32 v213, s78, v213\n" \
  "v_mul_f32 v214, s78, v214\n" \
  "v_mul_f32 v215, s78, v215\n" \
  "s_bitcmp1_b32 s66, 28\n" \
  "s_cbranch_scc1 Lp237_%=\n" \
  "buffer_store_dwordx4 v[212:215], v167, s[40:43], s51 offen offset:64 nt\n" \
  "Lq237_%=:\n" \
  "s_and_b32 s73, s65, 0xf00001ff\n" \
  "buffer_load_dwordx4 v[84:87], v166, s[36:39], s75 offen offset:1024\n" \
  "buffer_load_dwordx4 v[92:95], v166, s[36:39], s75 offen offset:3072\n" \
  "s_bfe_u32 m0, s65, 0x40018\n" \
  "s_nop 0\n" \
  "s_movreld_b32 s56, s77\n" \
  "s_branch Ls3_8_%=\n" \
  "Lsuf3_0_5_%=:\n" \
  "s_bitcmp1_b32 s66, 10\n" \
  "s_cbranch_scc1 Lsuf3_1_5_%=\n" \
  "ds_read_b128 v[144:147], v161 offset:61440\n" \
  "ds_read_b128 v[148:151], v161 offset:62464\n" \
  "ds_read_b128 v[152:155], v161 offset:63488\n" \
  "ds_read_b128 v[156:159], v161 offset:64512\n" \
  "s_add_u32 m0, s69, 86016\n" \
  "s_nop 0\n" \
  "buffer_load_dwordx4 v165, s[36:39], s61 offen lds\n" \
  "buffer_load_dwordx4 v[76:79], v165, s[36:39], s75 offen offset:3072\n" \
  "v_add_u32 v164, 4096, v163\n" \
  "s_cmp_gt_u32 s67, s68\n" \
  "s_cbranch_scc1 Lp238_%=\n" \
  "Lq238_%=:\n" \
  "buffer_load_dwordx4 v[72:75], v165, s[36:39], s75 offen offset:2048\n" \
  "buffer_load_dwordx4 v[80:83], v166, s[36:39], s75 offen offset:0\n" \
  "v_mul_f32 v196, s78, v196\n" \
  "v_mul_f32 v197, s78, v197\n" \
  "v_mul_f32 v198, s78, v198\n" \
  "v_mul_f32 v199, s78, v199\n" \
  "s_bitcmp1_b32 s66, 28\n" \
  "s_cbranch_scc1 Lp239_%=\n" \
  "buffer_store_dwordx4 v[196:199], v167, s[40:43], s51 offen offset:0 nt\n" \
  "Lq239_%=:\n" \
  "s_bfe_u32 s96, s65, 0x40014\n" \
  "s_mul_i32 s96, s96, Le0_1_%=-Le0_0_%=\n" \
  "s_add_u32 s96, s96, Le0_0_%=-Lpc_%=\n" \
  "s_add_u32 s96, s96, s94\n" \
  "s_addc_u32 s97, s95, 0\n" \
  "s_add_u32 m0, s69, 90112\n" \
  "s_nop 0\n" \
  "buffer_load_dwordx4 v165, s[36:39], s62 offen lds\n" \
  "buffer_load_dwordx4 v[88:91], v166, s[36:39], s75 offen offset:2048\n" \
  "v_mul_f32 v212, s78, v212\n" \
  "v_mul_f32 v213, s78, v213\n" \
  "v_mul_f32 v214, s78, v214\n" \
  "v_mul_f32 v215, s78, v215\n" \
  "s_bitcmp1_b32 s66, 28\n" \
  "s_cbranch_scc1 Lp240_%=\n" \
  "buffer_store_dwordx4 v[212:215], v167, s[40:43], s51 offen offset:64 nt\n" \
  "Lq240_%=:\n" \
  "s_and_b32 s73, s65, 0xf00001ff\n" \
  "buffer_load_dwordx4 v[84:87], v166, s[36:39], s75 offen offset:1024\n" \
  "buffer_load_dwordx4 v[92:95], v166, s[36:39], s75 offen offset:3072\n" \
  "s_bfe_u32 m0, s65, 0x40018\n" \
  "s_nop 0\n" \
  "s_movreld_b32 s56, s77\n" \
  "s_branch Ls3_8_%=\n" \
  "Lsuf3_0_6_%=:\n" \
  "s_bitcmp1_b32 s66, 10\n" \
  "s_cbranch_scc1 Lsuf3_1_6_%=\n" \
  "ds_read_b128 v[144:147], v161 offset:61440\n" \
  "ds_read_b128 v[148:151], v161 offset:62464\n" \
  "ds_read_b128 v[152:155], v161 offset:63488\n" \
  "ds_read_b128 v[156:159], v161 offset:64512\n" \
  "s_add_u32 m0, s69, 90112\n" \
  "s_nop 0\n" \
  "buffer_load_dwordx4 v165, s[36:39], s62 offen lds\n" \
  "buffer_load_dwordx4 v[88:91], v166, s[36:39], s75 offen offset:2048\n" \
  "v_mul_f32 v212, s78, v212\n" \
  "v_mul_f32 v213, s78, v213\n" \
  "v_mul_f32 v214, s78, v214\n" \
  "v_mul_f32 v215, s78, v215\n" \
  "s_bitcmp1_b32 s66, 28\n" \
  "s_cbranch_scc1 Lp241_%=\n" \
  "buffer_store_dwordx4 v[212:215], v167, s[40:43], s51 offen offset:64 nt\n" \
  "Lq241_%=:\n" \
  "s_and_b32 s73, s65, 0xf00001ff\n" \
  "buffer_load_dwordx4 v[84:87], v166, s[36:39], s75 offen offset:1024\n" \
  "buffer_load_dwordx4 v[92:95], v166, s[36:39], s75 offen offset:3072\n" \
  "s_bfe_u32 m0, s65, 0x40018\n" \
  "s_nop 0\n" \
  "s_movreld_b32 s56, s77\n" \
  "s_branch Ls3_8_%=\n" \
  "Lsuf3_1_1_%=:\n" \
  "buffer_load_dwordx4 v[68:71], v165, s[36:39], s75 offen offset:1024\n" \
  "buffer_load_dwordx4 v[64:67], v165, s[36:39], s75 offen offset:0\n" \
  "v_readlane_b32 s67, v175, s70\n" \
  "v_readlane_b32 s65, v178, s70\n" \
  "v_readlane_b32 s76, v169, s70\n" \
  "v_readlane_b32 s77, v172, s70\n" \
  "v_readlane_b32 s71, v181, s70\n" \
  "s_add_u32 s70, s70, 1\n" \
  "v_add_u32 v163, s76, v162\n" \
  "buffer_load_dwordx4 v[76:79], v165, s[36:39], s75 offen offset:3072\n" \
  "v_add_u32 v164, 4096, v163\n" \
  "s_cmp_gt_u32 s67, s68\n" \
  "s_cbranch_scc1 Lp242_%=\n" \
  "Lq242_%=:\n" \
  "buffer_load_dwordx4 v[72:75], v165, s[36:39], s75 offen offset:2048\n" \
  "buffer_load_dwordx4 v[80:83], v166, s[36:39], s75 offen offset:0\n" \
  "v_mul_f32 v196, s78, v196\n" \
  "v_mul_f32 v197, s78, v197\n" \
  "v_mul_f32 v198, s78, v198\n" \
  "v_mul_f32 v199, s78, v199\n" \
  "s_bitcmp1_b32 s66, 28\n" \
  "s_cbranch_scc1 Lp243_%=\n" \
  "buffer_store_dwordx4 v[196:199], v167, s[40:43], s51 offen offset:0 nt\n" \
  "Lq243_%=:\n" \
  "s_bfe_u32 s96, s65, 0x40014\n" \
  "s_mul_i32 s96, s96, Le0_1_%=-Le0_0_%=\n" \
  "s_add_u32 s96, s96, Le0_0_%=-Lpc_%=\n" \
  "s_add_u32 s96, s96, s94\n" \
  "s_addc_u32 s97, s95, 0\n" \
  "buffer_load_dwordx4 v[88:91], v166, s[36:39], s75 offen offset:2048\n" \
  "v_mul_f32 v212, s78, v212\n" \
  "v_mul_f32 v213, s78, v213\n" \
  "v_mul_f32 v214, s78, v214\n" \
  "v_mul_f32 v215, s78, v215\n" \
  "s_bitcmp1_b32 s66, 28\n" \
  "s_cbranch_scc1 Lp244_%=\n" \
  "buffer_store_dwordx4 v[212:215], v167, s[40:43], s51 offen offset:64 nt\n" \
  "Lq244_%=:\n" \
  "s_and_b32 s73, s65, 0xf00001ff\n" \
  "buffer_load_dwordx4 v[84:87], v166, s[36:39], s75 offen offset:1024\n" \
  "buffer_load_dwordx4 v[92:95], v166, s[36:39], s75 offen offset:3072\n" \
  "s_bfe_u32 m0, s65, 0x40018\n" \
  "s_nop 0\n" \
  "s_movreld_b32 s56, s77\n" \
  "s_branch Ls3_8_%=\n" \
  "Lsuf3_1_2_%=:\n" \
  "buffer_load_dwordx4 v[68:71], v165, s[36:39], s75 offen offset:1024\n" \
  "buffer_load_dwordx4 v[64:67], v165, s[36:39], s75 offen offset:0\n" \
  "v_readlane_b32 s67, v175, s70\n" \
  "v_readlane_b32 s65, v178, s70\n" \
  "v_readlane_b32 s76, v169, s70\n" \
  "v_readlane_b32 s77, v172, s70\n" \
  "v_readlane_b32 s71, v181, s70\n" \
  "s_add_u32 s70, s70, 1\n" \
  "v_add_u32 v163, s76, v162\n" \
  "buffer_load_dwordx4 v[76:79], v165, s[36:39], s75 offen offset:3072\n" \
  "v_add_u32 v164, 4096, v163\n" \
  "s_cmp_gt_u32 s67, s68\n" \
  "s_cbranch_scc1 Lp245_%=\n" \
  "Lq245_%=:\n" \
  "buffer_load_dwordx4 v[72:75], v165, s[36:39], s75 offen offset:2048\n" \
  "buffer_load_dwordx4 v[80:83], v166, s[36:39], s75 offen offset:0\n" \
  "v_mul_f32 v196, s78, v196\n" \
  "v_mul_f32 v197, s78, v197\n" \
  "v_mul_f32 v198, s78, v198\n" \
  "v_mul_f32 v199, s78, v199\n" \
  "s_bitcmp1_b32 s66, 28\n" \
  "s_cbranch_scc1 Lp246_%=\n" \
  "buffer_store_dwordx4 v[196:199], v167, s[40:43], s51 offen offset:0 nt\n" \
  "Lq246_%=:\n" \
  "s_bfe_u32 s96, s65, 0x40014\n" \
  "s_mul_i32 s96, s96, Le0_1_%=-Le0_0_%=\n" \
  "s_add_u32 s96, s96, Le0_0_%=-Lpc_%=\n" \
  "s_add_u32 s96, s96, s94\n" \
  "s_addc_u32 s97, s95, 0\n" \
  "buffer_load_dwordx4 v[88:91], v166, s[36:39], s75 offen offset:2048\n" \
  "v_mul_f32 v212, s78, v212\n" \
  "v_mul_f32 v213, s78, v213\n" \
  "v_mul_f32 v214, s78, v214\n" \
  "v_mul_f32 v215, s78, v215\n" \
  "s_bitcmp1_b32 s66, 28\n" \
  "s_cbranch_scc1 Lp247_%=\n" \
  "buffer_store_dwordx4 v[212:215], v167, s[40:43], s51 offen offset:64 nt\n" \
  "Lq247_%=:\n" \
  "s_and_b32 s73, s65, 0xf00001ff\n" \
  "buffer_load_dwordx4 v[84:87], v166, s[36:39], s75 offen offset:1024\n" \
  "buffer_load_dwordx4 v[92:95], v166, s[36:39], s75 offen offset:3072\n" \
  "s_bfe_u32 m0, s65, 0x40018\n" \
  "s_nop 0\n" \
  "s_movreld_b32 s56, s77\n" \
  "s_branch Ls3_8_%=\n" \
  "Lsuf3_1_3_%=:\n" \
  "buffer_load_dwordx4 v[68:71], v165, s[36:39], s75 offen offset:1024\n" \
  "buffer_load_dwordx4 v[64:67], v165, s[36:39], s75 offen offset:0\n" \
  "v_readlane_b32 s67, v175, s70\n" \
  "v_readlane_b32 s65, v178, s70\n" \
  "v_readlane_b32 s76, v169, s70\n" \
  "v_readlane_b32 s77, v172, s70\n" \
  "v_readlane_b32 s71, v181, s70\n" \
  "s_add_u32 s70, s70, 1\n" \
  "v_add_u32 v163, s76, v162\n" \
  "buffer_load_dwordx4 v[76:79], v165, s[36:39], s75 offen offset:3072\n" \
  "v_add_u32 v164, 4096, v163\n" \
  "s_cmp_gt_u32 s67, s68\n" \
  "s_cbranch_scc1 Lp248_%=\n" \
  "Lq248_%=:\n" \
  "buffer_load_dwordx4 v[72:75], v165, s[36:39], s75 offen offset:2048\n" \
  "buffer_load_dwordx4 v[80:83], v166, s[36:39], s75 offen offset:0\n" \
  "v_mul_f32 v196, s78, v196\n" \
  "v_mul_f32 v197, s78, v197\n" \
  "v_mul_f32 v198, s78, v198\n" \
  "v_mul_f32 v199, s78, v199\n" \
  "s_bitcmp1_b32 s66, 28\n" \
  "s_cbranch_scc1 Lp249_%=\n" \
  "buffer_store_dwordx4 v[196:199], v167, s[40:43], s51 offen offset:0 nt\n" \
  "Lq249_%=:\n" \
  "s_bfe_u32 s96, s65, 0x40014\n" \
  "s_mul_i32 s96, s96, Le0_1_%=-Le0_0_%=\n" \
  "s_add_u32 s96, s96, Le0_0_%=-Lpc_%=\n" \
  "s_add_u32 s96, s96, s94\n" \
  "s_addc_u32 s97, s95, 0\n" \
  "buffer_load_dwordx4 v[88:91], v166, s[36:39], s75 offen offset:2048\n" \
  "v_mul_f32 v212, s78, v212\n" \
  "v_mul_f32 v213, s78, v213\n" \
  "v_mul_f32 v214, s78, v214\n" \
  "v_mul_f32 v215, s78, v215\n" \
  "s_bitcmp1_b32 s66, 28\n" \
  "s_cbranch_scc1 Lp250_%=\n" \
  "buffer_store_dwordx4 v[212:215], v167, s[40:43], s51 offen offset:64 nt\n" \
  "Lq250_%=:\n" \
  "s_and_b32 s73, s65, 0xf00001ff\n" \
  "buffer_load_dwordx4 v[84:87], v166, s[36:39], s75 offen offset:1024\n" \
  "buffer_load_dwordx4 v[92:95], v166, s[36:39], s75 offen offset:3072\n" \
  "s_bfe_u32 m0, s65, 0x40018\n" \
  "s_nop 0\n" \
  "s_movreld_b32 s56, s77\n" \
  "s_branch Ls3_8_%=\n" \
  "Lsuf3_1_4_%=:\n" \
  "buffer_load_dwordx4 v[68:71], v165, s[36:39], s75 offen offset:1024\n" \
  "buffer_load_dwordx4 v[64:67], v165, s[36:39], s75 offen offset:0\n" \
  "v_readlane_b32 s67, v175, s70\n" \
  "v_readlane_b32 s65, v178, s70\n" \
  "v_readlane_b32 s76, v169, s70\n" \
  "v_readlane_b32 s77, v172, s70\n" \
  "v_readlane_b32 s71, v181, s70\n" \
  "s_add_u32 s70, s70, 1\n" \
  "v_add_u32 v163, s76, v162\n" \
  "buffer_load_dwordx4 v[76:79], v165, s[36:39], s75 offen offset:3072\n" \
  "v_add_u32 v164, 4096, v163\n" \
  "s_cmp_gt_u32 s67, s68\n" \
  "s_cbranch_scc1 Lp251_%=\n" \
  "Lq251_%=:\n" \
  "buffer_load_dwordx4 v[72:75], v165, s[36:39], s75 offen offset:2048\n" \
  "buffer_load_dwordx4 v[80:83], v166, s[36:39], s75 offen offset:0\n" \
  "v_mul_f32 v196, s78, v196\n" \
  "v_mul_f32 v197, s78, v197\n" \
  "v_mul_f32 v198, s78, v198\n" \
  "v_mul_f32 v199, s78, v199\n" \
  "s_bitcmp1_b32 s66, 28\n" \
  "s_cbranch_scc1 Lp252_%=\n" \
  "buffer_store_dwordx4 v[196:199], v167, s[40:43], s51 offen offset:0 nt\n" \
  "Lq252_%=:\n" \
  "s_bfe_u32 s96, s65, 0x40014\n" \
  "s_mul_i32 s96, s96, Le0_1_%=-Le0_0_%=\n" \
  "s_add_u32 s96, s96, Le0_0_%=-Lpc_%=\n" \
  "s_add_u32 s96, s96, s94\n" \
  "s_addc_u32 s97, s95, 0\n" \
  "buffer_load_dwordx4 v[88:91], v166, s[36:39], s75 offen offset:2048\n" \
  "v_mul_f32 v212, s78, v212\n" \
  "v_mul_f32 v213, s78, v213\n" \
  "v_mul_f32 v214, s78, v214\n" \
  "v_mul_f32 v215, s78, v215\n" \
  "s_bitcmp1_b32 s66, 28\n" \
  "s_cbranch_scc1 Lp253_%=\n" \
  "buffer_store_dwordx4 v[212:215], v167, s[40:43], s51 offen offset:64 nt\n" \
  "Lq253_%=:\n" \
  "s_and_b32 s73, s65, 0xf00001ff\n" \
  "buffer_load_dwordx4 v[84:87], v166, s[36:39], s75 offen offset:1024\n" \
  "buffer_load_dwordx4 v[92:95], v166, s[36:39], s75 offen offset:3072\n" \
  "s_bfe_u32 m0, s65, 0x40018\n" \
  "s_nop 0\n" \
  "s_movreld_b32 s56, s77\n" \
  "s_branch Ls3_8_%=\n" \
  "Lsuf3_1_5_%=:\n" \
  "buffer_load_dwordx4 v[76:79], v165, s[36:39], s75 offen offset:3072\n" \
  "v_add_u32 v164, 4096, v163\n" \
  "s_cmp_gt_u32 s67, s68\n" \
  "s_cbranch_scc1 Lp254_%=\n" \
  "Lq254_%=:\n" \
  "buffer_load_dwordx4 v[72:75], v165, s[36:39], s75 offen offset:2048\n" \
  "buffer_load_dwordx4 v[80:83], v166, s[36:39], s75 offen offset:0\n" \
  "v_mul_f32 v196, s78, v196\n" \
  "v_mul_f32 v197, s78, v197\n" \
  "v_mul_f32 v198, s78, v198\n" \
  "v_mul_f32 v199, s78, v199\n" \
  "s_bitcmp1_b32 s66, 28\n" \
  "s_cbranch_scc1 Lp255_%=\n" \
  "buffer_store_dwordx4 v[196:199], v167, s[40:43], s51 offen offset:0 nt\n" \
  "Lq255_%=:\n" \
  "s_bfe_u32 s96, s65, 0x40014\n" \
  "s_mul_i32 s96, s96, Le0_1_%=-Le0_0_%=\n" \
  "s_add_u32 s96, s96, Le0_0_%=-Lpc_%=\n" \
  "s_add_u32 s96, s96, s94\n" \
  "s_addc_u32 s97, s95, 0\n" \
  "buffer_load_dwordx4 v[88:91], v166, s[36:39], s75 offen offset:2048\n" \
  "v_mul_f32 v212, s78, v212\n" \
  "v_mul_f32 v213, s78, v213\n" \
  "v_mul_f32 v214, s78, v214\n" \
  "v_mul_f32 v215, s78, v215\n" \
  "s_bitcmp1_b32 s66, 28\n" \
  "s_cbranch_scc1 Lp256_%=\n" \
  "buffer_store_dwordx4 v[212:215], v167, s[40:43], s51 offen offset:64 nt\n" \
  "Lq256_%=:\n" \
  "s_and_b32 s73, s65, 0xf00001ff\n" \
  "buffer_load_dwordx4 v[84:87], v166, s[36:39], s75 offen offset:1024\n" \
  "buffer_load_dwordx4 v[92:95], v166, s[36:39], s75 offen offset:3072\n" \
  "s_bfe_u32 m0, s65, 0x40018\n" \
  "s_nop 0\n" \
  "s_movreld_b32 s56, s77\n" \
  "s_branch Ls3_8_%=\n" \
  "Lsuf3_1_6_%=:\n" \
  "buffer_load_dwordx4 v[88:91], v166, s[36:39], s75 offen offset:2048\n" \
  "v_mul_f32 v212, s78, v212\n" \
  "v_mul_f32 v213, s78, v213\n" \
  "v_mul_f32 v214, s78, v214\n" \
  "v_mul_f32 v215, s78, v215\n" \
  "s_bitcmp1_b32 s66, 28\n" \
  "s_cbranch_scc1 Lp257_%=\n" \
  "buffer_store_dwordx4 v[212:215], v167, s[40:43], s51 offen offset:64 nt\n" \
  "Lq257_%=:\n" \
  "s_and_b32 s73, s65, 0xf00001ff\n" \
  "buffer_load_dwordx4 v[84:87], v166, s[36:39], s75 offen offset:1024\n" \
  "buffer_load_dwordx4 v[92:95], v166, s[36:39], s75 offen offset:3072\n" \
  "s_bfe_u32 m0, s65, 0x40018\n" \
  "s_nop 0\n" \
  "s_movreld_b32 s56, s77\n" \
  "s_branch Ls3_8_%=\n" \
  ".p2align 6\n" \
  "Lpre3_0_0_%=:\n" \
  "s_branch Ls3_1_%=\n" \
  ".org Lpre3_0_0_%=+704\n" \
  "Lpre3_0_1_%=:\n" \
  "s_branch Ls3_1_%=\n" \
  ".org Lpre3_0_0_%=+1408\n" \
  "Lpre3_0_2_%=:\n" \
  "ds_read_b128 v[144:147], v161 offset:36864\n" \
  "ds_read_b128 v[148:151], v161 offset:37888\n" \
  "ds_read_b128 v[152:155], v161 offset:38912\n" \
  "ds_read_b128 v[156:159], v161 offset:39936\n" \
  "s_add_u32 m0, s69, 65536\n" \
  "s_nop 0\n" \
  "buffer_load_dwordx4 v165, s[36:39], s56 offen lds\n" \
  "s_branch Ls3_2_%=\n" \
  ".org Lpre3_0_0_%=+2112\n" \
  "Lpre3_0_3_%=:\n" \
  "ds_read_b128 v[128:131], v161 offset:40960\n" \
  "ds_read_b128 v[132:135], v161 offset:41984\n" \
  "ds_read_b128 v[136:139], v161 offset:43008\n" \
  "ds_read_b128 v[140:143], v161 offset:44032\n" \
  "s_add_u32 m0, s69, 65536\n" \
  "s_nop 0\n" \
  "buffer_load_dwordx4 v165, s[36:39], s56 offen lds\n" \
  "s_add_u32 m0, s69, 69632\n" \
  "s_nop 0\n" \
  "buffer_load_dwordx4 v165, s[36:39], s57 offen lds\n" \
  "s_branch Ls3_3_%=\n" \
  ".org Lpre3_0_0_%=+2816\n" \
  "Lpre3_0_4_%=:\n" \
  "ds_read_b128 v[144:147], v161 offset:45056\n" \
  "ds_read_b128 v[148:151], v161 offset:46080\n" \
  "ds_read_b128 v[152:155], v161 offset:47104\n" \
  "ds_read_b128 v[156:159], v161 offset:48128\n" \
  "s_add_u32 m0, s69, 65536\n" \
  "s_nop 0\n" \
  "buffer_load_dwordx4 v165, s[36:39], s56 offen lds\n" \
  "s_add_u32 m0, s69, 69632\n" \
  "s_nop 0\n" \
  "buffer_load_dwordx4 v165, s[36:39], s57 offen lds\n" \
  "s_add_u32 m0, s69, 73728\n" \
  "s_nop 0\n" \
  "buffer_load_dwordx4 v165, s[36:39], s58 offen lds\n" \
  "s_branch Ls3_4_%=\n" \
  ".org Lpre3_0_0_%=+3520\n" \
  "Lpre3_0_5_%=:\n" \
  "ds_read_b128 v[128:131], v161 offset:49152\n" \
  "ds_read_b128 v[132:135], v161 offset:50176\n" \
  "ds_read_b128 v[136:139], v161 offset:51200\n" \
  "ds_read_b128 v[140:143], v161 offset:52224\n" \
  "s_add_u32 m0, s69, 65536\n" \
  "s_nop 0\n" \
  "buffer_load_dwordx4 v165, s[36:39], s56 offen lds\n" \
  "s_add_u32 m0, s69, 69632\n" \
  "s_nop 0\n" \
  "buffer_load_dwordx4 v165, s[36:39], s57 offen lds\n" \
  "s_add_u32 m0, s69, 73728\n" \
  "s_nop 0\n" \
  "buffer_load_dwordx4 v165, s[36:39], s58 offen lds\n" \
  "s_add_u32 m0, s69, 77824\n" \
  "s_nop 0\n" \
  "buffer_load_dwordx4 v165, s[36:39], s59 offen lds\n" \
  "s_branch Ls3_5_%=\n" \
  ".org Lpre3_0_0_%=+4224\n" \
  "Lpre3_0_6_%=:\n" \
  "ds_read_b128 v[144:147], v161 offset:53248\n" \
  "ds_read_b128 v[148:151], v161 offset:54272\n" \
  "ds_read_b128 v[152:155], v161 offset:55296\n" \
  "ds_read_b128 v[156:159], v161 offset:56320\n" \
  "s_add_u32 m0, s69, 65536\n" \
  "s_nop 0\n" \
  "buffer_load_dwordx4 v165, s[36:39], s56 offen lds\n" \
  "s_add_u32 m0, s69, 69632\n" \
  "s_nop 0\n" \
  "buffer_load_dwordx4 v165, s[36:39], s57 offen lds\n" \
  "s_add_u32 m0, s69, 73728\n" \
  "s_nop 0\n" \
  "buffer_load_dwordx4 v165, s[36:39], s58 offen lds\n" \
  "s_add_u32 m0, s69, 77824\n" \
  "s_nop 0\n" \
  "buffer_load_dwordx4 v165, s[36:39], s59 offen lds\n" \
  "s_add_u32 m0, s69, 81920\n" \
  "s_nop 0\n" \
  "buffer_load_dwordx4 v165, s[36:39], s60 offen lds\n" \
  "buffer_load_dwordx4 v[68:71], v165, s[36:39], s75 offen offset:1024\n" \
  "buffer_load_dwordx4 v[64:67], v165, s[36:39], s75 offen offset:0\n" \
  "v_readlane_b32 s67, v175, s70\n" \
  "v_readlane_b32 s65, v178, s70\n" \
  "v_readlane_b32 s76, v169, s70\n" \
  "v_readlane_b32 s77, v172, s70\n" \
  "v_readlane_b32 s71, v181, s70\n" \
  "s_add_u32 s70, s70, 1\n" \
  "v_add_u32 v163, s76, v162\n" \
  "s_branch Ls3_6_%=\n" \
  ".org Lpre3_0_0_%=+4928\n" \
  "Lpre3_0_7_%=:\n" \
  "ds_read_b128 v[128:131], v161 offset:57344\n" \
  "ds_read_b128 v[132:135], v161 offset:58368\n" \
  "ds_read_b128 v[136:139], v161 offset:59392\n" \
  "ds_read_b128 v[140:143], v161 offset:60416\n" \
  "s_add_u32 m0, s69, 65536\n" \
  "s_nop 0\n" \
  "buffer_load_dwordx4 v165, s[36:39], s56 offen lds\n" \
  "s_add_u32 m0, s69, 69632\n" \
  "s_nop 0\n" \
  "buffer_load_dwordx4 v165, s[36:39], s57 offen lds\n" \
  "s_add_u32 m0, s69, 73728\n" \
  "s_nop 0\n" \
  "buffer_load_dwordx4 v165, s[36:39], s58 offen lds\n" \
  "s_add_u32 m0, s69, 77824\n" \
  "s_nop 0\n" \
  "buffer_load_dwordx4 v165, s[36:39], s59 offen lds\n" \
  "s_add_u32 m0, s69, 81920\n" \
  "s_nop 0\n" \
  "buffer_load_dwordx4 v165, s[36:39], s60 offen lds\n" \
  "buffer_load_dwordx4 v[68:71], v165, s[36:39], s75 offen offset:1024\n" \
  "buffer_load_dwordx4 v[64:67], v165, s[36:39], s75 offen offset:0\n" \
  "v_readlane_b32 s67, v175, s70\n" \
  "v_readlane_b32 s65, v178, s70\n" \
  "v_readlane_b32 s76, v169, s70\n" \
  "v_readlane_b32 s77, v172, s70\n" \
  "v_readlane_b32 s71, v181, s70\n" \
  "s_add_u32 s70, s70, 1\n" \
  "v_add_u32 v163, s76, v162\n" \
  "s_add_u32 m0, s69, 86016\n" \
  "s_nop 0\n" \
  "buffer_load_dwordx4 v165, s[36:39], s61 offen lds\n" \
  "buffer_load_dwordx4 v[76:79], v165, s[36:39], s75 offen offset:3072\n" \
  "v_add_u32 v164, 4096, v163\n" \
  "s_cmp_gt_u32 s67, s68\n" \
  "s_cbranch_scc1 Lp258_%=\n" \
  "Lq258_%=:\n" \
  "buffer_load_dwordx4 v[72:75], v165, s[36:39], s75 offen offset:2048\n" \
  "buffer_load_dwordx4 v[80:83], v166, s[36:39], s75 offen offset:0\n" \
  "v_mul_f32 v196, s78, v196\n" \
  "v_mul_f32 v197, s78, v197\n" \
  "v_mul_f32 v198, s78, v198\n" \
  "v_mul_f32 v199, s78, v199\n" \
  "s_bitcmp1_b32 s66, 28\n" \
  "s_cbranch_scc1 Lp259_%=\n" \
  "buffer_store_dwordx4 v[196:199], v167, s[40:43], s51 offen offset:0 nt\n" \
  "Lq259_%=:\n" \
  "s_bfe_u32 s96, s65, 0x40014\n" \
  "s_mul_i32 s96, s96, Le0_1_%=-Le0_0_%=\n" \
  "s_add_u32 s96, s96, Le0_0_%=-Lpc_%=\n" \
  "s_add_u32 s96, s96, s94\n" \
  "s_addc_u32 s97, s95, 0\n" \
  "s_branch Ls3_7_%=\n" \
  ".org Lpre3_0_0_%=+5632\n" \
  "Lpre3_0_8_%=:\n" \
  "ds_read_b128 v[144:147], v161 offset:61440\n" \
  "ds_read_b128 v[148:151], v161 offset:62464\n" \
  "ds_read_b128 v[152:155], v161 offset:63488\n" \
  "ds_read_b128 v[156:159], v161 offset:64512\n" \
  "s_add_u32 m0, s69, 65536\n" \
  "s_nop 0\n" \
  "buffer_load_dwordx4 v165, s[36:39], s56 offen lds\n" \
  "s_add_u32 m0, s69, 69632\n" \
  "s_nop 0\n" \
  "buffer_load_dwordx4 v165, s[36:39], s57 offen lds\n" \
  "s_add_u32 m0, s69, 73728\n" \
  "s_nop 0\n" \
  "buffer_load_dwordx4 v165, s[36:39], s58 offen lds\n" \
  "s_add_u32 m0, s69, 77824\n" \
  "s_nop 0\n" \
  "buffer_load_dwordx4 v165, s[36:39], s59 offen lds\n" \
  "s_add_u32 m0, s69, 81920\n" \
  "s_nop 0\n" \
  "buffer_load_dwordx4 v165, s[36:39], s60 offen lds\n" \
  "buffer_load_dwordx4 v[68:71], v165, s[36:39], s75 offen offset:1024\n" \
  "buffer_load_dwordx4 v[64:67], v165, s[36:39], s75 offen offset:0\n" \
  "v_readlane_b32 s67, v175, s70\n" \
  "v_readlane_b32 s65, v178, s70\n" \
  "v_readlane_b32 s76, v169, s70\n" \
  "v_readlane_b32 s77, v172, s70\n" \
  "v_readlane_b32 s71, v181, s70\n" \
  "s_add_u32 s70, s70, 1\n" \
  "v_add_u32 v163, s76, v162\n" \
  "s_add_u32 m0, s69, 86016\n" \
  "s_nop 0\n" \
  "buffer_load_dwordx4 v165, s[36:39], s61 offen lds\n" \
  "buffer_load_dwordx4 v[76:79], v165, s[36:39], s75 offen offset:3072\n" \
  "v_add_u32 v164, 4096, v163\n" \
  "s_cmp_gt_u32 s67, s68\n" \
  "s_cbranch_scc1 Lp260_%=\n" \
  "Lq260_%=:\n" \
  "buffer_load_dwordx4 v[72:75], v165, s[36:39], s75 offen offset:2048\n" \
  "buffer_load_dwordx4 v[80:83], v166, s[36:39], s75 offen offset:0\n" \
  "v_mul_f32 v196, s78, v196\n" \
  "v_mul_f32 v197, s78, v197\n" \
  "v_mul_f32 v198, s78, v198\n" \
  "v_mul_f32 v199, s78, v199\n" \
  "s_bitcmp1_b32 s66, 28\n" \
  "s_cbranch_scc1 Lp261_%=\n" \
  "buffer_store_dwordx4 v[196:199], v167, s[40:43], s51 offen offset:0 nt\n" \
  "Lq261_%=:\n" \
  "s_bfe_u32 s96, s65, 0x40014\n" \
  "s_mul_i32 s96, s96, Le0_1_%=-Le0_0_%=\n" \
  "s_add_u32 s96, s96, Le0_0_%=-Lpc_%=\n" \
  "s_add_u32 s96, s96, s94\n" \
  "s_addc_u32 s97, s95, 0\n" \
  "s_add_u32 m0, s69, 90112\n" \
  "s_nop 0\n" \
  "buffer_load_dwordx4 v165, s[36:39], s62 offen lds\n" \
  "buffer_load_dwordx4 v[88:91], v166, s[36:39], s75 offen offset:2048\n" \
  "v_mul_f32 v212, s78, v212\n" \
  "v_mul_f32 v213, s78, v213\n" \
  "v_mul_f32 v214, s78, v214\n" \
  "v_mul_f32 v215, s78, v215\n" \
  "s_bitcmp1_b32 s66, 28\n" \
  "s_cbranch_scc1 Lp262_%=\n" \
  "buffer_store_dwordx4 v[212:215], v167, s[40:43], s51 offen offset:64 nt\n" \
  "Lq262_%=:\n" \
  "s_and_b32 s73, s65, 0xf00001ff\n" \
  "buffer_load_dwordx4 v[84:87], v166, s[36:39], s75 offen offset:1024\n" \
  "buffer_load_dwordx4 v[92:95], v166, s[36:39], s75 offen offset:3072\n" \
  "s_bfe_u32 m0, s65, 0x40018\n" \
  "s_nop 0\n" \
  "s_movreld_b32 s56, s77\n" \
  "s_branch Ls3_8_%=\n" \
  ".org Lpre3_0_0_%=+6336\n" \
  "Lpre3_1_0_%=:\n" \
  "s_branch Ls3_1_%=\n" \
  ".org Lpre3_0_0_%=+7040\n" \
  "Lpre3_1_1_%=:\n" \
  "s_branch Ls3_1_%=\n" \
  ".org Lpre3_0_0_%=+7744\n" \
  "Lpre3_1_2_%=:\n" \
  "ds_read_b128 v[144:147], v161 offset:36864\n" \
  "ds_read_b128 v[148:151], v161 offset:37888\n" \
  "ds_read_b128 v[152:155], v161 offset:38912\n" \
  "ds_read_b128 v[156:159], v161 offset:39936\n" \
  "s_branch Ls3_2_%=\n" \
  ".org Lpre3_0_0_%=+8448\n" \
  "Lpre3_1_3_%=:\n" \
  "ds_read_b128 v[128:131], v161 offset:40960\n" \
  "ds_read_b128 v[132:135], v161 offset:41984\n" \
  "ds_read_b128 v[136:139], v161 offset:43008\n" \
  "ds_read_b128 v[140:143], v161 offset:44032\n" \
  "s_branch Ls3_3_%=\n" \
  ".org Lpre3_0_0_%=+9152\n" \
  "Lpre3_1_4_%=:\n" \
  "ds_read_b128 v[144:147], v161 offset:45056\n" \
  "ds_read_b128 v[148:151], v161 offset:46080\n" \
  "ds_read_b128 v[152:155], v161 offset:47104\n" \
  "ds_read_b128 v[156:159], v161 offset:48128\n" \
  "s_branch Ls3_4_%=\n" \
  ".org Lpre3_0_0_%=+9856\n" \
  "Lpre3_1_5_%=:\n" \
  "ds_read_b128 v[128:131], v161 offset:49152\n" \
  "ds_read_b128 v[132:135], v161 offset:50176\n" \
  "ds_read_b128 v[136:139], v161 offset:51200\n" \
  "ds_read_b128 v[140:143], v161 offset:52224\n" \
  "s_branch Ls3_5_%=\n" \
  ".org Lpre3_0_0_%=+10560\n" \
  "Lpre3_1_6_%=:\n" \
  "ds_read_b128 v[144:147], v161 offset:53248\n" \
  "ds_read_b128 v[148:151], v161 offset:54272\n" \
  "ds_read_b128 v[152:155], v161 offset:55296\n" \
  "ds_read_b128 v[156:159], v161 offset:56320\n" \
  "buffer_load_dwordx4 v[68:71], v165, s[36:39], s75 offen offset:1024\n" \
  "buffer_load_dwordx4 v[64:67], v165, s[36:39], s75 offen offset:0\n" \
  "v_readlane_b32 s67, v175, s70\n" \
  "v_readlane_b32 s65, v178, s70\n" \
  "v_readlane_b32 s76, v169, s70\n" \
  "v_readlane_b32 s77, v172, s70\n" \
  "v_readlane_b32 s71, v181, s70\n" \
  "s_add_u32 s70, s70, 1\n" \
  "v_add_u32 v163, s76, v162\n" \
  "s_branch Ls3_6_%=\n" \
  ".org Lpre3_0_0_%=+11264\n" \
  "Lpre3_1_7_%=:\n" \
  "ds_read_b128 v[128:131], v161 offset:57344\n" \
  "ds_read_b128 v[132:135], v161 offset:58368\n" \
  "ds_read_b128 v[136:139], v161 offset:59392\n" \
  "ds_read_b128 v[140:143], v161 offset:60416\n" \
  "buffer_load_dwordx4 v[68:71], v165, s[36:39], s75 offen offset:1024\n" \
  "buffer_load_dwordx4 v[64:67], v165, s[36:39], s75 offen offset:0\n" \
  "v_readlane_b32 s67, v175, s70\n" \
  "v_readlane_b32 s65, v178, s70\n" \
  "v_readlane_b32 s76, v169, s70\n" \
  "v_readlane_b32 s77, v172, s70\n" \
  "v_readlane_b32 s71, v181, s70\n" \
  "s_add_u32 s70, s70, 1\n" \
  "v_add_u32 v163, s76, v162\n" \
  "buffer_load_dwordx4 v[76:79], v165, s[36:39], s75 offen offset:3072\n" \
  "v_add_u32 v164, 4096, v163\n" \
  "s_cmp_gt_u32 s67, s68\n" \
  "s_cbranch_scc1 Lp263_%=\n" \
  "Lq263_%=:\n" \
  "buffer_load_dwordx4 v[72:75], v165, s[36:39], s75 offen offset:2048\n" \
  "buffer_load_dwordx4 v[80:83], v166, s[36:39], s75 offen offset:0\n" \
  "v_mul_f32 v196, s78, v196\n" \
  "v_mul_f32 v197, s78, v197\n" \
  "v_mul_f32 v198, s78, v198\n" \
  "v_mul_f32 v199, s78, v199\n" \
  "s_bitcmp1_b32 s66, 28\n" \
  "s_cbranch_scc1 Lp264_%=\n" \
  "buffer_store_dwordx4 v[196:199], v167, s[40:43], s51 offen offset:0 nt\n" \
  "Lq264_%=:\n" \
  "s_bfe_u32 s96, s65, 0x40014\n" \
  "s_mul_i32 s96, s96, Le0_1_%=-Le0_0_%=\n" \
  "s_add_u32 s96, s96, Le0_0_%=-Lpc_%=\n" \
  "s_add_u32 s96, s96, s94\n" \
  "s_addc_u32 s97, s95, 0\n" \
  "s_branch Ls3_7_%=\n" \
  ".org Lpre3_0_0_%=+11968\n" \
  "Lpre3_1_8_%=:\n" \
  "ds_read_b128 v[144:147], v161 offset:61440\n" \
  "ds_read_b128 v[148:151], v161 offset:62464\n" \
  "ds_read_b128 v[152:155], v161 offset:63488\n" \
  "ds_read_b128 v[156:159], v161 offset:64512\n" \
  "buffer_load_dwordx4 v[68:71], v165, s[36:39], s75 offen offset:1024\n" \
  "buffer_load_dwordx4 v[64:67], v165, s[36:39], s75 offen offset:0\n" \
  "v_readlane_b32 s67, v175, s70\n" \
  "v_readlane_b32 s65, v178, s70\n" \
  "v_readlane_b32 s76, v169, s70\n" \
  "v_readlane_b32 s77, v172, s70\n" \
  "v_readlane_b32 s71, v181, s70\n" \
  "s_add_u32 s70, s70, 1\n" \
  "v_add_u32 v163, s76, v162\n" \
  "buffer_load_dwordx4 v[76:79], v165, s[36:39], s75 offen offset:3072\n" \
  "v_add_u32 v164, 4096, v163\n" \
  "s_cmp_gt_u32 s67, s68\n" \
  "s_cbranch_scc1 Lp265_%=\n" \
  "Lq265_%=:\n" \
  "buffer_load_dwordx4 v[72:75], v165, s[36:39], s75 offen offset:2048\n" \
  "buffer_load_dwordx4 v[80:83], v166, s[36:39], s75 offen offset:0\n" \
  "v_mul_f32 v196, s78, v196\n" \
  "v_mul_f32 v197, s78, v197\n" \
  "v_mul_f32 v198, s78, v198\n" \
  "v_mul_f32 v199, s78, v199\n" \
  "s_bitcmp1_b32 s66, 28\n" \
  "s_cbranch_scc1 Lp266_%=\n" \
  "buffer_store_dwordx4 v[196:199], v167, s[40:43], s51 offen offset:0 nt\n" \
  "Lq266_%=:\n" \
  "s_bfe_u32 s96, s65, 0x40014\n" \
  "s_mul_i32 s96, s96, Le0_1_%=-Le0_0_%=\n" \
  "s_add_u32 s96, s96, Le0_0_%=-Lpc_%=\n" \
  "s_add_u32 s96, s96, s94\n" \
  "s_addc_u32 s97, s95, 0\n" \
  "buffer_load_dwordx4 v[88:91], v166, s[36:39], s75 offen offset:2048\n" \
  "v_mul_f32 v212, s78, v212\n" \
  "v_mul_f32 v213, s78, v213\n" \
  "v_mul_f32 v214, s78, v214\n" \
  "v_mul_f32 v215, s78, v215\n" \
  "s_bitcmp1_b32 s66, 28\n" \
  "s_cbranch_scc1 Lp267_%=\n" \
  "buffer_store_dwordx4 v[212:215], v167, s[40:43], s51 offen offset:64 nt\n" \
  "Lq267_%=:\n" \
  "s_and_b32 s73, s65, 0xf00001ff\n" \
  "buffer_load_dwordx4 v[84:87], v166, s[36:39], s75 offen offset:1024\n" \
  "buffer_load_dwordx4 v[92:95], v166, s[36:39], s75 offen offset:3072\n" \
  "s_bfe_u32 m0, s65, 0x40018\n" \
  "s_nop 0\n" \
  "s_movreld_b32 s56, s77\n" \
  "s_branch Ls3_8_%=\n" \
  ".p2align 6\n" \
  "Lev0_0_%=:\n" \
  "s_nop 7\n" \
  "v_accvgpr_read_b32 v184, a224\n" \
  "v_accvgpr_read_b32 v185, a225\n" \
  "v_accvgpr_read_b32 v186, a226\n" \
  "v_accvgpr_read_b32 v187, a227\n" \
  "v_accvgpr_write_b32 a224, 0\n" \
  "v_accvgpr_write_b32 a225, 0\n" \
  "v_accvgpr_write_b32 a226, 0\n" \
  "v_accvgpr_write_b32 a227, 0\n" \
  "v_accvgpr_read_b32 v188, a228\n" \
  "v_accvgpr_read_b32 v189, a229\n" \
  "v_accvgpr_read_b32 v190, a230\n" \
  "v_accvgpr_read_b32 v191, a231\n" \
  "v_accvgpr_write_b32 a228, 0\n" \
  "v_accvgpr_write_b32 a229, 0\n" \
  "v_accvgpr_write_b32 a230, 0\n" \
  "v_accvgpr_write_b32 a231, 0\n" \
  "v_accvgpr_read_b32 v192, a232\n" \
  "v_accvgpr_read_b32 v193, a233\n" \
  "v_accvgpr_read_b32 v194, a234\n" \
  "v_accvgpr_read_b32 v195, a235\n" \
  "v_accvgpr_write_b32 a232, 0\n" \
  "v_accvgpr_write_b32 a233, 0\n" \
  "v_accvgpr_write_b32 a234, 0\n" \
  "v_accvgpr_write_b32 a235, 0\n" \
  "v_accvgpr_read_b32 v196, a236\n" \
  "v_accvgpr_read_b32 v197, a237\n" \
  "v_accvgpr_read_b32 v198, a238\n" \
  "v_accvgpr_read_b32 v199, a239\n" \
  "v_accvgpr_write_b32 a236, 0\n" \
  "v_accvgpr_write_b32 a237, 0\n" \
  "v_accvgpr_write_b32 a238, 0\n" \
  "v_accvgpr_write_b32 a239, 0\n" \
  "v_accvgpr_read_b32 v200, a240\n" \
  "v_accvgpr_read_b32 v201, a241\n" \
  "v_accvgpr_read_b32 v202, a242\n" \
  "v_accvgpr_read_b32 v203, a243\n" \
  "v_accvgpr_write_b32 a240, 0\n" \
  "v_accvgpr_write_b32 a241, 0\n" \
  "v_accvgpr_write_b32 a242, 0\n" \
  "v_accvgpr_write_b32 a243, 0\n" \
  "v_accvgpr_read_b32 v204, a244\n" \
  "v_accvgpr_read_b32 v205, a245\n" \
  "v_accvgpr_read_b32 v206, a246\n" \
  "v_accvgpr_read_b32 v207, a247\n" \
  "v_accvgpr_write_b32 a244, 0\n" \
  "v_accvgpr_write_b32 a245, 0\n" \
  "v_accvgpr_write_b32 a246, 0\n" \
  "v_accvgpr_write_b32 a247, 0\n" \
  "v_accvgpr_read_b32 v208, a248\n" \
  "v_accvgpr_read_b32 v209, a249\n" \
  "v_accvgpr_read_b32 v210, a250\n" \
  "v_accvgpr_read_b32 v211, a251\n" \
  "v_accvgpr_write_b32 a248, 0\n" \
  "v_accvgpr_write_b32 a249, 0\n" \
  "v_accvgpr_write_b32 a250, 0\n" \
  "v_accvgpr_write_b32 a251, 0\n" \
  "v_accvgpr_read_b32 v212, a252\n" \
  "v_accvgpr_read_b32 v213, a253\n" \
  "v_accvgpr_read_b32 v214, a254\n" \
  "v_accvgpr_read_b32 v215, a255\n" \
  "v_accvgpr_write_b32 a252, 0\n" \
  "v_accvgpr_write_b32 a253, 0\n" \
  "v_accvgpr_write_b32 a254, 0\n" \
  "v_accvgpr_write_b32 a255, 0\n" \
  "s_setpc_b64 s[92:93]\n" \
  "Lev0_1_%=:\n" \
  "s_nop 7\n" \
  "v_accvgpr_read_b32 v184, a0\n" \
  "v_accvgpr_read_b32 v185, a1\n" \
  "v_accvgpr_read_b32 v186, a2\n" \
  "v_accvgpr_read_b32 v187, a3\n" \
  "v_accvgpr_write_b32 a0, 0\n" \
  "v_accvgpr_write_b32 a1, 0\n" \
  "v_accvgpr_write_b32 a2, 0\n" \
  "v_accvgpr_write_b32 a3, 0\n" \
  "v_accvgpr_read_b32 v188, a4\n" \
  "v_accvgpr_read_b32 v189, a5\n" \
  "v_accvgpr_read_b32 v190, a6\n" \
  "v_accvgpr_read_b32 v191, a7\n" \
  "v_accvgpr_write_b32 a4, 0\n" \
  "v_accvgpr_write_b32 a5, 0\n" \
  "v_accvgpr_write_b32 a6, 0\n" \
  "v_accvgpr_write_b32 a7, 0\n" \
  "v_accvgpr_read_b32 v192, a8\n" \
  "v_accvgpr_read_b32 v193, a9\n" \
  "v_accvgpr_read_b32 v194, a10\n" \
  "v_accvgpr_read_b32 v195, a11\n" \
  "v_accvgpr_write_b32 a8, 0\n" \
  "v_accvgpr_write_b32 a9, 0\n" \
  "v_accvgpr_write_b32 a10, 0\n" \
  "v_accvgpr_write_b32 a11, 0\n" \
  "v_accvgpr_read_b32 v196, a12\n" \
  "v_accvgpr_read_b32 v197, a13\n" \
  "v_accvgpr_read_b32 v198, a14\n" \
  "v_accvgpr_read_b32 v199, a15\n" \
  "v_accvgpr_write_b32 a12, 0\n" \
  "v_accvgpr_write_b32 a13, 0\n" \
  "v_accvgpr_write_b32 a14, 0\n" \
  "v_accvgpr_write_b32 a15, 0\n" \
  "v_accvgpr_read_b32 v200, a16\n" \
  "v_accvgpr_read_b32 v201, a17\n" \
  "v_accvgpr_read_b32 v202, a18\n" \
  "v_accvgpr_read_b32 v203, a19\n" \
  "v_accvgpr_write_b32 a16, 0\n" \
  "v_accvgpr_write_b32 a17, 0\n" \
  "v_accvgpr_write_b32 a18, 0\n" \
  "v_accvgpr_write_b32 a19, 0\n" \
  "v_accvgpr_read_b32 v204, a20\n" \
  "v_accvgpr_read_b32 v205, a21\n" \
  "v_accvgpr_read_b32 v206, a22\n" \
  "v_accvgpr_read_b32 v207, a23\n" \
  "v_accvgpr_write_b32 a20, 0\n" \
  "v_accvgpr_write_b32 a21, 0\n" \
  "v_accvgpr_write_b32 a22, 0\n" \
  "v_accvgpr_write_b32 a23, 0\n" \
  "v_accvgpr_read_b32 v208, a24\n" \
  "v_accvgpr_read_b32 v209, a25\n" \
  "v_accvgpr_read_b32 v210, a26\n" \
  "v_accvgpr_read_b32 v211, a27\n" \
  "v_accvgpr_write_b32 a24, 0\n" \
  "v_accvgpr_write_b32 a25, 0\n" \
  "v_accvgpr_write_b32 a26, 0\n" \
  "v_accvgpr_write_b32 a27, 0\n" \
  "v_accvgpr_read_b32 v212, a28\n" \
  "v_accvgpr_read_b32 v213, a29\n" \
  "v_accvgpr_read_b32 v214, a30\n" \
  "v_accvgpr_read_b32 v215, a31\n" \
  "v_accvgpr_write_b32 a28, 0\n" \
  "v_accvgpr_write_b32 a29, 0\n" \
  "v_accvgpr_write_b32 a30, 0\n" \
  "v_accvgpr_write_b32 a31, 0\n" \
  "s_setpc_b64 s[92:93]\n" \
  "Lev0_2_%=:\n" \
  "s_nop 7\n" \
  "v_accvgpr_read_b32 v184, a32\n" \
  "v_accvgpr_read_b32 v185, a33\n" \
  "v_accvgpr_read_b32 v186, a34\n" \
  "v_accvgpr_read_b32 v187, a35\n" \
  "v_accvgpr_write_b32 a32, 0\n" \
  "v_accvgpr_write_b32 a33, 0\n" \
  "v_accvgpr_write_b32 a34, 0\n" \
  "v_accvgpr_write_b32 a35, 0\n" \
  "v_accvgpr_read_b32 v188, a36\n" \
  "v_accvgpr_read_b32 v189, a37\n" \
  "v_accvgpr_read_b32 v190, a38\n" \
  "v_accvgpr_read_b32 v191, a39\n" \
  "v_accvgpr_write_b32 a36, 0\n" \
  "v_accvgpr_write_b32 a37, 0\n" \
  "v_accvgpr_write_b32 a38, 0\n" \
  "v_accvgpr_write_b32 a39, 0\n" \
  "v_accvgpr_read_b32 v192, a40\n" \
  "v_accvgpr_read_b32 v193, a41\n" \
  "v_accvgpr_read_b32 v194, a42\n" \
  "v_accvgpr_read_b32 v195, a43\n" \
  "v_accvgpr_write_b32 a40, 0\n" \
  "v_accvgpr_write_b32 a41, 0\n" \
  "v_accvgpr_write_b32 a42, 0\n" \
  "v_accvgpr_write_b32 a43, 0\n" \
  "v_accvgpr_read_b32 v196, a44\n" \
  "v_accvgpr_read_b32 v197, a45\n" \
  "v_accvgpr_read_b32 v198, a46\n" \
  "v_accvgpr_read_b32 v199, a47\n" \
  "v_accvgpr_write_b32 a44, 0\n" \
  "v_accvgpr_write_b32 a45, 0\n" \
  "v_accvgpr_write_b32 a46, 0\n" \
  "v_accvgpr_write_b32 a47, 0\n" \
  "v_accvgpr_read_b32 v200, a48\n" \
  "v_accvgpr_read_b32 v201, a49\n" \
  "v_accvgpr_read_b32 v202, a50\n" \
  "v_accvgpr_read_b32 v203, a51\n" \
  "v_accvgpr_write_b32 a48, 0\n" \
  "v_accvgpr_write_b32 a49, 0\n" \
  "v_accvgpr_write_b32 a50, 0\n" \
  "v_accvgpr_write_b32 a51, 0\n" \
  "v_accvgpr_read_b32 v204, a52\n" \
  "v_accvgpr_read_b32 v205, a53\n" \
  "v_accvgpr_read_b32 v206, a54\n" \
  "v_accvgpr_read_b32 v207, a55\n" \
  "v_accvgpr_write_b32 a52, 0\n" \
  "v_accvgpr_write_b32 a53, 0\n" \
  "v_accvgpr_write_b32 a54, 0\n" \
  "v_accvgpr_write_b32 a55, 0\n" \
  "v_accvgpr_read_b32 v208, a56\n" \
  "v_accvgpr_read_b32 v209, a57\n" \
  "v_accvgpr_read_b32 v210, a58\n" \
  "v_accvgpr_read_b32 v211, a59\n" \
  "v_accvgpr_write_b32 a56, 0\n" \
  "v_accvgpr_write_b32 a57, 0\n" \
  "v_accvgpr_write_b32 a58, 0\n" \
  "v_accvgpr_write_b32 a59, 0\n" \
  "v_accvgpr_read_b32 v212, a60\n" \
  "v_accvgpr_read_b32 v213, a61\n" \
  "v_accvgpr_read_b32 v214, a62\n" \
  "v_accvgpr_read_b32 v215, a63\n" \
  "v_accvgpr_write_b32 a60, 0\n" \
  "v_accvgpr_write_b32 a61, 0\n" \
  "v_accvgpr_write_b32 a62, 0\n" \
  "v_accvgpr_write_b32 a63, 0\n" \
  "s_setpc_b64 s[92:93]\n" \
  "Lev0_3_%=:\n" \
  "s_nop 7\n" \
  "v_accvgpr_read_b32 v184, a64\n" \
  "v_accvgpr_read_b32 v185, a65\n" \
  "v_accvgpr_read_b32 v186, a66\n" \
  "v_accvgpr_read_b32 v187, a67\n" \
  "v_accvgpr_write_b32 a64, 0\n" \
  "v_accvgpr_write_b32 a65, 0\n" \
  "v_accvgpr_write_b32 a66, 0\n" \
  "v_accvgpr_write_b32 a67, 0\n" \
  "v_accvgpr_read_b32 v188, a68\n" \
  "v_accvgpr_read_b32 v189, a69\n" \
  "v_accvgpr_read_b32 v190, a70\n" \
  "v_accvgpr_read_b32 v191, a71\n" \
  "v_accvgpr_write_b32 a68, 0\n" \
  "v_accvgpr_write_b32 a69, 0\n" \
  "v_accvgpr_write_b32 a70, 0\n" \
  "v_accvgpr_write_b32 a71, 0\n" \
  "v_accvgpr_read_b32 v192, a72\n" \
  "v_accvgpr_read_b32 v193, a73\n" \
  "v_accvgpr_read_b32 v194, a74\n" \
  "v_accvgpr_read_b32 v195, a75\n" \
  "v_accvgpr_write_b32 a72, 0\n" \
  "v_accvgpr_write_b32 a73, 0\n" \
  "v_accvgpr_write_b32 a74, 0\n" \
  "v_accvgpr_write_b32 a75, 0\n" \
  "v_accvgpr_read_b32 v196, a76\n" \
  "v_accvgpr_read_b32 v197, a77\n" \
  "v_accvgpr_read_b32 v198, a78\n" \
  "v_accvgpr_read_b32 v199, a79\n" \
  "v_accvgpr_write_b32 a76, 0\n" \
  "v_accvgpr_write_b32 a77, 0\n" \
  "v_accvgpr_write_b32 a78, 0\n" \
  "v_accvgpr_write_b32 a79, 0\n" \
  "v_accvgpr_read_b32 v200, a80\n" \
  "v_accvgpr_read_b32 v201, a81\n" \
  "v_accvgpr_read_b32 v202, a82\n" \
  "v_accvgpr_read_b32 v203, a83\n" \
  "v_accvgpr_write_b32 a80, 0\n" \
  "v_accvgpr_write_b32 a81, 0\n" \
  "v_accvgpr_write_b32 a82, 0\n" \
  "v_accvgpr_write_b32 a83, 0\n" \
  "v_accvgpr_read_b32 v204, a84\n" \
  "v_accvgpr_read_b32 v205, a85\n" \
  "v_accvgpr_read_b32 v206, a86\n" \
  "v_accvgpr_read_b32 v207, a87\n" \
  "v_accvgpr_write_b32 a84, 0\n" \
  "v_accvgpr_write_b32 a85, 0\n" \
  "v_accvgpr_write_b32 a86, 0\n" \
  "v_accvgpr_write_b32 a87, 0\n" \
  "v_accvgpr_read_b32 v208, a88\n" \
  "v_accvgpr_read_b32 v209, a89\n" \
  "v_accvgpr_read_b32 v210, a90\n" \
  "v_accvgpr_read_b32 v211, a91\n" \
  "v_accvgpr_write_b32 a88, 0\n" \
  "v_accvgpr_write_b32 a89, 0\n" \
  "v_accvgpr_write_b32 a90, 0\n" \
  "v_accvgpr_write_b32 a91, 0\n" \
  "v_accvgpr_read_b32 v212, a92\n" \
  "v_accvgpr_read_b32 v213, a93\n" \
  "v_accvgpr_read_b32 v214, a94\n" \
  "v_accvgpr_read_b32 v215, a95\n" \
  "v_accvgpr_write_b32 a92, 0\n" \
  "v_accvgpr_write_b32 a93, 0\n" \
  "v_accvgpr_write_b32 a94, 0\n" \
  "v_accvgpr_write_b32 a95, 0\n" \
  "s_setpc_b64 s[92:93]\n" \
  "Lev0_4_%=:\n" \
  "s_nop 7\n" \
  "v_accvgpr_read_b32 v184, a96\n" \
  "v_accvgpr_read_b32 v185, a97\n" \
  "v_accvgpr_read_b32 v186, a98\n" \
  "v_accvgpr_read_b32 v187, a99\n" \
  "v_accvgpr_write_b32 a96, 0\n" \
  "v_accvgpr_write_b32 a97, 0\n" \
  "v_accvgpr_write_b32 a98, 0\n" \
  "v_accvgpr_write_b32 a99, 0\n" \
  "v_accvgpr_read_b32 v188, a100\n" \
  "v_accvgpr_read_b32 v189, a101\n" \
  "v_accvgpr_read_b32 v190, a102\n" \
  "v_accvgpr_read_b32 v191, a103\n" \
  "v_accvgpr_write_b32 a100, 0\n" \
  "v_accvgpr_write_b32 a101, 0\n" \
  "v_accvgpr_write_b32 a102, 0\n" \
  "v_accvgpr_write_b32 a103, 0\n" \
  "v_accvgpr_read_b32 v192, a104\n" \
  "v_accvgpr_read_b32 v193, a105\n" \
  "v_accvgpr_read_b32 v194, a106\n" \
  "v_accvgpr_read_b32 v195, a107\n" \
  "v_accvgpr_write_b32 a104, 0\n" \
  "v_accvgpr_write_b32 a105, 0\n" \
  "v_accvgpr_write_b32 a106, 0\n" \
  "v_accvgpr_write_b32 a107, 0\n" \
  "v_accvgpr_read_b32 v196, a108\n" \
  "v_accvgpr_read_b32 v197, a109\n" \
  "v_accvgpr_read_b32 v198, a110\n" \
  "v_accvgpr_read_b32 v199, a111\n" \
  "v_accvgpr_write_b32 a108, 0\n" \
  "v_accvgpr_write_b32 a109, 0\n" \
  "v_accvgpr_write_b32 a110, 0\n" \
  "v_accvgpr_write_b32 a111, 0\n" \
  "v_accvgpr_read_b32 v200, a112\n" \
  "v_accvgpr_read_b32 v201, a113\n" \
  "v_accvgpr_read_b32 v202, a114\n" \
  "v_accvgpr_read_b32 v203, a115\n" \
  "v_accvgpr_write_b32 a112, 0\n" \
  "v_accvgpr_write_b32 a113, 0\n" \
  "v_accvgpr_write_b32 a114, 0\n" \
  "v_accvgpr_write_b32 a115, 0\n" \
  "v_accvgpr_read_b32 v204, a116\n" \
  "v_accvgpr_read_b32 v205, a117\n" \
  "v_accvgpr_read_b32 v206, a118\n" \
  "v_accvgpr_read_b32 v207, a119\n" \
  "v_accvgpr_write_b32 a116, 0\n" \
  "v_accvgpr_write_b32 a117, 0\n" \
  "v_accvgpr_write_b32 a118, 0\n" \
  "v_accvgpr_write_b32 a119, 0\n" \
  "v_accvgpr_read_b32 v208, a120\n" \
  "v_accvgpr_read_b32 v209, a121\n" \
  "v_accvgpr_read_b32 v210, a122\n" \
  "v_accvgpr_read_b32 v211, a123\n" \
  "v_accvgpr_write_b32 a120, 0\n" \
  "v_accvgpr_write_b32 a121, 0\n" \
  "v_accvgpr_write_b32 a122, 0\n" \
  "v_accvgpr_write_b32 a123, 0\n" \
  "v_accvgpr_read_b32 v212, a124\n" \
  "v_accvgpr_read_b32 v213, a125\n" \
  "v_accvgpr_read_b32 v214, a126\n" \
  "v_accvgpr_read_b32 v215, a127\n" \
  "v_accvgpr_write_b32 a124, 0\n" \
  "v_accvgpr_write_b32 a125, 0\n" \
  "v_accvgpr_write_b32 a126, 0\n" \
  "v_accvgpr_write_b32 a127, 0\n" \
  "s_setpc_b64 s[92:93]\n" \
  "Lev0_5_%=:\n" \
  "s_nop 7\n" \
  "v_accvgpr_read_b32 v184, a128\n" \
  "v_accvgpr_read_b32 v185, a129\n" \
  "v_accvgpr_read_b32 v186, a130\n" \
  "v_accvgpr_read_b32 v187, a131\n" \
  "v_accvgpr_write_b32 a128, 0\n" \
  "v_accvgpr_write_b32 a129, 0\n" \
  "v_accvgpr_write_b32 a130, 0\n" \
  "v_accvgpr_write_b32 a131, 0\n" \
  "v_accvgpr_read_b32 v188, a132\n" \
  "v_accvgpr_read_b32 v189, a133\n" \
  "v_accvgpr_read_b32 v190, a134\n" \
  "v_accvgpr_read_b32 v191, a135\n" \
  "v_accvgpr_write_b32 a132, 0\n" \
  "v_accvgpr_write_b32 a133, 0\n" \
  "v_accvgpr_write_b32 a134, 0\n" \
  "v_accvgpr_write_b32 a135, 0\n" \
  "v_accvgpr_read_b32 v192, a136\n" \
  "v_accvgpr_read_b32 v193, a137\n" \
  "v_accvgpr_read_b32 v194, a138\n" \
  "v_accvgpr_read_b32 v195, a139\n" \
  "v_accvgpr_write_b32 a136, 0\n" \
  "v_accvgpr_write_b32 a137, 0\n" \
  "v_accvgpr_write_b32 a138, 0\n" \
  "v_accvgpr_write_b32 a139, 0\n" \
  "v_accvgpr_read_b32 v196, a140\n" \
  "v_accvgpr_read_b32 v197, a141\n" \
  "v_accvgpr_read_b32 v198, a142\n" \
  "v_accvgpr_read_b32 v199, a143\n" \
  "v_accvgpr_write_b32 a140, 0\n" \
  "v_accvgpr_write_b32 a141, 0\n" \
  "v_accvgpr_write_b32 a142, 0\n" \
  "v_accvgpr_write_b32 a143, 0\n" \
  "v_accvgpr_read_b32 v200, a144\n" \
  "v_accvgpr_read_b32 v201, a145\n" \
  "v_accvgpr_read_b32 v202, a146\n" \
  "v_accvgpr_read_b32 v203, a147\n" \
  "v_accvgpr_write_b32 a144, 0\n" \
  "v_accvgpr_write_b32 a145, 0\n" \
  "v_accvgpr_write_b32 a146, 0\n" \
  "v_accvgpr_write_b32 a147, 0\n" \
  "v_accvgpr_read_b32 v204, a148\n" \
  "v_accvgpr_read_b32 v205, a149\n" \
  "v_accvgpr_read_b32 v206, a150\n" \
  "v_accvgpr_read_b32 v207, a151\n" \
  "v_accvgpr_write_b32 a148, 0\n" \
  "v_accvgpr_write_b32 a149, 0\n" \
  "v_accvgpr_write_b32 a150, 0\n" \
  "v_accvgpr_write_b32 a151, 0\n" \
  "v_accvgpr_read_b32 v208, a152\n" \
  "v_accvgpr_read_b32 v209, a153\n" \
  "v_accvgpr_read_b32 v210, a154\n" \
  "v_accvgpr_read_b32 v211, a155\n" \
  "v_accvgpr_write_b32 a152, 0\n" \
  "v_accvgpr_write_b32 a153, 0\n" \
  "v_accvgpr_write_b32 a154, 0\n" \
  "v_accvgpr_write_b32 a155, 0\n" \
  "v_accvgpr_read_b32 v212, a156\n" \
  "v_accvgpr_read_b32 v213, a157\n" \
  "v_accvgpr_read_b32 v214, a158\n" \
  "v_accvgpr_read_b32 v215, a159\n" \
  "v_accvgpr_write_b32 a156, 0\n" \
  "v_accvgpr_write_b32 a157, 0\n" \
  "v_accvgpr_write_b32 a158, 0\n" \
  "v_accvgpr_write_b32 a159, 0\n" \
  "s_setpc_b64 s[92:93]\n" \
  "Lev0_6_%=:\n" \
  "s_nop 7\n" \
  "v_accvgpr_read_b32 v184, a160\n" \
  "v_accvgpr_read_b32 v185, a161\n" \
  "v_accvgpr_read_b32 v186, a162\n" \
  "v_accvgpr_read_b32 v187, a163\n" \
  "v_accvgpr_write_b32 a160, 0\n" \
  "v_accvgpr_write_b32 a161, 0\n" \
  "v_accvgpr_write_b32 a162, 0\n" \
  "v_accvgpr_write_b32 a163, 0\n" \
  "v_accvgpr_read_b32 v188, a164\n" \
  "v_accvgpr_read_b32 v189, a165\n" \
  "v_accvgpr_read_b32 v190, a166\n" \
  "v_accvgpr_read_b32 v191, a167\n" \
  "v_accvgpr_write_b32 a164, 0\n" \
  "v_accvgpr_write_b32 a165, 0\n" \
  "v_accvgpr_write_b32 a166, 0\n" \
  "v_accvgpr_write_b32 a167, 0\n" \
  "v_accvgpr_read_b32 v192, a168\n" \
  "v_accvgpr_read_b32 v193, a169\n" \
  "v_accvgpr_read_b32 v194, a170\n" \
  "v_accvgpr_read_b32 v195, a171\n" \
  "v_accvgpr_write_b32 a168, 0\n" \
  "v_accvgpr_write_b32 a169, 0\n" \
  "v_accvgpr_write_b32 a170, 0\n" \
  "v_accvgpr_write_b32 a171, 0\n" \
  "v_accvgpr_read_b32 v196, a172\n" \
  "v_accvgpr_read_b32 v197, a173\n" \
  "v_accvgpr_read_b32 v198, a174\n" \
  "v_accvgpr_read_b32 v199, a175\n" \
  "v_accvgpr_write_b32 a172, 0\n" \
  "v_accvgpr_write_b32 a173, 0\n" \
  "v_accvgpr_write_b32 a174, 0\n" \
  "v_accvgpr_write_b32 a175, 0\n" \
  "v_accvgpr_read_b32 v200, a176\n" \
  "v_accvgpr_read_b32 v201, a177\n" \
  "v_accvgpr_read_b32 v202, a178\n" \
  "v_accvgpr_read_b32 v203, a179\n" \
  "v_accvgpr_write_b32 a176, 0\n" \
  "v_accvgpr_write_b32 a177, 0\n" \
  "v_accvgpr_write_b32 a178, 0\n" \
  "v_accvgpr_write_b32 a179, 0\n" \
  "v_accvgpr_read_b32 v204, a180\n" \
  "v_accvgpr_read_b32 v205, a181\n" \
  "v_accvgpr_read_b32 v206, a182\n" \
  "v_accvgpr_read_b32 v207, a183\n" \
  "v_accvgpr_write_b32 a180, 0\n" \
  "v_accvgpr_write_b32 a181, 0\n" \
  "v_accvgpr_write_b32 a182, 0\n" \
  "v_accvgpr_write_b32 a183, 0\n" \
  "v_accvgpr_read_b32 v208, a184\n" \
  "v_accvgpr_read_b32 v209, a185\n" \
  "v_accvgpr_read_b32 v210, a186\n" \
  "v_accvgpr_read_b32 v211, a187\n" \
  "v_accvgpr_write_b32 a184, 0\n" \
  "v_accvgpr_write_b32 a185, 0\n" \
  "v_accvgpr_write_b32 a186, 0\n" \
  "v_accvgpr_write_b32 a187, 0\n" \
  "v_accvgpr_read_b32 v212, a188\n" \
  "v_accvgpr_read_b32 v213, a189\n" \
  "v_accvgpr_read_b32 v214, a190\n" \
  "v_accvgpr_read_b32 v215, a191\n" \
  "v_accvgpr_write_b32 a188, 0\n" \
  "v_accvgpr_write_b32 a189, 0\n" \
  "v_accvgpr_write_b32 a190, 0\n" \
  "v_accvgpr_write_b32 a191, 0\n" \
  "s_setpc_b64 s[92:93]\n" \
  "Lev0_7_%=:\n" \
  "s_nop 7\n" \
  "v_accvgpr_read_b32 v184, a192\n" \
  "v_accvgpr_read_b32 v185, a193\n" \
  "v_accvgpr_read_b32 v186, a194\n" \
  "v_accvgpr_read_b32 v187, a195\n" \
  "v_accvgpr_write_b32 a192, 0\n" \
  "v_accvgpr_write_b32 a193, 0\n" \
  "v_accvgpr_write_b32 a194, 0\n" \
  "v_accvgpr_write_b32 a195, 0\n" \
  "v_accvgpr_read_b32 v188, a196\n" \
  "v_accvgpr_read_b32 v189, a197\n" \
  "v_accvgpr_read_b32 v190, a198\n" \
  "v_accvgpr_read_b32 v191, a199\n" \
  "v_accvgpr_write_b32 a196, 0\n" \
  "v_accvgpr_write_b32 a197, 0\n" \
  "v_accvgpr_write_b32 a198, 0\n" \
  "v_accvgpr_write_b32 a199, 0\n" \
  "v_accvgpr_read_b32 v192, a200\n" \
  "v_accvgpr_read_b32 v193, a201\n" \
  "v_accvgpr_read_b32 v194, a202\n" \
  "v_accvgpr_read_b32 v195, a203\n" \
  "v_accvgpr_write_b32 a200, 0\n" \
  "v_accvgpr_write_b32 a201, 0\n" \
  "v_accvgpr_write_b32 a202, 0\n" \
  "v_accvgpr_write_b32 a203, 0\n" \
  "v_accvgpr_read_b32 v196, a204\n" \
  "v_accvgpr_read_b32 v197, a205\n" \
  "v_accvgpr_read_b32 v198, a206\n" \
  "v_accvgpr_read_b32 v199, a207\n" \
  "v_accvgpr_write_b32 a204, 0\n" \
  "v_accvgpr_write_b32 a205, 0\n" \
  "v_accvgpr_write_b32 a206, 0\n" \
  "v_accvgpr_write_b32 a207, 0\n" \
  "v_accvgpr_read_b32 v200, a208\n" \
  "v_accvgpr_read_b32 v201, a209\n" \
  "v_accvgpr_read_b32 v202, a210\n" \
  "v_accvgpr_read_b32 v203, a211\n" \
  "v_accvgpr_write_b32 a208, 0\n" \
  "v_accvgpr_write_b32 a209, 0\n" \
  "v_accvgpr_write_b32 a210, 0\n" \
  "v_accvgpr_write_b32 a211, 0\n" \
  "v_accvgpr_read_b32 v204, a212\n" \
  "v_accvgpr_read_b32 v205, a213\n" \
  "v_accvgpr_read_b32 v206, a214\n" \
  "v_accvgpr_read_b32 v207, a215\n" \
  "v_accvgpr_write_b32 a212, 0\n" \
  "v_accvgpr_write_b32 a213, 0\n" \
  "v_accvgpr_write_b32 a214, 0\n" \
  "v_accvgpr_write_b32 a215, 0\n" \
  "v_accvgpr_read_b32 v208, a216\n" \
  "v_accvgpr_read_b32 v209, a217\n" \
  "v_accvgpr_read_b32 v210, a218\n" \
  "v_accvgpr_read_b32 v211, a219\n" \
  "v_accvgpr_write_b32 a216, 0\n" \
  "v_accvgpr_write_b32 a217, 0\n" \
  "v_accvgpr_write_b32 a218, 0\n" \
  "v_accvgpr_write_b32 a219, 0\n" \
  "v_accvgpr_read_b32 v212, a220\n" \
  "v_accvgpr_read_b32 v213, a221\n" \
  "v_accvgpr_read_b32 v214, a222\n" \
  "v_accvgpr_read_b32 v215, a223\n" \
  "v_accvgpr_write_b32 a220, 0\n" \
  "v_accvgpr_write_b32 a221, 0\n" \
  "v_accvgpr_write_b32 a222, 0\n" \
  "v_accvgpr_write_b32 a223, 0\n" \
  "s_setpc_b64 s[92:93]\n" \
  "Lp1_%=:\n" \
  "s_swappc_b64 s[92:93], s[90:91]\n" \
  "s_branch Lq1_%=\n" \
  "Lp2_%=:\n" \
  "s_swappc_b64 s[92:93], s[90:91]\n" \
  "s_branch Lq2_%=\n" \
  "Lp3_%=:\n" \
  "s_swappc_b64 s[92:93], s[90:91]\n" \
  "s_branch Lq3_%=\n" \
  "Lp4_%=:\n" \
  "s_swappc_b64 s[92:93], s[90:91]\n" \
  "s_branch Lq4_%=\n" \
  "Lp5_%=:\n" \
  "v_mov_b32 v169, v170\n" \
  "v_mov_b32 v170, v171\n" \
  "v_mov_b32 v172, v173\n" \
  "v_mov_b32 v173, v174\n" \
  "v_mov_b32 v175, v176\n" \
  "v_mov_b32 v176, v177\n" \
  "v_mov_b32 v178, v179\n" \
  "v_mov_b32 v179, v180\n" \
  "v_mov_b32 v181, v182\n" \
  "v_mov_b32 v182, v183\n" \
  "s_mov_b32 s70, 0\n" \
  "s_nop 0\n" \
  "s_branch Lq5_%=\n" \
  "Lp6_%=:\n" \
  "s_swappc_b64 s[92:93], s[90:91]\n" \
  "s_branch Lq6_%=\n" \
  "Lp7_%=:\n" \
  "buffer_store_dwordx4 v[184:187], v222, s[52:55], s48 offen offset:0 sc0 sc1\n" \
  "s_branch Lq7_%=\n" \
  "Lp8_%=:\n" \
  "buffer_store_dwordx4 v[200:203], v222, s[52:55], s48 offen offset:64 sc0 sc1\n" \
  "s_branch Lq8_%=\n" \
  "Lp9_%=:\n" \
  "s_swappc_b64 s[92:93], s[90:91]\n" \
  "s_branch Lq9_%=\n" \
  "Lp10_%=:\n" \
  "buffer_store_dwordx4 v[188:191], v222, s[52:55], s49 offen offset:0 sc0 sc1\n" \
  "s_branch Lq10_%=\n" \
  "Lp11_%=:\n" \
  "buffer_store_dwordx4 v[204:207], v222, s[52:55], s49 offen offset:64 sc0 sc1\n" \
  "s_branch Lq11_%=\n" \
  "Lp12_%=:\n" \
  "v_mov_b32 v216, 1\n" \
  "s_mov_b64 exec, 1\n" \
  "global_atomic_add v224, v216, s[80:81]\n" \
  "s_mov_b64 exec, -1\n" \
  "s_nop 1\n" \
  "s_branch Lq12_%=\n" \
  "Lp13_%=:\n" \
  "s_swappc_b64 s[92:93], s[90:91]\n" \
  "s_branch Lq13_%=\n" \
  "Lp14_%=:\n" \
  "buffer_store_dwordx4 v[192:195], v222, s[52:55], s50 offen offset:0 sc0 sc1\n" \
  "s_branch Lq14_%=\n" \
  "Lp15_%=:\n" \
  "buffer_store_dwordx4 v[208:211], v222, s[52:55], s50 offen offset:64 sc0 sc1\n" \
  "s_branch Lq15_%=\n" \
  "Lp16_%=:\n" \
  "s_swappc_b64 s[92:93], s[90:91]\n" \
  "s_branch Lq16_%=\n" \
  "Lp17_%=:\n" \
  "buffer_store_dwordx4 v[196:199], v222, s[52:55], s51 offen offset:0 sc0 sc1\n" \
  "s_branch Lq17_%=\n" \
  "Lp18_%=:\n" \
  "buffer_store_dwordx4 v[212:215], v222, s[52:55], s51 offen offset:64 sc0 sc1\n" \
  "s_branch Lq18_%=\n" \
  "Lp19_%=:\n" \
  "v_mov_b32 v169, v170\n" \
  "v_mov_b32 v170, v171\n" \
  "v_mov_b32 v172, v173\n" \
  "v_mov_b32 v173, v174\n" \
  "v_mov_b32 v175, v176\n" \
  "v_mov_b32 v176, v177\n" \
  "v_mov_b32 v178, v179\n" \
  "v_mov_b32 v179, v180\n" \
  "v_mov_b32 v181, v182\n" \
  "v_mov_b32 v182, v183\n" \
  "s_mov_b32 s70, 0\n" \
  "s_nop 0\n" \
  "s_branch Lq19_%=\n" \
  "Lp20_%=:\n" \
  "s_swappc_b64 s[92:93], s[90:91]\n" \
  "s_branch Lq20_%=\n" \
  "Lp21_%=:\n" \
  "v_mov_b32 v169, v170\n" \
  "v_mov_b32 v170, v171\n" \
  "v_mov_b32 v172, v173\n" \
  "v_mov_b32 v173, v174\n" \
  "v_mov_b32 v175, v176\n" \
  "v_mov_b32 v176, v177\n" \
  "v_mov_b32 v178, v179\n" \
  "v_mov_b32 v179, v180\n" \
  "v_mov_b32 v181, v182\n" \
  "v_mov_b32 v182, v183\n" \
  "s_mov_b32 s70, 0\n" \
  "s_nop 0\n" \
  "s_branch Lq21_%=\n" \
  "Lp22_%=:\n" \
  "s_swappc_b64 s[92:93], s[90:91]\n" \
  "s_branch Lq22_%=\n" \
  "Lp23_%=:\n" \
  "v_mov_b32 v169, v170\n" \
  "v_mov_b32 v170, v171\n" \
  "v_mov_b32 v172, v173\n" \
  "v_mov_b32 v173, v174\n" \
  "v_mov_b32 v175, v176\n" \
  "v_mov_b32 v176, v177\n" \
  "v_mov_b32 v178, v179\n" \
  "v_mov_b32 v179, v180\n" \
  "v_mov_b32 v181, v182\n" \
  "v_mov_b32 v182, v183\n" \
  "s_mov_b32 s70, 0\n" \
  "s_nop 0\n" \
  "s_branch Lq23_%=\n" \
  "Lp24_%=:\n" \
  "s_swappc_b64 s[92:93], s[90:91]\n" \
  "s_branch Lq24_%=\n" \
  "Lp25_%=:\n" \
  "v_mov_b32 v169, v170\n" \
  "v_mov_b32 v170, v171\n" \
  "v_mov_b32 v172, v173\n" \
  "v_mov_b32 v173, v174\n" \
  "v_mov_b32 v175, v176\n" \
  "v_mov_b32 v176, v177\n" \
  "v_mov_b32 v178, v179\n" \
  "v_mov_b32 v179, v180\n" \
  "v_mov_b32 v181, v182\n" \
  "v_mov_b32 v182, v183\n" \
  "s_mov_b32 s70, 0\n" \
  "s_nop 0\n" \
  "s_branch Lq25_%=\n" \
  "Lp26_%=:\n" \
  "s_swappc_b64 s[92:93], s[90:91]\n" \
  "s_branch Lq26_%=\n" \
  "Lp27_%=:\n" \
  "v_mov_b32 v169, v170\n" \
  "v_mov_b32 v170, v171\n" \
  "v_mov_b32 v172, v173\n" \
  "v_mov_b32 v173, v174\n" \
  "v_mov_b32 v175, v176\n" \
  "v_mov_b32 v176, v177\n" \
  "v_mov_b32 v178, v179\n" \
  "v_mov_b32 v179, v180\n" \
  "v_mov_b32 v181, v182\n" \
  "v_mov_b32 v182, v183\n" \
  "s_mov_b32 s70, 0\n" \
  "s_nop 0\n" \
  "s_branch Lq27_%=\n" \
  "Lp28_%=:\n" \
  "s_swappc_b64 s[92:93], s[90:91]\n" \
  "s_branch Lq28_%=\n" \
  "Lp29_%=:\n" \
  "v_mov_b32 v169, v170\n" \
  "v_mov_b32 v170, v171\n" \
  "v_mov_b32 v172, v173\n" \
  "v_mov_b32 v173, v174\n" \
  "v_mov_b32 v175, v176\n" \
  "v_mov_b32 v176, v177\n" \
  "v_mov_b32 v178, v179\n" \
  "v_mov_b32 v179, v180\n" \
  "v_mov_b32 v181, v182\n" \
  "v_mov_b32 v182, v183\n" \
  "s_mov_b32 s70, 0\n" \
  "s_nop 0\n" \
  "s_branch Lq29_%=\n" \
  "Lp30_%=:\n" \
  "s_swappc_b64 s[92:93], s[90:91]\n" \
  "s_branch Lq30_%=\n" \
  "Lp31_%=:\n" \
  "v_mov_b32 v169, v170\n" \
  "v_mov_b32 v170, v171\n" \
  "v_mov_b32 v172, v173\n" \
  "v_mov_b32 v173, v174\n" \
  "v_mov_b32 v175, v176\n" \
  "v_mov_b32 v176, v177\n" \
  "v_mov_b32 v178, v179\n" \
  "v_mov_b32 v179, v180\n" \
  "v_mov_b32 v181, v182\n" \
  "v_mov_b32 v182, v183\n" \
  "s_mov_b32 s70, 0\n" \
  "s_nop 0\n" \
  "s_branch Lq31_%=\n" \
  "Lp32_%=:\n" \
  "s_swappc_b64 s[92:93], s[90:91]\n" \
  "s_branch Lq32_%=\n" \
  "Lp33_%=:\n" \
  "v_mov_b32 v169, v170\n" \
  "v_mov_b32 v170, v171\n" \
  "v_mov_b32 v172, v173\n" \
  "v_mov_b32 v173, v174\n" \
  "v_mov_b32 v175, v176\n" \
  "v_mov_b32 v176, v177\n" \
  "v_mov_b32 v178, v179\n" \
  "v_mov_b32 v179, v180\n" \
  "v_mov_b32 v181, v182\n" \
  "v_mov_b32 v182, v183\n" \
  "s_mov_b32 s70, 0\n" \
  "s_nop 0\n" \
  "s_branch Lq33_%=\n" \
  "Lp34_%=:\n" \
  "s_swappc_b64 s[92:93], s[90:91]\n" \
  "s_branch Lq34_%=\n" \
  "Lp35_%=:\n" \
  "s_swappc_b64 s[92:93], s[90:91]\n" \
  "s_branch Lq35_%=\n" \
  "Lp36_%=:\n" \
  "s_swappc_b64 s[92:93], s[90:91]\n" \
  "s_branch Lq36_%=\n" \
  "Lp37_%=:\n" \
  "s_swappc_b64 s[92:93], s[90:91]\n" \
  "s_branch Lq37_%=\n" \
  "Lp38_%=:\n" \
  "v_mov_b32 v169, v170\n" \
  "v_mov_b32 v170, v171\n" \
  "v_mov_b32 v172, v173\n" \
  "v_mov_b32 v173, v174\n" \
  "v_mov_b32 v175, v176\n" \
  "v_mov_b32 v176, v177\n" \
  "v_mov_b32 v178, v179\n" \
  "v_mov_b32 v179, v180\n" \
  "v_mov_b32 v181, v182\n" \
  "v_mov_b32 v182, v183\n" \
  "s_mov_b32 s70, 0\n" \
  "s_nop 0\n" \
  "s_branch Lq38_%=\n" \
  "Lp39_%=:\n" \
  "s_swappc_b64 s[92:93], s[90:91]\n" \
  "s_branch Lq39_%=\n" \
  "Lp40_%=:\n" \
  "s_swappc_b64 s[92:93], s[90:91]\n" \
  "s_branch Lq40_%=\n" \
  "Lp41_%=:\n" \
  "s_swappc_b64 s[92:93], s[90:91]\n" \
  "s_branch Lq41_%=\n" \
  "Lp42_%=:\n" \
  "s_swappc_b64 s[92:93], s[90:91]\n" \
  "s_branch Lq42_%=\n" \
  "Lp43_%=:\n" \
  "v_mov_b32 v169, v170\n" \
  "v_mov_b32 v170, v171\n" \
  "v_mov_b32 v172, v173\n" \
  "v_mov_b32 v173, v174\n" \
  "v_mov_b32 v175, v176\n" \
  "v_mov_b32 v176, v177\n" \
  "v_mov_b32 v178, v179\n" \
  "v_mov_b32 v179, v180\n" \
  "v_mov_b32 v181, v182\n" \
  "v_mov_b32 v182, v183\n" \
  "s_mov_b32 s70, 0\n" \
  "s_nop 0\n" \
  "s_branch Lq43_%=\n" \
  "Lp44_%=:\n" \
  "s_swappc_b64 s[92:93], s[90:91]\n" \
  "s_branch Lq44_%=\n" \
  "Lp45_%=:\n" \
  "s_swappc_b64 s[92:93], s[90:91]\n" \
  "s_branch Lq45_%=\n" \
  "Lp46_%=:\n" \
  "s_swappc_b64 s[92:93], s[90:91]\n" \
  "s_branch Lq46_%=\n" \
  "Lp47_%=:\n" \
  "s_swappc_b64 s[92:93], s[90:91]\n" \
  "s_branch Lq47_%=\n" \
  "Lp48_%=:\n" \
  "v_mov_b32 v169, v170\n" \
  "v_mov_b32 v170, v171\n" \
  "v_mov_b32 v172, v173\n" \
  "v_mov_b32 v173, v174\n" \
  "v_mov_b32 v175, v176\n" \
  "v_mov_b32 v176, v177\n" \
  "v_mov_b32 v178, v179\n" \
  "v_mov_b32 v179, v180\n" \
  "v_mov_b32 v181, v182\n" \
  "v_mov_b32 v182, v183\n" \
  "s_mov_b32 s70, 0\n" \
  "s_nop 0\n" \
  "s_branch Lq48_%=\n" \
  "Lp49_%=:\n" \
  "s_swappc_b64 s[92:93], s[90:91]\n" \
  "s_branch Lq49_%=\n" \
  "Lp50_%=:\n" \
  "s_swappc_b64 s[92:93], s[90:91]\n" \
  "s_branch Lq50_%=\n" \
  "Lp51_%=:\n" \
  "s_swappc_b64 s[92:93], s[90:91]\n" \
  "s_branch Lq51_%=\n" \
  "Lp52_%=:\n" \
  "s_swappc_b64 s[92:93], s[90:91]\n" \
  "s_branch Lq52_%=\n" \
  "Lp53_%=:\n" \
  "v_mov_b32 v169, v170\n" \
  "v_mov_b32 v170, v171\n" \
  "v_mov_b32 v172, v173\n" \
  "v_mov_b32 v173, v174\n" \
  "v_mov_b32 v175, v176\n" \
  "v_mov_b32 v176, v177\n" \
  "v_mov_b32 v178, v179\n" \
  "v_mov_b32 v179, v180\n" \
  "v_mov_b32 v181, v182\n" \
  "v_mov_b32 v182, v183\n" \
  "s_mov_b32 s70, 0\n" \
  "s_nop 0\n" \
  "s_branch Lq53_%=\n" \
  "Lp54_%=:\n" \
  "s_swappc_b64 s[92:93], s[90:91]\n" \
  "s_branch Lq54_%=\n" \
  "Lp55_%=:\n" \
  "s_swappc_b64 s[92:93], s[90:91]\n" \
  "s_branch Lq55_%=\n" \
  "Lp56_%=:\n" \
  "s_swappc_b64 s[92:93], s[90:91]\n" \
  "s_branch Lq56_%=\n" \
  "Lp57_%=:\n" \
  "s_swappc_b64 s[92:93], s[90:91]\n" \
  "s_branch Lq57_%=\n" \
  "Lp58_%=:\n" \
  "v_mov_b32 v169, v170\n" \
  "v_mov_b32 v170, v171\n" \
  "v_mov_b32 v172, v173\n" \
  "v_mov_b32 v173, v174\n" \
  "v_mov_b32 v175, v176\n" \
  "v_mov_b32 v176, v177\n" \
  "v_mov_b32 v178, v179\n" \
  "v_mov_b32 v179, v180\n" \
  "v_mov_b32 v181, v182\n" \
  "v_mov_b32 v182, v183\n" \
  "s_mov_b32 s70, 0\n" \
  "s_nop 0\n" \
  "s_branch Lq58_%=\n" \
  "Lp59_%=:\n" \
  "s_swappc_b64 s[92:93], s[90:91]\n" \
  "s_branch Lq59_%=\n" \
  "Lp60_%=:\n" \
  "s_swappc_b64 s[92:93], s[90:91]\n" \
  "s_branch Lq60_%=\n" \
  "Lp61_%=:\n" \
  "s_swappc_b64 s[92:93], s[90:91]\n" \
  "s_branch Lq61_%=\n" \
  "Lp62_%=:\n" \
  "s_swappc_b64 s[92:93], s[90:91]\n" \
  "s_branch Lq62_%=\n" \
  "Lp63_%=:\n" \
  "v_mov_b32 v169, v170\n" \
  "v_mov_b32 v170, v171\n" \
  "v_mov_b32 v172, v173\n" \
  "v_mov_b32 v173, v174\n" \
  "v_mov_b32 v175, v176\n" \
  "v_mov_b32 v176, v177\n" \
  "v_mov_b32 v178, v179\n" \
  "v_mov_b32 v179, v180\n" \
  "v_mov_b32 v181, v182\n" \
  "v_mov_b32 v182, v183\n" \
  "s_mov_b32 s70, 0\n" \
  "s_nop 0\n" \
  "s_branch Lq63_%=\n" \
  "Lp64_%=:\n" \
  "s_swappc_b64 s[92:93], s[90:91]\n" \
  "s_branch Lq64_%=\n" \
  "Lp65_%=:\n" \
  "s_swappc_b64 s[92:93], s[90:91]\n" \
  "s_branch Lq65_%=\n" \
  "Lp66_%=:\n" \
  "s_swappc_b64 s[92:93], s[90:91]\n" \
  "s_branch Lq66_%=\n" \
  "Lp67_%=:\n" \
  "s_swappc_b64 s[92:93], s[90:91]\n" \
  "s_branch Lq67_%=\n" \
  "Lp68_%=:\n" \
  "v_mov_b32 v169, v170\n" \
  "v_mov_b32 v170, v171\n" \
  "v_mov_b32 v172, v173\n" \
  "v_mov_b32 v173, v174\n" \
  "v_mov_b32 v175, v176\n" \
  "v_mov_b32 v176, v177\n" \
  "v_mov_b32 v178, v179\n" \
  "v_mov_b32 v179, v180\n" \
  "v_mov_b32 v181, v182\n" \
  "v_mov_b32 v182, v183\n" \
  "s_mov_b32 s70, 0\n" \
  "s_nop 0\n" \
  "s_branch Lq68_%=\n" \
  "Lp69_%=:\n" \
  "s_swappc_b64 s[92:93], s[90:91]\n" \
  "s_branch Lq69_%=\n" \
  "Lp70_%=:\n" \
  "s_swappc_b64 s[92:93], s[90:91]\n" \
  "s_branch Lq70_%=\n" \
  "Lp71_%=:\n" \
  "s_swappc_b64 s[92:93], s[90:91]\n" \
  "s_branch Lq71_%=\n" \
  "Lp72_%=:\n" \
  "v_mov_b32 v169, v170\n" \
  "v_mov_b32 v170, v171\n" \
  "v_mov_b32 v172, v173\n" \
  "v_mov_b32 v173, v174\n" \
  "v_mov_b32 v175, v176\n" \
  "v_mov_b32 v176, v177\n" \
  "v_mov_b32 v178, v179\n" \
  "v_mov_b32 v179, v180\n" \
  "v_mov_b32 v181, v182\n" \
  "v_mov_b32 v182, v183\n" \
  "s_mov_b32 s70, 0\n" \
  "s_nop 0\n" \
  "s_branch Lq72_%=\n" \
  "Lp73_%=:\n" \
  "s_swappc_b64 s[92:93], s[90:91]\n" \
  "s_branch Lq73_%=\n" \
  "Lp74_%=:\n" \
  "buffer_store_dwordx4 v[184:187], v222, s[52:55], s48 offen offset:0 sc0 sc1\n" \
  "s_branch Lq74_%=\n" \
  "Lp75_%=:\n" \
  "buffer_store_dwordx4 v[200:203], v222, s[52:55], s48 offen offset:64 sc0 sc1\n" \
  "s_branch Lq75_%=\n" \
  "Lp76_%=:\n" \
  "v_mov_b32 v169, v170\n" \
  "v_mov_b32 v170, v171\n" \
  "v_mov_b32 v172, v173\n" \
  "v_mov_b32 v173, v174\n" \
  "v_mov_b32 v175, v176\n" \
  "v_mov_b32 v176, v177\n" \
  "v_mov_b32 v178, v179\n" \
  "v_mov_b32 v179, v180\n" \
  "v_mov_b32 v181, v182\n" \
  "v_mov_b32 v182, v183\n" \
  "s_mov_b32 s70, 0\n" \
  "s_nop 0\n" \
  "s_branch Lq76_%=\n" \
  "Lp77_%=:\n" \
  "s_swappc_b64 s[92:93], s[90:91]\n" \
  "s_branch Lq77_%=\n" \
  "Lp78_%=:\n" \
  "buffer_store_dwordx4 v[184:187], v222, s[52:55], s48 offen offset:0 sc0 sc1\n" \
  "s_branch Lq78_%=\n" \
  "Lp79_%=:\n" \
  "buffer_store_dwordx4 v[200:203], v222, s[52:55], s48 offen offset:64 sc0 sc1\n" \
  "s_branch Lq79_%=\n" \
  "Lp80_%=:\n" \
  "v_mov_b32 v169, v170\n" \
  "v_mov_b32 v170, v171\n" \
  "v_mov_b32 v172, v173\n" \
  "v_mov_b32 v173, v174\n" \
  "v_mov_b32 v175, v176\n" \
  "v_mov_b32 v176, v177\n" \
  "v_mov_b32 v178, v179\n" \
  "v_mov_b32 v179, v180\n" \
  "v_mov_b32 v181, v182\n" \
  "v_mov_b32 v182, v183\n" \
  "s_mov_b32 s70, 0\n" \
  "s_nop 0\n" \
  "s_branch Lq80_%=\n" \
  "Lp81_%=:\n" \
  "s_swappc_b64 s[92:93], s[90:91]\n" \
  "s_branch Lq81_%=\n" \
  "Lp82_%=:\n" \
  "buffer_store_dwordx4 v[184:187], v222, s[52:55], s48 offen offset:0 sc0 sc1\n" \
  "s_branch Lq82_%=\n" \
  "Lp83_%=:\n" \
  "buffer_store_dwordx4 v[200:203], v222, s[52:55], s48 offen offset:64 sc0 sc1\n" \
  "s_branch Lq83_%=\n" \
  "Lp84_%=:\n" \
  "v_mov_b32 v169, v170\n" \
  "v_mov_b32 v170, v171\n" \
  "v_mov_b32 v172, v173\n" \
  "v_mov_b32 v173, v174\n" \
  "v_mov_b32 v175, v176\n" \
  "v_mov_b32 v176, v177\n" \
  "v_mov_b32 v178, v179\n" \
  "v_mov_b32 v179, v180\n" \
  "v_mov_b32 v181, v182\n" \
  "v_mov_b32 v182, v183\n" \
  "s_mov_b32 s70, 0\n" \
  "s_nop 0\n" \
  "s_branch Lq84_%=\n" \
  "Lp85_%=:\n" \
  "s_swappc_b64 s[92:93], s[90:91]\n" \
  "s_branch Lq85_%=\n" \
  "Lp86_%=:\n" \
  "buffer_store_dwordx4 v[184:187], v222, s[52:55], s48 offen offset:0 sc0 sc1\n" \
  "s_branch Lq86_%=\n" \
  "Lp87_%=:\n" \
  "buffer_store_dwordx4 v[200:203], v222, s[52:55], s48 offen offset:64 sc0 sc1\n" \
  "s_branch Lq87_%=\n" \
  "Lp88_%=:\n" \
  "s_swappc_b64 s[92:93], s[90:91]\n" \
  "s_branch Lq88_%=\n" \
  "Lp89_%=:\n" \
  "buffer_store_dwordx4 v[184:187], v222, s[52:55], s48 offen offset:0 sc0 sc1\n" \
  "s_branch Lq89_%=\n" \
  "Lp90_%=:\n" \
  "buffer_store_dwordx4 v[200:203], v222, s[52:55], s48 offen offset:64 sc0 sc1\n" \
  "s_branch Lq90_%=\n" \
  "Lp91_%=:\n" \
  "buffer_store_dwordx4 v[184:187], v222, s[52:55], s48 offen offset:0 sc0 sc1\n" \
  "s_branch Lq91_%=\n" \
  "Lp92_%=:\n" \
  "buffer_store_dwordx4 v[200:203], v222, s[52:55], s48 offen offset:64 sc0 sc1\n" \
  "s_branch Lq92_%=\n" \
  "Lp93_%=:\n" \
  "v_mov_b32 v169, v170\n" \
  "v_mov_b32 v170, v171\n" \
  "v_mov_b32 v172, v173\n" \
  "v_mov_b32 v173, v174\n" \
  "v_mov_b32 v175, v176\n" \
  "v_mov_b32 v176, v177\n" \
  "v_mov_b32 v178, v179\n" \
  "v_mov_b32 v179, v180\n" \
  "v_mov_b32 v181, v182\n" \
  "v_mov_b32 v182, v183\n" \
  "s_mov_b32 s70, 0\n" \
  "s_nop 0\n" \
  "s_branch Lq93_%=\n" \
  "Lp94_%=:\n" \
  "s_swappc_b64 s[92:93], s[90:91]\n" \
  "s_branch Lq94_%=\n" \
  "Lp95_%=:\n" \
  "buffer_store_dwordx4 v[184:187], v222, s[52:55], s48 offen offset:0 sc0 sc1\n" \
  "s_branch Lq95_%=\n" \
  "Lp96_%=:\n" \
  "buffer_store_dwordx4 v[200:203], v222, s[52:55], s48 offen offset:64 sc0 sc1\n" \
  "s_branch Lq96_%=\n" \
  "Lp97_%=:\n" \
  "v_mov_b32 v169, v170\n" \
  "v_mov_b32 v170, v171\n" \
  "v_mov_b32 v172, v173\n" \
  "v_mov_b32 v173, v174\n" \
  "v_mov_b32 v175, v176\n" \
  "v_mov_b32 v176, v177\n" \
  "v_mov_b32 v178, v179\n" \
  "v_mov_b32 v179, v180\n" \
  "v_mov_b32 v181, v182\n" \
  "v_mov_b32 v182, v183\n" \
  "s_mov_b32 s70, 0\n" \
  "s_nop 0\n" \
  "s_branch Lq97_%=\n" \
  "Lp98_%=:\n" \
  "s_swappc_b64 s[92:93], s[90:91]\n" \
  "s_branch Lq98_%=\n" \
  "Lp99_%=:\n" \
  "buffer_store_dwordx4 v[184:187], v222, s[52:55], s48 offen offset:0 sc0 sc1\n" \
  "s_branch Lq99_%=\n" \
  "Lp100_%=:\n" \
  "buffer_store_dwordx4 v[200:203], v222, s[52:55], s48 offen offset:64 sc0 sc1\n" \
  "s_branch Lq100_%=\n" \
  "Lp101_%=:\n" \
  "v_mov_b32 v169, v170\n" \
  "v_mov_b32 v170, v171\n" \
  "v_mov_b32 v172, v173\n" \
  "v_mov_b32 v173, v174\n" \
  "v_mov_b32 v175, v176\n" \
  "v_mov_b32 v176, v177\n" \
  "v_mov_b32 v178, v179\n" \
  "v_mov_b32 v179, v180\n" \
  "v_mov_b32 v181, v182\n" \
  "v_mov_b32 v182, v183\n" \
  "s_mov_b32 s70, 0\n" \
  "s_nop 0\n" \
  "s_branch Lq101_%=\n" \
  "Lp102_%=:\n" \
  "s_swappc_b64 s[92:93], s[90:91]\n" \
  "s_branch Lq102_%=\n" \
  "Lp103_%=:\n" \
  "buffer_store_dwordx4 v[184:187], v222, s[52:55], s48 offen offset:0 sc0 sc1\n" \
  "s_branch Lq103_%=\n" \
  "Lp104_%=:\n" \
  "buffer_store_dwordx4 v[200:203], v222, s[52:55], s48 offen offset:64 sc0 sc1\n" \
  "s_branch Lq104_%=\n" \
  "Lp105_%=:\n" \
  "v_mov_b32 v169, v170\n" \
  "v_mov_b32 v170, v171\n" \
  "v_mov_b32 v172, v173\n" \
  "v_mov_b32 v173, v174\n" \
  "v_mov_b32 v175, v176\n" \
  "v_mov_b32 v176, v177\n" \
  "v_mov_b32 v178, v179\n" \
  "v_mov_b32 v179, v180\n" \
  "v_mov_b32 v181, v182\n" \
  "v_mov_b32 v182, v183\n" \
  "s_mov_b32 s70, 0\n" \
  "s_nop 0\n" \
  "s_branch Lq105_%=\n" \
  "Lp106_%=:\n" \
  "s_swappc_b64 s[92:93], s[90:91]\n" \
  "s_branch Lq106_%=\n" \
  "Lp107_%=:\n" \
  "buffer_store_dwordx4 v[184:187], v222, s[52:55], s48 offen offset:0 sc0 sc1\n" \
  "s_branch Lq107_%=\n" \
  "Lp108_%=:\n" \
  "buffer_store_dwordx4 v[200:203], v222, s[52:55], s48 offen offset:64 sc0 sc1\n" \
  "s_branch Lq108_%=\n" \
  "Lp109_%=:\n" \
  "s_swappc_b64 s[92:93], s[90:91]\n" \
  "s_branch Lq109_%=\n" \
  "Lp110_%=:\n" \
  "buffer_store_dwordx4 v[184:187], v222, s[52:55], s48 offen offset:0 sc0 sc1\n" \
  "s_branch Lq110_%=\n" \
  "Lp111_%=:\n" \
  "buffer_store_dwordx4 v[200:203], v222, s[52:55], s48 offen offset:64 sc0 sc1\n" \
  "s_branch Lq111_%=\n" \
  "Lp112_%=:\n" \
  "buffer_store_dwordx4 v[184:187], v222, s[52:55], s48 offen offset:0 sc0 sc1\n" \
  "s_branch Lq112_%=\n" \
  "Lp113_%=:\n" \
  "buffer_store_dwordx4 v[200:203], v222, s[52:55], s48 offen offset:64 sc0 sc1\n" \
  "s_branch Lq113_%=\n" \
  "Lp114_%=:\n" \
  "v_mov_b32 v169, v170\n" \
  "v_mov_b32 v170, v171\n" \
  "v_mov_b32 v172, v173\n" \
  "v_mov_b32 v173, v174\n" \
  "v_mov_b32 v175, v176\n" \
  "v_mov_b32 v176, v177\n" \
  "v_mov_b32 v178, v179\n" \
  "v_mov_b32 v179, v180\n" \
  "v_mov_b32 v181, v182\n" \
  "v_mov_b32 v182, v183\n" \
  "s_mov_b32 s70, 0\n" \
  "s_nop 0\n" \
  "s_branch Lq114_%=\n" \
  "Lp115_%=:\n" \
  "v_mov_b32 v169, v170\n" \
  "v_mov_b32 v170, v171\n" \
  "v_mov_b32 v172, v173\n" \
  "v_mov_b32 v173, v174\n" \
  "v_mov_b32 v175, v176\n" \
  "v_mov_b32 v176, v177\n" \
  "v_mov_b32 v178, v179\n" \
  "v_mov_b32 v179, v180\n" \
  "v_mov_b32 v181, v182\n" \
  "v_mov_b32 v182, v183\n" \
  "s_mov_b32 s70, 0\n" \
  "s_nop 0\n" \
  "s_branch Lq115_%=\n" \
  "Lp116_%=:\n" \
  "s_swappc_b64 s[92:93], s[90:91]\n" \
  "s_branch Lq116_%=\n" \
  "Lp117_%=:\n" \
  "v_mov_b32 v169, v170\n" \
  "v_mov_b32 v170, v171\n" \
  "v_mov_b32 v172, v173\n" \
  "v_mov_b32 v173, v174\n" \
  "v_mov_b32 v175, v176\n" \
  "v_mov_b32 v176, v177\n" \
  "v_mov_b32 v178, v179\n" \
  "v_mov_b32 v179, v180\n" \
  "v_mov_b32 v181, v182\n" \
  "v_mov_b32 v182, v183\n" \
  "s_mov_b32 s70, 0\n" \
  "s_nop 0\n" \
  "s_branch Lq117_%=\n" \
  "Lp118_%=:\n" \
  "s_swappc_b64 s[92:93], s[90:91]\n" \
  "s_branch Lq118_%=\n" \
  "Lp119_%=:\n" \
  "buffer_store_dwordx4 v[184:187], v222, s[52:55], s48 offen offset:0 sc0 sc1\n" \
  "s_branch Lq119_%=\n" \
  "Lp120_%=:\n" \
  "buffer_store_dwordx4 v[200:203], v222, s[52:55], s48 offen offset:64 sc0 sc1\n" \
  "s_branch Lq120_%=\n" \
  "Lp121_%=:\n" \
  "v_mov_b32 v169, v170\n" \
  "v_mov_b32 v170, v171\n" \
  "v_mov_b32 v172, v173\n" \
  "v_mov_b32 v173, v174\n" \
  "v_mov_b32 v175, v176\n" \
  "v_mov_b32 v176, v177\n" \
  "v_mov_b32 v178, v179\n" \
  "v_mov_b32 v179, v180\n" \
  "v_mov_b32 v181, v182\n" \
  "v_mov_b32 v182, v183\n" \
  "s_mov_b32 s70, 0\n" \
  "s_nop 0\n" \
  "s_branch Lq121_%=\n" \
  "Lp122_%=:\n" \
  "v_mov_b32 v169, v170\n" \
  "v_mov_b32 v170, v171\n" \
  "v_mov_b32 v172, v173\n" \
  "v_mov_b32 v173, v174\n" \
  "v_mov_b32 v175, v176\n" \
  "v_mov_b32 v176, v177\n" \
  "v_mov_b32 v178, v179\n" \
  "v_mov_b32 v179, v180\n" \
  "v_mov_b32 v181, v182\n" \
  "v_mov_b32 v182, v183\n" \
  "s_mov_b32 s70, 0\n" \
  "s_nop 0\n" \
  "s_branch Lq122_%=\n" \
  "Lp123_%=:\n" \
  "s_swappc_b64 s[92:93], s[90:91]\n" \
  "s_branch Lq123_%=\n" \
  "Lp124_%=:\n" \
  "v_mov_b32 v169, v170\n" \
  "v_mov_b32 v170, v171\n" \
  "v_mov_b32 v172, v173\n" \
  "v_mov_b32 v173, v174\n" \
  "v_mov_b32 v175, v176\n" \
  "v_mov_b32 v176, v177\n" \
  "v_mov_b32 v178, v179\n" \
  "v_mov_b32 v179, v180\n" \
  "v_mov_b32 v181, v182\n" \
  "v_mov_b32 v182, v183\n" \
  "s_mov_b32 s70, 0\n" \
  "s_nop 0\n" \
  "s_branch Lq124_%=\n" \
  "Lp125_%=:\n" \
  "s_swappc_b64 s[92:93], s[90:91]\n" \
  "s_branch Lq125_%=\n" \
  "Lp126_%=:\n" \
  "buffer_store_dwordx4 v[184:187], v222, s[52:55], s48 offen offset:0 sc0 sc1\n" \
  "s_branch Lq126_%=\n" \
  "Lp127_%=:\n" \
  "buffer_store_dwordx4 v[200:203], v222, s[52:55], s48 offen offset:64 sc0 sc1\n" \
  "s_branch Lq127_%=\n" \
  "Lp128_%=:\n" \
  "s_swappc_b64 s[92:93], s[90:91]\n" \
  "s_branch Lq128_%=\n" \
  "Lp129_%=:\n" \
  "buffer_store_dwordx4 v[188:191], v222, s[52:55], s49 offen offset:0 sc0 sc1\n" \
  "s_branch Lq129_%=\n" \
  "Lp130_%=:\n" \
  "buffer_store_dwordx4 v[204:207], v222, s[52:55], s49 offen offset:64 sc0 sc1\n" \
  "s_branch Lq130_%=\n" \
  "Lp131_%=:\n" \
  "s_swappc_b64 s[92:93], s[90:91]\n" \
  "s_branch Lq131_%=\n" \
  "Lp132_%=:\n" \
  "buffer_store_dwordx4 v[188:191], v222, s[52:55], s49 offen offset:0 sc0 sc1\n" \
  "s_branch Lq132_%=\n" \
  "Lp133_%=:\n" \
  "buffer_store_dwordx4 v[204:207], v222, s[52:55], s49 offen offset:64 sc0 sc1\n" \
  "s_branch Lq133_%=\n" \
  "Lp134_%=:\n" \
  "s_swappc_b64 s[92:93], s[90:91]\n" \
  "s_branch Lq134_%=\n" \
  "Lp135_%=:\n" \
  "buffer_store_dwordx4 v[188:191], v222, s[52:55], s49 offen offset:0 sc0 sc1\n" \
  "s_branch Lq135_%=\n" \
  "Lp136_%=:\n" \
  "buffer_store_dwordx4 v[204:207], v222, s[52:55], s49 offen offset:64 sc0 sc1\n" \
  "s_branch Lq136_%=\n" \
  "Lp137_%=:\n" \
  "s_swappc_b64 s[92:93], s[90:91]\n" \
  "s_branch Lq137_%=\n" \
  "Lp138_%=:\n" \
  "buffer_store_dwordx4 v[188:191], v222, s[52:55], s49 offen offset:0 sc0 sc1\n" \
  "s_branch Lq138_%=\n" \
  "Lp139_%=:\n" \
  "buffer_store_dwordx4 v[204:207], v222, s[52:55], s49 offen offset:64 sc0 sc1\n" \
  "s_branch Lq139_%=\n" \
  "Lp140_%=:\n" \
  "s_swappc_b64 s[92:93], s[90:91]\n" \
  "s_branch Lq140_%=\n" \
  "Lp141_%=:\n" \
  "buffer_store_dwordx4 v[188:191], v222, s[52:55], s49 offen offset:0 sc0 sc1\n" \
  "s_branch Lq141_%=\n" \
  "Lp142_%=:\n" \
  "buffer_store_dwordx4 v[204:207], v222, s[52:55], s49 offen offset:64 sc0 sc1\n" \
  "s_branch Lq142_%=\n" \
  "Lp143_%=:\n" \
  "buffer_store_dwordx4 v[204:207], v222, s[52:55], s49 offen offset:64 sc0 sc1\n" \
  "s_branch Lq143_%=\n" \
  "Lp144_%=:\n" \
  "s_swappc_b64 s[92:93], s[90:91]\n" \
  "s_branch Lq144_%=\n" \
  "Lp145_%=:\n" \
  "buffer_store_dwordx4 v[188:191], v222, s[52:55], s49 offen offset:0 sc0 sc1\n" \
  "s_branch Lq145_%=\n" \
  "Lp146_%=:\n" \
  "buffer_store_dwordx4 v[204:207], v222, s[52:55], s49 offen offset:64 sc0 sc1\n" \
  "s_branch Lq146_%=\n" \
  "Lp147_%=:\n" \
  "s_swappc_b64 s[92:93], s[90:91]\n" \
  "s_branch Lq147_%=\n" \
  "Lp148_%=:\n" \
  "buffer_store_dwordx4 v[188:191], v222, s[52:55], s49 offen offset:0 sc0 sc1\n" \
  "s_branch Lq148_%=\n" \
  "Lp149_%=:\n" \
  "buffer_store_dwordx4 v[204:207], v222, s[52:55], s49 offen offset:64 sc0 sc1\n" \
  "s_branch Lq149_%=\n" \
  "Lp150_%=:\n" \
  "s_swappc_b64 s[92:93], s[90:91]\n" \
  "s_branch Lq150_%=\n" \
  "Lp151_%=:\n" \
  "buffer_store_dwordx4 v[188:191], v222, s[52:55], s49 offen offset:0 sc0 sc1\n" \
  "s_branch Lq151_%=\n" \
  "Lp152_%=:\n" \
  "buffer_store_dwordx4 v[204:207], v222, s[52:55], s49 offen offset:64 sc0 sc1\n" \
  "s_branch Lq152_%=\n" \
  "Lp153_%=:\n" \
  "s_swappc_b64 s[92:93], s[90:91]\n" \
  "s_branch Lq153_%=\n" \
  "Lp154_%=:\n" \
  "buffer_store_dwordx4 v[188:191], v222, s[52:55], s49 offen offset:0 sc0 sc1\n" \
  "s_branch Lq154_%=\n" \
  "Lp155_%=:\n" \
  "buffer_store_dwordx4 v[204:207], v222, s[52:55], s49 offen offset:64 sc0 sc1\n" \
  "s_branch Lq155_%=\n" \
  "Lp156_%=:\n" \
  "s_swappc_b64 s[92:93], s[90:91]\n" \
  "s_branch Lq156_%=\n" \
  "Lp157_%=:\n" \
  "buffer_store_dwordx4 v[188:191], v222, s[52:55], s49 offen offset:0 sc0 sc1\n" \
  "s_branch Lq157_%=\n" \
  "Lp158_%=:\n" \
  "buffer_store_dwordx4 v[204:207], v222, s[52:55], s49 offen offset:64 sc0 sc1\n" \
  "s_branch Lq158_%=\n" \
  "Lp159_%=:\n" \
  "buffer_store_dwordx4 v[204:207], v222, s[52:55], s49 offen offset:64 sc0 sc1\n" \
  "s_branch Lq159_%=\n" \
  "Lp160_%=:\n" \
  "s_swappc_b64 s[92:93], s[90:91]\n" \
  "s_branch Lq160_%=\n" \
  "Lp161_%=:\n" \
  "buffer_store_dwordx4 v[188:191], v222, s[52:55], s49 offen offset:0 sc0 sc1\n" \
  "s_branch Lq161_%=\n" \
  "Lp162_%=:\n" \
  "s_swappc_b64 s[92:93], s[90:91]\n" \
  "s_branch Lq162_%=\n" \
  "Lp163_%=:\n" \
  "buffer_store_dwordx4 v[188:191], v222, s[52:55], s49 offen offset:0 sc0 sc1\n" \
  "s_branch Lq163_%=\n" \
  "Lp164_%=:\n" \
  "buffer_store_dwordx4 v[204:207], v222, s[52:55], s49 offen offset:64 sc0 sc1\n" \
  "s_branch Lq164_%=\n" \
  "Lp165_%=:\n" \
  "s_swappc_b64 s[92:93], s[90:91]\n" \
  "s_branch Lq165_%=\n" \
  "Lp166_%=:\n" \
  "buffer_store_dwordx4 v[188:191], v222, s[52:55], s49 offen offset:0 sc0 sc1\n" \
  "s_branch Lq166_%=\n" \
  "Lp167_%=:\n" \
  "s_swappc_b64 s[92:93], s[90:91]\n" \
  "s_branch Lq167_%=\n" \
  "Lp168_%=:\n" \
  "buffer_store_dwordx4 v[188:191], v222, s[52:55], s49 offen offset:0 sc0 sc1\n" \
  "s_branch Lq168_%=\n" \
  "Lp169_%=:\n" \
  "buffer_store_dwordx4 v[204:207], v222, s[52:55], s49 offen offset:64 sc0 sc1\n" \
  "s_branch Lq169_%=\n" \
  "Lp170_%=:\n" \
  "v_mov_b32 v216, 1\n" \
  "s_mov_b64 exec, 1\n" \
  "global_atomic_add v224, v216, s[80:81]\n" \
  "s_mov_b64 exec, -1\n" \
  "s_nop 1\n" \
  "s_branch Lq170_%=\n" \
  "Lp171_%=:\n" \
  "s_swappc_b64 s[92:93], s[90:91]\n" \
  "s_branch Lq171_%=\n" \
  "Lp172_%=:\n" \
  "buffer_store_dwordx4 v[192:195], v222, s[52:55], s50 offen offset:0 sc0 sc1\n" \
  "s_branch Lq172_%=\n" \
  "Lp173_%=:\n" \
  "buffer_store_dwordx4 v[208:211], v222, s[52:55], s50 offen offset:64 sc0 sc1\n" \
  "s_branch Lq173_%=\n" \
  "Lp174_%=:\n" \
  "v_mov_b32 v216, 1\n" \
  "s_mov_b64 exec, 1\n" \
  "global_atomic_add v224, v216, s[80:81]\n" \
  "s_mov_b64 exec, -1\n" \
  "s_nop 1\n" \
  "s_branch Lq174_%=\n" \
  "Lp175_%=:\n" \
  "s_swappc_b64 s[92:93], s[90:91]\n" \
  "s_branch Lq175_%=\n" \
  "Lp176_%=:\n" \
  "buffer_store_dwordx4 v[192:195], v222, s[52:55], s50 offen offset:0 sc0 sc1\n" \
  "s_branch Lq176_%=\n" \
  "Lp177_%=:\n" \
  "buffer_store_dwordx4 v[208:211], v222, s[52:55], s50 offen offset:64 sc0 sc1\n" \
  "s_branch Lq177_%=\n" \
  "Lp178_%=:\n" \
  "v_mov_b32 v216, 1\n" \
  "s_mov_b64 exec, 1\n" \
  "global_atomic_add v224, v216, s[80:81]\n" \
  "s_mov_b64 exec, -1\n" \
  "s_nop 1\n" \
  "s_branch Lq178_%=\n" \
  "Lp179_%=:\n" \
  "s_swappc_b64 s[92:93], s[90:91]\n" \
  "s_branch Lq179_%=\n" \
  "Lp180_%=:\n" \
  "buffer_store_dwordx4 v[192:195], v222, s[52:55], s50 offen offset:0 sc0 sc1\n" \
  "s_branch Lq180_%=\n" \
  "Lp181_%=:\n" \
  "buffer_store_dwordx4 v[208:211], v222, s[52:55], s50 offen offset:64 sc0 sc1\n" \
  "s_branch Lq181_%=\n" \
  "Lp182_%=:\n" \
  "v_mov_b32 v216, 1\n" \
  "s_mov_b64 exec, 1\n" \
  "global_atomic_add v224, v216, s[80:81]\n" \
  "s_mov_b64 exec, -1\n" \
  "s_nop 1\n" \
  "s_branch Lq182_%=\n" \
  "Lp183_%=:\n" \
  "s_swappc_b64 s[92:93], s[90:91]\n" \
  "s_branch Lq183_%=\n" \
  "Lp184_%=:\n" \
  "buffer_store_dwordx4 v[192:195], v222, s[52:55], s50 offen offset:0 sc0 sc1\n" \
  "s_branch Lq184_%=\n" \
  "Lp185_%=:\n" \
  "buffer_store_dwordx4 v[208:211], v222, s[52:55], s50 offen offset:64 sc0 sc1\n" \
  "s_branch Lq185_%=\n" \
  "Lp186_%=:\n" \
  "v_mov_b32 v216, 1\n" \
  "s_mov_b64 exec, 1\n" \
  "global_atomic_add v224, v216, s[80:81]\n" \
  "s_mov_b64 exec, -1\n" \
  "s_nop 1\n" \
  "s_branch Lq186_%=\n" \
  "Lp187_%=:\n" \
  "s_swappc_b64 s[92:93], s[90:91]\n" \
  "s_branch Lq187_%=\n" \
  "Lp188_%=:\n" \
  "buffer_store_dwordx4 v[192:195], v222, s[52:55], s50 offen offset:0 sc0 sc1\n" \
  "s_branch Lq188_%=\n" \
  "Lp189_%=:\n" \
  "buffer_store_dwordx4 v[208:211], v222, s[52:55], s50 offen offset:64 sc0 sc1\n" \
  "s_branch Lq189_%=\n" \
  "Lp190_%=:\n" \
  "buffer_store_dwordx4 v[208:211], v222, s[52:55], s50 offen offset:64 sc0 sc1\n" \
  "s_branch Lq190_%=\n" \
  "Lp191_%=:\n" \
  "v_mov_b32 v216, 1\n" \
  "s_mov_b64 exec, 1\n" \
  "global_atomic_add v224, v216, s[80:81]\n" \
  "s_mov_b64 exec, -1\n" \
  "s_nop 1\n" \
  "s_branch Lq191_%=\n" \
  "Lp192_%=:\n" \
  "s_swappc_b64 s[92:93], s[90:91]\n" \
  "s_branch Lq192_%=\n" \
  "Lp193_%=:\n" \
  "buffer_store_dwordx4 v[192:195], v222, s[52:55], s50 offen offset:0 sc0 sc1\n" \
  "s_branch Lq193_%=\n" \
  "Lp194_%=:\n" \
  "buffer_store_dwordx4 v[208:211], v222, s[52:55], s50 offen offset:64 sc0 sc1\n" \
  "s_branch Lq194_%=\n" \
  "Lp195_%=:\n" \
  "v_mov_b32 v216, 1\n" \
  "s_mov_b64 exec, 1\n" \
  "global_atomic_add v224, v216, s[80:81]\n" \
  "s_mov_b64 exec, -1\n" \
  "s_nop 1\n" \
  "s_branch Lq195_%=\n" \
  "Lp196_%=:\n" \
  "s_swappc_b64 s[92:93], s[90:91]\n" \
  "s_branch Lq196_%=\n" \
  "Lp197_%=:\n" \
  "buffer_store_dwordx4 v[192:195], v222, s[52:55], s50 offen offset:0 sc0 sc1\n" \
  "s_branch Lq197_%=\n" \
  "Lp198_%=:\n" \
  "buffer_store_dwordx4 v[208:211], v222, s[52:55], s50 offen offset:64 sc0 sc1\n" \
  "s_branch Lq198_%=\n" \
  "Lp199_%=:\n" \
  "v_mov_b32 v216, 1\n" \
  "s_mov_b64 exec, 1\n" \
  "global_atomic_add v224, v216, s[80:81]\n" \
  "s_mov_b64 exec, -1\n" \
  "s_nop 1\n" \
  "s_branch Lq199_%=\n" \
  "Lp200_%=:\n" \
  "s_swappc_b64 s[92:93], s[90:91]\n" \
  "s_branch Lq200_%=\n" \
  "Lp201_%=:\n" \
  "buffer_store_dwordx4 v[192:195], v222, s[52:55], s50 offen offset:0 sc0 sc1\n" \
  "s_branch Lq201_%=\n" \
  "Lp202_%=:\n" \
  "buffer_store_dwordx4 v[208:211], v222, s[52:55], s50 offen offset:64 sc0 sc1\n" \
  "s_branch Lq202_%=\n" \
  "Lp203_%=:\n" \
  "v_mov_b32 v216, 1\n" \
  "s_mov_b64 exec, 1\n" \
  "global_atomic_add v224, v216, s[80:81]\n" \
  "s_mov_b64 exec, -1\n" \
  "s_nop 1\n" \
  "s_branch Lq203_%=\n" \
  "Lp204_%=:\n" \
  "s_swappc_b64 s[92:93], s[90:91]\n" \
  "s_branch Lq204_%=\n" \
  "Lp205_%=:\n" \
  "buffer_store_dwordx4 v[192:195], v222, s[52:55], s50 offen offset:0 sc0 sc1\n" \
  "s_branch Lq205_%=\n" \
  "Lp206_%=:\n" \
  "buffer_store_dwordx4 v[208:211], v222, s[52:55], s50 offen offset:64 sc0 sc1\n" \
  "s_branch Lq206_%=\n" \
  "Lp207_%=:\n" \
  "v_mov_b32 v216, 1\n" \
  "s_mov_b64 exec, 1\n" \
  "global_atomic_add v224, v216, s[80:81]\n" \
  "s_mov_b64 exec, -1\n" \
  "s_nop 1\n" \
  "s_branch Lq207_%=\n" \
  "Lp208_%=:\n" \
  "s_swappc_b64 s[92:93], s[90:91]\n" \
  "s_branch Lq208_%=\n" \
  "Lp209_%=:\n" \
  "buffer_store_dwordx4 v[192:195], v222, s[52:55], s50 offen offset:0 sc0 sc1\n" \
  "s_branch Lq209_%=\n" \
  "Lp210_%=:\n" \
  "buffer_store_dwordx4 v[208:211], v222, s[52:55], s50 offen offset:64 sc0 sc1\n" \
  "s_branch Lq210_%=\n" \
  "Lp211_%=:\n" \
  "buffer_store_dwordx4 v[208:211], v222, s[52:55], s50 offen offset:64 sc0 sc1\n" \
  "s_branch Lq211_%=\n" \
  "Lp212_%=:\n" \
  "v_mov_b32 v216, 1\n" \
  "s_mov_b64 exec, 1\n" \
  "global_atomic_add v224, v216, s[80:81]\n" \
  "s_mov_b64 exec, -1\n" \
  "s_nop 1\n" \
  "s_branch Lq212_%=\n" \
  "Lp213_%=:\n" \
  "s_swappc_b64 s[92:93], s[90:91]\n" \
  "s_branch Lq213_%=\n" \
  "Lp214_%=:\n" \
  "buffer_store_dwordx4 v[192:195], v222, s[52:55], s50 offen offset:0 sc0 sc1\n" \
  "s_branch Lq214_%=\n" \
  "Lp215_%=:\n" \
  "v_mov_b32 v216, 1\n" \
  "s_mov_b64 exec, 1\n" \
  "global_atomic_add v224, v216, s[80:81]\n" \
  "s_mov_b64 exec, -1\n" \
  "s_nop 1\n" \
  "s_branch Lq215_%=\n" \
  "Lp216_%=:\n" \
  "s_swappc_b64 s[92:93], s[90:91]\n" \
  "s_branch Lq216_%=\n" \
  "Lp217_%=:\n" \
  "buffer_store_dwordx4 v[192:195], v222, s[52:55], s50 offen offset:0 sc0 sc1\n" \
  "s_branch Lq217_%=\n" \
  "Lp218_%=:\n" \
  "buffer_store_dwordx4 v[208:211], v222, s[52:55], s50 offen offset:64 sc0 sc1\n" \
  "s_branch Lq218_%=\n" \
  "Lp219_%=:\n" \
  "v_mov_b32 v216, 1\n" \
  "s_mov_b64 exec, 1\n" \
  "global_atomic_add v224, v216, s[80:81]\n" \
  "s_mov_b64 exec, -1\n" \
  "s_nop 1\n" \
  "s_branch Lq219_%=\n" \
  "Lp220_%=:\n" \
  "s_swappc_b64 s[92:93], s[90:91]\n" \
  "s_branch Lq220_%=\n" \
  "Lp221_%=:\n" \
  "buffer_store_dwordx4 v[192:195], v222, s[52:55], s50 offen offset:0 sc0 sc1\n" \
  "s_branch Lq221_%=\n" \
  "Lp222_%=:\n" \
  "v_mov_b32 v216, 1\n" \
  "s_mov_b64 exec, 1\n" \
  "global_atomic_add v224, v216, s[80:81]\n" \
  "s_mov_b64 exec, -1\n" \
  "s_nop 1\n" \
  "s_branch Lq222_%=\n" \
  "Lp223_%=:\n" \
  "s_swappc_b64 s[92:93], s[90:91]\n" \
  "s_branch Lq223_%=\n" \
  "Lp224_%=:\n" \
  "buffer_store_dwordx4 v[192:195], v222, s[52:55], s50 offen offset:0 sc0 sc1\n" \
  "s_branch Lq224_%=\n" \
  "Lp225_%=:\n" \
  "buffer_store_dwordx4 v[208:211], v222, s[52:55], s50 offen offset:64 sc0 sc1\n" \
  "s_branch Lq225_%=\n" \
  "Lp226_%=:\n" \
  "s_swappc_b64 s[92:93], s[90:91]\n" \
  "s_branch Lq226_%=\n" \
  "Lp227_%=:\n" \
  "buffer_store_dwordx4 v[196:199], v222, s[52:55], s51 offen offset:0 sc0 sc1\n" \
  "s_branch Lq227_%=\n" \
  "Lp228_%=:\n" \
  "buffer_store_dwordx4 v[212:215], v222, s[52:55], s51 offen offset:64 sc0 sc1\n" \
  "s_branch Lq228_%=\n" \
  "Lp229_%=:\n" \
  "s_swappc_b64 s[92:93], s[90:91]\n" \
  "s_branch Lq229_%=\n" \
  "Lp230_%=:\n" \
  "buffer_store_dwordx4 v[196:199], v222, s[52:55], s51 offen offset:0 sc0 sc1\n" \
  "s_branch Lq230_%=\n" \
  "Lp231_%=:\n" \
  "buffer_store_dwordx4 v[212:215], v222, s[52:55], s51 offen offset:64 sc0 sc1\n" \
  "s_branch Lq231_%=\n" \
  "Lp232_%=:\n" \
  "s_swappc_b64 s[92:93], s[90:91]\n" \
  "s_branch Lq232_%=\n" \
  "Lp233_%=:\n" \
  "buffer_store_dwordx4 v[196:199], v222, s[52:55], s51 offen offset:0 sc0 sc1\n" \
  "s_branch Lq233_%=\n" \
  "Lp234_%=:\n" \
  "buffer_store_dwordx4 v[212:215], v222, s[52:55], s51 offen offset:64 sc0 sc1\n" \
  "s_branch Lq234_%=\n" \
  "Lp235_%=:\n" \
  "s_swappc_b64 s[92:93], s[90:91]\n" \
  "s_branch Lq235_%=\n" \
  "Lp236_%=:\n" \
  "buffer_store_dwordx4 v[196:199], v222, s[52:55], s51 offen offset:0 sc0 sc1\n" \
  "s_branch Lq236_%=\n" \
  "Lp237_%=:\n" \
  "buffer_store_dwordx4 v[212:215], v222, s[52:55], s51 offen offset:64 sc0 sc1\n" \
  "s_branch Lq237_%=\n" \
  "Lp238_%=:\n" \
  "s_swappc_b64 s[92:93], s[90:91]\n" \
  "s_branch Lq238_%=\n" \
  "Lp239_%=:\n" \
  "buffer_store_dwordx4 v[196:199], v222, s[52:55], s51 offen offset:0 sc0 sc1\n" \
  "s_branch Lq239_%=\n" \
  "Lp240_%=:\n" \
  "buffer_store_dwordx4 v[212:215], v222, s[52:55], s51 offen offset:64 sc0 sc1\n" \
  "s_branch Lq240_%=\n" \
  "Lp241_%=:\n" \
  "buffer_store_dwordx4 v[212:215], v222, s[52:55], s51 offen offset:64 sc0 sc1\n" \
  "s_branch Lq241_%=\n" \
  "Lp242_%=:\n" \
  "s_swappc_b64 s[92:93], s[90:91]\n" \
  "s_branch Lq242_%=\n" \
  "Lp243_%=:\n" \
  "buffer_store_dwordx4 v[196:199], v222, s[52:55], s51 offen offset:0 sc0 sc1\n" \
  "s_branch Lq243_%=\n" \
  "Lp244_%=:\n" \
  "buffer_store_dwordx4 v[212:215], v222, s[52:55], s51 offen offset:64 sc0 sc1\n" \
  "s_branch Lq244_%=\n" \
  "Lp245_%=:\n" \
  "s_swappc_b64 s[92:93], s[90:91]\n" \
  "s_branch Lq245_%=\n" \
  "Lp246_%=:\n" \
  "buffer_store_dwordx4 v[196:199], v222, s[52:55], s51 offen offset:0 sc0 sc1\n" \
  "s_branch Lq246_%=\n" \
  "Lp247_%=:\n" \
  "buffer_store_dwordx4 v[212:215], v222, s[52:55], s51 offen offset:64 sc0 sc1\n" \
  "s_branch Lq247_%=\n" \
  "Lp248_%=:\n" \
  "s_swappc_b64 s[92:93], s[90:91]\n" \
  "s_branch Lq248_%=\n" \
  "Lp249_%=:\n" \
  "buffer_store_dwordx4 v[196:199], v222, s[52:55], s51 offen offset:0 sc0 sc1\n" \
  "s_branch Lq249_%=\n" \
  "Lp250_%=:\n" \
  "buffer_store_dwordx4 v[212:215], v222, s[52:55], s51 offen offset:64 sc0 sc1\n" \
  "s_branch Lq250_%=\n" \
  "Lp251_%=:\n" \
  "s_swappc_b64 s[92:93], s[90:91]\n" \
  "s_branch Lq251_%=\n" \
  "Lp252_%=:\n" \
  "buffer_store_dwordx4 v[196:199], v222, s[52:55], s51 offen offset:0 sc0 sc1\n" \
  "s_branch Lq252_%=\n" \
  "Lp253_%=:\n" \
  "buffer_store_dwordx4 v[212:215], v222, s[52:55], s51 offen offset:64 sc0 sc1\n" \
  "s_branch Lq253_%=\n" \
  "Lp254_%=:\n" \
  "s_swappc_b64 s[92:93], s[90:91]\n" \
  "s_branch Lq254_%=\n" \
  "Lp255_%=:\n" \
  "buffer_store_dwordx4 v[196:199], v222, s[52:55], s51 offen offset:0 sc0 sc1\n" \
  "s_branch Lq255_%=\n" \
  "Lp256_%=:\n" \
  "buffer_store_dwordx4 v[212:215], v222, s[52:55], s51 offen offset:64 sc0 sc1\n" \
  "s_branch Lq256_%=\n" \
  "Lp257_%=:\n" \
  "buffer_store_dwordx4 v[212:215], v222, s[52:55], s51 offen offset:64 sc0 sc1\n" \
  "s_branch Lq257_%=\n" \
  "Lp258_%=:\n" \
  "s_swappc_b64 s[92:93], s[90:91]\n" \
  "s_branch Lq258_%=\n" \
  "Lp259_%=:\n" \
  "buffer_store_dwordx4 v[196:199], v222, s[52:55], s51 offen offset:0 sc0 sc1\n" \
  "s_branch Lq259_%=\n" \
  "Lp260_%=:\n" \
  "s_swappc_b64 s[92:93], s[90:91]\n" \
  "s_branch Lq260_%=\n" \
  "Lp261_%=:\n" \
  "buffer_store_dwordx4 v[196:199], v222, s[52:55], s51 offen offset:0 sc0 sc1\n" \
  "s_branch Lq261_%=\n" \
  "Lp262_%=:\n" \
  "buffer_store_dwordx4 v[212:215], v222, s[52:55], s51 offen offset:64 sc0 sc1\n" \
  "s_branch Lq262_%=\n" \
  "Lp263_%=:\n" \
  "s_swappc_b64 s[92:93], s[90:91]\n" \
  "s_branch Lq263_%=\n" \
  "Lp264_%=:\n" \
  "buffer_store_dwordx4 v[196:199], v222, s[52:55], s51 offen offset:0 sc0 sc1\n" \
  "s_branch Lq264_%=\n" \
  "Lp265_%=:\n" \
  "s_swappc_b64 s[92:93], s[90:91]\n" \
  "s_branch Lq265_%=\n" \
  "Lp266_%=:\n" \
  "buffer_store_dwordx4 v[196:199], v222, s[52:55], s51 offen offset:0 sc0 sc1\n" \
  "s_branch Lq266_%=\n" \
  "Lp267_%=:\n" \
  "buffer_store_dwordx4 v[212:215], v222, s[52:55], s51 offen offset:64 sc0 sc1\n" \
  "s_branch Lq267_%=\n" \
  "Lpollr_%=:\n" \
  "global_load_dword v216, v168, s[80:81] sc1\n" \
  "s_memrealtime s[84:85]\n" \
  "s_waitcnt vmcnt(0) lgkmcnt(0)\n" \
  "v_readfirstlane_b32 s68, v216\n" \
  "s_sub_u32 s84, s84, s82\n" \
  "s_nop 3\n" \
  "s_cmp_le_u32 s67, s68\n" \
  "s_cbranch_scc1 Lpollx_%=\n" \
  "s_sleep 4\n" \
  "s_cmp_lt_u32 s84, 40000000\n" \
  "s_cbranch_scc1 Lpollr_%=\n" \
  "s_mov_b32 s68, -1\n" \
  "s_mov_b32 s79, -1\n" \
  "s_mov_b32 s78, 0x7fc00000\n" \
  "Lpollx_%=:\n" \
  "s_setpc_b64 s[92:93]\n" \
  "Ldone_%=:\n" \
  "s_waitcnt vmcnt(8)\n" \
  "v_readfirstlane_b32 %[tmo], v218\n" \
  "s_nop 3\n" \
  "s_or_b32 %[tmo], %[tmo], s79\n" \
  "s_waitcnt vmcnt(0) lgkmcnt(0)\n" \
  ""
#define GEMM_ASM_CLOBBERS "v0", "v1", "v2", "v3", "v4", "v5", "v6", "v7", "v8", "v9", "v10", "v11", "v12", "v13", "v14", "v15", "v16", "v17", "v18", "v19", "v20", "v21", "v22", "v23", "v24", "v25", "v26", "v27", "v28", "v29", "v30", "v31", "v32", "v33", "v34", "v35", "v36", "v37", "v38", "v39", "v40", "v41", "v42", "v43", "v44", "v45", "v46", "v47", "v48", "v49", "v50", "v51", "v52", "v53", "v54", "v55", "v56", "v57", "v58", "v59", "v60", "v61", "v62", "v63", "v64", "v65", "v66", "v67", "v68", "v69", "v70", "v71", "v72", "v73", "v74", "v75", "v76", "v77", "v78", "v79", "v80", "v81", "v82", "v83", "v84", "v85", "v86", "v87", "v88", "v89", "v90", "v91", "v92", "v93", "v94", "v95", "v96", "v97", "v98", "v99", "v100", "v101", "v102", "v103", "v104", "v105", "v106", "v107", "v108", "v109", "v110", "v111", "v112", "v113", "v114", "v115", "v116", "v117", "v118", "v119", "v120", "v121", "v122", "v123", "v124", "v125", "v126", "v127", "v128", "v129", "v130", "v131", "v132", "v133", "v134", "v135", "v136", "v137", "v138", "v139", "v140", "v141", "v142", "v143", "v144", "v145", "v146", "v147", "v148", "v149", "v150", "v151", "v152", "v153", "v154", "v155", "v156", "v157", "v158", "v159", "v160", "v161", "v162", "v163", "v164", "v165", "v166", "v167", "v168", "v169", "v170", "v171", "v172", "v173", "v174", "v175", "v176", "v177", "v178", "v179", "v180", "v181", "v182", "v183", "v184", "v185", "v186", "v187", "v188", "v189", "v190", "v191", "v192", "v193", "v194", "v195", "v196", "v197", "v198", "v199", "v200", "v201", "v202", "v203", "v204", "v205", "v206", "v207", "v208", "v209", "v210", "v211", "v212", "v213", "v214", "v215", "v216", "v217", "v218", "v219", "v220", "v221", "v222", "v223", "v224", "v225", "a0", "a1", "a2", "a3", "a4", "a5", "a6", "a7", "a8", "a9", "a10", "a11", "a12", "a13", "a14", "a15", "a16", "a17", "a18", "a19", "a20", "a21", "a22", "a23", "a24", "a25", "a26", "a27", "a28", "a29", "a30", "a31", "a32", "a33", "a34", "a35", "a36", "a37", "a38", "a39", "a40", "a41", "a42", "a43", "a44", "a45", "a46", "a47", "a48", "a49", "a50", "a51", "a52", "a53", "a54", "a55", "a56", "a57", "a58", "a59", "a60", "a61", "a62", "a63", "a64", "a65", "a66", "a67", "a68", "a69", "a70", "a71", "a72", "a73", "a74", "a75", "a76", "a77", "a78", "a79", "a80", "a81", "a82", "a83", "a84", "a85", "a86", "a87", "a88", "a89", "a90", "a91", "a92", "a93", "a94", "a95", "a96", "a97", "a98", "a99", "a100", "a101", "a102", "a103", "a104", "a105", "a106", "a107", "a108", "a109", "a110", "a111", "a112", "a113", "a114", "a115", "a116", "a117", "a118", "a119", "a120", "a121", "a122", "a123", "a124", "a125", "a126", "a127", "a128", "a129", "a130", "a131", "a132", "a133", "a134", "a135", "a136", "a137", "a138", "a139", "a140", "a141", "a142", "a143", "a144", "a145", "a146", "a147", "a148", "a149", "a150", "a151", "a152", "a153", "a154", "a155", "a156", "a157", "a158", "a159", "a160", "a161", "a162", "a163", "a164", "a165", "a166", "a167", "a168", "a169", "a170", "a171", "a172", "a173", "a174", "a175", "a176", "a177", "a178", "a179", "a180", "a181", "a182", "a183", "a184", "a185", "a186", "a187", "a188", "a189", "a190", "a191", "a192", "a193", "a194", "a195", "a196", "a197", "a198", "a199", "a200", "a201", "a202", "a203", "a204", "a205", "a206", "a207", "a208", "a209", "a210", "a211", "a212", "a213", "a214", "a215", "a216", "a217", "a218", "a219", "a220", "a221", "a222", "a223", "a224", "a225", "a226", "a227", "a228", "a229", "a230", "a231", "a232", "a233", "a234", "a235", "a236", "a237", "a238", "a239", "a240", "a241", "a242", "a243", "a244", "a245", "a246", "a247", "a248", "a249", "a250", "a251", "a252", "a253", "a254", "a255", "s36", "s37", "s38", "s39", "s40", "s41", "s42", "s43", "s44", "s45", "s46", "s47", "s48", "s49", "s50", "s51", "s52", "s53", "s54", "s55", "s56", "s57", "s58", "s59", "s60", "s61", "s62", "s63", "s64", "s65", "s66", "s67", "s68", "s69", "s70", "s71", "s72", "s73", "s74", "s75", "s76", "s77", "s78", "s79", "s80", "s81", "s82", "s83", "s84", "s85", "s86", "s87", "s88", "s89", "s90", "s91", "s92", "s93", "s94", "s95", "s96", "s97", "s98", "s99", "vcc", "scc", "memory"
#define PULL_ASM_TEXT \
  "s_mov_b64 s[36:37], %[rs]\n" \
  "s_mov_b32 s39, 0x00020000\n" \
  "s_mov_b64 s[40:41], %[rd]\n" \
  "s_mov_b32 s43, 0x00020000\n" \
  "s_mov_b32 s74, 3\n" \
  "s_mov_b64 s[44:45], %[rs]\n" \
  "s_mov_b32 s47, 0x00020000\n" \
  "s_mov_b64 s[48:49], %[rd]\n" \
  "s_mov_b32 s51, 0x00020000\n" \
  "s_mov_b32 s75, 3\n" \
  "s_mov_b64 s[52:53], %[pp]\n" \
  "s_mov_b32 s54, 0\n" \
  "s_mov_b32 s55, 0x00020000\n" \
  "s_mov_b32 s80, %[tag]\n" \
  "s_mul_i32 s81, %[call], 4\n" \
  "s_mov_b32 s86, %[col]\n" \
  "s_mov_b32 s82, 0\n" \
  "s_mov_b32 s84, 0\n" \
  "s_mov_b64 s[68:69], %[rdy]\n" \
  "s_mov_b32 %[ok], 1\n" \
  "s_memrealtime s[64:65]\n" \
  "v_mbcnt_lo_u32_b32 v32, -1, 0\n" \
  "v_mbcnt_hi_u32_b32 v32, -1, v32\n" \
  "v_and_b32 v50, 7, v32\n" \
  "v_lshlrev_b32 v51, 2, v50\n" \
  "v_lshlrev_b32 v32, 4, v32\n" \
  "v_mov_b32 v44, %[la]\n" \
  "v_and_b32 v45, -16, v44\n" \
  "v_mov_b32 v47, 1\n" \
  "v_mov_b32 v48, 0\n" \
  "v_add_u32 v33, 0, %[k0]\n" \
  "v_add_u32 v34, 16, %[k0]\n" \
  "v_add_u32 v35, 32, %[k0]\n" \
  "v_add_u32 v36, 48, %[k0]\n" \
  "s_waitcnt lgkmcnt(0)\n" \
  "v_readfirstlane_b32 s76, v33\n" \
  "s_nop 3\n" \
  "s_add_u32 s76, s76, 64\n" \
  "s_min_u32 s76, s76, 3160\n" \
  "s_lshl_b32 s83, s76, 1\n" \
  "v_mov_b32 v49, s83\n" \
  "ds_read_u16 v49, v49\n" \
  "s_cmp_lt_u32 s76, 3160\n" \
  "s_cselect_b32 s38, 0xa00000, 0\n" \
  "s_cselect_b32 s42, 0xa00000, 0\n" \
  "Lfq1_%=:\n" \
  "s_cmp_ge_u32 s76, s82\n" \
  "s_cbranch_scc1 Lfb1_%=\n" \
  "Lfr1_%=:\n" \
  "s_waitcnt lgkmcnt(0)\n" \
  "v_readfirstlane_b32 s70, v49\n" \
  "s_nop 3\n" \
  "s_bitcmp1_b32 s70, 15\n" \
  "s_cbranch_scc1 Llo1_%=\n" \
  "s_lshl_b32 s70, s70, 12\n" \
  "Llr1_%=:\n" \
  "buffer_load_dwordx4 v[0:3], v32, s[36:39], s70 offen offset:0 sc0 sc1\n" \
  "buffer_load_dwordx4 v[4:7], v32, s[36:39], s70 offen offset:1024 sc0 sc1\n" \
  "buffer_load_dwordx4 v[8:11], v32, s[36:39], s70 offen offset:2048 sc0 sc1\n" \
  "buffer_load_dwordx4 v[12:15], v32, s[36:39], s70 offen offset:3072 sc0 sc1\n" \
  "buffer_store_dword v48, off, s[52:55], 0\n" \
  "buffer_store_dword v48, off, s[52:55], 0\n" \
  "buffer_store_dword v48, off, s[52:55], 0\n" \
  "buffer_store_dword v48, off, s[52:55], 0\n" \
  "buffer_store_dword v48, off, s[52:55], 0\n" \
  "buffer_store_dword v48, off, s[52:55], 0\n" \
  "v_readfirstlane_b32 s77, v34\n" \
  "s_nop 3\n" \
  "s_add_u32 s77, s77, 64\n" \
  "s_min_u32 s77, s77, 3160\n" \
  "s_lshl_b32 s83, s77, 1\n" \
  "v_mov_b32 v49, s83\n" \
  "ds_read_u16 v49, v49\n" \
  "s_cmp_lt_u32 s77, 3160\n" \
  "s_cselect_b32 s46, 0xa00000, 0\n" \
  "s_cselect_b32 s50, 0xa00000, 0\n" \
  "Lfq2_%=:\n" \
  "s_cmp_ge_u32 s77, s82\n" \
  "s_cbranch_scc1 Lfb2_%=\n" \
  "Lfr2_%=:\n" \
  "s_waitcnt lgkmcnt(0)\n" \
  "v_readfirstlane_b32 s71, v49\n" \
  "s_nop 3\n" \
  "s_bitcmp1_b32 s71, 15\n" \
  "s_cbranch_scc1 Llo2_%=\n" \
  "s_lshl_b32 s71, s71, 12\n" \
  "Llr2_%=:\n" \
  "buffer_load_dwordx4 v[16:19], v32, s[44:47], s71 offen offset:0 sc0 sc1\n" \
  "buffer_load_dwordx4 v[20:23], v32, s[44:47], s71 offen offset:1024 sc0 sc1\n" \
  "buffer_load_dwordx4 v[24:27], v32, s[44:47], s71 offen offset:2048 sc0 sc1\n" \
  "buffer_load_dwordx4 v[28:31], v32, s[44:47], s71 offen offset:3072 sc0 sc1\n" \
  "buffer_store_dword v48, off, s[52:55], 0\n" \
  "s_mov_b32 s79, s76\n" \
  "s_mov_b32 s54, %[pn]\n" \
  "Lloop_%=:\n" \
  "s_cmp_ge_u32 s76, 3160\n" \
  "s_cbranch_scc1 Lend_%=\n" \
  "s_waitcnt vmcnt(11)\n" \
  "v_mov_b32 v46, s79\n" \
  "ds_write_b32 v44, v46\n" \
  "ds_read_b128 v[40:43], v45\n" \
  "s_cmp_lg_u32 s74, 3\n" \
  "s_cbranch_scc1 Lso1_%=\n" \
  "buffer_store_dwordx4 v[0:3], v32, s[40:43], s70 offen offset:0 sc1\n" \
  "buffer_store_dwordx4 v[4:7], v32, s[40:43], s70 offen offset:1024 sc1\n" \
  "buffer_store_dwordx4 v[8:11], v32, s[40:43], s70 offen offset:2048 sc1\n" \
  "buffer_store_dwordx4 v[12:15], v32, s[40:43], s70 offen offset:3072 sc1\n" \
  "Lsr1_%=:\n" \
  "v_readfirstlane_b32 s78, v35\n" \
  "s_nop 3\n" \
  "s_add_u32 s78, s78, 64\n" \
  "s_min_u32 s78, s78, 3160\n" \
  "s_lshl_b32 s83, s78, 1\n" \
  "v_mov_b32 v49, s83\n" \
  "ds_read_u16 v49, v49\n" \
  "s_cmp_lt_u32 s78, 3160\n" \
  "s_cselect_b32 s38, 0xa00000, 0\n" \
  "s_cselect_b32 s42, 0xa00000, 0\n" \
  "Lfq3_%=:\n" \
  "s_cmp_ge_u32 s78, s82\n" \
  "s_cbranch_scc1 Lfb3_%=\n" \
  "Lfr3_%=:\n" \
  "s_waitcnt lgkmcnt(0)\n" \
  "v_readfirstlane_b32 s70, v49\n" \
  "s_nop 3\n" \
  "s_bitcmp1_b32 s70, 15\n" \
  "s_cbranch_scc1 Llo3_%=\n" \
  "s_lshl_b32 s70, s70, 12\n" \
  "Llr3_%=:\n" \
  "s_mov_b64 exec, 1\n" \
  "global_atomic_add v33, v48, v47, %[ctr] sc0\n" \
  "s_mov_b64 exec, -1\n" \
  "buffer_load_dwordx4 v[0:3], v32, s[36:39], s70 offen offset:0 sc0 sc1\n" \
  "buffer_load_dwordx4 v[4:7], v32, s[36:39], s70 offen offset:1024 sc0 sc1\n" \
  "buffer_load_dwordx4 v[8:11], v32, s[36:39], s70 offen offset:2048 sc0 sc1\n" \
  "buffer_load_dwordx4 v[12:15], v32, s[36:39], s70 offen offset:3072 sc0 sc1\n" \
  "v_min3_u32 v40, v40, v41, v42\n" \
  "v_min_u32 v40, v40, v43\n" \
  "v_or_b32 v40, s80, v40\n" \
  "buffer_store_dword v40, off, s[52:55], 0 sc1\n" \
  "s_cmp_ge_u32 s77, 3160\n" \
  "s_cbranch_scc1 Lend_%=\n" \
  "s_waitcnt vmcnt(11)\n" \
  "v_mov_b32 v46, s76\n" \
  "ds_write_b32 v44, v46\n" \
  "ds_read_b128 v[40:43], v45\n" \
  "s_cmp_lg_u32 s75, 3\n" \
  "s_cbranch_scc1 Lso2_%=\n" \
  "buffer_store_dwordx4 v[16:19], v32, s[48:51], s71 offen offset:0 sc1\n" \
  "buffer_store_dwordx4 v[20:23], v32, s[48:51], s71 offen offset:1024 sc1\n" \
  "buffer_store_dwordx4 v[24:27], v32, s[48:51], s71 offen offset:2048 sc1\n" \
  "buffer_store_dwordx4 v[28:31], v32, s[48:51], s71 offen offset:3072 sc1\n" \
  "Lsr2_%=:\n" \
  "v_readfirstlane_b32 s79, v36\n" \
  "s_nop 3\n" \
  "s_add_u32 s79, s79, 64\n" \
  "s_min_u32 s79, s79, 3160\n" \
  "s_lshl_b32 s83, s79, 1\n" \
  "v_mov_b32 v49, s83\n" \
  "ds_read_u16 v49, v49\n" \
  "s_cmp_lt_u32 s79, 3160\n" \
  "s_cselect_b32 s46, 0xa00000, 0\n" \
  "s_cselect_b32 s50, 0xa00000, 0\n" \
  "Lfq4_%=:\n" \
  "s_cmp_ge_u32 s79, s82\n" \
  "s_cbranch_scc1 Lfb4_%=\n" \
  "Lfr4_%=:\n" \
  "s_waitcnt lgkmcnt(0)\n" \
  "v_readfirstlane_b32 s71, v49\n" \
  "s_nop 3\n" \
  "s_bitcmp1_b32 s71, 15\n" \
  "s_cbranch_scc1 Llo4_%=\n" \
  "s_lshl_b32 s71, s71, 12\n" \
  "Llr4_%=:\n" \
  "s_mov_b64 exec, 1\n" \
  "global_atomic_add v34, v48, v47, %[ctr] sc0\n" \
  "s_mov_b64 exec, -1\n" \
  "buffer_load_dwordx4 v[16:19], v32, s[44:47], s71 offen offset:0 sc0 sc1\n" \
  "buffer_load_dwordx4 v[20:23], v32, s[44:47], s71 offen offset:1024 sc0 sc1\n" \
  "buffer_load_dwordx4 v[24:27], v32, s[44:47], s71 offen offset:2048 sc0 sc1\n" \
  "buffer_load_dwordx4 v[28:31], v32, s[44:47], s71 offen offset:3072 sc0 sc1\n" \
  "v_min3_u32 v40, v40, v41, v42\n" \
  "v_min_u32 v40, v40, v43\n" \
  "v_or_b32 v40, s80, v40\n" \
  "buffer_store_dword v40, off, s[52:55], 0 sc1\n" \
  "s_cmp_ge_u32 s78, 3160\n" \
  "s_cbranch_scc1 Lend_%=\n" \
  "s_waitcnt vmcnt(11)\n" \
  "v_mov_b32 v46, s77\n" \
  "ds_write_b32 v44, v46\n" \
  "ds_read_b128 v[40:43], v45\n" \
  "s_cmp_lg_u32 s74, 3\n" \
  "s_cbranch_scc1 Lso3_%=\n" \
  "buffer_store_dwordx4 v[0:3], v32, s[40:43], s70 offen offset:0 sc1\n" \
  "buffer_store_dwordx4 v[4:7], v32, s[40:43], s70 offen offset:1024 sc1\n" \
  "buffer_store_dwordx4 v[8:11], v32, s[40:43], s70 offen offset:2048 sc1\n" \
  "buffer_store_dwordx4 v[12:15], v32, s[40:43], s70 offen offset:3072 sc1\n" \
  "Lsr3_%=:\n" \
  "v_readfirstlane_b32 s76, v33\n" \
  "s_nop 3\n" \
  "s_add_u32 s76, s76, 64\n" \
  "s_min_u32 s76, s76, 3160\n" \
  "s_lshl_b32 s83, s76, 1\n" \
  "v_mov_b32 v49, s83\n" \
  "ds_read_u16 v49, v49\n" \
  "s_cmp_lt_u32 s76, 3160\n" \
  "s_cselect_b32 s38, 0xa00000, 0\n" \
  "s_cselect_b32 s42, 0xa00000, 0\n" \
  "Lfq5_%=:\n" \
  "s_cmp_ge_u32 s76, s82\n" \
  "s_cbranch_scc1 Lfb5_%=\n" \
  "Lfr5_%=:\n" \
  "s_waitcnt lgkmcnt(0)\n" \
  "v_readfirstlane_b32 s70, v49\n" \
  "s_nop 3\n" \
  "s_bitcmp1_b32 s70, 15\n" \
  "s_cbranch_scc1 Llo5_%=\n" \
  "s_lshl_b32 s70, s70, 12\n" \
  "Llr5_%=:\n" \
  "s_mov_b64 exec, 1\n" \
  "global_atomic_add v35, v48, v47, %[ctr] sc0\n" \
  "s_mov_b64 exec, -1\n" \
  "buffer_load_dwordx4 v[0:3], v32, s[36:39], s70 offen offset:0 sc0 sc1\n" \
  "buffer_load_dwordx4 v[4:7], v32, s[36:39], s70 offen offset:1024 sc0 sc1\n" \
  "buffer_load_dwordx4 v[8:11], v32, s[36:39], s70 offen offset:2048 sc0 sc1\n" \
  "buffer_load_dwordx4 v[12:15], v32, s[36:39], s70 offen offset:3072 sc0 sc1\n" \
  "v_min3_u32 v40, v40, v41, v42\n" \
  "v_min_u32 v40, v40, v43\n" \
  "v_or_b32 v40, s80, v40\n" \
  "buffer_store_dword v40, off, s[52:55], 0 sc1\n" \
  "s_cmp_ge_u32 s79, 3160\n" \
  "s_cbranch_scc1 Lend_%=\n" \
  "s_waitcnt vmcnt(11)\n" \
  "v_mov_b32 v46, s78\n" \
  "ds_write_b32 v44, v46\n" \
  "ds_read_b128 v[40:43], v45\n" \
  "s_cmp_lg_u32 s75, 3\n" \
  "s_cbranch_scc1 Lso4_%=\n" \
  "buffer_store_dwordx4 v[16:19], v32, s[48:51], s71 offen offset:0 sc1\n" \
  "buffer_store_dwordx4 v[20:23], v32, s[48:51], s71 offen offset:1024 sc1\n" \
  "buffer_store_dwordx4 v[24:27], v32, s[48:51], s71 offen offset:2048 sc1\n" \
  "buffer_store_dwordx4 v[28:31], v32, s[48:51], s71 offen offset:3072 sc1\n" \
  "Lsr4_%=:\n" \
  "v_readfirstlane_b32 s77, v34\n" \
  "s_nop 3\n" \
  "s_add_u32 s77, s77, 64\n" \
  "s_min_u32 s77, s77, 3160\n" \
  "s_lshl_b32 s83, s77, 1\n" \
  "v_mov_b32 v49, s83\n" \
  "ds_read_u16 v49, v49\n" \
  "s_cmp_lt_u32 s77, 3160\n" \
  "s_cselect_b32 s46, 0xa00000, 0\n" \
  "s_cselect_b32 s50, 0xa00000, 0\n" \
  "Lfq6_%=:\n" \
  "s_cmp_ge_u32 s77, s82\n" \
  "s_cbranch_scc1 Lfb6_%=\n" \
  "Lfr6_%=:\n" \
  "s_waitcnt lgkmcnt(0)\n" \
  "v_readfirstlane_b32 s71, v49\n" \
  "s_nop 3\n" \
  "s_bitcmp1_b32 s71, 15\n" \
  "s_cbranch_scc1 Llo6_%=\n" \
  "s_lshl_b32 s71, s71, 12\n" \
  "Llr6_%=:\n" \
  "s_mov_b64 exec, 1\n" \
  "global_atomic_add v36, v48, v47, %[ctr] sc0\n" \
  "s_mov_b64 exec, -1\n" \
  "buffer_load_dwordx4 v[16:19], v32, s[44:47], s71 offen offset:0 sc0 sc1\n" \
  "buffer_load_dwordx4 v[20:23], v32, s[44:47], s71 offen offset:1024 sc0 sc1\n" \
  "buffer_load_dwordx4 v[24:27], v32, s[44:47], s71 offen offset:2048 sc0 sc1\n" \
  "buffer_load_dwordx4 v[28:31], v32, s[44:47], s71 offen offset:3072 sc0 sc1\n" \
  "v_min3_u32 v40, v40, v41, v42\n" \
  "v_min_u32 v40, v40, v43\n" \
  "v_or_b32 v40, s80, v40\n" \
  "buffer_store_dword v40, off, s[52:55], 0 sc1\n" \
  "s_branch Lloop_%=\n" \
  "Lfb1_%=:\n" \
  "s_load_dwordx8 s[56:63], s[68:69], s84\n" \
  "v_add_u32 v52, s84, v51\n" \
  "s_waitcnt lgkmcnt(0)\n" \
  "s_min_u32 s56, s56, s57\n" \
  "s_min_u32 s56, s56, s58\n" \
  "s_min_u32 s56, s56, s59\n" \
  "s_min_u32 s56, s56, s60\n" \
  "s_min_u32 s56, s56, s61\n" \
  "s_min_u32 s56, s56, s62\n" \
  "s_min_u32 s56, s56, s63\n" \
  "s_cmp_ge_u32 s56, s81\n" \
  "s_cbranch_scc1 Lfd1_%=\n" \
  "Lfc1_%=:\n" \
  "global_load_dword v50, v52, s[68:69] sc0 sc1\n" \
  "s_memrealtime s[66:67]\n" \
  "s_waitcnt vmcnt(0) lgkmcnt(0)\n" \
  "v_cmp_gt_u32 vcc, s81, v50\n" \
  "s_sub_u32 s66, s66, s64\n" \
  "s_nop 4\n" \
  "s_cmp_eq_u64 vcc, 0\n" \
  "s_cbranch_scc1 Lfd1_%=\n" \
  "s_cmp_lt_u32 s66, 40000000\n" \
  "s_cbranch_scc1 Lfc1_%=\n" \
  "s_mov_b32 %[ok], 0\n" \
  "Lfd1_%=:\n" \
  "s_lshr_b32 s83, s84, 3\n" \
  "s_lshr_b64 s[56:57], %[lims], s83\n" \
  "s_and_b32 s82, s56, 0xffff\n" \
  "s_lshr_b32 s83, s84, 4\n" \
  "s_lshr_b32 s81, 0x01041c, s83\n" \
  "s_and_b32 s81, s81, 0xff\n" \
  "s_mul_i32 s81, s81, %[call]\n" \
  "s_add_u32 s84, s84, 128\n" \
  "s_cmp_lg_u32 s84, 256\n" \
  "s_cbranch_scc1 Lfq1_%=\n" \
  "s_getpc_b64 s[66:67]\n" \
  "s_branch Lvo_%=\n" \
  "s_branch Lfq1_%=\n" \
  "Lfb2_%=:\n" \
  "s_load_dwordx8 s[56:63], s[68:69], s84\n" \
  "v_add_u32 v52, s84, v51\n" \
  "s_waitcnt lgkmcnt(0)\n" \
  "s_min_u32 s56, s56, s57\n" \
  "s_min_u32 s56, s56, s58\n" \
  "s_min_u32 s56, s56, s59\n" \
  "s_min_u32 s56, s56, s60\n" \
  "s_min_u32 s56, s56, s61\n" \
  "s_min_u32 s56, s56, s62\n" \
  "s_min_u32 s56, s56, s63\n" \
  "s_cmp_ge_u32 s56, s81\n" \
  "s_cbranch_scc1 Lfd2_%=\n" \
  "Lfc2_%=:\n" \
  "global_load_dword v50, v52, s[68:69] sc0 sc1\n" \
  "s_memrealtime s[66:67]\n" \
  "s_waitcnt vmcnt(0) lgkmcnt(0)\n" \
  "v_cmp_gt_u32 vcc, s81, v50\n" \
  "s_sub_u32 s66, s66, s64\n" \
  "s_nop 4\n" \
  "s_cmp_eq_u64 vcc, 0\n" \
  "s_cbranch_scc1 Lfd2_%=\n" \
  "s_cmp_lt_u32 s66, 40000000\n" \
  "s_cbranch_scc1 Lfc2_%=\n" \
  "s_mov_b32 %[ok], 0\n" \
  "Lfd2_%=:\n" \
  "s_lshr_b32 s83, s84, 3\n" \
  "s_lshr_b64 s[56:57], %[lims], s83\n" \
  "s_and_b32 s82, s56, 0xffff\n" \
  "s_lshr_b32 s83, s84, 4\n" \
  "s_lshr_b32 s81, 0x01041c, s83\n" \
  "s_and_b32 s81, s81, 0xff\n" \
  "s_mul_i32 s81, s81, %[call]\n" \
  "s_add_u32 s84, s84, 128\n" \
  "s_cmp_lg_u32 s84, 256\n" \
  "s_cbranch_scc1 Lfq2_%=\n" \
  "s_getpc_b64 s[66:67]\n" \
  "s_branch Lvo_%=\n" \
  "s_branch Lfq2_%=\n" \
  "Lfb3_%=:\n" \
  "s_load_dwordx8 s[56:63], s[68:69], s84\n" \
  "v_add_u32 v52, s84, v51\n" \
  "s_waitcnt lgkmcnt(0)\n" \
  "s_min_u32 s56, s56, s57\n" \
  "s_min_u32 s56, s56, s58\n" \
  "s_min_u32 s56, s56, s59\n" \
  "s_min_u32 s56, s56, s60\n" \
  "s_min_u32 s56, s56, s61\n" \
  "s_min_u32 s56, s56, s62\n" \
  "s_min_u32 s56, s56, s63\n" \
  "s_cmp_ge_u32 s56, s81\n" \
  "s_cbranch_scc1 Lfd3_%=\n" \
  "Lfc3_%=:\n" \
  "global_load_dword v50, v52, s[68:69] sc0 sc1\n" \
  "s_memrealtime s[66:67]\n" \
  "s_waitcnt vmcnt(0) lgkmcnt(0)\n" \
  "v_cmp_gt_u32 vcc, s81, v50\n" \
  "s_sub_u32 s66, s66, s64\n" \
  "s_nop 4\n" \
  "s_cmp_eq_u64 vcc, 0\n" \
  "s_cbranch_scc1 Lfd3_%=\n" \
  "s_cmp_lt_u32 s66, 40000000\n" \
  "s_cbranch_scc1 Lfc3_%=\n" \
  "s_mov_b32 %[ok], 0\n" \
  "Lfd3_%=:\n" \
  "s_lshr_b32 s83, s84, 3\n" \
  "s_lshr_b64 s[56:57], %[lims], s83\n" \
  "s_and_b32 s82, s56, 0xffff\n" \
  "s_lshr_b32 s83, s84, 4\n" \
  "s_lshr_b32 s81, 0x01041c, s83\n" \
  "s_and_b32 s81, s81, 0xff\n" \
  "s_mul_i32 s81, s81, %[call]\n" \
  "s_add_u32 s84, s84, 128\n" \
  "s_cmp_lg_u32 s84, 256\n" \
  "s_cbranch_scc1 Lfq3_%=\n" \
  "s_getpc_b64 s[66:67]\n" \
  "s_branch Lvo_%=\n" \
  "s_branch Lfq3_%=\n" \
  "Lfb4_%=:\n" \
  "s_load_dwordx8 s[56:63], s[68:69], s84\n" \
  "v_add_u32 v52, s84, v51\n" \
  "s_waitcnt lgkmcnt(0)\n" \
  "s_min_u32 s56, s56, s57\n" \
  "s_min_u32 s56, s56, s58\n" \
  "s_min_u32 s56, s56, s59\n" \
  "s_min_u32 s56, s56, s60\n" \
  "s_min_u32 s56, s56, s61\n" \
  "s_min_u32 s56, s56, s62\n" \
  "s_min_u32 s56, s56, s63\n" \
  "s_cmp_ge_u32 s56, s81\n" \
  "s_cbranch_scc1 Lfd4_%=\n" \
  "Lfc4_%=:\n" \
  "global_load_dword v50, v52, s[68:69] sc0 sc1\n" \
  "s_memrealtime s[66:67]\n" \
  "s_waitcnt vmcnt(0) lgkmcnt(0)\n" \
  "v_cmp_gt_u32 vcc, s81, v50\n" \
  "s_sub_u32 s66, s66, s64\n" \
  "s_nop 4\n" \
  "s_cmp_eq_u64 vcc, 0\n" \
  "s_cbranch_scc1 Lfd4_%=\n" \
  "s_cmp_lt_u32 s66, 40000000\n" \
  "s_cbranch_scc1 Lfc4_%=\n" \
  "s_mov_b32 %[ok], 0\n" \
  "Lfd4_%=:\n" \
  "s_lshr_b32 s83, s84, 3\n" \
  "s_lshr_b64 s[56:57], %[lims], s83\n" \
  "s_and_b32 s82, s56, 0xffff\n" \
  "s_lshr_b32 s83, s84, 4\n" \
  "s_lshr_b32 s81, 0x01041c, s83\n" \
  "s_and_b32 s81, s81, 0xff\n" \
  "s_mul_i32 s81, s81, %[call]\n" \
  "s_add_u32 s84, s84, 128\n" \
  "s_cmp_lg_u32 s84, 256\n" \
  "s_cbranch_scc1 Lfq4_%=\n" \
  "s_getpc_b64 s[66:67]\n" \
  "s_branch Lvo_%=\n" \
  "s_branch Lfq4_%=\n" \
  "Lfb5_%=:\n" \
  "s_load_dwordx8 s[56:63], s[68:69], s84\n" \
  "v_add_u32 v52, s84, v51\n" \
  "s_waitcnt lgkmcnt(0)\n" \
  "s_min_u32 s56, s56, s57\n" \
  "s_min_u32 s56, s56, s58\n" \
  "s_min_u32 s56, s56, s59\n" \
  "s_min_u32 s56, s56, s60\n" \
  "s_min_u32 s56, s56, s61\n" \
  "s_min_u32 s56, s56, s62\n" \
  "s_min_u32 s56, s56, s63\n" \
  "s_cmp_ge_u32 s56, s81\n" \
  "s_cbranch_scc1 Lfd5_%=\n" \
  "Lfc5_%=:\n" \
  "global_load_dword v50, v52, s[68:69] sc0 sc1\n" \
  "s_memrealtime s[66:67]\n" \
  "s_waitcnt vmcnt(0) lgkmcnt(0)\n" \
  "v_cmp_gt_u32 vcc, s81, v50\n" \
  "s_sub_u32 s66, s66, s64\n" \
  "s_nop 4\n" \
  "s_cmp_eq_u64 vcc, 0\n" \
  "s_cbranch_scc1 Lfd5_%=\n" \
  "s_cmp_lt_u32 s66, 40000000\n" \
  "s_cbranch_scc1 Lfc5_%=\n" \
  "s_mov_b32 %[ok], 0\n" \
  "Lfd5_%=:\n" \
  "s_lshr_b32 s83, s84, 3\n" \
  "s_lshr_b64 s[56:57], %[lims], s83\n" \
  "s_and_b32 s82, s56, 0xffff\n" \
  "s_lshr_b32 s83, s84, 4\n" \
  "s_lshr_b32 s81, 0x01041c, s83\n" \
  "s_and_b32 s81, s81, 0xff\n" \
  "s_mul_i32 s81, s81, %[call]\n" \
  "s_add_u32 s84, s84, 128\n" \
  "s_cmp_lg_u32 s84, 256\n" \
  "s_cbranch_scc1 Lfq5_%=\n" \
  "s_getpc_b64 s[66:67]\n" \
  "s_branch Lvo_%=\n" \
  "s_branch Lfq5_%=\n" \
  "Lfb6_%=:\n" \
  "s_load_dwordx8 s[56:63], s[68:69], s84\n" \
  "v_add_u32 v52, s84, v51\n" \
  "s_waitcnt lgkmcnt(0)\n" \
  "s_min_u32 s56, s56, s57\n" \
  "s_min_u32 s56, s56, s58\n" \
  "s_min_u32 s56, s56, s59\n" \
  "s_min_u32 s56, s56, s60\n" \
  "s_min_u32 s56, s56, s61\n" \
  "s_min_u32 s56, s56, s62\n" \
  "s_min_u32 s56, s56, s63\n" \
  "s_cmp_ge_u32 s56, s81\n" \
  "s_cbranch_scc1 Lfd6_%=\n" \
  "Lfc6_%=:\n" \
  "global_load_dword v50, v52, s[68:69] sc0 sc1\n" \
  "s_memrealtime s[66:67]\n" \
  "s_waitcnt vmcnt(0) lgkmcnt(0)\n" \
  "v_cmp_gt_u32 vcc, s81, v50\n" \
  "s_sub_u32 s66, s66, s64\n" \
  "s_nop 4\n" \
  "s_cmp_eq_u64 vcc, 0\n" \
  "s_cbranch_scc1 Lfd6_%=\n" \
  "s_cmp_lt_u32 s66, 40000000\n" \
  "s_cbranch_scc1 Lfc6_%=\n" \
  "s_mov_b32 %[ok], 0\n" \
  "Lfd6_%=:\n" \
  "s_lshr_b32 s83, s84, 3\n" \
  "s_lshr_b64 s[56:57], %[lims], s83\n" \
  "s_and_b32 s82, s56, 0xffff\n" \
  "s_lshr_b32 s83, s84, 4\n" \
  "s_lshr_b32 s81, 0x01041c, s83\n" \
  "s_and_b32 s81, s81, 0xff\n" \
  "s_mul_i32 s81, s81, %[call]\n" \
  "s_add_u32 s84, s84, 128\n" \
  "s_cmp_lg_u32 s84, 256\n" \
  "s_cbranch_scc1 Lfq6_%=\n" \
  "s_getpc_b64 s[66:67]\n" \
  "s_branch Lvo_%=\n" \
  "s_branch Lfq6_%=\n" \
  "Lvo_%=:\n" \
  "v_lshrrev_b32 v50, 4, v32\n" \
  "v_add_u32 v53, 0, v50\n" \
  "v_mul_u32_u24 v52, 0x5556, v53\n" \
  "v_lshrrev_b32 v52, 20, v52\n" \
  "v_mul_u32_u24 v46, 48, v52\n" \
  "v_sub_u32 v53, v53, v46\n" \
  "v_lshlrev_b32 v52, 16, v52\n" \
  "v_lshl_add_u32 v53, v53, 4, v52\n" \
  "v_add_u32 v54, 64, v50\n" \
  "v_mul_u32_u24 v52, 0x5556, v54\n" \
  "v_lshrrev_b32 v52, 20, v52\n" \
  "v_mul_u32_u24 v46, 48, v52\n" \
  "v_sub_u32 v54, v54, v46\n" \
  "v_lshlrev_b32 v52, 16, v52\n" \
  "v_lshl_add_u32 v54, v54, 4, v52\n" \
  "v_add_u32 v55, 128, v50\n" \
  "v_mul_u32_u24 v52, 0x5556, v55\n" \
  "v_lshrrev_b32 v52, 20, v52\n" \
  "v_mul_u32_u24 v46, 48, v52\n" \
  "v_sub_u32 v55, v55, v46\n" \
  "v_lshlrev_b32 v52, 16, v52\n" \
  "v_lshl_add_u32 v55, v55, 4, v52\n" \
  "v_add_u32 v56, 192, v50\n" \
  "v_mul_u32_u24 v52, 0x5556, v56\n" \
  "v_lshrrev_b32 v52, 20, v52\n" \
  "v_mul_u32_u24 v46, 48, v52\n" \
  "v_sub_u32 v56, v56, v46\n" \
  "v_lshlrev_b32 v52, 16, v52\n" \
  "v_lshl_add_u32 v56, v56, 4, v52\n" \
  "v_add_u32 v57, 256, v50\n" \
  "v_mul_u32_u24 v52, 0x5556, v57\n" \
  "v_lshrrev_b32 v52, 20, v52\n" \
  "v_mul_u32_u24 v46, 48, v52\n" \
  "v_sub_u32 v57, v57, v46\n" \
  "v_lshlrev_b32 v52, 16, v52\n" \
  "v_lshl_add_u32 v57, v57, 4, v52\n" \
  "v_add_u32 v58, 320, v50\n" \
  "v_mul_u32_u24 v52, 0x5556, v58\n" \
  "v_lshrrev_b32 v52, 20, v52\n" \
  "v_mul_u32_u24 v46, 48, v52\n" \
  "v_sub_u32 v58, v58, v46\n" \
  "v_lshlrev_b32 v52, 16, v52\n" \
  "v_lshl_add_u32 v58, v58, 4, v52\n" \
  "v_add_u32 v59, 384, v50\n" \
  "v_mul_u32_u24 v52, 0x5556, v59\n" \
  "v_lshrrev_b32 v52, 20, v52\n" \
  "v_mul_u32_u24 v46, 48, v52\n" \
  "v_sub_u32 v59, v59, v46\n" \
  "v_lshlrev_b32 v52, 16, v52\n" \
  "v_lshl_add_u32 v59, v59, 4, v52\n" \
  "v_add_u32 v60, 448, v50\n" \
  "v_mul_u32_u24 v52, 0x5556, v60\n" \
  "v_lshrrev_b32 v52, 20, v52\n" \
  "v_mul_u32_u24 v46, 48, v52\n" \
  "v_sub_u32 v60, v60, v46\n" \
  "v_lshlrev_b32 v52, 16, v52\n" \
  "v_lshl_add_u32 v60, v60, 4, v52\n" \
  "v_add_u32 v61, 512, v50\n" \
  "v_mul_u32_u24 v52, 0x5556, v61\n" \
  "v_lshrrev_b32 v52, 20, v52\n" \
  "v_mul_u32_u24 v46, 48, v52\n" \
  "v_sub_u32 v61, v61, v46\n" \
  "v_lshlrev_b32 v52, 16, v52\n" \
  "v_lshl_add_u32 v61, v61, 4, v52\n" \
  "v_add_u32 v62, 576, v50\n" \
  "v_mul_u32_u24 v52, 0x5556, v62\n" \
  "v_lshrrev_b32 v52, 20, v52\n" \
  "v_mul_u32_u24 v46, 48, v52\n" \
  "v_sub_u32 v62, v62, v46\n" \
  "v_lshlrev_b32 v52, 16, v52\n" \
  "v_lshl_add_u32 v62, v62, 4, v52\n" \
  "v_add_u32 v63, 640, v50\n" \
  "v_mul_u32_u24 v52, 0x5556, v63\n" \
  "v_lshrrev_b32 v52, 20, v52\n" \
  "v_mul_u32_u24 v46, 48, v52\n" \
  "v_sub_u32 v63, v63, v46\n" \
  "v_lshlrev_b32 v52, 16, v52\n" \
  "v_lshl_add_u32 v63, v63, 4, v52\n" \
  "v_add_u32 v64, 704, v50\n" \
  "v_mul_u32_u24 v52, 0x5556, v64\n" \
  "v_lshrrev_b32 v52, 20, v52\n" \
  "v_mul_u32_u24 v46, 48, v52\n" \
  "v_sub_u32 v64, v64, v46\n" \
  "v_lshlrev_b32 v52, 16, v52\n" \
  "v_lshl_add_u32 v64, v64, 4, v52\n" \
  "s_add_u32 s66, s66, 4\n" \
  "s_addc_u32 s67, s67, 0\n" \
  "s_setpc_b64 s[66:67]\n" \
  "Llo1_%=:\n" \
  "s_mov_b64 s[36:37], %[rs2]\n" \
  "s_mov_b64 s[40:41], %[out]\n" \
  "s_mov_b32 s42, 0x10000000\n" \
  "s_and_b32 s74, s70, 3\n" \
  "s_bfe_u32 s70, s70, 0x80002\n" \
  "s_lshl_b32 s72, s70, 20\n" \
  "s_add_u32 s72, s72, s86\n" \
  "s_mul_i32 s70, s70, 3\n" \
  "s_add_u32 s70, s70, s74\n" \
  "s_lshl_b32 s70, s70, 12\n" \
  "s_branch Llr1_%=\n" \
  "Llo2_%=:\n" \
  "s_mov_b64 s[44:45], %[rs2]\n" \
  "s_mov_b64 s[48:49], %[out]\n" \
  "s_mov_b32 s50, 0x10000000\n" \
  "s_and_b32 s75, s71, 3\n" \
  "s_bfe_u32 s71, s71, 0x80002\n" \
  "s_lshl_b32 s73, s71, 20\n" \
  "s_add_u32 s73, s73, s86\n" \
  "s_mul_i32 s71, s71, 3\n" \
  "s_add_u32 s71, s71, s75\n" \
  "s_lshl_b32 s71, s71, 12\n" \
  "s_branch Llr2_%=\n" \
  "Lso1_%=:\n" \
  "s_cmp_eq_u32 s74, 1\n" \
  "s_cbranch_scc1 Lso1_1_%=\n" \
  "s_cmp_eq_u32 s74, 2\n" \
  "s_cbranch_scc1 Lso1_2_%=\n" \
  "Lso1_0_%=:\n" \
  "buffer_store_dwordx4 v[0:3], v53, s[40:43], s72 offen sc1\n" \
  "buffer_store_dwordx4 v[4:7], v54, s[40:43], s72 offen sc1\n" \
  "buffer_store_dwordx4 v[8:11], v55, s[40:43], s72 offen sc1\n" \
  "buffer_store_dwordx4 v[12:15], v56, s[40:43], s72 offen sc1\n" \
  "s_branch Lsr1_%=\n" \
  "Lso1_1_%=:\n" \
  "buffer_store_dwordx4 v[0:3], v57, s[40:43], s72 offen sc1\n" \
  "buffer_store_dwordx4 v[4:7], v58, s[40:43], s72 offen sc1\n" \
  "buffer_store_dwordx4 v[8:11], v59, s[40:43], s72 offen sc1\n" \
  "buffer_store_dwordx4 v[12:15], v60, s[40:43], s72 offen sc1\n" \
  "s_branch Lsr1_%=\n" \
  "Lso1_2_%=:\n" \
  "buffer_store_dwordx4 v[0:3], v61, s[40:43], s72 offen sc1\n" \
  "buffer_store_dwordx4 v[4:7], v62, s[40:43], s72 offen sc1\n" \
  "buffer_store_dwordx4 v[8:11], v63, s[40:43], s72 offen sc1\n" \
  "buffer_store_dwordx4 v[12:15], v64, s[40:43], s72 offen sc1\n" \
  "s_branch Lsr1_%=\n" \
  "Llo3_%=:\n" \
  "s_mov_b64 s[36:37], %[rs2]\n" \
  "s_mov_b64 s[40:41], %[out]\n" \
  "s_mov_b32 s42, 0x10000000\n" \
  "s_and_b32 s74, s70, 3\n" \
  "s_bfe_u32 s70, s70, 0x80002\n" \
  "s_lshl_b32 s72, s70, 20\n" \
  "s_add_u32 s72, s72, s86\n" \
  "s_mul_i32 s70, s70, 3\n" \
  "s_add_u32 s70, s70, s74\n" \
  "s_lshl_b32 s70, s70, 12\n" \
  "s_branch Llr3_%=\n" \
  "Lso2_%=:\n" \
  "s_cmp_eq_u32 s75, 1\n" \
  "s_cbranch_scc1 Lso2_1_%=\n" \
  "s_cmp_eq_u32 s75, 2\n" \
  "s_cbranch_scc1 Lso2_2_%=\n" \
  "Lso2_0_%=:\n" \
  "buffer_store_dwordx4 v[16:19], v53, s[48:51], s73 offen sc1\n" \
  "buffer_store_dwordx4 v[20:23], v54, s[48:51], s73 offen sc1\n" \
  "buffer_store_dwordx4 v[24:27], v55, s[48:51], s73 offen sc1\n" \
  "buffer_store_dwordx4 v[28:31], v56, s[48:51], s73 offen sc1\n" \
  "s_branch Lsr2_%=\n" \
  "Lso2_1_%=:\n" \
  "buffer_store_dwordx4 v[16:19], v57, s[48:51], s73 offen sc1\n" \
  "buffer_store_dwordx4 v[20:23], v58, s[48:51], s73 offen sc1\n" \
  "buffer_store_dwordx4 v[24:27], v59, s[48:51], s73 offen sc1\n" \
  "buffer_store_dwordx4 v[28:31], v60, s[48:51], s73 offen sc1\n" \
  "s_branch Lsr2_%=\n" \
  "Lso2_2_%=:\n" \
  "buffer_store_dwordx4 v[16:19], v61, s[48:51], s73 offen sc1\n" \
  "buffer_store_dwordx4 v[20:23], v62, s[48:51], s73 offen sc1\n" \
  "buffer_store_dwordx4 v[24:27], v63, s[48:51], s73 offen sc1\n" \
  "buffer_store_dwordx4 v[28:31], v64, s[48:51], s73 offen sc1\n" \
  "s_branch Lsr2_%=\n" \
  "Llo4_%=:\n" \
  "s_mov_b64 s[44:45], %[rs2]\n" \
  "s_mov_b64 s[48:49], %[out]\n" \
  "s_mov_b32 s50, 0x10000000\n" \
  "s_and_b32 s75, s71, 3\n" \
  "s_bfe_u32 s71, s71, 0x80002\n" \
  "s_lshl_b32 s73, s71, 20\n" \
  "s_add_u32 s73, s73, s86\n" \
  "s_mul_i32 s71, s71, 3\n" \
  "s_add_u32 s71, s71, s75\n" \
  "s_lshl_b32 s71, s71, 12\n" \
  "s_branch Llr4_%=\n" \
  "Lso3_%=:\n" \
  "s_cmp_eq_u32 s74, 1\n" \
  "s_cbranch_scc1 Lso3_1_%=\n" \
  "s_cmp_eq_u32 s74, 2\n" \
  "s_cbranch_scc1 Lso3_2_%=\n" \
  "Lso3_0_%=:\n" \
  "buffer_store_dwordx4 v[0:3], v53, s[40:43], s72 offen sc1\n" \
  "buffer_store_dwordx4 v[4:7], v54, s[40:43], s72 offen sc1\n" \
  "buffer_store_dwordx4 v[8:11], v55, s[40:43], s72 offen sc1\n" \
  "buffer_store_dwordx4 v[12:15], v56, s[40:43], s72 offen sc1\n" \
  "s_branch Lsr3_%=\n" \
  "Lso3_1_%=:\n" \
  "buffer_store_dwordx4 v[0:3], v57, s[40:43], s72 offen sc1\n" \
  "buffer_store_dwordx4 v[4:7], v58, s[40:43], s72 offen sc1\n" \
  "buffer_store_dwordx4 v[8:11], v59, s[40:43], s72 offen sc1\n" \
  "buffer_store_dwordx4 v[12:15], v60, s[40:43], s72 offen sc1\n" \
  "s_branch Lsr3_%=\n" \
  "Lso3_2_%=:\n" \
  "buffer_store_dwordx4 v[0:3], v61, s[40:43], s72 offen sc1\n" \
  "buffer_store_dwordx4 v[4:7], v62, s[40:43], s72 offen sc1\n" \
  "buffer_store_dwordx4 v[8:11], v63, s[40:43], s72 offen sc1\n" \
  "buffer_store_dwordx4 v[12:15], v64, s[40:43], s72 offen sc1\n" \
  "s_branch Lsr3_%=\n" \
  "Llo5_%=:\n" \
  "s_mov_b64 s[36:37], %[rs2]\n" \
  "s_mov_b64 s[40:41], %[out]\n" \
  "s_mov_b32 s42, 0x10000000\n" \
  "s_and_b32 s74, s70, 3\n" \
  "s_bfe_u32 s70, s70, 0x80002\n" \
  "s_lshl_b32 s72, s70, 20\n" \
  "s_add_u32 s72, s72, s86\n" \
  "s_mul_i32 s70, s70, 3\n" \
  "s_add_u32 s70, s70, s74\n" \
  "s_lshl_b32 s70, s70, 12\n" \
  "s_branch Llr5_%=\n" \
  "Lso4_%=:\n" \
  "s_cmp_eq_u32 s75, 1\n" \
  "s_cbranch_scc1 Lso4_1_%=\n" \
  "s_cmp_eq_u32 s75, 2\n" \
  "s_cbranch_scc1 Lso4_2_%=\n" \
  "Lso4_0_%=:\n" \
  "buffer_store_dwordx4 v[16:19], v53, s[48:51], s73 offen sc1\n" \
  "buffer_store_dwordx4 v[20:23], v54, s[48:51], s73 offen sc1\n" \
  "buffer_store_dwordx4 v[24:27], v55, s[48:51], s73 offen sc1\n" \
  "buffer_store_dwordx4 v[28:31], v56, s[48:51], s73 offen sc1\n" \
  "s_branch Lsr4_%=\n" \
  "Lso4_1_%=:\n" \
  "buffer_store_dwordx4 v[16:19], v57, s[48:51], s73 offen sc1\n" \
  "buffer_store_dwordx4 v[20:23], v58, s[48:51], s73 offen sc1\n" \
  "buffer_store_dwordx4 v[24:27], v59, s[48:51], s73 offen sc1\n" \
  "buffer_store_dwordx4 v[28:31], v60, s[48:51], s73 offen sc1\n" \
  "s_branch Lsr4_%=\n" \
  "Lso4_2_%=:\n" \
  "buffer_store_dwordx4 v[16:19], v61, s[48:51], s73 offen sc1\n" \
  "buffer_store_dwordx4 v[20:23], v62, s[48:51], s73 offen sc1\n" \
  "buffer_store_dwordx4 v[24:27], v63, s[48:51], s73 offen sc1\n" \
  "buffer_store_dwordx4 v[28:31], v64, s[48:51], s73 offen sc1\n" \
  "s_branch Lsr4_%=\n" \
  "Llo6_%=:\n" \
  "s_mov_b64 s[44:45], %[rs2]\n" \
  "s_mov_b64 s[48:49], %[out]\n" \
  "s_mov_b32 s50, 0x10000000\n" \
  "s_and_b32 s75, s71, 3\n" \
  "s_bfe_u32 s71, s71, 0x80002\n" \
  "s_lshl_b32 s73, s71, 20\n" \
  "s_add_u32 s73, s73, s86\n" \
  "s_mul_i32 s71, s71, 3\n" \
  "s_add_u32 s71, s71, s75\n" \
  "s_lshl_b32 s71, s71, 12\n" \
  "s_branch Llr6_%=\n" \
  "Lend_%=:\n" \
  "s_waitcnt vmcnt(0) lgkmcnt(0)\n" \
  "v_mov_b32 v46, 0xfff\n" \
  "ds_write_b32 v44, v46\n" \
  "s_waitcnt lgkmcnt(0)\n" \
  ""
#define PULL_ASM_CLOBBERS "v0", "v1", "v2", "v3", "v4", "v5", "v6", "v7", "v8", "v9", "v10", "v11", "v12", "v13", "v14", "v15", "v16", "v17", "v18", "v19", "v20", "v21", "v22", "v23", "v24", "v25", "v26", "v27", "v28", "v29", "v30", "v31", "v32", "v33", "v34", "v35", "v36", "v37", "v38", "v39", "v40", "v41", "v42", "v43", "v44", "v45", "v46", "v47", "v48", "v49", "v50", "v51", "v52", "v53", "v54", "v55", "v56", "v57", "v58", "v59", "v60", "v61", "v62", "v63", "v64", "s36", "s37", "s38", "s39", "s40", "s41", "s42", "s43", "s44", "s45", "s46", "s47", "s48", "s49", "s50", "s51", "s52", "s53", "s54", "s55", "s56", "s57", "s58", "s59", "s60", "s61", "s62", "s63", "s64", "s65", "s66", "s67", "s68", "s69", "s70", "s71", "s72", "s73", "s74", "s75", "s76", "s77", "s78", "s79", "s80", "s81", "s82", "s83", "s84", "s85", "s86", "vcc", "scc", "memory"
#define PULL_R 4

constexpr unsigned PULL_TOT = 3160;
constexpr unsigned SHG = 6, SH0 = 464;
constexpr unsigned PULL_NOP = 2392;
constexpr unsigned PULL_LIM2 = 1703;
static __device__ const __attribute__((aligned(16))) unsigned short PULL_LIST[7 * 3328] = {
0,1,2,3,4,5,6,7,8,9,10,11,12,13,14,15,16,17,18,19,20,21,22,23,24,25,26,27,28,29,30,31,32,33,34,35,36,37,38,39,40,41,42,43,44,45,46,47,48,49,50,51,52,53,54,55,56,57,58,59,60,61,62,63,64,65,66,67,68,69,70,71,72,73,74,75,76,77,78,79,80,81,82,83,84,85,86,87,88,89,90,91,92,93,94,95,96,97,98,99,100,101,102,103,104,105,106,107,108,109,110,111,112,113,114,115,116,117,118,119,120,121,122,123,124,125,126,127,576,578,580,582,584,586,588,590,16976,16977,16978,16979,16980,16981,640,641,642,643,644,645,646,647,648,649,650,651,652,653,654,655,656,657,658,659,660,661,662,663,664,665,666,667,668,669,670,671,672,673,674,675,676,677,678,679,680,681,682,683,684,685,686,687,688,689,690,691,692,693,694,695,696,697,698,699,700,701,702,703,704,705,706,707,708,709,710,711,712,713,714,715,716,717,718,719,720,721,722,723,724,725,726,727,728,729,730,731,732,733,734,735,736,737,738,739,740,741,742,743,744,745,746,747,748,749,750,751,752,753,754,755,756,757,758,759,760,761,762,763,764,765,766,767,1216,1217,866,1218,1219,869,1220,1221,872,1222,1223,875,1224,1225,878,1226,1227,769,1228,1229,772,1230,1231,775,792,17616,778,795,17617,781,798,17618,784,801,17619,787,804,17620,790,807,17621,793,1280,1281,1282,1283,1284,1285,1286,1287,1288,1289,1290,1291,1292,1293,1294,1295,1296,1297,1298,1299,1300,1301,1302,1303,1304,1305,1306,1307,1308,1309,1310,1311,1312,1313,1314,1315,1316,1317,1318,1319,1320,1321,1322,1323,1324,1325,1326,1327,1328,1329,1330,1331,1332,1333,1334,1335,1336,1337,1338,1339,1340,1341,1342,1343,1344,1345,1346,1347,1348,1349,1350,1351,1352,1353,1354,1355,1356,1357,1358,1359,1360,1361,1362,1363,1364,1365,1366,1367,1368,1369,1370,1371,1372,1373,1374,1375,1376,1377,1378,1379,1380,1381,1382,1383,1384,1385,1386,1387,1388,1389,1390,1391,1392,1393,1394,1395,1396,1397,1398,1399,1400,1401,1402,1403,1404,1405,1406,1407,1408,1422,1856,1857,1506,1411,1425,1858,1859,1509,1414,1428,1860,1861,1512,1417,1431,1862,1863,1515,1420,1434,1864,1865,1518,1423,1437,1866,1867,1409,1426,1440,1868,1869,1412,1429,1443,1870,1871,1415,1432,1446,1460,18256,1418,1435,1449,1463,18257,1421,1438,1452,1466,18258,1424,1441,1455,1469,18259,1427,1444,1458,1472,18260,1430,1447,1461,1475,18261,1433,1920,1921,1922,1923,1924,1925,1926,1927,1928,1929,1930,1931,1932,1933,1934,1935,1936,1937,1938,1939,1940,1941,1942,1943,1944,1945,1946,1947,1948,1949,1950,1951,1952,1953,1954,1955,1956,1957,1958,1959,1960,1961,1962,1963,1964,1965,1966,1967,1968,1969,1970,1971,1972,1973,1974,1975,1976,1977,1978,1979,1980,1981,1982,1983,1984,1985,1986,1987,1988,1989,1990,1991,1992,1993,1994,1995,1996,1997,1998,1999,2000,2001,2002,2003,2004,2005,2006,2007,2008,2009,2010,2011,2012,2013,2014,2015,2016,2017,2018,2019,2020,2021,2022,2023,2024,2025,2026,2027,2028,2029,2030,2031,2032,2033,2034,2035,2036,2037,2038,2039,2040,2041,2042,2043,2044,2045,2046,2047,2048,2062,2076,2090,2496,2497,2146,2051,2065,2079,2093,2498,2499,2149,2054,2068,2082,2096,2500,2501,2152,2057,2071,2085,2099,2502,2503,2155,2060,2074,2088,2102,2504,2505,2158,2063,2077,2091,2105,2506,2507,2049,2066,2080,2094,2108,2508,2509,2052,2069,2083,2097,2111,2510,2511,2055,2072,2086,2100,2114,2128,18896,2058,2075,2089,2103,2117,2131,18897,2061,2078,2092,2106,2120,2134,18898,2064,2081,2095,2109,2123,2137,18899,2067,2084,2098,2112,2126,2140,18900,2070,2087,2101,2115,2129,2143,18901,2073,128,142,156,170,184,198,577,226,131,145,159,173,187,201,579,229,134,148,162,176,190,204,581,232,137,151,165,179,193,207,583,235,140,154,168,182,196,210,585,238,143,157,171,185,199,213,587,129,146,160,174,188,202,216,589,132,149,163,177,191,205,219,591,135,152,166,180,194,208,222,236,138,155,169,183,197,211,225,239,141,158,172,186,200,214,228,130,144,161,175,189,203,217,231,133,147,164,178,192,206,220,234,136,150,167,181,195,209,223,237,139,153,768,782,796,810,824,838,852,978,771,785,799,813,827,841,855,981,774,788,802,816,830,844,858,984,777,791,805,819,833,847,861,987,780,794,808,822,836,850,864,990,783,797,811,825,839,853,867,881,786,800,814,828,842,856,870,884,789,803,817,831,845,859,873,887,904,806,820,834,848,862,876,890,907,809,823,837,851,865,879,893,910,812,826,840,854,868,770,896,913,815,829,843,857,871,773,899,916,818,832,846,860,874,776,902,919,821,835,849,863,877,779,905,1520,1534,1436,1450,1464,1478,1492,1618,1523,1537,1439,1453,1467,1481,1495,1621,1526,1540,1442,1456,1470,1484,1498,1624,1529,1543,1445,1459,1473,1487,1501,1627,1532,1546,1448,1462,1476,1490,1504,1630,1535,1549,1451,1465,1479,1493,1507,1521,1538,1552,1454,1468,1482,1496,1510,1524,1541,1555,1457,1471,1485,1499,1513,1527,1544,1558,1572,1474,1488,1502,1516,1530,1547,1561,1575,1477,1491,1505,1519,1533,1550,1564,1578,1480,1494,1508,1410,1536,1553,1567,1581,1483,1497,1511,1413,1539,1556,1570,1584,1486,1500,1514,1416,1542,1559,1573,1587,1489,1503,1517,1419,1545,2160,2174,2188,2202,2104,2118,2132,2258,2163,2177,2191,2205,2107,2121,2135,2261,2166,2180,2194,2208,2110,2124,2138,2264,2169,2183,2197,2211,2113,2127,2141,2267,2172,2186,2200,2214,2116,2130,2144,2270,2175,2189,2203,2217,2119,2133,2147,2161,2178,2192,2206,2220,2122,2136,2150,2164,2181,2195,2209,2223,2125,2139,2153,2167,2184,2198,2212,2226,2240,2142,2156,2170,2187,2201,2215,2229,2243,2145,2159,2173,2190,2204,2218,2232,2246,2148,2050,2176,2193,2207,2221,2235,2249,2151,2053,2179,2196,2210,2224,2238,2252,2154,2056,2182,2199,2213,2227,2241,2255,2157,2059,2185,240,254,268,282,296,310,212,338,243,257,271,285,299,313,215,341,246,260,274,288,302,316,218,344,249,263,277,291,305,319,221,347,252,266,280,294,308,322,224,350,255,269,283,297,311,325,227,241,258,272,286,300,314,328,230,244,261,275,289,303,317,331,233,247,264,278,292,306,320,334,348,250,267,281,295,309,323,337,351,253,270,284,298,312,326,340,242,256,273,287,301,315,329,343,245,259,276,290,304,318,332,346,248,262,279,293,307,321,335,349,251,265,880,894,908,922,936,950,964,1090,883,897,911,925,939,953,967,1093,886,900,914,928,942,956,970,1096,889,903,917,931,945,959,973,1099,892,906,920,934,948,962,976,1102,895,909,923,937,951,965,979,993,898,912,926,940,954,968,982,996,901,915,929,943,957,971,985,999,1016,918,932,946,960,974,988,1002,1019,921,935,949,963,977,991,1005,1022,924,938,952,966,980,882,1008,1025,927,941,955,969,983,885,1011,1028,930,944,958,972,986,888,1014,1031,933,947,961,975,989,891,1017,1632,1646,1548,1562,1576,1590,1604,1730,1635,1649,1551,1565,1579,1593,1607,1733,1638,1652,1554,1568,1582,1596,1610,1736,1641,1655,1557,1571,1585,1599,1613,1739,1644,1658,1560,1574,1588,1602,1616,1742,1647,1661,1563,1577,1591,1605,1619,1633,1650,1664,1566,1580,1594,1608,1622,1636,1653,1667,1569,1583,1597,1611,1625,1639,1656,1670,1684,1586,1600,1614,1628,1642,1659,1673,1687,1589,1603,1617,1631,1645,1662,1676,1690,1592,1606,1620,1522,1648,1665,1679,1693,1595,1609,1623,1525,1651,1668,1682,1696,1598,1612,1626,1528,1654,1671,1685,1699,1601,1615,1629,1531,1657,2272,2286,2300,2314,2216,2230,2244,2370,2275,2289,2303,2317,2219,2233,2247,2373,2278,2292,2306,2320,2222,2236,2250,2376,2281,2295,2309,2323,2225,2239,2253,2379,2284,2298,2312,2326,2228,2242,2256,2382,2287,2301,2315,2329,2231,2245,2259,2273,2290,2304,2318,2332,2234,2248,2262,2276,2293,2307,2321,2335,2237,2251,2265,2279,2296,2310,2324,2338,2352,2254,2268,2282,2299,2313,2327,2341,2355,2257,2271,2285,2302,2316,2330,2344,2358,2260,2162,2288,2305,2319,2333,2347,2361,2263,2165,2291,2308,2322,2336,2350,2364,2266,2168,2294,2311,2325,2339,2353,2367,2269,2171,2297,352,366,380,394,408,422,324,450,355,369,383,397,411,425,327,453,358,372,386,400,414,428,330,456,361,375,389,403,417,431,333,459,364,378,392,406,420,434,336,462,367,381,395,409,423,437,339,353,370,384,398,412,426,440,342,356,373,387,401,415,429,443,345,359,376,390,404,418,432,446,460,362,379,393,407,421,435,449,463,365,382,396,410,424,438,452,354,368,385,399,413,427,441,455,357,371,388,402,416,430,444,458,360,374,391,405,419,433,447,461,363,377,992,1006,1020,1034,1048,1062,1076,1202,995,1009,1023,1037,1051,1065,1079,1205,998,1012,1026,1040,1054,1068,1082,1208,1001,1015,1029,1043,1057,1071,1085,1211,1004,1018,1032,1046,1060,1074,1088,1214,1007,1021,1035,1049,1063,1077,1091,1105,1010,1024,1038,1052,1066,1080,1094,1108,1013,1027,1041,1055,1069,1083,1097,1111,1128,1030,1044,1058,1072,1086,1100,1114,1131,1033,1047,1061,1075,1089,1103,1117,1134,1036,1050,1064,1078,1092,994,1120,1137,1039,1053,1067,1081,1095,997,1123,1140,1042,1056,1070,1084,1098,1000,1126,1143,1045,1059,1073,1087,1101,1003,1129,1744,1758,1660,1674,1688,1702,1716,1842,1747,1761,1663,1677,1691,1705,1719,1845,1750,1764,1666,1680,1694,1708,1722,1848,1753,1767,1669,1683,1697,1711,1725,1851,1756,1770,1672,1686,1700,1714,1728,1854,1759,1773,1675,1689,1703,1717,1731,1745,1762,1776,1678,1692,1706,1720,1734,1748,1765,1779,1681,1695,1709,1723,1737,1751,1768,1782,1796,1698,1712,1726,1740,1754,1771,1785,1799,1701,1715,1729,1743,1757,1774,1788,1802,1704,1718,1732,1634,1760,1777,1791,1805,1707,1721,1735,1637,1763,1780,1794,1808,1710,1724,1738,1640,1766,1783,1797,1811,1713,1727,1741,1643,1769,2384,2398,2412,2426,2328,2342,2356,2482,2387,2401,2415,2429,2331,2345,2359,2485,2390,2404,2418,2432,2334,2348,2362,2488,2393,2407,2421,2435,2337,2351,2365,2491,2396,2410,2424,2438,2340,2354,2368,2494,2399,2413,2427,2441,2343,2357,2371,2385,2402,2416,2430,2444,2346,2360,2374,2388,2405,2419,2433,2447,2349,2363,2377,2391,2408,2422,2436,2450,2464,2366,2380,2394,2411,2425,2439,2453,2467,2369,2383,2397,2414,2428,2442,2456,2470,2372,2274,2400,2417,2431,2445,2459,2473,2375,2277,2403,2420,2434,2448,2462,2476,2378,2280,2406,2423,2437,2451,2465,2479,2381,2283,2409,464,478,492,506,520,534,436,562,467,481,495,509,523,537,439,565,470,484,498,512,526,540,442,568,473,487,501,515,529,543,445,571,476,490,504,518,532,546,448,574,479,493,507,521,535,549,451,465,482,496,510,524,538,552,454,468,485,499,513,527,541,555,457,471,488,502,516,530,544,558,572,474,491,505,519,533,547,561,575,477,494,508,522,536,550,564,466,480,497,511,525,539,553,567,469,483,500,514,528,542,556,570,472,486,503,517,531,545,559,573,475,489,1104,1118,1132,1146,1160,1174,1188,1107,1121,1135,1149,1163,1177,1191,1110,1124,1138,1152,1166,1180,1194,1113,1127,1141,1155,1169,1183,1197,1116,1130,1144,1158,1172,1186,1200,1119,1133,1147,1161,1175,1189,1203,1122,1136,1150,1164,1178,1192,1206,1125,1139,1153,1167,1181,1195,1209,1142,1156,1170,1184,1198,1212,1145,1159,1173,1187,1201,1215,1148,1162,1176,1190,1204,1106,1151,1165,1179,1193,1207,1109,1154,1168,1182,1196,1210,1112,1157,1171,1185,1199,1213,1115,1772,1786,1800,1814,1828,1775,1789,1803,1817,1831,1778,1792,1806,1820,1834,1781,1795,1809,1823,1837,1784,1798,1812,1826,1840,1787,1801,1815,1829,1843,1790,1804,1818,1832,1846,1793,1807,1821,1835,1849,1810,1824,1838,1852,1813,1827,1841,1855,1816,1830,1844,1746,1819,1833,1847,1749,1822,1836,1850,1752,1825,1839,1853,1755,2440,2454,2468,2443,2457,2471,2446,2460,2474,2449,2463,2477,2452,2466,2480,2455,2469,2483,2458,2472,2486,2461,2475,2489,2478,2492,2481,2495,2484,2386,2487,2389,2490,2392,2493,2395,548,551,554,557,560,563,566,569,32768,32769,32770,32772,32773,32774,32776,32777,32778,32780,32781,32782,32784,32785,32786,32788,32789,32790,32792,32793,32794,32796,32797,32798,32800,32801,32802,32804,32805,32806,32808,32809,32810,32812,32813,32814,32816,32817,32818,32820,32821,32822,32824,32825,32826,32828,32829,32830,32832,32833,32834,32836,32837,32838,32840,32841,32842,32844,32845,32846,32848,32849,32850,32852,32853,32854,32856,32857,32858,32860,32861,32862,32864,32865,32866,32868,32869,32870,32872,32873,32874,32876,32877,32878,32880,32881,32882,32884,32885,32886,32888,32889,32890,32892,32893,32894,32896,32897,32898,32900,32901,32902,32904,32905,32906,32908,32909,32910,32912,32913,32914,32916,32917,32918,32920,32921,32922,32924,32925,32926,32928,32929,32930,32932,32933,32934,32936,32937,32938,32940,32941,32942,32944,32945,32946,32948,32949,32950,32952,32953,32954,32956,32957,32958,32960,32961,32962,32964,32965,32966,32968,32969,32970,32972,32973,32974,32976,32977,32978,32980,32981,32982,32984,32985,32986,32988,32989,32990,32992,32993,32994,32996,32997,32998,33000,33001,33002,33004,33005,33006,33008,33009,33010,33012,33013,33014,33016,33017,33018,33020,33021,33022,33024,33025,33026,33028,33029,33030,33032,33033,33034,33036,33037,33038,33040,33041,33042,33044,33045,33046,33048,33049,33050,33052,33053,33054,33056,33057,33058,33060,33061,33062,33064,33065,33066,33068,33069,33070,33072,33073,33074,33076,33077,33078,33080,33081,33082,33084,33085,33086,33088,33089,33090,33092,33093,33094,33096,33097,33098,33100,33101,33102,33104,33105,33106,33108,33109,33110,33112,33113,33114,33116,33117,33118,33120,33121,33122,33124,33125,33126,33128,33129,33130,33132,33133,33134,33136,33137,33138,33140,33141,33142,33144,33145,33146,33148,33149,33150,33152,33153,33154,33156,33157,33158,33160,33161,33162,33164,33165,33166,33168,33169,33170,33172,33173,33174,33176,33177,33178,33180,33181,33182,33184,33185,33186,33188,33189,33190,33192,33193,33194,33196,33197,33198,33200,33201,33202,33204,33205,33206,33208,33209,33210,33212,33213,33214,33216,33217,33218,33220,33221,33222,33224,33225,33226,33228,33229,33230,33232,33233,33234,33236,33237,33238,33240,33241,33242,33244,33245,33246,33248,33249,33250,33252,33253,33254,33256,33257,33258,33260,33261,33262,33264,33265,33266,33268,33269,33270,33272,33273,33274,33276,33277,33278,33280,33281,33282,33284,33285,33286,33288,33289,33290,33292,33293,33294,33296,33297,33298,33300,33301,33302,33304,33305,33306,33308,33309,33310,33312,33313,33314,33316,33317,33318,33320,33321,33322,33324,33325,33326,33328,33329,33330,33332,33333,33334,33336,33337,33338,33340,33341,33342,33344,33345,33346,33348,33349,33350,33352,33353,33354,33356,33357,33358,33360,33361,33362,33364,33365,33366,33368,33369,33370,33372,33373,33374,33376,33377,33378,33380,33381,33382,33384,33385,33386,33388,33389,33390,33392,33393,33394,33396,33397,33398,33400,33401,33402,33404,33405,33406,33408,33409,33410,33412,33413,33414,33416,33417,33418,33420,33421,33422,33424,33425,33426,33428,33429,33430,33432,33433,33434,33436,33437,33438,33440,33441,33442,33444,33445,33446,33448,33449,33450,33452,33453,33454,33456,33457,33458,33460,33461,33462,33464,33465,33466,33468,33469,33470,33472,33473,33474,33476,33477,33478,33480,33481,33482,33484,33485,33486,33488,33489,33490,33492,33493,33494,33496,33497,33498,33500,33501,33502,33504,33505,33506,33508,33509,33510,33512,33513,33514,33516,33517,33518,33520,33521,33522,33524,33525,33526,33528,33529,33530,33532,33533,33534,33536,33537,33538,33540,33541,33542,33544,33545,33546,33548,33549,33550,33552,33553,33554,33556,33557,33558,33560,33561,33562,33564,33565,33566,33568,33569,33570,33572,33573,33574,33576,33577,33578,33580,33581,33582,33584,33585,33586,33588,33589,33590,33592,33593,33594,33596,33597,33598,33600,33601,33602,33604,33605,33606,33608,33609,33610,33612,33613,33614,33616,33617,33618,33620,33621,33622,33624,33625,33626,33628,33629,33630,33632,33633,33634,33636,33637,33638,33640,33641,33642,33644,33645,33646,33648,33649,33650,33652,33653,33654,33656,33657,33658,33660,33661,33662,33664,33665,33666,33668,33669,33670,33672,33673,33674,33676,33677,33678,33680,33681,33682,33684,33685,33686,33688,33689,33690,33692,33693,33694,33696,33697,33698,33700,33701,33702,33704,33705,33706,33708,33709,33710,33712,33713,33714,33716,33717,33718,33720,33721,33722,33724,33725,33726,33728,33729,33730,33732,33733,33734,33736,33737,33738,33740,33741,33742,33744,33745,33746,33748,33749,33750,33752,33753,33754,33756,33757,33758,33760,33761,33762,33764,33765,33766,33768,33769,33770,33772,33773,33774,33776,33777,33778,33780,33781,33782,33784,33785,33786,33788,33789,33790,0,0,0,0,0,0,0,0,0,0,0,0,0,0,0,0,0,0,0,0,0,0,0,0,0,0,0,0,0,0,0,0,0,0,0,0,0,0,0,0,0,0,0,0,0,0,0,0,0,0,0,0,0,0,0,0,0,0,0,0,0,0,0,0,0,0,0,0,0,0,0,0,0,0,0,0,0,0,0,0,0,0,0,0,0,0,0,0,0,0,0,0,0,0,0,0,0,0,0,0,0,0,0,0,0,0,0,0,0,0,0,0,0,0,0,0,0,0,0,0,0,0,0,0,0,0,0,0,0,0,0,0,0,0,0,0,0,0,0,0,0,0,0,0,0,0,0,0,0,0,0,0,0,0,0,0,0,0,0,0,0,0,0,0,0,0,0,0,
0,1,2,3,4,5,6,7,8,9,10,11,12,13,14,15,16,17,18,19,20,21,22,23,24,25,26,27,28,29,30,31,32,33,34,35,36,37,38,39,40,41,42,43,44,45,46,47,48,49,50,51,52,53,54,55,56,57,58,59,60,61,62,63,64,65,66,67,68,69,70,71,72,73,74,75,76,77,78,79,80,81,82,83,84,85,86,87,88,89,90,91,92,93,94,95,96,97,98,99,100,101,102,103,104,105,106,107,108,109,110,111,112,113,114,115,116,117,118,119,120,121,122,123,124,125,126,127,576,578,580,582,584,586,588,590,16976,16977,16978,16979,16980,16981,640,641,642,643,644,645,646,647,648,649,650,651,652,653,654,655,656,657,658,659,660,661,662,663,664,665,666,667,668,669,670,671,672,673,674,675,676,677,678,679,680,681,682,683,684,685,686,687,688,689,690,691,692,693,694,695,696,697,698,699,700,701,702,703,704,705,706,707,708,709,710,711,712,713,714,715,716,717,718,719,720,721,722,723,724,725,726,727,728,729,730,731,732,733,734,735,736,737,738,739,740,741,742,743,744,745,746,747,748,749,750,751,752,753,754,755,756,757,758,759,760,761,762,763,764,765,766,767,1216,1217,866,1218,1219,869,1220,1221,872,1222,1223,875,1224,1225,878,1226,1227,769,1228,1229,772,1230,1231,775,792,17616,778,795,17617,781,798,17618,784,801,17619,787,804,17620,790,807,17621,793,1280,1281,1282,1283,1284,1285,1286,1287,1288,1289,1290,1291,1292,1293,1294,1295,1296,1297,1298,1299,1300,1301,1302,1303,1304,1305,1306,1307,1308,1309,1310,1311,1312,1313,1314,1315,1316,1317,1318,1319,1320,1321,1322,1323,1324,1325,1326,1327,1328,1329,1330,1331,1332,1333,1334,1335,1336,1337,1338,1339,1340,1341,1342,1343,1344,1345,1346,1347,1348,1349,1350,1351,1352,1353,1354,1355,1356,1357,1358,1359,1360,1361,1362,1363,1364,1365,1366,1367,1368,1369,1370,1371,1372,1373,1374,1375,1376,1377,1378,1379,1380,1381,1382,1383,1384,1385,1386,1387,1388,1389,1390,1391,1392,1393,1394,1395,1396,1397,1398,1399,1400,1401,1402,1403,1404,1405,1406,1407,1408,1422,1856,1857,1506,1411,1425,1858,1859,1509,1414,1428,1860,1861,1512,1417,1431,1862,1863,1515,1420,1434,1864,1865,1518,1423,1437,1866,1867,1409,1426,1440,1868,1869,1412,1429,1443,1870,1871,1415,1432,1446,1460,18256,1418,1435,1449,1463,18257,1421,1438,1452,1466,18258,1424,1441,1455,1469,18259,1427,1444,1458,1472,18260,1430,1447,1461,1475,18261,1433,1920,1921,1922,1923,1924,1925,1926,1927,1928,1929,1930,1931,1932,1933,1934,1935,1936,1937,1938,1939,1940,1941,1942,1943,1944,1945,1946,1947,1948,1949,1950,1951,1952,1953,1954,1955,1956,1957,1958,1959,1960,1961,1962,1963,1964,1965,1966,1967,1968,1969,1970,1971,1972,1973,1974,1975,1976,1977,1978,1979,1980,1981,1982,1983,1984,1985,1986,1987,1988,1989,1990,1991,1992,1993,1994,1995,1996,1997,1998,1999,2000,2001,2002,2003,2004,2005,2006,2007,2008,2009,2010,2011,2012,2013,2014,2015,2016,2017,2018,2019,2020,2021,2022,2023,2024,2025,2026,2027,2028,2029,2030,2031,2032,2033,2034,2035,2036,2037,2038,2039,2040,2041,2042,2043,2044,2045,2046,2047,2048,2062,2076,2090,2496,2497,2146,2051,2065,2079,2093,2498,2499,2149,2054,2068,2082,2096,2500,2501,2152,2057,2071,2085,2099,2502,2503,2155,2060,2074,2088,2102,2504,2505,2158,2063,2077,2091,2105,2506,2507,2049,2066,2080,2094,2108,2508,2509,2052,2069,2083,2097,2111,2510,2511,2055,2072,2086,2100,2114,2128,18896,2058,2075,2089,2103,2117,2131,18897,2061,2078,2092,2106,2120,2134,18898,2064,2081,2095,2109,2123,2137,18899,2067,2084,2098,2112,2126,2140,18900,2070,2087,2101,2115,2129,2143,18901,2073,128,142,156,170,184,198,577,226,131,145,159,173,187,201,579,229,134,148,162,176,190,204,581,232,137,151,165,179,193,207,583,235,140,154,168,182,196,210,585,238,143,157,171,185,199,213,587,129,146,160,174,188,202,216,589,132,149,163,177,191,205,219,591,135,152,166,180,194,208,222,236,138,155,169,183,197,211,225,239,141,158,172,186,200,214,228,130,144,161,175,189,203,217,231,133,147,164,178,192,206,220,234,136,150,167,181,195,209,223,237,139,153,768,782,796,810,824,838,852,978,771,785,799,813,827,841,855,981,774,788,802,816,830,844,858,984,777,791,805,819,833,847,861,987,780,794,808,822,836,850,864,990,783,797,811,825,839,853,867,881,786,800,814,828,842,856,870,884,789,803,817,831,845,859,873,887,904,806,820,834,848,862,876,890,907,809,823,837,851,865,879,893,910,812,826,840,854,868,770,896,913,815,829,843,857,871,773,899,916,818,832,846,860,874,776,902,919,821,835,849,863,877,779,905,1520,1534,1436,1450,1464,1478,1492,1618,1523,1537,1439,1453,1467,1481,1495,1621,1526,1540,1442,1456,1470,1484,1498,1624,1529,1543,1445,1459,1473,1487,1501,1627,1532,1546,1448,1462,1476,1490,1504,1630,1535,1549,1451,1465,1479,1493,1507,1521,1538,1552,1454,1468,1482,1496,1510,1524,1541,1555,1457,1471,1485,1499,1513,1527,1544,1558,1572,1474,1488,1502,1516,1530,1547,1561,1575,1477,1491,1505,1519,1533,1550,1564,1578,1480,1494,1508,1410,1536,1553,1567,1581,1483,1497,1511,1413,1539,1556,1570,1584,1486,1500,1514,1416,1542,1559,1573,1587,1489,1503,1517,1419,1545,2160,2174,2188,2202,2104,2118,2132,2258,2163,2177,2191,2205,2107,2121,2135,2261,2166,2180,2194,2208,2110,2124,2138,2264,2169,2183,2197,2211,2113,2127,2141,2267,2172,2186,2200,2214,2116,2130,2144,2270,2175,2189,2203,2217,2119,2133,2147,2161,2178,2192,2206,2220,2122,2136,2150,2164,2181,2195,2209,2223,2125,2139,2153,2167,2184,2198,2212,2226,2240,2142,2156,2170,2187,2201,2215,2229,2243,2145,2159,2173,2190,2204,2218,2232,2246,2148,2050,2176,2193,2207,2221,2235,2249,2151,2053,2179,2196,2210,2224,2238,2252,2154,2056,2182,2199,2213,2227,2241,2255,2157,2059,2185,240,254,268,282,296,310,212,338,243,257,271,285,299,313,215,341,246,260,274,288,302,316,218,344,249,263,277,291,305,319,221,347,252,266,280,294,308,322,224,350,255,269,283,297,311,325,227,241,258,272,286,300,314,328,230,244,261,275,289,303,317,331,233,247,264,278,292,306,320,334,348,250,267,281,295,309,323,337,351,253,270,284,298,312,326,340,242,256,273,287,301,315,329,343,245,259,276,290,304,318,332,346,248,262,279,293,307,321,335,349,251,265,880,894,908,922,936,950,964,1090,883,897,911,925,939,953,967,1093,886,900,914,928,942,956,970,1096,889,903,917,931,945,959,973,1099,892,906,920,934,948,962,976,1102,895,909,923,937,951,965,979,993,898,912,926,940,954,968,982,996,901,915,929,943,957,971,985,999,1016,918,932,946,960,974,988,1002,1019,921,935,949,963,977,991,1005,1022,924,938,952,966,980,882,1008,1025,927,941,955,969,983,885,1011,1028,930,944,958,972,986,888,1014,1031,933,947,961,975,989,891,1017,1632,1646,1548,1562,1576,1590,1604,1730,1635,1649,1551,1565,1579,1593,1607,1733,1638,1652,1554,1568,1582,1596,1610,1736,1641,1655,1557,1571,1585,1599,1613,1739,1644,1658,1560,1574,1588,1602,1616,1742,1647,1661,1563,1577,1591,1605,1619,1633,1650,1664,1566,1580,1594,1608,1622,1636,1653,1667,1569,1583,1597,1611,1625,1639,1656,1670,1684,1586,1600,1614,1628,1642,1659,1673,1687,1589,1603,1617,1631,1645,1662,1676,1690,1592,1606,1620,1522,1648,1665,1679,1693,1595,1609,1623,1525,1651,1668,1682,1696,1598,1612,1626,1528,1654,1671,1685,1699,1601,1615,1629,1531,1657,2272,2286,2300,2314,2216,2230,2244,2370,2275,2289,2303,2317,2219,2233,2247,2373,2278,2292,2306,2320,2222,2236,2250,2376,2281,2295,2309,2323,2225,2239,2253,2379,2284,2298,2312,2326,2228,2242,2256,2382,2287,2301,2315,2329,2231,2245,2259,2273,2290,2304,2318,2332,2234,2248,2262,2276,2293,2307,2321,2335,2237,2251,2265,2279,2296,2310,2324,2338,2352,2254,2268,2282,2299,2313,2327,2341,2355,2257,2271,2285,2302,2316,2330,2344,2358,2260,2162,2288,2305,2319,2333,2347,2361,2263,2165,2291,2308,2322,2336,2350,2364,2266,2168,2294,2311,2325,2339,2353,2367,2269,2171,2297,352,366,380,394,408,422,324,450,355,369,383,397,411,425,327,453,358,372,386,400,414,428,330,456,361,375,389,403,417,431,333,459,364,378,392,406,420,434,336,462,367,381,395,409,423,437,339,353,370,384,398,412,426,440,342,356,373,387,401,415,429,443,345,359,376,390,404,418,432,446,460,362,379,393,407,421,435,449,463,365,382,396,410,424,438,452,354,368,385,399,413,427,441,455,357,371,388,402,416,430,444,458,360,374,391,405,419,433,447,461,363,377,992,1006,1020,1034,1048,1062,1076,1202,995,1009,1023,1037,1051,1065,1079,1205,998,1012,1026,1040,1054,1068,1082,1208,1001,1015,1029,1043,1057,1071,1085,1211,1004,1018,1032,1046,1060,1074,1088,1214,1007,1021,1035,1049,1063,1077,1091,1105,1010,1024,1038,1052,1066,1080,1094,1108,1013,1027,1041,1055,1069,1083,1097,1111,1128,1030,1044,1058,1072,1086,1100,1114,1131,1033,1047,1061,1075,1089,1103,1117,1134,1036,1050,1064,1078,1092,994,1120,1137,1039,1053,1067,1081,1095,997,1123,1140,1042,1056,1070,1084,1098,1000,1126,1143,1045,1059,1073,1087,1101,1003,1129,1744,1758,1660,1674,1688,1702,1716,1842,1747,1761,1663,1677,1691,1705,1719,1845,1750,1764,1666,1680,1694,1708,1722,1848,1753,1767,1669,1683,1697,1711,1725,1851,1756,1770,1672,1686,1700,1714,1728,1854,1759,1773,1675,1689,1703,1717,1731,1745,1762,1776,1678,1692,1706,1720,1734,1748,1765,1779,1681,1695,1709,1723,1737,1751,1768,1782,1796,1698,1712,1726,1740,1754,1771,1785,1799,1701,1715,1729,1743,1757,1774,1788,1802,1704,1718,1732,1634,1760,1777,1791,1805,1707,1721,1735,1637,1763,1780,1794,1808,1710,1724,1738,1640,1766,1783,1797,1811,1713,1727,1741,1643,1769,2384,2398,2412,2426,2328,2342,2356,2482,2387,2401,2415,2429,2331,2345,2359,2485,2390,2404,2418,2432,2334,2348,2362,2488,2393,2407,2421,2435,2337,2351,2365,2491,2396,2410,2424,2438,2340,2354,2368,2494,2399,2413,2427,2441,2343,2357,2371,2385,2402,2416,2430,2444,2346,2360,2374,2388,2405,2419,2433,2447,2349,2363,2377,2391,2408,2422,2436,2450,2464,2366,2380,2394,2411,2425,2439,2453,2467,2369,2383,2397,2414,2428,2442,2456,2470,2372,2274,2400,2417,2431,2445,2459,2473,2375,2277,2403,2420,2434,2448,2462,2476,2378,2280,2406,2423,2437,2451,2465,2479,2381,2283,2409,464,478,492,506,520,534,436,562,467,481,495,509,523,537,439,565,470,484,498,512,526,540,442,568,473,487,501,515,529,543,445,571,476,490,504,518,532,546,448,574,479,493,507,521,535,549,451,465,482,496,510,524,538,552,454,468,485,499,513,527,541,555,457,471,488,502,516,530,544,558,572,474,491,505,519,533,547,561,575,477,494,508,522,536,550,564,466,480,497,511,525,539,553,567,469,483,500,514,528,542,556,570,472,486,503,517,531,545,559,573,475,489,1104,1118,1132,1146,1160,1174,1188,1107,1121,1135,1149,1163,1177,1191,1110,1124,1138,1152,1166,1180,1194,1113,1127,1141,1155,1169,1183,1197,1116,1130,1144,1158,1172,1186,1200,1119,1133,1147,1161,1175,1189,1203,1122,1136,1150,1164,1178,1192,1206,1125,1139,1153,1167,1181,1195,1209,1142,1156,1170,1184,1198,1212,1145,1159,1173,1187,1201,1215,1148,1162,1176,1190,1204,1106,1151,1165,1179,1193,1207,1109,1154,1168,1182,1196,1210,1112,1157,1171,1185,1199,1213,1115,1772,1786,1800,1814,1828,1775,1789,1803,1817,1831,1778,1792,1806,1820,1834,1781,1795,1809,1823,1837,1784,1798,1812,1826,1840,1787,1801,1815,1829,1843,1790,1804,1818,1832,1846,1793,1807,1821,1835,1849,1810,1824,1838,1852,1813,1827,1841,1855,1816,1830,1844,1746,1819,1833,1847,1749,1822,1836,1850,1752,1825,1839,1853,1755,2440,2454,2468,2443,2457,2471,2446,2460,2474,2449,2463,2477,2452,2466,2480,2455,2469,2483,2458,2472,2486,2461,2475,2489,2478,2492,2481,2495,2484,2386,2487,2389,2490,2392,2493,2395,548,551,554,557,560,563,566,569,32768,32769,32770,32772,32773,32774,32776,32777,32778,32780,32781,32782,32784,32785,32786,32788,32789,32790,32792,32793,32794,32796,32797,32798,32800,32801,32802,32804,32805,32806,32808,32809,32810,32812,32813,32814,32816,32817,32818,32820,32821,32822,32824,32825,32826,32828,32829,32830,32832,32833,32834,32836,32837,32838,32840,32841,32842,32844,32845,32846,32848,32849,32850,32852,32853,32854,32856,32857,32858,32860,32861,32862,32864,32865,32866,32868,32869,32870,32872,32873,32874,32876,32877,32878,32880,32881,32882,32884,32885,32886,32888,32889,32890,32892,32893,32894,32896,32897,32898,32900,32901,32902,32904,32905,32906,32908,32909,32910,32912,32913,32914,32916,32917,32918,32920,32921,32922,32924,32925,32926,32928,32929,32930,32932,32933,32934,32936,32937,32938,32940,32941,32942,32944,32945,32946,32948,32949,32950,32952,32953,32954,32956,32957,32958,32960,32961,32962,32964,32965,32966,32968,32969,32970,32972,32973,32974,32976,32977,32978,32980,32981,32982,32984,32985,32986,32988,32989,32990,32992,32993,32994,32996,32997,32998,33000,33001,33002,33004,33005,33006,33008,33009,33010,33012,33013,33014,33016,33017,33018,33020,33021,33022,33024,33025,33026,33028,33029,33030,33032,33033,33034,33036,33037,33038,33040,33041,33042,33044,33045,33046,33048,33049,33050,33052,33053,33054,33056,33057,33058,33060,33061,33062,33064,33065,33066,33068,33069,33070,33072,33073,33074,33076,33077,33078,33080,33081,33082,33084,33085,33086,33088,33089,33090,33092,33093,33094,33096,33097,33098,33100,33101,33102,33104,33105,33106,33108,33109,33110,33112,33113,33114,33116,33117,33118,33120,33121,33122,33124,33125,33126,33128,33129,33130,33132,33133,33134,33136,33137,33138,33140,33141,33142,33144,33145,33146,33148,33149,33150,33152,33153,33154,33156,33157,33158,33160,33161,33162,33164,33165,33166,33168,33169,33170,33172,33173,33174,33176,33177,33178,33180,33181,33182,33184,33185,33186,33188,33189,33190,33192,33193,33194,33196,33197,33198,33200,33201,33202,33204,33205,33206,33208,33209,33210,33212,33213,33214,33216,33217,33218,33220,33221,33222,33224,33225,33226,33228,33229,33230,33232,33233,33234,33236,33237,33238,33240,33241,33242,33244,33245,33246,33248,33249,33250,33252,33253,33254,33256,33257,33258,33260,33261,33262,33264,33265,33266,33268,33269,33270,33272,33273,33274,33276,33277,33278,33280,33281,33282,33284,33285,33286,33288,33289,33290,33292,33293,33294,33296,33297,33298,33300,33301,33302,33304,33305,33306,33308,33309,33310,33312,33313,33314,33316,33317,33318,33320,33321,33322,33324,33325,33326,33328,33329,33330,33332,33333,33334,33336,33337,33338,33340,33341,33342,33344,33345,33346,33348,33349,33350,33352,33353,33354,33356,33357,33358,33360,33361,33362,33364,33365,33366,33368,33369,33370,33372,33373,33374,33376,33377,33378,33380,33381,33382,33384,33385,33386,33388,33389,33390,33392,33393,33394,33396,33397,33398,33400,33401,33402,33404,33405,33406,33408,33409,33410,33412,33413,33414,33416,33417,33418,33420,33421,33422,33424,33425,33426,33428,33429,33430,33432,33433,33434,33436,33437,33438,33440,33441,33442,33444,33445,33446,33448,33449,33450,33452,33453,33454,33456,33457,33458,33460,33461,33462,33464,33465,33466,33468,33469,33470,33472,33473,33474,33476,33477,33478,33480,33481,33482,33484,33485,33486,33488,33489,33490,33492,33493,33494,33496,33497,33498,33500,33501,33502,33504,33505,33506,33508,33509,33510,33512,33513,33514,33516,33517,33518,33520,33521,33522,33524,33525,33526,33528,33529,33530,33532,33533,33534,33536,33537,33538,33540,33541,33542,33544,33545,33546,33548,33549,33550,33552,33553,33554,33556,33557,33558,33560,33561,33562,33564,33565,33566,33568,33569,33570,33572,33573,33574,33576,33577,33578,33580,33581,33582,33584,33585,33586,33588,33589,33590,33592,33593,33594,33596,33597,33598,33600,33601,33602,33604,33605,33606,33608,33609,33610,33612,33613,33614,33616,33617,33618,33620,33621,33622,33624,33625,33626,33628,33629,33630,33632,33633,33634,33636,33637,33638,33640,33641,33642,33644,33645,33646,33648,33649,33650,33652,33653,33654,33656,33657,33658,33660,33661,33662,33664,33665,33666,33668,33669,33670,33672,33673,33674,33676,33677,33678,33680,33681,33682,33684,33685,33686,33688,33689,33690,33692,33693,33694,33696,33697,33698,33700,33701,33702,33704,33705,33706,33708,33709,33710,33712,33713,33714,33716,33717,33718,33720,33721,33722,33724,33725,33726,33728,33729,33730,33732,33733,33734,33736,33737,33738,33740,33741,33742,33744,33745,33746,33748,33749,33750,33752,33753,33754,33756,33757,33758,33760,33761,33762,33764,33765,33766,33768,33769,33770,33772,33773,33774,33776,33777,33778,33780,33781,33782,33784,33785,33786,33788,33789,33790,0,0,0,0,0,0,0,0,0,0,0,0,0,0,0,0,0,0,0,0,0,0,0,0,0,0,0,0,0,0,0,0,0,0,0,0,0,0,0,0,0,0,0,0,0,0,0,0,0,0,0,0,0,0,0,0,0,0,0,0,0,0,0,0,0,0,0,0,0,0,0,0,0,0,0,0,0,0,0,0,0,0,0,0,0,0,0,0,0,0,0,0,0,0,0,0,0,0,0,0,0,0,0,0,0,0,0,0,0,0,0,0,0,0,0,0,0,0,0,0,0,0,0,0,0,0,0,0,0,0,0,0,0,0,0,0,0,0,0,0,0,0,0,0,0,0,0,0,0,0,0,0,0,0,0,0,0,0,0,0,0,0,0,0,0,0,0,0,
0,1,2,3,4,5,6,7,8,9,10,11,12,13,14,15,16,17,18,19,20,21,22,23,24,25,26,27,28,29,30,31,32,33,34,35,36,37,38,39,40,41,42,43,44,45,46,47,48,49,50,51,52,53,54,55,56,57,58,59,60,61,62,63,64,65,66,67,68,69,70,71,72,73,74,75,76,77,78,79,80,81,82,83,84,85,86,87,88,89,90,91,92,93,94,95,96,97,98,99,100,101,102,103,104,105,106,107,108,109,110,111,112,113,114,115,116,117,118,119,120,121,122,123,124,125,126,127,576,578,580,582,584,586,588,590,16976,138,16977,141,16978,144,16979,147,16980,150,16981,153,640,641,642,643,644,645,646,647,648,649,650,651,652,653,654,655,656,657,658,659,660,661,662,663,664,665,666,667,668,669,670,671,672,673,674,675,676,677,678,679,680,681,682,683,684,685,686,687,688,689,690,691,692,693,694,695,696,697,698,699,700,701,702,703,704,705,706,707,708,709,710,711,712,713,714,715,716,717,718,719,720,721,722,723,724,725,726,727,728,729,730,731,732,733,734,735,736,737,738,739,740,741,742,743,744,745,746,747,748,749,750,751,752,753,754,755,756,757,758,759,760,761,762,763,764,765,766,767,1216,1217,866,1218,1219,869,1220,1221,872,1222,1223,875,1224,1225,878,1226,1227,769,1228,1229,772,1230,1231,775,792,17616,778,795,17617,781,798,17618,784,801,17619,787,804,17620,790,807,17621,793,1280,1281,1282,1283,1284,1285,1286,1287,1288,1289,1290,1291,1292,1293,1294,1295,1296,1297,1298,1299,1300,1301,1302,1303,1304,1305,1306,1307,1308,1309,1310,1311,1312,1313,1314,1315,1316,1317,1318,1319,1320,1321,1322,1323,1324,1325,1326,1327,1328,1329,1330,1331,1332,1333,1334,1335,1336,1337,1338,1339,1340,1341,1342,1343,1344,1345,1346,1347,1348,1349,1350,1351,1352,1353,1354,1355,1356,1357,1358,1359,1360,1361,1362,1363,1364,1365,1366,1367,1368,1369,1370,1371,1372,1373,1374,1375,1376,1377,1378,1379,1380,1381,1382,1383,1384,1385,1386,1387,1388,1389,1390,1391,1392,1393,1394,1395,1396,1397,1398,1399,1400,1401,1402,1403,1404,1405,1406,1407,1408,1422,1856,1857,1506,1411,1425,1858,1859,1509,1414,1428,1860,1861,1512,1417,1431,1862,1863,1515,1420,1434,1864,1865,1518,1423,1437,1866,1867,1409,1426,1440,1868,1869,1412,1429,1443,1870,1871,1415,1432,1446,1460,18256,1418,1435,1449,1463,18257,1421,1438,1452,1466,18258,1424,1441,1455,1469,18259,1427,1444,1458,1472,18260,1430,1447,1461,1475,18261,1433,1920,1921,1922,1923,1924,1925,1926,1927,1928,1929,1930,1931,1932,1933,1934,1935,1936,1937,1938,1939,1940,1941,1942,1943,1944,1945,1946,1947,1948,1949,1950,1951,1952,1953,1954,1955,1956,1957,1958,1959,1960,1961,1962,1963,1964,1965,1966,1967,1968,1969,1970,1971,1972,1973,1974,1975,1976,1977,1978,1979,1980,1981,1982,1983,1984,1985,1986,1987,1988,1989,1990,1991,1992,1993,1994,1995,1996,1997,1998,1999,2000,2001,2002,2003,2004,2005,2006,2007,2008,2009,2010,2011,2012,2013,2014,2015,2016,2017,2018,2019,2020,2021,2022,2023,2024,2025,2026,2027,2028,2029,2030,2031,2032,2033,2034,2035,2036,2037,2038,2039,2040,2041,2042,2043,2044,2045,2046,2047,2048,2062,2076,2090,2496,2497,2146,2051,2065,2079,2093,2498,2499,2149,2054,2068,2082,2096,2500,2501,2152,2057,2071,2085,2099,2502,2503,2155,2060,2074,2088,2102,2504,2505,2158,2063,2077,2091,2105,2506,2507,2049,2066,2080,2094,2108,2508,2509,2052,2069,2083,2097,2111,2510,2511,2055,2072,2086,2100,2114,2128,18896,2058,2075,2089,2103,2117,2131,18897,2061,2078,2092,2106,2120,2134,18898,2064,2081,2095,2109,2123,2137,18899,2067,2084,2098,2112,2126,2140,18900,2070,2087,2101,2115,2129,2143,18901,2073,128,142,156,170,184,198,577,226,131,145,159,173,187,201,579,229,134,148,162,176,190,204,581,232,137,151,165,179,193,207,583,235,140,154,168,182,196,210,585,238,143,157,171,185,199,213,587,129,146,160,174,188,202,216,589,132,149,163,177,191,205,219,591,135,152,166,180,194,208,222,236,250,155,169,183,197,211,225,239,253,158,172,186,200,214,228,130,256,161,175,189,203,217,231,133,259,164,178,192,206,220,234,136,262,167,181,195,209,223,237,139,265,768,782,796,810,824,838,852,978,771,785,799,813,827,841,855,981,774,788,802,816,830,844,858,984,777,791,805,819,833,847,861,987,780,794,808,822,836,850,864,990,783,797,811,825,839,853,867,881,786,800,814,828,842,856,870,884,789,803,817,831,845,859,873,887,904,806,820,834,848,862,876,890,907,809,823,837,851,865,879,893,910,812,826,840,854,868,770,896,913,815,829,843,857,871,773,899,916,818,832,846,860,874,776,902,919,821,835,849,863,877,779,905,1520,1534,1436,1450,1464,1478,1492,1618,1523,1537,1439,1453,1467,1481,1495,1621,1526,1540,1442,1456,1470,1484,1498,1624,1529,1543,1445,1459,1473,1487,1501,1627,1532,1546,1448,1462,1476,1490,1504,1630,1535,1549,1451,1465,1479,1493,1507,1521,1538,1552,1454,1468,1482,1496,1510,1524,1541,1555,1457,1471,1485,1499,1513,1527,1544,1558,1572,1474,1488,1502,1516,1530,1547,1561,1575,1477,1491,1505,1519,1533,1550,1564,1578,1480,1494,1508,1410,1536,1553,1567,1581,1483,1497,1511,1413,1539,1556,1570,1584,1486,1500,1514,1416,1542,1559,1573,1587,1489,1503,1517,1419,1545,2160,2174,2188,2202,2104,2118,2132,2258,2163,2177,2191,2205,2107,2121,2135,2261,2166,2180,2194,2208,2110,2124,2138,2264,2169,2183,2197,2211,2113,2127,2141,2267,2172,2186,2200,2214,2116,2130,2144,2270,2175,2189,2203,2217,2119,2133,2147,2161,2178,2192,2206,2220,2122,2136,2150,2164,2181,2195,2209,2223,2125,2139,2153,2167,2184,2198,2212,2226,2240,2142,2156,2170,2187,2201,2215,2229,2243,2145,2159,2173,2190,2204,2218,2232,2246,2148,2050,2176,2193,2207,2221,2235,2249,2151,2053,2179,2196,2210,2224,2238,2252,2154,2056,2182,2199,2213,2227,2241,2255,2157,2059,2185,240,254,268,282,296,310,212,338,243,257,271,285,299,313,215,341,246,260,274,288,302,316,218,344,249,263,277,291,305,319,221,347,252,266,280,294,308,322,224,350,255,269,283,297,311,325,227,241,258,272,286,300,314,328,230,244,261,275,289,303,317,331,233,247,264,278,292,306,320,334,348,362,267,281,295,309,323,337,351,365,270,284,298,312,326,340,242,368,273,287,301,315,329,343,245,371,276,290,304,318,332,346,248,374,279,293,307,321,335,349,251,377,880,894,908,922,936,950,964,1090,883,897,911,925,939,953,967,1093,886,900,914,928,942,956,970,1096,889,903,917,931,945,959,973,1099,892,906,920,934,948,962,976,1102,895,909,923,937,951,965,979,993,898,912,926,940,954,968,982,996,901,915,929,943,957,971,985,999,1016,918,932,946,960,974,988,1002,1019,921,935,949,963,977,991,1005,1022,924,938,952,966,980,882,1008,1025,927,941,955,969,983,885,1011,1028,930,944,958,972,986,888,1014,1031,933,947,961,975,989,891,1017,1632,1646,1548,1562,1576,1590,1604,1730,1635,1649,1551,1565,1579,1593,1607,1733,1638,1652,1554,1568,1582,1596,1610,1736,1641,1655,1557,1571,1585,1599,1613,1739,1644,1658,1560,1574,1588,1602,1616,1742,1647,1661,1563,1577,1591,1605,1619,1633,1650,1664,1566,1580,1594,1608,1622,1636,1653,1667,1569,1583,1597,1611,1625,1639,1656,1670,1684,1586,1600,1614,1628,1642,1659,1673,1687,1589,1603,1617,1631,1645,1662,1676,1690,1592,1606,1620,1522,1648,1665,1679,1693,1595,1609,1623,1525,1651,1668,1682,1696,1598,1612,1626,1528,1654,1671,1685,1699,1601,1615,1629,1531,1657,2272,2286,2300,2314,2216,2230,2244,2370,2275,2289,2303,2317,2219,2233,2247,2373,2278,2292,2306,2320,2222,2236,2250,2376,2281,2295,2309,2323,2225,2239,2253,2379,2284,2298,2312,2326,2228,2242,2256,2382,2287,2301,2315,2329,2231,2245,2259,2273,2290,2304,2318,2332,2234,2248,2262,2276,2293,2307,2321,2335,2237,2251,2265,2279,2296,2310,2324,2338,2352,2254,2268,2282,2299,2313,2327,2341,2355,2257,2271,2285,2302,2316,2330,2344,2358,2260,2162,2288,2305,2319,2333,2347,2361,2263,2165,2291,2308,2322,2336,2350,2364,2266,2168,2294,2311,2325,2339,2353,2367,2269,2171,2297,352,366,380,394,408,422,324,450,355,369,383,397,411,425,327,453,358,372,386,400,414,428,330,456,361,375,389,403,417,431,333,459,364,378,392,406,420,434,336,462,367,381,395,409,423,437,339,353,370,384,398,412,426,440,342,356,373,387,401,415,429,443,345,359,376,390,404,418,432,446,460,474,379,393,407,421,435,449,463,477,382,396,410,424,438,452,354,480,385,399,413,427,441,455,357,483,388,402,416,430,444,458,360,486,391,405,419,433,447,461,363,489,992,1006,1020,1034,1048,1062,1076,1202,995,1009,1023,1037,1051,1065,1079,1205,998,1012,1026,1040,1054,1068,1082,1208,1001,1015,1029,1043,1057,1071,1085,1211,1004,1018,1032,1046,1060,1074,1088,1214,1007,1021,1035,1049,1063,1077,1091,1105,1010,1024,1038,1052,1066,1080,1094,1108,1013,1027,1041,1055,1069,1083,1097,1111,1128,1030,1044,1058,1072,1086,1100,1114,1131,1033,1047,1061,1075,1089,1103,1117,1134,1036,1050,1064,1078,1092,994,1120,1137,1039,1053,1067,1081,1095,997,1123,1140,1042,1056,1070,1084,1098,1000,1126,1143,1045,1059,1073,1087,1101,1003,1129,1744,1758,1660,1674,1688,1702,1716,1842,1747,1761,1663,1677,1691,1705,1719,1845,1750,1764,1666,1680,1694,1708,1722,1848,1753,1767,1669,1683,1697,1711,1725,1851,1756,1770,1672,1686,1700,1714,1728,1854,1759,1773,1675,1689,1703,1717,1731,1745,1762,1776,1678,1692,1706,1720,1734,1748,1765,1779,1681,1695,1709,1723,1737,1751,1768,1782,1796,1698,1712,1726,1740,1754,1771,1785,1799,1701,1715,1729,1743,1757,1774,1788,1802,1704,1718,1732,1634,1760,1777,1791,1805,1707,1721,1735,1637,1763,1780,1794,1808,1710,1724,1738,1640,1766,1783,1797,1811,1713,1727,1741,1643,1769,2384,2398,2412,2426,2328,2342,2356,2482,2387,2401,2415,2429,2331,2345,2359,2485,2390,2404,2418,2432,2334,2348,2362,2488,2393,2407,2421,2435,2337,2351,2365,2491,2396,2410,2424,2438,2340,2354,2368,2494,2399,2413,2427,2441,2343,2357,2371,2385,2402,2416,2430,2444,2346,2360,2374,2388,2405,2419,2433,2447,2349,2363,2377,2391,2408,2422,2436,2450,2464,2366,2380,2394,2411,2425,2439,2453,2467,2369,2383,2397,2414,2428,2442,2456,2470,2372,2274,2400,2417,2431,2445,2459,2473,2375,2277,2403,2420,2434,2448,2462,2476,2378,2280,2406,2423,2437,2451,2465,2479,2381,2283,2409,464,478,492,506,520,534,436,562,467,481,495,509,523,537,439,565,470,484,498,512,526,540,442,568,473,487,501,515,529,543,445,571,476,490,504,518,532,546,448,574,479,493,507,521,535,549,451,465,482,496,510,524,538,552,454,468,485,499,513,527,541,555,457,471,488,502,516,530,544,558,572,491,505,519,533,547,561,575,494,508,522,536,550,564,466,497,511,525,539,553,567,469,500,514,528,542,556,570,472,503,517,531,545,559,573,475,1104,1118,1132,1146,1160,1174,1188,1107,1121,1135,1149,1163,1177,1191,1110,1124,1138,1152,1166,1180,1194,1113,1127,1141,1155,1169,1183,1197,1116,1130,1144,1158,1172,1186,1200,1119,1133,1147,1161,1175,1189,1203,1122,1136,1150,1164,1178,1192,1206,1125,1139,1153,1167,1181,1195,1209,1142,1156,1170,1184,1198,1212,1145,1159,1173,1187,1201,1215,1148,1162,1176,1190,1204,1106,1151,1165,1179,1193,1207,1109,1154,1168,1182,1196,1210,1112,1157,1171,1185,1199,1213,1115,1772,1786,1800,1814,1828,1775,1789,1803,1817,1831,1778,1792,1806,1820,1834,1781,1795,1809,1823,1837,1784,1798,1812,1826,1840,1787,1801,1815,1829,1843,1790,1804,1818,1832,1846,1793,1807,1821,1835,1849,1810,1824,1838,1852,1813,1827,1841,1855,1816,1830,1844,1746,1819,1833,1847,1749,1822,1836,1850,1752,1825,1839,1853,1755,2440,2454,2468,2443,2457,2471,2446,2460,2474,2449,2463,2477,2452,2466,2480,2455,2469,2483,2458,2472,2486,2461,2475,2489,2478,2492,2481,2495,2484,2386,2487,2389,2490,2392,2493,2395,548,551,554,557,560,563,566,569,32768,32769,32770,32772,32773,32774,32776,32777,32778,32780,32781,32782,32784,32785,32786,32788,32789,32790,32792,32793,32794,32796,32797,32798,32800,32801,32802,32804,32805,32806,32808,32809,32810,32812,32813,32814,32816,32817,32818,32820,32821,32822,32824,32825,32826,32828,32829,32830,32832,32833,32834,32836,32837,32838,32840,32841,32842,32844,32845,32846,32848,32849,32850,32852,32853,32854,32856,32857,32858,32860,32861,32862,32864,32865,32866,32868,32869,32870,32872,32873,32874,32876,32877,32878,32880,32881,32882,32884,32885,32886,32888,32889,32890,32892,32893,32894,32896,32897,32898,32900,32901,32902,32904,32905,32906,32908,32909,32910,32912,32913,32914,32916,32917,32918,32920,32921,32922,32924,32925,32926,32928,32929,32930,32932,32933,32934,32936,32937,32938,32940,32941,32942,32944,32945,32946,32948,32949,32950,32952,32953,32954,32956,32957,32958,32960,32961,32962,32964,32965,32966,32968,32969,32970,32972,32973,32974,32976,32977,32978,32980,32981,32982,32984,32985,32986,32988,32989,32990,32992,32993,32994,32996,32997,32998,33000,33001,33002,33004,33005,33006,33008,33009,33010,33012,33013,33014,33016,33017,33018,33020,33021,33022,33024,33025,33026,33028,33029,33030,33032,33033,33034,33036,33037,33038,33040,33041,33042,33044,33045,33046,33048,33049,33050,33052,33053,33054,33056,33057,33058,33060,33061,33062,33064,33065,33066,33068,33069,33070,33072,33073,33074,33076,33077,33078,33080,33081,33082,33084,33085,33086,33088,33089,33090,33092,33093,33094,33096,33097,33098,33100,33101,33102,33104,33105,33106,33108,33109,33110,33112,33113,33114,33116,33117,33118,33120,33121,33122,33124,33125,33126,33128,33129,33130,33132,33133,33134,33136,33137,33138,33140,33141,33142,33144,33145,33146,33148,33149,33150,33152,33153,33154,33156,33157,33158,33160,33161,33162,33164,33165,33166,33168,33169,33170,33172,33173,33174,33176,33177,33178,33180,33181,33182,33184,33185,33186,33188,33189,33190,33192,33193,33194,33196,33197,33198,33200,33201,33202,33204,33205,33206,33208,33209,33210,33212,33213,33214,33216,33217,33218,33220,33221,33222,33224,33225,33226,33228,33229,33230,33232,33233,33234,33236,33237,33238,33240,33241,33242,33244,33245,33246,33248,33249,33250,33252,33253,33254,33256,33257,33258,33260,33261,33262,33264,33265,33266,33268,33269,33270,33272,33273,33274,33276,33277,33278,33280,33281,33282,33284,33285,33286,33288,33289,33290,33292,33293,33294,33296,33297,33298,33300,33301,33302,33304,33305,33306,33308,33309,33310,33312,33313,33314,33316,33317,33318,33320,33321,33322,33324,33325,33326,33328,33329,33330,33332,33333,33334,33336,33337,33338,33340,33341,33342,33344,33345,33346,33348,33349,33350,33352,33353,33354,33356,33357,33358,33360,33361,33362,33364,33365,33366,33368,33369,33370,33372,33373,33374,33376,33377,33378,33380,33381,33382,33384,33385,33386,33388,33389,33390,33392,33393,33394,33396,33397,33398,33400,33401,33402,33404,33405,33406,33408,33409,33410,33412,33413,33414,33416,33417,33418,33420,33421,33422,33424,33425,33426,33428,33429,33430,33432,33433,33434,33436,33437,33438,33440,33441,33442,33444,33445,33446,33448,33449,33450,33452,33453,33454,33456,33457,33458,33460,33461,33462,33464,33465,33466,33468,33469,33470,33472,33473,33474,33476,33477,33478,33480,33481,33482,33484,33485,33486,33488,33489,33490,33492,33493,33494,33496,33497,33498,33500,33501,33502,33504,33505,33506,33508,33509,33510,33512,33513,33514,33516,33517,33518,33520,33521,33522,33524,33525,33526,33528,33529,33530,33532,33533,33534,33536,33537,33538,33540,33541,33542,33544,33545,33546,33548,33549,33550,33552,33553,33554,33556,33557,33558,33560,33561,33562,33564,33565,33566,33568,33569,33570,33572,33573,33574,33576,33577,33578,33580,33581,33582,33584,33585,33586,33588,33589,33590,33592,33593,33594,33596,33597,33598,33600,33601,33602,33604,33605,33606,33608,33609,33610,33612,33613,33614,33616,33617,33618,33620,33621,33622,33624,33625,33626,33628,33629,33630,33632,33633,33634,33636,33637,33638,33640,33641,33642,33644,33645,33646,33648,33649,33650,33652,33653,33654,33656,33657,33658,33660,33661,33662,33664,33665,33666,33668,33669,33670,33672,33673,33674,33676,33677,33678,33680,33681,33682,33684,33685,33686,33688,33689,33690,33692,33693,33694,33696,33697,33698,33700,33701,33702,33704,33705,33706,33708,33709,33710,33712,33713,33714,33716,33717,33718,33720,33721,33722,33724,33725,33726,33728,33729,33730,33732,33733,33734,33736,33737,33738,33740,33741,33742,33744,33745,33746,33748,33749,33750,33752,33753,33754,33756,33757,33758,33760,33761,33762,33764,33765,33766,33768,33769,33770,33772,33773,33774,33776,33777,33778,33780,33781,33782,33784,33785,33786,33788,33789,33790,0,0,0,0,0,0,0,0,0,0,0,0,0,0,0,0,0,0,0,0,0,0,0,0,0,0,0,0,0,0,0,0,0,0,0,0,0,0,0,0,0,0,0,0,0,0,0,0,0,0,0,0,0,0,0,0,0,0,0,0,0,0,0,0,0,0,0,0,0,0,0,0,0,0,0,0,0,0,0,0,0,0,0,0,0,0,0,0,0,0,0,0,0,0,0,0,0,0,0,0,0,0,0,0,0,0,0,0,0,0,0,0,0,0,0,0,0,0,0,0,0,0,0,0,0,0,0,0,0,0,0,0,0,0,0,0,0,0,0,0,0,0,0,0,0,0,0,0,0,0,0,0,0,0,0,0,0,0,0,0,0,0,0,0,0,0,0,0,
0,1,2,3,4,5,6,7,8,9,10,11,12,13,14,15,16,17,18,19,20,21,22,23,24,25,26,27,28,29,30,31,32,33,34,35,36,37,38,39,40,41,42,43,44,45,46,47,48,49,50,51,52,53,54,55,56,57,58,59,60,61,62,63,64,65,66,67,68,69,70,71,72,73,74,75,76,77,78,79,80,81,82,83,84,85,86,87,88,89,90,91,92,93,94,95,96,97,98,99,100,101,102,103,104,105,106,107,108,109,110,111,112,113,114,115,116,117,118,119,120,121,122,123,124,125,126,127,576,577,578,579,580,581,582,583,584,585,586,587,588,589,590,591,16976,138,16977,141,16978,144,16979,147,16980,150,16981,153,640,641,642,643,644,645,646,647,648,649,650,651,652,653,654,655,656,657,658,659,660,661,662,663,664,665,666,667,668,669,670,671,672,673,674,675,676,677,678,679,680,681,682,683,684,685,686,687,688,689,690,691,692,693,694,695,696,697,698,699,700,701,702,703,704,705,706,707,708,709,710,711,712,713,714,715,716,717,718,719,720,721,722,723,724,725,726,727,728,729,730,731,732,733,734,735,736,737,738,739,740,741,742,743,744,745,746,747,748,749,750,751,752,753,754,755,756,757,758,759,760,761,762,763,764,765,766,767,768,1216,1217,866,771,1218,1219,869,774,1220,1221,872,777,1222,1223,875,780,1224,1225,878,783,1226,1227,769,786,1228,1229,772,789,1230,1231,775,792,806,17616,778,795,809,17617,781,798,812,17618,784,801,815,17619,787,804,818,17620,790,807,821,17621,793,1280,1281,1282,1283,1284,1285,1286,1287,1288,1289,1290,1291,1292,1293,1294,1295,1296,1297,1298,1299,1300,1301,1302,1303,1304,1305,1306,1307,1308,1309,1310,1311,1312,1313,1314,1315,1316,1317,1318,1319,1320,1321,1322,1323,1324,1325,1326,1327,1328,1329,1330,1331,1332,1333,1334,1335,1336,1337,1338,1339,1340,1341,1342,1343,1344,1345,1346,1347,1348,1349,1350,1351,1352,1353,1354,1355,1356,1357,1358,1359,1360,1361,1362,1363,1364,1365,1366,1367,1368,1369,1370,1371,1372,1373,1374,1375,1376,1377,1378,1379,1380,1381,1382,1383,1384,1385,1386,1387,1388,1389,1390,1391,1392,1393,1394,1395,1396,1397,1398,1399,1400,1401,1402,1403,1404,1405,1406,1407,1408,1422,1436,1856,1857,1506,1411,1425,1439,1858,1859,1509,1414,1428,1442,1860,1861,1512,1417,1431,1445,1862,1863,1515,1420,1434,1448,1864,1865,1518,1423,1437,1451,1866,1867,1409,1426,1440,1454,1868,1869,1412,1429,1443,1457,1870,1871,1415,1432,1446,1460,1474,18256,1418,1435,1449,1463,1477,18257,1421,1438,1452,1466,1480,18258,1424,1441,1455,1469,1483,18259,1427,1444,1458,1472,1486,18260,1430,1447,1461,1475,1489,18261,1433,1920,1921,1922,1923,1924,1925,1926,1927,1928,1929,1930,1931,1932,1933,1934,1935,1936,1937,1938,1939,1940,1941,1942,1943,1944,1945,1946,1947,1948,1949,1950,1951,1952,1953,1954,1955,1956,1957,1958,1959,1960,1961,1962,1963,1964,1965,1966,1967,1968,1969,1970,1971,1972,1973,1974,1975,1976,1977,1978,1979,1980,1981,1982,1983,1984,1985,1986,1987,1988,1989,1990,1991,1992,1993,1994,1995,1996,1997,1998,1999,2000,2001,2002,2003,2004,2005,2006,2007,2008,2009,2010,2011,2012,2013,2014,2015,2016,2017,2018,2019,2020,2021,2022,2023,2024,2025,2026,2027,2028,2029,2030,2031,2032,2033,2034,2035,2036,2037,2038,2039,2040,2041,2042,2043,2044,2045,2046,2047,2048,2062,2076,2090,2104,2496,2497,2146,2051,2065,2079,2093,2107,2498,2499,2149,2054,2068,2082,2096,2110,2500,2501,2152,2057,2071,2085,2099,2113,2502,2503,2155,2060,2074,2088,2102,2116,2504,2505,2158,2063,2077,2091,2105,2119,2506,2507,2049,2066,2080,2094,2108,2122,2508,2509,2052,2069,2083,2097,2111,2125,2510,2511,2055,2072,2086,2100,2114,2128,2142,18896,2058,2075,2089,2103,2117,2131,2145,18897,2061,2078,2092,2106,2120,2134,2148,18898,2064,2081,2095,2109,2123,2137,2151,18899,2067,2084,2098,2112,2126,2140,2154,18900,2070,2087,2101,2115,2129,2143,2157,18901,2073,128,142,156,170,184,198,212,226,131,145,159,173,187,201,215,229,134,148,162,176,190,204,218,232,137,151,165,179,193,207,221,235,140,154,168,182,196,210,224,238,143,157,171,185,199,213,227,129,146,160,174,188,202,216,230,132,149,163,177,191,205,219,233,135,152,166,180,194,208,222,236,250,155,169,183,197,211,225,239,253,158,172,186,200,214,228,130,256,161,175,189,203,217,231,133,259,164,178,192,206,220,234,136,262,167,181,195,209,223,237,139,265,880,782,796,810,824,838,852,978,883,785,799,813,827,841,855,981,886,788,802,816,830,844,858,984,889,791,805,819,833,847,861,987,892,794,808,822,836,850,864,990,895,797,811,825,839,853,867,881,898,800,814,828,842,856,870,884,901,803,817,831,845,859,873,887,904,918,820,834,848,862,876,890,907,921,823,837,851,865,879,893,910,924,826,840,854,868,770,896,913,927,829,843,857,871,773,899,916,930,832,846,860,874,776,902,919,933,835,849,863,877,779,905,1520,1534,1548,1450,1464,1478,1492,1618,1523,1537,1551,1453,1467,1481,1495,1621,1526,1540,1554,1456,1470,1484,1498,1624,1529,1543,1557,1459,1473,1487,1501,1627,1532,1546,1560,1462,1476,1490,1504,1630,1535,1549,1563,1465,1479,1493,1507,1521,1538,1552,1566,1468,1482,1496,1510,1524,1541,1555,1569,1471,1485,1499,1513,1527,1544,1558,1572,1586,1488,1502,1516,1530,1547,1561,1575,1589,1491,1505,1519,1533,1550,1564,1578,1592,1494,1508,1410,1536,1553,1567,1581,1595,1497,1511,1413,1539,1556,1570,1584,1598,1500,1514,1416,1542,1559,1573,1587,1601,1503,1517,1419,1545,2160,2174,2188,2202,2216,2118,2132,2258,2163,2177,2191,2205,2219,2121,2135,2261,2166,2180,2194,2208,2222,2124,2138,2264,2169,2183,2197,2211,2225,2127,2141,2267,2172,2186,2200,2214,2228,2130,2144,2270,2175,2189,2203,2217,2231,2133,2147,2161,2178,2192,2206,2220,2234,2136,2150,2164,2181,2195,2209,2223,2237,2139,2153,2167,2184,2198,2212,2226,2240,2254,2156,2170,2187,2201,2215,2229,2243,2257,2159,2173,2190,2204,2218,2232,2246,2260,2050,2176,2193,2207,2221,2235,2249,2263,2053,2179,2196,2210,2224,2238,2252,2266,2056,2182,2199,2213,2227,2241,2255,2269,2059,2185,240,254,268,282,296,310,324,338,243,257,271,285,299,313,327,341,246,260,274,288,302,316,330,344,249,263,277,291,305,319,333,347,252,266,280,294,308,322,336,350,255,269,283,297,311,325,339,241,258,272,286,300,314,328,342,244,261,275,289,303,317,331,345,247,264,278,292,306,320,334,348,362,267,281,295,309,323,337,351,365,270,284,298,312,326,340,242,368,273,287,301,315,329,343,245,371,276,290,304,318,332,346,248,374,279,293,307,321,335,349,251,377,992,894,908,922,936,950,964,1090,995,897,911,925,939,953,967,1093,998,900,914,928,942,956,970,1096,1001,903,917,931,945,959,973,1099,1004,906,920,934,948,962,976,1102,1007,909,923,937,951,965,979,993,1010,912,926,940,954,968,982,996,1013,915,929,943,957,971,985,999,1016,1030,932,946,960,974,988,1002,1019,1033,935,949,963,977,991,1005,1022,1036,938,952,966,980,882,1008,1025,1039,941,955,969,983,885,1011,1028,1042,944,958,972,986,888,1014,1031,1045,947,961,975,989,891,1017,1632,1646,1660,1562,1576,1590,1604,1730,1635,1649,1663,1565,1579,1593,1607,1733,1638,1652,1666,1568,1582,1596,1610,1736,1641,1655,1669,1571,1585,1599,1613,1739,1644,1658,1672,1574,1588,1602,1616,1742,1647,1661,1675,1577,1591,1605,1619,1633,1650,1664,1678,1580,1594,1608,1622,1636,1653,1667,1681,1583,1597,1611,1625,1639,1656,1670,1684,1698,1600,1614,1628,1642,1659,1673,1687,1701,1603,1617,1631,1645,1662,1676,1690,1704,1606,1620,1522,1648,1665,1679,1693,1707,1609,1623,1525,1651,1668,1682,1696,1710,1612,1626,1528,1654,1671,1685,1699,1713,1615,1629,1531,1657,2272,2286,2300,2314,2328,2230,2244,2370,2275,2289,2303,2317,2331,2233,2247,2373,2278,2292,2306,2320,2334,2236,2250,2376,2281,2295,2309,2323,2337,2239,2253,2379,2284,2298,2312,2326,2340,2242,2256,2382,2287,2301,2315,2329,2343,2245,2259,2273,2290,2304,2318,2332,2346,2248,2262,2276,2293,2307,2321,2335,2349,2251,2265,2279,2296,2310,2324,2338,2352,2366,2268,2282,2299,2313,2327,2341,2355,2369,2271,2285,2302,2316,2330,2344,2358,2372,2162,2288,2305,2319,2333,2347,2361,2375,2165,2291,2308,2322,2336,2350,2364,2378,2168,2294,2311,2325,2339,2353,2367,2381,2171,2297,352,366,380,394,408,422,436,450,355,369,383,397,411,425,439,453,358,372,386,400,414,428,442,456,361,375,389,403,417,431,445,459,364,378,392,406,420,434,448,462,367,381,395,409,423,437,451,353,370,384,398,412,426,440,454,356,373,387,401,415,429,443,457,359,376,390,404,418,432,446,460,474,379,393,407,421,435,449,463,477,382,396,410,424,438,452,354,480,385,399,413,427,441,455,357,483,388,402,416,430,444,458,360,486,391,405,419,433,447,461,363,489,1104,1006,1020,1034,1048,1062,1076,1202,1107,1009,1023,1037,1051,1065,1079,1205,1110,1012,1026,1040,1054,1068,1082,1208,1113,1015,1029,1043,1057,1071,1085,1211,1116,1018,1032,1046,1060,1074,1088,1214,1119,1021,1035,1049,1063,1077,1091,1105,1122,1024,1038,1052,1066,1080,1094,1108,1125,1027,1041,1055,1069,1083,1097,1111,1128,1142,1044,1058,1072,1086,1100,1114,1131,1145,1047,1061,1075,1089,1103,1117,1134,1148,1050,1064,1078,1092,994,1120,1137,1151,1053,1067,1081,1095,997,1123,1140,1154,1056,1070,1084,1098,1000,1126,1143,1157,1059,1073,1087,1101,1003,1129,1744,1758,1772,1674,1688,1702,1716,1842,1747,1761,1775,1677,1691,1705,1719,1845,1750,1764,1778,1680,1694,1708,1722,1848,1753,1767,1781,1683,1697,1711,1725,1851,1756,1770,1784,1686,1700,1714,1728,1854,1759,1773,1787,1689,1703,1717,1731,1745,1762,1776,1790,1692,1706,1720,1734,1748,1765,1779,1793,1695,1709,1723,1737,1751,1768,1782,1796,1810,1712,1726,1740,1754,1771,1785,1799,1813,1715,1729,1743,1757,1774,1788,1802,1816,1718,1732,1634,1760,1777,1791,1805,1819,1721,1735,1637,1763,1780,1794,1808,1822,1724,1738,1640,1766,1783,1797,1811,1825,1727,1741,1643,1769,2384,2398,2412,2426,2440,2342,2356,2482,2387,2401,2415,2429,2443,2345,2359,2485,2390,2404,2418,2432,2446,2348,2362,2488,2393,2407,2421,2435,2449,2351,2365,2491,2396,2410,2424,2438,2452,2354,2368,2494,2399,2413,2427,2441,2455,2357,2371,2385,2402,2416,2430,2444,2458,2360,2374,2388,2405,2419,2433,2447,2461,2363,2377,2391,2408,2422,2436,2450,2464,2478,2380,2394,2411,2425,2439,2453,2467,2481,2383,2397,2414,2428,2442,2456,2470,2484,2274,2400,2417,2431,2445,2459,2473,2487,2277,2403,2420,2434,2448,2462,2476,2490,2280,2406,2423,2437,2451,2465,2479,2493,2283,2409,464,478,492,506,520,534,548,562,467,481,495,509,523,537,551,565,470,484,498,512,526,540,554,568,473,487,501,515,529,543,557,571,476,490,504,518,532,546,560,574,479,493,507,521,535,549,563,465,482,496,510,524,538,552,566,468,485,499,513,527,541,555,569,471,488,502,516,530,544,558,572,491,505,519,533,547,561,575,494,508,522,536,550,564,466,497,511,525,539,553,567,469,500,514,528,542,556,570,472,503,517,531,545,559,573,475,1118,1132,1146,1160,1174,1188,1121,1135,1149,1163,1177,1191,1124,1138,1152,1166,1180,1194,1127,1141,1155,1169,1183,1197,1130,1144,1158,1172,1186,1200,1133,1147,1161,1175,1189,1203,1136,1150,1164,1178,1192,1206,1139,1153,1167,1181,1195,1209,1156,1170,1184,1198,1212,1159,1173,1187,1201,1215,1162,1176,1190,1204,1106,1165,1179,1193,1207,1109,1168,1182,1196,1210,1112,1171,1185,1199,1213,1115,1786,1800,1814,1828,1789,1803,1817,1831,1792,1806,1820,1834,1795,1809,1823,1837,1798,1812,1826,1840,1801,1815,1829,1843,1804,1818,1832,1846,1807,1821,1835,1849,1824,1838,1852,1827,1841,1855,1830,1844,1746,1833,1847,1749,1836,1850,1752,1839,1853,1755,2454,2468,2457,2471,2460,2474,2463,2477,2466,2480,2469,2483,2472,2486,2475,2489,2492,2495,2386,2389,2392,2395,32768,32769,32770,32772,32773,32774,32776,32777,32778,32780,32781,32782,32784,32785,32786,32788,32789,32790,32792,32793,32794,32796,32797,32798,32800,32801,32802,32804,32805,32806,32808,32809,32810,32812,32813,32814,32816,32817,32818,32820,32821,32822,32824,32825,32826,32828,32829,32830,32832,32833,32834,32836,32837,32838,32840,32841,32842,32844,32845,32846,32848,32849,32850,32852,32853,32854,32856,32857,32858,32860,32861,32862,32864,32865,32866,32868,32869,32870,32872,32873,32874,32876,32877,32878,32880,32881,32882,32884,32885,32886,32888,32889,32890,32892,32893,32894,32896,32897,32898,32900,32901,32902,32904,32905,32906,32908,32909,32910,32912,32913,32914,32916,32917,32918,32920,32921,32922,32924,32925,32926,32928,32929,32930,32932,32933,32934,32936,32937,32938,32940,32941,32942,32944,32945,32946,32948,32949,32950,32952,32953,32954,32956,32957,32958,32960,32961,32962,32964,32965,32966,32968,32969,32970,32972,32973,32974,32976,32977,32978,32980,32981,32982,32984,32985,32986,32988,32989,32990,32992,32993,32994,32996,32997,32998,33000,33001,33002,33004,33005,33006,33008,33009,33010,33012,33013,33014,33016,33017,33018,33020,33021,33022,33024,33025,33026,33028,33029,33030,33032,33033,33034,33036,33037,33038,33040,33041,33042,33044,33045,33046,33048,33049,33050,33052,33053,33054,33056,33057,33058,33060,33061,33062,33064,33065,33066,33068,33069,33070,33072,33073,33074,33076,33077,33078,33080,33081,33082,33084,33085,33086,33088,33089,33090,33092,33093,33094,33096,33097,33098,33100,33101,33102,33104,33105,33106,33108,33109,33110,33112,33113,33114,33116,33117,33118,33120,33121,33122,33124,33125,33126,33128,33129,33130,33132,33133,33134,33136,33137,33138,33140,33141,33142,33144,33145,33146,33148,33149,33150,33152,33153,33154,33156,33157,33158,33160,33161,33162,33164,33165,33166,33168,33169,33170,33172,33173,33174,33176,33177,33178,33180,33181,33182,33184,33185,33186,33188,33189,33190,33192,33193,33194,33196,33197,33198,33200,33201,33202,33204,33205,33206,33208,33209,33210,33212,33213,33214,33216,33217,33218,33220,33221,33222,33224,33225,33226,33228,33229,33230,33232,33233,33234,33236,33237,33238,33240,33241,33242,33244,33245,33246,33248,33249,33250,33252,33253,33254,33256,33257,33258,33260,33261,33262,33264,33265,33266,33268,33269,33270,33272,33273,33274,33276,33277,33278,33280,33281,33282,33284,33285,33286,33288,33289,33290,33292,33293,33294,33296,33297,33298,33300,33301,33302,33304,33305,33306,33308,33309,33310,33312,33313,33314,33316,33317,33318,33320,33321,33322,33324,33325,33326,33328,33329,33330,33332,33333,33334,33336,33337,33338,33340,33341,33342,33344,33345,33346,33348,33349,33350,33352,33353,33354,33356,33357,33358,33360,33361,33362,33364,33365,33366,33368,33369,33370,33372,33373,33374,33376,33377,33378,33380,33381,33382,33384,33385,33386,33388,33389,33390,33392,33393,33394,33396,33397,33398,33400,33401,33402,33404,33405,33406,33408,33409,33410,33412,33413,33414,33416,33417,33418,33420,33421,33422,33424,33425,33426,33428,33429,33430,33432,33433,33434,33436,33437,33438,33440,33441,33442,33444,33445,33446,33448,33449,33450,33452,33453,33454,33456,33457,33458,33460,33461,33462,33464,33465,33466,33468,33469,33470,33472,33473,33474,33476,33477,33478,33480,33481,33482,33484,33485,33486,33488,33489,33490,33492,33493,33494,33496,33497,33498,33500,33501,33502,33504,33505,33506,33508,33509,33510,33512,33513,33514,33516,33517,33518,33520,33521,33522,33524,33525,33526,33528,33529,33530,33532,33533,33534,33536,33537,33538,33540,33541,33542,33544,33545,33546,33548,33549,33550,33552,33553,33554,33556,33557,33558,33560,33561,33562,33564,33565,33566,33568,33569,33570,33572,33573,33574,33576,33577,33578,33580,33581,33582,33584,33585,33586,33588,33589,33590,33592,33593,33594,33596,33597,33598,33600,33601,33602,33604,33605,33606,33608,33609,33610,33612,33613,33614,33616,33617,33618,33620,33621,33622,33624,33625,33626,33628,33629,33630,33632,33633,33634,33636,33637,33638,33640,33641,33642,33644,33645,33646,33648,33649,33650,33652,33653,33654,33656,33657,33658,33660,33661,33662,33664,33665,33666,33668,33669,33670,33672,33673,33674,33676,33677,33678,33680,33681,33682,33684,33685,33686,33688,33689,33690,33692,33693,33694,33696,33697,33698,33700,33701,33702,33704,33705,33706,33708,33709,33710,33712,33713,33714,33716,33717,33718,33720,33721,33722,33724,33725,33726,33728,33729,33730,33732,33733,33734,33736,33737,33738,33740,33741,33742,33744,33745,33746,33748,33749,33750,33752,33753,33754,33756,33757,33758,33760,33761,33762,33764,33765,33766,33768,33769,33770,33772,33773,33774,33776,33777,33778,33780,33781,33782,33784,33785,33786,33788,33789,33790,0,0,0,0,0,0,0,0,0,0,0,0,0,0,0,0,0,0,0,0,0,0,0,0,0,0,0,0,0,0,0,0,0,0,0,0,0,0,0,0,0,0,0,0,0,0,0,0,0,0,0,0,0,0,0,0,0,0,0,0,0,0,0,0,0,0,0,0,0,0,0,0,0,0,0,0,0,0,0,0,0,0,0,0,0,0,0,0,0,0,0,0,0,0,0,0,0,0,0,0,0,0,0,0,0,0,0,0,0,0,0,0,0,0,0,0,0,0,0,0,0,0,0,0,0,0,0,0,0,0,0,0,0,0,0,0,0,0,0,0,0,0,0,0,0,0,0,0,0,0,0,0,0,0,0,0,0,0,0,0,0,0,0,0,0,0,0,0,
0,1,2,3,4,5,6,7,8,9,10,11,12,13,14,15,16,17,18,19,20,21,22,23,24,25,26,27,28,29,30,31,32,33,34,35,36,37,38,39,40,41,42,43,44,45,46,47,48,49,50,51,52,53,54,55,56,57,58,59,60,61,62,63,64,65,66,67,68,69,70,71,72,73,74,75,76,77,78,79,80,81,82,83,84,85,86,87,88,89,90,91,92,93,94,95,96,97,98,99,100,101,102,103,104,105,106,107,108,109,110,111,112,113,114,115,116,117,118,119,120,121,122,123,124,125,126,127,576,577,578,579,580,581,582,583,584,585,586,587,588,589,590,591,16976,138,16977,141,16978,144,16979,147,16980,150,16981,153,640,641,642,643,644,645,646,647,648,649,650,651,652,653,654,655,656,657,658,659,660,661,662,663,664,665,666,667,668,669,670,671,672,673,674,675,676,677,678,679,680,681,682,683,684,685,686,687,688,689,690,691,692,693,694,695,696,697,698,699,700,701,702,703,704,705,706,707,708,709,710,711,712,713,714,715,716,717,718,719,720,721,722,723,724,725,726,727,728,729,730,731,732,733,734,735,736,737,738,739,740,741,742,743,744,745,746,747,748,749,750,751,752,753,754,755,756,757,758,759,760,761,762,763,764,765,766,767,768,1216,1217,866,771,1218,1219,869,774,1220,1221,872,777,1222,1223,875,780,1224,1225,878,783,1226,1227,769,786,1228,1229,772,789,1230,1231,775,792,806,17616,778,795,809,17617,781,798,812,17618,784,801,815,17619,787,804,818,17620,790,807,821,17621,793,1280,1281,1282,1283,1284,1285,1286,1287,1288,1289,1290,1291,1292,1293,1294,1295,1296,1297,1298,1299,1300,1301,1302,1303,1304,1305,1306,1307,1308,1309,1310,1311,1312,1313,1314,1315,1316,1317,1318,1319,1320,1321,1322,1323,1324,1325,1326,1327,1328,1329,1330,1331,1332,1333,1334,1335,1336,1337,1338,1339,1340,1341,1342,1343,1344,1345,1346,1347,1348,1349,1350,1351,1352,1353,1354,1355,1356,1357,1358,1359,1360,1361,1362,1363,1364,1365,1366,1367,1368,1369,1370,1371,1372,1373,1374,1375,1376,1377,1378,1379,1380,1381,1382,1383,1384,1385,1386,1387,1388,1389,1390,1391,1392,1393,1394,1395,1396,1397,1398,1399,1400,1401,1402,1403,1404,1405,1406,1407,1408,1422,1436,1856,1857,1506,1411,1425,1439,1858,1859,1509,1414,1428,1442,1860,1861,1512,1417,1431,1445,1862,1863,1515,1420,1434,1448,1864,1865,1518,1423,1437,1451,1866,1867,1409,1426,1440,1454,1868,1869,1412,1429,1443,1457,1870,1871,1415,1432,1446,1460,1474,18256,1418,1435,1449,1463,1477,18257,1421,1438,1452,1466,1480,18258,1424,1441,1455,1469,1483,18259,1427,1444,1458,1472,1486,18260,1430,1447,1461,1475,1489,18261,1433,1920,1921,1922,1923,1924,1925,1926,1927,1928,1929,1930,1931,1932,1933,1934,1935,1936,1937,1938,1939,1940,1941,1942,1943,1944,1945,1946,1947,1948,1949,1950,1951,1952,1953,1954,1955,1956,1957,1958,1959,1960,1961,1962,1963,1964,1965,1966,1967,1968,1969,1970,1971,1972,1973,1974,1975,1976,1977,1978,1979,1980,1981,1982,1983,1984,1985,1986,1987,1988,1989,1990,1991,1992,1993,1994,1995,1996,1997,1998,1999,2000,2001,2002,2003,2004,2005,2006,2007,2008,2009,2010,2011,2012,2013,2014,2015,2016,2017,2018,2019,2020,2021,2022,2023,2024,2025,2026,2027,2028,2029,2030,2031,2032,2033,2034,2035,2036,2037,2038,2039,2040,2041,2042,2043,2044,2045,2046,2047,2048,2062,2076,2090,2104,2496,2497,2146,2051,2065,2079,2093,2107,2498,2499,2149,2054,2068,2082,2096,2110,2500,2501,2152,2057,2071,2085,2099,2113,2502,2503,2155,2060,2074,2088,2102,2116,2504,2505,2158,2063,2077,2091,2105,2119,2506,2507,2049,2066,2080,2094,2108,2122,2508,2509,2052,2069,2083,2097,2111,2125,2510,2511,2055,2072,2086,2100,2114,2128,2142,18896,2058,2075,2089,2103,2117,2131,2145,18897,2061,2078,2092,2106,2120,2134,2148,18898,2064,2081,2095,2109,2123,2137,2151,18899,2067,2084,2098,2112,2126,2140,2154,18900,2070,2087,2101,2115,2129,2143,2157,18901,2073,128,142,156,170,184,198,212,226,131,145,159,173,187,201,215,229,134,148,162,176,190,204,218,232,137,151,165,179,193,207,221,235,140,154,168,182,196,210,224,238,143,157,171,185,199,213,227,129,146,160,174,188,202,216,230,132,149,163,177,191,205,219,233,135,152,166,180,194,208,222,236,250,155,169,183,197,211,225,239,253,158,172,186,200,214,228,130,256,161,175,189,203,217,231,133,259,164,178,192,206,220,234,136,262,167,181,195,209,223,237,139,265,880,782,796,810,824,838,852,978,883,785,799,813,827,841,855,981,886,788,802,816,830,844,858,984,889,791,805,819,833,847,861,987,892,794,808,822,836,850,864,990,895,797,811,825,839,853,867,881,898,800,814,828,842,856,870,884,901,803,817,831,845,859,873,887,904,918,820,834,848,862,876,890,907,921,823,837,851,865,879,893,910,924,826,840,854,868,770,896,913,927,829,843,857,871,773,899,916,930,832,846,860,874,776,902,919,933,835,849,863,877,779,905,1520,1534,1548,1450,1464,1478,1492,1618,1523,1537,1551,1453,1467,1481,1495,1621,1526,1540,1554,1456,1470,1484,1498,1624,1529,1543,1557,1459,1473,1487,1501,1627,1532,1546,1560,1462,1476,1490,1504,1630,1535,1549,1563,1465,1479,1493,1507,1521,1538,1552,1566,1468,1482,1496,1510,1524,1541,1555,1569,1471,1485,1499,1513,1527,1544,1558,1572,1586,1488,1502,1516,1530,1547,1561,1575,1589,1491,1505,1519,1533,1550,1564,1578,1592,1494,1508,1410,1536,1553,1567,1581,1595,1497,1511,1413,1539,1556,1570,1584,1598,1500,1514,1416,1542,1559,1573,1587,1601,1503,1517,1419,1545,2160,2174,2188,2202,2216,2118,2132,2258,2163,2177,2191,2205,2219,2121,2135,2261,2166,2180,2194,2208,2222,2124,2138,2264,2169,2183,2197,2211,2225,2127,2141,2267,2172,2186,2200,2214,2228,2130,2144,2270,2175,2189,2203,2217,2231,2133,2147,2161,2178,2192,2206,2220,2234,2136,2150,2164,2181,2195,2209,2223,2237,2139,2153,2167,2184,2198,2212,2226,2240,2254,2156,2170,2187,2201,2215,2229,2243,2257,2159,2173,2190,2204,2218,2232,2246,2260,2050,2176,2193,2207,2221,2235,2249,2263,2053,2179,2196,2210,2224,2238,2252,2266,2056,2182,2199,2213,2227,2241,2255,2269,2059,2185,240,254,268,282,296,310,324,338,243,257,271,285,299,313,327,341,246,260,274,288,302,316,330,344,249,263,277,291,305,319,333,347,252,266,280,294,308,322,336,350,255,269,283,297,311,325,339,241,258,272,286,300,314,328,342,244,261,275,289,303,317,331,345,247,264,278,292,306,320,334,348,362,267,281,295,309,323,337,351,365,270,284,298,312,326,340,242,368,273,287,301,315,329,343,245,371,276,290,304,318,332,346,248,374,279,293,307,321,335,349,251,377,992,894,908,922,936,950,964,1090,995,897,911,925,939,953,967,1093,998,900,914,928,942,956,970,1096,1001,903,917,931,945,959,973,1099,1004,906,920,934,948,962,976,1102,1007,909,923,937,951,965,979,993,1010,912,926,940,954,968,982,996,1013,915,929,943,957,971,985,999,1016,1030,932,946,960,974,988,1002,1019,1033,935,949,963,977,991,1005,1022,1036,938,952,966,980,882,1008,1025,1039,941,955,969,983,885,1011,1028,1042,944,958,972,986,888,1014,1031,1045,947,961,975,989,891,1017,1632,1646,1660,1562,1576,1590,1604,1730,1635,1649,1663,1565,1579,1593,1607,1733,1638,1652,1666,1568,1582,1596,1610,1736,1641,1655,1669,1571,1585,1599,1613,1739,1644,1658,1672,1574,1588,1602,1616,1742,1647,1661,1675,1577,1591,1605,1619,1633,1650,1664,1678,1580,1594,1608,1622,1636,1653,1667,1681,1583,1597,1611,1625,1639,1656,1670,1684,1698,1600,1614,1628,1642,1659,1673,1687,1701,1603,1617,1631,1645,1662,1676,1690,1704,1606,1620,1522,1648,1665,1679,1693,1707,1609,1623,1525,1651,1668,1682,1696,1710,1612,1626,1528,1654,1671,1685,1699,1713,1615,1629,1531,1657,2272,2286,2300,2314,2328,2230,2244,2370,2275,2289,2303,2317,2331,2233,2247,2373,2278,2292,2306,2320,2334,2236,2250,2376,2281,2295,2309,2323,2337,2239,2253,2379,2284,2298,2312,2326,2340,2242,2256,2382,2287,2301,2315,2329,2343,2245,2259,2273,2290,2304,2318,2332,2346,2248,2262,2276,2293,2307,2321,2335,2349,2251,2265,2279,2296,2310,2324,2338,2352,2366,2268,2282,2299,2313,2327,2341,2355,2369,2271,2285,2302,2316,2330,2344,2358,2372,2162,2288,2305,2319,2333,2347,2361,2375,2165,2291,2308,2322,2336,2350,2364,2378,2168,2294,2311,2325,2339,2353,2367,2381,2171,2297,352,366,380,394,408,422,436,450,355,369,383,397,411,425,439,453,358,372,386,400,414,428,442,456,361,375,389,403,417,431,445,459,364,378,392,406,420,434,448,462,367,381,395,409,423,437,451,353,370,384,398,412,426,440,454,356,373,387,401,415,429,443,457,359,376,390,404,418,432,446,460,474,379,393,407,421,435,449,463,477,382,396,410,424,438,452,354,480,385,399,413,427,441,455,357,483,388,402,416,430,444,458,360,486,391,405,419,433,447,461,363,489,1104,1006,1020,1034,1048,1062,1076,1202,1107,1009,1023,1037,1051,1065,1079,1205,1110,1012,1026,1040,1054,1068,1082,1208,1113,1015,1029,1043,1057,1071,1085,1211,1116,1018,1032,1046,1060,1074,1088,1214,1119,1021,1035,1049,1063,1077,1091,1105,1122,1024,1038,1052,1066,1080,1094,1108,1125,1027,1041,1055,1069,1083,1097,1111,1128,1142,1044,1058,1072,1086,1100,1114,1131,1145,1047,1061,1075,1089,1103,1117,1134,1148,1050,1064,1078,1092,994,1120,1137,1151,1053,1067,1081,1095,997,1123,1140,1154,1056,1070,1084,1098,1000,1126,1143,1157,1059,1073,1087,1101,1003,1129,1744,1758,1772,1674,1688,1702,1716,1842,1747,1761,1775,1677,1691,1705,1719,1845,1750,1764,1778,1680,1694,1708,1722,1848,1753,1767,1781,1683,1697,1711,1725,1851,1756,1770,1784,1686,1700,1714,1728,1854,1759,1773,1787,1689,1703,1717,1731,1745,1762,1776,1790,1692,1706,1720,1734,1748,1765,1779,1793,1695,1709,1723,1737,1751,1768,1782,1796,1810,1712,1726,1740,1754,1771,1785,1799,1813,1715,1729,1743,1757,1774,1788,1802,1816,1718,1732,1634,1760,1777,1791,1805,1819,1721,1735,1637,1763,1780,1794,1808,1822,1724,1738,1640,1766,1783,1797,1811,1825,1727,1741,1643,1769,2384,2398,2412,2426,2440,2342,2356,2482,2387,2401,2415,2429,2443,2345,2359,2485,2390,2404,2418,2432,2446,2348,2362,2488,2393,2407,2421,2435,2449,2351,2365,2491,2396,2410,2424,2438,2452,2354,2368,2494,2399,2413,2427,2441,2455,2357,2371,2385,2402,2416,2430,2444,2458,2360,2374,2388,2405,2419,2433,2447,2461,2363,2377,2391,2408,2422,2436,2450,2464,2478,2380,2394,2411,2425,2439,2453,2467,2481,2383,2397,2414,2428,2442,2456,2470,2484,2274,2400,2417,2431,2445,2459,2473,2487,2277,2403,2420,2434,2448,2462,2476,2490,2280,2406,2423,2437,2451,2465,2479,2493,2283,2409,464,478,492,506,520,534,548,562,467,481,495,509,523,537,551,565,470,484,498,512,526,540,554,568,473,487,501,515,529,543,557,571,476,490,504,518,532,546,560,574,479,493,507,521,535,549,563,465,482,496,510,524,538,552,566,468,485,499,513,527,541,555,569,471,488,502,516,530,544,558,572,491,505,519,533,547,561,575,494,508,522,536,550,564,466,497,511,525,539,553,567,469,500,514,528,542,556,570,472,503,517,531,545,559,573,475,1118,1132,1146,1160,1174,1188,1121,1135,1149,1163,1177,1191,1124,1138,1152,1166,1180,1194,1127,1141,1155,1169,1183,1197,1130,1144,1158,1172,1186,1200,1133,1147,1161,1175,1189,1203,1136,1150,1164,1178,1192,1206,1139,1153,1167,1181,1195,1209,1156,1170,1184,1198,1212,1159,1173,1187,1201,1215,1162,1176,1190,1204,1106,1165,1179,1193,1207,1109,1168,1182,1196,1210,1112,1171,1185,1199,1213,1115,1786,1800,1814,1828,1789,1803,1817,1831,1792,1806,1820,1834,1795,1809,1823,1837,1798,1812,1826,1840,1801,1815,1829,1843,1804,1818,1832,1846,1807,1821,1835,1849,1824,1838,1852,1827,1841,1855,1830,1844,1746,1833,1847,1749,1836,1850,1752,1839,1853,1755,2454,2468,2457,2471,2460,2474,2463,2477,2466,2480,2469,2483,2472,2486,2475,2489,2492,2495,2386,2389,2392,2395,32768,32769,32770,32772,32773,32774,32776,32777,32778,32780,32781,32782,32784,32785,32786,32788,32789,32790,32792,32793,32794,32796,32797,32798,32800,32801,32802,32804,32805,32806,32808,32809,32810,32812,32813,32814,32816,32817,32818,32820,32821,32822,32824,32825,32826,32828,32829,32830,32832,32833,32834,32836,32837,32838,32840,32841,32842,32844,32845,32846,32848,32849,32850,32852,32853,32854,32856,32857,32858,32860,32861,32862,32864,32865,32866,32868,32869,32870,32872,32873,32874,32876,32877,32878,32880,32881,32882,32884,32885,32886,32888,32889,32890,32892,32893,32894,32896,32897,32898,32900,32901,32902,32904,32905,32906,32908,32909,32910,32912,32913,32914,32916,32917,32918,32920,32921,32922,32924,32925,32926,32928,32929,32930,32932,32933,32934,32936,32937,32938,32940,32941,32942,32944,32945,32946,32948,32949,32950,32952,32953,32954,32956,32957,32958,32960,32961,32962,32964,32965,32966,32968,32969,32970,32972,32973,32974,32976,32977,32978,32980,32981,32982,32984,32985,32986,32988,32989,32990,32992,32993,32994,32996,32997,32998,33000,33001,33002,33004,33005,33006,33008,33009,33010,33012,33013,33014,33016,33017,33018,33020,33021,33022,33024,33025,33026,33028,33029,33030,33032,33033,33034,33036,33037,33038,33040,33041,33042,33044,33045,33046,33048,33049,33050,33052,33053,33054,33056,33057,33058,33060,33061,33062,33064,33065,33066,33068,33069,33070,33072,33073,33074,33076,33077,33078,33080,33081,33082,33084,33085,33086,33088,33089,33090,33092,33093,33094,33096,33097,33098,33100,33101,33102,33104,33105,33106,33108,33109,33110,33112,33113,33114,33116,33117,33118,33120,33121,33122,33124,33125,33126,33128,33129,33130,33132,33133,33134,33136,33137,33138,33140,33141,33142,33144,33145,33146,33148,33149,33150,33152,33153,33154,33156,33157,33158,33160,33161,33162,33164,33165,33166,33168,33169,33170,33172,33173,33174,33176,33177,33178,33180,33181,33182,33184,33185,33186,33188,33189,33190,33192,33193,33194,33196,33197,33198,33200,33201,33202,33204,33205,33206,33208,33209,33210,33212,33213,33214,33216,33217,33218,33220,33221,33222,33224,33225,33226,33228,33229,33230,33232,33233,33234,33236,33237,33238,33240,33241,33242,33244,33245,33246,33248,33249,33250,33252,33253,33254,33256,33257,33258,33260,33261,33262,33264,33265,33266,33268,33269,33270,33272,33273,33274,33276,33277,33278,33280,33281,33282,33284,33285,33286,33288,33289,33290,33292,33293,33294,33296,33297,33298,33300,33301,33302,33304,33305,33306,33308,33309,33310,33312,33313,33314,33316,33317,33318,33320,33321,33322,33324,33325,33326,33328,33329,33330,33332,33333,33334,33336,33337,33338,33340,33341,33342,33344,33345,33346,33348,33349,33350,33352,33353,33354,33356,33357,33358,33360,33361,33362,33364,33365,33366,33368,33369,33370,33372,33373,33374,33376,33377,33378,33380,33381,33382,33384,33385,33386,33388,33389,33390,33392,33393,33394,33396,33397,33398,33400,33401,33402,33404,33405,33406,33408,33409,33410,33412,33413,33414,33416,33417,33418,33420,33421,33422,33424,33425,33426,33428,33429,33430,33432,33433,33434,33436,33437,33438,33440,33441,33442,33444,33445,33446,33448,33449,33450,33452,33453,33454,33456,33457,33458,33460,33461,33462,33464,33465,33466,33468,33469,33470,33472,33473,33474,33476,33477,33478,33480,33481,33482,33484,33485,33486,33488,33489,33490,33492,33493,33494,33496,33497,33498,33500,33501,33502,33504,33505,33506,33508,33509,33510,33512,33513,33514,33516,33517,33518,33520,33521,33522,33524,33525,33526,33528,33529,33530,33532,33533,33534,33536,33537,33538,33540,33541,33542,33544,33545,33546,33548,33549,33550,33552,33553,33554,33556,33557,33558,33560,33561,33562,33564,33565,33566,33568,33569,33570,33572,33573,33574,33576,33577,33578,33580,33581,33582,33584,33585,33586,33588,33589,33590,33592,33593,33594,33596,33597,33598,33600,33601,33602,33604,33605,33606,33608,33609,33610,33612,33613,33614,33616,33617,33618,33620,33621,33622,33624,33625,33626,33628,33629,33630,33632,33633,33634,33636,33637,33638,33640,33641,33642,33644,33645,33646,33648,33649,33650,33652,33653,33654,33656,33657,33658,33660,33661,33662,33664,33665,33666,33668,33669,33670,33672,33673,33674,33676,33677,33678,33680,33681,33682,33684,33685,33686,33688,33689,33690,33692,33693,33694,33696,33697,33698,33700,33701,33702,33704,33705,33706,33708,33709,33710,33712,33713,33714,33716,33717,33718,33720,33721,33722,33724,33725,33726,33728,33729,33730,33732,33733,33734,33736,33737,33738,33740,33741,33742,33744,33745,33746,33748,33749,33750,33752,33753,33754,33756,33757,33758,33760,33761,33762,33764,33765,33766,33768,33769,33770,33772,33773,33774,33776,33777,33778,33780,33781,33782,33784,33785,33786,33788,33789,33790,0,0,0,0,0,0,0,0,0,0,0,0,0,0,0,0,0,0,0,0,0,0,0,0,0,0,0,0,0,0,0,0,0,0,0,0,0,0,0,0,0,0,0,0,0,0,0,0,0,0,0,0,0,0,0,0,0,0,0,0,0,0,0,0,0,0,0,0,0,0,0,0,0,0,0,0,0,0,0,0,0,0,0,0,0,0,0,0,0,0,0,0,0,0,0,0,0,0,0,0,0,0,0,0,0,0,0,0,0,0,0,0,0,0,0,0,0,0,0,0,0,0,0,0,0,0,0,0,0,0,0,0,0,0,0,0,0,0,0,0,0,0,0,0,0,0,0,0,0,0,0,0,0,0,0,0,0,0,0,0,0,0,0,0,0,0,0,0,
0,1,2,3,4,5,6,7,8,9,10,11,12,13,14,15,16,17,18,19,20,21,22,23,24,25,26,27,28,29,30,31,32,33,34,35,36,37,38,39,40,41,42,43,44,45,46,47,48,49,50,51,52,53,54,55,56,57,58,59,60,61,62,63,64,65,66,67,68,69,70,71,72,73,74,75,76,77,78,79,80,81,82,83,84,85,86,87,88,89,90,91,92,93,94,95,96,97,98,99,100,101,102,103,104,105,106,107,108,109,110,111,112,113,114,115,116,117,118,119,120,121,122,123,124,125,126,127,576,577,578,579,580,581,582,583,584,585,586,587,588,589,590,591,16976,138,16977,141,16978,144,16979,147,16980,150,16981,153,640,641,642,643,644,645,646,647,648,649,650,651,652,653,654,655,656,657,658,659,660,661,662,663,664,665,666,667,668,669,670,671,672,673,674,675,676,677,678,679,680,681,682,683,684,685,686,687,688,689,690,691,692,693,694,695,696,697,698,699,700,701,702,703,704,705,706,707,708,709,710,711,712,713,714,715,716,717,718,719,720,721,722,723,724,725,726,727,728,729,730,731,732,733,734,735,736,737,738,739,740,741,742,743,744,745,746,747,748,749,750,751,752,753,754,755,756,757,758,759,760,761,762,763,764,765,766,767,768,1216,1217,866,771,1218,1219,869,774,1220,1221,872,777,1222,1223,875,780,1224,1225,878,783,1226,1227,769,786,1228,1229,772,789,1230,1231,775,792,806,17616,778,795,809,17617,781,798,812,17618,784,801,815,17619,787,804,818,17620,790,807,821,17621,793,1280,1281,1282,1283,1284,1285,1286,1287,1288,1289,1290,1291,1292,1293,1294,1295,1296,1297,1298,1299,1300,1301,1302,1303,1304,1305,1306,1307,1308,1309,1310,1311,1312,1313,1314,1315,1316,1317,1318,1319,1320,1321,1322,1323,1324,1325,1326,1327,1328,1329,1330,1331,1332,1333,1334,1335,1336,1337,1338,1339,1340,1341,1342,1343,1344,1345,1346,1347,1348,1349,1350,1351,1352,1353,1354,1355,1356,1357,1358,1359,1360,1361,1362,1363,1364,1365,1366,1367,1368,1369,1370,1371,1372,1373,1374,1375,1376,1377,1378,1379,1380,1381,1382,1383,1384,1385,1386,1387,1388,1389,1390,1391,1392,1393,1394,1395,1396,1397,1398,1399,1400,1401,1402,1403,1404,1405,1406,1407,1408,1422,1436,1856,1857,1506,1411,1425,1439,1858,1859,1509,1414,1428,1442,1860,1861,1512,1417,1431,1445,1862,1863,1515,1420,1434,1448,1864,1865,1518,1423,1437,1451,1866,1867,1409,1426,1440,1454,1868,1869,1412,1429,1443,1457,1870,1871,1415,1432,1446,1460,1474,18256,1418,1435,1449,1463,1477,18257,1421,1438,1452,1466,1480,18258,1424,1441,1455,1469,1483,18259,1427,1444,1458,1472,1486,18260,1430,1447,1461,1475,1489,18261,1433,1920,1921,1922,1923,1924,1925,1926,1927,1928,1929,1930,1931,1932,1933,1934,1935,1936,1937,1938,1939,1940,1941,1942,1943,1944,1945,1946,1947,1948,1949,1950,1951,1952,1953,1954,1955,1956,1957,1958,1959,1960,1961,1962,1963,1964,1965,1966,1967,1968,1969,1970,1971,1972,1973,1974,1975,1976,1977,1978,1979,1980,1981,1982,1983,1984,1985,1986,1987,1988,1989,1990,1991,1992,1993,1994,1995,1996,1997,1998,1999,2000,2001,2002,2003,2004,2005,2006,2007,2008,2009,2010,2011,2012,2013,2014,2015,2016,2017,2018,2019,2020,2021,2022,2023,2024,2025,2026,2027,2028,2029,2030,2031,2032,2033,2034,2035,2036,2037,2038,2039,2040,2041,2042,2043,2044,2045,2046,2047,2048,2062,2076,2090,2104,2496,2497,2146,2051,2065,2079,2093,2107,2498,2499,2149,2054,2068,2082,2096,2110,2500,2501,2152,2057,2071,2085,2099,2113,2502,2503,2155,2060,2074,2088,2102,2116,2504,2505,2158,2063,2077,2091,2105,2119,2506,2507,2049,2066,2080,2094,2108,2122,2508,2509,2052,2069,2083,2097,2111,2125,2510,2511,2055,2072,2086,2100,2114,2128,2142,18896,2058,2075,2089,2103,2117,2131,2145,18897,2061,2078,2092,2106,2120,2134,2148,18898,2064,2081,2095,2109,2123,2137,2151,18899,2067,2084,2098,2112,2126,2140,2154,18900,2070,2087,2101,2115,2129,2143,2157,18901,2073,128,142,156,170,184,198,212,226,131,145,159,173,187,201,215,229,134,148,162,176,190,204,218,232,137,151,165,179,193,207,221,235,140,154,168,182,196,210,224,238,143,157,171,185,199,213,227,129,146,160,174,188,202,216,230,132,149,163,177,191,205,219,233,135,152,166,180,194,208,222,236,250,155,169,183,197,211,225,239,253,158,172,186,200,214,228,130,256,161,175,189,203,217,231,133,259,164,178,192,206,220,234,136,262,167,181,195,209,223,237,139,265,880,782,796,810,824,838,852,978,883,785,799,813,827,841,855,981,886,788,802,816,830,844,858,984,889,791,805,819,833,847,861,987,892,794,808,822,836,850,864,990,895,797,811,825,839,853,867,881,898,800,814,828,842,856,870,884,901,803,817,831,845,859,873,887,904,918,820,834,848,862,876,890,907,921,823,837,851,865,879,893,910,924,826,840,854,868,770,896,913,927,829,843,857,871,773,899,916,930,832,846,860,874,776,902,919,933,835,849,863,877,779,905,1520,1534,1548,1450,1464,1478,1492,1618,1523,1537,1551,1453,1467,1481,1495,1621,1526,1540,1554,1456,1470,1484,1498,1624,1529,1543,1557,1459,1473,1487,1501,1627,1532,1546,1560,1462,1476,1490,1504,1630,1535,1549,1563,1465,1479,1493,1507,1521,1538,1552,1566,1468,1482,1496,1510,1524,1541,1555,1569,1471,1485,1499,1513,1527,1544,1558,1572,1586,1488,1502,1516,1530,1547,1561,1575,1589,1491,1505,1519,1533,1550,1564,1578,1592,1494,1508,1410,1536,1553,1567,1581,1595,1497,1511,1413,1539,1556,1570,1584,1598,1500,1514,1416,1542,1559,1573,1587,1601,1503,1517,1419,1545,2160,2174,2188,2202,2216,2118,2132,2258,2163,2177,2191,2205,2219,2121,2135,2261,2166,2180,2194,2208,2222,2124,2138,2264,2169,2183,2197,2211,2225,2127,2141,2267,2172,2186,2200,2214,2228,2130,2144,2270,2175,2189,2203,2217,2231,2133,2147,2161,2178,2192,2206,2220,2234,2136,2150,2164,2181,2195,2209,2223,2237,2139,2153,2167,2184,2198,2212,2226,2240,2254,2156,2170,2187,2201,2215,2229,2243,2257,2159,2173,2190,2204,2218,2232,2246,2260,2050,2176,2193,2207,2221,2235,2249,2263,2053,2179,2196,2210,2224,2238,2252,2266,2056,2182,2199,2213,2227,2241,2255,2269,2059,2185,240,254,268,282,296,310,324,338,243,257,271,285,299,313,327,341,246,260,274,288,302,316,330,344,249,263,277,291,305,319,333,347,252,266,280,294,308,322,336,350,255,269,283,297,311,325,339,241,258,272,286,300,314,328,342,244,261,275,289,303,317,331,345,247,264,278,292,306,320,334,348,362,267,281,295,309,323,337,351,365,270,284,298,312,326,340,242,368,273,287,301,315,329,343,245,371,276,290,304,318,332,346,248,374,279,293,307,321,335,349,251,377,992,894,908,922,936,950,964,1090,995,897,911,925,939,953,967,1093,998,900,914,928,942,956,970,1096,1001,903,917,931,945,959,973,1099,1004,906,920,934,948,962,976,1102,1007,909,923,937,951,965,979,993,1010,912,926,940,954,968,982,996,1013,915,929,943,957,971,985,999,1016,1030,932,946,960,974,988,1002,1019,1033,935,949,963,977,991,1005,1022,1036,938,952,966,980,882,1008,1025,1039,941,955,969,983,885,1011,1028,1042,944,958,972,986,888,1014,1031,1045,947,961,975,989,891,1017,1632,1646,1660,1562,1576,1590,1604,1730,1635,1649,1663,1565,1579,1593,1607,1733,1638,1652,1666,1568,1582,1596,1610,1736,1641,1655,1669,1571,1585,1599,1613,1739,1644,1658,1672,1574,1588,1602,1616,1742,1647,1661,1675,1577,1591,1605,1619,1633,1650,1664,1678,1580,1594,1608,1622,1636,1653,1667,1681,1583,1597,1611,1625,1639,1656,1670,1684,1698,1600,1614,1628,1642,1659,1673,1687,1701,1603,1617,1631,1645,1662,1676,1690,1704,1606,1620,1522,1648,1665,1679,1693,1707,1609,1623,1525,1651,1668,1682,1696,1710,1612,1626,1528,1654,1671,1685,1699,1713,1615,1629,1531,1657,2272,2286,2300,2314,2328,2230,2244,2370,2275,2289,2303,2317,2331,2233,2247,2373,2278,2292,2306,2320,2334,2236,2250,2376,2281,2295,2309,2323,2337,2239,2253,2379,2284,2298,2312,2326,2340,2242,2256,2382,2287,2301,2315,2329,2343,2245,2259,2273,2290,2304,2318,2332,2346,2248,2262,2276,2293,2307,2321,2335,2349,2251,2265,2279,2296,2310,2324,2338,2352,2366,2268,2282,2299,2313,2327,2341,2355,2369,2271,2285,2302,2316,2330,2344,2358,2372,2162,2288,2305,2319,2333,2347,2361,2375,2165,2291,2308,2322,2336,2350,2364,2378,2168,2294,2311,2325,2339,2353,2367,2381,2171,2297,352,366,380,394,408,422,436,450,355,369,383,397,411,425,439,453,358,372,386,400,414,428,442,456,361,375,389,403,417,431,445,459,364,378,392,406,420,434,448,462,367,381,395,409,423,437,451,353,370,384,398,412,426,440,454,356,373,387,401,415,429,443,457,359,376,390,404,418,432,446,460,474,379,393,407,421,435,449,463,477,382,396,410,424,438,452,354,480,385,399,413,427,441,455,357,483,388,402,416,430,444,458,360,486,391,405,419,433,447,461,363,489,1104,1006,1020,1034,1048,1062,1076,1202,1107,1009,1023,1037,1051,1065,1079,1205,1110,1012,1026,1040,1054,1068,1082,1208,1113,1015,1029,1043,1057,1071,1085,1211,1116,1018,1032,1046,1060,1074,1088,1214,1119,1021,1035,1049,1063,1077,1091,1105,1122,1024,1038,1052,1066,1080,1094,1108,1125,1027,1041,1055,1069,1083,1097,1111,1128,1142,1044,1058,1072,1086,1100,1114,1131,1145,1047,1061,1075,1089,1103,1117,1134,1148,1050,1064,1078,1092,994,1120,1137,1151,1053,1067,1081,1095,997,1123,1140,1154,1056,1070,1084,1098,1000,1126,1143,1157,1059,1073,1087,1101,1003,1129,1744,1758,1772,1674,1688,1702,1716,1842,1747,1761,1775,1677,1691,1705,1719,1845,1750,1764,1778,1680,1694,1708,1722,1848,1753,1767,1781,1683,1697,1711,1725,1851,1756,1770,1784,1686,1700,1714,1728,1854,1759,1773,1787,1689,1703,1717,1731,1745,1762,1776,1790,1692,1706,1720,1734,1748,1765,1779,1793,1695,1709,1723,1737,1751,1768,1782,1796,1810,1712,1726,1740,1754,1771,1785,1799,1813,1715,1729,1743,1757,1774,1788,1802,1816,1718,1732,1634,1760,1777,1791,1805,1819,1721,1735,1637,1763,1780,1794,1808,1822,1724,1738,1640,1766,1783,1797,1811,1825,1727,1741,1643,1769,2384,2398,2412,2426,2440,2342,2356,2482,2387,2401,2415,2429,2443,2345,2359,2485,2390,2404,2418,2432,2446,2348,2362,2488,2393,2407,2421,2435,2449,2351,2365,2491,2396,2410,2424,2438,2452,2354,2368,2494,2399,2413,2427,2441,2455,2357,2371,2385,2402,2416,2430,2444,2458,2360,2374,2388,2405,2419,2433,2447,2461,2363,2377,2391,2408,2422,2436,2450,2464,2478,2380,2394,2411,2425,2439,2453,2467,2481,2383,2397,2414,2428,2442,2456,2470,2484,2274,2400,2417,2431,2445,2459,2473,2487,2277,2403,2420,2434,2448,2462,2476,2490,2280,2406,2423,2437,2451,2465,2479,2493,2283,2409,464,478,492,506,520,534,548,562,467,481,495,509,523,537,551,565,470,484,498,512,526,540,554,568,473,487,501,515,529,543,557,571,476,490,504,518,532,546,560,574,479,493,507,521,535,549,563,465,482,496,510,524,538,552,566,468,485,499,513,527,541,555,569,471,488,502,516,530,544,558,572,491,505,519,533,547,561,575,494,508,522,536,550,564,466,497,511,525,539,553,567,469,500,514,528,542,556,570,472,503,517,531,545,559,573,475,1118,1132,1146,1160,1174,1188,1121,1135,1149,1163,1177,1191,1124,1138,1152,1166,1180,1194,1127,1141,1155,1169,1183,1197,1130,1144,1158,1172,1186,1200,1133,1147,1161,1175,1189,1203,1136,1150,1164,1178,1192,1206,1139,1153,1167,1181,1195,1209,1156,1170,1184,1198,1212,1159,1173,1187,1201,1215,1162,1176,1190,1204,1106,1165,1179,1193,1207,1109,1168,1182,1196,1210,1112,1171,1185,1199,1213,1115,1786,1800,1814,1828,1789,1803,1817,1831,1792,1806,1820,1834,1795,1809,1823,1837,1798,1812,1826,1840,1801,1815,1829,1843,1804,1818,1832,1846,1807,1821,1835,1849,1824,1838,1852,1827,1841,1855,1830,1844,1746,1833,1847,1749,1836,1850,1752,1839,1853,1755,2454,2468,2457,2471,2460,2474,2463,2477,2466,2480,2469,2483,2472,2486,2475,2489,2492,2495,2386,2389,2392,2395,32768,32769,32770,32772,32773,32774,32776,32777,32778,32780,32781,32782,32784,32785,32786,32788,32789,32790,32792,32793,32794,32796,32797,32798,32800,32801,32802,32804,32805,32806,32808,32809,32810,32812,32813,32814,32816,32817,32818,32820,32821,32822,32824,32825,32826,32828,32829,32830,32832,32833,32834,32836,32837,32838,32840,32841,32842,32844,32845,32846,32848,32849,32850,32852,32853,32854,32856,32857,32858,32860,32861,32862,32864,32865,32866,32868,32869,32870,32872,32873,32874,32876,32877,32878,32880,32881,32882,32884,32885,32886,32888,32889,32890,32892,32893,32894,32896,32897,32898,32900,32901,32902,32904,32905,32906,32908,32909,32910,32912,32913,32914,32916,32917,32918,32920,32921,32922,32924,32925,32926,32928,32929,32930,32932,32933,32934,32936,32937,32938,32940,32941,32942,32944,32945,32946,32948,32949,32950,32952,32953,32954,32956,32957,32958,32960,32961,32962,32964,32965,32966,32968,32969,32970,32972,32973,32974,32976,32977,32978,32980,32981,32982,32984,32985,32986,32988,32989,32990,32992,32993,32994,32996,32997,32998,33000,33001,33002,33004,33005,33006,33008,33009,33010,33012,33013,33014,33016,33017,33018,33020,33021,33022,33024,33025,33026,33028,33029,33030,33032,33033,33034,33036,33037,33038,33040,33041,33042,33044,33045,33046,33048,33049,33050,33052,33053,33054,33056,33057,33058,33060,33061,33062,33064,33065,33066,33068,33069,33070,33072,33073,33074,33076,33077,33078,33080,33081,33082,33084,33085,33086,33088,33089,33090,33092,33093,33094,33096,33097,33098,33100,33101,33102,33104,33105,33106,33108,33109,33110,33112,33113,33114,33116,33117,33118,33120,33121,33122,33124,33125,33126,33128,33129,33130,33132,33133,33134,33136,33137,33138,33140,33141,33142,33144,33145,33146,33148,33149,33150,33152,33153,33154,33156,33157,33158,33160,33161,33162,33164,33165,33166,33168,33169,33170,33172,33173,33174,33176,33177,33178,33180,33181,33182,33184,33185,33186,33188,33189,33190,33192,33193,33194,33196,33197,33198,33200,33201,33202,33204,33205,33206,33208,33209,33210,33212,33213,33214,33216,33217,33218,33220,33221,33222,33224,33225,33226,33228,33229,33230,33232,33233,33234,33236,33237,33238,33240,33241,33242,33244,33245,33246,33248,33249,33250,33252,33253,33254,33256,33257,33258,33260,33261,33262,33264,33265,33266,33268,33269,33270,33272,33273,33274,33276,33277,33278,33280,33281,33282,33284,33285,33286,33288,33289,33290,33292,33293,33294,33296,33297,33298,33300,33301,33302,33304,33305,33306,33308,33309,33310,33312,33313,33314,33316,33317,33318,33320,33321,33322,33324,33325,33326,33328,33329,33330,33332,33333,33334,33336,33337,33338,33340,33341,33342,33344,33345,33346,33348,33349,33350,33352,33353,33354,33356,33357,33358,33360,33361,33362,33364,33365,33366,33368,33369,33370,33372,33373,33374,33376,33377,33378,33380,33381,33382,33384,33385,33386,33388,33389,33390,33392,33393,33394,33396,33397,33398,33400,33401,33402,33404,33405,33406,33408,33409,33410,33412,33413,33414,33416,33417,33418,33420,33421,33422,33424,33425,33426,33428,33429,33430,33432,33433,33434,33436,33437,33438,33440,33441,33442,33444,33445,33446,33448,33449,33450,33452,33453,33454,33456,33457,33458,33460,33461,33462,33464,33465,33466,33468,33469,33470,33472,33473,33474,33476,33477,33478,33480,33481,33482,33484,33485,33486,33488,33489,33490,33492,33493,33494,33496,33497,33498,33500,33501,33502,33504,33505,33506,33508,33509,33510,33512,33513,33514,33516,33517,33518,33520,33521,33522,33524,33525,33526,33528,33529,33530,33532,33533,33534,33536,33537,33538,33540,33541,33542,33544,33545,33546,33548,33549,33550,33552,33553,33554,33556,33557,33558,33560,33561,33562,33564,33565,33566,33568,33569,33570,33572,33573,33574,33576,33577,33578,33580,33581,33582,33584,33585,33586,33588,33589,33590,33592,33593,33594,33596,33597,33598,33600,33601,33602,33604,33605,33606,33608,33609,33610,33612,33613,33614,33616,33617,33618,33620,33621,33622,33624,33625,33626,33628,33629,33630,33632,33633,33634,33636,33637,33638,33640,33641,33642,33644,33645,33646,33648,33649,33650,33652,33653,33654,33656,33657,33658,33660,33661,33662,33664,33665,33666,33668,33669,33670,33672,33673,33674,33676,33677,33678,33680,33681,33682,33684,33685,33686,33688,33689,33690,33692,33693,33694,33696,33697,33698,33700,33701,33702,33704,33705,33706,33708,33709,33710,33712,33713,33714,33716,33717,33718,33720,33721,33722,33724,33725,33726,33728,33729,33730,33732,33733,33734,33736,33737,33738,33740,33741,33742,33744,33745,33746,33748,33749,33750,33752,33753,33754,33756,33757,33758,33760,33761,33762,33764,33765,33766,33768,33769,33770,33772,33773,33774,33776,33777,33778,33780,33781,33782,33784,33785,33786,33788,33789,33790,0,0,0,0,0,0,0,0,0,0,0,0,0,0,0,0,0,0,0,0,0,0,0,0,0,0,0,0,0,0,0,0,0,0,0,0,0,0,0,0,0,0,0,0,0,0,0,0,0,0,0,0,0,0,0,0,0,0,0,0,0,0,0,0,0,0,0,0,0,0,0,0,0,0,0,0,0,0,0,0,0,0,0,0,0,0,0,0,0,0,0,0,0,0,0,0,0,0,0,0,0,0,0,0,0,0,0,0,0,0,0,0,0,0,0,0,0,0,0,0,0,0,0,0,0,0,0,0,0,0,0,0,0,0,0,0,0,0,0,0,0,0,0,0,0,0,0,0,0,0,0,0,0,0,0,0,0,0,0,0,0,0,0,0,0,0,0,0,
0,1,2,3,4,5,6,7,8,9,10,11,12,13,14,15,16,17,18,19,20,21,22,23,24,25,26,27,28,29,30,31,32,33,34,35,36,37,38,39,40,41,42,43,44,45,46,47,48,49,50,51,52,53,54,55,56,57,58,59,60,61,62,63,64,65,66,67,68,69,70,71,72,73,74,75,76,77,78,79,80,81,82,83,84,85,86,87,88,89,90,91,92,93,94,95,96,97,98,99,100,101,102,103,104,105,106,107,108,109,110,111,112,113,114,115,116,117,118,119,120,121,122,123,124,125,126,127,576,577,226,578,579,229,580,581,232,582,583,235,584,585,238,586,587,129,588,589,132,590,591,135,16976,138,16977,141,16978,144,16979,147,16980,150,16981,153,640,641,642,643,644,645,646,647,648,649,650,651,652,653,654,655,656,657,658,659,660,661,662,663,664,665,666,667,668,669,670,671,672,673,674,675,676,677,678,679,680,681,682,683,684,685,686,687,688,689,690,691,692,693,694,695,696,697,698,699,700,701,702,703,704,705,706,707,708,709,710,711,712,713,714,715,716,717,718,719,720,721,722,723,724,725,726,727,728,729,730,731,732,733,734,735,736,737,738,739,740,741,742,743,744,745,746,747,748,749,750,751,752,753,754,755,756,757,758,759,760,761,762,763,764,765,766,767,768,1216,1217,866,771,1218,1219,869,774,1220,1221,872,777,1222,1223,875,780,1224,1225,878,783,1226,1227,769,786,1228,1229,772,789,1230,1231,775,792,806,17616,778,795,809,17617,781,798,812,17618,784,801,815,17619,787,804,818,17620,790,807,821,17621,793,1280,1281,1282,1283,1284,1285,1286,1287,1288,1289,1290,1291,1292,1293,1294,1295,1296,1297,1298,1299,1300,1301,1302,1303,1304,1305,1306,1307,1308,1309,1310,1311,1312,1313,1314,1315,1316,1317,1318,1319,1320,1321,1322,1323,1324,1325,1326,1327,1328,1329,1330,1331,1332,1333,1334,1335,1336,1337,1338,1339,1340,1341,1342,1343,1344,1345,1346,1347,1348,1349,1350,1351,1352,1353,1354,1355,1356,1357,1358,1359,1360,1361,1362,1363,1364,1365,1366,1367,1368,1369,1370,1371,1372,1373,1374,1375,1376,1377,1378,1379,1380,1381,1382,1383,1384,1385,1386,1387,1388,1389,1390,1391,1392,1393,1394,1395,1396,1397,1398,1399,1400,1401,1402,1403,1404,1405,1406,1407,1408,1422,1436,1856,1857,1506,1411,1425,1439,1858,1859,1509,1414,1428,1442,1860,1861,1512,1417,1431,1445,1862,1863,1515,1420,1434,1448,1864,1865,1518,1423,1437,1451,1866,1867,1409,1426,1440,1454,1868,1869,1412,1429,1443,1457,1870,1871,1415,1432,1446,1460,1474,18256,1418,1435,1449,1463,1477,18257,1421,1438,1452,1466,1480,18258,1424,1441,1455,1469,1483,18259,1427,1444,1458,1472,1486,18260,1430,1447,1461,1475,1489,18261,1433,1920,1921,1922,1923,1924,1925,1926,1927,1928,1929,1930,1931,1932,1933,1934,1935,1936,1937,1938,1939,1940,1941,1942,1943,1944,1945,1946,1947,1948,1949,1950,1951,1952,1953,1954,1955,1956,1957,1958,1959,1960,1961,1962,1963,1964,1965,1966,1967,1968,1969,1970,1971,1972,1973,1974,1975,1976,1977,1978,1979,1980,1981,1982,1983,1984,1985,1986,1987,1988,1989,1990,1991,1992,1993,1994,1995,1996,1997,1998,1999,2000,2001,2002,2003,2004,2005,2006,2007,2008,2009,2010,2011,2012,2013,2014,2015,2016,2017,2018,2019,2020,2021,2022,2023,2024,2025,2026,2027,2028,2029,2030,2031,2032,2033,2034,2035,2036,2037,2038,2039,2040,2041,2042,2043,2044,2045,2046,2047,2048,2062,2076,2090,2104,2496,2497,2146,2051,2065,2079,2093,2107,2498,2499,2149,2054,2068,2082,2096,2110,2500,2501,2152,2057,2071,2085,2099,2113,2502,2503,2155,2060,2074,2088,2102,2116,2504,2505,2158,2063,2077,2091,2105,2119,2506,2507,2049,2066,2080,2094,2108,2122,2508,2509,2052,2069,2083,2097,2111,2125,2510,2511,2055,2072,2086,2100,2114,2128,2142,18896,2058,2075,2089,2103,2117,2131,2145,18897,2061,2078,2092,2106,2120,2134,2148,18898,2064,2081,2095,2109,2123,2137,2151,18899,2067,2084,2098,2112,2126,2140,2154,18900,2070,2087,2101,2115,2129,2143,2157,18901,2073,128,142,156,170,184,198,212,338,131,145,159,173,187,201,215,341,134,148,162,176,190,204,218,344,137,151,165,179,193,207,221,347,140,154,168,182,196,210,224,350,143,157,171,185,199,213,227,241,146,160,174,188,202,216,230,244,149,163,177,191,205,219,233,247,152,166,180,194,208,222,236,250,155,169,183,197,211,225,239,253,158,172,186,200,214,228,130,256,161,175,189,203,217,231,133,259,164,178,192,206,220,234,136,262,167,181,195,209,223,237,139,265,880,782,796,810,824,838,852,978,883,785,799,813,827,841,855,981,886,788,802,816,830,844,858,984,889,791,805,819,833,847,861,987,892,794,808,822,836,850,864,990,895,797,811,825,839,853,867,881,898,800,814,828,842,856,870,884,901,803,817,831,845,859,873,887,904,918,820,834,848,862,876,890,907,921,823,837,851,865,879,893,910,924,826,840,854,868,770,896,913,927,829,843,857,871,773,899,916,930,832,846,860,874,776,902,919,933,835,849,863,877,779,905,1520,1534,1548,1450,1464,1478,1492,1618,1523,1537,1551,1453,1467,1481,1495,1621,1526,1540,1554,1456,1470,1484,1498,1624,1529,1543,1557,1459,1473,1487,1501,1627,1532,1546,1560,1462,1476,1490,1504,1630,1535,1549,1563,1465,1479,1493,1507,1521,1538,1552,1566,1468,1482,1496,1510,1524,1541,1555,1569,1471,1485,1499,1513,1527,1544,1558,1572,1586,1488,1502,1516,1530,1547,1561,1575,1589,1491,1505,1519,1533,1550,1564,1578,1592,1494,1508,1410,1536,1553,1567,1581,1595,1497,1511,1413,1539,1556,1570,1584,1598,1500,1514,1416,1542,1559,1573,1587,1601,1503,1517,1419,1545,2160,2174,2188,2202,2216,2118,2132,2258,2163,2177,2191,2205,2219,2121,2135,2261,2166,2180,2194,2208,2222,2124,2138,2264,2169,2183,2197,2211,2225,2127,2141,2267,2172,2186,2200,2214,2228,2130,2144,2270,2175,2189,2203,2217,2231,2133,2147,2161,2178,2192,2206,2220,2234,2136,2150,2164,2181,2195,2209,2223,2237,2139,2153,2167,2184,2198,2212,2226,2240,2254,2156,2170,2187,2201,2215,2229,2243,2257,2159,2173,2190,2204,2218,2232,2246,2260,2050,2176,2193,2207,2221,2235,2249,2263,2053,2179,2196,2210,2224,2238,2252,2266,2056,2182,2199,2213,2227,2241,2255,2269,2059,2185,240,254,268,282,296,310,324,450,243,257,271,285,299,313,327,453,246,260,274,288,302,316,330,456,249,263,277,291,305,319,333,459,252,266,280,294,308,322,336,462,255,269,283,297,311,325,339,353,258,272,286,300,314,328,342,356,261,275,289,303,317,331,345,359,264,278,292,306,320,334,348,362,267,281,295,309,323,337,351,365,270,284,298,312,326,340,242,368,273,287,301,315,329,343,245,371,276,290,304,318,332,346,248,374,279,293,307,321,335,349,251,377,992,894,908,922,936,950,964,1090,995,897,911,925,939,953,967,1093,998,900,914,928,942,956,970,1096,1001,903,917,931,945,959,973,1099,1004,906,920,934,948,962,976,1102,1007,909,923,937,951,965,979,993,1010,912,926,940,954,968,982,996,1013,915,929,943,957,971,985,999,1016,1030,932,946,960,974,988,1002,1019,1033,935,949,963,977,991,1005,1022,1036,938,952,966,980,882,1008,1025,1039,941,955,969,983,885,1011,1028,1042,944,958,972,986,888,1014,1031,1045,947,961,975,989,891,1017,1632,1646,1660,1562,1576,1590,1604,1730,1635,1649,1663,1565,1579,1593,1607,1733,1638,1652,1666,1568,1582,1596,1610,1736,1641,1655,1669,1571,1585,1599,1613,1739,1644,1658,1672,1574,1588,1602,1616,1742,1647,1661,1675,1577,1591,1605,1619,1633,1650,1664,1678,1580,1594,1608,1622,1636,1653,1667,1681,1583,1597,1611,1625,1639,1656,1670,1684,1698,1600,1614,1628,1642,1659,1673,1687,1701,1603,1617,1631,1645,1662,1676,1690,1704,1606,1620,1522,1648,1665,1679,1693,1707,1609,1623,1525,1651,1668,1682,1696,1710,1612,1626,1528,1654,1671,1685,1699,1713,1615,1629,1531,1657,2272,2286,2300,2314,2328,2230,2244,2370,2275,2289,2303,2317,2331,2233,2247,2373,2278,2292,2306,2320,2334,2236,2250,2376,2281,2295,2309,2323,2337,2239,2253,2379,2284,2298,2312,2326,2340,2242,2256,2382,2287,2301,2315,2329,2343,2245,2259,2273,2290,2304,2318,2332,2346,2248,2262,2276,2293,2307,2321,2335,2349,2251,2265,2279,2296,2310,2324,2338,2352,2366,2268,2282,2299,2313,2327,2341,2355,2369,2271,2285,2302,2316,2330,2344,2358,2372,2162,2288,2305,2319,2333,2347,2361,2375,2165,2291,2308,2322,2336,2350,2364,2378,2168,2294,2311,2325,2339,2353,2367,2381,2171,2297,352,366,380,394,408,422,436,562,355,369,383,397,411,425,439,565,358,372,386,400,414,428,442,568,361,375,389,403,417,431,445,571,364,378,392,406,420,434,448,574,367,381,395,409,423,437,451,465,370,384,398,412,426,440,454,468,373,387,401,415,429,443,457,471,376,390,404,418,432,446,460,474,379,393,407,421,435,449,463,477,382,396,410,424,438,452,354,480,385,399,413,427,441,455,357,483,388,402,416,430,444,458,360,486,391,405,419,433,447,461,363,489,1104,1006,1020,1034,1048,1062,1076,1202,1107,1009,1023,1037,1051,1065,1079,1205,1110,1012,1026,1040,1054,1068,1082,1208,1113,1015,1029,1043,1057,1071,1085,1211,1116,1018,1032,1046,1060,1074,1088,1214,1119,1021,1035,1049,1063,1077,1091,1105,1122,1024,1038,1052,1066,1080,1094,1108,1125,1027,1041,1055,1069,1083,1097,1111,1128,1142,1044,1058,1072,1086,1100,1114,1131,1145,1047,1061,1075,1089,1103,1117,1134,1148,1050,1064,1078,1092,994,1120,1137,1151,1053,1067,1081,1095,997,1123,1140,1154,1056,1070,1084,1098,1000,1126,1143,1157,1059,1073,1087,1101,1003,1129,1744,1758,1772,1674,1688,1702,1716,1842,1747,1761,1775,1677,1691,1705,1719,1845,1750,1764,1778,1680,1694,1708,1722,1848,1753,1767,1781,1683,1697,1711,1725,1851,1756,1770,1784,1686,1700,1714,1728,1854,1759,1773,1787,1689,1703,1717,1731,1745,1762,1776,1790,1692,1706,1720,1734,1748,1765,1779,1793,1695,1709,1723,1737,1751,1768,1782,1796,1810,1712,1726,1740,1754,1771,1785,1799,1813,1715,1729,1743,1757,1774,1788,1802,1816,1718,1732,1634,1760,1777,1791,1805,1819,1721,1735,1637,1763,1780,1794,1808,1822,1724,1738,1640,1766,1783,1797,1811,1825,1727,1741,1643,1769,2384,2398,2412,2426,2440,2342,2356,2482,2387,2401,2415,2429,2443,2345,2359,2485,2390,2404,2418,2432,2446,2348,2362,2488,2393,2407,2421,2435,2449,2351,2365,2491,2396,2410,2424,2438,2452,2354,2368,2494,2399,2413,2427,2441,2455,2357,2371,2385,2402,2416,2430,2444,2458,2360,2374,2388,2405,2419,2433,2447,2461,2363,2377,2391,2408,2422,2436,2450,2464,2478,2380,2394,2411,2425,2439,2453,2467,2481,2383,2397,2414,2428,2442,2456,2470,2484,2274,2400,2417,2431,2445,2459,2473,2487,2277,2403,2420,2434,2448,2462,2476,2490,2280,2406,2423,2437,2451,2465,2479,2493,2283,2409,464,478,492,506,520,534,548,467,481,495,509,523,537,551,470,484,498,512,526,540,554,473,487,501,515,529,543,557,476,490,504,518,532,546,560,479,493,507,521,535,549,563,482,496,510,524,538,552,566,485,499,513,527,541,555,569,488,502,516,530,544,558,572,491,505,519,533,547,561,575,494,508,522,536,550,564,466,497,511,525,539,553,567,469,500,514,528,542,556,570,472,503,517,531,545,559,573,475,1118,1132,1146,1160,1174,1188,1121,1135,1149,1163,1177,1191,1124,1138,1152,1166,1180,1194,1127,1141,1155,1169,1183,1197,1130,1144,1158,1172,1186,1200,1133,1147,1161,1175,1189,1203,1136,1150,1164,1178,1192,1206,1139,1153,1167,1181,1195,1209,1156,1170,1184,1198,1212,1159,1173,1187,1201,1215,1162,1176,1190,1204,1106,1165,1179,1193,1207,1109,1168,1182,1196,1210,1112,1171,1185,1199,1213,1115,1786,1800,1814,1828,1789,1803,1817,1831,1792,1806,1820,1834,1795,1809,1823,1837,1798,1812,1826,1840,1801,1815,1829,1843,1804,1818,1832,1846,1807,1821,1835,1849,1824,1838,1852,1827,1841,1855,1830,1844,1746,1833,1847,1749,1836,1850,1752,1839,1853,1755,2454,2468,2457,2471,2460,2474,2463,2477,2466,2480,2469,2483,2472,2486,2475,2489,2492,2495,2386,2389,2392,2395,32768,32769,32770,32772,32773,32774,32776,32777,32778,32780,32781,32782,32784,32785,32786,32788,32789,32790,32792,32793,32794,32796,32797,32798,32800,32801,32802,32804,32805,32806,32808,32809,32810,32812,32813,32814,32816,32817,32818,32820,32821,32822,32824,32825,32826,32828,32829,32830,32832,32833,32834,32836,32837,32838,32840,32841,32842,32844,32845,32846,32848,32849,32850,32852,32853,32854,32856,32857,32858,32860,32861,32862,32864,32865,32866,32868,32869,32870,32872,32873,32874,32876,32877,32878,32880,32881,32882,32884,32885,32886,32888,32889,32890,32892,32893,32894,32896,32897,32898,32900,32901,32902,32904,32905,32906,32908,32909,32910,32912,32913,32914,32916,32917,32918,32920,32921,32922,32924,32925,32926,32928,32929,32930,32932,32933,32934,32936,32937,32938,32940,32941,32942,32944,32945,32946,32948,32949,32950,32952,32953,32954,32956,32957,32958,32960,32961,32962,32964,32965,32966,32968,32969,32970,32972,32973,32974,32976,32977,32978,32980,32981,32982,32984,32985,32986,32988,32989,32990,32992,32993,32994,32996,32997,32998,33000,33001,33002,33004,33005,33006,33008,33009,33010,33012,33013,33014,33016,33017,33018,33020,33021,33022,33024,33025,33026,33028,33029,33030,33032,33033,33034,33036,33037,33038,33040,33041,33042,33044,33045,33046,33048,33049,33050,33052,33053,33054,33056,33057,33058,33060,33061,33062,33064,33065,33066,33068,33069,33070,33072,33073,33074,33076,33077,33078,33080,33081,33082,33084,33085,33086,33088,33089,33090,33092,33093,33094,33096,33097,33098,33100,33101,33102,33104,33105,33106,33108,33109,33110,33112,33113,33114,33116,33117,33118,33120,33121,33122,33124,33125,33126,33128,33129,33130,33132,33133,33134,33136,33137,33138,33140,33141,33142,33144,33145,33146,33148,33149,33150,33152,33153,33154,33156,33157,33158,33160,33161,33162,33164,33165,33166,33168,33169,33170,33172,33173,33174,33176,33177,33178,33180,33181,33182,33184,33185,33186,33188,33189,33190,33192,33193,33194,33196,33197,33198,33200,33201,33202,33204,33205,33206,33208,33209,33210,33212,33213,33214,33216,33217,33218,33220,33221,33222,33224,33225,33226,33228,33229,33230,33232,33233,33234,33236,33237,33238,33240,33241,33242,33244,33245,33246,33248,33249,33250,33252,33253,33254,33256,33257,33258,33260,33261,33262,33264,33265,33266,33268,33269,33270,33272,33273,33274,33276,33277,33278,33280,33281,33282,33284,33285,33286,33288,33289,33290,33292,33293,33294,33296,33297,33298,33300,33301,33302,33304,33305,33306,33308,33309,33310,33312,33313,33314,33316,33317,33318,33320,33321,33322,33324,33325,33326,33328,33329,33330,33332,33333,33334,33336,33337,33338,33340,33341,33342,33344,33345,33346,33348,33349,33350,33352,33353,33354,33356,33357,33358,33360,33361,33362,33364,33365,33366,33368,33369,33370,33372,33373,33374,33376,33377,33378,33380,33381,33382,33384,33385,33386,33388,33389,33390,33392,33393,33394,33396,33397,33398,33400,33401,33402,33404,33405,33406,33408,33409,33410,33412,33413,33414,33416,33417,33418,33420,33421,33422,33424,33425,33426,33428,33429,33430,33432,33433,33434,33436,33437,33438,33440,33441,33442,33444,33445,33446,33448,33449,33450,33452,33453,33454,33456,33457,33458,33460,33461,33462,33464,33465,33466,33468,33469,33470,33472,33473,33474,33476,33477,33478,33480,33481,33482,33484,33485,33486,33488,33489,33490,33492,33493,33494,33496,33497,33498,33500,33501,33502,33504,33505,33506,33508,33509,33510,33512,33513,33514,33516,33517,33518,33520,33521,33522,33524,33525,33526,33528,33529,33530,33532,33533,33534,33536,33537,33538,33540,33541,33542,33544,33545,33546,33548,33549,33550,33552,33553,33554,33556,33557,33558,33560,33561,33562,33564,33565,33566,33568,33569,33570,33572,33573,33574,33576,33577,33578,33580,33581,33582,33584,33585,33586,33588,33589,33590,33592,33593,33594,33596,33597,33598,33600,33601,33602,33604,33605,33606,33608,33609,33610,33612,33613,33614,33616,33617,33618,33620,33621,33622,33624,33625,33626,33628,33629,33630,33632,33633,33634,33636,33637,33638,33640,33641,33642,33644,33645,33646,33648,33649,33650,33652,33653,33654,33656,33657,33658,33660,33661,33662,33664,33665,33666,33668,33669,33670,33672,33673,33674,33676,33677,33678,33680,33681,33682,33684,33685,33686,33688,33689,33690,33692,33693,33694,33696,33697,33698,33700,33701,33702,33704,33705,33706,33708,33709,33710,33712,33713,33714,33716,33717,33718,33720,33721,33722,33724,33725,33726,33728,33729,33730,33732,33733,33734,33736,33737,33738,33740,33741,33742,33744,33745,33746,33748,33749,33750,33752,33753,33754,33756,33757,33758,33760,33761,33762,33764,33765,33766,33768,33769,33770,33772,33773,33774,33776,33777,33778,33780,33781,33782,33784,33785,33786,33788,33789,33790,0,0,0,0,0,0,0,0,0,0,0,0,0,0,0,0,0,0,0,0,0,0,0,0,0,0,0,0,0,0,0,0,0,0,0,0,0,0,0,0,0,0,0,0,0,0,0,0,0,0,0,0,0,0,0,0,0,0,0,0,0,0,0,0,0,0,0,0,0,0,0,0,0,0,0,0,0,0,0,0,0,0,0,0,0,0,0,0,0,0,0,0,0,0,0,0,0,0,0,0,0,0,0,0,0,0,0,0,0,0,0,0,0,0,0,0,0,0,0,0,0,0,0,0,0,0,0,0,0,0,0,0,0,0,0,0,0,0,0,0,0,0,0,0,0,0,0,0,0,0,0,0,0,0,0,0,0,0,0,0,0,0,0,0,0,0,0,0,
};
static __device__ const unsigned GEMM_TAB[14 * 192 * 2] = {
0x5000208,0x1c0,0x8000a08,0x2040000,0x8001208,0x2040000,0x8681a66,0x2040000,0x6002266,0x20801c1,0x8002a66,0x2080000,0x8003266,0x2080000,0x7783a76,0x20c0062,0x8004276,0x0,0x8004a76,0x4440000,0x8005276,0x4440000,0x8085b76,0x45c0000,0x6376,0x4800000,0x8006b76,0x4800000,0x8007376,0x4800000,0x8187971,0x4a00000,0x1008171,0xe,0x8008971,0x6f40000,0x8009171,0x6f40000,0x8289971,0x70c0000,0x200a171,0x768001c,0x800a971,0x7680000,0x800b171,0x7680000,0x838b971,0x7880000,0x300c171,0x2a,0x8000171,0x0,0x8000171,0x0,0x8480171,0x0,0x8000308,0x0,0x800ca08,0xa140000,0x800d208,0xa140000,0x800da08,0xa2c0000,0x400e071,0xac00038,0x800e971,0xac00000,0x800f171,0xac00000,0x858f971,0xae00000,0x5000171,0x46,0x8000971,0xba00000,0x8001171,0xba00000,0x86c1971,0xbbb8000,0x6002171,0xc800054,0x8002971,0xc800000,0x8003171,0xc800000,0x77c3971,0xca382d2,0x8004171,0x0,0x8004971,0xd600000,0x8005171,0xd600000,0x80c5971,0xd78c400,0x6171,0xe400070,0x8006971,0xe400000,0x8007171,0xe400000,0x81c7971,0xe600000,0x1008171,0x7e,0x8008971,0xf200000,0x8009171,0xf200000,0x82c9971,0xf381c00,0x200a171,0x1000008c,0x800a971,0x10000000,0x800b171,0x10000000,0x83cb971,0x10203800,0x300c171,0x9a,0x800c971,0x10e00000,0x800d171,0x10e00000,0x84cd971,0x10f85400,0x400e171,0x11c000a8,0x800e971,0x11c00000,0x800f171,0x11c00000,0x85cf971,0x11e07000,0x5000171,0xb6,0x8000971,0x12a00000,0x8001171,0x12a00000,0x86c1971,0x12b88c00,0x6002171,0x138000c4,0x8002971,0x13800000,0x8003171,0x13800000,0x77c3971,0x13a0a942,0x8004171,0x0,0x8004971,0x14600000,0x8005171,0x14600000,0x80c5971,0x1479a400,0x6171,0x154000e0,0x8006971,0x15400000,0x8007171,0x15400000,0x81c7971,0x1560e000,0x1008171,0xee,0x8008971,0x16200000,0x8009171,0x16200000,0x82c9971,0x1638fc00,0x200a171,0x170000fc,0x800a971,0x17000000,0x800b171,0x17000000,0x83cb971,0x17211800,0x300c171,0x10a,0x800c971,0x17e00000,0x800d171,0x17e00000,0x84cd971,0x17f93400,0x400e171,0x18c00118,0x800e971,0x18c00000,0x800f171,0x18c00000,0x85cf971,0x18e15000,0x5000171,0x126,0x8000971,0x19a00000,0x8001171,0x19a00000,0x86c1971,0x19b96c00,0x6002171,0x1a800134,0x8002971,0x1a800000,0x8003171,0x1a800000,0x77c3971,0x1aa189b2,0x8004171,0x0,0x8004971,0x1b600000,0x8005171,0x1b600000,0x80c5971,0x1b7a8400,0x6171,0x1c400150,0x8006971,0x1c400000,0x8007171,0x1c400000,0x81c7971,0x1c61c000,0x1008171,0x15e,0x8008971,0x1d200000,0x8009171,0x1d200000,0x82c9971,0x1d39dc00,0x200a171,0x1e00016c,0x800a971,0x1e000000,0x800b171,0x1e000000,0x83cb971,0x1e21f800,0x300c171,0x17a,0x800c971,0x1ee00000,0x800d171,0x1ee00000,0x84cd971,0x1efa1400,0x400e171,0x1fc00188,0x800e971,0x1fc00000,0x800f171,0x1fc00000,0x85cf971,0x1fe23000,0x5000171,0x196,0x8000971,0x20a00000,0x8001171,0x20a00000,0x86c1971,0x20ba4c00,0x6002171,0x218001a4,0x8002971,0x21800000,0x8003171,0x21800000,0x87c3971,0x219e6800,0x8004171,0x0,0x8004971,0x225c0000,0x8005171,0x225c0000,0x280c5871,0x225f6400,0x8006071,0x23200000,0x8006871,0x23200000,0x8007071,0x23200000,0x81c7a72,0x2322a000,0x8008272,0x0,0x8008a72,0x23c40000,0x8009272,0x23c40000,0x82c9a73,0x23c6bc00,0x800a273,0x24500000,0x800aa73,0x24500000,0x800b273,0x24500000,0x83cba74,0x2452d800,0x800c274,0x0,0x800ca74,0x24bc0000,0x800d274,0x24bc0000,0x84cda75,0x24bef400,0x800e275,0x25100000,0x800ea75,0x25100000,0x800f275,0x25100000,0x85cfa76,0x25131000,0x8000276,0x0,0x8000a76,0x25440000,0x8001276,0x25440000,0x86c1a77,0x25472c00,0x8000277,0x0,0x8000277,0x0,0x8000277,0x0,0x87c0208,0x34800,0x8000208,0x0,0x8000208,0x0,0x8000208,0x0,0x88000000,0x0,0x88000000,0x0,0x88000000,0x0,0x88000000,0x0,0x88000000,0x0,0x88000000,0x0,0x88000000,0x0,0x88000000,0x0,0x88000000,0x0,0x88000000,0x0,0x88000000,0x0,0x88000000,0x0,0x88000000,0x0,0x88000000,0x0,0x88000000,0x0,0x88000000,0x0,0x88000000,0x0,
0x5000208,0x1c2,0x8000a08,0x2080000,0x8001208,0x2080000,0x8681a66,0x2080000,0x6002266,0x21001c3,0x8002a66,0x2100000,0x8003266,0x2100000,0x7783a76,0x2180065,0x8004276,0x0,0x8004a76,0x4500000,0x8005276,0x4500000,0x8085b76,0x4680000,0x6376,0x4900003,0x8006b76,0x4900000,0x8007376,0x4900000,0x8187971,0x4b00000,0x1008171,0x11,0x8008971,0x7080000,0x8009171,0x7080000,0x8289971,0x7200000,0x200a171,0x780001f,0x800a971,0x7800000,0x800b171,0x7800000,0x838b971,0x7a00000,0x300c171,0x2d,0x8000171,0x0,0x8000171,0x0,0x8480171,0x0,0x8000308,0x0,0x800ca08,0xa300000,0x800d208,0xa300000,0x800da08,0xa480000,0x400e071,0xae0003b,0x800e971,0xae00000,0x800f171,0xae00000,0x858f971,0xb000000,0x5000171,0x49,0x8000971,0xbc00000,0x8001171,0xbc00000,0x86c1971,0xbdb8400,0x6002171,0xca00057,0x8002971,0xca00000,0x8003171,0xca00000,0x77c3971,0xcc386d5,0x8004171,0x0,0x8004971,0xd800000,0x8005171,0xd800000,0x80c5971,0xd98ca00,0x6171,0xe600073,0x8006971,0xe600000,0x8007171,0xe600000,0x81c7971,0xe800600,0x1008171,0x81,0x8008971,0xf400000,0x8009171,0xf400000,0x82c9971,0xf582200,0x200a171,0x1020008f,0x800a971,0x10200000,0x800b171,0x10200000,0x83cb971,0x10403e00,0x300c171,0x9d,0x800c971,0x11000000,0x800d171,0x11000000,0x84cd971,0x11185a00,0x400e171,0x11e000ab,0x800e971,0x11e00000,0x800f171,0x11e00000,0x85cf971,0x12007600,0x5000171,0xb9,0x8000971,0x12c00000,0x8001171,0x12c00000,0x86c1971,0x12d89200,0x6002171,0x13a000c7,0x8002971,0x13a00000,0x8003171,0x13a00000,0x77c3971,0x13c0af45,0x8004171,0x0,0x8004971,0x14800000,0x8005171,0x14800000,0x80c5971,0x1499aa00,0x6171,0x156000e3,0x8006971,0x15600000,0x8007171,0x15600000,0x81c7971,0x1580e600,0x1008171,0xf1,0x8008971,0x16400000,0x8009171,0x16400000,0x82c9971,0x16590200,0x200a171,0x172000ff,0x800a971,0x17200000,0x800b171,0x17200000,0x83cb971,0x17411e00,0x300c171,0x10d,0x800c971,0x18000000,0x800d171,0x18000000,0x84cd971,0x18193a00,0x400e171,0x18e0011b,0x800e971,0x18e00000,0x800f171,0x18e00000,0x85cf971,0x19015600,0x5000171,0x129,0x8000971,0x19c00000,0x8001171,0x19c00000,0x86c1971,0x19d97200,0x6002171,0x1aa00137,0x8002971,0x1aa00000,0x8003171,0x1aa00000,0x77c3971,0x1ac18fb5,0x8004171,0x0,0x8004971,0x1b800000,0x8005171,0x1b800000,0x80c5971,0x1b9a8a00,0x6171,0x1c600153,0x8006971,0x1c600000,0x8007171,0x1c600000,0x81c7971,0x1c81c600,0x1008171,0x161,0x8008971,0x1d400000,0x8009171,0x1d400000,0x82c9971,0x1d59e200,0x200a171,0x1e20016f,0x800a971,0x1e200000,0x800b171,0x1e200000,0x83cb971,0x1e41fe00,0x300c171,0x17d,0x800c971,0x1f000000,0x800d171,0x1f000000,0x84cd971,0x1f1a1a00,0x400e171,0x1fe0018b,0x800e971,0x1fe00000,0x800f171,0x1fe00000,0x85cf971,0x20023600,0x5000171,0x199,0x8000971,0x20c00000,0x8001171,0x20c00000,0x86c1971,0x20da5200,0x6002171,0x21a001a7,0x8002971,0x21a00000,0x8003171,0x21a00000,0x87c3971,0x21ba6e00,0x8004171,0x0,0x8004971,0x22780000,0x8005171,0x22780000,0x280c5871,0x227b6a00,0x8006071,0x23380000,0x8006871,0x23380000,0x8007071,0x23380000,0x81c7a72,0x233aa600,0x8008272,0x0,0x8008a72,0x23d80000,0x8009272,0x23d80000,0x82c9a73,0x23dac200,0x800a273,0x24600000,0x800aa73,0x24600000,0x800b273,0x24600000,0x83cba74,0x2462de00,0x800c274,0x0,0x800ca74,0x24c80000,0x800d274,0x24c80000,0x84cda75,0x24cafa00,0x800e275,0x25180000,0x800ea75,0x25180000,0x800f275,0x25180000,0x85cfa76,0x251b1600,0x8000276,0x0,0x8000a76,0x25480000,0x8001276,0x25480000,0x86c1a77,0x254b3200,0x8000277,0x0,0x8000277,0x0,0x8000277,0x0,0x87c0208,0x34e00,0x8000208,0x0,0x8000208,0x0,0x8000208,0x0,0x88000000,0x0,0x88000000,0x0,0x88000000,0x0,0x88000000,0x0,0x88000000,0x0,0x88000000,0x0,0x88000000,0x0,0x88000000,0x0,0x88000000,0x0,0x88000000,0x0,0x88000000,0x0,0x88000000,0x0,0x88000000,0x0,0x88000000,0x0,0x88000000,0x0,0x88000000,0x0,0x88000000,0x0,
0x5000208,0x1c4,0x8000a08,0x20c0000,0x8001208,0x20c0000,0x8681a66,0x20c0000,0x6002266,0x21801c5,0x8002a66,0x2180000,0x8003266,0x2180000,0x7783a76,0x2240068,0x8004276,0x0,0x8004a76,0x45c0000,0x8005276,0x45c0000,0x8085b76,0x4740000,0x6376,0x4a00006,0x8006b76,0x4a00000,0x8007376,0x4a00000,0x8187971,0x4c00000,0x1008171,0x14,0x8008971,0x71c0000,0x8009171,0x71c0000,0x8289971,0x7340000,0x200a171,0x7980022,0x800a971,0x7980000,0x800b171,0x7980000,0x838b971,0x7b80000,0x300c171,0x30,0x8000171,0x0,0x8000171,0x0,0x8480171,0x0,0x8000308,0x0,0x800ca08,0xa4c0000,0x800d208,0xa4c0000,0x800da08,0xa640000,0x400e071,0xb00003e,0x800e971,0xb000000,0x800f171,0xb000000,0x858f971,0xb200000,0x5000171,0x4c,0x8000971,0xbe00000,0x8001171,0xbe00000,0x86c1971,0xbfb8800,0x6002171,0xcc0005a,0x8002971,0xcc00000,0x8003171,0xcc00000,0x77c3971,0xce38ad8,0x8004171,0x0,0x8004971,0xda00000,0x8005171,0xda00000,0x80c5971,0xdb8d000,0x6171,0xe800076,0x8006971,0xe800000,0x8007171,0xe800000,0x81c7971,0xea00c00,0x1008171,0x84,0x8008971,0xf600000,0x8009171,0xf600000,0x82c9971,0xf782800,0x200a171,0x10400092,0x800a971,0x10400000,0x800b171,0x10400000,0x83cb971,0x10604400,0x300c171,0xa0,0x800c971,0x11200000,0x800d171,0x11200000,0x84cd971,0x11386000,0x400e171,0x120000ae,0x800e971,0x12000000,0x800f171,0x12000000,0x85cf971,0x12207c00,0x5000171,0xbc,0x8000971,0x12e00000,0x8001171,0x12e00000,0x86c1971,0x12f89800,0x6002171,0x13c000ca,0x8002971,0x13c00000,0x8003171,0x13c00000,0x77c3971,0x13e0b548,0x8004171,0x0,0x8004971,0x14a00000,0x8005171,0x14a00000,0x80c5971,0x14b9b000,0x6171,0x158000e6,0x8006971,0x15800000,0x8007171,0x15800000,0x81c7971,0x15a0ec00,0x1008171,0xf4,0x8008971,0x16600000,0x8009171,0x16600000,0x82c9971,0x16790800,0x200a171,0x17400102,0x800a971,0x17400000,0x800b171,0x17400000,0x83cb971,0x17612400,0x300c171,0x110,0x800c971,0x18200000,0x800d171,0x18200000,0x84cd971,0x18394000,0x400e171,0x1900011e,0x800e971,0x19000000,0x800f171,0x19000000,0x85cf971,0x19215c00,0x5000171,0x12c,0x8000971,0x19e00000,0x8001171,0x19e00000,0x86c1971,0x19f97800,0x6002171,0x1ac0013a,0x8002971,0x1ac00000,0x8003171,0x1ac00000,0x77c3971,0x1ae195b8,0x8004171,0x0,0x8004971,0x1ba00000,0x8005171,0x1ba00000,0x80c5971,0x1bba9000,0x6171,0x1c800156,0x8006971,0x1c800000,0x8007171,0x1c800000,0x81c7971,0x1ca1cc00,0x1008171,0x164,0x8008971,0x1d600000,0x8009171,0x1d600000,0x82c9971,0x1d79e800,0x200a171,0x1e400172,0x800a971,0x1e400000,0x800b171,0x1e400000,0x83cb971,0x1e620400,0x300c171,0x180,0x800c971,0x1f200000,0x800d171,0x1f200000,0x84cd971,0x1f3a2000,0x400e171,0x2000018e,0x800e971,0x20000000,0x800f171,0x20000000,0x85cf971,0x20223c00,0x5000171,0x19c,0x8000971,0x20e00000,0x8001171,0x20e00000,0x86c1971,0x20fa5800,0x6002171,0x21c001aa,0x8002971,0x21c00000,0x8003171,0x21c00000,0x87c3971,0x21d67400,0x8004171,0x0,0x8004971,0x22940000,0x8005171,0x22940000,0x280c5871,0x22977000,0x8006071,0x23500000,0x8006871,0x23500000,0x8007071,0x23500000,0x81c7a72,0x2352ac00,0x8008272,0x0,0x8008a72,0x23ec0000,0x8009272,0x23ec0000,0x82c9a73,0x23eec800,0x800a273,0x24700000,0x800aa73,0x24700000,0x800b273,0x24700000,0x83cba74,0x2472e400,0x800c274,0x0,0x800ca74,0x24d40000,0x800d274,0x24d40000,0x84cda75,0x24d70000,0x800e275,0x25200000,0x800ea75,0x25200000,0x800f275,0x25200000,0x85cfa76,0x25231c00,0x8000276,0x0,0x8000a76,0x254c0000,0x8001276,0x254c0000,0x86c1a77,0x254f3800,0x8000277,0x0,0x8000277,0x0,0x8000277,0x0,0x87c0208,0x35400,0x8000208,0x0,0x8000208,0x0,0x8000208,0x0,0x88000000,0x0,0x88000000,0x0,0x88000000,0x0,0x88000000,0x0,0x88000000,0x0,0x88000000,0x0,0x88000000,0x0,0x88000000,0x0,0x88000000,0x0,0x88000000,0x0,0x88000000,0x0,0x88000000,0x0,0x88000000,0x0,0x88000000,0x0,0x88000000,0x0,0x88000000,0x0,0x88000000,0x0,
0x5000208,0x1c6,0x8000a08,0x2100000,0x8001208,0x2100000,0x8681a66,0x2100000,0x6002266,0x22001c7,0x8002a66,0x2200000,0x8003266,0x2200000,0x7783a76,0x230006b,0x8004276,0x0,0x8004a76,0x4680000,0x8005276,0x4680000,0x8085b76,0x4800000,0x6376,0x4b00009,0x8006b76,0x4b00000,0x8007376,0x4b00000,0x8187971,0x4d00000,0x1008171,0x17,0x8008971,0x7300000,0x8009171,0x7300000,0x8289971,0x7480000,0x200a171,0x7b00025,0x800a971,0x7b00000,0x800b171,0x7b00000,0x838b971,0x7d00000,0x300c171,0x33,0x8000171,0x0,0x8000171,0x0,0x8480171,0x0,0x8000308,0x0,0x800ca08,0xa680000,0x800d208,0xa680000,0x800da08,0xa800000,0x400e071,0xb200041,0x800e971,0xb200000,0x800f171,0xb200000,0x858f971,0xb400000,0x5000171,0x4f,0x8000971,0xc000000,0x8001171,0xc000000,0x86c1971,0xc1b8c00,0x6002171,0xce0005d,0x8002971,0xce00000,0x8003171,0xce00000,0x77c3971,0xd038edb,0x8004171,0x0,0x8004971,0xdc00000,0x8005171,0xdc00000,0x80c5971,0xdd8d600,0x6171,0xea00079,0x8006971,0xea00000,0x8007171,0xea00000,0x81c7971,0xec01200,0x1008171,0x87,0x8008971,0xf800000,0x8009171,0xf800000,0x82c9971,0xf982e00,0x200a171,0x10600095,0x800a971,0x10600000,0x800b171,0x10600000,0x83cb971,0x10804a00,0x300c171,0xa3,0x800c971,0x11400000,0x800d171,0x11400000,0x84cd971,0x11586600,0x400e171,0x122000b1,0x800e971,0x12200000,0x800f171,0x12200000,0x85cf971,0x12408200,0x5000171,0xbf,0x8000971,0x13000000,0x8001171,0x13000000,0x86c1971,0x13189e00,0x6002171,0x13e000cd,0x8002971,0x13e00000,0x8003171,0x13e00000,0x77c3971,0x1400bb4b,0x8004171,0x0,0x8004971,0x14c00000,0x8005171,0x14c00000,0x80c5971,0x14d9b600,0x6171,0x15a000e9,0x8006971,0x15a00000,0x8007171,0x15a00000,0x81c7971,0x15c0f200,0x1008171,0xf7,0x8008971,0x16800000,0x8009171,0x16800000,0x82c9971,0x16990e00,0x200a171,0x17600105,0x800a971,0x17600000,0x800b171,0x17600000,0x83cb971,0x17812a00,0x300c171,0x113,0x800c971,0x18400000,0x800d171,0x18400000,0x84cd971,0x18594600,0x400e171,0x19200121,0x800e971,0x19200000,0x800f171,0x19200000,0x85cf971,0x19416200,0x5000171,0x12f,0x8000971,0x1a000000,0x8001171,0x1a000000,0x86c1971,0x1a197e00,0x6002171,0x1ae0013d,0x8002971,0x1ae00000,0x8003171,0x1ae00000,0x77c3971,0x1b019bbb,0x8004171,0x0,0x8004971,0x1bc00000,0x8005171,0x1bc00000,0x80c5971,0x1bda9600,0x6171,0x1ca00159,0x8006971,0x1ca00000,0x8007171,0x1ca00000,0x81c7971,0x1cc1d200,0x1008171,0x167,0x8008971,0x1d800000,0x8009171,0x1d800000,0x82c9971,0x1d99ee00,0x200a171,0x1e600175,0x800a971,0x1e600000,0x800b171,0x1e600000,0x83cb971,0x1e820a00,0x300c171,0x183,0x800c971,0x1f400000,0x800d171,0x1f400000,0x84cd971,0x1f5a2600,0x400e171,0x20200191,0x800e971,0x20200000,0x800f171,0x20200000,0x85cf971,0x20424200,0x5000171,0x19f,0x8000971,0x21000000,0x8001171,0x21000000,0x86c1971,0x211a5e00,0x6002171,0x21e001ad,0x8002971,0x21e00000,0x8003171,0x21e00000,0x87c3971,0x21f27a00,0x8004171,0x0,0x8004971,0x22b00000,0x8005171,0x22b00000,0x280c5871,0x22b37600,0x8006071,0x23680000,0x8006871,0x23680000,0x8007071,0x23680000,0x81c7a72,0x236ab200,0x8008272,0x0,0x8008a72,0x24000000,0x8009272,0x24000000,0x82c9a73,0x2402ce00,0x800a273,0x24800000,0x800aa73,0x24800000,0x800b273,0x24800000,0x83cba74,0x2482ea00,0x800c274,0x0,0x800ca74,0x24e00000,0x800d274,0x24e00000,0x84cda75,0x24e30600,0x800e275,0x25280000,0x800ea75,0x25280000,0x800f275,0x25280000,0x85cfa76,0x252b2200,0x8000276,0x0,0x8000a76,0x25500000,0x8001276,0x25500000,0x86c1a77,0x25533e00,0x8000277,0x0,0x8000277,0x0,0x8000277,0x0,0x87c0208,0x35a00,0x8000208,0x0,0x8000208,0x0,0x8000208,0x0,0x88000000,0x0,0x88000000,0x0,0x88000000,0x0,0x88000000,0x0,0x88000000,0x0,0x88000000,0x0,0x88000000,0x0,0x88000000,0x0,0x88000000,0x0,0x88000000,0x0,0x88000000,0x0,0x88000000,0x0,0x88000000,0x0,0x88000000,0x0,0x88000000,0x0,0x88000000,0x0,0x88000000,0x0,
0x5000208,0x1c8,0x8000a08,0x2140000,0x8001208,0x2140000,0x8681a66,0x2140000,0x6002266,0x22801c9,0x8002a66,0x2280000,0x8003266,0x2280000,0x7783a76,0x23c006e,0x8004276,0x0,0x8004a76,0x4740000,0x8005276,0x4740000,0x8085b76,0x48c0000,0x6376,0x4c0000c,0x8006b76,0x4c00000,0x8007376,0x4c00000,0x8187971,0x4e00000,0x1008171,0x1a,0x8008971,0x7440000,0x8009171,0x7440000,0x8289971,0x75c0000,0x200a171,0x7c80028,0x800a971,0x7c80000,0x800b171,0x7c80000,0x838b971,0x7e80000,0x300c171,0x36,0x8000171,0x0,0x8000171,0x0,0x8480171,0x0,0x8000308,0x0,0x800ca08,0xa840000,0x800d208,0xa840000,0x800da08,0xa9c0000,0x400e071,0xb400044,0x800e971,0xb400000,0x800f171,0xb400000,0x858f971,0xb600000,0x5000171,0x52,0x8000971,0xc200000,0x8001171,0xc200000,0x86c1971,0xc3b9000,0x6002171,0xd000060,0x8002971,0xd000000,0x8003171,0xd000000,0x77c3971,0xd2392de,0x8004171,0x0,0x8004971,0xde00000,0x8005171,0xde00000,0x80c5971,0xdf8dc00,0x6171,0xec0007c,0x8006971,0xec00000,0x8007171,0xec00000,0x81c7971,0xee01800,0x1008171,0x8a,0x8008971,0xfa00000,0x8009171,0xfa00000,0x82c9971,0xfb83400,0x200a171,0x10800098,0x800a971,0x10800000,0x800b171,0x10800000,0x83cb971,0x10a05000,0x300c171,0xa6,0x800c971,0x11600000,0x800d171,0x11600000,0x84cd971,0x11786c00,0x400e171,0x124000b4,0x800e971,0x12400000,0x800f171,0x12400000,0x85cf971,0x12608800,0x5000171,0xc2,0x8000971,0x13200000,0x8001171,0x13200000,0x86c1971,0x1338a400,0x6002171,0x140000d0,0x8002971,0x14000000,0x8003171,0x14000000,0x77c3971,0x1420c14e,0x8004171,0x0,0x8004971,0x14e00000,0x8005171,0x14e00000,0x80c5971,0x14f9bc00,0x6171,0x15c000ec,0x8006971,0x15c00000,0x8007171,0x15c00000,0x81c7971,0x15e0f800,0x1008171,0xfa,0x8008971,0x16a00000,0x8009171,0x16a00000,0x82c9971,0x16b91400,0x200a171,0x17800108,0x800a971,0x17800000,0x800b171,0x17800000,0x83cb971,0x17a13000,0x300c171,0x116,0x800c971,0x18600000,0x800d171,0x18600000,0x84cd971,0x18794c00,0x400e171,0x19400124,0x800e971,0x19400000,0x800f171,0x19400000,0x85cf971,0x19616800,0x5000171,0x132,0x8000971,0x1a200000,0x8001171,0x1a200000,0x86c1971,0x1a398400,0x6002171,0x1b000140,0x8002971,0x1b000000,0x8003171,0x1b000000,0x77c3971,0x1b21a1be,0x8004171,0x0,0x8004971,0x1be00000,0x8005171,0x1be00000,0x80c5971,0x1bfa9c00,0x6171,0x1cc0015c,0x8006971,0x1cc00000,0x8007171,0x1cc00000,0x81c7971,0x1ce1d800,0x1008171,0x16a,0x8008971,0x1da00000,0x8009171,0x1da00000,0x82c9971,0x1db9f400,0x200a171,0x1e800178,0x800a971,0x1e800000,0x800b171,0x1e800000,0x83cb971,0x1ea21000,0x300c171,0x186,0x800c971,0x1f600000,0x800d171,0x1f600000,0x84cd971,0x1f7a2c00,0x400e171,0x20400194,0x800e971,0x20400000,0x800f171,0x20400000,0x85cf971,0x20624800,0x5000171,0x1a2,0x8000971,0x21200000,0x8001171,0x21200000,0x86c1971,0x213a6400,0x6002171,0x220001b0,0x8002971,0x22000000,0x8003171,0x22000000,0x87c3971,0x220e8000,0x8004171,0x0,0x8004971,0x22cc0000,0x8005171,0x22cc0000,0x280c5871,0x22cf7c00,0x8006071,0x23800000,0x8006871,0x23800000,0x8007071,0x23800000,0x81c7a72,0x2382b800,0x8008272,0x0,0x8008a72,0x24140000,0x8009272,0x24140000,0x82c9a73,0x2416d400,0x800a273,0x24900000,0x800aa73,0x24900000,0x800b273,0x24900000,0x83cba74,0x2492f000,0x800c274,0x0,0x800ca74,0x24ec0000,0x800d274,0x24ec0000,0x84cda75,0x24ef0c00,0x800e275,0x25300000,0x800ea75,0x25300000,0x800f275,0x25300000,0x85cfa76,0x25332800,0x8000276,0x0,0x8000a76,0x25540000,0x8001276,0x25540000,0x86c1a77,0x25574400,0x8000277,0x0,0x8000277,0x0,0x8000277,0x0,0x87c0208,0x36000,0x8000208,0x0,0x8000208,0x0,0x8000208,0x0,0x88000000,0x0,0x88000000,0x0,0x88000000,0x0,0x88000000,0x0,0x88000000,0x0,0x88000000,0x0,0x88000000,0x0,0x88000000,0x0,0x88000000,0x0,0x88000000,0x0,0x88000000,0x0,0x88000000,0x0,0x88000000,0x0,0x88000000,0x0,0x88000000,0x0,0x88000000,0x0,0x88000000,0x0,
0x5000208,0x1ca,0x8000a08,0x2180000,0x8001208,0x2180000,0x8681a66,0x2180000,0x6002266,0x23001cb,0x8002a66,0x2300000,0x8003266,0x2300000,0x7783a76,0x2480001,0x8004276,0x0,0x8004a76,0x4800000,0x8005276,0x4800000,0x8085b76,0x4980000,0x6376,0x4d0000f,0x8006b76,0x4d00000,0x8007376,0x4d00000,0x8187971,0x4f00000,0x1008171,0x1d,0x8008971,0x7580000,0x8009171,0x7580000,0x8289971,0x7700000,0x200a171,0x7e0002b,0x800a971,0x7e00000,0x800b171,0x7e00000,0x838b971,0x8000000,0x300c171,0x39,0x8000171,0x0,0x8000171,0x0,0x8480171,0x0,0x8000308,0x0,0x800ca08,0xaa00000,0x800d208,0xaa00000,0x800da08,0xab80000,0x400e071,0xb600047,0x800e971,0xb600000,0x800f171,0xb600000,0x858f971,0xb800000,0x5000171,0x55,0x8000971,0xc400000,0x8001171,0xc400000,0x86c1971,0xc5b9400,0x6002171,0xd200063,0x8002971,0xd200000,0x8003171,0xd200000,0x77c3971,0xd439671,0x8004171,0x0,0x8004971,0xe000000,0x8005171,0xe000000,0x80c5971,0xe180200,0x6171,0xee0007f,0x8006971,0xee00000,0x8007171,0xee00000,0x81c7971,0xf001e00,0x1008171,0x8d,0x8008971,0xfc00000,0x8009171,0xfc00000,0x82c9971,0xfd83a00,0x200a171,0x10a0009b,0x800a971,0x10a00000,0x800b171,0x10a00000,0x83cb971,0x10c05600,0x300c171,0xa9,0x800c971,0x11800000,0x800d171,0x11800000,0x84cd971,0x11987200,0x400e171,0x126000b7,0x800e971,0x12600000,0x800f171,0x12600000,0x85cf971,0x12808e00,0x5000171,0xc5,0x8000971,0x13400000,0x8001171,0x13400000,0x86c1971,0x1358aa00,0x6002171,0x142000d3,0x8002971,0x14200000,0x8003171,0x14200000,0x77c3971,0x1440c6e1,0x8004171,0x0,0x8004971,0x15000000,0x8005171,0x15000000,0x80c5971,0x1518e200,0x6171,0x15e000ef,0x8006971,0x15e00000,0x8007171,0x15e00000,0x81c7971,0x1600fe00,0x1008171,0xfd,0x8008971,0x16c00000,0x8009171,0x16c00000,0x82c9971,0x16d91a00,0x200a171,0x17a0010b,0x800a971,0x17a00000,0x800b171,0x17a00000,0x83cb971,0x17c13600,0x300c171,0x119,0x800c971,0x18800000,0x800d171,0x18800000,0x84cd971,0x18995200,0x400e171,0x19600127,0x800e971,0x19600000,0x800f171,0x19600000,0x85cf971,0x19816e00,0x5000171,0x135,0x8000971,0x1a400000,0x8001171,0x1a400000,0x86c1971,0x1a598a00,0x6002171,0x1b200143,0x8002971,0x1b200000,0x8003171,0x1b200000,0x77c3971,0x1b41a751,0x8004171,0x0,0x8004971,0x1c000000,0x8005171,0x1c000000,0x80c5971,0x1c19c200,0x6171,0x1ce0015f,0x8006971,0x1ce00000,0x8007171,0x1ce00000,0x81c7971,0x1d01de00,0x1008171,0x16d,0x8008971,0x1dc00000,0x8009171,0x1dc00000,0x82c9971,0x1dd9fa00,0x200a171,0x1ea0017b,0x800a971,0x1ea00000,0x800b171,0x1ea00000,0x83cb971,0x1ec21600,0x300c171,0x189,0x800c971,0x1f800000,0x800d171,0x1f800000,0x84cd971,0x1f9a3200,0x400e171,0x20600197,0x800e971,0x20600000,0x800f171,0x20600000,0x85cf971,0x20824e00,0x5000171,0x1a5,0x8000971,0x21400000,0x8001171,0x21400000,0x86c1971,0x215a6a00,0x6002171,0x222001b3,0x8002971,0x22200000,0x8003171,0x22200000,0x87c3971,0x222a8600,0x8004171,0x0,0x8004971,0x22e80000,0x8005171,0x22e80000,0x280c5871,0x22eaa200,0x8006071,0x23980000,0x8006871,0x23980000,0x8007071,0x23980000,0x81c7a72,0x239abe00,0x8008272,0x0,0x8008a72,0x24280000,0x8009272,0x24280000,0x82c9a73,0x242ada00,0x800a273,0x24a00000,0x800aa73,0x24a00000,0x800b273,0x24a00000,0x83cba74,0x24a2f600,0x800c274,0x0,0x800ca74,0x24f80000,0x800d274,0x24f80000,0x84cda75,0x24fb1200,0x800e275,0x25380000,0x800ea75,0x25380000,0x800f275,0x25380000,0x85cfa76,0x253b2e00,0x8000276,0x0,0x8000a76,0x25580000,0x8001276,0x25580000,0x86c1a77,0x255b4a00,0x8000277,0x0,0x8000277,0x0,0x8000277,0x0,0x87c0208,0x36600,0x8000208,0x0,0x8000208,0x0,0x8000208,0x0,0x88000000,0x0,0x88000000,0x0,0x88000000,0x0,0x88000000,0x0,0x88000000,0x0,0x88000000,0x0,0x88000000,0x0,0x88000000,0x0,0x88000000,0x0,0x88000000,0x0,0x88000000,0x0,0x88000000,0x0,0x88000000,0x0,0x88000000,0x0,0x88000000,0x0,0x88000000,0x0,0x88000000,0x0,
0x5000208,0x1cc,0x8000a08,0x21c0000,0x8001208,0x21c0000,0x8681a66,0x21c0000,0x6002266,0x23801cd,0x8002a66,0x2380000,0x8003266,0x2380000,0x7783a76,0x2540004,0x8004276,0x0,0x8004a76,0x48c0000,0x8005276,0x48c0000,0x8085b76,0x4a40000,0x6376,0x4e00012,0x8006b76,0x4e00000,0x8007376,0x4e00000,0x8187971,0x5000000,0x1008171,0x20,0x8008971,0x76c0000,0x8009171,0x76c0000,0x8289971,0x7840000,0x200a171,0x7f8002e,0x800a971,0x7f80000,0x800b171,0x7f80000,0x838b971,0x8180000,0x300c171,0x3c,0x8000171,0x0,0x8000171,0x0,0x8480171,0x0,0x8000308,0x0,0x800ca08,0xabc0000,0x800d208,0xabc0000,0x800da08,0xad40000,0x400e071,0xb80004a,0x800e971,0xb800000,0x800f171,0xb800000,0x858f971,0xba00000,0x5000171,0x58,0x8000971,0xc600000,0x8001171,0xc600000,0x86c1971,0xc7b9800,0x6002171,0xd400066,0x8002971,0xd400000,0x8003171,0xd400000,0x77c3971,0xd639a74,0x8004171,0x0,0x8004971,0xe200000,0x8005171,0xe200000,0x80c5971,0xe380800,0x6171,0xf000082,0x8006971,0xf000000,0x8007171,0xf000000,0x81c7971,0xf202400,0x1008171,0x90,0x8008971,0xfe00000,0x8009171,0xfe00000,0x82c9971,0xff84000,0x200a171,0x10c0009e,0x800a971,0x10c00000,0x800b171,0x10c00000,0x83cb971,0x10e05c00,0x300c171,0xac,0x800c971,0x11a00000,0x800d171,0x11a00000,0x84cd971,0x11b87800,0x400e171,0x128000ba,0x800e971,0x12800000,0x800f171,0x12800000,0x85cf971,0x12a09400,0x5000171,0xc8,0x8000971,0x13600000,0x8001171,0x13600000,0x86c1971,0x1378b000,0x6002171,0x144000d6,0x8002971,0x14400000,0x8003171,0x14400000,0x77c3971,0x1460cce4,0x8004171,0x0,0x8004971,0x15200000,0x8005171,0x15200000,0x80c5971,0x1538e800,0x6171,0x160000f2,0x8006971,0x16000000,0x8007171,0x16000000,0x81c7971,0x16210400,0x1008171,0x100,0x8008971,0x16e00000,0x8009171,0x16e00000,0x82c9971,0x16f92000,0x200a171,0x17c0010e,0x800a971,0x17c00000,0x800b171,0x17c00000,0x83cb971,0x17e13c00,0x300c171,0x11c,0x800c971,0x18a00000,0x800d171,0x18a00000,0x84cd971,0x18b95800,0x400e171,0x1980012a,0x800e971,0x19800000,0x800f171,0x19800000,0x85cf971,0x19a17400,0x5000171,0x138,0x8000971,0x1a600000,0x8001171,0x1a600000,0x86c1971,0x1a799000,0x6002171,0x1b400146,0x8002971,0x1b400000,0x8003171,0x1b400000,0x77c3971,0x1b61ad54,0x8004171,0x0,0x8004971,0x1c200000,0x8005171,0x1c200000,0x80c5971,0x1c39c800,0x6171,0x1d000162,0x8006971,0x1d000000,0x8007171,0x1d000000,0x81c7971,0x1d21e400,0x1008171,0x170,0x8008971,0x1de00000,0x8009171,0x1de00000,0x82c9971,0x1dfa0000,0x200a171,0x1ec0017e,0x800a971,0x1ec00000,0x800b171,0x1ec00000,0x83cb971,0x1ee21c00,0x300c171,0x18c,0x800c971,0x1fa00000,0x800d171,0x1fa00000,0x84cd971,0x1fba3800,0x400e171,0x2080019a,0x800e971,0x20800000,0x800f171,0x20800000,0x85cf971,0x20a25400,0x5000171,0x1a8,0x8000971,0x21600000,0x8001171,0x21600000,0x86c1971,0x217a7000,0x6002171,0x224001b6,0x8002971,0x22400000,0x8003171,0x22400000,0x87c3971,0x22468c00,0x8004171,0x0,0x8004971,0x23040000,0x8005171,0x23040000,0x280c5871,0x2306a800,0x8006071,0x23b00000,0x8006871,0x23b00000,0x8007071,0x23b00000,0x81c7a72,0x23b2c400,0x8008272,0x0,0x8008a72,0x243c0000,0x8009272,0x243c0000,0x82c9a73,0x243ee000,0x800a273,0x24b00000,0x800aa73,0x24b00000,0x800b273,0x24b00000,0x83cba74,0x24b2fc00,0x800c274,0x0,0x800ca74,0x25040000,0x800d274,0x25040000,0x84cda75,0x25071800,0x800e275,0x25400000,0x800ea75,0x25400000,0x800f275,0x25400000,0x85cfa76,0x25433400,0x8000276,0x0,0x8000a76,0x255c0000,0x8001276,0x255c0000,0x86c1a77,0x255f5000,0x8000277,0x0,0x8000277,0x0,0x8000277,0x0,0x87c0208,0x36c00,0x8000208,0x0,0x8000208,0x0,0x8000208,0x0,0x88000000,0x0,0x88000000,0x0,0x88000000,0x0,0x88000000,0x0,0x88000000,0x0,0x88000000,0x0,0x88000000,0x0,0x88000000,0x0,0x88000000,0x0,0x88000000,0x0,0x88000000,0x0,0x88000000,0x0,0x88000000,0x0,0x88000000,0x0,0x88000000,0x0,0x88000000,0x0,0x88000000,0x0,
0x5000208,0x1ce,0x8000a08,0x2200000,0x8001208,0x2200000,0x8681a66,0x2200000,0x6002266,0x24001cf,0x8002a66,0x2400000,0x8003266,0x2400000,0x7783a76,0x2600007,0x8004276,0x0,0x8004a76,0x4980000,0x8005276,0x4980000,0x8085b76,0x4b00000,0x6376,0x4f00015,0x8006b76,0x4f00000,0x8007376,0x4f00000,0x8187971,0x5100000,0x1008171,0x23,0x8008971,0x7800000,0x8009171,0x7800000,0x8289971,0x7980000,0x200a171,0x8100031,0x800a971,0x8100000,0x800b171,0x8100000,0x838b971,0x8300000,0x300c171,0x3f,0x8000171,0x0,0x8000171,0x0,0x8480171,0x0,0x8000308,0x0,0x800ca08,0xad80000,0x800d208,0xad80000,0x800da08,0xaf00000,0x400e071,0xba0004d,0x800e971,0xba00000,0x800f171,0xba00000,0x858f971,0xbc00000,0x5000171,0x5b,0x8000971,0xc800000,0x8001171,0xc800000,0x86c1971,0xc9b9c00,0x6002171,0xd600069,0x8002971,0xd600000,0x8003171,0xd600000,0x77c3971,0xd839e77,0x8004171,0x0,0x8004971,0xe400000,0x8005171,0xe400000,0x80c5971,0xe580e00,0x6171,0xf200085,0x8006971,0xf200000,0x8007171,0xf200000,0x81c7971,0xf402a00,0x1008171,0x93,0x8008971,0x10000000,0x8009171,0x10000000,0x82c9971,0x10184600,0x200a171,0x10e000a1,0x800a971,0x10e00000,0x800b171,0x10e00000,0x83cb971,0x11006200,0x300c171,0xaf,0x800c971,0x11c00000,0x800d171,0x11c00000,0x84cd971,0x11d87e00,0x400e171,0x12a000bd,0x800e971,0x12a00000,0x800f171,0x12a00000,0x85cf971,0x12c09a00,0x5000171,0xcb,0x8000971,0x13800000,0x8001171,0x13800000,0x86c1971,0x1398b600,0x6002171,0x146000d9,0x8002971,0x14600000,0x8003171,0x14600000,0x77c3971,0x1480d2e7,0x8004171,0x0,0x8004971,0x15400000,0x8005171,0x15400000,0x80c5971,0x1558ee00,0x6171,0x162000f5,0x8006971,0x16200000,0x8007171,0x16200000,0x81c7971,0x16410a00,0x1008171,0x103,0x8008971,0x17000000,0x8009171,0x17000000,0x82c9971,0x17192600,0x200a171,0x17e00111,0x800a971,0x17e00000,0x800b171,0x17e00000,0x83cb971,0x18014200,0x300c171,0x11f,0x800c971,0x18c00000,0x800d171,0x18c00000,0x84cd971,0x18d95e00,0x400e171,0x19a0012d,0x800e971,0x19a00000,0x800f171,0x19a00000,0x85cf971,0x19c17a00,0x5000171,0x13b,0x8000971,0x1a800000,0x8001171,0x1a800000,0x86c1971,0x1a999600,0x6002171,0x1b600149,0x8002971,0x1b600000,0x8003171,0x1b600000,0x77c3971,0x1b81b357,0x8004171,0x0,0x8004971,0x1c400000,0x8005171,0x1c400000,0x80c5971,0x1c59ce00,0x6171,0x1d200165,0x8006971,0x1d200000,0x8007171,0x1d200000,0x81c7971,0x1d41ea00,0x1008171,0x173,0x8008971,0x1e000000,0x8009171,0x1e000000,0x82c9971,0x1e1a0600,0x200a171,0x1ee00181,0x800a971,0x1ee00000,0x800b171,0x1ee00000,0x83cb971,0x1f022200,0x300c171,0x18f,0x800c971,0x1fc00000,0x800d171,0x1fc00000,0x84cd971,0x1fda3e00,0x400e171,0x20a0019d,0x800e971,0x20a00000,0x800f171,0x20a00000,0x85cf971,0x20c25a00,0x5000171,0x1ab,0x8000971,0x21800000,0x8001171,0x21800000,0x86c1971,0x219a7600,0x6002171,0x226001b9,0x8002971,0x22600000,0x8003171,0x22600000,0x87c3971,0x22629200,0x8004171,0x0,0x8004971,0x23200000,0x8005171,0x23200000,0x280c5871,0x2322ae00,0x8006071,0x23c80000,0x8006871,0x23c80000,0x8007071,0x23c80000,0x81c7a72,0x23caca00,0x8008272,0x0,0x8008a72,0x24500000,0x8009272,0x24500000,0x82c9a73,0x2452e600,0x800a273,0x24c00000,0x800aa73,0x24c00000,0x800b273,0x24c00000,0x83cba74,0x24c30200,0x800c274,0x0,0x800ca74,0x25100000,0x800d274,0x25100000,0x84cda75,0x25131e00,0x800e275,0x25480000,0x800ea75,0x25480000,0x800f275,0x25480000,0x85cfa76,0x254b3a00,0x8000276,0x0,0x8000a76,0x25600000,0x8001276,0x25600000,0x86c1a77,0x25635600,0x8000277,0x0,0x8000277,0x0,0x8000277,0x0,0x87c0208,0x37200,0x8000208,0x0,0x8000208,0x0,0x8000208,0x0,0x88000000,0x0,0x88000000,0x0,0x88000000,0x0,0x88000000,0x0,0x88000000,0x0,0x88000000,0x0,0x88000000,0x0,0x88000000,0x0,0x88000000,0x0,0x88000000,0x0,0x88000000,0x0,0x88000000,0x0,0x88000000,0x0,0x88000000,0x0,0x88000000,0x0,0x88000000,0x0,0x88000000,0x0,
0x6000208,0x1d0,0x8000a08,0x2240000,0x8001208,0x2240000,0x7781a77,0x228000a,0x8002277,0x2480000,0x8002a77,0x2480000,0x8003277,0x2480000,0x8083b77,0x2680000,0x4377,0x18,0x8004b77,0x4a40000,0x8005377,0x4a40000,0x8185971,0x4bc0000,0x1006171,0x5000026,0x8006971,0x5000000,0x8007171,0x5000000,0x8287971,0x5200000,0x2008171,0x34,0x8008971,0x7940000,0x8009171,0x7940000,0x8389971,0x7ac0000,0x300a171,0x8280042,0x800a971,0x8280000,0x800b171,0x8280000,0x848b971,0x8480000,0x400c171,0x50,0x8000171,0x0,0x8000171,0x0,0x8580171,0x0,0x8000308,0x0,0x800ca08,0xaf40000,0x800d208,0xaf40000,0x800da08,0xb0c0000,0x500e071,0xbc0005e,0x800e971,0xbc00000,0x800f171,0xbc00000,0x868f971,0xbe00000,0x6000171,0x6c,0x8000971,0xca00000,0x8001171,0xca00000,0x177c1971,0xcbba07a,0x18002171,0xd800000,0x18002971,0xd800000,0x18003171,0xd800000,0x80c3971,0xda01400,0x4171,0x88,0x48804971,0xe600000,0x8005171,0xe600000,0x81c5971,0xe783000,0x1006171,0xf400096,0x8006971,0xf400000,0x8007171,0xf400000,0x82c7971,0xf604c00,0x2008171,0xa4,0x8008971,0x10200000,0x8009171,0x10200000,0x83c9971,0x10386800,0x300a171,0x110000b2,0x800a971,0x11000000,0x800b171,0x11000000,0x84cb971,0x11208400,0x400c171,0xc0,0x800c971,0x11e00000,0x800d171,0x11e00000,0x85cd971,0x11f8a000,0x500e171,0x12c000ce,0x800e971,0x12c00000,0x800f171,0x12c00000,0x86cf971,0x12e0bc00,0x6000171,0xdc,0x8000971,0x13a00000,0x8001171,0x13a00000,0x77c1971,0x13b8d8ea,0x8002171,0x14800000,0x8002971,0x14800000,0x8003171,0x14800000,0x80c3971,0x14a0f400,0x4171,0xf8,0x8004971,0x15600000,0x8005171,0x15600000,0x81c5971,0x15791000,0x1006171,0x16400106,0x8006971,0x16400000,0x8007171,0x16400000,0x82c7971,0x16612c00,0x2008171,0x114,0x8008971,0x17200000,0x8009171,0x17200000,0x83c9971,0x17394800,0x300a171,0x18000122,0x800a971,0x18000000,0x800b171,0x18000000,0x84cb971,0x18216400,0x400c171,0x130,0x800c971,0x18e00000,0x800d171,0x18e00000,0x85cd971,0x18f98000,0x500e171,0x19c0013e,0x800e971,0x19c00000,0x800f171,0x19c00000,0x86cf971,0x19e19c00,0x6000171,0x14c,0x8000971,0x1aa00000,0x8001171,0x1aa00000,0x77c1971,0x1ab9b95a,0x8002171,0x1b800000,0x8002971,0x1b800000,0x8003171,0x1b800000,0x80c3971,0x1ba1d400,0x4171,0x168,0x8004971,0x1c600000,0x8005171,0x1c600000,0x81c5971,0x1c79f000,0x1006171,0x1d400176,0x8006971,0x1d400000,0x8007171,0x1d400000,0x82c7971,0x1d620c00,0x2008171,0x184,0x8008971,0x1e200000,0x8009171,0x1e200000,0x83c9971,0x1e3a2800,0x300a171,0x1f000192,0x800a971,0x1f000000,0x800b171,0x1f000000,0x84cb971,0x1f224400,0x400c171,0x1a0,0x800c971,0x1fe00000,0x800d171,0x1fe00000,0x85cd971,0x1ffa6000,0x500e171,0x20c001ae,0x800e971,0x20c00000,0x800f171,0x20c00000,0x86cf971,0x20e27c00,0x6000171,0x1bc,0x8000971,0x21a00000,0x8001171,0x21a00000,0x87c1971,0x21b69800,0x8002171,0x227c0000,0x8002971,0x227c0000,0x8003171,0x227c0000,0x280c3871,0x227eb400,0x8004071,0x0,0x8004871,0x23380000,0x8005071,0x23380000,0x81c5a72,0x233ad000,0x8006272,0x23dc0000,0x8006a72,0x23dc0000,0x8007272,0x23dc0000,0x82c7a73,0x23deec00,0x8008273,0x0,0x8008a73,0x24600000,0x8009273,0x24600000,0x83c9a74,0x24630800,0x800a274,0x24cc0000,0x800aa74,0x24cc0000,0x800b274,0x24cc0000,0x84cba75,0x24cf2400,0x800c275,0x0,0x800ca75,0x25180000,0x800d275,0x25180000,0x85cda76,0x251b4000,0x800e276,0x254c0000,0x800ea76,0x254c0000,0x800f276,0x254c0000,0x86cfa77,0x254f5c00,0x8000277,0x0,0x8000277,0x0,0x8000277,0x0,0x87c0208,0x37800,0x8000208,0x0,0x8000208,0x0,0x8000208,0x0,0x88000000,0x0,0x88000000,0x0,0x88000000,0x0,0x88000000,0x0,0x88000000,0x0,0x88000000,0x0,0x88000000,0x0,0x88000000,0x0,0x88000000,0x0,0x88000000,0x0,0x88000000,0x0,0x88000000,0x0,0x88000000,0x0,0x88000000,0x0,0x88000000,0x0,0x88000000,0x0,0x88000000,0x0,0x88000000,0x0,0x88000000,0x0,0x88000000,0x0,0x88000000,0x0,
0x6000208,0x1d1,0x8000a08,0x2280000,0x8001208,0x2280000,0x7781a77,0x230000d,0x8002277,0x2500000,0x8002a77,0x2500000,0x8003277,0x2500000,0x8083b77,0x2700000,0x4377,0x1b,0x8004b77,0x4b00000,0x8005377,0x4b00000,0x8185971,0x4c80000,0x1006171,0x5100029,0x8006971,0x5100000,0x8007171,0x5100000,0x8287971,0x5300000,0x2008171,0x37,0x8008971,0x7a80000,0x8009171,0x7a80000,0x8389971,0x7c00000,0x300a171,0x8400045,0x800a971,0x8400000,0x800b171,0x8400000,0x848b971,0x8600000,0x400c171,0x53,0x8000171,0x0,0x8000171,0x0,0x8580171,0x0,0x8000308,0x0,0x800ca08,0xb100000,0x800d208,0xb100000,0x800da08,0xb280000,0x500e071,0xbe00061,0x800e971,0xbe00000,0x800f171,0xbe00000,0x868f971,0xc000000,0x6000171,0x6f,0x8000971,0xcc00000,0x8001171,0xcc00000,0x177c1971,0xcdba27d,0x18002171,0xda00000,0x18002971,0xda00000,0x18003171,0xda00000,0x80c3971,0xdc01a00,0x4171,0x8b,0x48804971,0xe800000,0x8005171,0xe800000,0x81c5971,0xe983600,0x1006171,0xf600099,0x8006971,0xf600000,0x8007171,0xf600000,0x82c7971,0xf805200,0x2008171,0xa7,0x8008971,0x10400000,0x8009171,0x10400000,0x83c9971,0x10586e00,0x300a171,0x112000b5,0x800a971,0x11200000,0x800b171,0x11200000,0x84cb971,0x11408a00,0x400c171,0xc3,0x800c971,0x12000000,0x800d171,0x12000000,0x85cd971,0x1218a600,0x500e171,0x12e000d1,0x800e971,0x12e00000,0x800f171,0x12e00000,0x86cf971,0x1300c200,0x6000171,0xdf,0x8000971,0x13c00000,0x8001171,0x13c00000,0x77c1971,0x13d8deed,0x8002171,0x14a00000,0x8002971,0x14a00000,0x8003171,0x14a00000,0x80c3971,0x14c0fa00,0x4171,0xfb,0x8004971,0x15800000,0x8005171,0x15800000,0x81c5971,0x15991600,0x1006171,0x16600109,0x8006971,0x16600000,0x8007171,0x16600000,0x82c7971,0x16813200,0x2008171,0x117,0x8008971,0x17400000,0x8009171,0x17400000,0x83c9971,0x17594e00,0x300a171,0x18200125,0x800a971,0x18200000,0x800b171,0x18200000,0x84cb971,0x18416a00,0x400c171,0x133,0x800c971,0x19000000,0x800d171,0x19000000,0x85cd971,0x19198600,0x500e171,0x19e00141,0x800e971,0x19e00000,0x800f171,0x19e00000,0x86cf971,0x1a01a200,0x6000171,0x14f,0x8000971,0x1ac00000,0x8001171,0x1ac00000,0x77c1971,0x1ad9bf5d,0x8002171,0x1ba00000,0x8002971,0x1ba00000,0x8003171,0x1ba00000,0x80c3971,0x1bc1da00,0x4171,0x16b,0x8004971,0x1c800000,0x8005171,0x1c800000,0x81c5971,0x1c99f600,0x1006171,0x1d600179,0x8006971,0x1d600000,0x8007171,0x1d600000,0x82c7971,0x1d821200,0x2008171,0x187,0x8008971,0x1e400000,0x8009171,0x1e400000,0x83c9971,0x1e5a2e00,0x300a171,0x1f200195,0x800a971,0x1f200000,0x800b171,0x1f200000,0x84cb971,0x1f424a00,0x400c171,0x1a3,0x800c971,0x20000000,0x800d171,0x20000000,0x85cd971,0x201a6600,0x500e171,0x20e001b1,0x800e971,0x20e00000,0x800f171,0x20e00000,0x86cf971,0x21028200,0x6000171,0x1bf,0x8000971,0x21c00000,0x8001171,0x21c00000,0x87c1971,0x21d29e00,0x8002171,0x22980000,0x8002971,0x22980000,0x8003171,0x22980000,0x280c3871,0x229aba00,0x8004071,0x0,0x8004871,0x23500000,0x8005071,0x23500000,0x81c5a72,0x2352d600,0x8006272,0x23f00000,0x8006a72,0x23f00000,0x8007272,0x23f00000,0x82c7a73,0x23f2f200,0x8008273,0x0,0x8008a73,0x24700000,0x8009273,0x24700000,0x83c9a74,0x24730e00,0x800a274,0x24d80000,0x800aa74,0x24d80000,0x800b274,0x24d80000,0x84cba75,0x24db2a00,0x800c275,0x0,0x800ca75,0x25200000,0x800d275,0x25200000,0x85cda76,0x25234600,0x800e276,0x25500000,0x800ea76,0x25500000,0x800f276,0x25500000,0x86cfa77,0x25536200,0x8000277,0x0,0x8000277,0x0,0x8000277,0x0,0x87c0208,0x37e00,0x8000208,0x0,0x8000208,0x0,0x8000208,0x0,0x88000000,0x0,0x88000000,0x0,0x88000000,0x0,0x88000000,0x0,0x88000000,0x0,0x88000000,0x0,0x88000000,0x0,0x88000000,0x0,0x88000000,0x0,0x88000000,0x0,0x88000000,0x0,0x88000000,0x0,0x88000000,0x0,0x88000000,0x0,0x88000000,0x0,0x88000000,0x0,0x88000000,0x0,0x88000000,0x0,0x88000000,0x0,0x88000000,0x0,0x88000000,0x0,
0x6000208,0x1d2,0x8000a08,0x22c0000,0x8001208,0x22c0000,0x7781a77,0x2380010,0x8002277,0x2580000,0x8002a77,0x2580000,0x8003277,0x2580000,0x8083b77,0x2780000,0x4377,0x1e,0x8004b77,0x4bc0000,0x8005377,0x4bc0000,0x8185971,0x4d40000,0x1006171,0x520002c,0x8006971,0x5200000,0x8007171,0x5200000,0x8287971,0x5400000,0x2008171,0x3a,0x8008971,0x7bc0000,0x8009171,0x7bc0000,0x8389971,0x7d40000,0x300a171,0x8580048,0x800a971,0x8580000,0x800b171,0x8580000,0x848b971,0x8780000,0x400c171,0x56,0x8000171,0x0,0x8000171,0x0,0x8580171,0x0,0x8000308,0x0,0x800ca08,0xb2c0000,0x800d208,0xb2c0000,0x800da08,0xb440000,0x500e071,0xc000064,0x800e971,0xc000000,0x800f171,0xc000000,0x868f971,0xc200000,0x6000171,0x2,0x8000971,0xce00000,0x8001171,0xce00000,0x177c1971,0xcfba480,0x18002171,0xdc00000,0x18002971,0xdc00000,0x18003171,0xdc00000,0x80c3971,0xde02000,0x4171,0x8e,0x48804971,0xea00000,0x8005171,0xea00000,0x81c5971,0xeb83c00,0x1006171,0xf80009c,0x8006971,0xf800000,0x8007171,0xf800000,0x82c7971,0xfa05800,0x2008171,0xaa,0x8008971,0x10600000,0x8009171,0x10600000,0x83c9971,0x10787400,0x300a171,0x114000b8,0x800a971,0x11400000,0x800b171,0x11400000,0x84cb971,0x11609000,0x400c171,0xc6,0x800c971,0x12200000,0x800d171,0x12200000,0x85cd971,0x1238ac00,0x500e171,0x130000d4,0x800e971,0x13000000,0x800f171,0x13000000,0x86cf971,0x1320c800,0x6000171,0x72,0x8000971,0x13e00000,0x8001171,0x13e00000,0x77c1971,0x13f804f0,0x8002171,0x14c00000,0x8002971,0x14c00000,0x8003171,0x14c00000,0x80c3971,0x14e10000,0x4171,0xfe,0x8004971,0x15a00000,0x8005171,0x15a00000,0x81c5971,0x15b91c00,0x1006171,0x1680010c,0x8006971,0x16800000,0x8007171,0x16800000,0x82c7971,0x16a13800,0x2008171,0x11a,0x8008971,0x17600000,0x8009171,0x17600000,0x83c9971,0x17795400,0x300a171,0x18400128,0x800a971,0x18400000,0x800b171,0x18400000,0x84cb971,0x18617000,0x400c171,0x136,0x800c971,0x19200000,0x800d171,0x19200000,0x85cd971,0x19398c00,0x500e171,0x1a000144,0x800e971,0x1a000000,0x800f171,0x1a000000,0x86cf971,0x1a21a800,0x6000171,0xe2,0x8000971,0x1ae00000,0x8001171,0x1ae00000,0x77c1971,0x1af8e560,0x8002171,0x1bc00000,0x8002971,0x1bc00000,0x8003171,0x1bc00000,0x80c3971,0x1be1e000,0x4171,0x16e,0x8004971,0x1ca00000,0x8005171,0x1ca00000,0x81c5971,0x1cb9fc00,0x1006171,0x1d80017c,0x8006971,0x1d800000,0x8007171,0x1d800000,0x82c7971,0x1da21800,0x2008171,0x18a,0x8008971,0x1e600000,0x8009171,0x1e600000,0x83c9971,0x1e7a3400,0x300a171,0x1f400198,0x800a971,0x1f400000,0x800b171,0x1f400000,0x84cb971,0x1f625000,0x400c171,0x1a6,0x800c971,0x20200000,0x800d171,0x20200000,0x85cd971,0x203a6c00,0x500e171,0x210001b4,0x800e971,0x21000000,0x800f171,0x21000000,0x86cf971,0x21228800,0x6000171,0x152,0x8000971,0x21e00000,0x8001171,0x21e00000,0x87c1971,0x21edc400,0x8002171,0x22b40000,0x8002971,0x22b40000,0x8003171,0x22b40000,0x280c3871,0x22b6c000,0x8004071,0x0,0x8004871,0x23680000,0x8005071,0x23680000,0x81c5a72,0x236adc00,0x8006272,0x24040000,0x8006a72,0x24040000,0x8007272,0x24040000,0x82c7a73,0x2406f800,0x8008273,0x0,0x8008a73,0x24800000,0x8009273,0x24800000,0x83c9a74,0x24831400,0x800a274,0x24e40000,0x800aa74,0x24e40000,0x800b274,0x24e40000,0x84cba75,0x24e73000,0x800c275,0x0,0x800ca75,0x25280000,0x800d275,0x25280000,0x85cda76,0x252b4c00,0x800e276,0x25540000,0x800ea76,0x25540000,0x800f276,0x25540000,0x86cfa77,0x25576800,0x8000277,0x0,0x8000277,0x0,0x8000277,0x0,0x87c0208,0x2a400,0x8000208,0x0,0x8000208,0x0,0x8000208,0x0,0x88000000,0x0,0x88000000,0x0,0x88000000,0x0,0x88000000,0x0,0x88000000,0x0,0x88000000,0x0,0x88000000,0x0,0x88000000,0x0,0x88000000,0x0,0x88000000,0x0,0x88000000,0x0,0x88000000,0x0,0x88000000,0x0,0x88000000,0x0,0x88000000,0x0,0x88000000,0x0,0x88000000,0x0,0x88000000,0x0,0x88000000,0x0,0x88000000,0x0,0x88000000,0x0,
0x6000208,0x1d3,0x8000a08,0x2300000,0x8001208,0x2300000,0x7781a77,0x2400013,0x8002277,0x2600000,0x8002a77,0x2600000,0x8003277,0x2600000,0x8083b77,0x2800000,0x4377,0x21,0x8004b77,0x4c80000,0x8005377,0x4c80000,0x8185971,0x4e00000,0x1006171,0x530002f,0x8006971,0x5300000,0x8007171,0x5300000,0x8287971,0x5500000,0x2008171,0x3d,0x8008971,0x7d00000,0x8009171,0x7d00000,0x8389971,0x7e80000,0x300a171,0x870004b,0x800a971,0x8700000,0x800b171,0x8700000,0x848b971,0x8900000,0x400c171,0x59,0x8000171,0x0,0x8000171,0x0,0x8580171,0x0,0x8000308,0x0,0x800ca08,0xb480000,0x800d208,0xb480000,0x800da08,0xb600000,0x500e071,0xc200067,0x800e971,0xc200000,0x800f171,0xc200000,0x868f971,0xc400000,0x6000171,0x5,0x8000971,0xd000000,0x8001171,0xd000000,0x177c1971,0xd1ba683,0x18002171,0xde00000,0x18002971,0xde00000,0x18003171,0xde00000,0x80c3971,0xe002600,0x4171,0x91,0x48804971,0xec00000,0x8005171,0xec00000,0x81c5971,0xed84200,0x1006171,0xfa0009f,0x8006971,0xfa00000,0x8007171,0xfa00000,0x82c7971,0xfc05e00,0x2008171,0xad,0x8008971,0x10800000,0x8009171,0x10800000,0x83c9971,0x10987a00,0x300a171,0x116000bb,0x800a971,0x11600000,0x800b171,0x11600000,0x84cb971,0x11809600,0x400c171,0xc9,0x800c971,0x12400000,0x800d171,0x12400000,0x85cd971,0x1258b200,0x500e171,0x132000d7,0x800e971,0x13200000,0x800f171,0x13200000,0x86cf971,0x1340ce00,0x6000171,0x75,0x8000971,0x14000000,0x8001171,0x14000000,0x77c1971,0x14180af3,0x8002171,0x14e00000,0x8002971,0x14e00000,0x8003171,0x14e00000,0x80c3971,0x15010600,0x4171,0x101,0x8004971,0x15c00000,0x8005171,0x15c00000,0x81c5971,0x15d92200,0x1006171,0x16a0010f,0x8006971,0x16a00000,0x8007171,0x16a00000,0x82c7971,0x16c13e00,0x2008171,0x11d,0x8008971,0x17800000,0x8009171,0x17800000,0x83c9971,0x17995a00,0x300a171,0x1860012b,0x800a971,0x18600000,0x800b171,0x18600000,0x84cb971,0x18817600,0x400c171,0x139,0x800c971,0x19400000,0x800d171,0x19400000,0x85cd971,0x19599200,0x500e171,0x1a200147,0x800e971,0x1a200000,0x800f171,0x1a200000,0x86cf971,0x1a41ae00,0x6000171,0xe5,0x8000971,0x1b000000,0x8001171,0x1b000000,0x77c1971,0x1b18eb63,0x8002171,0x1be00000,0x8002971,0x1be00000,0x8003171,0x1be00000,0x80c3971,0x1c01e600,0x4171,0x171,0x8004971,0x1cc00000,0x8005171,0x1cc00000,0x81c5971,0x1cda0200,0x1006171,0x1da0017f,0x8006971,0x1da00000,0x8007171,0x1da00000,0x82c7971,0x1dc21e00,0x2008171,0x18d,0x8008971,0x1e800000,0x8009171,0x1e800000,0x83c9971,0x1e9a3a00,0x300a171,0x1f60019b,0x800a971,0x1f600000,0x800b171,0x1f600000,0x84cb971,0x1f825600,0x400c171,0x1a9,0x800c971,0x20400000,0x800d171,0x20400000,0x85cd971,0x205a7200,0x500e171,0x212001b7,0x800e971,0x21200000,0x800f171,0x21200000,0x86cf971,0x21428e00,0x6000171,0x155,0x8000971,0x22000000,0x8001171,0x22000000,0x87c1971,0x2209ca00,0x8002171,0x22d00000,0x8002971,0x22d00000,0x8003171,0x22d00000,0x280c3871,0x22d2c600,0x8004071,0x0,0x8004871,0x23800000,0x8005071,0x23800000,0x81c5a72,0x2382e200,0x8006272,0x24180000,0x8006a72,0x24180000,0x8007272,0x24180000,0x82c7a73,0x241afe00,0x8008273,0x0,0x8008a73,0x24900000,0x8009273,0x24900000,0x83c9a74,0x24931a00,0x800a274,0x24f00000,0x800aa74,0x24f00000,0x800b274,0x24f00000,0x84cba75,0x24f33600,0x800c275,0x0,0x800ca75,0x25300000,0x800d275,0x25300000,0x85cda76,0x25335200,0x800e276,0x25580000,0x800ea76,0x25580000,0x800f276,0x25580000,0x86cfa77,0x255b6e00,0x8000277,0x0,0x8000277,0x0,0x8000277,0x0,0x87c0208,0x2aa00,0x8000208,0x0,0x8000208,0x0,0x8000208,0x0,0x88000000,0x0,0x88000000,0x0,0x88000000,0x0,0x88000000,0x0,0x88000000,0x0,0x88000000,0x0,0x88000000,0x0,0x88000000,0x0,0x88000000,0x0,0x88000000,0x0,0x88000000,0x0,0x88000000,0x0,0x88000000,0x0,0x88000000,0x0,0x88000000,0x0,0x88000000,0x0,0x88000000,0x0,0x88000000,0x0,0x88000000,0x0,0x88000000,0x0,0x88000000,0x0,
0x6000208,0x1d4,0x8000a08,0x2340000,0x8001208,0x2340000,0x7781a77,0x2480016,0x8002277,0x2680000,0x8002a77,0x2680000,0x8003277,0x2680000,0x8083b77,0x2880000,0x4377,0x24,0x8004b77,0x4d40000,0x8005377,0x4d40000,0x8185971,0x4ec0000,0x1006171,0x5400032,0x8006971,0x5400000,0x8007171,0x5400000,0x8287971,0x5600000,0x2008171,0x40,0x8008971,0x7e40000,0x8009171,0x7e40000,0x8389971,0x7fc0000,0x300a171,0x888004e,0x800a971,0x8880000,0x800b171,0x8880000,0x848b971,0x8a80000,0x400c171,0x5c,0x8000171,0x0,0x8000171,0x0,0x8580171,0x0,0x8000308,0x0,0x800ca08,0xb640000,0x800d208,0xb640000,0x800da08,0xb7c0000,0x500e071,0xc40006a,0x800e971,0xc400000,0x800f171,0xc400000,0x868f971,0xc600000,0x6000171,0x8,0x8000971,0xd200000,0x8001171,0xd200000,0x177c1971,0xd3ba886,0x18002171,0xe000000,0x18002971,0xe000000,0x18003171,0xe000000,0x80c3971,0xe202c00,0x4171,0x94,0x48804971,0xee00000,0x8005171,0xee00000,0x81c5971,0xef84800,0x1006171,0xfc000a2,0x8006971,0xfc00000,0x8007171,0xfc00000,0x82c7971,0xfe06400,0x2008171,0xb0,0x8008971,0x10a00000,0x8009171,0x10a00000,0x83c9971,0x10b88000,0x300a171,0x118000be,0x800a971,0x11800000,0x800b171,0x11800000,0x84cb971,0x11a09c00,0x400c171,0xcc,0x800c971,0x12600000,0x800d171,0x12600000,0x85cd971,0x1278b800,0x500e171,0x134000da,0x800e971,0x13400000,0x800f171,0x13400000,0x86cf971,0x1360d400,0x6000171,0x78,0x8000971,0x14200000,0x8001171,0x14200000,0x77c1971,0x143810f6,0x8002171,0x15000000,0x8002971,0x15000000,0x8003171,0x15000000,0x80c3971,0x15210c00,0x4171,0x104,0x8004971,0x15e00000,0x8005171,0x15e00000,0x81c5971,0x15f92800,0x1006171,0x16c00112,0x8006971,0x16c00000,0x8007171,0x16c00000,0x82c7971,0x16e14400,0x2008171,0x120,0x8008971,0x17a00000,0x8009171,0x17a00000,0x83c9971,0x17b96000,0x300a171,0x1880012e,0x800a971,0x18800000,0x800b171,0x18800000,0x84cb971,0x18a17c00,0x400c171,0x13c,0x800c971,0x19600000,0x800d171,0x19600000,0x85cd971,0x19799800,0x500e171,0x1a40014a,0x800e971,0x1a400000,0x800f171,0x1a400000,0x86cf971,0x1a61b400,0x6000171,0xe8,0x8000971,0x1b200000,0x8001171,0x1b200000,0x77c1971,0x1b38f166,0x8002171,0x1c000000,0x8002971,0x1c000000,0x8003171,0x1c000000,0x80c3971,0x1c21ec00,0x4171,0x174,0x8004971,0x1ce00000,0x8005171,0x1ce00000,0x81c5971,0x1cfa0800,0x1006171,0x1dc00182,0x8006971,0x1dc00000,0x8007171,0x1dc00000,0x82c7971,0x1de22400,0x2008171,0x190,0x8008971,0x1ea00000,0x8009171,0x1ea00000,0x83c9971,0x1eba4000,0x300a171,0x1f80019e,0x800a971,0x1f800000,0x800b171,0x1f800000,0x84cb971,0x1fa25c00,0x400c171,0x1ac,0x800c971,0x20600000,0x800d171,0x20600000,0x85cd971,0x207a7800,0x500e171,0x214001ba,0x800e971,0x21400000,0x800f171,0x21400000,0x86cf971,0x21629400,0x6000171,0x158,0x8000971,0x22200000,0x8001171,0x22200000,0x87c1971,0x2225d000,0x8002171,0x22ec0000,0x8002971,0x22ec0000,0x8003171,0x22ec0000,0x280c3871,0x22eecc00,0x8004071,0x0,0x8004871,0x23980000,0x8005071,0x23980000,0x81c5a72,0x239ae800,0x8006272,0x242c0000,0x8006a72,0x242c0000,0x8007272,0x242c0000,0x82c7a73,0x242f0400,0x8008273,0x0,0x8008a73,0x24a00000,0x8009273,0x24a00000,0x83c9a74,0x24a32000,0x800a274,0x24fc0000,0x800aa74,0x24fc0000,0x800b274,0x24fc0000,0x84cba75,0x24ff3c00,0x800c275,0x0,0x800ca75,0x25380000,0x800d275,0x25380000,0x85cda76,0x253b5800,0x800e276,0x255c0000,0x800ea76,0x255c0000,0x800f276,0x255c0000,0x86cfa77,0x255f7400,0x8000277,0x0,0x8000277,0x0,0x8000277,0x0,0x87c0208,0x2b000,0x8000208,0x0,0x8000208,0x0,0x8000208,0x0,0x88000000,0x0,0x88000000,0x0,0x88000000,0x0,0x88000000,0x0,0x88000000,0x0,0x88000000,0x0,0x88000000,0x0,0x88000000,0x0,0x88000000,0x0,0x88000000,0x0,0x88000000,0x0,0x88000000,0x0,0x88000000,0x0,0x88000000,0x0,0x88000000,0x0,0x88000000,0x0,0x88000000,0x0,0x88000000,0x0,0x88000000,0x0,0x88000000,0x0,0x88000000,0x0,
0x6000208,0x1d5,0x8000a08,0x2380000,0x8001208,0x2380000,0x7781a77,0x2500019,0x8002277,0x2700000,0x8002a77,0x2700000,0x8003277,0x2700000,0x8083b77,0x2900000,0x4377,0x27,0x8004b77,0x4e00000,0x8005377,0x4e00000,0x8185971,0x4f80000,0x1006171,0x5500035,0x8006971,0x5500000,0x8007171,0x5500000,0x8287971,0x5700000,0x2008171,0x43,0x8008971,0x7f80000,0x8009171,0x7f80000,0x8389971,0x8100000,0x300a171,0x8a00051,0x800a971,0x8a00000,0x800b171,0x8a00000,0x848b971,0x8c00000,0x400c171,0x5f,0x8000171,0x0,0x8000171,0x0,0x8580171,0x0,0x8000308,0x0,0x800ca08,0xb800000,0x800d208,0xb800000,0x800da08,0xb980000,0x500e071,0xc60006d,0x800e971,0xc600000,0x800f171,0xc600000,0x868f971,0xc800000,0x6000171,0xb,0x8000971,0xd400000,0x8001171,0xd400000,0x177c1971,0xd5baa89,0x18002171,0xe200000,0x18002971,0xe200000,0x18003171,0xe200000,0x80c3971,0xe403200,0x4171,0x97,0x48804971,0xf000000,0x8005171,0xf000000,0x81c5971,0xf184e00,0x1006171,0xfe000a5,0x8006971,0xfe00000,0x8007171,0xfe00000,0x82c7971,0x10006a00,0x2008171,0xb3,0x8008971,0x10c00000,0x8009171,0x10c00000,0x83c9971,0x10d88600,0x300a171,0x11a000c1,0x800a971,0x11a00000,0x800b171,0x11a00000,0x84cb971,0x11c0a200,0x400c171,0xcf,0x800c971,0x12800000,0x800d171,0x12800000,0x85cd971,0x1298be00,0x500e171,0x136000dd,0x800e971,0x13600000,0x800f171,0x13600000,0x86cf971,0x1380da00,0x6000171,0x7b,0x8000971,0x14400000,0x8001171,0x14400000,0x77c1971,0x145816f9,0x8002171,0x15200000,0x8002971,0x15200000,0x8003171,0x15200000,0x80c3971,0x15411200,0x4171,0x107,0x8004971,0x16000000,0x8005171,0x16000000,0x81c5971,0x16192e00,0x1006171,0x16e00115,0x8006971,0x16e00000,0x8007171,0x16e00000,0x82c7971,0x17014a00,0x2008171,0x123,0x8008971,0x17c00000,0x8009171,0x17c00000,0x83c9971,0x17d96600,0x300a171,0x18a00131,0x800a971,0x18a00000,0x800b171,0x18a00000,0x84cb971,0x18c18200,0x400c171,0x13f,0x800c971,0x19800000,0x800d171,0x19800000,0x85cd971,0x19999e00,0x500e171,0x1a60014d,0x800e971,0x1a600000,0x800f171,0x1a600000,0x86cf971,0x1a81ba00,0x6000171,0xeb,0x8000971,0x1b400000,0x8001171,0x1b400000,0x77c1971,0x1b58f769,0x8002171,0x1c200000,0x8002971,0x1c200000,0x8003171,0x1c200000,0x80c3971,0x1c41f200,0x4171,0x177,0x8004971,0x1d000000,0x8005171,0x1d000000,0x81c5971,0x1d1a0e00,0x1006171,0x1de00185,0x8006971,0x1de00000,0x8007171,0x1de00000,0x82c7971,0x1e022a00,0x2008171,0x193,0x8008971,0x1ec00000,0x8009171,0x1ec00000,0x83c9971,0x1eda4600,0x300a171,0x1fa001a1,0x800a971,0x1fa00000,0x800b171,0x1fa00000,0x84cb971,0x1fc26200,0x400c171,0x1af,0x800c971,0x20800000,0x800d171,0x20800000,0x85cd971,0x209a7e00,0x500e171,0x216001bd,0x800e971,0x21600000,0x800f171,0x21600000,0x86cf971,0x21829a00,0x6000171,0x15b,0x8000971,0x22400000,0x8001171,0x22400000,0x87c1971,0x2241d600,0x8002171,0x23080000,0x8002971,0x23080000,0x8003171,0x23080000,0x280c3871,0x230ad200,0x8004071,0x0,0x8004871,0x23b00000,0x8005071,0x23b00000,0x81c5a72,0x23b2ee00,0x8006272,0x24400000,0x8006a72,0x24400000,0x8007272,0x24400000,0x82c7a73,0x24430a00,0x8008273,0x0,0x8008a73,0x24b00000,0x8009273,0x24b00000,0x83c9a74,0x24b32600,0x800a274,0x25080000,0x800aa74,0x25080000,0x800b274,0x25080000,0x84cba75,0x250b4200,0x800c275,0x0,0x800ca75,0x25400000,0x800d275,0x25400000,0x85cda76,0x25435e00,0x800e276,0x25600000,0x800ea76,0x25600000,0x800f276,0x25600000,0x86cfa77,0x25637a00,0x8000277,0x0,0x8000277,0x0,0x8000277,0x0,0x87c0208,0x2b600,0x8000208,0x0,0x8000208,0x0,0x8000208,0x0,0x88000000,0x0,0x88000000,0x0,0x88000000,0x0,0x88000000,0x0,0x88000000,0x0,0x88000000,0x0,0x88000000,0x0,0x88000000,0x0,0x88000000,0x0,0x88000000,0x0,0x88000000,0x0,0x88000000,0x0,0x88000000,0x0,0x88000000,0x0,0x88000000,0x0,0x88000000,0x0,0x88000000,0x0,0x88000000,0x0,0x88000000,0x0,0x88000000,0x0,0x88000000,0x0,
};

#define HIP_CHECK(expr) do { hipError_t _e = (expr); if (_e != hipSuccess) { \
    fprintf(stderr, "HIP error %s at %s:%d\n", hipGetErrorString(_e), __FILE__, __LINE__); \
    return; } } while (0)

typedef unsigned u32x4 __attribute__((ext_vector_type(4)));
typedef unsigned u32x2 __attribute__((ext_vector_type(2)));

constexpr int M = 4096, N = 16384, KS = 512, NR = 8;
constexpr unsigned SHARD = 10u << 20, JSTR = 2560u << 10, WOFF = 512u << 10, STAGE = 80u << 20, CTRL = 1u << 20;
constexpr unsigned DEADLINE = 40000000u;
constexpr unsigned C_CALL = 0  , C_TMO = 45056, C_READY = 4096  , C_TICK = 40960  , C_PROG = 32768  , C_REP = 16384  ;
constexpr unsigned SHB = CTRL + 2 * STAGE, SHSZ = 3u << 20;
constexpr unsigned DBG = SHB + 2 * SHSZ;
#define STAMP(i) do {} while (0)
#define MON_SLEEP 4
constexpr int LDS_BYTES = 131072;

struct Params {
  const unsigned char* x; const unsigned char* w; const float* sx; const float* sw; float* out;
  unsigned char* ws[NR];
  int rank;
};

__device__ __forceinline__ bool wait_words(const unsigned* words, int n, unsigned v, int lane, unsigned* tmo, unsigned call, unsigned long long t0, int slp = 1) {
  for (;;) {
    const unsigned cur = lane < n ? __hip_atomic_load(words + lane, __ATOMIC_RELAXED, __HIP_MEMORY_SCOPE_SYSTEM) : v;
    if (__all(cur >= v)) return true;
    if (__hip_atomic_load(tmo, __ATOMIC_RELAXED, __HIP_MEMORY_SCOPE_AGENT) == call || __builtin_amdgcn_s_memrealtime() - t0 > DEADLINE) {
      if (lane == 0) __hip_atomic_store(tmo, call, __ATOMIC_RELAXED, __HIP_MEMORY_SCOPE_AGENT);
      return false;
    }
    if (slp > 1) __builtin_amdgcn_s_sleep(64); else __builtin_amdgcn_s_sleep(1);
  }
}

__global__ void __launch_bounds__(256, 1) fused(Params p) {
  extern __shared__ __attribute__((aligned(16))) char smem[];
  const unsigned long long t0 = __builtin_amdgcn_s_memrealtime();
  const int tid = threadIdx.x, lane = tid & 63, wave = __builtin_amdgcn_readfirstlane(tid >> 6);
  const int bid = blockIdx.x, rank = p.rank, xcd = bid & 7, c = bid >> 3;
  unsigned char* const wsl = p.ws[rank];
  unsigned* const ctl = (unsigned*)wsl;
  unsigned* const tmo = ctl + C_TMO / 4;
  unsigned* const lcall = (unsigned*)(smem + 65536);

  const int jj = tid >> 7;
  const int pi = (c - 28) * 8 + xcd, sig = (pi + 28) % 7 + 1;
  const int ti = (xcd & 3) * 4 + (c & 3), b = (xcd >> 2) * 7 + (c >> 2);
  unsigned wlo = 0, whi = 0;
#pragma unroll
  for (int i = 0; i < NR; ++i) {
    asm("v_writelane_b32 %0, %1, %2" : "+v"(wlo) : "s"((unsigned)(unsigned long long)p.ws[i]), "i"(i));
    asm("v_writelane_b32 %0, %1, %2" : "+v"(whi) : "s"((unsigned)((unsigned long long)p.ws[i] >> 32)), "i"(i));
  }
  unsigned* const rdyp = (unsigned*)((((unsigned long long)whi << 32) | wlo) + C_READY + rank * 512) + xcd;
  if (wave == 0) asm volatile("s_mov_b64 exec, 1\n\tglobal_atomic_add v255, %0, %1, %2 sc0\n\ts_mov_b64 exec, -1" :: "v"(C_CALL + xcd * 128), "v"(1u), "s"(ctl) : "memory", "v255");
  __builtin_amdgcn_sched_barrier(0);
  unsigned tick;
  auto arrive = [&](int ph) {
    if (lane < NR) __hip_atomic_fetch_add((__attribute__((address_space(1))) unsigned*)(rdyp + ph * 32), 1u, __ATOMIC_RELAXED, __HIP_MEMORY_SCOPE_SYSTEM);
  };
  const unsigned char* const xsrc = p.x + (size_t)(bid * 16 + (lane & 15)) * KS + jj * 128 + (lane >> 4) * 32 + ((tid >> 6) & 1) * 16;
  u32x2 tw[3]; u32x4 pl[2];
  float scv = 0.f;
  unsigned call;
  if (c < 28) {
    const int q8 = tid >> 5, c8 = tid & 31;
    auto wload = [&](u32x2 (&r)[16], int un) {
      const unsigned char* src = p.w + (size_t)((un >> 6) * 128 + q8 * 16) * N + (un & 63) * 256 + c8 * 8;
#pragma unroll
      for (int i = 0; i < 16; ++i) r[i] = *(const u32x2*)(src + (size_t)i * N);
    };
    auto wstore = [&](const u32x2 (&r)[16], int un, char* lb, __amdgpu_buffer_rsrc_t rdsc) {
#pragma unroll
      for (int d = 0; d < 2; ++d) {
        unsigned o[4][4];
#pragma unroll
        for (int kq = 0; kq < 4; ++kq) {
          const unsigned a = r[kq * 4][d], bb = r[kq * 4 + 1][d], cc = r[kq * 4 + 2][d], e = r[kq * 4 + 3][d];
          const unsigned ab_lo = __builtin_amdgcn_perm(bb, a, 0x05010400u), ab_hi = __builtin_amdgcn_perm(bb, a, 0x07030602u);
          const unsigned ce_lo = __builtin_amdgcn_perm(e, cc, 0x05010400u), ce_hi = __builtin_amdgcn_perm(e, cc, 0x07030602u);
          o[0][kq] = __builtin_amdgcn_perm(ce_lo, ab_lo, 0x05040100u); o[1][kq] = __builtin_amdgcn_perm(ce_lo, ab_lo, 0x07060302u);
          o[2][kq] = __builtin_amdgcn_perm(ce_hi, ab_hi, 0x05040100u); o[3][kq] = __builtin_amdgcn_perm(ce_hi, ab_hi, 0x07060302u);
        }
#pragma unroll
        for (int cc = 0; cc < 4; ++cc) { const int n = c8 * 8 + d * 4 + cc;
          *(u32x4*)(lb + (n >> 4) * 2048 + (q8 & 1) * 1024 + ((q8 >> 1) * 16 + (n & 15)) * 16) = (u32x4){o[cc][0], o[cc][1], o[cc][2], o[cc][3]}; }
      }
      __syncthreads();
#pragma unroll
      for (int i = 0; i < 8; ++i) __builtin_amdgcn_raw_buffer_store_b128(*(const u32x4*)(lb + i * 4096 + tid * 16), rdsc, tid * 16 + i * 4096, (un >> 6) * JSTR + WOFF + (un & 63) * 32768, 17);
    };
    u32x2 r[16], r2[16];
    const int nt1 = bid % 56, un1 = (bid / 56) * 64 + (nt1 < 48 ? nt1 : nt1 + 8), un2 = (bid >> 3) * 64 + 48 + (bid & 7);
    const u32x4 xv[2] = {*(const u32x4*)xsrc, *(const u32x4*)(xsrc + 256)};
    wload(r, un1);
    __builtin_amdgcn_sched_barrier(0);
    asm volatile("s_waitcnt vmcnt(16)\n\tv_mov_b32 %0, v255" : "=v"(tick) :: "memory");
    if (tid == 0) lcall[0] = tick / 32u + 1u;
    if (bid < 32) wload(r2, un2);
#pragma unroll
    for (int k = 0; k < 3; ++k) tw[k] = *(const u32x2*)(GEMM_TAB + (b * 192 + k * 64 + lane) * 2);
    scv = p.sx[0] * p.sw[0];
    __syncthreads();
    call = __builtin_amdgcn_readfirstlane(lcall[0]);
    const __amdgpu_buffer_rsrc_t rdsc = __builtin_amdgcn_make_buffer_rsrc(wsl + CTRL + (size_t)(call & 1) * STAGE + (size_t)rank * SHARD, 0, SHARD, 0x00020000);
#pragma unroll
    for (int rd = 0; rd < 2; ++rd) __builtin_amdgcn_raw_buffer_store_b128(xv[rd], rdsc, jj * JSTR + (tid & 127) * 16, rd * 2 * JSTR + bid * 2048, 17);
    STAMP(4);
    wstore(r, un1, smem, rdsc);
    STAMP(7);
    asm volatile("s_waitcnt vmcnt(0)" ::: "memory");
    STAMP(8);
    __syncthreads();
    if (wave == 1) { arrive(1); STAMP(9); }
    if (bid < 32) {
      wstore(r2, un2, smem + 32768, rdsc);
      asm volatile("s_waitcnt vmcnt(0)" ::: "memory");
      __syncthreads();
      if (wave == 2) arrive(2);
    }
  } else {
    u32x4 x0[4];
#pragma unroll
    for (int i = 0; i < 4; ++i) x0[i] = *(const u32x4*)(p.x + (size_t)((pi * 8 + i * 2 + jj) * 16 + (lane & 15)) * KS + (lane >> 4) * 32 + ((tid >> 6) & 1) * 16);
    const u32x4 xv[2] = {*(const u32x4*)xsrc, *(const u32x4*)(xsrc + 256)};
    __builtin_amdgcn_sched_barrier(0);
    asm volatile("s_waitcnt vmcnt(6)\n\tv_mov_b32 %0, v255" : "=v"(tick) :: "memory");
    if (tid == 0) lcall[0] = tick / 32u + 1u;
    __syncthreads();
    call = __builtin_amdgcn_readfirstlane(lcall[0]);
    const __amdgpu_buffer_rsrc_t rdsc = __builtin_amdgcn_make_buffer_rsrc(wsl + CTRL + (size_t)(call & 1) * STAGE + (size_t)rank * SHARD, 0, SHARD, 0x00020000);
#pragma unroll
    for (int i = 0; i < 4; ++i) __builtin_amdgcn_raw_buffer_store_b128(x0[i], rdsc, tid * 16, pi * 16384 + i * 4096, 17);
#pragma unroll
    for (int rd = 0; rd < 2; ++rd) __builtin_amdgcn_raw_buffer_store_b128(xv[rd], rdsc, jj * JSTR + (tid & 127) * 16, rd * 2 * JSTR + bid * 2048, 17);
    __builtin_amdgcn_sched_barrier(0);
    asm volatile("buffer_inv sc0 sc1" ::: "memory");
    const __amdgpu_buffer_rsrc_t rl = __builtin_amdgcn_make_buffer_rsrc((void*)(PULL_LIST + (sig - 1) * 3328), 0, 6656, 0x00020000);
    pl[0] = __builtin_amdgcn_raw_buffer_load_b128(rl, tid * 16, 0, 1);
    pl[1] = __builtin_amdgcn_raw_buffer_load_b128(rl, tid * 16, 4096, 1);
    __builtin_amdgcn_sched_barrier(0);
    asm volatile("s_waitcnt vmcnt(3)" ::: "memory");
    __builtin_amdgcn_s_barrier();
    if (wave == 3) arrive(0);
    STAMP(5);
  }
  STAMP(1);
  call = __builtin_amdgcn_readfirstlane(call);
  const unsigned tag = call << 12;
  unsigned char* const stage = wsl + CTRL + (size_t)(call & 1) * STAGE;

  if (c >= 28) {
    if (pi == 28 && wave == 1 && lane < 8)
      __hip_atomic_store(ctl + (C_TICK + (~call & 1) * 1024 + lane * 128) / 4, 0u, __ATOMIC_RELAXED, __HIP_MEMORY_SCOPE_AGENT);
    if (pi >= 30) {
      if (wait_words(ctl + (C_READY + rank * 512 + 384) / 4, 8, call, lane, tmo, call, t0, 64)) {
        const __amdgpu_buffer_rsrc_t rs = __builtin_amdgcn_make_buffer_rsrc(wsl + SHB + (call & 1) * SHSZ, 0, SHSZ, 0x00020000);
        const __amdgpu_buffer_rsrc_t ro = __builtin_amdgcn_make_buffer_rsrc(p.out, 0, 256u << 20, 0x00020000);
        for (int q = (pi - 30) * 4 + wave; q < 256; q += 8) {
          u32x4 v[12];
#pragma unroll
          for (int i = 0; i < 12; ++i) v[i] = __builtin_amdgcn_raw_buffer_load_b128(rs, lane * 16 + i * 1024, q * 12288, 16);
#pragma unroll
          for (int i = 0; i < 12; ++i) { const unsigned ch = i * 64 + lane; __builtin_amdgcn_raw_buffer_store_b128(v[i], ro, (ch / 48u) * 65536u + (ch % 48u) * 16u, q * 1048576u + (SH0 + rank * SHG) * 128u, 2); }
        }
      } else if (tid == 0) p.out[(SH0 + rank * SHG) * 32] = __builtin_nanf("");
      return;
    }
    if (pi >= 28) {
      if (pi == 28 && wave == 2) {
        if (wait_words(ctl + (C_TICK + (call & 1) * 1024 + 7 * 128) / 4, 1, SHG * 64u, lane, tmo, call, t0, 64))
          __hip_atomic_store((unsigned*)(p.ws[lane >> 3] + C_READY + rank * 512 + 384) + (lane & 7), call, __ATOMIC_RELAXED, __HIP_MEMORY_SCOPE_SYSTEM);
      }
      if (wave != 0) return;
      unsigned lastv = 0;
      for (;;) {
        unsigned m = __hip_atomic_load(lane < 32 ? ctl + (C_PROG + (lane < 28 ? lane : 0) * 128) / 4 : ctl + (C_READY + rank * 512 + (lane < 56 ? (lane >> 3) - 4 : 0) * 128) / 4 + (lane & 7), __ATOMIC_RELAXED, __HIP_MEMORY_SCOPE_SYSTEM);
        if (lane >= 32) m = m >= call * (lane >= 40 && lane < 48 ? 28u : 4u) ? (tag | 0xfffu) : 0u;
#pragma unroll
        for (int o = 32; o >= 1; o >>= 1) m = min(m, (unsigned)__shfl_xor((int)m, o));
        m = __builtin_amdgcn_readfirstlane(m);
        const bool dead = __builtin_amdgcn_s_memrealtime() - t0 > DEADLINE;
        if (dead) { m = tag | 0xfffu; if (lane == 0) __hip_atomic_store(tmo, call, __ATOMIC_RELAXED, __HIP_MEMORY_SCOPE_AGENT); asm volatile("s_waitcnt vmcnt(0)" ::: "memory"); }
        if (m >= (tag | PULL_NOP)) m = tag | 0xfffu;
        if (m >= tag && m != lastv) { lastv = m; if (lane < 16) __hip_atomic_store(ctl + (C_REP + ((pi - 28) * 16 + lane) * 128) / 4, m, __ATOMIC_RELAXED, __HIP_MEMORY_SCOPE_AGENT); }
        if (m == (tag | 0xfffu)) return;
        __builtin_amdgcn_s_sleep(MON_SLEEP);
      }
    }
    const unsigned peer = (unsigned)(rank + sig) & 7u;
    unsigned* const myprog = ctl + (C_PROG + pi * 128) / 4;
#pragma unroll
    for (int h = 0; h < 2; ++h)
#pragma unroll
      for (int i = 0; i < 4; ++i) pl[h][i] = (pl[h][i] & 0xbfffbfffu) + ((pl[h][i] >> 14) & 0x00010001u) * (rank * SHG);
    *(u32x4*)(smem + tid * 16) = pl[0];
    if (tid < 160) *(u32x4*)(smem + 4096 + tid * 16) = pl[1];
    if (lane == 0) *(unsigned*)(smem + 66560 + wave * 4) = 0;
    if (tid == 0) __hip_atomic_store(myprog, tag, __ATOMIC_RELAXED, __HIP_MEMORY_SCOPE_AGENT);
    unsigned* const ctr = ctl + (C_TICK + (call & 1) * 1024 + (sig - 1) * 128) / 4;
    const unsigned k0 = (unsigned)((pi / 7) * 4 + wave) - 16u * PULL_R;
    __syncthreads();
    const unsigned* const rdy = ctl + (C_READY + peer * 512) / 4;
    unsigned ok;
    STAMP(2);
    {
      asm volatile(PULL_ASM_TEXT
        : [ok] "=&s"(ok)
        : [rs] "s"(p.ws[peer] + CTRL + (size_t)(call & 1) * STAGE + (size_t)peer * SHARD), [rd] "s"(stage + (size_t)peer * SHARD), [pp] "s"(myprog), [pn] "s"(wave == 0 ? 4u : 0u),
          [tag] "s"(tag), [call] "s"(call), [lims] "s"(128ull | ((unsigned long long)PULL_LIM2 << 16) | ((unsigned long long)PULL_NOP << 32) | (0xffffull << 48)), [rdy] "s"(rdy), [ctr] "s"(ctr), [col] "s"((SH0 + peer * SHG) * 128u), [rs2] "s"(p.ws[peer] + SHB + (call & 1) * SHSZ), [out] "s"(p.out),
          [k0] "v"(k0), [la] "v"(66560 + wave * 4)
        : PULL_ASM_CLOBBERS);
      if (!ok && lane == 0) {
        __hip_atomic_store(tmo, call, __ATOMIC_RELAXED, __HIP_MEMORY_SCOPE_AGENT);
        p.out[0] = __builtin_nanf("");
      }
    }
    STAMP(3);
    return;
  }

  {
    const float warm = __builtin_nontemporal_load(p.out + (size_t)(ti * 256 + tid) * N + b * 256);
    asm volatile("" :: "v"(warm));
  }
  unsigned tk[3], twn[3], tn[3], to[3];
#pragma unroll
  for (int k = 0; k < 3; ++k) {
    tk[k] = (((unsigned)rank + (tw[k][0] >> 11)) & 7u) * SHARD + ((tw[k][0] >> 14) & 3u) * JSTR;
    twn[k] = WOFF + ((tw[k][1] & 511u) + ((tw[k][1] & 511u) >= SH0 ? rank * SHG : 0u)) * 4096u;
    tn[k] = wave == 0 ? (tag | (tw[k][1] >> 18)) : 0u;
    const unsigned os = (tw[k][1] >> 9) & 511u;
    to[k] = os >= SH0 ? (ti * 256u + wave * 64u) * 768u + (os - SH0) * 128u : ((unsigned)(ti * 256 + wave * 64) * N + os * 32u) * 4u;
  }
  const float sc = __builtin_bit_cast(float, __builtin_amdgcn_readfirstlane(__builtin_bit_cast(int, scv)));
  unsigned tmof;
  STAMP(2);
  asm volatile(GEMM_ASM_TEXT
    : [tmo] "=s"(tmof)
    : [rs] "s"(stage), [ro] "s"(p.out), [sb] "s"(wsl + SHB + (call & 1) * SHSZ), [m0b] "s"(wave * 1024), [xb] "s"(__builtin_amdgcn_readfirstlane(ti * 32768 + wave * 7168)), [sc] "s"(sc),
      [prog] "s"(ctl + (C_REP + (bid & 31) * 128) / 4), [pdn] "s"(wave ? 0u : 4u), [tmoo] "v"(C_TMO - C_REP - (bid & 31) * 128), [sg0] "v"(C_TICK + (call & 1) * 1024 + 7 * 128 - C_REP - (bid & 31) * 128),
      [t0] "v"(tk[0]), [t1] "v"(tk[1]), [t2] "v"(tk[2]), [t3] "v"(twn[0]), [t4] "v"(twn[1]), [t5] "v"(twn[2]), [t6] "v"(tn[0]), [t7] "v"(tn[1]), [t8] "v"(tn[2]),
      [t9] "v"(tw[0][0]), [t10] "v"(tw[1][0]), [t11] "v"(tw[2][0]), [t12] "v"(to[0]), [t13] "v"(to[1]), [t14] "v"(to[2])
    : GEMM_ASM_CLOBBERS);
  STAMP(3);
  if (tid == 0) {
    if (tmof == 0xffffffffu) __hip_atomic_store(tmo, call, __ATOMIC_RELAXED, __HIP_MEMORY_SCOPE_AGENT);
    if (tmof == 0xffffffffu || tmof == call) p.out[(size_t)(ti * 256) * N + b * 32] = __builtin_nanf("");
  }
}

extern "C" void kernel_launch(int rank, int nranks, ncclComm_t comm, hipStream_t stream,
                              void* const* d_in, const int* in_sizes, int n_in,
                              void* d_out, int out_size, const int* in_dtypes, int out_dtype,
                              void* const* d_out_all, void* const* d_ws_all, size_t ws_bytes) {
  if (nranks != NR || (unsigned)rank >= (unsigned)NR || n_in < 4 || out_size != M * N || in_sizes[0] != M * KS || in_sizes[1] != KS * N || ws_bytes < (size_t)DBG + SHARD) {
    fprintf(stderr, "unexpected problem geometry\n"); return; }
  static bool once = false;
  if (!once) { HIP_CHECK(hipFuncSetAttribute((const void*)fused, hipFuncAttributeMaxDynamicSharedMemorySize, LDS_BYTES)); once = true; }
  Params p;
  p.x = (const unsigned char*)d_in[0]; p.w = (const unsigned char*)d_in[1]; p.sx = (const float*)d_in[2]; p.sw = (const float*)d_in[3];
  p.out = (float*)d_out; p.rank = rank;
  for (int i = 0; i < NR; ++i) p.ws[i] = (unsigned char*)d_ws_all[i];
  hipLaunchKernelGGL(fused, dim3(256), dim3(256), LDS_BYTES, stream, p);
  HIP_CHECK(hipGetLastError());
}
```
